```python
import numpy as np

kernel_name = "gpu_gemm_ar_m4096_k4096_n32768_bf16_silu"


ABSMAX_RTOL = 0.008

COMPUTE_DTYPE = "bfloat16"
EPILOGUE = "silu"
IN_QUANT = None
OUT_QUANT = None
IN_DTYPES = {"x": "bfloat16", "w_mat": "bfloat16"}
WORLD_SIZE = 8

def _to_bf16(x):
    u = np.ascontiguousarray(x, dtype=np.float32).view(np.uint32)
    return ((u + 0x7FFF + ((u >> 16) & 1)) & 0xFFFF0000).view(np.float32)

def setup_inputs():

    x = _to_bf16(np.random.randn(4096, 4096).astype(np.float32))
    w_mat = _to_bf16((np.random.randn(4096, 32768) / np.sqrt(4096)).astype(np.float32))
    return {"x": x, "w_mat": w_mat}

def _epilogue(y):
    return (y / (1.0 + np.exp(-np.clip(y, -60.0, 60.0)))).astype(np.float32)

def reference(x, w_mat):
    y = (x @ w_mat).astype(np.float32)
    return _epilogue(y)
```

```cpp
#include <hip/hip_runtime.h>
#include <rccl/rccl.h>
#include <cstdio>

#define HIP_CHECK(expr) do { hipError_t _e = (expr); if (_e != hipSuccess) { \
    fprintf(stderr, "HIP error %s at %s:%d\n", hipGetErrorString(_e), __FILE__, __LINE__); \
    return; } } while (0)

#define LAS __attribute__((address_space(3)))
typedef short bf16x8 __attribute__((ext_vector_type(8)));
typedef float f32x4 __attribute__((ext_vector_type(4)));
typedef unsigned u32x4 __attribute__((ext_vector_type(4)));
typedef __amdgpu_buffer_rsrc_t rsrc_t;

namespace {
constexpr int NRANK = 8;
constexpr int MM = 4096, KK = 4096, NN = 32768, KSH = KK / NRANK;
constexpr int KT = KK / 64, KTS = KSH / 64, NTM = MM / 256, NPAN = NN / 256;
#define CFG_NS 16
#define CFG_PW 8
constexpr int NS = CFG_NS;
constexpr int NRP = NPAN - NRANK * NS;
constexpr int NIN = NTM + NS + NRP;
constexpr int NOUT = NS * NTM;
constexpr int NFLAG = NIN * 4 + NOUT;
constexpr int NUNIT = NTM * 4 + NPAN * 4;
constexpr int NTILE = (NS + NRP) * NTM;
constexpr int GRID = 256;
constexpr int PW = CFG_PW;
constexpr int NPULL = PW * (NRANK - 1);
constexpr int NW = PW * 8;
constexpr int NG = GRID - NPULL;

constexpr unsigned KTB = 32768, HTB = 16384;
constexpr unsigned PANB = KT * KTB;
constexpr unsigned CHB = KTS * KTB;
constexpr unsigned WS_TICKET = 0;
constexpr unsigned WS_TMO = WS_TICKET + GRID * 128;
constexpr unsigned WS_FLAG = WS_TMO + 128;
constexpr unsigned FLAGB = 4096;
constexpr unsigned WS_DONE = WS_FLAG + NRANK * FLAGB;
constexpr unsigned WS_CNT = WS_DONE + NRANK * 512;
constexpr unsigned WS_CTRL_END = WS_CNT + (NTM + NPAN) * NRANK * 128;
constexpr unsigned WS_DBG = 512u << 10;
constexpr unsigned WS_PAN = 1u << 20;
constexpr unsigned WS_END = WS_PAN + (NTM + NPAN) * PANB;
constexpr unsigned OUTB = (unsigned)MM * NN * 2;
static_assert(WS_CTRL_END <= WS_DBG, "ctrl region");
static_assert((NFLAG + 64) * 4 <= (int)FLAGB, "flag region");
static_assert(NW <= 128, "done words");
constexpr unsigned STAGE_BYTES = 8 * HTB;
constexpr unsigned LDS_BYTES = STAGE_BYTES + 64;
constexpr unsigned long long DEADLINE = 50000000ull;
constexpr int AUX_SYS = 17, AUX_SYS_NT = 19, AUX_AGENT = 16;

struct Params {
    const void* x; const void* w;
    char* out[NRANK]; char* ws[NRANK];
    int rank;
};

__device__ __forceinline__ rsrc_t mk_rsrc(const void* p, unsigned bytes) {
    const unsigned long long v = (unsigned long long)p;
    const unsigned lo = __builtin_amdgcn_readfirstlane((unsigned)v), hi = __builtin_amdgcn_readfirstlane((unsigned)(v >> 32));
    return __builtin_amdgcn_make_buffer_rsrc((void*)(((unsigned long long)hi << 32) | lo), 0, bytes, 0x00020000);
}
__device__ __forceinline__ unsigned ld_sys(const unsigned* p) { return __hip_atomic_load(p, __ATOMIC_RELAXED, __HIP_MEMORY_SCOPE_SYSTEM); }
__device__ __forceinline__ unsigned ld_agent(const unsigned* p) { return __hip_atomic_load(p, __ATOMIC_RELAXED, __HIP_MEMORY_SCOPE_AGENT); }
__device__ __forceinline__ void st_sys(unsigned* p, unsigned v) { __hip_atomic_store(p, v, __ATOMIC_RELAXED, __HIP_MEMORY_SCOPE_SYSTEM); }
__device__ __forceinline__ void st_agent(unsigned* p, unsigned v) { __hip_atomic_store(p, v, __ATOMIC_RELAXED, __HIP_MEMORY_SCOPE_AGENT); }
__device__ __forceinline__ void add_agent(unsigned* p, unsigned v) { (void)__hip_atomic_fetch_add(p, v, __ATOMIC_RELAXED, __HIP_MEMORY_SCOPE_AGENT); }
#define WAIT_VM0() asm volatile("s_waitcnt vmcnt(0)" ::: "memory")

__device__ __forceinline__ bool give_up(unsigned* tmo, unsigned call, unsigned long long t0) {
    if ((unsigned)__builtin_amdgcn_readfirstlane(ld_agent(tmo)) == call) return true;
    if (__builtin_amdgcn_s_memrealtime() - t0 > DEADLINE) { st_agent(tmo, call); return true; }
    return false;
}

__device__ __forceinline__ int panel_of_pos(int pos, int consumer) {
    if (pos < 2 * NS) return (pos & 1) ? NTM + consumer * NS + (pos >> 1) : (pos >> 1);
    if (pos < NS + NTM) return pos - NS;
    return NTM + NRANK * NS + (pos - NS - NTM);
}
__device__ __forceinline__ void stile_of(int ord, int& tm, int& s) {
    if (ord < NS * NS) {
        const int L = (int)__builtin_amdgcn_sqrtf((float)ord + 0.5f), i = ord - L * L;
        if (i < L) { tm = L; s = i; } else { tm = i - L; s = L; }
    } else { const int o = ord - NS * NS; tm = NS + o / NS; s = o % NS; }
    tm = __builtin_amdgcn_readfirstlane(tm); s = __builtin_amdgcn_readfirstlane(s);
}

__device__ __forceinline__ unsigned cvt_pk_bf16(float lo, float hi) { unsigned r; asm("v_cvt_pk_bf16_f32 %0, %1, %2" : "=v"(r) : "v"(lo), "v"(hi)); return r; }
__device__ __forceinline__ float silu(float y) {
    const float c = __builtin_amdgcn_fmed3f(y, -60.0f, 60.0f);
    const float e = __builtin_amdgcn_exp2f(c * -1.44269504088896f);
    return y * __builtin_amdgcn_rcpf(1.0f + e);
}

#define STAMP(P, slot) do {} while (0)

struct FlagWin { unsigned long long mask; int base; };

__device__ __forceinline__ void pull_role(const Params& P, unsigned call, int peer, int widx, int lane) {
    const int rank = P.rank;
    char* wsl = P.ws[rank];
    const rsrc_t rwr = mk_rsrc(P.ws[peer], WS_END), rwl = mk_rsrc(wsl, WS_END);
    const rsrc_t ror = mk_rsrc(P.out[peer], OUTB), rol = mk_rsrc(P.out[rank], OUTB);
    const unsigned* flags = (const unsigned*)(wsl + WS_FLAG + peer * FLAGB);
    unsigned* tmo = (unsigned*)(wsl + WS_TMO);
    unsigned* cnt = (unsigned*)(wsl + WS_CNT);
    unsigned long long mask = 0; int base = -1000; int pend = -1; bool bad = false;

    auto wait_flag = [&](int f) {
        const unsigned d = (unsigned)(f - base);
        if (d < 64u && ((mask >> d) & 1ull)) return;
        if (pend >= 0) { WAIT_VM0(); if (lane == 0) add_agent(cnt + pend * 32, 1u); pend = -1; }
        base = f;
        const unsigned long long t0 = __builtin_amdgcn_s_memrealtime();
        for (;;) {
            const unsigned v = ld_sys(flags + f + lane);
            mask = __ballot((int)(v - call) >= 0);
            if (mask & 1ull) break;
            if (give_up(tmo, call, t0)) { bad = true; break; }
            __builtin_amdgcn_s_sleep(2);
        }
        asm volatile("" ::: "memory");
    };

    STAMP(P, 0);
    for (int b = widx; b < NIN * 64; b += NW) {
        wait_flag(b >> 4);
        if (bad) break;
        if (b == widx) STAMP(P, 1);
        const int pan = panel_of_pos(b >> 6, rank);
        const unsigned soff = WS_PAN + (unsigned)pan * PANB + (unsigned)peer * CHB + (unsigned)(b & 63) * 4096u;
#pragma unroll
        for (int i = 0; i < 4; ++i) {
            const u32x4 v = __builtin_amdgcn_raw_buffer_load_b128(rwr, lane * 16 + i * 1024, soff, AUX_SYS_NT);
            WAIT_VM0();
            __builtin_amdgcn_raw_buffer_store_b128(v, rwl, lane * 16 + i * 1024, soff, AUX_AGENT);
            if (i == 0 && pend >= 0) { if (lane == 0) add_agent(cnt + pend * 32, 1u); pend = -1; }
        }
        pend = pan * NRANK + peer;
    }
    STAMP(P, 2);
    const unsigned voff = (unsigned)(lane >> 5) * (NN * 2) + (unsigned)(lane & 31) * 16u;
    if (pend >= 0) { WAIT_VM0(); if (lane == 0) add_agent(cnt + pend * 32, 1u); pend = -1; }
    auto out_off = [&](int t) {
        int tm, s; stile_of(t >> 7, tm, s);
        return (unsigned)(tm * 256 + (t & 127) * 2) * (NN * 2) + (unsigned)(peer * NS + s) * 512u;
    };
    unsigned soff = out_off(widx);
    for (int t = widx; t < NOUT * 128; t += NW) {
        if (!bad) wait_flag(NIN * 4 + (t >> 7));
        if (t == widx) STAMP(P, 3);
        u32x4 v = {0x7fc07fc0u, 0x7fc07fc0u, 0x7fc07fc0u, 0x7fc07fc0u};
        if (!bad) v = __builtin_amdgcn_raw_buffer_load_b128(ror, voff, soff, AUX_SYS_NT);
        const unsigned soffn = out_off(t + NW < NOUT * 128 ? t + NW : t);
        __builtin_amdgcn_raw_buffer_store_b128(v, rol, voff, soff, AUX_AGENT);
        soff = soffn;
    }
    WAIT_VM0();
    STAMP(P, 4);
    if (lane == 0) st_sys((unsigned*)(P.ws[peer] + WS_DONE + rank * 512) + widx, call);
    if (widx == 0) {
        const unsigned* dn = (const unsigned*)(wsl + WS_DONE + peer * 512);
        const unsigned long long t0 = __builtin_amdgcn_s_memrealtime();
        for (;;) {
            bool ok = true;
            if (lane < NW) ok = (int)(ld_sys(dn + lane) - call) >= 0;
            if (NW > 64 && lane + 64 < NW) ok = ok && (int)(ld_sys(dn + 64 + lane) - call) >= 0;
            if (__all(ok)) break;
            if (give_up(tmo, call, t0)) {
                if (lane == 0) __builtin_amdgcn_raw_buffer_store_b32(0x7fc07fc0u, rol, (unsigned)(rank * (NS > 0 ? NS : 0)) * 512u, 0, AUX_SYS);
                break;
            }
            __builtin_amdgcn_s_sleep(2);
        }
        STAMP(P, 5);
    }
}

__device__ __forceinline__ void pack_role(const Params& P, unsigned tk, int g, int wid, int lane) {
    const int rank = P.rank;
    char* wsl = P.ws[rank];
    const rsrc_t rwl = mk_rsrc(wsl, WS_END);
    const rsrc_t rx = mk_rsrc(P.x, (unsigned)MM * KSH * 2), rw = mk_rsrc(P.w, (unsigned)KSH * NN * 2);
    const int fr = lane & 15, fq = lane >> 4;
    for (int u = g; u < NUNIT; u += NG) {
        int pos, q, dest;
        if (u < 36 * NS) {
            const int s = u / 36, r = u % 36;
            if (r < 4) { pos = 2 * s; q = r; dest = -1; }
            else { pos = 2 * s + 1; q = (r - 4) & 3; dest = (rank + 1 + ((r - 4) >> 2)) & (NRANK - 1); }
        } else { const int v = u - 36 * NS; pos = 2 * NS + (v >> 2); q = v & 3; dest = -1; }
        const int pan = panel_of_pos(pos, dest < 0 ? 0 : dest);
        const int ktl = q * 2 + (wid >> 2);
        const unsigned dst = WS_PAN + (unsigned)pan * PANB + (unsigned)(rank * KTS + ktl) * KTB;
        u32x4 v[8];
        if (pan < NTM) {
            const int rb0 = (wid & 3) * 4;
            const unsigned vo = (unsigned)fr * (KSH * 2) + (unsigned)fq * 16u;
            const unsigned so = (unsigned)(pan * 256 + rb0 * 16) * (KSH * 2) + (unsigned)ktl * 128u;
#pragma unroll
            for (int i = 0; i < 8; ++i) v[i] = __builtin_amdgcn_raw_buffer_load_b128(rx, vo + (i & 1) * 64, so + (unsigned)(i >> 1) * (16 * KSH * 2), 0);
#pragma unroll
            for (int i = 0; i < 8; ++i) __builtin_amdgcn_raw_buffer_store_b128(v[i], rwl, lane * 16 + (i & 1) * 1024, dst + (unsigned)(rb0 + (i >> 1)) * 2048u, AUX_SYS);
        } else {
            const int ks = (wid >> 1) & 1, wr = wid & 1;
            const unsigned vo = (unsigned)(fq * 8) * (NN * 2) + (unsigned)fr * 16u;
            const unsigned so = (unsigned)(ktl * 64 + ks * 32) * (NN * 2) + (unsigned)((pan - NTM) * 256 + wr * 128) * 2u;
#pragma unroll
            for (int j = 0; j < 8; ++j) v[j] = __builtin_amdgcn_raw_buffer_load_b128(rw, vo, so + (unsigned)j * (NN * 2), 2);
#pragma unroll
            for (int i = 0; i < 8; ++i) {
                u32x4 o;
#pragma unroll
                for (int e = 0; e < 4; ++e) o[e] = __builtin_amdgcn_perm(v[2 * e + 1][i >> 1], v[2 * e][i >> 1], (i & 1) ? 0x07060302u : 0x05040100u);
                __builtin_amdgcn_raw_buffer_store_b128(o, rwl, lane * 16, dst + (unsigned)(i >> 2) * HTB + (unsigned)wr * 8192u + (unsigned)(i & 3) * 2048u + (unsigned)ks * 1024u, AUX_SYS);
            }
        }
        WAIT_VM0();
        __builtin_amdgcn_s_barrier();
        if (wid == 0 && lane < NRANK && (dest < 0 || dest == lane)) {
            if (lane == rank) add_agent((unsigned*)(wsl + WS_CNT) + (pan * NRANK + rank) * 32, 16u);
            else { unsigned c = tk; asm volatile("" : "+v"(c)); st_sys((unsigned*)(P.ws[lane] + WS_FLAG + rank * FLAGB) + pos * 4 + q, c + 1u); }
        }
    }
}

struct Tile { int tm, pan, ord; };
__device__ __forceinline__ bool tile_of(int idx, int rank, Tile& t) {
    if (idx >= NTILE) return false;
    if (idx < NOUT) { int s; stile_of(idx, t.tm, s); t.pan = rank * NS + s; t.ord = idx; }
    else { const int o = idx - NOUT; t.tm = o & (NTM - 1); t.pan = NRANK * NS + (o >> 4); t.ord = -1; }
    return true;
}

__device__ __forceinline__ void gemm_role(const Params& P, unsigned call, int g, LAS unsigned char* lds) {
    const int tid = threadIdx.x, wid = __builtin_amdgcn_readfirstlane(tid >> 6), lane = tid & 63, wr = wid >> 2, wc = wid & 3;
    const int rank = P.rank;
    char* wsl = P.ws[rank];
    const rsrc_t rws = mk_rsrc(wsl, WS_END);
    LAS unsigned* ctl = (LAS unsigned*)(lds + STAGE_BYTES);
    const unsigned voff = (unsigned)tid * 16u;
    const unsigned ldsw = (unsigned)wid * 1024u;
    const int aoff = wr * 8192 + lane * 16, boff = wc * 4096 + lane * 16;

    auto ready = [&](const Tile& t) {
        if (wid == 0) {
            const unsigned* c = (const unsigned*)(wsl + WS_CNT) + ((((lane & 8) ? NTM + t.pan : t.tm) * NRANK + (lane & 7)) * 32);
            unsigned* tmo = (unsigned*)(wsl + WS_TMO);
            const unsigned need = call * 64u;
            const unsigned long long t0 = __builtin_amdgcn_s_memrealtime();
            for (;;) {
                const bool ok = (int)(ld_agent(c) - need) >= 0;
                if (__all(ok)) break;
                if (give_up(tmo, call, t0)) { ctl[0] = 1u; break; }
                __builtin_amdgcn_s_sleep(4);
            }
            __builtin_amdgcn_fence(__ATOMIC_ACQUIRE, "agent");
            asm volatile("s_waitcnt vmcnt(0) lgkmcnt(0)" ::: "memory");
        }
        asm volatile("" ::: "memory"); __builtin_amdgcn_s_barrier(); asm volatile("" ::: "memory");
    };

#define SA(b, h) (((b) * 2 + (h)) * HTB)
#define SB(b, h) ((4 + (b) * 2 + (h)) * HTB)
#define STAGE(bufoff, soff) do { _Pragma("unroll") for (int _i = 0; _i < 2; ++_i) \
        __builtin_amdgcn_raw_ptr_buffer_load_lds(rws, (LAS unsigned*)(lds + (bufoff) + ldsw + _i * 8192), 16, voff, (soff) + _i * 8192u, 0, 0); } while (0)
#define LDA(dst, b, h) do { _Pragma("unroll") for (int m = 0; m < 4; ++m) _Pragma("unroll") for (int k = 0; k < 2; ++k) dst[m][k] = *(const LAS bf16x8*)(lds + SA(b, h) + aoff + m * 2048 + k * 1024); } while (0)
#define LDB(dst, b, h) do { _Pragma("unroll") for (int n = 0; n < 2; ++n) _Pragma("unroll") for (int k = 0; k < 2; ++k) dst[n][k] = *(const LAS bf16x8*)(lds + SB(b, h) + boff + n * 2048 + k * 1024); } while (0)
#define MMA(ai, bj, At, Bt) do { __builtin_amdgcn_s_setprio(1); _Pragma("unroll") for (int m = 0; m < 4; ++m) _Pragma("unroll") for (int n = 0; n < 2; ++n) _Pragma("unroll") for (int k = 0; k < 2; ++k) \
        acc[ai][bj][m][n] = __builtin_amdgcn_mfma_f32_16x16x32_bf16(Bt[n][k], At[m][k], acc[ai][bj][m][n], 0, 0, 0); __builtin_amdgcn_s_setprio(0); } while (0)
#define WAIT_V(n) asm volatile("s_waitcnt vmcnt(" #n ")" ::: "memory")
#define WAIT_L(n) asm volatile("s_waitcnt lgkmcnt(" #n ")" ::: "memory")
#define BAR __builtin_amdgcn_s_barrier()
#define SCHED __builtin_amdgcn_sched_barrier(0)

    Tile cur, nxt; int ui = g;
    if (!tile_of(ui, rank, cur)) return;
    f32x4 acc[2][2][4][2];
#pragma unroll
    for (int a = 0; a < 2; ++a)
#pragma unroll
        for (int b = 0; b < 2; ++b)
#pragma unroll
            for (int m = 0; m < 4; ++m)
#pragma unroll
                for (int n = 0; n < 2; ++n) acc[a][b][m][n] = (f32x4){0.f, 0.f, 0.f, 0.f};
    bf16x8 At[4][2], B0[2][2], B1[2][2];
    unsigned cA = WS_PAN + (unsigned)(NTM + cur.pan) * PANB, cB = WS_PAN + (unsigned)cur.tm * PANB;
    ready(cur);
    STAMP(P, 2);
    STAGE(SB(0, 0), cB); STAGE(SA(0, 0), cA); STAGE(SB(0, 1), cB + HTB); STAGE(SA(0, 1), cA + HTB);
    if (wr == 1) BAR;
    WAIT_V(4); BAR;
    STAGE(SB(1, 0), cB + KTB); STAGE(SA(1, 0), cA + KTB); STAGE(SB(1, 1), cB + HTB + KTB);
    WAIT_V(6); BAR;
    for (;;) {
        const bool has_next = tile_of(ui + NG, rank, nxt);
        const unsigned nA = has_next ? WS_PAN + (unsigned)(NTM + nxt.pan) * PANB : cA, nB = has_next ? WS_PAN + (unsigned)nxt.tm * PANB : cB;
        for (int t = 0; t < KT; t += 2) {
            const bool last = (t == KT - 2);
            const unsigned a1 = cA + (unsigned)(t + 1) * KTB;
            const unsigned a2 = last ? nA : cA + (unsigned)(t + 2) * KTB, b2 = last ? nB : cB + (unsigned)(t + 2) * KTB;
            const unsigned a3 = a2 + KTB, b3 = b2 + KTB;
            if (last && has_next) ready(nxt);
            LDB(B0, 0, 0); SCHED; LDA(At, 0, 0); STAGE(SA(1, 1), a1 + HTB);
            WAIT_L(8); BAR; WAIT_L(0); MMA(0, 0, At, B0); BAR; SCHED;
            LDB(B1, 0, 1); STAGE(SB(0, 0), b2);
            BAR; WAIT_L(0); MMA(0, 1, At, B1); BAR;
            LDA(At, 0, 1); STAGE(SA(0, 0), a2);
            BAR; WAIT_L(0); MMA(1, 0, At, B0); BAR; SCHED;
            STAGE(SB(0, 1), b2 + HTB);
            WAIT_V(6); BAR; MMA(1, 1, At, B1); BAR;
            LDB(B0, 1, 0); SCHED; LDA(At, 1, 0); STAGE(SA(0, 1), a2 + HTB);
            WAIT_L(8); BAR; WAIT_L(0); MMA(0, 0, At, B0); BAR; SCHED;
            LDB(B1, 1, 1); STAGE(SB(1, 0), b3);
            BAR; WAIT_L(0); MMA(0, 1, At, B1); BAR;
            LDA(At, 1, 1); STAGE(SA(1, 0), a3);
            BAR; WAIT_L(0); MMA(1, 0, At, B0); BAR; SCHED;
            STAGE(SB(1, 1), b3 + HTB);
            WAIT_V(6); BAR; MMA(1, 1, At, B1); BAR;
        }
        {
            int tz = threadIdx.x; asm volatile("" : "+v"(tz));
            const int ln = tz & 63, fr = ln & 15, fq = ln >> 4;
            const rsrc_t rol = mk_rsrc(P.out[rank], OUTB);
            const bool bad = ctl[0] != 0u;
            const unsigned vo = (unsigned)(wc * 32 + fq * 4) * (NN * 2) + (unsigned)(wr * 256 + fr * 16);
            const unsigned so = (unsigned)(cur.tm * 256) * (NN * 2) + (unsigned)cur.pan * 512u;
#pragma unroll
            for (int bj = 0; bj < 2; ++bj)
#pragma unroll
                for (int n = 0; n < 2; ++n)
#pragma unroll
                    for (int r = 0; r < 4; ++r) {
                        u32x4 o;
                        o[0] = cvt_pk_bf16(silu(acc[0][bj][0][n][r]), silu(acc[0][bj][1][n][r]));
                        o[1] = cvt_pk_bf16(silu(acc[0][bj][2][n][r]), silu(acc[0][bj][3][n][r]));
                        o[2] = cvt_pk_bf16(silu(acc[1][bj][0][n][r]), silu(acc[1][bj][1][n][r]));
                        o[3] = cvt_pk_bf16(silu(acc[1][bj][2][n][r]), silu(acc[1][bj][3][n][r]));
                        if (bad) o = (u32x4){0x7fc07fc0u, 0x7fc07fc0u, 0x7fc07fc0u, 0x7fc07fc0u};
                        __builtin_amdgcn_raw_buffer_store_b128(o, rol, vo, so + (unsigned)(bj * 128 + n * 16 + r) * (NN * 2), AUX_SYS);
                    }
            if (cur.ord >= 0) {
                WAIT_VM0();
                unsigned old = 0;
                if (ln == 0) old = __hip_atomic_fetch_add(ctl + 1, 1u, __ATOMIC_RELAXED, __HIP_MEMORY_SCOPE_WORKGROUP);
                old = __builtin_amdgcn_readfirstlane(old);
                if ((old & 7u) == 7u && ln < NRANK && ln != rank)
                    st_sys((unsigned*)(P.ws[ln] + WS_FLAG + rank * FLAGB) + NIN * 4 + cur.ord, call);
            }
        }
        STAMP(P, 3 + (ui >= NG ? 1 : 0));
        if (!has_next) break;
#pragma unroll
        for (int a = 0; a < 2; ++a)
#pragma unroll
            for (int b = 0; b < 2; ++b)
#pragma unroll
                for (int m = 0; m < 4; ++m)
#pragma unroll
                    for (int n = 0; n < 2; ++n) acc[a][b][m][n] = (f32x4){0.f, 0.f, 0.f, 0.f};
        cur = nxt; cA = nA; cB = nB; ui += NG;
    }
    WAIT_V(0);
    if (wr == 0) BAR;
    BAR;
}

__global__ __launch_bounds__(512, 2) void fused_gemm_ar(Params P) {
    extern __shared__ __attribute__((aligned(16))) unsigned char shm[];
    LAS unsigned char* lds = (LAS unsigned char*)shm;
    const int wid = __builtin_amdgcn_readfirstlane(threadIdx.x >> 6), lane = threadIdx.x & 63;
    const int b = __builtin_amdgcn_readfirstlane(blockIdx.x);
    unsigned* tkp = (unsigned*)(P.ws[P.rank] + WS_TICKET + b * 128 + wid * 16);
    const unsigned tk = ld_agent(tkp);
    if (b < NPULL) {
        const unsigned call = __builtin_amdgcn_readfirstlane(tk) + 1u;
        st_agent(tkp, call);
        const int pi = b / PW, slot = b % PW;
        const int peer = (P.rank + 1 + pi) & (NRANK - 1);
        pull_role(P, call, peer, slot * 8 + wid, lane);
    } else {
        const int g = b - NPULL;
        ((LAS unsigned*)(lds + STAGE_BYTES))[lane & 1] = 0u;
        STAMP(P, 0);
        pack_role(P, tk, g, wid, lane);
        const unsigned call = __builtin_amdgcn_readfirstlane(tk) + 1u;
        st_agent(tkp, call);
        STAMP(P, 1);
        asm volatile("s_waitcnt lgkmcnt(0)" ::: "memory");
        __builtin_amdgcn_s_barrier();
        gemm_role(P, call, g, lds);
    }
}
}

extern "C" void kernel_launch(int rank, int nranks, ncclComm_t comm,
                              hipStream_t stream,
                              void* const* d_in, const int* in_sizes, int n_in,
                              void* d_out, int out_size,
                              const int* in_dtypes, int out_dtype,
                              void* const* d_out_all,
                              void* const* d_ws_all, size_t ws_bytes) {
    if (nranks != NRANK || (unsigned)rank >= (unsigned)NRANK || ws_bytes < (size_t)WS_END || n_in < 2 ||
        in_sizes[0] != MM * KSH || in_sizes[1] != KSH * NN || out_size != MM * NN) {
        fprintf(stderr, "unexpected problem geometry\n");
        return;
    }
    static bool attr_done[NRANK] = {};
    if (!attr_done[rank]) {
        HIP_CHECK(hipFuncSetAttribute((const void*)fused_gemm_ar, hipFuncAttributeMaxDynamicSharedMemorySize, LDS_BYTES));
        attr_done[rank] = true;
    }
    Params P;
    P.x = d_in[0]; P.w = d_in[1]; P.rank = rank;
    for (int p = 0; p < NRANK; ++p) { P.out[p] = (char*)d_out_all[p]; P.ws[p] = (char*)d_ws_all[p]; }
    hipLaunchKernelGGL(fused_gemm_ar, dim3(GRID), dim3(512), LDS_BYTES, stream, P);
    HIP_CHECK(hipGetLastError());
}
```

```cpp
#include <hip/hip_runtime.h>
#include <rccl/rccl.h>
#include <cstdio>

#define HIP_CHECK(expr) do { hipError_t _e = (expr); if (_e != hipSuccess) { \
    fprintf(stderr, "HIP error %s at %s:%d\n", hipGetErrorString(_e), __FILE__, __LINE__); \
    return; } } while (0)

#define LAS __attribute__((address_space(3)))
typedef short bf16x8 __attribute__((ext_vector_type(8)));
typedef float f32x4 __attribute__((ext_vector_type(4)));
typedef unsigned u32x4 __attribute__((ext_vector_type(4)));
typedef __amdgpu_buffer_rsrc_t rsrc_t;

namespace {
constexpr int NRANK = 8;
constexpr int MM = 4096, KK = 4096, NN = 32768, KSH = KK / NRANK;
constexpr int KT = KK / 64, KTS = KSH / 64, NTM = MM / 256, NPAN = NN / 256;
#define CFG_NS 8
#define CFG_PW 10
constexpr int NS = CFG_NS;
constexpr int NRP = NPAN - NRANK * NS;
constexpr int NIN = NTM + NS + NRP;
constexpr int NOUT = NS * NTM;
constexpr int NFLAG = NIN * 4 + NOUT;
constexpr int NUNIT = NTM * 4 + NPAN * 4;
constexpr int NTILE = (NS + NRP) * NTM;
constexpr int GRID = 256;
constexpr int PW = CFG_PW;
constexpr int NPULL = PW * (NRANK - 1);
constexpr int NW = PW * 8;
constexpr int NG = GRID - NPULL;
#define CFG_OB 2
#define CFG_U 1
constexpr int OB = CFG_OB;
constexpr int UU = CFG_U;
constexpr int NBIN = NIN * 64;
constexpr int NBLK = NBIN + NOUT * 128 / OB;

constexpr unsigned KTB = 32768, HTB = 16384;
constexpr unsigned PANB = KT * KTB;
constexpr unsigned CHB = KTS * KTB;
constexpr unsigned WS_TICKET = 0;
constexpr unsigned WS_TMO = WS_TICKET + GRID * 128;
constexpr unsigned WS_FLAG = WS_TMO + 128;
constexpr unsigned FLAGB = 4096;
constexpr unsigned WS_DONE = WS_FLAG + NRANK * FLAGB;
constexpr unsigned WS_HEAD = WS_DONE + NRANK * 512;
constexpr unsigned WS_CNT = WS_HEAD + NRANK * 128;
constexpr unsigned WS_CTRL_END = WS_CNT + (NTM + NPAN) * NRANK * 128;
constexpr unsigned WS_DBG = 512u << 10;
constexpr unsigned WS_PAN = 1u << 20;
constexpr unsigned WS_END = WS_PAN + (NTM + NPAN) * PANB;
constexpr unsigned OUTB = (unsigned)MM * NN * 2;
static_assert(WS_CTRL_END <= WS_DBG, "ctrl region");
static_assert((NFLAG + 64) * 4 <= (int)FLAGB, "flag region");
static_assert(NW <= 128, "done words");
constexpr unsigned STAGE_BYTES = 8 * HTB;
constexpr unsigned LDS_BYTES = STAGE_BYTES + 64;
constexpr unsigned long long DEADLINE = 50000000ull;
constexpr int AUX_SYS = 17, AUX_SYS_NT = 19, AUX_AGENT = 16;

struct Params {
    const void* x; const void* w;
    char* out[NRANK]; char* ws[NRANK];
    int rank;
};

__device__ __forceinline__ rsrc_t mk_rsrc(const void* p, unsigned bytes) {
    const unsigned long long v = (unsigned long long)p;
    const unsigned lo = __builtin_amdgcn_readfirstlane((unsigned)v), hi = __builtin_amdgcn_readfirstlane((unsigned)(v >> 32));
    return __builtin_amdgcn_make_buffer_rsrc((void*)(((unsigned long long)hi << 32) | lo), 0, bytes, 0x00020000);
}
__device__ __forceinline__ unsigned ld_sys(const unsigned* p) { return __hip_atomic_load(p, __ATOMIC_RELAXED, __HIP_MEMORY_SCOPE_SYSTEM); }
__device__ __forceinline__ unsigned ld_agent(const unsigned* p) { return __hip_atomic_load(p, __ATOMIC_RELAXED, __HIP_MEMORY_SCOPE_AGENT); }
__device__ __forceinline__ void st_sys(unsigned* p, unsigned v) { __hip_atomic_store(p, v, __ATOMIC_RELAXED, __HIP_MEMORY_SCOPE_SYSTEM); }
__device__ __forceinline__ void st_agent(unsigned* p, unsigned v) { __hip_atomic_store(p, v, __ATOMIC_RELAXED, __HIP_MEMORY_SCOPE_AGENT); }
__device__ __forceinline__ void add_agent(unsigned* p, unsigned v) { (void)__hip_atomic_fetch_add(p, v, __ATOMIC_RELAXED, __HIP_MEMORY_SCOPE_AGENT); }
#define WAIT_VM0() asm volatile("s_waitcnt vmcnt(0)" ::: "memory")

__device__ __forceinline__ bool give_up(unsigned* tmo, unsigned call, unsigned long long t0) {
    if ((unsigned)__builtin_amdgcn_readfirstlane(ld_agent(tmo)) == call) return true;
    if (__builtin_amdgcn_s_memrealtime() - t0 > DEADLINE) { st_agent(tmo, call); return true; }
    return false;
}

__device__ __forceinline__ int panel_of_pos(int pos, int consumer) {
    if (pos < 2 * NS) return (pos & 1) ? NTM + consumer * NS + (pos >> 1) : (pos >> 1);
    if (pos < NS + NTM) return pos - NS;
    return NTM + NRANK * NS + (pos - NS - NTM);
}
__device__ __forceinline__ void stile_of(int ord, int& tm, int& s) {
    if (ord < NS * NS) {
        const int L = (int)__builtin_amdgcn_sqrtf((float)ord + 0.5f), i = ord - L * L;
        if (i < L) { tm = L; s = i; } else { tm = i - L; s = L; }
    } else { const int o = ord - NS * NS; tm = NS + o / NS; s = o % NS; }
    tm = __builtin_amdgcn_readfirstlane(tm); s = __builtin_amdgcn_readfirstlane(s);
}

__device__ __forceinline__ unsigned cvt_pk_bf16(float lo, float hi) { unsigned r; asm("v_cvt_pk_bf16_f32 %0, %1, %2" : "=v"(r) : "v"(lo), "v"(hi)); return r; }
__device__ __forceinline__ float silu(float y) {
    const float c = __builtin_amdgcn_fmed3f(y, -60.0f, 60.0f);
    const float e = __builtin_amdgcn_exp2f(c * -1.44269504088896f);
    return y * __builtin_amdgcn_rcpf(1.0f + e);
}

#define STAMP(P, slot) do {} while (0)

struct FlagWin { unsigned long long mask; int base; };

__device__ __forceinline__ void pull_role(const Params& P, unsigned call, int peer, int widx, int lane) {
    const int rank = P.rank;
    char* wsl = P.ws[rank];
    const rsrc_t rwr = mk_rsrc(P.ws[peer], WS_END), rwl = mk_rsrc(wsl, WS_END);
    const rsrc_t ror = mk_rsrc(P.out[peer], OUTB), rol = mk_rsrc(P.out[rank], OUTB);
    const unsigned* flags = (const unsigned*)(wsl + WS_FLAG + peer * FLAGB);
    unsigned* tmo = (unsigned*)(wsl + WS_TMO);
    unsigned* cnt = (unsigned*)(wsl + WS_CNT);
    unsigned long long mask = 0; int base = -1000; int pend = -1; bool bad = false;

    auto wait_flag = [&](int f) {
        const unsigned d = (unsigned)(f - base);
        if (d < 64u && ((mask >> d) & 1ull)) return;
        if (pend >= 0) { WAIT_VM0(); if (lane == 0) add_agent(cnt + pend * 32, 1u); pend = -1; }
        base = f;
        const unsigned long long t0 = __builtin_amdgcn_s_memrealtime();
        for (;;) {
            const unsigned v = ld_sys(flags + f + lane);
            mask = __ballot((int)(v - call) >= 0);
            if (mask & 1ull) break;
            if (give_up(tmo, call, t0)) { bad = true; break; }
            __builtin_amdgcn_s_sleep(2);
        }
        asm volatile("" ::: "memory");
    };

    STAMP(P, 0);
    unsigned* head = (unsigned*)(wsl + WS_HEAD + peer * 128);
    const unsigned hbase = (call - 1u) * (unsigned)(NBLK + NW);
    auto draw = [&]() { unsigned v = 0; if (lane == 0) v = __hip_atomic_fetch_add(head, 1u, __ATOMIC_RELAXED, __HIP_MEMORY_SCOPE_AGENT); return v; };
    const unsigned voff = (unsigned)(lane >> 5) * (NN * 2) + (unsigned)(lane & 31) * 16u;
    unsigned nv = draw();
    bool first = true, firsto = true;
    for (;;) {
        const int b = (int)(__builtin_amdgcn_readfirstlane(nv) - hbase);
        if ((unsigned)b >= (unsigned)NBLK) break;
        if (b < NBIN) {
            if (!bad) wait_flag(b >> 4);
            if (bad) { nv = draw(); continue; }
            if (first) { STAMP(P, 1); first = false; }
            const int pan = panel_of_pos(b >> 6, rank);
            const unsigned soff = WS_PAN + (unsigned)pan * PANB + (unsigned)peer * CHB + (unsigned)(b & 63) * 4096u;
#pragma unroll
            for (int i = 0; i < 4; i += UU) {
                u32x4 v[UU];
#pragma unroll
                for (int k = 0; k < UU; ++k) v[k] = __builtin_amdgcn_raw_buffer_load_b128(rwr, lane * 16 + (i + k) * 1024, soff, AUX_SYS_NT);
                if (i + UU >= 4) nv = draw();
                WAIT_VM0();
#pragma unroll
                for (int k = 0; k < UU; ++k) __builtin_amdgcn_raw_buffer_store_b128(v[k], rwl, lane * 16 + (i + k) * 1024, soff, AUX_AGENT);
                if (i == 0 && pend >= 0) { if (lane == 0) add_agent(cnt + pend * 32, 1u); pend = -1; }
            }
            pend = pan * NRANK + peer;
        } else {
            if (pend >= 0) { WAIT_VM0(); if (lane == 0) add_agent(cnt + pend * 32, 1u); pend = -1; }
            if (firsto) STAMP(P, 2);
            const int t = (b - NBIN) * OB;
            if (!bad) wait_flag(NIN * 4 + (t >> 7));
            if (firsto) { STAMP(P, 3); firsto = false; }
            int tm, s; stile_of(t >> 7, tm, s);
            const unsigned soff = (unsigned)(tm * 256 + (t & 127) * 2) * (NN * 2) + (unsigned)(peer * NS + s) * 512u;
            constexpr int UO = UU < OB ? UU : OB;
#pragma unroll
            for (int i = 0; i < OB; i += UO) {
                u32x4 v[UO];
#pragma unroll
                for (int k = 0; k < UO; ++k) {
                    v[k] = (u32x4){0x7fc07fc0u, 0x7fc07fc0u, 0x7fc07fc0u, 0x7fc07fc0u};
                    if (!bad) v[k] = __builtin_amdgcn_raw_buffer_load_b128(ror, voff, soff + (unsigned)(i + k) * (2 * NN * 2), AUX_SYS_NT);
                }
                if (i + UO >= OB) nv = draw();
                WAIT_VM0();
#pragma unroll
                for (int k = 0; k < UO; ++k) __builtin_amdgcn_raw_buffer_store_b128(v[k], rol, voff, soff + (unsigned)(i + k) * (2 * NN * 2), AUX_AGENT);
            }
        }
    }
    if (pend >= 0) { WAIT_VM0(); if (lane == 0) add_agent(cnt + pend * 32, 1u); pend = -1; }
    WAIT_VM0();
    STAMP(P, 4);
    if (lane == 0) st_sys((unsigned*)(P.ws[peer] + WS_DONE + rank * 512) + widx, call);
    if (widx == 0) {
        const unsigned* dn = (const unsigned*)(wsl + WS_DONE + peer * 512);
        const unsigned long long t0 = __builtin_amdgcn_s_memrealtime();
        for (;;) {
            bool ok = true;
            if (lane < NW) ok = (int)(ld_sys(dn + lane) - call) >= 0;
            if (NW > 64 && lane + 64 < NW) ok = ok && (int)(ld_sys(dn + 64 + lane) - call) >= 0;
            if (__all(ok)) break;
            if (give_up(tmo, call, t0)) {
                if (lane == 0) __builtin_amdgcn_raw_buffer_store_b32(0x7fc07fc0u, rol, (unsigned)(rank * (NS > 0 ? NS : 0)) * 512u, 0, AUX_SYS);
                break;
            }
            __builtin_amdgcn_s_sleep(2);
        }
        STAMP(P, 5);
    }
}

__device__ __forceinline__ void pack_role(const Params& P, unsigned tk, int g, int wid, int lane) {
    const int rank = P.rank;
    char* wsl = P.ws[rank];
    const rsrc_t rwl = mk_rsrc(wsl, WS_END);
    const rsrc_t rx = mk_rsrc(P.x, (unsigned)MM * KSH * 2), rw = mk_rsrc(P.w, (unsigned)KSH * NN * 2);
    const int fr = lane & 15, fq = lane >> 4;
    for (int u = g; u < NUNIT; u += NG) {
        int pos, q, dest;
        if (u < 36 * NS) {
            const int s = u / 36, r = u % 36;
            if (r < 4) { pos = 2 * s; q = r; dest = -1; }
            else { pos = 2 * s + 1; q = (r - 4) & 3; dest = (rank + 1 + ((r - 4) >> 2)) & (NRANK - 1); }
        } else { const int v = u - 36 * NS; pos = 2 * NS + (v >> 2); q = v & 3; dest = -1; }
        const int pan = panel_of_pos(pos, dest < 0 ? 0 : dest);
        const int ktl = q * 2 + (wid >> 2);
        const unsigned dst = WS_PAN + (unsigned)pan * PANB + (unsigned)(rank * KTS + ktl) * KTB;
        u32x4 v[8];
        if (pan < NTM) {
            const int rb0 = (wid & 3) * 4;
            const unsigned vo = (unsigned)fr * (KSH * 2) + (unsigned)fq * 16u;
            const unsigned so = (unsigned)(pan * 256 + rb0 * 16) * (KSH * 2) + (unsigned)ktl * 128u;
#pragma unroll
            for (int i = 0; i < 8; ++i) v[i] = __builtin_amdgcn_raw_buffer_load_b128(rx, vo + (i & 1) * 64, so + (unsigned)(i >> 1) * (16 * KSH * 2), 0);
#pragma unroll
            for (int i = 0; i < 8; ++i) __builtin_amdgcn_raw_buffer_store_b128(v[i], rwl, lane * 16 + (i & 1) * 1024, dst + (unsigned)(rb0 + (i >> 1)) * 2048u, AUX_SYS);
        } else {
            const int ks = (wid >> 1) & 1, wr = wid & 1;
            const unsigned vo = (unsigned)(fq * 8) * (NN * 2) + (unsigned)fr * 16u;
            const unsigned so = (unsigned)(ktl * 64 + ks * 32) * (NN * 2) + (unsigned)((pan - NTM) * 256 + wr * 128) * 2u;
#pragma unroll
            for (int j = 0; j < 8; ++j) v[j] = __builtin_amdgcn_raw_buffer_load_b128(rw, vo, so + (unsigned)j * (NN * 2), 2);
#pragma unroll
            for (int i = 0; i < 8; ++i) {
                u32x4 o;
#pragma unroll
                for (int e = 0; e < 4; ++e) o[e] = __builtin_amdgcn_perm(v[2 * e + 1][i >> 1], v[2 * e][i >> 1], (i & 1) ? 0x07060302u : 0x05040100u);
                __builtin_amdgcn_raw_buffer_store_b128(o, rwl, lane * 16, dst + (unsigned)(i >> 2) * HTB + (unsigned)wr * 8192u + (unsigned)(i & 3) * 2048u + (unsigned)ks * 1024u, AUX_SYS);
            }
        }
        WAIT_VM0();
        __builtin_amdgcn_s_barrier();
        if (wid == 0 && lane < NRANK && (dest < 0 || dest == lane)) {
            if (lane == rank) add_agent((unsigned*)(wsl + WS_CNT) + (pan * NRANK + rank) * 32, 16u);
            else { unsigned c = tk; asm volatile("" : "+v"(c)); st_sys((unsigned*)(P.ws[lane] + WS_FLAG + rank * FLAGB) + pos * 4 + q, c + 1u); }
        }
    }
}

struct Tile { int tm, pan, ord; };
__device__ __forceinline__ bool tile_of(int idx, int rank, Tile& t) {
    if (idx >= NTILE) return false;
    if (idx < NOUT) { int s; stile_of(idx, t.tm, s); t.pan = rank * NS + s; t.ord = idx; }
    else { const int o = idx - NOUT; t.tm = o & (NTM - 1); t.pan = NRANK * NS + (o >> 4); t.ord = -1; }
    return true;
}

__device__ __forceinline__ void gemm_role(const Params& P, unsigned call, int g, LAS unsigned char* lds) {
    const int tid = threadIdx.x, wid = __builtin_amdgcn_readfirstlane(tid >> 6), lane = tid & 63, wr = wid >> 2, wc = wid & 3;
    const int rank = P.rank;
    char* wsl = P.ws[rank];
    const rsrc_t rws = mk_rsrc(wsl, WS_END);
    LAS unsigned* ctl = (LAS unsigned*)(lds + STAGE_BYTES);
    const unsigned voff = (unsigned)tid * 16u;
    const unsigned ldsw = (unsigned)wid * 1024u;
    const int aoff = wr * 8192 + lane * 16, boff = wc * 4096 + lane * 16;

    auto ready = [&](const Tile& t, bool block) -> bool {
        if (wid == 0) {
            const unsigned* c = (const unsigned*)(wsl + WS_CNT) + ((((lane & 8) ? NTM + t.pan : t.tm) * NRANK + (lane & 7)) * 32);
            unsigned* tmo = (unsigned*)(wsl + WS_TMO);
            const unsigned need = call * 64u;
            const unsigned long long t0 = __builtin_amdgcn_s_memrealtime();
            unsigned got = 1u;
            for (;;) {
                const bool ok = (int)(ld_agent(c) - need) >= 0;
                if (__all(ok)) break;
                if (!block) { got = 0u; break; }
                if (give_up(tmo, call, t0)) { ctl[0] = 1u; break; }
                __builtin_amdgcn_s_sleep(4);
            }
            ctl[2] = got;
            __builtin_amdgcn_fence(__ATOMIC_ACQUIRE, "agent");
            asm volatile("s_waitcnt vmcnt(0) lgkmcnt(0)" ::: "memory");
        }
        asm volatile("" ::: "memory"); __builtin_amdgcn_s_barrier(); asm volatile("" ::: "memory");
        const unsigned r = __builtin_amdgcn_readfirstlane(ctl[2]);
        asm volatile("s_waitcnt lgkmcnt(0)" ::: "memory");
        return r != 0u;
    };

#define SA(b, h) (((b) * 2 + (h)) * HTB)
#define SB(b, h) ((4 + (b) * 2 + (h)) * HTB)
#define STAGE(bufoff, soff) do { _Pragma("unroll") for (int _i = 0; _i < 2; ++_i) \
        __builtin_amdgcn_raw_ptr_buffer_load_lds(rws, (LAS unsigned*)(lds + (bufoff) + ldsw + _i * 8192), 16, voff, (soff) + _i * 8192u, 0, 0); } while (0)
#define LDA(dst, b, h) do { _Pragma("unroll") for (int m = 0; m < 4; ++m) _Pragma("unroll") for (int k = 0; k < 2; ++k) dst[m][k] = *(const LAS bf16x8*)(lds + SA(b, h) + aoff + m * 2048 + k * 1024); } while (0)
#define LDB(dst, b, h) do { _Pragma("unroll") for (int n = 0; n < 2; ++n) _Pragma("unroll") for (int k = 0; k < 2; ++k) dst[n][k] = *(const LAS bf16x8*)(lds + SB(b, h) + boff + n * 2048 + k * 1024); } while (0)
#define MMA(ai, bj, At, Bt) do { __builtin_amdgcn_s_setprio(1); _Pragma("unroll") for (int m = 0; m < 4; ++m) _Pragma("unroll") for (int n = 0; n < 2; ++n) _Pragma("unroll") for (int k = 0; k < 2; ++k) \
        acc[ai][bj][m][n] = __builtin_amdgcn_mfma_f32_16x16x32_bf16(Bt[n][k], At[m][k], acc[ai][bj][m][n], 0, 0, 0); __builtin_amdgcn_s_setprio(0); } while (0)
#define WAIT_V(n) asm volatile("s_waitcnt vmcnt(" #n ")" ::: "memory")
#define WAIT_L(n) asm volatile("s_waitcnt lgkmcnt(" #n ")" ::: "memory")
#define BAR __builtin_amdgcn_s_barrier()
#define SCHED __builtin_amdgcn_sched_barrier(0)

    Tile cur, nxt; int ui = g;
    if (!tile_of(ui, rank, cur)) return;
    f32x4 acc[2][2][4][2];
#pragma unroll
    for (int a = 0; a < 2; ++a)
#pragma unroll
        for (int b = 0; b < 2; ++b)
#pragma unroll
            for (int m = 0; m < 4; ++m)
#pragma unroll
                for (int n = 0; n < 2; ++n) acc[a][b][m][n] = (f32x4){0.f, 0.f, 0.f, 0.f};
    bf16x8 At[4][2], B0[2][2], B1[2][2];
    unsigned cA = WS_PAN + (unsigned)(NTM + cur.pan) * PANB, cB = WS_PAN + (unsigned)cur.tm * PANB;
    bool fresh = true;
    for (;;) {
        if (fresh) {
            (void)ready(cur, true);
            if (ui < NG) STAMP(P, 2);
            STAGE(SB(0, 0), cB); STAGE(SA(0, 0), cA); STAGE(SB(0, 1), cB + HTB); STAGE(SA(0, 1), cA + HTB);
            if (wr == 1) BAR;
            WAIT_V(4); BAR;
            STAGE(SB(1, 0), cB + KTB); STAGE(SA(1, 0), cA + KTB); STAGE(SB(1, 1), cB + HTB + KTB);
            WAIT_V(6); BAR;
        }
        const bool has_next = tile_of(ui + NG, rank, nxt);
        const unsigned nA = has_next ? WS_PAN + (unsigned)(NTM + nxt.pan) * PANB : cA, nB = has_next ? WS_PAN + (unsigned)nxt.tm * PANB : cB;
        bool pre = false;
        for (int t = 0; t < KT; t += 2) {
            const bool last = (t == KT - 2);
            if (last && has_next) pre = ready(nxt, false);
            const unsigned a1 = cA + (unsigned)(t + 1) * KTB;
            const unsigned a2 = last ? (pre ? nA : cA) : cA + (unsigned)(t + 2) * KTB, b2 = last ? (pre ? nB : cB) : cB + (unsigned)(t + 2) * KTB;
            const unsigned a3 = a2 + KTB, b3 = b2 + KTB;
            LDB(B0, 0, 0); SCHED; LDA(At, 0, 0); STAGE(SA(1, 1), a1 + HTB);
            WAIT_L(8); BAR; WAIT_L(0); MMA(0, 0, At, B0); BAR; SCHED;
            LDB(B1, 0, 1); STAGE(SB(0, 0), b2);
            BAR; WAIT_L(0); MMA(0, 1, At, B1); BAR;
            LDA(At, 0, 1); STAGE(SA(0, 0), a2);
            BAR; WAIT_L(0); MMA(1, 0, At, B0); BAR; SCHED;
            STAGE(SB(0, 1), b2 + HTB);
            WAIT_V(6); BAR; MMA(1, 1, At, B1); BAR;
            LDB(B0, 1, 0); SCHED; LDA(At, 1, 0); STAGE(SA(0, 1), a2 + HTB);
            WAIT_L(8); BAR; WAIT_L(0); MMA(0, 0, At, B0); BAR; SCHED;
            LDB(B1, 1, 1); STAGE(SB(1, 0), b3);
            BAR; WAIT_L(0); MMA(0, 1, At, B1); BAR;
            LDA(At, 1, 1); STAGE(SA(1, 0), a3);
            BAR; WAIT_L(0); MMA(1, 0, At, B0); BAR; SCHED;
            STAGE(SB(1, 1), b3 + HTB);
            WAIT_V(6); BAR; MMA(1, 1, At, B1); BAR;
        }
        {
            int tz = threadIdx.x; asm volatile("" : "+v"(tz));
            const int ln = tz & 63, fr = ln & 15, fq = ln >> 4;
            const rsrc_t rol = mk_rsrc(P.out[rank], OUTB);
            const bool bad = ctl[0] != 0u;
            const unsigned vo = (unsigned)(wc * 32 + fq * 4) * (NN * 2) + (unsigned)(wr * 256 + fr * 16);
            const unsigned so = (unsigned)(cur.tm * 256) * (NN * 2) + (unsigned)cur.pan * 512u;
#pragma unroll
            for (int bj = 0; bj < 2; ++bj)
#pragma unroll
                for (int n = 0; n < 2; ++n)
#pragma unroll
                    for (int r = 0; r < 4; ++r) {
                        u32x4 o;
                        o[0] = cvt_pk_bf16(silu(acc[0][bj][0][n][r]), silu(acc[0][bj][1][n][r]));
                        o[1] = cvt_pk_bf16(silu(acc[0][bj][2][n][r]), silu(acc[0][bj][3][n][r]));
                        o[2] = cvt_pk_bf16(silu(acc[1][bj][0][n][r]), silu(acc[1][bj][1][n][r]));
                        o[3] = cvt_pk_bf16(silu(acc[1][bj][2][n][r]), silu(acc[1][bj][3][n][r]));
                        if (bad) o = (u32x4){0x7fc07fc0u, 0x7fc07fc0u, 0x7fc07fc0u, 0x7fc07fc0u};
                        __builtin_amdgcn_raw_buffer_store_b128(o, rol, vo, so + (unsigned)(bj * 128 + n * 16 + r) * (NN * 2), AUX_SYS);
                    }
            if (cur.ord >= 0) {
                WAIT_VM0();
                unsigned old = 0;
                if (ln == 0) old = __hip_atomic_fetch_add(ctl + 1, 1u, __ATOMIC_RELAXED, __HIP_MEMORY_SCOPE_WORKGROUP);
                old = __builtin_amdgcn_readfirstlane(old);
                if ((old & 7u) == 7u && ln < NRANK && ln != rank)
                    st_sys((unsigned*)(P.ws[ln] + WS_FLAG + rank * FLAGB) + NIN * 4 + cur.ord, call);
            }
        }
        STAMP(P, 3 + (ui >= NG ? 1 : 0));
        if (!pre) {
            WAIT_V(0);
            if (wr == 0) BAR;
            BAR;
        }
        if (!has_next) break;
        fresh = !pre;
#pragma unroll
        for (int a = 0; a < 2; ++a)
#pragma unroll
            for (int b = 0; b < 2; ++b)
#pragma unroll
                for (int m = 0; m < 4; ++m)
#pragma unroll
                    for (int n = 0; n < 2; ++n) acc[a][b][m][n] = (f32x4){0.f, 0.f, 0.f, 0.f};
        cur = nxt; cA = nA; cB = nB; ui += NG;
    }
}

__global__ __launch_bounds__(512, 2) void fused_gemm_ar(Params P) {
    extern __shared__ __attribute__((aligned(16))) unsigned char shm[];
    LAS unsigned char* lds = (LAS unsigned char*)shm;
    const int wid = __builtin_amdgcn_readfirstlane(threadIdx.x >> 6), lane = threadIdx.x & 63;
    const int b = __builtin_amdgcn_readfirstlane(blockIdx.x);
    unsigned* tkp = (unsigned*)(P.ws[P.rank] + WS_TICKET + b * 128 + wid * 16);
    const unsigned tk = ld_agent(tkp);
    if (b < NPULL) {
        const unsigned call = __builtin_amdgcn_readfirstlane(tk) + 1u;
        st_agent(tkp, call);
        const int pi = b / PW, slot = b % PW;
        const int peer = (P.rank + 1 + pi) & (NRANK - 1);
        pull_role(P, call, peer, slot * 8 + wid, lane);
    } else {
        const int g = b - NPULL;
        ((LAS unsigned*)(lds + STAGE_BYTES))[lane & 3] = 0u;
        STAMP(P, 0);
        pack_role(P, tk, g, wid, lane);
        const unsigned call = __builtin_amdgcn_readfirstlane(tk) + 1u;
        st_agent(tkp, call);
        STAMP(P, 1);
        asm volatile("s_waitcnt lgkmcnt(0)" ::: "memory");
        __builtin_amdgcn_s_barrier();
        gemm_role(P, call, g, lds);
    }
}
}

extern "C" void kernel_launch(int rank, int nranks, ncclComm_t comm,
                              hipStream_t stream,
                              void* const* d_in, const int* in_sizes, int n_in,
                              void* d_out, int out_size,
                              const int* in_dtypes, int out_dtype,
                              void* const* d_out_all,
                              void* const* d_ws_all, size_t ws_bytes) {
    if (nranks != NRANK || (unsigned)rank >= (unsigned)NRANK || ws_bytes < (size_t)WS_END || n_in < 2 ||
        in_sizes[0] != MM * KSH || in_sizes[1] != KSH * NN || out_size != MM * NN) {
        fprintf(stderr, "unexpected problem geometry\n");
        return;
    }
    static bool attr_done[NRANK] = {};
    if (!attr_done[rank]) {
        HIP_CHECK(hipFuncSetAttribute((const void*)fused_gemm_ar, hipFuncAttributeMaxDynamicSharedMemorySize, LDS_BYTES));
        attr_done[rank] = true;
    }
    Params P;
    P.x = d_in[0]; P.w = d_in[1]; P.rank = rank;
    for (int p = 0; p < NRANK; ++p) { P.out[p] = (char*)d_out_all[p]; P.ws[p] = (char*)d_ws_all[p]; }
    hipLaunchKernelGGL(fused_gemm_ar, dim3(GRID), dim3(512), LDS_BYTES, stream, P);
    HIP_CHECK(hipGetLastError());
}
```

```cpp
#include <hip/hip_runtime.h>
#include <rccl/rccl.h>
#include <cstdio>

#define HIP_CHECK(expr) do { hipError_t _e = (expr); if (_e != hipSuccess) { \
    fprintf(stderr, "HIP error %s at %s:%d\n", hipGetErrorString(_e), __FILE__, __LINE__); \
    return; } } while (0)

#define LAS __attribute__((address_space(3)))
typedef short bf16x8 __attribute__((ext_vector_type(8)));
typedef float f32x4 __attribute__((ext_vector_type(4)));
typedef unsigned u32x4 __attribute__((ext_vector_type(4)));
typedef __amdgpu_buffer_rsrc_t rsrc_t;

namespace {
constexpr int NRANK = 8;
constexpr int MM = 4096, KK = 4096, NN = 32768, KSH = KK / NRANK;
constexpr int KT = KK / 64, KTS = KSH / 64, NTM = MM / 256, NPAN = NN / 256;
#define CFG_NS 8
#define CFG_PW 9
constexpr int NS = CFG_NS;
constexpr int NRP = NPAN - NRANK * NS;
constexpr int NIN = NTM + NS + NRP;
constexpr int NOUT = NS * NTM;
constexpr int NFLAG = NIN * 4 + NOUT;
constexpr int NUNIT = NTM * 4 + NPAN * 4;
constexpr int NTILE = (NS + NRP) * NTM;
constexpr int GRID = 256;
constexpr int PW = CFG_PW;
constexpr int NPULL = PW * (NRANK - 1);
constexpr int NW = PW * 8;
constexpr int NG = GRID - NPULL;
#define CFG_PU 4
constexpr int PU = CFG_PU, PSH = PU == 1 ? 0 : PU == 2 ? 1 : PU == 4 ? 2 : 3;
constexpr int NPIN = NIN * 256 / PU;
constexpr int NPC = NPIN + NOUT * 128 / PU;
constexpr unsigned CNT_FULL = 256;

constexpr unsigned KTB = 32768, HTB = 16384;
constexpr unsigned PANB = KT * KTB;
constexpr unsigned CHB = KTS * KTB;
constexpr unsigned WS_TICKET = 0;
constexpr unsigned WS_TMO = WS_TICKET + GRID * 128;
constexpr unsigned WS_FLAG = WS_TMO + 128;
constexpr unsigned FLAGB = 4096;
constexpr unsigned WS_DONE = WS_FLAG + NRANK * FLAGB;
constexpr unsigned WS_HEAD = WS_DONE + NRANK * 512;
constexpr unsigned WS_CNT = WS_HEAD + NRANK * 128;
constexpr unsigned WS_CTRL_END = WS_CNT + (NTM + NPAN) * NRANK * 128;
constexpr unsigned WS_DBG = 512u << 10;
constexpr unsigned WS_PAN = 1u << 20;
constexpr unsigned WS_END = WS_PAN + (NTM + NPAN) * PANB;
constexpr unsigned OUTB = (unsigned)MM * NN * 2;
static_assert(WS_CTRL_END <= WS_DBG, "ctrl region");
static_assert((NFLAG + 64) * 4 <= (int)FLAGB, "flag region");
static_assert(NW <= 128, "done words");
static_assert(PU >= 2 && PU <= 8, "block size");
constexpr unsigned STAGE_BYTES = 8 * HTB;
constexpr unsigned LDS_BYTES = STAGE_BYTES + 64;
constexpr unsigned long long DEADLINE = 50000000ull;
constexpr int AUX_SYS = 17, AUX_SYS_NT = 19, AUX_AGENT = 16;
#define CFG_OST 16
#define CFG_RST 17

struct Params {
    const void* x; const void* w;
    char* out[NRANK]; char* ws[NRANK];
    int rank;
};

__device__ __forceinline__ rsrc_t mk_rsrc(const void* p, unsigned bytes) {
    const unsigned long long v = (unsigned long long)p;
    const unsigned lo = __builtin_amdgcn_readfirstlane((unsigned)v), hi = __builtin_amdgcn_readfirstlane((unsigned)(v >> 32));
    return __builtin_amdgcn_make_buffer_rsrc((void*)(((unsigned long long)hi << 32) | lo), 0, bytes, 0x00020000);
}
__device__ __forceinline__ unsigned ld_sys(const unsigned* p) { return __hip_atomic_load(p, __ATOMIC_RELAXED, __HIP_MEMORY_SCOPE_SYSTEM); }
__device__ __forceinline__ unsigned ld_agent(const unsigned* p) { return __hip_atomic_load(p, __ATOMIC_RELAXED, __HIP_MEMORY_SCOPE_AGENT); }
__device__ __forceinline__ void st_sys(unsigned* p, unsigned v) { __hip_atomic_store(p, v, __ATOMIC_RELAXED, __HIP_MEMORY_SCOPE_SYSTEM); }
__device__ __forceinline__ void st_agent(unsigned* p, unsigned v) { __hip_atomic_store(p, v, __ATOMIC_RELAXED, __HIP_MEMORY_SCOPE_AGENT); }
__device__ __forceinline__ void add_agent(unsigned* p, unsigned v) { (void)__hip_atomic_fetch_add(p, v, __ATOMIC_RELAXED, __HIP_MEMORY_SCOPE_AGENT); }
#define WAIT_VM0() asm volatile("s_waitcnt vmcnt(0)" ::: "memory")
#define WAIT_VMN(n) do { asm volatile("" ::: "memory"); __builtin_amdgcn_s_waitcnt(0x0F70 | (n)); asm volatile("" ::: "memory"); } while (0)

__device__ __forceinline__ bool give_up(unsigned* tmo, unsigned call, unsigned long long t0) {
    if ((unsigned)__builtin_amdgcn_readfirstlane(ld_agent(tmo)) == call) return true;
    if (__builtin_amdgcn_s_memrealtime() - t0 > DEADLINE) { st_agent(tmo, call); return true; }
    return false;
}

__device__ __forceinline__ int panel_of_pos(int pos, int consumer) {
    if (pos < 2 * NS) return (pos & 1) ? NTM + consumer * NS + (pos >> 1) : (pos >> 1);
    if (pos < NS + NTM) return pos - NS;
    return NTM + NRANK * NS + (pos - NS - NTM);
}
__device__ __forceinline__ void stile_of(int ord, int& tm, int& s) {
    if (ord < NS * NS) {
        const int L = (int)__builtin_amdgcn_sqrtf((float)ord + 0.5f), i = ord - L * L;
        if (i < L) { tm = L; s = i; } else { tm = i - L; s = L; }
    } else { const int o = ord - NS * NS; tm = NS + o / NS; s = o % NS; }
    tm = __builtin_amdgcn_readfirstlane(tm); s = __builtin_amdgcn_readfirstlane(s);
}

__device__ __forceinline__ unsigned cvt_pk_bf16(float lo, float hi) { unsigned r; asm("v_cvt_pk_bf16_f32 %0, %1, %2" : "=v"(r) : "v"(lo), "v"(hi)); return r; }
__device__ __forceinline__ float silu(float y) {
    const float c = __builtin_amdgcn_fmed3f(y, -60.0f, 60.0f);
    const float e = __builtin_amdgcn_exp2f(c * -1.44269504088896f);
    return y * __builtin_amdgcn_rcpf(1.0f + e);
}

#define STAMP1(P, slot) do {} while (0)
#define STAMP(P, slot) do {} while (0)

__device__ __forceinline__ void pull_role(const Params& P, unsigned call, int peer, int widx, int lane) {
    const int rank = P.rank;
    char* wsl = P.ws[rank];
    const rsrc_t rwr = mk_rsrc(P.ws[peer], WS_END), rwl = mk_rsrc(wsl, WS_END);
    const rsrc_t ror = mk_rsrc(P.out[peer], OUTB), rol = mk_rsrc(P.out[rank], OUTB);
    const unsigned* flags = (const unsigned*)(wsl + WS_FLAG + peer * FLAGB);
    unsigned* tmo = (unsigned*)(wsl + WS_TMO);
    constexpr unsigned OOB = 0xfffffff0u;
    const unsigned lane0 = lane == 0 ? 0u : OOB;
    const unsigned hbase = (call - 1u) * (unsigned)(NPC + NW);
    auto draw = [&]() { return (unsigned)__builtin_amdgcn_raw_ptr_buffer_atomic_add_i32(1, rwl, lane0, WS_HEAD + (unsigned)peer * 128u, 0); };
    auto arrive = [&](int pan, unsigned n) { (void)__builtin_amdgcn_raw_ptr_buffer_atomic_add_i32((int)n, rwl, lane0, WS_CNT + (unsigned)(pan * NRANK + peer) * 128u, 0); };
    unsigned long long mask = 0; int base = -1000; bool bad = false; int stamped = 0; (void)stamped;
    auto known = [&](int f) { const unsigned d = (unsigned)(f - base); return d < 64u && ((mask >> d) & 1ull); };
    auto poll = [&](int f) {
        base = f;
        const unsigned long long t0 = __builtin_amdgcn_s_memrealtime();
        for (;;) {
            const unsigned v = ld_sys(flags + f + lane);
            mask = __ballot((int)(v - call) >= 0);
            if (mask & 1ull) break;
            if (give_up(tmo, call, t0)) { bad = true; break; }
            __builtin_amdgcn_s_sleep(2);
        }
        asm volatile("" ::: "memory");
    };
    STAMP(P, 0);
    auto index_of = [&](unsigned v) { return (int)((unsigned)__builtin_amdgcn_readfirstlane(v) - hbase); };
    int cur = index_of(draw());
    {
        const unsigned vo = (unsigned)lane * 16u;
        auto off_of = [&](int i, int& pan) { i <<= PSH; pan = panel_of_pos(i >> 8, rank); return WS_PAN + (unsigned)pan * PANB + (unsigned)peer * CHB + (unsigned)(i & 255) * 1024u; };
        auto flag_of = [&](int i) { return i >> (6 - PSH); };
        while ((unsigned)cur < (unsigned)NPIN) {
            if (!bad && !known(flag_of(cur))) poll(flag_of(cur));
            if (bad) { cur = index_of(draw()); continue; }
            STAMP1(P, 1);
            int pprev = -1; unsigned n = 0;
            unsigned oa = 0, vs = OOB;
            u32x4 va = {0u, 0u, 0u, 0u};
            do {
                int pb; const unsigned ob = off_of(cur, pb);
                unsigned nv = 0;
#pragma unroll
                for (int k = 0; k < PU; ++k) {
                    const u32x4 vb = __builtin_amdgcn_raw_buffer_load_b128(rwr, vo, ob + k * 1024u, AUX_SYS_NT);
                    if (k == 0) nv = draw();
                    __builtin_amdgcn_raw_buffer_store_b128(va, rwl, k == 0 ? vs : vo, k == 0 ? oa : ob + (k - 1) * 1024u, AUX_AGENT);
                    if (k == 0) WAIT_VMN(2); else WAIT_VMN(1);
                    if (k == 1 && pprev != pb) { if (pprev >= 0) arrive(pprev, n); n = 0; pprev = pb; }
                    va = vb;
                }
                n += PU; oa = ob + (PU - 1) * 1024u; vs = vo;
                cur = index_of(nv);
            } while ((unsigned)cur < (unsigned)NPIN && known(flag_of(cur)));
            __builtin_amdgcn_raw_buffer_store_b128(va, rwl, vo, oa, AUX_AGENT);
            WAIT_VM0();
            arrive(pprev, n);
        }
    }
    STAMP(P, 2);
    {
        const unsigned vo = (unsigned)(lane >> 5) * (NN * 2) + (unsigned)(lane & 31) * 16u;
        auto off_of = [&](int i) { const int t = (i - NPIN) << PSH; int tm, s; stile_of(t >> 7, tm, s);
                                   return (unsigned)(tm * 256 + (t & 127) * 2) * (NN * 2) + (unsigned)(peer * NS + s) * 512u; };
        auto flag_of = [&](int i) { return NIN * 4 + ((i - NPIN) >> (7 - PSH)); };
        while ((unsigned)cur < (unsigned)NPC) {
            if (!bad && !known(flag_of(cur))) poll(flag_of(cur));
            if (bad) {
                const unsigned ob = off_of(cur);
#pragma unroll
                for (int k = 0; k < PU; ++k) __builtin_amdgcn_raw_buffer_store_b128((u32x4){0x7fc07fc0u, 0x7fc07fc0u, 0x7fc07fc0u, 0x7fc07fc0u}, rol, vo, ob + k * (NN * 4u), CFG_OST);
                cur = index_of(draw()); continue;
            }
            STAMP1(P, 3);
            unsigned oa = 0, vs = OOB;
            u32x4 va = {0u, 0u, 0u, 0u};
            do {
                const unsigned ob = off_of(cur);
                unsigned nv = 0;
#pragma unroll
                for (int k = 0; k < PU; ++k) {
                    const u32x4 vb = __builtin_amdgcn_raw_buffer_load_b128(ror, vo, ob + k * (NN * 4u), AUX_SYS_NT);
                    if (k == 0) nv = draw();
                    __builtin_amdgcn_raw_buffer_store_b128(va, rol, k == 0 ? vs : vo, k == 0 ? oa : ob + (k - 1) * (NN * 4u), CFG_OST);
                    if (k == 0) WAIT_VMN(2); else WAIT_VMN(1);
                    va = vb;
                }
                oa = ob + (PU - 1) * (NN * 4u); vs = vo;
                cur = index_of(nv);
            } while ((unsigned)cur < (unsigned)NPC && known(flag_of(cur)));
            __builtin_amdgcn_raw_buffer_store_b128(va, rol, vo, oa, CFG_OST);
        }
    }
    WAIT_VM0();
    STAMP(P, 4);
    if (lane == 0) st_sys((unsigned*)(P.ws[peer] + WS_DONE + rank * 512) + widx, call);
    if (widx == 0) {
        const unsigned* dn = (const unsigned*)(wsl + WS_DONE + peer * 512);
        const unsigned long long t0 = __builtin_amdgcn_s_memrealtime();
        for (;;) {
            bool ok = true;
            if (lane < NW) ok = (int)(ld_sys(dn + lane) - call) >= 0;
            if (NW > 64 && lane + 64 < NW) ok = ok && (int)(ld_sys(dn + 64 + lane) - call) >= 0;
            if (__all(ok)) break;
            if (give_up(tmo, call, t0)) {
                if (lane == 0) __builtin_amdgcn_raw_buffer_store_b32(0x7fc07fc0u, rol, (unsigned)(rank * NS) * 512u, 0, AUX_SYS);
                break;
            }
            __builtin_amdgcn_s_sleep(2);
        }
        STAMP(P, 5);
    }
}

__device__ __forceinline__ void pack_role(const Params& P, unsigned tk, int g, int wid, int lane) {
    const int rank = P.rank;
    char* wsl = P.ws[rank];
    const rsrc_t rwl = mk_rsrc(wsl, WS_END);
    const rsrc_t rx = mk_rsrc(P.x, (unsigned)MM * KSH * 2), rw = mk_rsrc(P.w, (unsigned)KSH * NN * 2);
    const int fr = lane & 15, fq = lane >> 4;
    for (int u = g; u < NUNIT; u += NG) {
        int pos, q, dest;
        if (u < 36 * NS) {
            const int s = u / 36, r = u % 36;
            if (r < 4) { pos = 2 * s; q = r; dest = -1; }
            else { pos = 2 * s + 1; q = (r - 4) & 3; dest = (rank + 1 + ((r - 4) >> 2)) & (NRANK - 1); }
        } else { const int v = u - 36 * NS; pos = 2 * NS + (v >> 2); q = v & 3; dest = -1; }
        const int pan = panel_of_pos(pos, dest < 0 ? 0 : dest);
        const int ktl = q * 2 + (wid >> 2);
        const unsigned dst = WS_PAN + (unsigned)pan * PANB + (unsigned)(rank * KTS + ktl) * KTB;
        u32x4 v[8];
        if (pan < NTM) {
            const int rb0 = (wid & 3) * 4;
            const unsigned vo = (unsigned)fr * (KSH * 2) + (unsigned)fq * 16u;
            const unsigned so = (unsigned)(pan * 256 + rb0 * 16) * (KSH * 2) + (unsigned)ktl * 128u;
#pragma unroll
            for (int i = 0; i < 8; ++i) v[i] = __builtin_amdgcn_raw_buffer_load_b128(rx, vo + (i & 1) * 64, so + (unsigned)(i >> 1) * (16 * KSH * 2), 0);
#pragma unroll
            for (int i = 0; i < 8; ++i) __builtin_amdgcn_raw_buffer_store_b128(v[i], rwl, lane * 16 + (i & 1) * 1024, dst + (unsigned)(rb0 + (i >> 1)) * 2048u, AUX_SYS);
        } else {
            const int ks = (wid >> 1) & 1, wr = wid & 1;
            const unsigned vo = (unsigned)(fq * 8) * (NN * 2) + (unsigned)fr * 16u;
            const unsigned so = (unsigned)(ktl * 64 + ks * 32) * (NN * 2) + (unsigned)((pan - NTM) * 256 + wr * 128) * 2u;
#pragma unroll
            for (int j = 0; j < 8; ++j) v[j] = __builtin_amdgcn_raw_buffer_load_b128(rw, vo, so + (unsigned)j * (NN * 2), 2);
#pragma unroll
            for (int i = 0; i < 8; ++i) {
                u32x4 o;
#pragma unroll
                for (int e = 0; e < 4; ++e) o[e] = __builtin_amdgcn_perm(v[2 * e + 1][i >> 1], v[2 * e][i >> 1], (i & 1) ? 0x07060302u : 0x05040100u);
                __builtin_amdgcn_raw_buffer_store_b128(o, rwl, lane * 16, dst + (unsigned)(i >> 2) * HTB + (unsigned)wr * 8192u + (unsigned)(i & 3) * 2048u + (unsigned)ks * 1024u, AUX_SYS);
            }
        }
        WAIT_VM0();
        __builtin_amdgcn_s_barrier();
        if (wid == 0 && lane < NRANK && (dest < 0 || dest == lane)) {
            if (lane == rank) add_agent((unsigned*)(wsl + WS_CNT) + (pan * NRANK + rank) * 32, CNT_FULL / 4);
            else { unsigned c = tk; asm volatile("" : "+v"(c)); st_sys((unsigned*)(P.ws[lane] + WS_FLAG + rank * FLAGB) + pos * 4 + q, c + 1u); }
        }
    }
}

struct Tile { int tm, pan, ord; };
__device__ __forceinline__ bool tile_of(int idx, int rank, Tile& t) {
    if (idx >= NTILE) return false;
    if (idx < NOUT) { int s; stile_of(idx, t.tm, s); t.pan = rank * NS + s; t.ord = idx; }
    else { const int o = idx - NOUT; t.tm = o & (NTM - 1); t.pan = NRANK * NS + (o >> 4); t.ord = -1; }
    return true;
}

__device__ __forceinline__ void gemm_role(const Params& P, unsigned call, int g, LAS unsigned char* lds) {
    const int tid = threadIdx.x, wid = __builtin_amdgcn_readfirstlane(tid >> 6), lane = tid & 63, wr = wid >> 2, wc = wid & 3;
    const int rank = P.rank;
    char* wsl = P.ws[rank];
    const rsrc_t rws = mk_rsrc(wsl, WS_END);
    LAS unsigned* ctl = (LAS unsigned*)(lds + STAGE_BYTES);
    const unsigned voff = (unsigned)tid * 16u;
    const unsigned ldsw = (unsigned)wid * 1024u;
    const int aoff = wr * 8192 + lane * 16, boff = wc * 4096 + lane * 16;

    auto ready = [&](const Tile& t, bool block) -> bool {
        if (wid == 0) {
            const unsigned* c = (const unsigned*)(wsl + WS_CNT) + ((((lane & 8) ? NTM + t.pan : t.tm) * NRANK + (lane & 7)) * 32);
            unsigned* tmo = (unsigned*)(wsl + WS_TMO);
            const unsigned need = call * CNT_FULL;
            const unsigned long long t0 = __builtin_amdgcn_s_memrealtime();
            unsigned got = 1u;
            for (;;) {
                const bool ok = (int)(ld_agent(c) - need) >= 0;
                if (__all(ok)) break;
                if (!block) { got = 0u; break; }
                if (give_up(tmo, call, t0)) { ctl[0] = 1u; break; }
                __builtin_amdgcn_s_sleep(4);
            }
            ctl[2] = got;
            __builtin_amdgcn_fence(__ATOMIC_ACQUIRE, "agent");
            asm volatile("s_waitcnt vmcnt(0) lgkmcnt(0)" ::: "memory");
        }
        asm volatile("" ::: "memory"); __builtin_amdgcn_s_barrier(); asm volatile("" ::: "memory");
        const unsigned r = __builtin_amdgcn_readfirstlane(ctl[2]);
        asm volatile("s_waitcnt lgkmcnt(0)" ::: "memory");
        return r != 0u;
    };

#define SA(b, h) (((b) * 2 + (h)) * HTB)
#define SB(b, h) ((4 + (b) * 2 + (h)) * HTB)
#define STAGE(bufoff, soff) do { _Pragma("unroll") for (int _i = 0; _i < 2; ++_i) \
        __builtin_amdgcn_raw_ptr_buffer_load_lds(rws, (LAS unsigned*)(lds + (bufoff) + ldsw + _i * 8192), 16, voff, (soff) + _i * 8192u, 0, 0); } while (0)
#define LDA(dst, b, h) do { _Pragma("unroll") for (int m = 0; m < 4; ++m) _Pragma("unroll") for (int k = 0; k < 2; ++k) dst[m][k] = *(const LAS bf16x8*)(lds + SA(b, h) + aoff + m * 2048 + k * 1024); } while (0)
#define LDB(dst, b, h) do { _Pragma("unroll") for (int n = 0; n < 2; ++n) _Pragma("unroll") for (int k = 0; k < 2; ++k) dst[n][k] = *(const LAS bf16x8*)(lds + SB(b, h) + boff + n * 2048 + k * 1024); } while (0)
#define MMA(ai, bj, At, Bt) do { __builtin_amdgcn_s_setprio(1); _Pragma("unroll") for (int m = 0; m < 4; ++m) _Pragma("unroll") for (int n = 0; n < 2; ++n) _Pragma("unroll") for (int k = 0; k < 2; ++k) \
        acc[ai][bj][m][n] = __builtin_amdgcn_mfma_f32_16x16x32_bf16(Bt[n][k], At[m][k], acc[ai][bj][m][n], 0, 0, 0); __builtin_amdgcn_s_setprio(0); } while (0)
#define WAIT_V(n) asm volatile("s_waitcnt vmcnt(" #n ")" ::: "memory")
#define WAIT_L(n) asm volatile("s_waitcnt lgkmcnt(" #n ")" ::: "memory")
#define BAR __builtin_amdgcn_s_barrier()
#define SCHED __builtin_amdgcn_sched_barrier(0)

    Tile cur, nxt; int ui = g;
    if (!tile_of(ui, rank, cur)) return;
    f32x4 acc[2][2][4][2];
#pragma unroll
    for (int a = 0; a < 2; ++a)
#pragma unroll
        for (int b = 0; b < 2; ++b)
#pragma unroll
            for (int m = 0; m < 4; ++m)
#pragma unroll
                for (int n = 0; n < 2; ++n) acc[a][b][m][n] = (f32x4){0.f, 0.f, 0.f, 0.f};
    bf16x8 At[4][2], B0[2][2], B1[2][2];
    unsigned cA = WS_PAN + (unsigned)(NTM + cur.pan) * PANB, cB = WS_PAN + (unsigned)cur.tm * PANB;
    bool fresh = true;
    for (;;) {
        if (fresh) {
            (void)ready(cur, true);
            if (ui < NG) STAMP(P, 2);
            STAGE(SB(0, 0), cB); STAGE(SA(0, 0), cA); STAGE(SB(0, 1), cB + HTB); STAGE(SA(0, 1), cA + HTB);
            if (wr == 1) BAR;
            WAIT_V(4); BAR;
            STAGE(SB(1, 0), cB + KTB); STAGE(SA(1, 0), cA + KTB); STAGE(SB(1, 1), cB + HTB + KTB);
            WAIT_V(6); BAR;
        }
        const bool has_next = tile_of(ui + NG, rank, nxt);
        const unsigned nA = has_next ? WS_PAN + (unsigned)(NTM + nxt.pan) * PANB : cA, nB = has_next ? WS_PAN + (unsigned)nxt.tm * PANB : cB;
        bool pre = false;
        for (int t = 0; t < KT; t += 2) {
            const bool last = (t == KT - 2);
            if (last && has_next) pre = ready(nxt, false);
            const unsigned a1 = cA + (unsigned)(t + 1) * KTB;
            const unsigned a2 = last ? (pre ? nA : cA) : cA + (unsigned)(t + 2) * KTB, b2 = last ? (pre ? nB : cB) : cB + (unsigned)(t + 2) * KTB;
            const unsigned a3 = a2 + KTB, b3 = b2 + KTB;
            LDB(B0, 0, 0); SCHED; LDA(At, 0, 0); STAGE(SA(1, 1), a1 + HTB);
            WAIT_L(8); BAR; WAIT_L(0); MMA(0, 0, At, B0); BAR; SCHED;
            LDB(B1, 0, 1); STAGE(SB(0, 0), b2);
            BAR; WAIT_L(0); MMA(0, 1, At, B1); BAR;
            LDA(At, 0, 1); STAGE(SA(0, 0), a2);
            BAR; WAIT_L(0); MMA(1, 0, At, B0); BAR; SCHED;
            STAGE(SB(0, 1), b2 + HTB);
            WAIT_V(6); BAR; MMA(1, 1, At, B1); BAR;
            LDB(B0, 1, 0); SCHED; LDA(At, 1, 0); STAGE(SA(0, 1), a2 + HTB);
            WAIT_L(8); BAR; WAIT_L(0); MMA(0, 0, At, B0); BAR; SCHED;
            LDB(B1, 1, 1); STAGE(SB(1, 0), b3);
            BAR; WAIT_L(0); MMA(0, 1, At, B1); BAR;
            LDA(At, 1, 1); STAGE(SA(1, 0), a3);
            BAR; WAIT_L(0); MMA(1, 0, At, B0); BAR; SCHED;
            STAGE(SB(1, 1), b3 + HTB);
            WAIT_V(6); BAR; MMA(1, 1, At, B1); BAR;
        }
        {
            int tz = threadIdx.x; asm volatile("" : "+v"(tz));
            const int ln = tz & 63, fr = ln & 15, fq = ln >> 4;
            const rsrc_t rol = mk_rsrc(P.out[rank], OUTB);
            const bool bad = ctl[0] != 0u;
            const unsigned vo = (unsigned)(wc * 32 + fq * 4) * (NN * 2) + (unsigned)(wr * 256 + fr * 16);
            const unsigned so = (unsigned)(cur.tm * 256) * (NN * 2) + (unsigned)cur.pan * 512u;
#pragma unroll
            for (int bj = 0; bj < 2; ++bj)
#pragma unroll
                for (int n = 0; n < 2; ++n)
#pragma unroll
                    for (int r = 0; r < 4; ++r) {
                        u32x4 o;
                        o[0] = cvt_pk_bf16(silu(acc[0][bj][0][n][r]), silu(acc[0][bj][1][n][r]));
                        o[1] = cvt_pk_bf16(silu(acc[0][bj][2][n][r]), silu(acc[0][bj][3][n][r]));
                        o[2] = cvt_pk_bf16(silu(acc[1][bj][0][n][r]), silu(acc[1][bj][1][n][r]));
                        o[3] = cvt_pk_bf16(silu(acc[1][bj][2][n][r]), silu(acc[1][bj][3][n][r]));
                        if (bad) o = (u32x4){0x7fc07fc0u, 0x7fc07fc0u, 0x7fc07fc0u, 0x7fc07fc0u};
                        if (CFG_RST == AUX_SYS || cur.ord >= 0) __builtin_amdgcn_raw_buffer_store_b128(o, rol, vo, so + (unsigned)(bj * 128 + n * 16 + r) * (NN * 2), AUX_SYS);
                        else __builtin_amdgcn_raw_buffer_store_b128(o, rol, vo, so + (unsigned)(bj * 128 + n * 16 + r) * (NN * 2), CFG_RST);
                    }
            if (cur.ord >= 0) {
                WAIT_VM0();
                unsigned old = 0;
                if (ln == 0) old = __hip_atomic_fetch_add(ctl + 1, 1u, __ATOMIC_RELAXED, __HIP_MEMORY_SCOPE_WORKGROUP);
                old = __builtin_amdgcn_readfirstlane(old);
                if ((old & 7u) == 7u && ln < NRANK && ln != rank)
                    st_sys((unsigned*)(P.ws[ln] + WS_FLAG + rank * FLAGB) + NIN * 4 + cur.ord, call);
            }
        }
        STAMP(P, 3 + (ui >= NG ? 1 : 0));
        if (!pre) {
            WAIT_V(0);
            if (wr == 0) BAR;
            BAR;
        }
        if (!has_next) break;
        fresh = !pre;
#pragma unroll
        for (int a = 0; a < 2; ++a)
#pragma unroll
            for (int b = 0; b < 2; ++b)
#pragma unroll
                for (int m = 0; m < 4; ++m)
#pragma unroll
                    for (int n = 0; n < 2; ++n) acc[a][b][m][n] = (f32x4){0.f, 0.f, 0.f, 0.f};
        cur = nxt; cA = nA; cB = nB; ui += NG;
    }
}

__global__ __launch_bounds__(512, 2) void fused_gemm_ar(Params P) {
    extern __shared__ __attribute__((aligned(16))) unsigned char shm[];
    LAS unsigned char* lds = (LAS unsigned char*)shm;
    const int wid = __builtin_amdgcn_readfirstlane(threadIdx.x >> 6), lane = threadIdx.x & 63;
    const int b = __builtin_amdgcn_readfirstlane(blockIdx.x);
    unsigned* tkp = (unsigned*)(P.ws[P.rank] + WS_TICKET + b * 128 + wid * 16);
    const unsigned tk = ld_agent(tkp);
    if (b < NPULL) {
        const unsigned call = __builtin_amdgcn_readfirstlane(tk) + 1u;
        st_agent(tkp, call);
        const int pi = b / PW, slot = b % PW;
        const int peer = (P.rank + 1 + pi) & (NRANK - 1);
        pull_role(P, call, peer, slot * 8 + wid, lane);
    } else {
        const int g = b - NPULL;
        ((LAS unsigned*)(lds + STAGE_BYTES))[lane & 3] = 0u;
        STAMP(P, 0);
        pack_role(P, tk, g, wid, lane);
        const unsigned call = __builtin_amdgcn_readfirstlane(tk) + 1u;
        st_agent(tkp, call);
        STAMP(P, 1);
        asm volatile("s_waitcnt lgkmcnt(0)" ::: "memory");
        __builtin_amdgcn_s_barrier();
        gemm_role(P, call, g, lds);
    }
}
}

extern "C" void kernel_launch(int rank, int nranks, ncclComm_t comm,
                              hipStream_t stream,
                              void* const* d_in, const int* in_sizes, int n_in,
                              void* d_out, int out_size,
                              const int* in_dtypes, int out_dtype,
                              void* const* d_out_all,
                              void* const* d_ws_all, size_t ws_bytes) {
    if (nranks != NRANK || (unsigned)rank >= (unsigned)NRANK || ws_bytes < (size_t)WS_END || n_in < 2 ||
        in_sizes[0] != MM * KSH || in_sizes[1] != KSH * NN || out_size != MM * NN) {
        fprintf(stderr, "unexpected problem geometry\n");
        return;
    }
    static bool attr_done[NRANK] = {};
    if (!attr_done[rank]) {
        HIP_CHECK(hipFuncSetAttribute((const void*)fused_gemm_ar, hipFuncAttributeMaxDynamicSharedMemorySize, LDS_BYTES));
        attr_done[rank] = true;
    }
    Params P;
    P.x = d_in[0]; P.w = d_in[1]; P.rank = rank;
    for (int p = 0; p < NRANK; ++p) { P.out[p] = (char*)d_out_all[p]; P.ws[p] = (char*)d_ws_all[p]; }
    hipLaunchKernelGGL(fused_gemm_ar, dim3(GRID), dim3(512), LDS_BYTES, stream, P);
    HIP_CHECK(hipGetLastError());
}
```

```cpp
#include <hip/hip_runtime.h>
#include <rccl/rccl.h>
#include <cstdio>

#define HIP_CHECK(expr) do { hipError_t _e = (expr); if (_e != hipSuccess) { \
    fprintf(stderr, "HIP error %s at %s:%d\n", hipGetErrorString(_e), __FILE__, __LINE__); \
    return; } } while (0)

#define LAS __attribute__((address_space(3)))
typedef short bf16x8 __attribute__((ext_vector_type(8)));
typedef float f32x4 __attribute__((ext_vector_type(4)));
typedef unsigned u32x4 __attribute__((ext_vector_type(4)));
typedef __amdgpu_buffer_rsrc_t rsrc_t;

namespace {
constexpr int NRANK = 8;
constexpr int MM = 4096, KK = 4096, NN = 32768, KSH = KK / NRANK;
constexpr int KT = KK / 64, KTS = KSH / 64, NTM = MM / 256, NPAN = NN / 256;
#define CFG_NS 7
#define CFG_PW 8
constexpr int NS = CFG_NS;
constexpr int NRP = NPAN - NRANK * NS;
constexpr int NWP = NS + NRP;
constexpr int NIN = NTM + NWP;
constexpr int NOUT = NS * NTM;
constexpr int NFLAG = NIN * 4 + NOUT;
constexpr int NUNIT = NTM * 4 + NPAN * 4;
constexpr int GRID = 256;
constexpr int PW = CFG_PW;
constexpr int NPULL = PW * (NRANK - 1);
constexpr int NW = PW * 8;
constexpr int NPK = GRID - NPULL;
constexpr int NPS = (NPULL + 7) / 8 * 8;
constexpr int KX = (GRID - NPS) / 8;
constexpr int GW = 2 * ((KX + 3) / 4);
constexpr int G0 = GW + NTM;
constexpr int NGF = (NWP - GW) / GW, GL = (NWP - GW) % GW;
static_assert(NS <= GW && NWP > GW, "shared panels sit in the first group");
#define CFG_PU 4
constexpr int PU = CFG_PU, PSH = PU == 1 ? 0 : PU == 2 ? 1 : PU == 4 ? 2 : 3;
constexpr int NPIN = NIN * 256 / PU;
constexpr int NPC = NPIN + NOUT * 128 / PU;
constexpr unsigned CNT_FULL = 64;

constexpr unsigned KTB = 32768, HTB = 16384;
constexpr unsigned PANB = KT * KTB;
constexpr unsigned CHB = KTS * KTB;
constexpr unsigned UNB = 2 * KTB;
constexpr unsigned WS_TICKET = 0;
constexpr unsigned WS_TMO = WS_TICKET + GRID * 128;
constexpr unsigned WS_FLAG = WS_TMO + 128;
constexpr unsigned FLAGB = 4096;
constexpr unsigned WS_DONE = WS_FLAG + NRANK * FLAGB;
constexpr unsigned WS_HEAD = WS_DONE + NRANK * 512;
constexpr unsigned WS_CNT = WS_HEAD + NRANK * 128;
constexpr unsigned WS_CTRL_END = WS_CNT + (NTM + NPAN) * NRANK * 128;
constexpr unsigned WS_DBG = 512u << 10;
constexpr unsigned WS_PAN = 1u << 20;
constexpr unsigned WS_END = WS_PAN + (NTM + NPAN) * PANB;
constexpr unsigned OUTB = (unsigned)MM * NN * 2;
static_assert(WS_CTRL_END <= WS_DBG, "ctrl region");
static_assert((NFLAG + 64) * 4 <= (int)FLAGB, "flag region");
static_assert(NW <= 128, "done words");
static_assert(PU == 4, "block size");
#define CFG_DEEP 2
constexpr unsigned STAGE_BYTES = 8 * HTB;
constexpr unsigned LDS_BYTES = STAGE_BYTES + 64;
constexpr unsigned long long DEADLINE = 50000000ull;
constexpr int AUX_SYS = 17, AUX_SYS_NT = 19, AUX_AGENT = 16;
#define CFG_OST 16
#define CFG_RST 0

struct Params {
    const void* x; const void* w;
    char* out[NRANK]; char* ws[NRANK];
    int rank;
};

__device__ __forceinline__ rsrc_t mk_rsrc(const void* p, unsigned bytes) {
    const unsigned long long v = (unsigned long long)p;
    const unsigned lo = __builtin_amdgcn_readfirstlane((unsigned)v), hi = __builtin_amdgcn_readfirstlane((unsigned)(v >> 32));
    return __builtin_amdgcn_make_buffer_rsrc((void*)(((unsigned long long)hi << 32) | lo), 0, bytes, 0x00020000);
}
__device__ __forceinline__ unsigned ld_sys(const unsigned* p) { return __hip_atomic_load(p, __ATOMIC_RELAXED, __HIP_MEMORY_SCOPE_SYSTEM); }
__device__ __forceinline__ unsigned ld_agent(const unsigned* p) { return __hip_atomic_load(p, __ATOMIC_RELAXED, __HIP_MEMORY_SCOPE_AGENT); }
__device__ __forceinline__ void st_sys(unsigned* p, unsigned v) { __hip_atomic_store(p, v, __ATOMIC_RELAXED, __HIP_MEMORY_SCOPE_SYSTEM); }
__device__ __forceinline__ void st_agent(unsigned* p, unsigned v) { __hip_atomic_store(p, v, __ATOMIC_RELAXED, __HIP_MEMORY_SCOPE_AGENT); }
__device__ __forceinline__ void add_agent(unsigned* p, unsigned v) { (void)__hip_atomic_fetch_add(p, v, __ATOMIC_RELAXED, __HIP_MEMORY_SCOPE_AGENT); }
#define WAIT_VM0() asm volatile("s_waitcnt vmcnt(0)" ::: "memory")
#define WAIT_VMN(n) do { asm volatile("" ::: "memory"); __builtin_amdgcn_s_waitcnt(0x0F70 | (n)); asm volatile("" ::: "memory"); } while (0)

__device__ __forceinline__ bool give_up(unsigned* tmo, unsigned call, unsigned long long t0) {
    if ((unsigned)__builtin_amdgcn_readfirstlane(ld_agent(tmo)) == call) return true;
    if (__builtin_amdgcn_s_memrealtime() - t0 > DEADLINE) { st_agent(tmo, call); return true; }
    return false;
}

__device__ __forceinline__ int gpan_of(int v, int consumer) { return v < NS ? consumer * NS + v : (NRANK - 1) * NS + v; }
__device__ __forceinline__ int unit_of(int o, int consumer, int& q) {
    if (o < 4 * G0) { q = o / G0; const int m = o % G0; return m < GW ? NTM + gpan_of(m, consumer) : m - GW; }
    o -= 4 * G0;
    const int g = o / (4 * GW), r = o % (4 * GW);
    int i;
    if (GL == 0 || g < NGF) { q = r / GW; i = r % GW; } else { q = r / (GL ? GL : 1); i = r % (GL ? GL : 1); }
    return NTM + (NRANK - 1) * NS + GW * (g + 1) + i;
}

__device__ __forceinline__ unsigned cvt_pk_bf16(float lo, float hi) { unsigned r; asm("v_cvt_pk_bf16_f32 %0, %1, %2" : "=v"(r) : "v"(lo), "v"(hi)); return r; }
__device__ __forceinline__ float silu(float y) {
    const float c = __builtin_amdgcn_fmed3f(y, -60.0f, 60.0f);
    const float e = __builtin_amdgcn_exp2f(c * -1.44269504088896f);
    return y * __builtin_amdgcn_rcpf(1.0f + e);
}

#define STAMP1(P, slot) do {} while (0)
#define STAMP(P, slot) do {} while (0)

__device__ __forceinline__ void pull_role(const Params& P, unsigned call, int peer, int widx, int lane) {
    const int rank = P.rank;
    char* wsl = P.ws[rank];
    const rsrc_t rwr = mk_rsrc(P.ws[peer], WS_END), rwl = mk_rsrc(wsl, WS_END);
    const rsrc_t ror = mk_rsrc(P.out[peer], OUTB), rol = mk_rsrc(P.out[rank], OUTB);
    const unsigned* flags = (const unsigned*)(wsl + WS_FLAG + peer * FLAGB);
    unsigned* tmo = (unsigned*)(wsl + WS_TMO);
    constexpr unsigned OOB = 0xfffffff0u;
    const unsigned lane0 = lane == 0 ? 0u : OOB;
    const unsigned hbase = (call - 1u) * (unsigned)(NPC + NW);
    auto draw = [&]() { return (unsigned)__builtin_amdgcn_raw_ptr_buffer_atomic_add_i32(1, rwl, lane0, WS_HEAD + (unsigned)peer * 128u, 0); };
    auto arrive = [&](unsigned vo, unsigned cnt) { (void)__builtin_amdgcn_raw_ptr_buffer_atomic_add_i32(PU, rwl, vo, cnt, 0); };
    const bool deep = (widx & 7) < CFG_DEEP;
    unsigned long long mask = 0; int base = -1000; bool bad = false; int stamped = 0; (void)stamped;
    auto known = [&](int f, unsigned bits) { const unsigned d = (unsigned)(f - base); return d < 64u && ((unsigned)(mask >> d) & bits) == bits; };
    auto poll = [&](int f, unsigned bits) {
        base = f;
        const unsigned long long t0 = __builtin_amdgcn_s_memrealtime();
        for (;;) {
            const unsigned v = ld_sys(flags + f + lane);
            mask = __ballot((int)(v - call) >= 0);
            if (((unsigned)mask & bits) == bits) break;
            if (give_up(tmo, call, t0)) { bad = true; break; }
            __builtin_amdgcn_s_sleep(2);
        }
        asm volatile("" ::: "memory");
    };
    STAMP(P, 0);
    auto index_of = [&](unsigned v) { return (int)((unsigned)__builtin_amdgcn_readfirstlane(v) - hbase); };
    int cur = index_of(draw());
    {
        const unsigned vo = (unsigned)lane * 16u;
        auto flag_of = [&](int i) { return i >> (6 - PSH); };
        auto off_of = [&](int i, unsigned& cnt) { int q; const int pan = unit_of(flag_of(i), rank, q);
                                                  cnt = WS_CNT + (unsigned)(pan * NRANK + peer) * 128u + (unsigned)q * 4u;
                                                  return WS_PAN + (unsigned)pan * PANB + (unsigned)peer * CHB + (unsigned)q * UNB + (unsigned)((i << PSH) & 63) * 1024u; };
        while ((unsigned)cur < (unsigned)NPIN) {
            if (!bad && !known(flag_of(cur), 1u)) poll(flag_of(cur), 1u);
            if (bad) { cur = index_of(draw()); continue; }
            STAMP1(P, 1);
            unsigned cprev = 0, oprev = 0;
            unsigned vs = OOB, va = OOB;
            u32x4 v[4] = {};
            do {
                unsigned cb; const unsigned ob = off_of(cur, cb);
                unsigned nv = 0;
#pragma unroll
                for (int k = 0; k < 4; ++k) {
                    v[k] = __builtin_amdgcn_raw_buffer_load_b128(rwr, vo, ob + k * 1024u, AUX_SYS_NT);
                    if (k == 0) nv = draw();
                    __builtin_amdgcn_raw_buffer_store_b128(v[k ^ 2], rwl, k < 2 ? vs : vo, (k < 2 ? oprev : ob) + (k ^ 2) * 1024u, AUX_AGENT);
                    if (k == 0) WAIT_VMN(5); else if (k == 1) WAIT_VMN(4); else WAIT_VMN(3);
                    if (!deep) { if (k == 0) WAIT_VMN(2); else WAIT_VMN(1); }
                }
                arrive(va, cprev);
                cprev = cb; oprev = ob; vs = vo; va = lane0;
                cur = index_of(nv);
            } while ((unsigned)cur < (unsigned)NPIN && known(flag_of(cur), 1u));
            __builtin_amdgcn_raw_buffer_store_b128(v[2], rwl, vo, oprev + 2048u, AUX_AGENT);
            __builtin_amdgcn_raw_buffer_store_b128(v[3], rwl, vo, oprev + 3072u, AUX_AGENT);
            WAIT_VM0();
            arrive(lane0, cprev);
        }
    }
    STAMP(P, 2);
    {
        constexpr unsigned BITS = (1u << NS) - 1u;
        const unsigned vb0 = (unsigned)(peer * NS) * 512u + (unsigned)(lane & 31) * 16u;
        auto off_of = [&](int i, int k) { const unsigned sg = (unsigned)(((i - NPIN) << PSH) + k) * 2u + (unsigned)(lane >> 5);
                                          return vb0 + sg / NS * (NN * 2u) + sg % NS * 512u; };
        auto flag_of = [&](int i) { return NIN * 4 + ((i - NPIN) >> (7 - PSH)) / NS * NS; };
        while ((unsigned)cur < (unsigned)NPC) {
            if (!bad && !known(flag_of(cur), BITS)) poll(flag_of(cur), BITS);
            if (bad) {
#pragma unroll
                for (int k = 0; k < PU; ++k) __builtin_amdgcn_raw_buffer_store_b128((u32x4){0x7fc07fc0u, 0x7fc07fc0u, 0x7fc07fc0u, 0x7fc07fc0u}, rol, off_of(cur, k), 0, CFG_OST);
                cur = index_of(draw()); continue;
            }
            STAMP1(P, 3);
            unsigned o[4] = {OOB, OOB, OOB, OOB};
            u32x4 v[4] = {};
            do {
                unsigned nv = 0;
#pragma unroll
                for (int k = 0; k < 4; ++k) {
                    o[k] = off_of(cur, k);
                    v[k] = __builtin_amdgcn_raw_buffer_load_b128(ror, o[k], 0, AUX_SYS_NT);
                    if (k == 0) nv = draw();
                    __builtin_amdgcn_raw_buffer_store_b128(v[k ^ 2], rol, o[k ^ 2], 0, CFG_OST);
                    if (k < 2) WAIT_VMN(4); else WAIT_VMN(3);
                    if (!deep) { if (k == 0) WAIT_VMN(2); else WAIT_VMN(1); }
                }
                cur = index_of(nv);
            } while ((unsigned)cur < (unsigned)NPC && known(flag_of(cur), BITS));
            __builtin_amdgcn_raw_buffer_store_b128(v[2], rol, o[2], 0, CFG_OST);
            __builtin_amdgcn_raw_buffer_store_b128(v[3], rol, o[3], 0, CFG_OST);
        }
    }
    WAIT_VM0();
    STAMP(P, 4);
    if (lane == 0) st_sys((unsigned*)(P.ws[peer] + WS_DONE + rank * 512) + widx, call);
    if (widx == 0) {
        const unsigned* dn = (const unsigned*)(wsl + WS_DONE + peer * 512);
        const unsigned long long t0 = __builtin_amdgcn_s_memrealtime();
        for (;;) {
            bool ok = true;
            if (lane < NW) ok = (int)(ld_sys(dn + lane) - call) >= 0;
            if (NW > 64 && lane + 64 < NW) ok = ok && (int)(ld_sys(dn + 64 + lane) - call) >= 0;
            if (__all(ok)) break;
            if (give_up(tmo, call, t0)) {
                if (lane == 0) __builtin_amdgcn_raw_buffer_store_b32(0x7fc07fc0u, rol, (unsigned)(rank * NS) * 512u, 0, AUX_SYS);
                break;
            }
            __builtin_amdgcn_s_sleep(2);
        }
        STAMP(P, 5);
    }
}

__device__ __forceinline__ void pack_role(const Params& P, unsigned tk, int g, int wid, int lane) {
    const int rank = P.rank;
    char* wsl = P.ws[rank];
    const rsrc_t rwl = mk_rsrc(wsl, WS_END);
    const rsrc_t rx = mk_rsrc(P.x, (unsigned)MM * KSH * 2), rw = mk_rsrc(P.w, (unsigned)KSH * NN * 2);
    const int fr = lane & 15, fq = lane >> 4;
    constexpr int U0 = G0 + (NRANK - 1) * NS;
    for (int u = g; u < NUNIT; u += NPK) {
        int o, dest = -1;
        if (u < 4 * U0) {
            const int r = u % U0;
            if (r < NRANK * NS) { o = r >> 3; dest = (rank + 1 + r) & (NRANK - 1); } else o = r - (NRANK - 1) * NS;
            o += u / U0 * G0;
        } else o = u - 4 * (NRANK - 1) * NS;
        int q; const int pan = unit_of(o, dest < 0 ? 0 : dest, q);
        const int ktl = q * 2 + (wid >> 2);
        const unsigned dst = WS_PAN + (unsigned)pan * PANB + (unsigned)(rank * KTS + ktl) * KTB;
        u32x4 v[8];
        if (pan < NTM) {
            const int rb0 = (wid & 3) * 4;
            const unsigned vo = (unsigned)fr * (KSH * 2) + (unsigned)fq * 16u;
            const unsigned so = (unsigned)(pan * 256 + rb0 * 16) * (KSH * 2) + (unsigned)ktl * 128u;
#pragma unroll
            for (int i = 0; i < 8; ++i) v[i] = __builtin_amdgcn_raw_buffer_load_b128(rx, vo + (i & 1) * 64, so + (unsigned)(i >> 1) * (16 * KSH * 2), 0);
#pragma unroll
            for (int i = 0; i < 8; ++i) __builtin_amdgcn_raw_buffer_store_b128(v[i], rwl, lane * 16 + (i & 1) * 1024, dst + (unsigned)(rb0 + (i >> 1)) * 2048u, AUX_SYS);
        } else {
            const int ks = (wid >> 1) & 1, wr = wid & 1;
            const unsigned vo = (unsigned)(fq * 8) * (NN * 2) + (unsigned)fr * 16u;
            const unsigned so = (unsigned)(ktl * 64 + ks * 32) * (NN * 2) + (unsigned)((pan - NTM) * 256 + wr * 128) * 2u;
#pragma unroll
            for (int j = 0; j < 8; ++j) v[j] = __builtin_amdgcn_raw_buffer_load_b128(rw, vo, so + (unsigned)j * (NN * 2), 2);
#pragma unroll
            for (int i = 0; i < 8; ++i) {
                u32x4 o;
#pragma unroll
                for (int e = 0; e < 4; ++e) o[e] = __builtin_amdgcn_perm(v[2 * e + 1][i >> 1], v[2 * e][i >> 1], (i & 1) ? 0x07060302u : 0x05040100u);
                __builtin_amdgcn_raw_buffer_store_b128(o, rwl, lane * 16, dst + (unsigned)(i >> 2) * HTB + (unsigned)wr * 8192u + (unsigned)(i & 3) * 2048u + (unsigned)ks * 1024u, AUX_SYS);
            }
        }
        WAIT_VM0();
        __builtin_amdgcn_s_barrier();
        if (wid == 0 && lane < NRANK && (dest < 0 || dest == lane)) {
            unsigned c = tk; asm volatile("" : "+v"(c));
            if (lane == rank) st_agent((unsigned*)(wsl + WS_CNT) + (pan * NRANK + rank) * 32 + q, (c + 1u) * CNT_FULL);
            else st_sys((unsigned*)(P.ws[lane] + WS_FLAG + rank * FLAGB) + o, c + 1u);
        }
    }
}

struct Tile { int tm, pan, ord; };
__device__ __forceinline__ bool tile_of(int idx, int col, int rank, Tile& t) {
    const int v = 2 * (idx >> 2) + (col >> 2);
    if (v >= NWP) return false;
    t.tm = 4 * (col & 3) + (idx & 3); t.pan = gpan_of(v, rank); t.ord = v < NS ? t.tm * NS + v : -1;
    return true;
}

__device__ __forceinline__ void gemm_role(const Params& P, unsigned call, int col, int slot, LAS unsigned char* lds) {
    const int tid = threadIdx.x, wid = __builtin_amdgcn_readfirstlane(tid >> 6), lane = tid & 63, wr = wid >> 2, wc = wid & 3;
    const int rank = P.rank;
    char* wsl = P.ws[rank];
    const rsrc_t rws = mk_rsrc(wsl, WS_END);
    LAS unsigned* ctl = (LAS unsigned*)(lds + STAGE_BYTES);
    const unsigned voff = (unsigned)tid * 16u;
    const unsigned ldsw = (unsigned)wid * 1024u;
    const int aoff = wr * 8192 + lane * 16, boff = wc * 4096 + lane * 16;

    auto ready = [&](const Tile& t, int want) -> int {
        if (wid == 0) {
            const unsigned* c = (const unsigned*)(wsl + WS_CNT) + ((((lane & 32) ? NTM + t.pan : t.tm) * NRANK + ((lane >> 2) & 7)) * 32 + (lane & 3));
            unsigned* tmo = (unsigned*)(wsl + WS_TMO);
            const unsigned need = call * CNT_FULL;
            const unsigned long long t0 = __builtin_amdgcn_s_memrealtime();
            int got;
            for (;;) {
                const unsigned long long miss = __ballot((int)(ld_agent(c) - need) < 0);
                unsigned m = (unsigned)miss | (unsigned)(miss >> 32);
                const bool own = ((m >> (rank * 4)) & 15u) == 0u;
                m |= m >> 16; m |= m >> 8; m |= m >> 4;
                got = own ? 1 + __builtin_ctz(m | 16u) : 0;
                if (got >= want) break;
                if (give_up(tmo, call, t0)) { ctl[0] = 1u; got = 5; break; }
                __builtin_amdgcn_s_sleep(4);
            }
            ctl[2] = (unsigned)got;
            __builtin_amdgcn_fence(__ATOMIC_ACQUIRE, "agent");
            asm volatile("s_waitcnt vmcnt(0) lgkmcnt(0)" ::: "memory");
        }
        asm volatile("" ::: "memory"); __builtin_amdgcn_s_barrier(); asm volatile("" ::: "memory");
        const int r = __builtin_amdgcn_readfirstlane(ctl[2]);
        asm volatile("s_waitcnt lgkmcnt(0)" ::: "memory");
        return r;
    };

#define SA(b, h) (((b) * 2 + (h)) * HTB)
#define SB(b, h) ((4 + (b) * 2 + (h)) * HTB)
#define STAGE(bufoff, soff) do { _Pragma("unroll") for (int _i = 0; _i < 2; ++_i) \
        __builtin_amdgcn_raw_ptr_buffer_load_lds(rws, (LAS unsigned*)(lds + (bufoff) + ldsw + _i * 8192), 16, voff, (soff) + _i * 8192u, 0, 0); } while (0)
#define LDA(dst, b, h) do { _Pragma("unroll") for (int m = 0; m < 4; ++m) _Pragma("unroll") for (int k = 0; k < 2; ++k) dst[m][k] = *(const LAS bf16x8*)(lds + SA(b, h) + aoff + m * 2048 + k * 1024); } while (0)
#define LDB(dst, b, h) do { _Pragma("unroll") for (int n = 0; n < 2; ++n) _Pragma("unroll") for (int k = 0; k < 2; ++k) dst[n][k] = *(const LAS bf16x8*)(lds + SB(b, h) + boff + n * 2048 + k * 1024); } while (0)
#define MMA(ai, bj, At, Bt) do { __builtin_amdgcn_s_setprio(1); _Pragma("unroll") for (int m = 0; m < 4; ++m) _Pragma("unroll") for (int n = 0; n < 2; ++n) _Pragma("unroll") for (int k = 0; k < 2; ++k) \
        acc[ai][bj][m][n] = __builtin_amdgcn_mfma_f32_16x16x32_bf16(Bt[n][k], At[m][k], acc[ai][bj][m][n], 0, 0, 0); __builtin_amdgcn_s_setprio(0); } while (0)
#define WAIT_V(n) asm volatile("s_waitcnt vmcnt(" #n ")" ::: "memory")
#define WAIT_L(n) asm volatile("s_waitcnt lgkmcnt(" #n ")" ::: "memory")
#define BAR __builtin_amdgcn_s_barrier()
#define SCHED __builtin_amdgcn_sched_barrier(0)

    Tile cur, nxt; int ui = slot;
    if (!tile_of(ui, col, rank, cur)) return;
    bf16x8 At[4][2], B0[2][2], B1[2][2];
    unsigned cA = WS_PAN + (unsigned)(NTM + cur.pan) * PANB, cB = WS_PAN + (unsigned)cur.tm * PANB;
    const unsigned k0 = (unsigned)rank * CHB;
    int lvl = ready(cur, 1);
    STAMP(P, 2);
    STAGE(SB(0, 0), cB + k0); STAGE(SA(0, 0), cA + k0); STAGE(SB(0, 1), cB + k0 + HTB); STAGE(SA(0, 1), cA + k0 + HTB);
    if (wr == 1) BAR;
    WAIT_V(4); BAR;
    STAGE(SB(1, 0), cB + k0 + KTB); STAGE(SA(1, 0), cA + k0 + KTB); STAGE(SB(1, 1), cB + k0 + HTB + KTB);
    WAIT_V(6); BAR;
#pragma unroll 1
    for (;;) {
        f32x4 acc[2][2][4][2];
#pragma unroll
        for (int a = 0; a < 2; ++a)
#pragma unroll
            for (int b = 0; b < 2; ++b)
#pragma unroll
                for (int m = 0; m < 4; ++m)
#pragma unroll
                    for (int n = 0; n < 2; ++n) acc[a][b][m][n] = (f32x4){0.f, 0.f, 0.f, 0.f};
        const bool has_next = tile_of(ui + KX, col, rank, nxt);
        const unsigned nA = has_next ? WS_PAN + (unsigned)(NTM + nxt.pan) * PANB : cA, nB = has_next ? WS_PAN + (unsigned)nxt.tm * PANB : cB;
        unsigned ko = k0;
        int jchk = lvl >= 5 ? 31 : 7 * lvl - 4;
        for (int j = 0; j < KT / 2; ++j) {
            const bool last = (j == KT / 2 - 1);
            if (j == jchk && (has_next || !last)) {
                lvl = ready(last ? nxt : cur, last ? 1 : lvl + 1);
                jchk = lvl >= 5 ? 31 : 7 * lvl - 4;
            }
            unsigned kn;
            if (j < 3) kn = ko + UNB;
            else {
                kn = ((j == 3 ? k0 : ko) + CHB) & (PANB - 1);
                if ((kn ^ k0) < CHB) kn = (kn + CHB + UNB) & (PANB - 1);
            }
            const unsigned a1 = cA + ko + KTB;
            const unsigned a2 = last ? nA + k0 : cA + kn, b2 = last ? nB + k0 : cB + kn;
            const unsigned a3 = a2 + KTB, b3 = b2 + KTB;
            ko = kn;
            LDB(B0, 0, 0); SCHED; LDA(At, 0, 0); STAGE(SA(1, 1), a1 + HTB);
            WAIT_L(8); BAR; WAIT_L(0); MMA(0, 0, At, B0); BAR; SCHED;
            LDB(B1, 0, 1); STAGE(SB(0, 0), b2);
            BAR; WAIT_L(0); MMA(0, 1, At, B1); BAR;
            LDA(At, 0, 1); STAGE(SA(0, 0), a2);
            BAR; WAIT_L(0); MMA(1, 0, At, B0); BAR; SCHED;
            STAGE(SB(0, 1), b2 + HTB);
            WAIT_V(6); BAR; MMA(1, 1, At, B1); BAR;
            LDB(B0, 1, 0); SCHED; LDA(At, 1, 0); STAGE(SA(0, 1), a2 + HTB);
            WAIT_L(8); BAR; WAIT_L(0); MMA(0, 0, At, B0); BAR; SCHED;
            LDB(B1, 1, 1); STAGE(SB(1, 0), b3);
            BAR; WAIT_L(0); MMA(0, 1, At, B1); BAR;
            LDA(At, 1, 1); STAGE(SA(1, 0), a3);
            BAR; WAIT_L(0); MMA(1, 0, At, B0); BAR; SCHED;
            STAGE(SB(1, 1), b3 + HTB);
            WAIT_V(6); BAR; MMA(1, 1, At, B1); BAR;
        }
        {
            int tz = threadIdx.x; asm volatile("" : "+v"(tz));
            const int ln = tz & 63, fr = ln & 15, fq = ln >> 4;
            const rsrc_t rol = mk_rsrc(P.out[rank], OUTB);
            const bool bad = ctl[0] != 0u;
            const unsigned vo = (unsigned)(wc * 32 + fq * 4) * (NN * 2) + (unsigned)(wr * 256 + fr * 16);
            const unsigned so = (unsigned)(cur.tm * 256) * (NN * 2) + (unsigned)cur.pan * 512u;
#pragma unroll
            for (int bj = 0; bj < 2; ++bj)
#pragma unroll
                for (int n = 0; n < 2; ++n)
#pragma unroll
                    for (int r = 0; r < 4; ++r) {
                        u32x4 o;
                        o[0] = cvt_pk_bf16(silu(acc[0][bj][0][n][r]), silu(acc[0][bj][1][n][r]));
                        o[1] = cvt_pk_bf16(silu(acc[0][bj][2][n][r]), silu(acc[0][bj][3][n][r]));
                        o[2] = cvt_pk_bf16(silu(acc[1][bj][0][n][r]), silu(acc[1][bj][1][n][r]));
                        o[3] = cvt_pk_bf16(silu(acc[1][bj][2][n][r]), silu(acc[1][bj][3][n][r]));
                        if (bad) o = (u32x4){0x7fc07fc0u, 0x7fc07fc0u, 0x7fc07fc0u, 0x7fc07fc0u};
                        if (CFG_RST == AUX_SYS || cur.ord >= 0) __builtin_amdgcn_raw_buffer_store_b128(o, rol, vo, so + (unsigned)(bj * 128 + n * 16 + r) * (NN * 2), AUX_SYS);
                        else __builtin_amdgcn_raw_buffer_store_b128(o, rol, vo, so + (unsigned)(bj * 128 + n * 16 + r) * (NN * 2), CFG_RST);
                    }
            if (cur.ord >= 0) {
                WAIT_VM0();
                unsigned old = 0;
                if (ln == 0) old = __hip_atomic_fetch_add(ctl + 1, 1u, __ATOMIC_RELAXED, __HIP_MEMORY_SCOPE_WORKGROUP);
                old = __builtin_amdgcn_readfirstlane(old);
                if ((old & 7u) == 7u && ln < NRANK && ln != rank)
                    st_sys((unsigned*)(P.ws[ln] + WS_FLAG + rank * FLAGB) + NIN * 4 + cur.ord, call);
            }
        }
        STAMP(P, 3 + (ui >= KX ? 1 : 0));
        if (!has_next) break;
        cur = nxt; cA = nA; cB = nB; ui += KX;
    }
    WAIT_V(0);
    if (wr == 0) BAR;
}

__global__ __launch_bounds__(512, 2) void fused_gemm_ar(Params P) {
    extern __shared__ __attribute__((aligned(16))) unsigned char shm[];
    LAS unsigned char* lds = (LAS unsigned char*)shm;
    const int wid = __builtin_amdgcn_readfirstlane(threadIdx.x >> 6), lane = threadIdx.x & 63;
    const int b = __builtin_amdgcn_readfirstlane(blockIdx.x);
    unsigned* tkp = (unsigned*)(P.ws[P.rank] + WS_TICKET + b * 128 + wid * 16);
    const unsigned tk = ld_agent(tkp);
    if (b < NPULL) {
        const unsigned call = __builtin_amdgcn_readfirstlane(tk) + 1u;
        st_agent(tkp, call);
        const int pi = b / PW, slot = b % PW;
        const int peer = (P.rank + 1 + pi) & (NRANK - 1);
        pull_role(P, call, peer, slot * 8 + wid, lane);
    } else {
        const int g = b - NPULL;
        ((LAS unsigned*)(lds + STAGE_BYTES))[lane & 3] = 0u;
        STAMP(P, 0);
        pack_role(P, tk, g, wid, lane);
        const unsigned call = __builtin_amdgcn_readfirstlane(tk) + 1u;
        st_agent(tkp, call);
        STAMP(P, 1);
        asm volatile("s_waitcnt lgkmcnt(0)" ::: "memory");
        __builtin_amdgcn_s_barrier();
        if (b >= NPS) gemm_role(P, call, b & 7, (b - NPS) >> 3, lds);
    }
}
}

extern "C" void kernel_launch(int rank, int nranks, ncclComm_t comm,
                              hipStream_t stream,
                              void* const* d_in, const int* in_sizes, int n_in,
                              void* d_out, int out_size,
                              const int* in_dtypes, int out_dtype,
                              void* const* d_out_all,
                              void* const* d_ws_all, size_t ws_bytes) {
    if (nranks != NRANK || (unsigned)rank >= (unsigned)NRANK || ws_bytes < (size_t)WS_END || n_in < 2 ||
        in_sizes[0] != MM * KSH || in_sizes[1] != KSH * NN || out_size != MM * NN) {
        fprintf(stderr, "unexpected problem geometry\n");
        return;
    }
    static bool attr_done[NRANK] = {};
    if (!attr_done[rank]) {
        HIP_CHECK(hipFuncSetAttribute((const void*)fused_gemm_ar, hipFuncAttributeMaxDynamicSharedMemorySize, LDS_BYTES));
        attr_done[rank] = true;
    }
    Params P;
    P.x = d_in[0]; P.w = d_in[1]; P.rank = rank;
    for (int p = 0; p < NRANK; ++p) { P.out[p] = (char*)d_out_all[p]; P.ws[p] = (char*)d_ws_all[p]; }
    hipLaunchKernelGGL(fused_gemm_ar, dim3(GRID), dim3(512), LDS_BYTES, stream, P);
    HIP_CHECK(hipGetLastError());
}
```

```cpp
#include <hip/hip_runtime.h>
#include <rccl/rccl.h>
#include <cstdio>

#define HIP_CHECK(expr) do { hipError_t _e = (expr); if (_e != hipSuccess) { \
    fprintf(stderr, "HIP error %s at %s:%d\n", hipGetErrorString(_e), __FILE__, __LINE__); \
    return; } } while (0)

#define LAS __attribute__((address_space(3)))
typedef short bf16x8 __attribute__((ext_vector_type(8)));
typedef float f32x4 __attribute__((ext_vector_type(4)));
typedef unsigned u32x4 __attribute__((ext_vector_type(4)));
typedef __amdgpu_buffer_rsrc_t rsrc_t;

namespace {
constexpr int NRANK = 8;
constexpr int MM = 4096, KK = 4096, NN = 32768, KSH = KK / NRANK;
constexpr int KT = KK / 64, KTS = KSH / 64, NTM = MM / 256, NPAN = NN / 256;
#define CFG_NS 8
#define CFG_PW 8
constexpr int NS = CFG_NS;
constexpr int NRP = NPAN - NRANK * NS;
constexpr int NWP = NS + NRP;
constexpr int NIN = NTM + NWP;
constexpr int NOUT = NS * NTM;
constexpr int NFLAG = NIN * 4 + NOUT;
constexpr int NUNIT = NTM * 4 + NPAN * 4;
constexpr int GRID = 256;
constexpr int PW = CFG_PW;
constexpr int NPULL = PW * (NRANK - 1);
constexpr int NW = PW * 8;
constexpr int NPK = GRID - NPULL;
constexpr int NPS = (NPULL + 7) / 8 * 8;
constexpr int KX = (GRID - NPS) / 8;
constexpr int GW = 2 * ((KX + 3) / 4);
constexpr int G0 = GW + NTM;
constexpr int NGF = (NWP - GW) / GW, GL = (NWP - GW) % GW;
static_assert(NS <= GW && NWP > GW, "shared panels sit in the first group");
#define CFG_PU 4
constexpr int PU = CFG_PU, PSH = PU == 1 ? 0 : PU == 2 ? 1 : PU == 4 ? 2 : 3;
#define CFG_PUSH 2
constexpr int NPUSH = CFG_PUSH;
constexpr int NPIN = (NIN * 4 - NPUSH) * 64 / PU;
static_assert(NPUSH <= NS, "pushed units have a single consumer");
constexpr int NPC = NPIN + NOUT * 128 / PU;
constexpr unsigned CNT_FULL = 64;

constexpr unsigned KTB = 32768, HTB = 16384;
constexpr unsigned PANB = KT * KTB;
constexpr unsigned CHB = KTS * KTB;
constexpr unsigned UNB = 2 * KTB;
constexpr unsigned WS_TICKET = 0;
constexpr unsigned WS_TMO = WS_TICKET + GRID * 128;
constexpr unsigned WS_FLAG = WS_TMO + 128;
constexpr unsigned FLAGB = 4096;
constexpr unsigned WS_DONE = WS_FLAG + NRANK * FLAGB;
constexpr unsigned WS_HEAD = WS_DONE + NRANK * 512;
constexpr unsigned WS_CNT = WS_HEAD + NRANK * 128;
constexpr unsigned WS_CTRL_END = WS_CNT + (NTM + NPAN) * NRANK * 128;
constexpr unsigned WS_DBG = 512u << 10;
constexpr unsigned WS_PAN = 1u << 20;
constexpr unsigned WS_END = WS_PAN + (NTM + NPAN) * PANB;
constexpr unsigned OUTB = (unsigned)MM * NN * 2;
static_assert(WS_CTRL_END <= WS_DBG, "ctrl region");
static_assert(NFLAG <= 512 && 2048 <= (int)FLAGB, "flag region");
static_assert(NW <= 128, "done words");
static_assert(PU == 4, "block size");
#define CFG_DEEP 2
constexpr unsigned STAGE_BYTES = 8 * HTB;
constexpr unsigned LDS_BYTES = STAGE_BYTES + 64;
constexpr unsigned long long DEADLINE = 50000000ull;
constexpr int AUX_SYS = 17, AUX_SYS_NT = 19, AUX_AGENT = 16;
#define CFG_PACE 40
#define CFG_LOOK 256
#define CFG_LAST 8
#define CFG_RST 0

struct Params {
    const void* x; const void* w;
    char* out[NRANK]; char* ws[NRANK];
    int rank;
};

__device__ __forceinline__ rsrc_t mk_rsrc(const void* p, unsigned bytes) {
    const unsigned long long v = (unsigned long long)p;
    const unsigned lo = __builtin_amdgcn_readfirstlane((unsigned)v), hi = __builtin_amdgcn_readfirstlane((unsigned)(v >> 32));
    return __builtin_amdgcn_make_buffer_rsrc((void*)(((unsigned long long)hi << 32) | lo), 0, bytes, 0x00020000);
}
__device__ __forceinline__ unsigned ld_sys(const unsigned* p) { return __hip_atomic_load(p, __ATOMIC_RELAXED, __HIP_MEMORY_SCOPE_SYSTEM); }
__device__ __forceinline__ unsigned ld_agent(const unsigned* p) { return __hip_atomic_load(p, __ATOMIC_RELAXED, __HIP_MEMORY_SCOPE_AGENT); }
__device__ __forceinline__ void st_sys(unsigned* p, unsigned v) { __hip_atomic_store(p, v, __ATOMIC_RELAXED, __HIP_MEMORY_SCOPE_SYSTEM); }
__device__ __forceinline__ void st_agent(unsigned* p, unsigned v) { __hip_atomic_store(p, v, __ATOMIC_RELAXED, __HIP_MEMORY_SCOPE_AGENT); }
__device__ __forceinline__ void add_agent(unsigned* p, unsigned v) { (void)__hip_atomic_fetch_add(p, v, __ATOMIC_RELAXED, __HIP_MEMORY_SCOPE_AGENT); }
#define WAIT_VM0() asm volatile("s_waitcnt vmcnt(0)" ::: "memory")
#define WAIT_VMN(n) do { asm volatile("" ::: "memory"); __builtin_amdgcn_s_waitcnt(0x0F70 | (n)); asm volatile("" ::: "memory"); } while (0)

__device__ __forceinline__ bool give_up(unsigned* tmo, unsigned call, unsigned long long t0) {
    if ((unsigned)__builtin_amdgcn_readfirstlane(ld_agent(tmo)) == call) return true;
    if (__builtin_amdgcn_s_memrealtime() - t0 > DEADLINE) { st_agent(tmo, call); return true; }
    return false;
}

__device__ __forceinline__ int gpan_of(int v, int consumer) { return v < NS ? consumer * NS + v : (NRANK - 1) * NS + v; }
__device__ __forceinline__ int unit_of(int o, int consumer, int& q) {
    if (o < 4 * G0) { q = o / G0; const int m = o % G0; return m < GW ? NTM + gpan_of(m, consumer) : m - GW; }
    o -= 4 * G0;
    const int g = o / (4 * GW), r = o % (4 * GW);
    int i;
    if (GL == 0 || g < NGF) { q = r / GW; i = r % GW; } else { q = r / (GL ? GL : 1); i = r % (GL ? GL : 1); }
    return NTM + (NRANK - 1) * NS + GW * (g + 1) + i;
}

__device__ __forceinline__ unsigned cvt_pk_bf16(float lo, float hi) { unsigned r; asm("v_cvt_pk_bf16_f32 %0, %1, %2" : "=v"(r) : "v"(lo), "v"(hi)); return r; }
__device__ __forceinline__ float silu(float y) {
    const float c = __builtin_amdgcn_fmed3f(y, -60.0f, 60.0f);
    const float e = __builtin_amdgcn_exp2f(c * -1.44269504088896f);
    return y * __builtin_amdgcn_rcpf(1.0f + e);
}

#define STAMP1(P, slot) do {} while (0)
#define STAMP(P, slot) do {} while (0)

__device__ __forceinline__ void pull_role(const Params& P, unsigned call, int peer, int widx, int lane) {
    const int rank = P.rank;
    char* wsl = P.ws[rank];
    const rsrc_t rwr = mk_rsrc(P.ws[peer], WS_END), rwl = mk_rsrc(wsl, WS_END);
    const rsrc_t ror = mk_rsrc(P.out[peer], OUTB), rol = mk_rsrc(P.out[rank], OUTB);
    unsigned* tmo = (unsigned*)(wsl + WS_TMO);
    constexpr unsigned OOB = 0xfffffff0u;
    const unsigned lane0 = lane == 0 ? 0u : OOB;
    const unsigned hbase = (call - 1u) * (unsigned)(NPC + NW);
    auto draw = [&]() { return (unsigned)__builtin_amdgcn_raw_ptr_buffer_atomic_add_i32(1, rwl, lane0, WS_HEAD + (unsigned)peer * 128u, 0); };
    auto arrive = [&](unsigned vo, unsigned cnt) { (void)__builtin_amdgcn_raw_ptr_buffer_atomic_add_i32(PU, rwl, vo, cnt, 0); };
    const bool deep = (widx & 7) < CFG_DEEP;
    bool bad = false; int stamped = 0; (void)stamped;
    STAMP(P, 0);
    auto index_of = [&](unsigned v) { return (int)((unsigned)__builtin_amdgcn_readfirstlane(v) - hbase); };
    int cur = index_of(draw());
    constexpr int BPB = NS * 128 / PU;
    const unsigned fso = WS_FLAG + (unsigned)peer * FLAGB;
    auto first_of = [&](int i) { return i < NPIN ? (i >> (6 - PSH)) + NPUSH : NIN * 4 + (i - NPIN) / BPB * NS; };
    auto limit_of = [&](const u32x4& f0, const u32x4& f1, int i) {
        const int first = first_of(i);
        unsigned un = 0;
#pragma unroll
        for (int j = 0; j < 8; ++j) un |= ((int)((j < 4 ? f0[j & 3] : f1[j & 3]) - call) < 0 && lane * 8 + j >= first ? 1u : 0u) << j;
        const int l = __builtin_ctzll(__ballot(un != 0u) | (1ull << 63));
        const int u = l * 8 + __builtin_ctz(__builtin_amdgcn_readlane(un, l) | 256u);
        return u <= NIN * 4 ? (u - NPUSH) << (6 - PSH) : u >= NFLAG ? NPC : NPIN + (u - NIN * 4) / NS * BPB;
    };
    auto offs_of = [&](int i, unsigned (&o)[4], unsigned& cnt) {
        if (i < NPIN) {
            int q; const int pan = unit_of((i >> (6 - PSH)) + NPUSH, rank, q);
            cnt = WS_CNT + (unsigned)(pan * NRANK + peer) * 128u + (unsigned)q * 4u;
            const unsigned b = WS_PAN + (unsigned)pan * PANB + (unsigned)peer * CHB + (unsigned)q * UNB + (unsigned)((i << PSH) & 63) * 1024u + (unsigned)lane * 16u;
#pragma unroll
            for (int k = 0; k < 4; ++k) o[k] = b + k * 1024u;
        } else {
            cnt = 0;
#pragma unroll
            for (int k = 0; k < 4; ++k) {
                const unsigned sg = (unsigned)(((i - NPIN) << PSH) + k) * 2u + (unsigned)(lane >> 5);
                o[k] = (unsigned)(peer * NS) * 512u + (unsigned)(lane & 31) * 16u + sg / NS * (NN * 2u) + sg % NS * 512u;
            }
        }
    };
    int lim = 0;
    while ((unsigned)cur < (unsigned)NPC) {
        if (!bad) {
            const unsigned long long t0 = __builtin_amdgcn_s_memrealtime();
            const unsigned pvo = (unsigned)(lane - (first_of(cur) >> 3)) < 4u ? (unsigned)lane * 32u : OOB - 16u;
            for (;;) {
                const u32x4 f0 = __builtin_amdgcn_raw_buffer_load_b128(rwl, pvo, fso, AUX_SYS), f1 = __builtin_amdgcn_raw_buffer_load_b128(rwl, pvo + 16u, fso, AUX_SYS);
                lim = limit_of(f0, f1, cur);
                if (cur < lim) break;
                if (give_up(tmo, call, t0)) { bad = true; break; }
                __builtin_amdgcn_s_sleep(2);
            }
            asm volatile("" ::: "memory");
        }
        unsigned o[4], p2 = OOB, p3 = OOB, va = OOB, cprev = 0;
        if (bad) {
            offs_of(cur, o, cprev);
            if (cur >= NPIN) {
#pragma unroll
                for (int k = 0; k < PU; ++k) __builtin_amdgcn_raw_buffer_store_b128((u32x4){0x7fc07fc0u, 0x7fc07fc0u, 0x7fc07fc0u, 0x7fc07fc0u}, rol, o[k], 0, AUX_AGENT);
            }
            cur = index_of(draw()); continue;
        }
        STAMP1(P, 1);
        rsrc_t rp = rwl;
        u32x4 v[4] = {};
        do {
            unsigned cb; offs_of(cur, o, cb);
            const bool in = cur < NPIN;
            const rsrc_t rs = in ? rwr : ror, rd = in ? rwl : rol;
            const bool look = lim < NPC && cur + CFG_LOOK >= lim;
            const unsigned fvo = look ? (unsigned)lane * 32u : OOB - 16u;
            unsigned nv = 0; u32x4 f0, f1;
#pragma unroll
            for (int k = 0; k < 4; ++k) {
                v[k] = __builtin_amdgcn_raw_buffer_load_b128(rs, o[k], 0, AUX_SYS_NT);
                if (k == 0) {
                    nv = draw();
                    f0 = __builtin_amdgcn_raw_buffer_load_b128(rwl, fvo, fso, AUX_SYS); f1 = __builtin_amdgcn_raw_buffer_load_b128(rwl, fvo + 16u, fso, AUX_SYS);
                }
                __builtin_amdgcn_raw_buffer_store_b128(v[k ^ 2], k < 2 ? rp : rd, k == 0 ? p2 : k == 1 ? p3 : o[k ^ 2], 0, AUX_AGENT);
                if (k == 0) WAIT_VMN(7); else if (k == 1) WAIT_VMN(6); else WAIT_VMN(3);
                if (!deep) { if (k == 0) WAIT_VMN(4); else WAIT_VMN(1); }
            }
            arrive(va, cprev);
            cprev = cb; va = in ? lane0 : OOB; rp = rd; p2 = o[2]; p3 = o[3];
            cur = index_of(nv);
            if (look) { const int l2 = limit_of(f0, f1, cur); lim = l2 > lim ? l2 : lim; }
        } while ((unsigned)cur < (unsigned)lim);
        __builtin_amdgcn_raw_buffer_store_b128(v[2], rp, p2, 0, AUX_AGENT);
        __builtin_amdgcn_raw_buffer_store_b128(v[3], rp, p3, 0, AUX_AGENT);
        WAIT_VM0();
        arrive(va, cprev);
    }
    WAIT_VM0();
    STAMP(P, 4);
    if (lane == 0) st_sys((unsigned*)(P.ws[peer] + WS_DONE + rank * 512) + widx, call);
    if (widx == 0) {
        const unsigned* dn = (const unsigned*)(wsl + WS_DONE + peer * 512);
        const unsigned long long t0 = __builtin_amdgcn_s_memrealtime();
        for (;;) {
            bool ok = true;
            if (lane < NW) ok = (int)(ld_sys(dn + lane) - call) >= 0;
            if (NW > 64 && lane + 64 < NW) ok = ok && (int)(ld_sys(dn + 64 + lane) - call) >= 0;
            if (__all(ok)) break;
            if (give_up(tmo, call, t0)) {
                if (lane == 0) __builtin_amdgcn_raw_buffer_store_b32(0x7fc07fc0u, rol, (unsigned)(rank * NS) * 512u, 0, AUX_SYS);
                break;
            }
            __builtin_amdgcn_s_sleep(2);
        }
        STAMP(P, 5);
    }
}

__device__ __forceinline__ void pack_role(const Params& P, unsigned tk, int g, int wid, int lane) {
    const int rank = P.rank;
    char* wsl = P.ws[rank];
    const rsrc_t rx = mk_rsrc(P.x, (unsigned)MM * KSH * 2), rw = mk_rsrc(P.w, (unsigned)KSH * NN * 2);
    const int fr = lane & 15, fq = lane >> 4;
    constexpr int U0 = G0 + (NRANK - 1) * NS;
    const unsigned long long t0 = __builtin_amdgcn_s_memrealtime();
    for (int u = g; u < NUNIT; u += NPK) {
        int o, dest = -1;
        if (u < 4 * U0) {
            const int r = u % U0;
            if (r < NRANK * NS) { o = r >> 3; dest = (rank + 1 + r) & (NRANK - 1); } else o = r - (NRANK - 1) * NS;
            o += u / U0 * G0;
        } else o = u - 4 * (NRANK - 1) * NS;
        int q; const int pan = unit_of(o, dest < 0 ? 0 : dest, q);
        const bool push = o < NPUSH;
        const rsrc_t rwl = mk_rsrc(P.ws[push ? dest & (NRANK - 1) : rank], WS_END);
        while (CFG_PACE > 0 && (unsigned)(__builtin_amdgcn_s_memrealtime() - t0) < (unsigned)(u * CFG_PACE / 10)) __builtin_amdgcn_s_sleep(1);
        const int ktl = q * 2 + (wid >> 2);
        const unsigned dst = WS_PAN + (unsigned)pan * PANB + (unsigned)(rank * KTS + ktl) * KTB;
        u32x4 v[8];
        if (pan < NTM) {
            const int rb0 = (wid & 3) * 4;
            const unsigned vo = (unsigned)fr * (KSH * 2) + (unsigned)fq * 16u;
            const unsigned so = (unsigned)(pan * 256 + rb0 * 16) * (KSH * 2) + (unsigned)ktl * 128u;
#pragma unroll
            for (int i = 0; i < 8; ++i) v[i] = __builtin_amdgcn_raw_buffer_load_b128(rx, vo + (i & 1) * 64, so + (unsigned)(i >> 1) * (16 * KSH * 2), 0);
#pragma unroll
            for (int i = 0; i < 8; ++i) __builtin_amdgcn_raw_buffer_store_b128(v[i], rwl, lane * 16 + (i & 1) * 1024, dst + (unsigned)(rb0 + (i >> 1)) * 2048u, AUX_SYS);
        } else {
            const int ks = (wid >> 1) & 1, wr = wid & 1;
            const unsigned vo = (unsigned)(fq * 8) * (NN * 2) + (unsigned)fr * 16u;
            const unsigned so = (unsigned)(ktl * 64 + ks * 32) * (NN * 2) + (unsigned)((pan - NTM) * 256 + wr * 128) * 2u;
#pragma unroll
            for (int j = 0; j < 8; ++j) v[j] = __builtin_amdgcn_raw_buffer_load_b128(rw, vo, so + (unsigned)j * (NN * 2), 2);
#pragma unroll
            for (int i = 0; i < 8; ++i) {
                u32x4 o;
#pragma unroll
                for (int e = 0; e < 4; ++e) o[e] = __builtin_amdgcn_perm(v[2 * e + 1][i >> 1], v[2 * e][i >> 1], (i & 1) ? 0x07060302u : 0x05040100u);
                __builtin_amdgcn_raw_buffer_store_b128(o, rwl, lane * 16, dst + (unsigned)(i >> 2) * HTB + (unsigned)wr * 8192u + (unsigned)(i & 3) * 2048u + (unsigned)ks * 1024u, AUX_SYS);
            }
        }
        WAIT_VM0();
        __builtin_amdgcn_s_barrier();
        if (wid == 0 && lane < NRANK && (dest < 0 || dest == lane)) {
            unsigned c = tk; asm volatile("" : "+v"(c));
            if (lane == rank || push) st_sys((unsigned*)(P.ws[lane] + WS_CNT) + (pan * NRANK + rank) * 32 + q, (c + 1u) * CNT_FULL);
            else st_sys((unsigned*)(P.ws[lane] + WS_FLAG + rank * FLAGB) + o, c + 1u);
        }
    }
}

struct Tile { int tm, pan, ord; };
__device__ __forceinline__ bool tile_of(int idx, int col, int rank, Tile& t) {
    const int v = 2 * (idx >> 2) + (col >> 2);
    if (v >= NWP) return false;
    t.tm = 4 * (col & 3) + (idx & 3); t.pan = gpan_of(v, rank); t.ord = v < NS ? t.tm * NS + v : -1;
    return true;
}

__device__ __forceinline__ void gemm_role(const Params& P, unsigned call, int col, int slot, LAS unsigned char* lds) {
    const int tid = threadIdx.x, wid = __builtin_amdgcn_readfirstlane(tid >> 6), lane = tid & 63, wr = wid >> 2, wc = wid & 3;
    const int rank = P.rank;
    char* wsl = P.ws[rank];
    const rsrc_t rws = mk_rsrc(wsl, WS_END);
    LAS unsigned* ctl = (LAS unsigned*)(lds + STAGE_BYTES);
    const unsigned voff = (unsigned)tid * 16u;
    const unsigned ldsw = (unsigned)wid * 1024u;
    const int aoff = wr * 8192 + lane * 16, boff = wc * 4096 + lane * 16;

    auto ready = [&](const Tile& t, int want) -> int {
        if (wid == 0) {
            const unsigned* c = (const unsigned*)(wsl + WS_CNT) + ((((lane & 32) ? NTM + t.pan : t.tm) * NRANK + ((lane >> 2) & 7)) * 32 + (lane & 3));
            unsigned* tmo = (unsigned*)(wsl + WS_TMO);
            const unsigned need = call * CNT_FULL;
            const unsigned long long t0 = __builtin_amdgcn_s_memrealtime();
            int got;
            for (;;) {
                const unsigned long long miss = __ballot((int)(ld_sys(c) - need) < 0);
                unsigned m = (unsigned)miss | (unsigned)(miss >> 32);
                const bool own = ((m >> (rank * 4)) & 15u) == 0u;
                m |= m >> 16; m |= m >> 8; m |= m >> 4;
                got = own ? 1 + __builtin_ctz(m | 16u) : 0;
                if (got >= want) break;
                if (give_up(tmo, call, t0)) { ctl[0] = 1u; got = 5; break; }
                __builtin_amdgcn_s_sleep(4);
            }
            ctl[2] = (unsigned)got;
            __builtin_amdgcn_fence(__ATOMIC_ACQUIRE, "agent");
            asm volatile("s_waitcnt vmcnt(0) lgkmcnt(0)" ::: "memory");
        }
        asm volatile("" ::: "memory"); __builtin_amdgcn_s_barrier(); asm volatile("" ::: "memory");
        const int r = __builtin_amdgcn_readfirstlane(ctl[2]);
        asm volatile("s_waitcnt lgkmcnt(0)" ::: "memory");
        return r;
    };

#define SA(b, h) (((b) * 2 + (h)) * HTB)
#define SB(b, h) ((4 + (b) * 2 + (h)) * HTB)
#define STAGE(bufoff, soff) do { _Pragma("unroll") for (int _i = 0; _i < 2; ++_i) \
        __builtin_amdgcn_raw_ptr_buffer_load_lds(rws, (LAS unsigned*)(lds + (bufoff) + ldsw + _i * 8192), 16, voff, (soff) + _i * 8192u, 0, 0); } while (0)
#define LDA(dst, b, h) do { _Pragma("unroll") for (int m = 0; m < 4; ++m) _Pragma("unroll") for (int k = 0; k < 2; ++k) dst[m][k] = *(const LAS bf16x8*)(lds + SA(b, h) + aoff + m * 2048 + k * 1024); } while (0)
#define LDB(dst, b, h) do { _Pragma("unroll") for (int n = 0; n < 2; ++n) _Pragma("unroll") for (int k = 0; k < 2; ++k) dst[n][k] = *(const LAS bf16x8*)(lds + SB(b, h) + boff + n * 2048 + k * 1024); } while (0)
#define MMA(ai, bj, At, Bt) do { __builtin_amdgcn_s_setprio(1); _Pragma("unroll") for (int m = 0; m < 4; ++m) _Pragma("unroll") for (int n = 0; n < 2; ++n) _Pragma("unroll") for (int k = 0; k < 2; ++k) \
        acc[ai][bj][m][n] = __builtin_amdgcn_mfma_f32_16x16x32_bf16(Bt[n][k], At[m][k], acc[ai][bj][m][n], 0, 0, 0); __builtin_amdgcn_s_setprio(0); } while (0)
#define WAIT_V(n) asm volatile("s_waitcnt vmcnt(" #n ")" ::: "memory")
#define WAIT_L(n) asm volatile("s_waitcnt lgkmcnt(" #n ")" ::: "memory")
#define BAR __builtin_amdgcn_s_barrier()
#define SCHED __builtin_amdgcn_sched_barrier(0)

    Tile cur, nxt; int ui = slot;
    if (!tile_of(ui, col, rank, cur)) return;
    bf16x8 At[4][2], B0[2][2], B1[2][2];
    unsigned cA = WS_PAN + (unsigned)(NTM + cur.pan) * PANB, cB = WS_PAN + (unsigned)cur.tm * PANB;
    const unsigned k0 = (unsigned)rank * CHB;
    int lvl = ready(cur, 1);
    STAMP(P, 2);
    STAGE(SB(0, 0), cB + k0); STAGE(SA(0, 0), cA + k0); STAGE(SB(0, 1), cB + k0 + HTB); STAGE(SA(0, 1), cA + k0 + HTB);
    if (wr == 1) BAR;
    WAIT_V(4); BAR;
    STAGE(SB(1, 0), cB + k0 + KTB); STAGE(SA(1, 0), cA + k0 + KTB); STAGE(SB(1, 1), cB + k0 + HTB + KTB);
    WAIT_V(6); BAR;
#pragma unroll 1
    for (;;) {
        f32x4 acc[2][2][4][2];
#pragma unroll
        for (int a = 0; a < 2; ++a)
#pragma unroll
            for (int b = 0; b < 2; ++b)
#pragma unroll
                for (int m = 0; m < 4; ++m)
#pragma unroll
                    for (int n = 0; n < 2; ++n) acc[a][b][m][n] = (f32x4){0.f, 0.f, 0.f, 0.f};
        const bool has_next = tile_of(ui + KX, col, rank, nxt);
        const unsigned nA = has_next ? WS_PAN + (unsigned)(NTM + nxt.pan) * PANB : cA, nB = has_next ? WS_PAN + (unsigned)nxt.tm * PANB : cB;
        unsigned ko = k0;
        int jchk = lvl >= 5 ? 31 : 7 * lvl - 4;
        for (int j = 0; j < KT / 2; ++j) {
            const bool last = (j == KT / 2 - 1);
            if (j == jchk && (has_next || !last)) {
                lvl = ready(last ? nxt : cur, last ? 1 : lvl + 1);
                jchk = lvl >= 5 ? 31 : 7 * lvl - 4;
            }
            unsigned kn;
            if (j < 3) kn = ko + UNB;
            else {
                kn = ((j == 3 ? k0 : ko) + CHB) & (PANB - 1);
                if ((kn ^ k0) < CHB) kn = (kn + CHB + UNB) & (PANB - 1);
            }
            const unsigned a1 = cA + ko + KTB;
            const unsigned a2 = last ? nA + k0 : cA + kn, b2 = last ? nB + k0 : cB + kn;
            const unsigned a3 = a2 + KTB, b3 = b2 + KTB;
            ko = kn;
            LDB(B0, 0, 0); SCHED; LDA(At, 0, 0); STAGE(SA(1, 1), a1 + HTB);
            WAIT_L(8); BAR; WAIT_L(0); MMA(0, 0, At, B0); BAR; SCHED;
            LDB(B1, 0, 1); STAGE(SB(0, 0), b2);
            BAR; WAIT_L(0); MMA(0, 1, At, B1); BAR;
            LDA(At, 0, 1); STAGE(SA(0, 0), a2);
            BAR; WAIT_L(0); MMA(1, 0, At, B0); BAR; SCHED;
            STAGE(SB(0, 1), b2 + HTB);
            WAIT_V(6); BAR; MMA(1, 1, At, B1); BAR;
            LDB(B0, 1, 0); SCHED; LDA(At, 1, 0); STAGE(SA(0, 1), a2 + HTB);
            WAIT_L(8); BAR; WAIT_L(0); MMA(0, 0, At, B0); BAR; SCHED;
            LDB(B1, 1, 1); STAGE(SB(1, 0), b3);
            BAR; WAIT_L(0); MMA(0, 1, At, B1); BAR;
            LDA(At, 1, 1); STAGE(SA(1, 0), a3);
            BAR; WAIT_L(0); MMA(1, 0, At, B0); BAR; SCHED;
            STAGE(SB(1, 1), b3 + HTB);
            WAIT_V(6); BAR; MMA(1, 1, At, B1); BAR;
        }
        {
            int tz = threadIdx.x; asm volatile("" : "+v"(tz));
            const int ln = tz & 63, fr = ln & 15, fq = ln >> 4;
            const rsrc_t rol = mk_rsrc(P.out[rank], OUTB);
            const bool bad = ctl[0] != 0u;
            const bool thru = cur.ord >= 0 || (CFG_LAST && !has_next);
            const unsigned vo = (unsigned)(wc * 32 + fq * 4) * (NN * 2) + (unsigned)(wr * 256 + fr * 16);
            const unsigned so = (unsigned)(cur.tm * 256) * (NN * 2) + (unsigned)cur.pan * 512u;
#pragma unroll
            for (int bj = 0; bj < 2; ++bj)
#pragma unroll
                for (int n = 0; n < 2; ++n)
#pragma unroll
                    for (int r = 0; r < 4; ++r) {
                        u32x4 o;
                        o[0] = cvt_pk_bf16(silu(acc[0][bj][0][n][r]), silu(acc[0][bj][1][n][r]));
                        o[1] = cvt_pk_bf16(silu(acc[0][bj][2][n][r]), silu(acc[0][bj][3][n][r]));
                        o[2] = cvt_pk_bf16(silu(acc[1][bj][0][n][r]), silu(acc[1][bj][1][n][r]));
                        o[3] = cvt_pk_bf16(silu(acc[1][bj][2][n][r]), silu(acc[1][bj][3][n][r]));
                        if (bad) o = (u32x4){0x7fc07fc0u, 0x7fc07fc0u, 0x7fc07fc0u, 0x7fc07fc0u};
                        if (thru) { __builtin_amdgcn_raw_buffer_store_b128(o, rol, vo, so + (unsigned)(bj * 128 + n * 16 + r) * (NN * 2), AUX_SYS);
                                    if (CFG_LAST > 1 && !has_next) { WAIT_VM0(); __builtin_amdgcn_s_sleep(CFG_LAST > 1 ? CFG_LAST - 1 : 1); } }
                        else __builtin_amdgcn_raw_buffer_store_b128(o, rol, vo, so + (unsigned)(bj * 128 + n * 16 + r) * (NN * 2), CFG_RST);
                    }
            if (cur.ord >= 0) {
                WAIT_VM0();
                unsigned old = 0;
                if (ln == 0) old = __hip_atomic_fetch_add(ctl + 1, 1u, __ATOMIC_RELAXED, __HIP_MEMORY_SCOPE_WORKGROUP);
                old = __builtin_amdgcn_readfirstlane(old);
                if ((old & 7u) == 7u && ln < NRANK && ln != rank)
                    st_sys((unsigned*)(P.ws[ln] + WS_FLAG + rank * FLAGB) + NIN * 4 + cur.ord, call);
            }
        }
        STAMP(P, 3 + (ui >= KX ? 1 : 0));
        if (!has_next) break;
        cur = nxt; cA = nA; cB = nB; ui += KX;
    }
    WAIT_V(0);
    if (wr == 0) BAR;
}

__global__ __launch_bounds__(512, 2) void fused_gemm_ar(Params P) {
    extern __shared__ __attribute__((aligned(16))) unsigned char shm[];
    LAS unsigned char* lds = (LAS unsigned char*)shm;
    const int wid = __builtin_amdgcn_readfirstlane(threadIdx.x >> 6), lane = threadIdx.x & 63;
    const int b = __builtin_amdgcn_readfirstlane(blockIdx.x);
    unsigned* tkp = (unsigned*)(P.ws[P.rank] + WS_TICKET + b * 128 + wid * 16);
    const unsigned tk = ld_agent(tkp);
    if (b < NPULL) {
        const unsigned call = __builtin_amdgcn_readfirstlane(tk) + 1u;
        st_agent(tkp, call);
        const int pi = b / PW, slot = b % PW;
        const int peer = (P.rank + 1 + pi) & (NRANK - 1);
        pull_role(P, call, peer, slot * 8 + wid, lane);
    } else {
        const int g = b - NPULL;
        ((LAS unsigned*)(lds + STAGE_BYTES))[lane & 3] = 0u;
        STAMP(P, 0);
        pack_role(P, tk, g, wid, lane);
        const unsigned call = __builtin_amdgcn_readfirstlane(tk) + 1u;
        st_agent(tkp, call);
        STAMP(P, 1);
        asm volatile("s_waitcnt lgkmcnt(0)" ::: "memory");
        __builtin_amdgcn_s_barrier();
        if (b >= NPS) gemm_role(P, call, b & 7, (b - NPS) >> 3, lds);
    }
}
}

extern "C" void kernel_launch(int rank, int nranks, ncclComm_t comm,
                              hipStream_t stream,
                              void* const* d_in, const int* in_sizes, int n_in,
                              void* d_out, int out_size,
                              const int* in_dtypes, int out_dtype,
                              void* const* d_out_all,
                              void* const* d_ws_all, size_t ws_bytes) {
    if (nranks != NRANK || (unsigned)rank >= (unsigned)NRANK || ws_bytes < (size_t)WS_END || n_in < 2 ||
        in_sizes[0] != MM * KSH || in_sizes[1] != KSH * NN || out_size != MM * NN) {
        fprintf(stderr, "unexpected problem geometry\n");
        return;
    }
    static bool attr_done[NRANK] = {};
    if (!attr_done[rank]) {
        HIP_CHECK(hipFuncSetAttribute((const void*)fused_gemm_ar, hipFuncAttributeMaxDynamicSharedMemorySize, LDS_BYTES));
        attr_done[rank] = true;
    }
    Params P;
    P.x = d_in[0]; P.w = d_in[1]; P.rank = rank;
    for (int p = 0; p < NRANK; ++p) { P.out[p] = (char*)d_out_all[p]; P.ws[p] = (char*)d_ws_all[p]; }
    hipLaunchKernelGGL(fused_gemm_ar, dim3(GRID), dim3(512), LDS_BYTES, stream, P);
    HIP_CHECK(hipGetLastError());
}
```

```cpp
#include <hip/hip_runtime.h>
#include <rccl/rccl.h>
#include <cstdio>

#define HIP_CHECK(expr) do { hipError_t _e = (expr); if (_e != hipSuccess) { \
    fprintf(stderr, "HIP error %s at %s:%d\n", hipGetErrorString(_e), __FILE__, __LINE__); \
    return; } } while (0)

#define LAS __attribute__((address_space(3)))
typedef short bf16x8 __attribute__((ext_vector_type(8)));
typedef float f32x4 __attribute__((ext_vector_type(4)));
typedef unsigned u32x4 __attribute__((ext_vector_type(4)));
typedef __amdgpu_buffer_rsrc_t rsrc_t;

namespace {
constexpr int NRANK = 8;
constexpr int MM = 4096, KK = 4096, NN = 32768, KSH = KK / NRANK;
constexpr int KT = KK / 64, KTS = KSH / 64, NTM = MM / 256, NPAN = NN / 256;
#define CFG_NS 8
#define CFG_PW 8
constexpr int NS = CFG_NS;
constexpr int NRP = NPAN - NRANK * NS;
constexpr int NWP = NS + NRP;
constexpr int NIN = NTM + NWP;
constexpr int NOUT = NS * NTM;
constexpr int NFLAG = NIN * 4 + NOUT;
constexpr int NUNIT = NTM * 4 + NPAN * 4;
constexpr int GRID = 256;
constexpr int PW = CFG_PW;
constexpr int NPULL = PW * (NRANK - 1);
constexpr int NW = PW * 8;
constexpr int NPK = GRID - NPULL;
constexpr int NPS = (NPULL + 7) / 8 * 8;
constexpr int KX = (GRID - NPS) / 8;
#define CFG_GW (2 * (KX / 4))
constexpr int GW = CFG_GW;
constexpr int G0 = GW + NTM;
constexpr int NGF = (NWP - GW) / GW, GL = (NWP - GW) % GW;
static_assert(NS <= GW && NWP > GW, "shared panels sit in the first group");
#define CFG_PU 4
constexpr int PU = CFG_PU, PSH = PU == 1 ? 0 : PU == 2 ? 1 : PU == 4 ? 2 : 3;
#define CFG_PUSH 2
constexpr int NPUSH = CFG_PUSH;
constexpr int NPIN = (NIN * 4 - NPUSH) * 64 / PU;
static_assert(NPUSH <= NS, "pushed units have a single consumer");
static_assert((GRID - NPS + 13) / 14 <= (NRANK - 1) * NS, "swap space: unused chunks of other ranks' shared panels");
constexpr int NPC = NPIN + NOUT * 128 / PU;
constexpr unsigned CNT_FULL = 64;

constexpr unsigned KTB = 32768, HTB = 16384;
constexpr unsigned PANB = KT * KTB;
constexpr unsigned CHB = KTS * KTB;
constexpr unsigned UNB = 2 * KTB;
constexpr unsigned WS_TICKET = 0;
constexpr unsigned WS_TMO = WS_TICKET + GRID * 128;
constexpr unsigned WS_FLAG = WS_TMO + 128;
constexpr unsigned FLAGB = 4096;
constexpr unsigned WS_DONE = WS_FLAG + NRANK * FLAGB;
constexpr unsigned WS_HEAD = WS_DONE + NRANK * 512;
constexpr unsigned WS_CNT = WS_HEAD + NRANK * 128;
constexpr unsigned WS_CTRL_END = WS_CNT + (NTM + NPAN) * NRANK * 128;
constexpr unsigned WS_DBG = 512u << 10;
constexpr unsigned WS_PAN = 1u << 20;
constexpr unsigned WS_END = WS_PAN + (NTM + NPAN) * PANB;
constexpr unsigned OUTB = (unsigned)MM * NN * 2;
static_assert(WS_CTRL_END <= WS_DBG, "ctrl region");
static_assert(NFLAG <= 512 && 2048 <= (int)FLAGB, "flag region");
static_assert(NW <= 128, "done words");
static_assert(PU == 4, "block size");
#define CFG_DEEP 2
constexpr unsigned STAGE_BYTES = 8 * HTB;
constexpr unsigned LDS_BYTES = STAGE_BYTES + 64;
constexpr unsigned long long DEADLINE = 50000000ull;
constexpr int AUX_SYS = 17, AUX_SYS_NT = 19, AUX_AGENT = 16;
#define CFG_PACE 40
#define CFG_LOOK 256
#define CFG_TAIL NW
#define CFG_SPLIT 1
#define CFG_LAST 8
#define CFG_RST 0

struct Params {
    const void* x; const void* w;
    char* out[NRANK]; char* ws[NRANK];
    int rank;
};

__device__ __forceinline__ rsrc_t mk_rsrc(const void* p, unsigned bytes) {
    const unsigned long long v = (unsigned long long)p;
    const unsigned lo = __builtin_amdgcn_readfirstlane((unsigned)v), hi = __builtin_amdgcn_readfirstlane((unsigned)(v >> 32));
    return __builtin_amdgcn_make_buffer_rsrc((void*)(((unsigned long long)hi << 32) | lo), 0, bytes, 0x00020000);
}
__device__ __forceinline__ unsigned ld_sys(const unsigned* p) { return __hip_atomic_load(p, __ATOMIC_RELAXED, __HIP_MEMORY_SCOPE_SYSTEM); }
__device__ __forceinline__ unsigned ld_agent(const unsigned* p) { return __hip_atomic_load(p, __ATOMIC_RELAXED, __HIP_MEMORY_SCOPE_AGENT); }
__device__ __forceinline__ void st_sys(unsigned* p, unsigned v) { __hip_atomic_store(p, v, __ATOMIC_RELAXED, __HIP_MEMORY_SCOPE_SYSTEM); }
__device__ __forceinline__ void st_agent(unsigned* p, unsigned v) { __hip_atomic_store(p, v, __ATOMIC_RELAXED, __HIP_MEMORY_SCOPE_AGENT); }
__device__ __forceinline__ void add_agent(unsigned* p, unsigned v) { (void)__hip_atomic_fetch_add(p, v, __ATOMIC_RELAXED, __HIP_MEMORY_SCOPE_AGENT); }
#define STORE16(v, rs, vo, so, aux) do { const u32x4 _d = (v); __builtin_amdgcn_raw_buffer_store_b128(_d, rs, vo, so, aux); asm volatile("s_nop 1" :: "v"(_d) : "memory"); } while (0)
#define WAIT_VM0() asm volatile("s_waitcnt vmcnt(0)" ::: "memory")
#define WAIT_VMN(n) do { asm volatile("" ::: "memory"); __builtin_amdgcn_s_waitcnt(0x0F70 | (n)); asm volatile("" ::: "memory"); } while (0)

__device__ __forceinline__ bool give_up(unsigned* tmo, unsigned call, unsigned long long t0) {
    const unsigned long long dt = __builtin_amdgcn_s_memrealtime() - t0;
    if (dt < 2000ull) return false;
    if ((unsigned)__builtin_amdgcn_readfirstlane(ld_agent(tmo)) == call) return true;
    if (dt > DEADLINE) { st_agent(tmo, call); return true; }
    return false;
}

__device__ __forceinline__ int gpan_of(int v, int consumer) { return v < NS ? consumer * NS + v : (NRANK - 1) * NS + v; }
__device__ __forceinline__ int unit_of(int o, int consumer, int& q) {
    if (o < 4 * G0) { q = o / G0; const int m = o % G0; return m < GW ? NTM + gpan_of(m, consumer) : m - GW; }
    o -= 4 * G0;
    const int g = o / (4 * GW), r = o % (4 * GW);
    int i;
    if (GL == 0 || g < NGF) { q = r / GW; i = r % GW; } else { q = r / (GL ? GL : 1); i = r % (GL ? GL : 1); }
    return NTM + (NRANK - 1) * NS + GW * (g + 1) + i;
}

__device__ __forceinline__ unsigned cvt_pk_bf16(float lo, float hi) { unsigned r; asm("v_cvt_pk_bf16_f32 %0, %1, %2" : "=v"(r) : "v"(lo), "v"(hi)); return r; }
__device__ __forceinline__ float silu(float y) {
    const float c = __builtin_amdgcn_fmed3f(y, -60.0f, 60.0f);
    const float e = __builtin_amdgcn_exp2f(c * -1.44269504088896f);
    return y * __builtin_amdgcn_rcpf(1.0f + e);
}

#define STAMP1(P, slot) do {} while (0)
#define STAMP(P, slot) do {} while (0)

__device__ __forceinline__ void pull_role(const Params& P, unsigned call, int peer, int widx, int lane) {
    const int rank = P.rank;
    char* wsl = P.ws[rank];
    const rsrc_t rwr = mk_rsrc(P.ws[peer], WS_END), rwl = mk_rsrc(wsl, WS_END);
    const rsrc_t ror = mk_rsrc(P.out[peer], OUTB), rol = mk_rsrc(P.out[rank], OUTB);
    unsigned* tmo = (unsigned*)(wsl + WS_TMO);
    constexpr unsigned OOB = 0xfffffff0u;
    const unsigned lane0 = lane == 0 ? 0u : OOB;
    const unsigned hbase = (call - 1u) * (unsigned)(NPC + NW);
    auto draw = [&]() { return (unsigned)__builtin_amdgcn_raw_ptr_buffer_atomic_add_i32(1, rwl, lane0, WS_HEAD + (unsigned)peer * 128u, 0); };
    auto arrive = [&](unsigned vo, unsigned cnt) { (void)__builtin_amdgcn_raw_ptr_buffer_atomic_add_i32(PU, rwl, vo, cnt, 0); };
    const bool deep0 = (widx & 7) < CFG_DEEP;
    bool bad = false; int stamped = 0; (void)stamped;
    STAMP(P, 0);
    auto index_of = [&](unsigned v) { return (int)((unsigned)__builtin_amdgcn_readfirstlane(v) - hbase); };
    int cur = index_of(draw());
    constexpr int BPB = NS * 128 / PU;
    const unsigned fso = WS_FLAG + (unsigned)peer * FLAGB;
    auto first_of = [&](int i) { return i < NPIN ? (i >> (6 - PSH)) + NPUSH : NIN * 4 + (i - NPIN) / BPB * NS; };
    auto limit_of = [&](const u32x4& f0, const u32x4& f1, int i) {
        const int first = first_of(i);
        unsigned un = 0;
#pragma unroll
        for (int j = 0; j < 8; ++j) un |= ((int)((j < 4 ? f0[j & 3] : f1[j & 3]) - call) < 0 && lane * 8 + j >= first ? 1u : 0u) << j;
        const int l = __builtin_ctzll(__ballot(un != 0u) | (1ull << 63));
        const int u = l * 8 + __builtin_ctz(__builtin_amdgcn_readlane(un, l) | 256u);
        return u <= NIN * 4 ? (u - NPUSH) << (6 - PSH) : u >= NFLAG ? NPC : NPIN + (u - NIN * 4) / NS * BPB;
    };
    auto offs_of = [&](int i, unsigned (&o)[4], unsigned& cnt) {
        if (i < NPIN) {
            int q; const int pan = unit_of((i >> (6 - PSH)) + NPUSH, rank, q);
            cnt = WS_CNT + (unsigned)(pan * NRANK + peer) * 128u + (unsigned)q * 4u;
            const unsigned b = WS_PAN + (unsigned)pan * PANB + (unsigned)peer * CHB + (unsigned)q * UNB + (unsigned)((i << PSH) & 63) * 1024u + (unsigned)lane * 16u;
#pragma unroll
            for (int k = 0; k < 4; ++k) o[k] = b + k * 1024u;
        } else {
            cnt = 0;
#pragma unroll
            for (int k = 0; k < 4; ++k) {
                const unsigned sg = (unsigned)(((i - NPIN) << PSH) + k) * 2u + (unsigned)(lane >> 5);
                o[k] = (unsigned)(peer * NS) * 512u + (unsigned)(lane & 31) * 16u + sg / NS * (NN * 2u) + sg % NS * 512u;
            }
        }
    };
    int lim = 0;
    while ((unsigned)cur < (unsigned)NPC) {
        if (!bad) {
            const unsigned long long t0 = __builtin_amdgcn_s_memrealtime();
            const unsigned pvo = (unsigned)(lane - (first_of(cur) >> 3)) < 4u ? (unsigned)lane * 32u : OOB - 16u;
            for (;;) {
                const u32x4 f0 = __builtin_amdgcn_raw_buffer_load_b128(rwl, pvo, fso, AUX_SYS), f1 = __builtin_amdgcn_raw_buffer_load_b128(rwl, pvo + 16u, fso, AUX_SYS);
                lim = limit_of(f0, f1, cur);
                if (cur < lim) break;
                if (give_up(tmo, call, t0)) { bad = true; break; }
                __builtin_amdgcn_s_sleep(2);
            }
            asm volatile("" ::: "memory");
        }
        unsigned o[4], p2 = OOB, p3 = OOB, va = OOB, cprev = 0;
        if (bad) {
            offs_of(cur, o, cprev);
            if (cur >= NPIN) {
#pragma unroll
                for (int k = 0; k < PU; ++k) __builtin_amdgcn_raw_buffer_store_b128((u32x4){0x7fc07fc0u, 0x7fc07fc0u, 0x7fc07fc0u, 0x7fc07fc0u}, rol, o[k], 0, AUX_AGENT);
            }
            cur = index_of(draw()); continue;
        }
        STAMP1(P, 1);
        rsrc_t rp = rwl;
        u32x4 v[4] = {};
        do {
            unsigned cb; offs_of(cur, o, cb);
            const bool in = cur < NPIN, deep = deep0 || cur >= NPC - CFG_TAIL;
            const rsrc_t rs = in ? rwr : ror, rd = in ? rwl : rol;
            const bool look = lim < NPC && cur + CFG_LOOK >= lim;
            const unsigned fvo = look ? (unsigned)lane * 32u : OOB - 16u;
            unsigned nv = 0; u32x4 f0, f1;
#pragma unroll
            for (int k = 0; k < 4; ++k) {
                v[k] = __builtin_amdgcn_raw_buffer_load_b128(rs, o[k], 0, AUX_SYS_NT);
                if (k == 0) {
                    nv = draw();
                    f0 = __builtin_amdgcn_raw_buffer_load_b128(rwl, fvo, fso, AUX_SYS); f1 = __builtin_amdgcn_raw_buffer_load_b128(rwl, fvo + 16u, fso, AUX_SYS);
                }
                __builtin_amdgcn_raw_buffer_store_b128(v[k ^ 2], k < 2 ? rp : rd, k == 0 ? p2 : k == 1 ? p3 : o[k ^ 2], 0, AUX_AGENT);
                if (k == 0) WAIT_VMN(7); else if (k == 1) WAIT_VMN(6); else WAIT_VMN(3);
                if (!deep) { if (k == 0) WAIT_VMN(4); else WAIT_VMN(1); }
            }
            arrive(va, cprev);
            cprev = cb; va = in ? lane0 : OOB; rp = rd; p2 = o[2]; p3 = o[3];
            cur = index_of(nv);
            if (look) { const int l2 = limit_of(f0, f1, cur); lim = l2 > lim ? l2 : lim; }
        } while ((unsigned)cur < (unsigned)lim);
        __builtin_amdgcn_raw_buffer_store_b128(v[2], rp, p2, 0, AUX_AGENT);
        __builtin_amdgcn_raw_buffer_store_b128(v[3], rp, p3, 0, AUX_AGENT);
        if (cprev) { WAIT_VM0(); arrive(va, cprev); }
    }
    STAMP(P, 4);
    asm volatile("" ::: "memory");
    if (lane == 0) st_sys((unsigned*)(P.ws[peer] + WS_DONE + rank * 512) + widx, call);
    if (widx == 0) {
        const unsigned* dn = (const unsigned*)(wsl + WS_DONE + peer * 512);
        const unsigned long long t0 = __builtin_amdgcn_s_memrealtime();
        for (;;) {
            bool ok = true;
            if (lane < NW) ok = (int)(ld_sys(dn + lane) - call) >= 0;
            if (NW > 64 && lane + 64 < NW) ok = ok && (int)(ld_sys(dn + 64 + lane) - call) >= 0;
            if (__all(ok)) break;
            if (give_up(tmo, call, t0)) {
                if (lane == 0) __builtin_amdgcn_raw_buffer_store_b32(0x7fc07fc0u, rol, (unsigned)(rank * NS) * 512u, 0, AUX_SYS);
                break;
            }
            __builtin_amdgcn_s_sleep(2);
        }
        STAMP(P, 5);
    }
}

__device__ __forceinline__ void pack_role(const Params& P, unsigned tk, int g, int wid, int lane) {
    const int rank = P.rank;
    char* wsl = P.ws[rank];
    const rsrc_t rx = mk_rsrc(P.x, (unsigned)MM * KSH * 2), rw = mk_rsrc(P.w, (unsigned)KSH * NN * 2);
    const int fr = lane & 15, fq = lane >> 4;
    constexpr int U0 = G0 + (NRANK - 1) * NS;
    const unsigned long long t0 = __builtin_amdgcn_s_memrealtime();
    for (int u = g; u < NUNIT; u += NPK) {
        int o, dest = -1;
        if (u < 4 * U0) {
            const int r = u % U0;
            if (r < NRANK * NS) { o = r >> 3; dest = (rank + 1 + r) & (NRANK - 1); } else o = r - (NRANK - 1) * NS;
            o += u / U0 * G0;
        } else o = u - 4 * (NRANK - 1) * NS;
        int q; const int pan = unit_of(o, dest < 0 ? 0 : dest, q);
        const bool push = o < NPUSH;
        const rsrc_t rwl = mk_rsrc(P.ws[push ? dest & (NRANK - 1) : rank], WS_END);
        while (CFG_PACE > 0 && (unsigned)(__builtin_amdgcn_s_memrealtime() - t0) < (unsigned)(u * CFG_PACE / 10)) __builtin_amdgcn_s_sleep(1);
        const int ktl = q * 2 + (wid >> 2);
        const unsigned dst = WS_PAN + (unsigned)pan * PANB + (unsigned)(rank * KTS + ktl) * KTB;
        u32x4 v[8];
        if (pan < NTM) {
            const int rb0 = (wid & 3) * 4;
            const unsigned vo = (unsigned)fr * (KSH * 2) + (unsigned)fq * 16u;
            const unsigned so = (unsigned)(pan * 256 + rb0 * 16) * (KSH * 2) + (unsigned)ktl * 128u;
#pragma unroll
            for (int i = 0; i < 8; ++i) v[i] = __builtin_amdgcn_raw_buffer_load_b128(rx, vo + (i & 1) * 64, so + (unsigned)(i >> 1) * (16 * KSH * 2), 0);
#pragma unroll
            for (int i = 0; i < 8; ++i) STORE16(v[i], rwl, lane * 16 + (i & 1) * 1024, dst + (unsigned)(rb0 + (i >> 1)) * 2048u, AUX_SYS);
        } else {
            const int ks = (wid >> 1) & 1, wr = wid & 1;
            const unsigned vo = (unsigned)(fq * 8) * (NN * 2) + (unsigned)fr * 16u;
            const unsigned so = (unsigned)(ktl * 64 + ks * 32) * (NN * 2) + (unsigned)((pan - NTM) * 256 + wr * 128) * 2u;
#pragma unroll
            for (int j = 0; j < 8; ++j) v[j] = __builtin_amdgcn_raw_buffer_load_b128(rw, vo, so + (unsigned)j * (NN * 2), 2);
#pragma unroll
            for (int i = 0; i < 8; ++i) {
                u32x4 o;
#pragma unroll
                for (int e = 0; e < 4; ++e) o[e] = __builtin_amdgcn_perm(v[2 * e + 1][i >> 1], v[2 * e][i >> 1], (i & 1) ? 0x07060302u : 0x05040100u);
                STORE16(o, rwl, lane * 16, dst + (unsigned)(i >> 2) * HTB + (unsigned)wr * 8192u + (unsigned)(i & 3) * 2048u + (unsigned)ks * 1024u, AUX_SYS);
            }
        }
        WAIT_VM0();
        __builtin_amdgcn_s_barrier();
        if (wid == 0 && lane < NRANK && (dest < 0 || dest == lane)) {
            unsigned c = tk; asm volatile("" : "+v"(c));
            if (lane == rank || push) st_sys((unsigned*)(P.ws[lane] + WS_CNT) + (pan * NRANK + rank) * 32 + q, (c + 1u) * CNT_FULL);
            else st_sys((unsigned*)(P.ws[lane] + WS_FLAG + rank * FLAGB) + o, c + 1u);
        }
    }
}

struct Tile { int tm, pan, ord, half; };
__device__ __forceinline__ bool tile_of(int first, int slot, int col, int rank, Tile& t) {
    const int nt = 4 * ((NWP + 1 - (col >> 2)) >> 1);
    const bool split = CFG_SPLIT && 2 * (nt - first) <= KX;
    const int idx = first + (split ? slot >> 1 : slot);
    const int v = 2 * (idx >> 2) + (col >> 2);
    if (idx >= nt) return false;
    t.tm = 4 * (col & 3) + (idx & 3); t.pan = gpan_of(v, rank); t.ord = v < NS ? t.tm * NS + v : -1; t.half = split ? slot & 1 : -1;
    return true;
}

__device__ __forceinline__ void gemm_role(const Params& P, unsigned call, int col, int slot, LAS unsigned char* lds) {
    const int tid = threadIdx.x, wid = __builtin_amdgcn_readfirstlane(tid >> 6), lane = tid & 63, wr = wid >> 2, wc = wid & 3;
    const int rank = P.rank;
    char* wsl = P.ws[rank];
    const rsrc_t rws = mk_rsrc(wsl, WS_END);
    LAS unsigned* ctl = (LAS unsigned*)(lds + STAGE_BYTES);
    const unsigned voff = (unsigned)tid * 16u;
    const unsigned ldsw = (unsigned)wid * 1024u;
    const int aoff = wr * 8192 + lane * 16, boff = wc * 4096 + lane * 16;

    auto ready = [&](const Tile& t, int want) -> int {
        if (wid == 0) {
            const unsigned* c = (const unsigned*)(wsl + WS_CNT) + ((((lane & 32) ? NTM + t.pan : t.tm) * NRANK + ((lane >> 2) & 7)) * 32 + (lane & 3));
            unsigned* tmo = (unsigned*)(wsl + WS_TMO);
            const unsigned need = call * CNT_FULL;
            const unsigned long long t0 = __builtin_amdgcn_s_memrealtime();
            int got;
            for (;;) {
                const unsigned long long miss = __ballot((int)(ld_sys(c) - need) < 0);
                unsigned m = (unsigned)miss | (unsigned)(miss >> 32);
                const bool own = ((m >> (rank * 4)) & 15u) == 0u;
                m |= m >> 16; m |= m >> 8; m |= m >> 4;
                got = own ? 1 + __builtin_ctz(m | 16u) : 0;
                if (got >= want) break;
                if (give_up(tmo, call, t0)) { ctl[0] = 1u; got = 5; break; }
                __builtin_amdgcn_s_sleep(4);
            }
            ctl[2] = (unsigned)got;
            __builtin_amdgcn_fence(__ATOMIC_ACQUIRE, "agent");
            asm volatile("s_waitcnt vmcnt(0) lgkmcnt(0)" ::: "memory");
        }
        asm volatile("" ::: "memory"); __builtin_amdgcn_s_barrier(); asm volatile("" ::: "memory");
        const int r = __builtin_amdgcn_readfirstlane(ctl[2]);
        asm volatile("s_waitcnt lgkmcnt(0)" ::: "memory");
        return r;
    };

#define SA(b, h) (((b) * 2 + (h)) * HTB)
#define SB(b, h) ((4 + (b) * 2 + (h)) * HTB)
#define STAGE(bufoff, soff) do { _Pragma("unroll") for (int _i = 0; _i < 2; ++_i) \
        __builtin_amdgcn_raw_ptr_buffer_load_lds(rws, (LAS unsigned*)(lds + (bufoff) + ldsw + _i * 8192), 16, voff, (soff) + _i * 8192u, 0, 0); } while (0)
#define LDA(dst, b, h) do { _Pragma("unroll") for (int m = 0; m < 4; ++m) _Pragma("unroll") for (int k = 0; k < 2; ++k) dst[m][k] = *(const LAS bf16x8*)(lds + SA(b, h) + aoff + m * 2048 + k * 1024); } while (0)
#define LDB(dst, b, h) do { _Pragma("unroll") for (int n = 0; n < 2; ++n) _Pragma("unroll") for (int k = 0; k < 2; ++k) dst[n][k] = *(const LAS bf16x8*)(lds + SB(b, h) + boff + n * 2048 + k * 1024); } while (0)
#define MMA(ai, bj, At, Bt) do { __builtin_amdgcn_s_setprio(1); _Pragma("unroll") for (int m = 0; m < 4; ++m) _Pragma("unroll") for (int n = 0; n < 2; ++n) _Pragma("unroll") for (int k = 0; k < 2; ++k) \
        acc[ai][bj][m][n] = __builtin_amdgcn_mfma_f32_16x16x32_bf16(Bt[n][k], At[m][k], acc[ai][bj][m][n], 0, 0, 0); __builtin_amdgcn_s_setprio(0); } while (0)
#define WAIT_V(n) asm volatile("s_waitcnt vmcnt(" #n ")" ::: "memory")
#define WAIT_L(n) asm volatile("s_waitcnt lgkmcnt(" #n ")" ::: "memory")
#define BAR __builtin_amdgcn_s_barrier()
#define SCHED __builtin_amdgcn_sched_barrier(0)

    Tile cur, nxt; int ui = 0;
    if (!tile_of(ui, slot, col, rank, cur)) return;
    bf16x8 At[4][2], B0[2][2], B1[2][2];
    unsigned cA = WS_PAN + (unsigned)(NTM + cur.pan) * PANB, cB = WS_PAN + (unsigned)cur.tm * PANB;
    const unsigned k0 = (unsigned)rank * CHB, k16 = (unsigned)((rank + 6) & (NRANK - 1)) * CHB + UNB;
    int lvl = ready(cur, 1);
    STAMP(P, 2);
    STAGE(SB(0, 0), cB + k0); STAGE(SA(0, 0), cA + k0); STAGE(SB(0, 1), cB + k0 + HTB); STAGE(SA(0, 1), cA + k0 + HTB);
    if (wr == 1) BAR;
    WAIT_V(4); BAR;
    STAGE(SB(1, 0), cB + k0 + KTB); STAGE(SA(1, 0), cA + k0 + KTB); STAGE(SB(1, 1), cB + k0 + HTB + KTB);
    WAIT_V(6); BAR;
#pragma unroll 1
    for (;;) {
        f32x4 acc[2][2][4][2];
#pragma unroll
        for (int a = 0; a < 2; ++a)
#pragma unroll
            for (int b = 0; b < 2; ++b)
#pragma unroll
                for (int m = 0; m < 4; ++m)
#pragma unroll
                    for (int n = 0; n < 2; ++n) acc[a][b][m][n] = (f32x4){0.f, 0.f, 0.f, 0.f};
        const bool has_next = tile_of(ui + KX, slot, col, rank, nxt);
        const unsigned nA = has_next ? WS_PAN + (unsigned)(NTM + nxt.pan) * PANB : cA, nB = has_next ? WS_PAN + (unsigned)nxt.tm * PANB : cB;
        const unsigned nk = has_next && nxt.half == 1 ? k16 : k0;
        unsigned ko = cur.half == 1 ? k16 : k0;
        const int jend = cur.half == 0 ? KT / 4 : KT / 2;
        int jchk = lvl >= 5 ? 31 : 7 * lvl - 4;
        for (int j = cur.half == 1 ? KT / 4 : 0; j < jend; ++j) {
            const bool last = (j == jend - 1);
            if (j == jchk && (has_next || !last)) {
                lvl = ready(last ? nxt : cur, last ? (nxt.half >= 0 ? 5 : 1) : lvl + 1);
                jchk = lvl >= 5 ? 31 : 7 * lvl - 4;
            }
            unsigned kn;
            if (j < 3) kn = ko + UNB;
            else {
                kn = ((j == 3 ? k0 : ko) + CHB) & (PANB - 1);
                if ((kn ^ k0) < CHB) kn = (kn + CHB + UNB) & (PANB - 1);
            }
            const unsigned a1 = cA + ko + KTB;
            const unsigned a2 = last ? nA + nk : cA + kn, b2 = last ? nB + nk : cB + kn;
            const unsigned a3 = a2 + KTB, b3 = b2 + KTB;
            ko = kn;
            LDB(B0, 0, 0); SCHED; LDA(At, 0, 0); STAGE(SA(1, 1), a1 + HTB);
            WAIT_L(8); BAR; WAIT_L(0); MMA(0, 0, At, B0); BAR; SCHED;
            LDB(B1, 0, 1); STAGE(SB(0, 0), b2);
            BAR; WAIT_L(0); MMA(0, 1, At, B1); BAR;
            LDA(At, 0, 1); STAGE(SA(0, 0), a2);
            BAR; WAIT_L(0); MMA(1, 0, At, B0); BAR; SCHED;
            STAGE(SB(0, 1), b2 + HTB);
            WAIT_V(6); BAR; MMA(1, 1, At, B1); BAR;
            LDB(B0, 1, 0); SCHED; LDA(At, 1, 0); STAGE(SA(0, 1), a2 + HTB);
            WAIT_L(8); BAR; WAIT_L(0); MMA(0, 0, At, B0); BAR; SCHED;
            LDB(B1, 1, 1); STAGE(SB(1, 0), b3);
            BAR; WAIT_L(0); MMA(0, 1, At, B1); BAR;
            LDA(At, 1, 1); STAGE(SA(1, 0), a3);
            BAR; WAIT_L(0); MMA(1, 0, At, B0); BAR; SCHED;
            STAGE(SB(1, 1), b3 + HTB);
            WAIT_V(6); BAR; MMA(1, 1, At, B1);
            if (!(last && wr == 1)) BAR;
        }
        {
            int tz = threadIdx.x; asm volatile("" : "+v"(tz));
            const int ln = tz & 63, fr = ln & 15, fq = ln >> 4;
            const rsrc_t rol = mk_rsrc(P.out[rank], OUTB);
            bool bad = ctl[0] != 0u;
            if (CFG_SPLIT && cur.half >= 0) {
                auto xoff = [&](int e) { return WS_PAN + (unsigned)(NTM + ((rank + 1) * NS + e / 14) % (NRANK * NS)) * PANB
                                                + (unsigned)((rank + 1 + e % 14 / 2) & (NRANK - 1)) * CHB + (unsigned)(e & 1) * (CHB / 2) + (unsigned)wid * 16384u; };
                int me = slot * 8 + col; asm volatile("" : "+s"(me));
                const int other = me ^ 8;
                const unsigned xs = xoff(me), xr = xoff(other);
#pragma unroll
                for (int bj = 0; bj < 2; ++bj) if (bj != cur.half) {
#pragma unroll
                    for (int i = 0; i < 16; ++i) STORE16(__builtin_bit_cast(u32x4, acc[i >> 3][bj][(i >> 1) & 3][i & 1]), rws, ln * 16, xs + i * 1024u, AUX_AGENT);
                }
                WAIT_VM0();
                unsigned* xf = (unsigned*)(wsl + WS_TICKET + wid * 16 + 4);
                if (ln == 0) st_agent(xf + (NPS + me) * 32, call);
                const unsigned long long t0 = __builtin_amdgcn_s_memrealtime();
                while ((unsigned)__builtin_amdgcn_readfirstlane(ld_agent(xf + (NPS + other) * 32)) != call) {
                    if (give_up((unsigned*)(wsl + WS_TMO), call, t0)) { bad = true; break; }
                    __builtin_amdgcn_s_sleep(2);
                }
                asm volatile("" ::: "memory");
#pragma unroll
                for (int bj = 0; bj < 2; ++bj) if (bj == cur.half) {
#pragma unroll
                    for (int i = 0; i < 16; ++i) acc[i >> 3][bj][(i >> 1) & 3][i & 1] += __builtin_bit_cast(f32x4, __builtin_amdgcn_raw_buffer_load_b128(rws, ln * 16, xr + i * 1024u, AUX_AGENT));
                }
            }
            const bool thru = cur.ord >= 0 || (CFG_LAST && !has_next);
            const unsigned vo = (unsigned)(wc * 32 + fq * 4) * (NN * 2) + (unsigned)(wr * 256 + fr * 16);
            const unsigned so = (unsigned)(cur.tm * 256) * (NN * 2) + (unsigned)cur.pan * 512u;
#pragma unroll
            for (int bj = 0; bj < 2; ++bj) if (cur.half != 1 - bj)
#pragma unroll
                for (int n = 0; n < 2; ++n)
#pragma unroll
                    for (int r = 0; r < 4; ++r) {
                        u32x4 o;
                        o[0] = cvt_pk_bf16(silu(acc[0][bj][0][n][r]), silu(acc[0][bj][1][n][r]));
                        o[1] = cvt_pk_bf16(silu(acc[0][bj][2][n][r]), silu(acc[0][bj][3][n][r]));
                        o[2] = cvt_pk_bf16(silu(acc[1][bj][0][n][r]), silu(acc[1][bj][1][n][r]));
                        o[3] = cvt_pk_bf16(silu(acc[1][bj][2][n][r]), silu(acc[1][bj][3][n][r]));
                        if (bad) o = (u32x4){0x7fc07fc0u, 0x7fc07fc0u, 0x7fc07fc0u, 0x7fc07fc0u};
                        if (thru) { STORE16(o, rol, vo, so + (unsigned)(bj * 128 + n * 16 + r) * (NN * 2), AUX_SYS);
                                    if (CFG_LAST > 1 && !has_next) { WAIT_VM0(); __builtin_amdgcn_s_sleep(CFG_LAST > 1 ? CFG_LAST - 1 : 1); } }
                        else STORE16(o, rol, vo, so + (unsigned)(bj * 128 + n * 16 + r) * (NN * 2), CFG_RST);
                    }
            if (cur.ord >= 0) {
                WAIT_VM0();
                unsigned old = 0;
                if (ln == 0) old = __hip_atomic_fetch_add(ctl + 1, 1u, __ATOMIC_RELAXED, __HIP_MEMORY_SCOPE_WORKGROUP);
                old = __builtin_amdgcn_readfirstlane(old);
                if ((old & 7u) == 7u && ln < NRANK && ln != rank)
                    st_sys((unsigned*)(P.ws[ln] + WS_FLAG + rank * FLAGB) + NIN * 4 + cur.ord, call);
            }
        }
        STAMP(P, 3 + (ui >= KX ? 1 : 0));
        if (!has_next) break;
        if (wr == 1) BAR;
        cur = nxt; cA = nA; cB = nB; ui += KX;
    }
    WAIT_V(0);
}

__global__ __launch_bounds__(512, 2) void fused_gemm_ar(Params P) {
    extern __shared__ __attribute__((aligned(16))) unsigned char shm[];
    LAS unsigned char* lds = (LAS unsigned char*)shm;
    const int wid = __builtin_amdgcn_readfirstlane(threadIdx.x >> 6), lane = threadIdx.x & 63;
    const int b = __builtin_amdgcn_readfirstlane(blockIdx.x);
    unsigned* tkp = (unsigned*)(P.ws[P.rank] + WS_TICKET + b * 128 + wid * 16);
    const unsigned tk = ld_agent(tkp);
    if (b < NPULL) {
        const unsigned call = __builtin_amdgcn_readfirstlane(tk) + 1u;
        st_agent(tkp, call);
        const int pi = b / PW, slot = b % PW;
        const int peer = (P.rank + 1 + pi) & (NRANK - 1);
        pull_role(P, call, peer, slot * 8 + wid, lane);
    } else {
        const int g = b - NPULL;
        ((LAS unsigned*)(lds + STAGE_BYTES))[lane & 3] = 0u;
        STAMP(P, 0);
        pack_role(P, tk, g, wid, lane);
        const unsigned call = __builtin_amdgcn_readfirstlane(tk) + 1u;
        st_agent(tkp, call);
        STAMP(P, 1);
        asm volatile("s_waitcnt lgkmcnt(0)" ::: "memory");
        __builtin_amdgcn_s_barrier();
        if (b >= NPS) gemm_role(P, call, b & 7, (b - NPS) >> 3, lds);
    }
}
}

extern "C" void kernel_launch(int rank, int nranks, ncclComm_t comm,
                              hipStream_t stream,
                              void* const* d_in, const int* in_sizes, int n_in,
                              void* d_out, int out_size,
                              const int* in_dtypes, int out_dtype,
                              void* const* d_out_all,
                              void* const* d_ws_all, size_t ws_bytes) {
    if (nranks != NRANK || (unsigned)rank >= (unsigned)NRANK || ws_bytes < (size_t)WS_END || n_in < 2 ||
        in_sizes[0] != MM * KSH || in_sizes[1] != KSH * NN || out_size != MM * NN) {
        fprintf(stderr, "unexpected problem geometry\n");
        return;
    }
    static bool attr_done[NRANK] = {};
    if (!attr_done[rank]) {
        HIP_CHECK(hipFuncSetAttribute((const void*)fused_gemm_ar, hipFuncAttributeMaxDynamicSharedMemorySize, LDS_BYTES));
        attr_done[rank] = true;
    }
    Params P;
    P.x = d_in[0]; P.w = d_in[1]; P.rank = rank;
    for (int p = 0; p < NRANK; ++p) { P.out[p] = (char*)d_out_all[p]; P.ws[p] = (char*)d_ws_all[p]; }
    hipLaunchKernelGGL(fused_gemm_ar, dim3(GRID), dim3(512), LDS_BYTES, stream, P);
    HIP_CHECK(hipGetLastError());
}
```

```cpp
#include <hip/hip_runtime.h>
#include <rccl/rccl.h>
#include <cstdio>

#define HIP_CHECK(expr) do { hipError_t _e = (expr); if (_e != hipSuccess) { \
    fprintf(stderr, "HIP error %s at %s:%d\n", hipGetErrorString(_e), __FILE__, __LINE__); \
    return; } } while (0)

#define LAS __attribute__((address_space(3)))
typedef short bf16x8 __attribute__((ext_vector_type(8)));
typedef float f32x4 __attribute__((ext_vector_type(4)));
typedef unsigned u32x4 __attribute__((ext_vector_type(4)));
typedef __amdgpu_buffer_rsrc_t rsrc_t;

namespace {
constexpr int NRANK = 8;
constexpr int MM = 4096, KK = 4096, NN = 32768, KSH = KK / NRANK;
constexpr int KT = KK / 64, KTS = KSH / 64, NTM = MM / 256, NPAN = NN / 256;
#define CFG_NS 8
#define CFG_PW 8
constexpr int NS = CFG_NS;
constexpr int NRP = NPAN - NRANK * NS;
constexpr int NWP = NS + NRP;
constexpr int NIN = NTM + NWP;
constexpr int NOUT = NS * NTM;
constexpr int NFLAG = NIN * 4 + NOUT;
constexpr int NUNIT = NTM * 4 + NPAN * 4;
constexpr int GRID = 256;
constexpr int PW = CFG_PW;
constexpr int NPULL = PW * (NRANK - 1);
constexpr int NW = PW * 8;
constexpr int NPK = GRID - NPULL;
constexpr int NPS = (NPULL + 7) / 8 * 8;
constexpr int KX = (GRID - NPS) / 8;
#define CFG_GW (2 * (KX / 4))
constexpr int GW = CFG_GW;
constexpr int G0 = GW + NTM;
constexpr int NGF = (NWP - GW) / GW, GL = (NWP - GW) % GW;
static_assert(NS <= GW && NWP > GW, "shared panels sit in the first group");
#define CFG_PU 4
constexpr int PU = CFG_PU, PSH = PU == 1 ? 0 : PU == 2 ? 1 : PU == 4 ? 2 : 3;
#define CFG_PUSH 2
constexpr int NPUSH = CFG_PUSH;
constexpr int NPIN = (NIN * 4 - NPUSH) * 64 / PU;
static_assert(NPUSH <= NS, "pushed units have a single consumer");
constexpr int NPC = NPIN + NOUT * 128 / PU;
constexpr unsigned CNT_FULL = 64;

constexpr unsigned KTB = 32768, HTB = 16384;
constexpr unsigned PANB = KT * KTB;
constexpr unsigned CHB = KTS * KTB;
constexpr unsigned UNB = 2 * KTB;
constexpr unsigned WS_TICKET = 0;
constexpr unsigned WS_TMO = WS_TICKET + GRID * 128;
constexpr unsigned WS_FLAG = WS_TMO + 128;
constexpr unsigned FLAGB = 4096;
constexpr unsigned WS_DONE = WS_FLAG + NRANK * FLAGB;
constexpr unsigned WS_HEAD = WS_DONE + NRANK * 512;
constexpr unsigned WS_CNT = WS_HEAD + NRANK * 128;
constexpr unsigned WS_CTRL_END = WS_CNT + NIN * NRANK * 128;
constexpr unsigned WS_DBG = 512u << 10;
constexpr unsigned WS_SND = 1u << 20;
constexpr unsigned ROWB = NS * 512;
constexpr unsigned WS_OSN = WS_SND + NUNIT * UNB;
constexpr unsigned WS_PAN = (WS_OSN + MM * ROWB + PANB - 1) / PANB * PANB;
constexpr unsigned WS_END = WS_PAN + NIN * PANB;
constexpr int U0 = G0 + (NRANK - 1) * NS;
constexpr unsigned OUTB = (unsigned)MM * NN * 2;
static_assert(WS_CTRL_END <= WS_DBG, "ctrl region");
static_assert(NFLAG <= 512 && 2048 <= (int)FLAGB, "flag region");
static_assert(NW <= 128, "done words");
static_assert(PU == 4, "block size");
#define CFG_DEEP 3
constexpr unsigned STAGE_BYTES = 8 * HTB;
constexpr unsigned LDS_BYTES = STAGE_BYTES + 64;
constexpr unsigned long long DEADLINE = 50000000ull;
constexpr int AUX_SYS = 17, AUX_SYS_NT = 19, AUX_AGENT = 16;
#define CFG_PACE 40
#define CFG_LOOK 256
#define CFG_TAIL NW
#define CFG_LAST 8
#define CFG_RST 0
#define CFG_RATE 1725
#define CFG_BURST 1
constexpr unsigned PACE16 = CFG_RATE * PW * (8 + CFG_DEEP) / 64;

struct Params {
    const void* x; const void* w;
    char* out[NRANK]; char* ws[NRANK];
    int rank;
};

__device__ __forceinline__ rsrc_t mk_rsrc(const void* p, unsigned bytes) {
    const unsigned long long v = (unsigned long long)p;
    const unsigned lo = __builtin_amdgcn_readfirstlane((unsigned)v), hi = __builtin_amdgcn_readfirstlane((unsigned)(v >> 32));
    return __builtin_amdgcn_make_buffer_rsrc((void*)(((unsigned long long)hi << 32) | lo), 0, bytes, 0x00020000);
}
__device__ __forceinline__ unsigned ld_sys(const unsigned* p) { return __hip_atomic_load(p, __ATOMIC_RELAXED, __HIP_MEMORY_SCOPE_SYSTEM); }
__device__ __forceinline__ unsigned ld_agent(const unsigned* p) { return __hip_atomic_load(p, __ATOMIC_RELAXED, __HIP_MEMORY_SCOPE_AGENT); }
__device__ __forceinline__ void st_sys(unsigned* p, unsigned v) { __hip_atomic_store(p, v, __ATOMIC_RELAXED, __HIP_MEMORY_SCOPE_SYSTEM); }
__device__ __forceinline__ void st_agent(unsigned* p, unsigned v) { __hip_atomic_store(p, v, __ATOMIC_RELAXED, __HIP_MEMORY_SCOPE_AGENT); }
__device__ __forceinline__ void add_agent(unsigned* p, unsigned v) { (void)__hip_atomic_fetch_add(p, v, __ATOMIC_RELAXED, __HIP_MEMORY_SCOPE_AGENT); }
#define STORE16(v, rs, vo, so, aux) do { const u32x4 _d = (v); __builtin_amdgcn_raw_buffer_store_b128(_d, rs, vo, so, aux); asm volatile("s_nop 1" :: "v"(_d) : "memory"); } while (0)
#define WAIT_VM0() asm volatile("s_waitcnt vmcnt(0)" ::: "memory")
#define WAIT_VMN(n) do { asm volatile("" ::: "memory"); __builtin_amdgcn_s_waitcnt(0x0F70 | (n)); asm volatile("" ::: "memory"); } while (0)

__device__ __forceinline__ bool give_up(unsigned* tmo, unsigned call, unsigned long long t0) {
    const unsigned long long dt = __builtin_amdgcn_s_memrealtime() - t0;
    if (dt < 2000ull) return false;
    if ((unsigned)__builtin_amdgcn_readfirstlane(ld_agent(tmo)) == call) return true;
    if (dt > DEADLINE) { st_agent(tmo, call); return true; }
    return false;
}

__device__ __forceinline__ int gpan_of(int v, int consumer) { return v < NS ? consumer * NS + v : (NRANK - 1) * NS + v; }
__device__ __forceinline__ int unit_of(int o, int consumer, int& q, int& pos) {
    if (o < 4 * G0) { q = o / G0; const int m = o % G0; pos = q * U0 + gpan_of(m, consumer); return m < GW ? NTM + m : m - GW; }
    pos = o + 4 * (NRANK - 1) * NS;
    o -= 4 * G0;
    const int g = o / (4 * GW), r = o % (4 * GW);
    int i;
    if (GL == 0 || g < NGF) { q = r / GW; i = r % GW; } else { q = r / (GL ? GL : 1); i = r % (GL ? GL : 1); }
    return NTM + GW * (g + 1) + i;
}

__device__ __forceinline__ unsigned cvt_pk_bf16(float lo, float hi) { unsigned r; asm("v_cvt_pk_bf16_f32 %0, %1, %2" : "=v"(r) : "v"(lo), "v"(hi)); return r; }
__device__ __forceinline__ float silu(float y) {
    const float c = __builtin_amdgcn_fmed3f(y, -60.0f, 60.0f);
    const float e = __builtin_amdgcn_exp2f(c * -1.44269504088896f);
    return y * __builtin_amdgcn_rcpf(1.0f + e);
}

#define STAMP1(P, slot) do {} while (0)
#define STAMP(P, slot) do {} while (0)

__device__ __forceinline__ void pull_role(const Params& P, unsigned call, int peer, int widx, int lane) {
    const int rank = P.rank;
    char* wsl = P.ws[rank];
    const rsrc_t rwr = mk_rsrc(P.ws[peer], WS_END), rwl = mk_rsrc(wsl, WS_END);
    const rsrc_t rol = mk_rsrc(P.out[rank], OUTB);
    unsigned* tmo = (unsigned*)(wsl + WS_TMO);
    constexpr unsigned OOB = 0xfffffff0u;
    const unsigned lane0 = lane == 0 ? 0u : OOB;
    const unsigned hbase = (call - 1u) * (unsigned)(NPC + NW);
    auto draw = [&]() { return (unsigned)__builtin_amdgcn_raw_ptr_buffer_atomic_add_i32(1, rwl, lane0, WS_HEAD + (unsigned)peer * 128u, 0); };
    auto arrive = [&](unsigned vo, unsigned cnt) { (void)__builtin_amdgcn_raw_ptr_buffer_atomic_add_i32(PU, rwl, vo, cnt, 0); };
    const bool deep0 = (widx & 7) < CFG_DEEP;
    bool bad = false; int stamped = 0; (void)stamped;
    STAMP(P, 0);
    auto index_of = [&](unsigned v) { return (int)((unsigned)__builtin_amdgcn_readfirstlane(v) - hbase); };
    int cur = index_of(draw());
    constexpr int BPB = NS * 128 / PU;
    const unsigned fso = WS_FLAG + (unsigned)peer * FLAGB;
    auto first_of = [&](int i) { return i < NPIN ? (i >> (6 - PSH)) + NPUSH : NIN * 4 + (i - NPIN) / BPB * NS; };
    auto limit_of = [&](const u32x4& f0, const u32x4& f1, int i) {
        const int first = first_of(i);
        unsigned un = 0;
#pragma unroll
        for (int j = 0; j < 8; ++j) un |= ((int)((j < 4 ? f0[j & 3] : f1[j & 3]) - call) < 0 && lane * 8 + j >= first ? 1u : 0u) << j;
        const int l = __builtin_ctzll(__ballot(un != 0u) | (1ull << 63));
        const int u = l * 8 + __builtin_ctz(__builtin_amdgcn_readlane(un, l) | 256u);
        return u <= NIN * 4 ? (u - NPUSH) << (6 - PSH) : u >= NFLAG ? NPC : NPIN + (u - NIN * 4) / NS * BPB;
    };
    auto offs_of = [&](int i, unsigned (&o)[4], unsigned& cnt) -> unsigned {
        if (i < NPIN) {
            int q, pos; const int pan = unit_of((i >> (6 - PSH)) + NPUSH, rank, q, pos);
            cnt = WS_CNT + (unsigned)(pan * NRANK + peer) * 128u + (unsigned)q * 4u;
            const unsigned sub = (unsigned)((i << PSH) & 63) * 1024u;
            const unsigned b = WS_PAN + (unsigned)pan * PANB + (unsigned)peer * CHB + (unsigned)q * UNB + sub + (unsigned)lane * 16u;
#pragma unroll
            for (int k = 0; k < 4; ++k) o[k] = b + k * 1024u;
            return WS_SND + (unsigned)pos * UNB + sub;
        }
        cnt = 0;
#pragma unroll
        for (int k = 0; k < 4; ++k) {
            const unsigned sg = (unsigned)(((i - NPIN) << PSH) + k) * 2u + (unsigned)(lane >> 5);
            o[k] = (unsigned)peer * ROWB + (unsigned)(lane & 31) * 16u + sg / NS * (NN * 2u) + sg % NS * 512u;
        }
        return WS_OSN + (unsigned)((i - NPIN) << PSH) * 1024u;
    };
    int lim = 0;
    while ((unsigned)cur < (unsigned)NPC) {
        if (!bad) {
            const unsigned long long t0 = __builtin_amdgcn_s_memrealtime();
            const unsigned pvo = (unsigned)(lane - (first_of(cur) >> 3)) < 4u ? (unsigned)lane * 32u : OOB - 16u;
            for (;;) {
                const u32x4 f0 = __builtin_amdgcn_raw_buffer_load_b128(rwl, pvo, fso, AUX_SYS), f1 = __builtin_amdgcn_raw_buffer_load_b128(rwl, pvo + 16u, fso, AUX_SYS);
                lim = limit_of(f0, f1, cur);
                if (cur < lim) break;
                if (give_up(tmo, call, t0)) { bad = true; break; }
                __builtin_amdgcn_s_sleep(2);
            }
            asm volatile("" ::: "memory");
        }
        unsigned o[4], p2 = OOB, p3 = OOB, va = OOB, cprev = 0;
        if (bad) {
            (void)offs_of(cur, o, cprev);
            if (cur >= NPIN) {
#pragma unroll
                for (int k = 0; k < PU; ++k) __builtin_amdgcn_raw_buffer_store_b128((u32x4){0x7fc07fc0u, 0x7fc07fc0u, 0x7fc07fc0u, 0x7fc07fc0u}, rol, o[k], 0, AUX_AGENT);
            }
            cur = index_of(draw()); continue;
        }
        STAMP1(P, 1);
        rsrc_t rp = rwl;
        u32x4 v[4] = {};
        const unsigned per = deep0 ? PACE16 / 2 : PACE16;
        unsigned slot = 0;
        if (CFG_RATE) { const unsigned now = (unsigned)__builtin_amdgcn_s_memrealtime() << 4; slot = now - (now & 0xffffffu) % PACE16 + (unsigned)((widx & 7) * PW + (widx >> 3)) * (PACE16 / NW) - PACE16; }
        do {
            unsigned cb; const unsigned src = offs_of(cur, o, cb);
            const bool in = cur < NPIN, deep = deep0 || cur >= NPC - CFG_TAIL;
            const rsrc_t rd = in ? rwl : rol;
            const bool look = lim < NPC && cur + CFG_LOOK >= lim;
            const unsigned fvo = look ? (unsigned)lane * 32u : OOB - 16u;
            unsigned nv = 0; u32x4 f0, f1;
#pragma unroll
            for (int k = 0; k < 4; ++k) {
                if (CFG_RATE && cur < NPC - CFG_TAIL) {
                    slot += per;
                    for (;;) {
                        const int d = (int)(((unsigned)__builtin_amdgcn_s_memrealtime() << 4) - slot);
                        if (d > (int)(PACE16 / 2 * CFG_BURST)) slot += (unsigned)d - PACE16 / 2 * CFG_BURST;
                        if (d >= 0) break;
                        __builtin_amdgcn_s_sleep(1);
                    }
                }
                v[k] = __builtin_amdgcn_raw_buffer_load_b128(rwr, lane * 16 + k * 1024, src, AUX_SYS_NT);
                if (k == 0) {
                    nv = draw();
                    f0 = __builtin_amdgcn_raw_buffer_load_b128(rwl, fvo, fso, AUX_SYS); f1 = __builtin_amdgcn_raw_buffer_load_b128(rwl, fvo + 16u, fso, AUX_SYS);
                }
                __builtin_amdgcn_raw_buffer_store_b128(v[k ^ 2], k < 2 ? rp : rd, k == 0 ? p2 : k == 1 ? p3 : o[k ^ 2], 0, AUX_AGENT);
                if (k == 0) WAIT_VMN(7); else if (k == 1) WAIT_VMN(6); else WAIT_VMN(3);
                if (!deep) { if (k == 0) WAIT_VMN(4); else WAIT_VMN(1); }
            }
            arrive(va, cprev);
            cprev = cb; va = in ? lane0 : OOB; rp = rd; p2 = o[2]; p3 = o[3];
            cur = index_of(nv);
            if (look) { const int l2 = limit_of(f0, f1, cur); lim = l2 > lim ? l2 : lim; }
        } while ((unsigned)cur < (unsigned)lim);
        __builtin_amdgcn_raw_buffer_store_b128(v[2], rp, p2, 0, AUX_AGENT);
        __builtin_amdgcn_raw_buffer_store_b128(v[3], rp, p3, 0, AUX_AGENT);
        if (cprev) { WAIT_VM0(); arrive(va, cprev); }
    }
    STAMP(P, 4);
    asm volatile("" ::: "memory");
    if (lane == 0) st_sys((unsigned*)(P.ws[peer] + WS_DONE + rank * 512) + widx, call);
    if (widx == 0) {
        const unsigned* dn = (const unsigned*)(wsl + WS_DONE + peer * 512);
        const unsigned long long t0 = __builtin_amdgcn_s_memrealtime();
        for (;;) {
            bool ok = true;
            if (lane < NW) ok = (int)(ld_sys(dn + lane) - call) >= 0;
            if (NW > 64 && lane + 64 < NW) ok = ok && (int)(ld_sys(dn + 64 + lane) - call) >= 0;
            if (__all(ok)) break;
            if (give_up(tmo, call, t0)) {
                if (lane == 0) __builtin_amdgcn_raw_buffer_store_b32(0x7fc07fc0u, rol, (unsigned)(rank * NS) * 512u, 0, AUX_SYS);
                break;
            }
            __builtin_amdgcn_s_sleep(2);
        }
        STAMP(P, 5);
    }
}

__device__ __forceinline__ void pack_role(const Params& P, unsigned tk, int g, int wid, int lane) {
    const int rank = P.rank;
    char* wsl = P.ws[rank];
    const rsrc_t rx = mk_rsrc(P.x, (unsigned)MM * KSH * 2), rw = mk_rsrc(P.w, (unsigned)KSH * NN * 2);
    const int fr = lane & 15, fq = lane >> 4;
    const unsigned long long t0 = __builtin_amdgcn_s_memrealtime();
    for (int u = g; u < NUNIT; u += NPK) {
        int o, dest = -1;
        if (u < 4 * U0) {
            const int r = u % U0;
            if (r < NRANK * NS) { o = r >> 3; dest = (rank + 1 + r) & (NRANK - 1); } else o = r - (NRANK - 1) * NS;
            o += u / U0 * G0;
        } else o = u - 4 * (NRANK - 1) * NS;
        int q, pos; const int pan = unit_of(o, dest < 0 ? 0 : dest, q, pos);
        const bool push = o < NPUSH && dest != rank;
        const rsrc_t rwl = mk_rsrc(P.ws[push ? dest & (NRANK - 1) : rank], WS_END);
        while (CFG_PACE > 0 && (unsigned)(__builtin_amdgcn_s_memrealtime() - t0) < (unsigned)(u * CFG_PACE / 10)) __builtin_amdgcn_s_sleep(1);
        const int ktl = q * 2 + (wid >> 2);
        const unsigned dst = push ? WS_PAN + (unsigned)pan * PANB + (unsigned)(rank * KTS + ktl) * KTB : WS_SND + (unsigned)pos * UNB + (unsigned)(wid >> 2) * KTB;
        u32x4 v[8];
        if (pan < NTM) {
            const int rb0 = (wid & 3) * 4;
            const unsigned vo = (unsigned)fr * (KSH * 2) + (unsigned)fq * 16u;
            const unsigned so = (unsigned)(pan * 256 + rb0 * 16) * (KSH * 2) + (unsigned)ktl * 128u;
#pragma unroll
            for (int i = 0; i < 8; ++i) v[i] = __builtin_amdgcn_raw_buffer_load_b128(rx, vo + (i & 1) * 64, so + (unsigned)(i >> 1) * (16 * KSH * 2), 0);
#pragma unroll
            for (int i = 0; i < 8; ++i) STORE16(v[i], rwl, lane * 16 + (i & 1) * 1024, dst + (unsigned)(rb0 + (i >> 1)) * 2048u, AUX_SYS);
        } else {
            const int ks = (wid >> 1) & 1, wr = wid & 1;
            const unsigned vo = (unsigned)(fq * 8) * (NN * 2) + (unsigned)fr * 16u;
            const unsigned so = (unsigned)(ktl * 64 + ks * 32) * (NN * 2) + (unsigned)(gpan_of(pan - NTM, dest < 0 ? 0 : dest) * 256 + wr * 128) * 2u;
#pragma unroll
            for (int j = 0; j < 8; ++j) v[j] = __builtin_amdgcn_raw_buffer_load_b128(rw, vo, so + (unsigned)j * (NN * 2), 2);
#pragma unroll
            for (int i = 0; i < 8; ++i) {
                u32x4 o;
#pragma unroll
                for (int e = 0; e < 4; ++e) o[e] = __builtin_amdgcn_perm(v[2 * e + 1][i >> 1], v[2 * e][i >> 1], (i & 1) ? 0x07060302u : 0x05040100u);
                STORE16(o, rwl, lane * 16, dst + (unsigned)(i >> 2) * HTB + (unsigned)wr * 8192u + (unsigned)(i & 3) * 2048u + (unsigned)ks * 1024u, AUX_SYS);
            }
        }
        WAIT_VM0();
        __builtin_amdgcn_s_barrier();
        if (wid == 0 && lane < NRANK && (dest < 0 || dest == lane)) {
            unsigned c = tk; asm volatile("" : "+v"(c));
            if (lane == rank || push) st_sys((unsigned*)(P.ws[lane] + WS_CNT) + (pan * NRANK + rank) * 32 + q, (c + 1u) * CNT_FULL);
            else st_sys((unsigned*)(P.ws[lane] + WS_FLAG + rank * FLAGB) + o, c + 1u);
        }
    }
}

struct Tile { int tm, v; unsigned sA, stA; };
__device__ __forceinline__ bool tile_of(int idx, int col, int rank, Tile& t) {
    const int nt = 4 * ((NWP + 1 - (col >> 2)) >> 1);
    const int v = 2 * (idx >> 2) + (col >> 2);
    if (idx >= nt) return false;
    t.tm = 4 * (col & 3) + (idx & 3); t.v = v;
    const int g = v / GW, i = v % GW;
    t.sA = WS_SND + (unsigned)(g == 0 ? gpan_of(v, rank) : 4 * U0 + (g - 1) * 4 * GW + i) * UNB;
    t.stA = (unsigned)(g == 0 ? U0 : g <= NGF ? GW : GL) * UNB;
    return true;
}

__device__ __forceinline__ void gemm_role(const Params& P, unsigned call, int col, int slot, LAS unsigned char* lds) {
    const int tid = threadIdx.x, wid = __builtin_amdgcn_readfirstlane(tid >> 6), lane = tid & 63, wr = wid >> 2, wc = wid & 3;
    const int rank = P.rank;
    char* wsl = P.ws[rank];
    const rsrc_t rws = mk_rsrc(wsl, WS_END);
    LAS unsigned* ctl = (LAS unsigned*)(lds + STAGE_BYTES);
    const unsigned voff = (unsigned)tid * 16u;
    const unsigned ldsw = (unsigned)wid * 1024u;
    const int aoff = wr * 8192 + lane * 16, boff = wc * 4096 + lane * 16;

    auto ready = [&](const Tile& t, int want) -> int {
        if (wid == 0) {
            const unsigned* c = (const unsigned*)(wsl + WS_CNT) + ((((lane & 32) ? NTM + t.v : t.tm) * NRANK + ((lane >> 2) & 7)) * 32 + (lane & 3));
            unsigned* tmo = (unsigned*)(wsl + WS_TMO);
            const unsigned need = call * CNT_FULL;
            const unsigned long long t0 = __builtin_amdgcn_s_memrealtime();
            int got;
            for (;;) {
                const unsigned long long miss = __ballot((int)(ld_sys(c) - need) < 0);
                unsigned m = (unsigned)miss | (unsigned)(miss >> 32);
                const bool own = ((m >> (rank * 4)) & 15u) == 0u;
                m |= m >> 16; m |= m >> 8; m |= m >> 4;
                got = own ? 1 + __builtin_ctz(m | 16u) : 0;
                if (got >= want) break;
                if (give_up(tmo, call, t0)) { ctl[0] = 1u; got = 5; break; }
                __builtin_amdgcn_s_sleep(4);
            }
            ctl[2] = (unsigned)got;
            __builtin_amdgcn_fence(__ATOMIC_ACQUIRE, "agent");
            asm volatile("s_waitcnt vmcnt(0) lgkmcnt(0)" ::: "memory");
        }
        asm volatile("" ::: "memory"); __builtin_amdgcn_s_barrier(); asm volatile("" ::: "memory");
        const int r = __builtin_amdgcn_readfirstlane(ctl[2]);
        asm volatile("s_waitcnt lgkmcnt(0)" ::: "memory");
        return r;
    };

#define SA(b, h) (((b) * 2 + (h)) * HTB)
#define SB(b, h) ((4 + (b) * 2 + (h)) * HTB)
#define STAGE(bufoff, soff) do { _Pragma("unroll") for (int _i = 0; _i < 2; ++_i) \
        __builtin_amdgcn_raw_ptr_buffer_load_lds(rws, (LAS unsigned*)(lds + (bufoff) + ldsw + _i * 8192), 16, voff, (soff) + _i * 8192u, 0, 0); } while (0)
#define LDA(dst, b, h) do { _Pragma("unroll") for (int m = 0; m < 4; ++m) _Pragma("unroll") for (int k = 0; k < 2; ++k) dst[m][k] = *(const LAS bf16x8*)(lds + SA(b, h) + aoff + m * 2048 + k * 1024); } while (0)
#define LDB(dst, b, h) do { _Pragma("unroll") for (int n = 0; n < 2; ++n) _Pragma("unroll") for (int k = 0; k < 2; ++k) dst[n][k] = *(const LAS bf16x8*)(lds + SB(b, h) + boff + n * 2048 + k * 1024); } while (0)
#define MMA(ai, bj, At, Bt) do { __builtin_amdgcn_s_setprio(1); _Pragma("unroll") for (int m = 0; m < 4; ++m) _Pragma("unroll") for (int n = 0; n < 2; ++n) _Pragma("unroll") for (int k = 0; k < 2; ++k) \
        acc[ai][bj][m][n] = __builtin_amdgcn_mfma_f32_16x16x32_bf16(Bt[n][k], At[m][k], acc[ai][bj][m][n], 0, 0, 0); __builtin_amdgcn_s_setprio(0); } while (0)
#define WAIT_V(n) asm volatile("s_waitcnt vmcnt(" #n ")" ::: "memory")
#define WAIT_L(n) asm volatile("s_waitcnt lgkmcnt(" #n ")" ::: "memory")
#define BAR __builtin_amdgcn_s_barrier()
#define SCHED __builtin_amdgcn_sched_barrier(0)

    Tile cur, nxt; int ui = slot;
    if (!tile_of(ui, col, rank, cur)) return;
    bf16x8 At[4][2], B0[2][2], B1[2][2];
    constexpr unsigned SB0 = WS_SND + ((NRANK - 1) * NS + GW) * UNB, STB = U0 * UNB;
    const unsigned k0 = (unsigned)rank * CHB;
    unsigned ua = cur.sA, ub = SB0 + (unsigned)cur.tm * UNB;
    int lvl = ready(cur, 1);
    STAMP(P, 2);
    STAGE(SB(0, 0), ub); STAGE(SA(0, 0), ua); STAGE(SB(0, 1), ub + HTB); STAGE(SA(0, 1), ua + HTB);
    if (wr == 1) BAR;
    WAIT_V(4); BAR;
    STAGE(SB(1, 0), ub + KTB); STAGE(SA(1, 0), ua + KTB); STAGE(SB(1, 1), ub + HTB + KTB);
    WAIT_V(6); BAR;
#pragma unroll 1
    for (;;) {
        f32x4 acc[2][2][4][2];
#pragma unroll
        for (int a = 0; a < 2; ++a)
#pragma unroll
            for (int b = 0; b < 2; ++b)
#pragma unroll
                for (int m = 0; m < 4; ++m)
#pragma unroll
                    for (int n = 0; n < 2; ++n) acc[a][b][m][n] = (f32x4){0.f, 0.f, 0.f, 0.f};
        const bool has_next = tile_of(ui + KX, col, rank, nxt);
        if (!has_next) nxt = cur;
        const unsigned cA = WS_PAN + (unsigned)(NTM + cur.v) * PANB, cB = WS_PAN + (unsigned)cur.tm * PANB;
        unsigned ko = k0;
        int jchk = lvl >= 5 ? 31 : 7 * lvl - 4;
        for (int j = 0; j < KT / 2; ++j) {
            const bool last = (j == KT / 2 - 1);
            if (j == jchk && (has_next || !last)) {
                lvl = ready(last ? nxt : cur, last ? 1 : lvl + 1);
                jchk = lvl >= 5 ? 31 : 7 * lvl - 4;
            }
            const unsigned a1 = ua + KTB;
            if (j < 3) { ua += cur.stA; ub += STB; }
            else if (last) { ua = nxt.sA; ub = SB0 + (unsigned)nxt.tm * UNB; }
            else {
                ko = (ko + CHB) & (PANB - 1);
                if ((ko ^ k0) < CHB) ko = (ko + CHB + UNB) & (PANB - 1);
                ua = cA + ko; ub = cB + ko;
            }
            const unsigned a2 = ua, b2 = ub, a3 = a2 + KTB, b3 = b2 + KTB;
            LDB(B0, 0, 0); SCHED; LDA(At, 0, 0); STAGE(SA(1, 1), a1 + HTB);
            WAIT_L(8); BAR; WAIT_L(0); MMA(0, 0, At, B0); BAR; SCHED;
            LDB(B1, 0, 1); STAGE(SB(0, 0), b2);
            BAR; WAIT_L(0); MMA(0, 1, At, B1); BAR;
            LDA(At, 0, 1); STAGE(SA(0, 0), a2);
            BAR; WAIT_L(0); MMA(1, 0, At, B0); BAR; SCHED;
            STAGE(SB(0, 1), b2 + HTB);
            WAIT_V(6); BAR; MMA(1, 1, At, B1); BAR;
            LDB(B0, 1, 0); SCHED; LDA(At, 1, 0); STAGE(SA(0, 1), a2 + HTB);
            WAIT_L(8); BAR; WAIT_L(0); MMA(0, 0, At, B0); BAR; SCHED;
            LDB(B1, 1, 1); STAGE(SB(1, 0), b3);
            BAR; WAIT_L(0); MMA(0, 1, At, B1); BAR;
            LDA(At, 1, 1); STAGE(SA(1, 0), a3);
            BAR; WAIT_L(0); MMA(1, 0, At, B0); BAR; SCHED;
            STAGE(SB(1, 1), b3 + HTB);
            WAIT_V(6); BAR; MMA(1, 1, At, B1);
            if (!(last && wr == 1)) BAR;
        }
        {
            int tz = threadIdx.x; asm volatile("" : "+v"(tz));
            const int ln = tz & 63, fr = ln & 15, fq = ln >> 4;
            const rsrc_t rol = mk_rsrc(P.out[rank], OUTB);
            const bool bad = ctl[0] != 0u, shared = cur.v < NS;
            const bool thru = CFG_LAST && !has_next;
            const unsigned vo = (unsigned)(wc * 32 + fq * 4) * (NN * 2) + (unsigned)(wr * 256 + fr * 16);
            const unsigned so = (unsigned)(cur.tm * 256) * (NN * 2) + (unsigned)gpan_of(cur.v, rank) * 512u;
            const unsigned vs = (unsigned)(wc * 32 + fq * 4) * ROWB + (unsigned)(wr * 256 + fr * 16);
            const unsigned ss = WS_OSN + (unsigned)(cur.tm * 256) * ROWB + (unsigned)cur.v * 512u;
#pragma unroll
            for (int bj = 0; bj < 2; ++bj)
#pragma unroll
                for (int n = 0; n < 2; ++n)
#pragma unroll
                    for (int r = 0; r < 4; ++r) {
                        u32x4 o;
                        o[0] = cvt_pk_bf16(silu(acc[0][bj][0][n][r]), silu(acc[0][bj][1][n][r]));
                        o[1] = cvt_pk_bf16(silu(acc[0][bj][2][n][r]), silu(acc[0][bj][3][n][r]));
                        o[2] = cvt_pk_bf16(silu(acc[1][bj][0][n][r]), silu(acc[1][bj][1][n][r]));
                        o[3] = cvt_pk_bf16(silu(acc[1][bj][2][n][r]), silu(acc[1][bj][3][n][r]));
                        if (bad) o = (u32x4){0x7fc07fc0u, 0x7fc07fc0u, 0x7fc07fc0u, 0x7fc07fc0u};
                        if (shared) STORE16(o, rws, vs, ss + (unsigned)(bj * 128 + n * 16 + r) * ROWB, AUX_SYS);
                        if (thru) { STORE16(o, rol, vo, so + (unsigned)(bj * 128 + n * 16 + r) * (NN * 2), AUX_SYS);
                                    if (CFG_LAST > 1) { WAIT_VM0(); __builtin_amdgcn_s_sleep(CFG_LAST > 1 ? CFG_LAST - 1 : 1); } }
                        else STORE16(o, rol, vo, so + (unsigned)(bj * 128 + n * 16 + r) * (NN * 2), CFG_RST);
                    }
            if (shared) {
                WAIT_VM0();
                unsigned old = 0;
                if (ln == 0) old = __hip_atomic_fetch_add(ctl + 1, 1u, __ATOMIC_RELAXED, __HIP_MEMORY_SCOPE_WORKGROUP);
                old = __builtin_amdgcn_readfirstlane(old);
                if ((old & 7u) == 7u && ln < NRANK && ln != rank)
                    st_sys((unsigned*)(P.ws[ln] + WS_FLAG + rank * FLAGB) + NIN * 4 + cur.tm * NS + cur.v, call);
            }
        }
        STAMP(P, 3 + (ui >= KX ? 1 : 0));
        if (!has_next) break;
        if (wr == 1) BAR;
        cur = nxt; ui += KX;
    }
    WAIT_V(0);
}

__global__ __launch_bounds__(512, 2) void fused_gemm_ar(Params P) {
    extern __shared__ __attribute__((aligned(16))) unsigned char shm[];
    LAS unsigned char* lds = (LAS unsigned char*)shm;
    const int wid = __builtin_amdgcn_readfirstlane(threadIdx.x >> 6), lane = threadIdx.x & 63;
    const int b = __builtin_amdgcn_readfirstlane(blockIdx.x);
    unsigned* tkp = (unsigned*)(P.ws[P.rank] + WS_TICKET + b * 128 + wid * 16);
    const unsigned tk = ld_agent(tkp);
    if (b < NPULL) {
        const unsigned call = __builtin_amdgcn_readfirstlane(tk) + 1u;
        st_agent(tkp, call);
        const int pi = b / PW, slot = b % PW;
        const int peer = (P.rank + 1 + pi) & (NRANK - 1);
        pull_role(P, call, peer, slot * 8 + wid, lane);
    } else {
        const int g = b - NPULL;
        ((LAS unsigned*)(lds + STAGE_BYTES))[lane & 3] = 0u;
        STAMP(P, 0);
        pack_role(P, tk, g, wid, lane);
        const unsigned call = __builtin_amdgcn_readfirstlane(tk) + 1u;
        st_agent(tkp, call);
        STAMP(P, 1);
        asm volatile("s_waitcnt lgkmcnt(0)" ::: "memory");
        __builtin_amdgcn_s_barrier();
        if (b >= NPS) gemm_role(P, call, b & 7, (b - NPS) >> 3, lds);
    }
}
}

extern "C" void kernel_launch(int rank, int nranks, ncclComm_t comm,
                              hipStream_t stream,
                              void* const* d_in, const int* in_sizes, int n_in,
                              void* d_out, int out_size,
                              const int* in_dtypes, int out_dtype,
                              void* const* d_out_all,
                              void* const* d_ws_all, size_t ws_bytes) {
    if (nranks != NRANK || (unsigned)rank >= (unsigned)NRANK || ws_bytes < (size_t)WS_END || n_in < 2 ||
        in_sizes[0] != MM * KSH || in_sizes[1] != KSH * NN || out_size != MM * NN) {
        fprintf(stderr, "unexpected problem geometry\n");
        return;
    }
    static bool attr_done[NRANK] = {};
    if (!attr_done[rank]) {
        HIP_CHECK(hipFuncSetAttribute((const void*)fused_gemm_ar, hipFuncAttributeMaxDynamicSharedMemorySize, LDS_BYTES));
        attr_done[rank] = true;
    }
    Params P;
    P.x = d_in[0]; P.w = d_in[1]; P.rank = rank;
    for (int p = 0; p < NRANK; ++p) { P.out[p] = (char*)d_out_all[p]; P.ws[p] = (char*)d_ws_all[p]; }
    hipLaunchKernelGGL(fused_gemm_ar, dim3(GRID), dim3(512), LDS_BYTES, stream, P);
    HIP_CHECK(hipGetLastError());
}
```

```cpp
#include <hip/hip_runtime.h>
#include <rccl/rccl.h>
#include <cstdio>

#define HIP_CHECK(expr) do { hipError_t _e = (expr); if (_e != hipSuccess) { \
    fprintf(stderr, "HIP error %s at %s:%d\n", hipGetErrorString(_e), __FILE__, __LINE__); \
    return; } } while (0)

#define LAS __attribute__((address_space(3)))
typedef short bf16x8 __attribute__((ext_vector_type(8)));
typedef float f32x4 __attribute__((ext_vector_type(4)));
typedef unsigned u32x4 __attribute__((ext_vector_type(4)));
typedef __amdgpu_buffer_rsrc_t rsrc_t;

namespace {
constexpr int NRANK = 8;
constexpr int MM = 4096, KK = 4096, NN = 32768, KSH = KK / NRANK;
constexpr int KT = KK / 64, KTS = KSH / 64, NTM = MM / 256, NPAN = NN / 256;
#define CFG_NS 8
#define CFG_PW 8
constexpr int NS = CFG_NS;
constexpr int NRP = NPAN - NRANK * NS;
constexpr int NWP = NS + NRP;
constexpr int NIN = NTM + NWP;
constexpr int NOUT = NS * NTM;
constexpr int NFLAG = NIN * 4 + NOUT;
constexpr int NUNIT = NTM * 4 + NPAN * 4;
constexpr int GRID = 256;
constexpr int PW = CFG_PW;
constexpr int NPULL = PW * (NRANK - 1);
constexpr int NW = PW * 8;
constexpr int NPK = GRID - NPULL;
constexpr int NPS = (NPULL + 7) / 8 * 8;
constexpr int KX = (GRID - NPS) / 8;
#define CFG_GW (2 * (KX / 4))
constexpr int GW = CFG_GW;
constexpr int G0 = GW + NTM;
constexpr int NGF = (NWP - GW) / GW, GL = (NWP - GW) % GW;
static_assert(NS <= GW && NWP > GW, "shared panels sit in the first group");
#define CFG_PU 4
constexpr int PU = CFG_PU, PSH = PU == 1 ? 0 : PU == 2 ? 1 : PU == 4 ? 2 : 3;
#define CFG_PUSH 3
constexpr int NPUSH = CFG_PUSH;
constexpr int NPIN = (NIN * 4 - NPUSH) * 64 / PU;
static_assert(NPUSH <= NS, "pushed units have a single consumer");
constexpr int NPC = NPIN + NOUT * 128 / PU;
constexpr unsigned CNT_FULL = 64;

constexpr unsigned KTB = 32768, HTB = 16384;
constexpr unsigned PANB = KT * KTB;
constexpr unsigned CHB = KTS * KTB;
constexpr unsigned UNB = 2 * KTB;
constexpr unsigned WS_TICKET = 0;
constexpr unsigned WS_TMO = WS_TICKET + GRID * 128;
constexpr unsigned WS_FLAG = WS_TMO + 128;
constexpr unsigned FLAGB = 4096;
constexpr unsigned WS_DONE = WS_FLAG + NRANK * FLAGB;
constexpr unsigned WS_HEAD = WS_DONE + NRANK * 512;
constexpr unsigned WS_CNT = WS_HEAD + NRANK * 128;
constexpr unsigned WS_CTRL_END = WS_CNT + NIN * NRANK * 128;
constexpr unsigned WS_DBG = 512u << 10;
constexpr unsigned WS_SND = 1u << 20;
constexpr unsigned ROWB = NS * 512;
constexpr unsigned WS_OSN = WS_SND + NUNIT * UNB;
constexpr unsigned WS_PAN = (WS_OSN + MM * ROWB + PANB - 1) / PANB * PANB;
constexpr unsigned WS_END = WS_PAN + NIN * PANB;
constexpr int U0 = G0 + (NRANK - 1) * NS;
constexpr unsigned OUTB = (unsigned)MM * NN * 2;
static_assert(WS_CTRL_END <= WS_DBG, "ctrl region");
static_assert(NFLAG <= 512 && 2048 <= (int)FLAGB, "flag region");
static_assert(NW <= 128, "done words");
static_assert(PU == 4, "block size");
#define CFG_DEEP 3
constexpr unsigned STAGE_BYTES = 8 * HTB;
constexpr unsigned LDS_BYTES = STAGE_BYTES + 64;
constexpr unsigned long long DEADLINE = 50000000ull;
constexpr int AUX_SYS = 17, AUX_SYS_NT = 19, AUX_AGENT = 16;
#define CFG_PACE 40
#define CFG_PACE0 40
#define CFG_HOLD 0
#define CFG_STH 0
#define CFG_ETH 0
#define CFG_STAG 300
#define CFG_P3 320
#define CFG_P3N 2
#define CFG_LOOK 256
#define CFG_TAIL NW
#define CFG_LAST 8
#define CFG_RST 0
#define CFG_RATE 1720
#define CFG_CRED 40
constexpr unsigned PACE16 = CFG_RATE * PW * (8 + CFG_DEEP) / 64;

struct Params {
    const void* x; const void* w;
    char* out[NRANK]; char* ws[NRANK];
    int rank;
};

__device__ __forceinline__ rsrc_t mk_rsrc(const void* p, unsigned bytes) {
    const unsigned long long v = (unsigned long long)p;
    const unsigned lo = __builtin_amdgcn_readfirstlane((unsigned)v), hi = __builtin_amdgcn_readfirstlane((unsigned)(v >> 32));
    return __builtin_amdgcn_make_buffer_rsrc((void*)(((unsigned long long)hi << 32) | lo), 0, bytes, 0x00020000);
}
__device__ __forceinline__ unsigned ld_sys(const unsigned* p) { return __hip_atomic_load(p, __ATOMIC_RELAXED, __HIP_MEMORY_SCOPE_SYSTEM); }
__device__ __forceinline__ unsigned ld_agent(const unsigned* p) { return __hip_atomic_load(p, __ATOMIC_RELAXED, __HIP_MEMORY_SCOPE_AGENT); }
__device__ __forceinline__ void st_sys(unsigned* p, unsigned v) { __hip_atomic_store(p, v, __ATOMIC_RELAXED, __HIP_MEMORY_SCOPE_SYSTEM); }
__device__ __forceinline__ void st_agent(unsigned* p, unsigned v) { __hip_atomic_store(p, v, __ATOMIC_RELAXED, __HIP_MEMORY_SCOPE_AGENT); }
__device__ __forceinline__ void add_agent(unsigned* p, unsigned v) { (void)__hip_atomic_fetch_add(p, v, __ATOMIC_RELAXED, __HIP_MEMORY_SCOPE_AGENT); }
#define STORE16(v, rs, vo, so, aux) do { const u32x4 _d = (v); __builtin_amdgcn_raw_buffer_store_b128(_d, rs, vo, so, aux); asm volatile("s_nop 1" :: "v"(_d) : "memory"); } while (0)
#define WAIT_VM0() asm volatile("s_waitcnt vmcnt(0)" ::: "memory")
#define WAIT_VMN(n) do { asm volatile("" ::: "memory"); __builtin_amdgcn_s_waitcnt(0x0F70 | (n)); asm volatile("" ::: "memory"); } while (0)

__device__ __forceinline__ bool give_up(unsigned* tmo, unsigned call, unsigned long long t0) {
    const unsigned long long dt = __builtin_amdgcn_s_memrealtime() - t0;
    if (dt < 2000ull) return false;
    if ((unsigned)__builtin_amdgcn_readfirstlane(ld_agent(tmo)) == call) return true;
    if (dt > DEADLINE) { st_agent(tmo, call); return true; }
    return false;
}

__device__ __forceinline__ int gpan_of(int v, int consumer) { return v < NS ? consumer * NS + v : (NRANK - 1) * NS + v; }
__device__ __forceinline__ int unit_of(int o, int consumer, int& q, int& pos) {
    if (o < 4 * G0) { q = o / G0; const int m = o % G0; pos = q * U0 + gpan_of(m, consumer); return m < GW ? NTM + m : m - GW; }
    pos = o + 4 * (NRANK - 1) * NS;
    o -= 4 * G0;
    const int g = o / (4 * GW), r = o % (4 * GW);
    int i;
    if (GL == 0 || g < NGF) { q = r / GW; i = r % GW; } else { q = r / (GL ? GL : 1); i = r % (GL ? GL : 1); }
    return NTM + GW * (g + 1) + i;
}

__device__ __forceinline__ unsigned cvt_pk_bf16(float lo, float hi) { unsigned r; asm("v_cvt_pk_bf16_f32 %0, %1, %2" : "=v"(r) : "v"(lo), "v"(hi)); return r; }
__device__ __forceinline__ float silu(float y) {
    const float c = __builtin_amdgcn_fmed3f(y, -60.0f, 60.0f);
    const float e = __builtin_amdgcn_exp2f(c * -1.44269504088896f);
    return y * __builtin_amdgcn_rcpf(1.0f + e);
}

#define STAMP1(P, slot) do {} while (0)
#define STAMP(P, slot) do {} while (0)

__device__ __forceinline__ void pull_role(const Params& P, unsigned call, int peer, int widx, int lane) {
    const int rank = P.rank;
    char* wsl = P.ws[rank];
    const rsrc_t rwr = mk_rsrc(P.ws[peer], WS_END), rwl = mk_rsrc(wsl, WS_END);
    const rsrc_t rol = mk_rsrc(P.out[rank], OUTB);
    unsigned* tmo = (unsigned*)(wsl + WS_TMO);
    constexpr unsigned OOB = 0xfffffff0u;
    const unsigned lane0 = lane == 0 ? 0u : OOB;
    const unsigned hbase = (call - 1u) * (unsigned)(NPC + NW);
    auto draw = [&]() { return (unsigned)__builtin_amdgcn_raw_ptr_buffer_atomic_add_i32(1, rwl, lane0, WS_HEAD + (unsigned)peer * 128u, 0); };
    auto arrive = [&](unsigned vo, unsigned cnt) { (void)__builtin_amdgcn_raw_ptr_buffer_atomic_add_i32(PU, rwl, vo, cnt, 0); };
    const bool deep0 = (widx & 7) < CFG_DEEP;
    bool bad = false; int stamped = 0; (void)stamped;
    STAMP(P, 0);
    auto index_of = [&](unsigned v) { return (int)((unsigned)__builtin_amdgcn_readfirstlane(v) - hbase); };
    const unsigned long long tb = __builtin_amdgcn_s_memrealtime();
    int cur = index_of(draw());
    while (CFG_HOLD > 0 && __builtin_amdgcn_s_memrealtime() - tb < (unsigned long long)CFG_HOLD) __builtin_amdgcn_s_sleep(2);
    constexpr int BPB = NS * 128 / PU;
    const unsigned fso = WS_FLAG + (unsigned)peer * FLAGB;
    auto first_of = [&](int i) { return i < NPIN ? (i >> (6 - PSH)) + NPUSH : NIN * 4 + (i - NPIN) / BPB * NS; };
    auto limit_of = [&](const u32x4& f0, const u32x4& f1, int i) {
        const int first = first_of(i);
        unsigned un = 0;
#pragma unroll
        for (int j = 0; j < 8; ++j) un |= ((int)((j < 4 ? f0[j & 3] : f1[j & 3]) - call) < 0 && lane * 8 + j >= first ? 1u : 0u) << j;
        const int l = __builtin_ctzll(__ballot(un != 0u) | (1ull << 63));
        const int u = l * 8 + __builtin_ctz(__builtin_amdgcn_readlane(un, l) | 256u);
        return u <= NIN * 4 ? (u - NPUSH) << (6 - PSH) : u >= NFLAG ? NPC : NPIN + (u - NIN * 4) / NS * BPB;
    };
    auto offs_of = [&](int i, unsigned (&o)[4], unsigned& cnt) -> unsigned {
        if (i < NPIN) {
            int q, pos; const int pan = unit_of((i >> (6 - PSH)) + NPUSH, rank, q, pos);
            cnt = WS_CNT + (unsigned)(pan * NRANK + peer) * 128u + (unsigned)q * 4u;
            const unsigned sub = (unsigned)((i << PSH) & 63) * 1024u;
            const unsigned b = WS_PAN + (unsigned)pan * PANB + (unsigned)peer * CHB + (unsigned)q * UNB + sub + (unsigned)lane * 16u;
#pragma unroll
            for (int k = 0; k < 4; ++k) o[k] = b + k * 1024u;
            return WS_SND + (unsigned)pos * UNB + sub;
        }
        cnt = 0;
#pragma unroll
        for (int k = 0; k < 4; ++k) {
            const unsigned sg = (unsigned)(((i - NPIN) << PSH) + k) * 2u + (unsigned)(lane >> 5);
            o[k] = (unsigned)peer * ROWB + (unsigned)(lane & 31) * 16u + sg / NS * (NN * 2u) + sg % NS * 512u;
        }
        return WS_OSN + (unsigned)((i - NPIN) << PSH) * 1024u;
    };
    int lim = 0;
    while ((unsigned)cur < (unsigned)NPC) {
        if (!bad) {
            const unsigned long long t0 = __builtin_amdgcn_s_memrealtime();
            const unsigned pvo = (unsigned)(lane - (first_of(cur) >> 3)) < 4u ? (unsigned)lane * 32u : OOB - 16u;
            for (;;) {
                const u32x4 f0 = __builtin_amdgcn_raw_buffer_load_b128(rwl, pvo, fso, AUX_SYS), f1 = __builtin_amdgcn_raw_buffer_load_b128(rwl, pvo + 16u, fso, AUX_SYS);
                lim = limit_of(f0, f1, cur);
                if (cur < lim) break;
                if (give_up(tmo, call, t0)) { bad = true; break; }
                __builtin_amdgcn_s_sleep(2);
            }
            asm volatile("" ::: "memory");
        }
        unsigned o[4], p2 = OOB, p3 = OOB, va = OOB, cprev = 0;
        if (bad) {
            (void)offs_of(cur, o, cprev);
            if (cur >= NPIN) {
#pragma unroll
                for (int k = 0; k < PU; ++k) __builtin_amdgcn_raw_buffer_store_b128((u32x4){0x7fc07fc0u, 0x7fc07fc0u, 0x7fc07fc0u, 0x7fc07fc0u}, rol, o[k], 0, AUX_AGENT);
            }
            cur = index_of(draw()); continue;
        }
        STAMP1(P, 1);
        rsrc_t rp = rwl;
        u32x4 v[4] = {};
        const unsigned per = deep0 ? PACE16 / 2 : PACE16;
        unsigned slot = 0;
        if (CFG_RATE) { const unsigned now = (unsigned)__builtin_amdgcn_s_memrealtime() << 4; slot = now - (now & 0xffffffu) % PACE16 + (unsigned)((widx & 7) * PW + (widx >> 3)) * (PACE16 / NW) - PACE16; }
        do {
            unsigned cb; const unsigned src = offs_of(cur, o, cb);
            const bool in = cur < NPIN, deep = deep0 || cur >= NPC - CFG_TAIL;
            const rsrc_t rd = in ? rwl : rol;
            const bool look = lim < NPC && cur + CFG_LOOK >= lim;
            const unsigned fvo = look ? (unsigned)lane * 32u : OOB - 16u;
            unsigned nv = 0; u32x4 f0, f1;
#pragma unroll
            for (int k = 0; k < 4; ++k) {
                if (CFG_RATE && cur < NPC - CFG_TAIL) {
                    slot += per;
                    for (;;) {
                        const int d = (int)(((unsigned)__builtin_amdgcn_s_memrealtime() << 4) - slot);
                        if (d > CFG_CRED * 16) slot += (unsigned)(d - CFG_CRED * 16);
                        if (d >= 0) break;
                        __builtin_amdgcn_s_sleep(1);
                    }
                }
                v[k] = __builtin_amdgcn_raw_buffer_load_b128(rwr, lane * 16 + k * 1024, src, AUX_SYS_NT);
                if (k == 0) {
                    nv = draw();
                    f0 = __builtin_amdgcn_raw_buffer_load_b128(rwl, fvo, fso, AUX_SYS); f1 = __builtin_amdgcn_raw_buffer_load_b128(rwl, fvo + 16u, fso, AUX_SYS);
                }
                __builtin_amdgcn_raw_buffer_store_b128(v[k ^ 2], k < 2 ? rp : rd, k == 0 ? p2 : k == 1 ? p3 : o[k ^ 2], 0, AUX_AGENT);
                if (k == 0) WAIT_VMN(7); else if (k == 1) WAIT_VMN(6); else WAIT_VMN(3);
                if (!deep) { if (k == 0) WAIT_VMN(4); else WAIT_VMN(1); }
            }
            arrive(va, cprev);
            cprev = cb; va = in ? lane0 : OOB; rp = rd; p2 = o[2]; p3 = o[3];
            cur = index_of(nv);
            if (look) { const int l2 = limit_of(f0, f1, cur); lim = l2 > lim ? l2 : lim; }
        } while ((unsigned)cur < (unsigned)lim);
        __builtin_amdgcn_raw_buffer_store_b128(v[2], rp, p2, 0, AUX_AGENT);
        __builtin_amdgcn_raw_buffer_store_b128(v[3], rp, p3, 0, AUX_AGENT);
        if (cprev) { WAIT_VM0(); arrive(va, cprev); }
    }
    STAMP(P, 4);
    asm volatile("" ::: "memory");
    if (lane == 0) st_sys((unsigned*)(P.ws[peer] + WS_DONE + rank * 512) + widx, call);
    if (widx == 0) {
        const unsigned* dn = (const unsigned*)(wsl + WS_DONE + peer * 512);
        const unsigned long long t0 = __builtin_amdgcn_s_memrealtime();
        for (;;) {
            bool ok = true;
            if (lane < NW) ok = (int)(ld_sys(dn + lane) - call) >= 0;
            if (NW > 64 && lane + 64 < NW) ok = ok && (int)(ld_sys(dn + 64 + lane) - call) >= 0;
            if (__all(ok)) break;
            if (give_up(tmo, call, t0)) {
                if (lane == 0) __builtin_amdgcn_raw_buffer_store_b32(0x7fc07fc0u, rol, (unsigned)(rank * NS) * 512u, 0, AUX_SYS);
                break;
            }
            __builtin_amdgcn_s_sleep(2);
        }
        STAMP(P, 5);
    }
}

__device__ __forceinline__ void pack_role(const Params& P, unsigned tk, int g, int wid, int lane) {
    const int rank = P.rank;
    char* wsl = P.ws[rank];
    const rsrc_t rx = mk_rsrc(P.x, (unsigned)MM * KSH * 2), rw = mk_rsrc(P.w, (unsigned)KSH * NN * 2);
    const int fr = lane & 15, fq = lane >> 4;
    const unsigned long long t0 = __builtin_amdgcn_s_memrealtime();
    for (int u = g; u < NUNIT; u += NPK) {
        int o, dest = -1;
        if (u < 4 * U0) {
            const int r = u % U0;
            if (r < NRANK * NS) { o = r >> 3; dest = (rank + 1 + r) & (NRANK - 1); } else o = r - (NRANK - 1) * NS;
            o += u / U0 * G0;
        } else o = u - 4 * (NRANK - 1) * NS;
        int q, pos; const int pan = unit_of(o, dest < 0 ? 0 : dest, q, pos);
        const bool push = o < NPUSH && dest != rank;
        const rsrc_t rwl = mk_rsrc(P.ws[push ? dest & (NRANK - 1) : rank], WS_END);
        while (CFG_PACE > 0 && (unsigned)(__builtin_amdgcn_s_memrealtime() - t0) < (unsigned)((u < CFG_PACE0 ? 0 : u - CFG_PACE0) * CFG_PACE / 10)) __builtin_amdgcn_s_sleep(1);
        const int ktl = q * 2 + (wid >> 2);
        const unsigned dst = push ? WS_PAN + (unsigned)pan * PANB + (unsigned)(rank * KTS + ktl) * KTB : WS_SND + (unsigned)pos * UNB + (unsigned)(wid >> 2) * KTB;
        u32x4 v[8];
        if (pan < NTM) {
            const int rb0 = (wid & 3) * 4;
            const unsigned vo = (unsigned)fr * (KSH * 2) + (unsigned)fq * 16u;
            const unsigned so = (unsigned)(pan * 256 + rb0 * 16) * (KSH * 2) + (unsigned)ktl * 128u;
#pragma unroll
            for (int i = 0; i < 8; ++i) v[i] = __builtin_amdgcn_raw_buffer_load_b128(rx, vo + (i & 1) * 64, so + (unsigned)(i >> 1) * (16 * KSH * 2), 0);
#pragma unroll
            for (int i = 0; i < 8; ++i) STORE16(v[i], rwl, lane * 16 + (i & 1) * 1024, dst + (unsigned)(rb0 + (i >> 1)) * 2048u, AUX_SYS);
        } else {
            const int ks = (wid >> 1) & 1, wr = wid & 1;
            const unsigned vo = (unsigned)(fq * 8) * (NN * 2) + (unsigned)fr * 16u;
            const unsigned so = (unsigned)(ktl * 64 + ks * 32) * (NN * 2) + (unsigned)(gpan_of(pan - NTM, dest < 0 ? 0 : dest) * 256 + wr * 128) * 2u;
#pragma unroll
            for (int j = 0; j < 8; ++j) v[j] = __builtin_amdgcn_raw_buffer_load_b128(rw, vo, so + (unsigned)j * (NN * 2), 2);
            while (CFG_P3 && push && o >= CFG_P3N && (unsigned)(__builtin_amdgcn_s_memrealtime() - t0) < (unsigned)(CFG_P3 + (o - CFG_P3N) * 110)) __builtin_amdgcn_s_sleep(1);
#pragma unroll
            for (int i = 0; i < 8; ++i) {
                u32x4 o;
#pragma unroll
                for (int e = 0; e < 4; ++e) o[e] = __builtin_amdgcn_perm(v[2 * e + 1][i >> 1], v[2 * e][i >> 1], (i & 1) ? 0x07060302u : 0x05040100u);
                STORE16(o, rwl, lane * 16, dst + (unsigned)(i >> 2) * HTB + (unsigned)wr * 8192u + (unsigned)(i & 3) * 2048u + (unsigned)ks * 1024u, AUX_SYS);
            }
        }
        WAIT_VM0();
        if (u == g) STAMP(P, 6);
        __builtin_amdgcn_s_barrier();
        if (wid == 0 && lane < NRANK && (dest < 0 || dest == lane)) {
            unsigned c = tk; asm volatile("" : "+v"(c));
            if (lane == rank || push) st_sys((unsigned*)(P.ws[lane] + WS_CNT) + (pan * NRANK + rank) * 32 + q, (c + 1u) * CNT_FULL);
            else st_sys((unsigned*)(P.ws[lane] + WS_FLAG + rank * FLAGB) + o, c + 1u);
        }
    }
}

struct Tile { int tm, v; unsigned sA, stA; };
__device__ __forceinline__ bool tile_of(int idx, int col, int rank, Tile& t) {
    const int nt = 4 * ((NWP + 1 - (col >> 2)) >> 1);
    const int v = 2 * (idx >> 2) + (col >> 2);
    if (idx >= nt) return false;
    t.tm = 4 * (col & 3) + (idx & 3); t.v = v;
    const int g = v / GW, i = v % GW;
    t.sA = WS_SND + (unsigned)(g == 0 ? gpan_of(v, rank) : 4 * U0 + (g - 1) * 4 * GW + i) * UNB;
    t.stA = (unsigned)(g == 0 ? U0 : g <= NGF ? GW : GL) * UNB;
    return true;
}

__device__ __forceinline__ void gemm_role(const Params& P, unsigned call, int col, int slot, LAS unsigned char* lds) {
    const int tid = threadIdx.x, wid = __builtin_amdgcn_readfirstlane(tid >> 6), lane = tid & 63, wr = wid >> 2, wc = wid & 3;
    const int rank = P.rank;
    char* wsl = P.ws[rank];
    const rsrc_t rws = mk_rsrc(wsl, WS_END);
    LAS unsigned* ctl = (LAS unsigned*)(lds + STAGE_BYTES);
    const unsigned voff = (unsigned)tid * 16u;
    const unsigned ldsw = (unsigned)wid * 1024u;
    const int aoff = wr * 8192 + lane * 16, boff = wc * 4096 + lane * 16;

    auto ready = [&](const Tile& t, int want) -> int {
        if (wid == 0) {
            const unsigned* c = (const unsigned*)(wsl + WS_CNT) + ((((lane & 32) ? NTM + t.v : t.tm) * NRANK + ((lane >> 2) & 7)) * 32 + (lane & 3));
            unsigned* tmo = (unsigned*)(wsl + WS_TMO);
            const unsigned need = call * CNT_FULL;
            const unsigned long long t0 = __builtin_amdgcn_s_memrealtime();
            int got;
            for (;;) {
                const unsigned long long miss = __ballot((int)(ld_sys(c) - need) < 0);
                unsigned m = (unsigned)miss | (unsigned)(miss >> 32);
                const bool own = ((m >> (rank * 4)) & 15u) == 0u;
                m |= m >> 16; m |= m >> 8; m |= m >> 4;
                got = own ? 1 + __builtin_ctz(m | 16u) : 0;
                if (got >= want) break;
                if (give_up(tmo, call, t0)) { ctl[0] = 1u; got = 5; break; }
                __builtin_amdgcn_s_sleep(4);
            }
            ctl[2] = (unsigned)got;
            __builtin_amdgcn_fence(__ATOMIC_ACQUIRE, "agent");
            asm volatile("s_waitcnt vmcnt(0) lgkmcnt(0)" ::: "memory");
        }
        asm volatile("" ::: "memory"); __builtin_amdgcn_s_barrier(); asm volatile("" ::: "memory");
        const int r = __builtin_amdgcn_readfirstlane(ctl[2]);
        asm volatile("s_waitcnt lgkmcnt(0)" ::: "memory");
        return r;
    };

#define SA(b, h) (((b) * 2 + (h)) * HTB)
#define SB(b, h) ((4 + (b) * 2 + (h)) * HTB)
#define STAGE(bufoff, soff) do { _Pragma("unroll") for (int _i = 0; _i < 2; ++_i) \
        __builtin_amdgcn_raw_ptr_buffer_load_lds(rws, (LAS unsigned*)(lds + (bufoff) + ldsw + _i * 8192), 16, voff, (soff) + _i * 8192u, 0, 0); } while (0)
#define LDA(dst, b, h) do { _Pragma("unroll") for (int m = 0; m < 4; ++m) _Pragma("unroll") for (int k = 0; k < 2; ++k) dst[m][k] = *(const LAS bf16x8*)(lds + SA(b, h) + aoff + m * 2048 + k * 1024); } while (0)
#define LDB(dst, b, h) do { _Pragma("unroll") for (int n = 0; n < 2; ++n) _Pragma("unroll") for (int k = 0; k < 2; ++k) dst[n][k] = *(const LAS bf16x8*)(lds + SB(b, h) + boff + n * 2048 + k * 1024); } while (0)
#define MMA(ai, bj, At, Bt) do { __builtin_amdgcn_s_setprio(1); _Pragma("unroll") for (int m = 0; m < 4; ++m) _Pragma("unroll") for (int n = 0; n < 2; ++n) _Pragma("unroll") for (int k = 0; k < 2; ++k) \
        acc[ai][bj][m][n] = __builtin_amdgcn_mfma_f32_16x16x32_bf16(Bt[n][k], At[m][k], acc[ai][bj][m][n], 0, 0, 0); __builtin_amdgcn_s_setprio(0); } while (0)
#define WAIT_V(n) asm volatile("s_waitcnt vmcnt(" #n ")" ::: "memory")
#define WAIT_L(n) asm volatile("s_waitcnt lgkmcnt(" #n ")" ::: "memory")
#define BAR __builtin_amdgcn_s_barrier()
#define SCHED __builtin_amdgcn_sched_barrier(0)

    Tile cur, nxt; int ui = slot;
    if (!tile_of(ui, col, rank, cur)) return;
    bf16x8 At[4][2], B0[2][2], B1[2][2];
    constexpr unsigned SB0 = WS_SND + ((NRANK - 1) * NS + GW) * UNB, STB = U0 * UNB;
    const unsigned k0 = (unsigned)rank * CHB;
    unsigned ua = cur.sA, ub = SB0 + (unsigned)cur.tm * UNB;
    int lvl = ready(cur, 1);
    STAMP(P, 2);
    STAGE(SB(0, 0), ub); STAGE(SA(0, 0), ua); STAGE(SB(0, 1), ub + HTB); STAGE(SA(0, 1), ua + HTB);
    if (wr == 1) BAR;
    WAIT_V(4); BAR;
    STAGE(SB(1, 0), ub + KTB); STAGE(SA(1, 0), ua + KTB); STAGE(SB(1, 1), ub + HTB + KTB);
    WAIT_V(6); BAR;
#pragma unroll 1
    for (;;) {
        f32x4 acc[2][2][4][2];
#pragma unroll
        for (int a = 0; a < 2; ++a)
#pragma unroll
            for (int b = 0; b < 2; ++b)
#pragma unroll
                for (int m = 0; m < 4; ++m)
#pragma unroll
                    for (int n = 0; n < 2; ++n) acc[a][b][m][n] = (f32x4){0.f, 0.f, 0.f, 0.f};
        const bool has_next = tile_of(ui + KX, col, rank, nxt);
        if (!has_next) nxt = cur;
        const unsigned cA = WS_PAN + (unsigned)(NTM + cur.v) * PANB, cB = WS_PAN + (unsigned)cur.tm * PANB;
        unsigned ko = k0;
        int jchk = lvl >= 5 ? 31 : 7 * lvl - 4;
        for (int j = 0; j < KT / 2; ++j) {
            const bool last = (j == KT / 2 - 1);
            if (j == jchk && (has_next || !last)) {
                lvl = ready(last ? nxt : cur, last ? 1 : lvl + 1);
                jchk = lvl >= 5 ? 31 : 7 * lvl - 4;
            }
            const unsigned a1 = ua + KTB;
            if (j < 3) { ua += cur.stA; ub += STB; }
            else if (last) { ua = nxt.sA; ub = SB0 + (unsigned)nxt.tm * UNB; }
            else {
                ko = (ko + CHB) & (PANB - 1);
                if ((ko ^ k0) < CHB) ko = (ko + CHB + UNB) & (PANB - 1);
                ua = cA + ko; ub = cB + ko;
            }
            const unsigned a2 = ua, b2 = ub, a3 = a2 + KTB, b3 = b2 + KTB;
            LDB(B0, 0, 0); SCHED; LDA(At, 0, 0); STAGE(SA(1, 1), a1 + HTB);
            WAIT_L(8); BAR; WAIT_L(0); MMA(0, 0, At, B0); BAR; SCHED;
            LDB(B1, 0, 1); STAGE(SB(0, 0), b2);
            BAR; WAIT_L(0); MMA(0, 1, At, B1); BAR;
            LDA(At, 0, 1); STAGE(SA(0, 0), a2);
            BAR; WAIT_L(0); MMA(1, 0, At, B0); BAR; SCHED;
            STAGE(SB(0, 1), b2 + HTB);
            WAIT_V(6); BAR; MMA(1, 1, At, B1); BAR;
            LDB(B0, 1, 0); SCHED; LDA(At, 1, 0); STAGE(SA(0, 1), a2 + HTB);
            WAIT_L(8); BAR; WAIT_L(0); MMA(0, 0, At, B0); BAR; SCHED;
            LDB(B1, 1, 1); STAGE(SB(1, 0), b3);
            BAR; WAIT_L(0); MMA(0, 1, At, B1); BAR;
            LDA(At, 1, 1); STAGE(SA(1, 0), a3);
            BAR; WAIT_L(0); MMA(1, 0, At, B0); BAR; SCHED;
            STAGE(SB(1, 1), b3 + HTB);
            WAIT_V(6); BAR; MMA(1, 1, At, B1);
            if (!(last && wr == 1)) BAR;
        }
        {
            int tz = threadIdx.x; asm volatile("" : "+v"(tz));
            const int ln = tz & 63, fr = ln & 15, fq = ln >> 4;
            const rsrc_t rol = mk_rsrc(P.out[rank], OUTB);
            const bool bad = ctl[0] != 0u, shared = cur.v < NS;
            if (CFG_STAG && ui < KX - 1) {
                const unsigned long long t0 = __builtin_amdgcn_s_memrealtime();
                while (__builtin_amdgcn_s_memrealtime() - t0 < (unsigned long long)(((col & 3) * 2 + (col >> 2)) * CFG_STAG)) __builtin_amdgcn_s_sleep(8);
            }
            const bool thru = CFG_LAST && !has_next;
            const unsigned vo = (unsigned)(wc * 32 + fq * 4) * (NN * 2) + (unsigned)(wr * 256 + fr * 16);
            const unsigned so = (unsigned)(cur.tm * 256) * (NN * 2) + (unsigned)gpan_of(cur.v, rank) * 512u;
            const unsigned vs = (unsigned)(wc * 32 + fq * 4) * ROWB + (unsigned)(wr * 256 + fr * 16);
            const unsigned ss = WS_OSN + (unsigned)(cur.tm * 256) * ROWB + (unsigned)cur.v * 512u;
#pragma unroll
            for (int bj = 0; bj < 2; ++bj)
#pragma unroll
                for (int n = 0; n < 2; ++n)
#pragma unroll
                    for (int r = 0; r < 4; ++r) {
                        u32x4 o;
                        o[0] = cvt_pk_bf16(silu(acc[0][bj][0][n][r]), silu(acc[0][bj][1][n][r]));
                        o[1] = cvt_pk_bf16(silu(acc[0][bj][2][n][r]), silu(acc[0][bj][3][n][r]));
                        o[2] = cvt_pk_bf16(silu(acc[1][bj][0][n][r]), silu(acc[1][bj][1][n][r]));
                        o[3] = cvt_pk_bf16(silu(acc[1][bj][2][n][r]), silu(acc[1][bj][3][n][r]));
                        if (bad) o = (u32x4){0x7fc07fc0u, 0x7fc07fc0u, 0x7fc07fc0u, 0x7fc07fc0u};
                        if (shared) { STORE16(o, rws, vs, ss + (unsigned)(bj * 128 + n * 16 + r) * ROWB, AUX_SYS);
                                      if (CFG_STH) { WAIT_VM0(); __builtin_amdgcn_s_sleep(CFG_STH ? CFG_STH - 1 : 0); } }
                        if (CFG_ETH) __builtin_amdgcn_s_sleep(CFG_ETH);
                        if (thru) { STORE16(o, rol, vo, so + (unsigned)(bj * 128 + n * 16 + r) * (NN * 2), AUX_SYS);
                                    if (CFG_LAST > 1) { WAIT_VM0(); __builtin_amdgcn_s_sleep(CFG_LAST > 1 ? CFG_LAST - 1 : 1); } }
                        else STORE16(o, rol, vo, so + (unsigned)(bj * 128 + n * 16 + r) * (NN * 2), CFG_RST);
                    }
            if (shared) {
                WAIT_VM0();
                unsigned old = 0;
                if (ln == 0) old = __hip_atomic_fetch_add(ctl + 1, 1u, __ATOMIC_RELAXED, __HIP_MEMORY_SCOPE_WORKGROUP);
                old = __builtin_amdgcn_readfirstlane(old);
                if ((old & 7u) == 7u && ln < NRANK && ln != rank)
                    st_sys((unsigned*)(P.ws[ln] + WS_FLAG + rank * FLAGB) + NIN * 4 + cur.tm * NS + cur.v, call);
            }
        }
        STAMP(P, 3 + (ui >= KX ? 1 : 0));
        if (!has_next) break;
        if (wr == 1) BAR;
        cur = nxt; ui += KX;
    }
    WAIT_V(0);
}

__global__ __launch_bounds__(512, 2) void fused_gemm_ar(Params P) {
    extern __shared__ __attribute__((aligned(16))) unsigned char shm[];
    LAS unsigned char* lds = (LAS unsigned char*)shm;
    const int wid = __builtin_amdgcn_readfirstlane(threadIdx.x >> 6), lane = threadIdx.x & 63;
    const int b = __builtin_amdgcn_readfirstlane(blockIdx.x);
    unsigned* tkp = (unsigned*)(P.ws[P.rank] + WS_TICKET + b * 128 + wid * 16);
    const unsigned tk = ld_agent(tkp);
    if (b < NPULL) {
        const unsigned call = __builtin_amdgcn_readfirstlane(tk) + 1u;
        st_agent(tkp, call);
        const int pi = b / PW, slot = b % PW;
        const int peer = (P.rank + 1 + pi) & (NRANK - 1);
        pull_role(P, call, peer, slot * 8 + wid, lane);
    } else {
        const int g = b - NPULL;
        ((LAS unsigned*)(lds + STAGE_BYTES))[lane & 3] = 0u;
        STAMP(P, 0);
        pack_role(P, tk, g, wid, lane);
        const unsigned call = __builtin_amdgcn_readfirstlane(tk) + 1u;
        st_agent(tkp, call);
        STAMP(P, 1);
        asm volatile("s_waitcnt lgkmcnt(0)" ::: "memory");
        __builtin_amdgcn_s_barrier();
        if (b >= NPS) gemm_role(P, call, b & 7, (b - NPS) >> 3, lds);
    }
}
}

extern "C" void kernel_launch(int rank, int nranks, ncclComm_t comm,
                              hipStream_t stream,
                              void* const* d_in, const int* in_sizes, int n_in,
                              void* d_out, int out_size,
                              const int* in_dtypes, int out_dtype,
                              void* const* d_out_all,
                              void* const* d_ws_all, size_t ws_bytes) {
    if (nranks != NRANK || (unsigned)rank >= (unsigned)NRANK || ws_bytes < (size_t)WS_END || n_in < 2 ||
        in_sizes[0] != MM * KSH || in_sizes[1] != KSH * NN || out_size != MM * NN) {
        fprintf(stderr, "unexpected problem geometry\n");
        return;
    }
    static bool attr_done[NRANK] = {};
    if (!attr_done[rank]) {
        HIP_CHECK(hipFuncSetAttribute((const void*)fused_gemm_ar, hipFuncAttributeMaxDynamicSharedMemorySize, LDS_BYTES));
        attr_done[rank] = true;
    }
    Params P;
    P.x = d_in[0]; P.w = d_in[1]; P.rank = rank;
    for (int p = 0; p < NRANK; ++p) { P.out[p] = (char*)d_out_all[p]; P.ws[p] = (char*)d_ws_all[p]; }
    hipLaunchKernelGGL(fused_gemm_ar, dim3(GRID), dim3(512), LDS_BYTES, stream, P);
    HIP_CHECK(hipGetLastError());
}
```

```cpp
#include <hip/hip_runtime.h>
#include <rccl/rccl.h>
#include <cstdio>

#define HIP_CHECK(expr) do { hipError_t _e = (expr); if (_e != hipSuccess) { \
    fprintf(stderr, "HIP error %s at %s:%d\n", hipGetErrorString(_e), __FILE__, __LINE__); \
    return; } } while (0)

#define LAS __attribute__((address_space(3)))
typedef short bf16x8 __attribute__((ext_vector_type(8)));
typedef float f32x4 __attribute__((ext_vector_type(4)));
typedef unsigned u32x4 __attribute__((ext_vector_type(4)));
typedef __amdgpu_buffer_rsrc_t rsrc_t;

namespace {
constexpr int NRANK = 8;
constexpr int MM = 4096, KK = 4096, NN = 32768, KSH = KK / NRANK;
constexpr int KT = KK / 64, KTS = KSH / 64, NTM = MM / 256, NPAN = NN / 256;
#define CFG_NS 8
#define CFG_PW 8
constexpr int NS = CFG_NS;
constexpr int NRP = NPAN - NRANK * NS;
constexpr int NWP = NS + NRP;
constexpr int NIN = NTM + NWP;
constexpr int NOUT = NS * NTM;
constexpr int NFLAG = NIN * 4 + NOUT;
constexpr int NUNIT = NTM * 4 + NPAN * 4;
constexpr int GRID = 256;
constexpr int PW = CFG_PW;
constexpr int NPULL = PW * (NRANK - 1);
constexpr int NW = PW * 8;
constexpr int NPK = GRID - NPULL;
constexpr int NPS = (NPULL + 7) / 8 * 8;
constexpr int KX = (GRID - NPS) / 8;
#define CFG_GW (2 * (KX / 4))
constexpr int GW = CFG_GW;
constexpr int G0 = GW + NTM;
constexpr int NGF = (NWP - GW) / GW, GL = (NWP - GW) % GW;
static_assert(NS <= GW && NWP > GW, "shared panels sit in the first group");
#define CFG_PU 4
constexpr int PU = CFG_PU, PSH = PU == 1 ? 0 : PU == 2 ? 1 : PU == 4 ? 2 : 3;
#define CFG_PUSH 3
constexpr int NPUSH = CFG_PUSH;
constexpr int NPIN = (NIN * 4 - NPUSH) * 64 / PU;
static_assert(NPUSH <= NS, "pushed units have a single consumer");
constexpr int NPC = NPIN + NOUT * 128 / PU;
constexpr unsigned CNT_FULL = 64;

constexpr bool SPLIT = 2 * (4 * ((NWP + 1) / 2) % KX) <= KX && 4 * ((NWP + 1) / 2) % KX != 0;
constexpr unsigned KTB = 32768, HTB = 16384;
constexpr unsigned PANB = KT * KTB;
constexpr unsigned CHB = KTS * KTB;
constexpr unsigned UNB = 2 * KTB;
constexpr unsigned WS_TICKET = 0;
constexpr unsigned WS_TMO = WS_TICKET + GRID * 128;
constexpr unsigned WS_FLAG = WS_TMO + 128;
constexpr unsigned FLAGB = 4096;
constexpr unsigned WS_DONE = WS_FLAG + NRANK * FLAGB;
constexpr unsigned WS_HEAD = WS_DONE + NRANK * 512;
constexpr unsigned WS_CNT = WS_HEAD + NRANK * 128;
constexpr unsigned WS_CTRL_END = WS_CNT + NIN * NRANK * 128;
constexpr unsigned WS_DBG = 512u << 10;
constexpr unsigned WS_SND = 1u << 20;
constexpr unsigned ROWB = NS * 512;
constexpr unsigned WS_OSN = WS_SND + NUNIT * UNB;
constexpr unsigned WS_PAN = (WS_OSN + MM * ROWB + PANB - 1) / PANB * PANB;
constexpr unsigned WS_END = WS_PAN + NIN * PANB;
constexpr int U0 = G0 + (NRANK - 1) * NS;
constexpr unsigned OUTB = (unsigned)MM * NN * 2;
static_assert(WS_CTRL_END <= WS_DBG, "ctrl region");
static_assert(NFLAG <= 512 && 2048 <= (int)FLAGB, "flag region");
static_assert(NW <= 128, "done words");
static_assert(PU == 4, "block size");
#define CFG_DEEP 3
constexpr unsigned STAGE_BYTES = 8 * HTB;
constexpr unsigned LDS_BYTES = STAGE_BYTES + 64;
constexpr unsigned long long DEADLINE = 50000000ull;
constexpr int AUX_SYS = 17, AUX_SYS_NT = 19, AUX_AGENT = 16;
#define CFG_PACE 40
#define CFG_PACE0 40
#define CFG_HOLD 0
#define CFG_STH 0
#define CFG_ETH 0
#define CFG_STAG 300
#define CFG_P3 320
#define CFG_P3N 2
#define CFG_PH 4
#define CFG_EB 0
#define CFG_SILU 1
#define CFG_LOOK 256
#define CFG_TAIL NW
#define CFG_LAST 8
#define CFG_RST 0
#define CFG_RATE 1720
#define CFG_CRED 40
constexpr unsigned PACE16 = CFG_RATE * PW * (8 + CFG_DEEP) / 64;

struct Params {
    const void* x; const void* w;
    char* out[NRANK]; char* ws[NRANK];
    int rank;
};

__device__ __forceinline__ rsrc_t mk_rsrc(const void* p, unsigned bytes) {
    const unsigned long long v = (unsigned long long)p;
    const unsigned lo = __builtin_amdgcn_readfirstlane((unsigned)v), hi = __builtin_amdgcn_readfirstlane((unsigned)(v >> 32));
    return __builtin_amdgcn_make_buffer_rsrc((void*)(((unsigned long long)hi << 32) | lo), 0, bytes, 0x00020000);
}
__device__ __forceinline__ unsigned ld_sys(const unsigned* p) { return __hip_atomic_load(p, __ATOMIC_RELAXED, __HIP_MEMORY_SCOPE_SYSTEM); }
__device__ __forceinline__ unsigned ld_agent(const unsigned* p) { return __hip_atomic_load(p, __ATOMIC_RELAXED, __HIP_MEMORY_SCOPE_AGENT); }
__device__ __forceinline__ void st_sys(unsigned* p, unsigned v) { __hip_atomic_store(p, v, __ATOMIC_RELAXED, __HIP_MEMORY_SCOPE_SYSTEM); }
__device__ __forceinline__ void st_agent(unsigned* p, unsigned v) { __hip_atomic_store(p, v, __ATOMIC_RELAXED, __HIP_MEMORY_SCOPE_AGENT); }
__device__ __forceinline__ void add_agent(unsigned* p, unsigned v) { (void)__hip_atomic_fetch_add(p, v, __ATOMIC_RELAXED, __HIP_MEMORY_SCOPE_AGENT); }
#define STORE16(v, rs, vo, so, aux) do { const u32x4 _d = (v); __builtin_amdgcn_raw_buffer_store_b128(_d, rs, vo, so, aux); asm volatile("s_nop 1" :: "v"(_d) : "memory"); } while (0)
#define WAIT_VM0() asm volatile("s_waitcnt vmcnt(0)" ::: "memory")
#define WAIT_VMN(n) do { asm volatile("" ::: "memory"); __builtin_amdgcn_s_waitcnt(0x0F70 | (n)); asm volatile("" ::: "memory"); } while (0)

__device__ __forceinline__ bool give_up(unsigned* tmo, unsigned call, unsigned long long t0) {
    const unsigned long long dt = __builtin_amdgcn_s_memrealtime() - t0;
    if (dt < 2000ull) return false;
    if ((unsigned)__builtin_amdgcn_readfirstlane(ld_agent(tmo)) == call) return true;
    if (dt > DEADLINE) { st_agent(tmo, call); return true; }
    return false;
}

__device__ __forceinline__ int gpan_of(int v, int consumer) { return v < NS ? consumer * NS + v : (NRANK - 1) * NS + v; }
__device__ __forceinline__ int unit_of(int o, int consumer, int& q, int& pos) {
    if (o < 4 * G0) { q = o / G0; const int m = o % G0; pos = q * U0 + gpan_of(m, consumer); return m < GW ? NTM + m : m - GW; }
    pos = o + 4 * (NRANK - 1) * NS;
    o -= 4 * G0;
    const int g = o / (4 * GW), r = o % (4 * GW);
    int i;
    if (GL == 0 || g < NGF) { q = r / GW; i = r % GW; } else { q = r / (GL ? GL : 1); i = r % (GL ? GL : 1); }
    return NTM + GW * (g + 1) + i;
}

__device__ __forceinline__ unsigned cvt_pk_bf16(float lo, float hi) { unsigned r; asm("v_cvt_pk_bf16_f32 %0, %1, %2" : "=v"(r) : "v"(lo), "v"(hi)); return r; }
__device__ __forceinline__ float silu(float y) {
    const float c = __builtin_amdgcn_fmed3f(y, -60.0f, 60.0f);
    const float e = __builtin_amdgcn_exp2f(c * -1.44269504088896f);
    return y * __builtin_amdgcn_rcpf(1.0f + e);
}

typedef float f32x2 __attribute__((ext_vector_type(2)));
__device__ __forceinline__ f32x2 silu2(f32x2 y) {
    const f32x2 t = __builtin_elementwise_fma(y, (f32x2){-1.44269504088896f, -1.44269504088896f}, (f32x2){-86.5617024533378f, -86.5617024533378f});
    const f32x2 e = {__builtin_amdgcn_fmed3f(__builtin_amdgcn_exp2f(t[0]), 0.0f, 1.0f), __builtin_amdgcn_fmed3f(__builtin_amdgcn_exp2f(t[1]), 0.0f, 1.0f)};
    const f32x2 d = __builtin_elementwise_fma(e, (f32x2){1.1420073898156842e26f, 1.1420073898156842e26f}, (f32x2){1.0f, 1.0f});
    return y * (f32x2){__builtin_amdgcn_rcpf(d[0]), __builtin_amdgcn_rcpf(d[1])};
}

#define STAMP1(P, slot) do {} while (0)
#define STAMP(P, slot) do {} while (0)

__device__ __forceinline__ void pull_role(const Params& P, unsigned call, int peer, int widx, int lane) {
    const int rank = P.rank;
    char* wsl = P.ws[rank];
    const rsrc_t rwr = mk_rsrc(P.ws[peer], WS_END), rwl = mk_rsrc(wsl, WS_END);
    const rsrc_t rol = mk_rsrc(P.out[rank], OUTB);
    unsigned* tmo = (unsigned*)(wsl + WS_TMO);
    constexpr unsigned OOB = 0xfffffff0u;
    const unsigned lane0 = lane == 0 ? 0u : OOB;
    const unsigned hbase = (call - 1u) * (unsigned)(NPC + NW);
    auto draw = [&]() { return (unsigned)__builtin_amdgcn_raw_ptr_buffer_atomic_add_i32(1, rwl, lane0, WS_HEAD + (unsigned)peer * 128u, 0); };
    auto arrive = [&](unsigned vo, unsigned cnt) { (void)__builtin_amdgcn_raw_ptr_buffer_atomic_add_i32(PU, rwl, vo, cnt, 0); };
    const bool deep0 = (widx & 7) < CFG_DEEP;
    bool bad = false; int stamped = 0; (void)stamped;
    STAMP(P, 0);
    auto index_of = [&](unsigned v) { return (int)((unsigned)__builtin_amdgcn_readfirstlane(v) - hbase); };
    const unsigned long long tb = __builtin_amdgcn_s_memrealtime();
    int cur = index_of(draw());
    while (CFG_HOLD > 0 && __builtin_amdgcn_s_memrealtime() - tb < (unsigned long long)CFG_HOLD) __builtin_amdgcn_s_sleep(2);
    constexpr int BPB = NS * 128 / PU;
    const unsigned fso = WS_FLAG + (unsigned)peer * FLAGB;
    auto first_of = [&](int i) { return i < NPIN ? (i >> (6 - PSH)) + NPUSH : NIN * 4 + (i - NPIN) / BPB * NS; };
    auto limit_of = [&](const u32x4& f0, const u32x4& f1, int i) {
        const int first = first_of(i);
        unsigned un = 0;
#pragma unroll
        for (int j = 0; j < 8; ++j) un |= ((int)((j < 4 ? f0[j & 3] : f1[j & 3]) - call) < 0 && lane * 8 + j >= first ? 1u : 0u) << j;
        const int l = __builtin_ctzll(__ballot(un != 0u) | (1ull << 63));
        const int u = l * 8 + __builtin_ctz(__builtin_amdgcn_readlane(un, l) | 256u);
        return u <= NIN * 4 ? (u - NPUSH) << (6 - PSH) : u >= NFLAG ? NPC : NPIN + (u - NIN * 4) / NS * BPB;
    };
    auto offs_of = [&](int i, unsigned (&o)[4], unsigned& cnt) -> unsigned {
        if (i < NPIN) {
            int q, pos; const int pan = unit_of((i >> (6 - PSH)) + NPUSH, rank, q, pos);
            cnt = WS_CNT + (unsigned)(pan * NRANK + peer) * 128u + (unsigned)q * 4u;
            const unsigned sub = (unsigned)((i << PSH) & 63) * 1024u;
            const unsigned b = WS_PAN + (unsigned)pan * PANB + (unsigned)peer * CHB + (unsigned)q * UNB + sub + (unsigned)lane * 16u;
#pragma unroll
            for (int k = 0; k < 4; ++k) o[k] = b + k * 1024u;
            return WS_SND + (unsigned)pos * UNB + sub;
        }
        cnt = 0;
#pragma unroll
        for (int k = 0; k < 4; ++k) {
            const unsigned sg = (unsigned)(((i - NPIN) << PSH) + k) * 2u + (unsigned)(lane >> 5);
            o[k] = (unsigned)peer * ROWB + (unsigned)(lane & 31) * 16u + sg / NS * (NN * 2u) + sg % NS * 512u;
        }
        return WS_OSN + (unsigned)((i - NPIN) << PSH) * 1024u;
    };
    int lim = 0;
    while ((unsigned)cur < (unsigned)NPC) {
        if (!bad) {
            const unsigned long long t0 = __builtin_amdgcn_s_memrealtime();
            const unsigned pvo = (unsigned)(lane - (first_of(cur) >> 3)) < 4u ? (unsigned)lane * 32u : OOB - 16u;
            for (;;) {
                const u32x4 f0 = __builtin_amdgcn_raw_buffer_load_b128(rwl, pvo, fso, AUX_SYS), f1 = __builtin_amdgcn_raw_buffer_load_b128(rwl, pvo + 16u, fso, AUX_SYS);
                lim = limit_of(f0, f1, cur);
                if (cur < lim) break;
                if (give_up(tmo, call, t0)) { bad = true; break; }
                __builtin_amdgcn_s_sleep(2);
            }
            asm volatile("" ::: "memory");
        }
        unsigned o[4], p2 = OOB, p3 = OOB, va = OOB, cprev = 0;
        if (bad) {
            (void)offs_of(cur, o, cprev);
            if (cur >= NPIN) {
#pragma unroll
                for (int k = 0; k < PU; ++k) __builtin_amdgcn_raw_buffer_store_b128((u32x4){0x7fc07fc0u, 0x7fc07fc0u, 0x7fc07fc0u, 0x7fc07fc0u}, rol, o[k], 0, AUX_AGENT);
            }
            cur = index_of(draw()); continue;
        }
        STAMP1(P, 1);
        rsrc_t rp = rwl;
        u32x4 v[4] = {};
        const unsigned per = deep0 ? PACE16 / 2 : PACE16;
        unsigned slot = 0;
        if (CFG_RATE) { const unsigned now = (unsigned)__builtin_amdgcn_s_memrealtime() << 4; slot = now - (now & 0xffffffu) % PACE16 + (unsigned)((widx & 7) * PW + (widx >> 3)) * (PACE16 / NW) - PACE16; }
        do {
            unsigned cb; const unsigned src = offs_of(cur, o, cb);
            const bool in = cur < NPIN, deep = deep0 || cur >= NPC - CFG_TAIL;
            const rsrc_t rd = in ? rwl : rol;
            const bool look = lim < NPC && cur + CFG_LOOK >= lim;
            const unsigned fvo = look ? (unsigned)lane * 32u : OOB - 16u;
            unsigned nv = 0; u32x4 f0, f1;
#pragma unroll
            for (int k = 0; k < 4; ++k) {
                if (CFG_RATE && cur < NPC - CFG_TAIL) {
                    slot += per;
                    for (;;) {
                        const int d = (int)(((unsigned)__builtin_amdgcn_s_memrealtime() << 4) - slot);
                        if (d > CFG_CRED * 16) slot += (unsigned)(d - CFG_CRED * 16);
                        if (d >= 0) break;
                        __builtin_amdgcn_s_sleep(1);
                    }
                }
                v[k] = __builtin_amdgcn_raw_buffer_load_b128(rwr, lane * 16 + k * 1024, src, AUX_SYS_NT);
                if (k == 0) {
                    nv = draw();
                    f0 = __builtin_amdgcn_raw_buffer_load_b128(rwl, fvo, fso, AUX_SYS); f1 = __builtin_amdgcn_raw_buffer_load_b128(rwl, fvo + 16u, fso, AUX_SYS);
                }
                __builtin_amdgcn_raw_buffer_store_b128(v[k ^ 2], k < 2 ? rp : rd, k == 0 ? p2 : k == 1 ? p3 : o[k ^ 2], 0, AUX_AGENT);
                if (k == 0) WAIT_VMN(7); else if (k == 1) WAIT_VMN(6); else WAIT_VMN(3);
                if (!deep) { if (k == 0) WAIT_VMN(4); else WAIT_VMN(1); }
            }
            arrive(va, cprev);
            cprev = cb; va = in ? lane0 : OOB; rp = rd; p2 = o[2]; p3 = o[3];
            cur = index_of(nv);
            if (look) { const int l2 = limit_of(f0, f1, cur); lim = l2 > lim ? l2 : lim; }
        } while ((unsigned)cur < (unsigned)lim);
        __builtin_amdgcn_raw_buffer_store_b128(v[2], rp, p2, 0, AUX_AGENT);
        __builtin_amdgcn_raw_buffer_store_b128(v[3], rp, p3, 0, AUX_AGENT);
        if (cprev) { WAIT_VM0(); arrive(va, cprev); }
    }
    STAMP(P, 4);
    asm volatile("" ::: "memory");
    if (lane == 0) st_sys((unsigned*)(P.ws[peer] + WS_DONE + rank * 512) + widx, call);
    if (widx == 0) {
        const unsigned* dn = (const unsigned*)(wsl + WS_DONE + peer * 512);
        const unsigned long long t0 = __builtin_amdgcn_s_memrealtime();
        for (;;) {
            bool ok = true;
            if (lane < NW) ok = (int)(ld_sys(dn + lane) - call) >= 0;
            if (NW > 64 && lane + 64 < NW) ok = ok && (int)(ld_sys(dn + 64 + lane) - call) >= 0;
            if (__all(ok)) break;
            if (give_up(tmo, call, t0)) {
                if (lane == 0) __builtin_amdgcn_raw_buffer_store_b32(0x7fc07fc0u, rol, (unsigned)(rank * NS) * 512u, 0, AUX_SYS);
                break;
            }
            __builtin_amdgcn_s_sleep(2);
        }
        STAMP(P, 5);
    }
}

__device__ __forceinline__ void pack_role(const Params& P, unsigned tk, int g, int wid, int lane) {
    const int rank = P.rank;
    char* wsl = P.ws[rank];
    const rsrc_t rx = mk_rsrc(P.x, (unsigned)MM * KSH * 2), rw = mk_rsrc(P.w, (unsigned)KSH * NN * 2);
    const int fr = lane & 15, fq = lane >> 4;
    const unsigned long long t0 = __builtin_amdgcn_s_memrealtime();
    for (int u = g; u < NUNIT; u += NPK) {
        int o, dest = -1;
        if (u < 4 * U0) {
            const int r = u % U0;
            if (r < NRANK * NS) { o = r >> 3; dest = (rank + 1 + r) & (NRANK - 1); } else o = r - (NRANK - 1) * NS;
            o += u / U0 * G0;
        } else o = u - 4 * (NRANK - 1) * NS;
        int q, pos; const int pan = unit_of(o, dest < 0 ? 0 : dest, q, pos);
        const bool push = o < NPUSH && dest != rank;
        const rsrc_t rwl = mk_rsrc(P.ws[push ? dest & (NRANK - 1) : rank], WS_END);
        while (CFG_PACE > 0 && (unsigned)(__builtin_amdgcn_s_memrealtime() - t0) < (unsigned)((u < CFG_PACE0 ? 0 : u - CFG_PACE0) * CFG_PACE / 10)) __builtin_amdgcn_s_sleep(1);
        const int ktl = q * 2 + (wid >> 2);
        const unsigned dst = push ? WS_PAN + (unsigned)pan * PANB + (unsigned)(rank * KTS + ktl) * KTB : WS_SND + (unsigned)pos * UNB + (unsigned)(wid >> 2) * KTB;
        u32x4 v[8];
        if (pan < NTM) {
            const int rb0 = (wid & 3) * 4;
            const unsigned vo = (unsigned)fr * (KSH * 2) + (unsigned)fq * 16u;
            const unsigned so = (unsigned)(pan * 256 + rb0 * 16) * (KSH * 2) + (unsigned)ktl * 128u;
#pragma unroll
            for (int i = 0; i < 8; ++i) v[i] = __builtin_amdgcn_raw_buffer_load_b128(rx, vo + (i & 1) * 64, so + (unsigned)(i >> 1) * (16 * KSH * 2), 0);
#pragma unroll
            for (int i = 0; i < 8; ++i) STORE16(v[i], rwl, lane * 16 + (i & 1) * 1024, dst + (unsigned)(rb0 + (i >> 1)) * 2048u, AUX_SYS);
        } else {
            const int ks = (wid >> 1) & 1, wr = wid & 1;
            const unsigned vo = (unsigned)(fq * 8) * (NN * 2) + (unsigned)fr * 16u;
            const unsigned so = (unsigned)(ktl * 64 + ks * 32) * (NN * 2) + (unsigned)(gpan_of(pan - NTM, dest < 0 ? 0 : dest) * 256 + wr * 128) * 2u;
#pragma unroll
            for (int j = 0; j < 8; ++j) v[j] = __builtin_amdgcn_raw_buffer_load_b128(rw, vo, so + (unsigned)j * (NN * 2), 2);
            while (CFG_P3 && push && o >= CFG_P3N && (unsigned)(__builtin_amdgcn_s_memrealtime() - t0) < (unsigned)(CFG_P3 + (o - CFG_P3N) * 110)) __builtin_amdgcn_s_sleep(1);
#pragma unroll
            for (int i = 0; i < 8; ++i) {
                u32x4 o;
#pragma unroll
                for (int e = 0; e < 4; ++e) o[e] = __builtin_amdgcn_perm(v[2 * e + 1][i >> 1], v[2 * e][i >> 1], (i & 1) ? 0x07060302u : 0x05040100u);
                STORE16(o, rwl, lane * 16, dst + (unsigned)(i >> 2) * HTB + (unsigned)wr * 8192u + (unsigned)(i & 3) * 2048u + (unsigned)ks * 1024u, AUX_SYS);
            }
        }
        WAIT_VM0();
        if (u == g) STAMP(P, 6);
        __builtin_amdgcn_s_barrier();
        if (wid == 0 && lane < NRANK && (dest < 0 || dest == lane)) {
            unsigned c = tk; asm volatile("" : "+v"(c));
            if (lane == rank || push) st_sys((unsigned*)(P.ws[lane] + WS_CNT) + (pan * NRANK + rank) * 32 + q, (c + 1u) * CNT_FULL);
            else st_sys((unsigned*)(P.ws[lane] + WS_FLAG + rank * FLAGB) + o, c + 1u);
        }
    }
}

struct Tile { int tm, v, half; unsigned sA, stA; };
__device__ __forceinline__ bool tile_of(int first, int slot, int col, int rank, Tile& t) {
    const int nt = 4 * ((NWP + 1 - (col >> 2)) >> 1);
    const bool split = SPLIT && 2 * (nt - first) <= KX;
    const int idx = first + (split ? slot >> 1 : slot);
    t.half = split ? slot & 1 : -1;
    const int v = 2 * (idx >> 2) + (col >> 2);
    if (idx >= nt) return false;
    t.tm = 4 * (col & 3) + (idx & 3); t.v = v;
    const int g = v / GW, i = v % GW;
    t.sA = WS_SND + (unsigned)(g == 0 ? gpan_of(v, rank) : 4 * U0 + (g - 1) * 4 * GW + i) * UNB;
    t.stA = (unsigned)(g == 0 ? U0 : g <= NGF ? GW : GL) * UNB;
    return true;
}

__device__ __forceinline__ void gemm_role(const Params& P, unsigned call, int col, int slot, LAS unsigned char* lds) {
    const int tid = threadIdx.x, wid = __builtin_amdgcn_readfirstlane(tid >> 6), lane = tid & 63, wr = wid >> 2, wc = wid & 3;
    const int rank = P.rank;
    char* wsl = P.ws[rank];
    const rsrc_t rws = mk_rsrc(wsl, WS_END);
    LAS unsigned* ctl = (LAS unsigned*)(lds + STAGE_BYTES);
    const unsigned voff = (unsigned)tid * 16u;
    const unsigned ldsw = (unsigned)wid * 1024u;
    const int aoff = wr * 8192 + lane * 16, boff = wc * 4096 + lane * 16;

    auto ready = [&](const Tile& t, int want) -> int {
        if (wid == 0) {
            const unsigned* c = (const unsigned*)(wsl + WS_CNT) + ((((lane & 32) ? NTM + t.v : t.tm) * NRANK + ((lane >> 2) & 7)) * 32 + (lane & 3));
            unsigned* tmo = (unsigned*)(wsl + WS_TMO);
            const unsigned need = call * CNT_FULL;
            const unsigned long long t0 = __builtin_amdgcn_s_memrealtime();
            int got;
            for (;;) {
                const unsigned long long miss = __ballot((int)(ld_sys(c) - need) < 0);
                unsigned m = (unsigned)miss | (unsigned)(miss >> 32);
                const bool own = ((m >> (rank * 4)) & 15u) == 0u;
                m |= m >> 16; m |= m >> 8; m |= m >> 4;
                got = own ? 1 + __builtin_ctz(m | 16u) : 0;
                if (got >= want) break;
                if (give_up(tmo, call, t0)) { ctl[0] = 1u; got = 5; break; }
                __builtin_amdgcn_s_sleep(4);
            }
            ctl[2] = (unsigned)got;
            __builtin_amdgcn_fence(__ATOMIC_ACQUIRE, "agent");
            asm volatile("s_waitcnt vmcnt(0) lgkmcnt(0)" ::: "memory");
        }
        asm volatile("" ::: "memory"); __builtin_amdgcn_s_barrier(); asm volatile("" ::: "memory");
        const int r = __builtin_amdgcn_readfirstlane(ctl[2]);
        asm volatile("s_waitcnt lgkmcnt(0)" ::: "memory");
        return r;
    };

#define SA(b, h) (((b) * 2 + (h)) * HTB)
#define SB(b, h) ((4 + (b) * 2 + (h)) * HTB)
#define STAGE(bufoff, soff) do { _Pragma("unroll") for (int _i = 0; _i < 2; ++_i) \
        __builtin_amdgcn_raw_ptr_buffer_load_lds(rws, (LAS unsigned*)(lds + (bufoff) + ldsw + _i * 8192), 16, voff, (soff) + _i * 8192u, 0, 0); } while (0)
#define LDA(dst, b, h) do { _Pragma("unroll") for (int m = 0; m < 4; ++m) _Pragma("unroll") for (int k = 0; k < 2; ++k) dst[m][k] = *(const LAS bf16x8*)(lds + SA(b, h) + aoff + m * 2048 + k * 1024); } while (0)
#define LDB(dst, b, h) do { _Pragma("unroll") for (int n = 0; n < 2; ++n) _Pragma("unroll") for (int k = 0; k < 2; ++k) dst[n][k] = *(const LAS bf16x8*)(lds + SB(b, h) + boff + n * 2048 + k * 1024); } while (0)
#define MMA(ai, bj, At, Bt) do { __builtin_amdgcn_s_setprio(1); _Pragma("unroll") for (int m = 0; m < 4; ++m) _Pragma("unroll") for (int n = 0; n < 2; ++n) _Pragma("unroll") for (int k = 0; k < 2; ++k) \
        acc[ai][bj][m][n] = __builtin_amdgcn_mfma_f32_16x16x32_bf16(Bt[n][k], At[m][k], acc[ai][bj][m][n], 0, 0, 0); __builtin_amdgcn_s_setprio(0); } while (0)
#define MMA2(ai, bar) do { __builtin_amdgcn_s_setprio(1); \
        if (!SPLIT || cur.half != 1) { _Pragma("unroll") for (int i = 0; i < 16; ++i) { const int m = i >> 2, n = (i >> 1) & 1, k = i & 1; \
            acc[ai][0][m][n] = __builtin_amdgcn_mfma_f32_16x16x32_bf16(B0[n][k], At[m][k], acc[ai][0][m][n], 0, 0, 0); } } \
        if (!SPLIT || cur.half != 0) { _Pragma("unroll") for (int i = 0; i < 16; ++i) { const int m = i >> 2, n = (i >> 1) & 1, k = i & 1; \
            if (CFG_EB && i == 16 - CFG_EB) { SCHED; __builtin_amdgcn_s_setprio(3); if (bar) BAR; SCHED; } \
            acc[ai][1][m][n] = __builtin_amdgcn_mfma_f32_16x16x32_bf16(B1[n][k], At[m][k], acc[ai][1][m][n], 0, 0, 0); } } \
        __builtin_amdgcn_s_setprio(0); if ((CFG_EB == 0 || (SPLIT && cur.half == 0)) && (bar)) BAR; SCHED; } while (0)
#define WAIT_L0() do { asm volatile("" ::: "memory"); __builtin_amdgcn_s_waitcnt(0xC07F); asm volatile("" ::: "memory"); } while (0)
#define WAIT_V(n) asm volatile("s_waitcnt vmcnt(" #n ")" ::: "memory")
#define WAIT_L(n) asm volatile("s_waitcnt lgkmcnt(" #n ")" ::: "memory")
#define BAR __builtin_amdgcn_s_barrier()
#define SCHED __builtin_amdgcn_sched_barrier(0)

    Tile cur, nxt; int ui = 0;
    if (!tile_of(ui, slot, col, rank, cur)) return;
    bf16x8 At[4][2], B0[2][2], B1[2][2];
    constexpr unsigned SB0 = WS_SND + ((NRANK - 1) * NS + GW) * UNB, STB = U0 * UNB;
    const unsigned k0 = (unsigned)rank * CHB;
    unsigned ua = cur.sA, ub = SB0 + (unsigned)cur.tm * UNB;
    int lvl = ready(cur, 1);
    STAMP(P, 2);
    STAGE(SB(0, 0), ub); STAGE(SA(0, 0), ua); STAGE(SB(0, 1), ub + HTB); STAGE(SA(0, 1), ua + HTB);
    if (wr == 1) BAR;
    WAIT_V(4); BAR;
    STAGE(SB(1, 0), ub + KTB); STAGE(SA(1, 0), ua + KTB); STAGE(SB(1, 1), ub + HTB + KTB);
    WAIT_V(0); BAR;
#pragma unroll 1
    for (;;) {
        f32x4 acc[2][2][4][2];
#pragma unroll
        for (int a = 0; a < 2; ++a)
#pragma unroll
            for (int b = 0; b < 2; ++b)
#pragma unroll
                for (int m = 0; m < 4; ++m)
#pragma unroll
                    for (int n = 0; n < 2; ++n) acc[a][b][m][n] = (f32x4){0.f, 0.f, 0.f, 0.f};
        const bool has_next = tile_of(ui + KX, slot, col, rank, nxt);
        if (!has_next) nxt = cur;
        const unsigned cA = WS_PAN + (unsigned)(NTM + cur.v) * PANB, cB = WS_PAN + (unsigned)cur.tm * PANB;
        unsigned ko = k0;
        int jchk = lvl >= 5 ? 31 : 7 * lvl - 4;
        for (int j = 0; j < KT / 2; ++j) {
            const bool last = (j == KT / 2 - 1);
            if (j == jchk && (has_next || !last)) {
                lvl = ready(last ? nxt : cur, last ? 1 : lvl + 1);
                jchk = lvl >= 5 ? 31 : 7 * lvl - 4;
            }
            const unsigned a1 = ua + KTB;
            if (j < 3) { ua += cur.stA; ub += STB; }
            else if (last) { ua = nxt.sA; ub = SB0 + (unsigned)nxt.tm * UNB; }
            else {
                ko = (ko + CHB) & (PANB - 1);
                if ((ko ^ k0) < CHB) ko = (ko + CHB + UNB) & (PANB - 1);
                ua = cA + ko; ub = cB + ko;
            }
            const unsigned a2 = ua, b2 = ub, a3 = a2 + KTB, b3 = b2 + KTB;
            LDB(B0, 0, 0); LDA(At, 0, 0); LDB(B1, 0, 1); STAGE(SA(1, 1), a1 + HTB);
            if (j) WAIT_V(8);
            WAIT_L0(); BAR; MMA2(0, true);
            LDA(At, 0, 1); STAGE(SB(0, 0), b2); STAGE(SA(0, 0), a2); STAGE(SB(0, 1), b2 + HTB);
            if (j) WAIT_V(8);
            WAIT_L0(); BAR; MMA2(1, true);
            LDB(B0, 1, 0); LDA(At, 1, 0); LDB(B1, 1, 1); STAGE(SA(0, 1), a2 + HTB);
            WAIT_V(8); WAIT_L0(); BAR; MMA2(0, true);
            LDA(At, 1, 1); STAGE(SB(1, 0), b3); STAGE(SA(1, 0), a3); STAGE(SB(1, 1), b3 + HTB);
            WAIT_V(8); WAIT_L0(); BAR; MMA2(1, !(last && wr == 1));
        }
        {
            int tz = threadIdx.x; asm volatile("" : "+v"(tz));
            const int ln = tz & 63, fr = ln & 15, fq = ln >> 4;
            const rsrc_t rol = mk_rsrc(P.out[rank], OUTB);
            const bool shared = cur.v < NS;
            if (__builtin_amdgcn_readfirstlane(ctl[0]) != 0u) {
#pragma unroll
                for (int i = 0; i < 32; ++i) acc[i >> 4][(i >> 3) & 1][(i >> 1) & 3][i & 1] = __builtin_bit_cast(f32x4, (u32x4){0x7fc00000u, 0x7fc00000u, 0x7fc00000u, 0x7fc00000u});
            }
            if (CFG_STAG && ui < KX && cur.v < GW) {
                const unsigned long long t0 = __builtin_amdgcn_s_memrealtime();
                while (__builtin_amdgcn_s_memrealtime() - t0 < (unsigned long long)(((col & 3) * 2 + (col >> 2)) * CFG_STAG)) __builtin_amdgcn_s_sleep(8);
            }
            const bool thru = CFG_LAST && !has_next;
            const unsigned vo = (unsigned)(wc * 32 + fq * 4) * (NN * 2) + (unsigned)(wr * 256 + fr * 16);
            const unsigned so = (unsigned)(cur.tm * 256) * (NN * 2) + (unsigned)gpan_of(cur.v, rank) * 512u;
            const unsigned vs = (unsigned)(wc * 32 + fq * 4) * ROWB + (unsigned)(wr * 256 + fr * 16);
            const unsigned ss = WS_OSN + (unsigned)(cur.tm * 256) * ROWB + (unsigned)cur.v * 512u;
#pragma unroll
            for (int bj = 0; bj < 2; ++bj) if (!SPLIT || cur.half != 1 - bj)
#pragma unroll
                for (int n = 0; n < 2; ++n)
#pragma unroll
                    for (int r = 0; r < 4; ++r) {
                        u32x4 o;
                        if ((r & 1) == 0) {
#pragma unroll
                            for (int i = 0; i < 8; ++i) { f32x4& a = acc[i >> 2][bj][i & 3][n]; const f32x2 z = silu2((f32x2){a[r], a[r + 1]}); a[r] = z[0]; a[r + 1] = z[1]; }
                        }
#pragma unroll
                        for (int i = 0; i < 4; ++i) o[i] = cvt_pk_bf16(acc[i >> 1][bj][(i & 1) * 2][n][r], acc[i >> 1][bj][(i & 1) * 2 + 1][n][r]);
                        if (CFG_PH == 4 && bj + n + r == 0) { asm volatile("" : "+v"(o)); WAIT_V(0); }
                        if (shared) { STORE16(o, rws, vs, ss + (unsigned)(bj * 128 + n * 16 + r) * ROWB, AUX_SYS);
                                      if (CFG_STH) { WAIT_VM0(); __builtin_amdgcn_s_sleep(CFG_STH ? CFG_STH - 1 : 0); } }
                        if (CFG_ETH) __builtin_amdgcn_s_sleep(CFG_ETH);
                        if (thru) { STORE16(o, rol, vo, so + (unsigned)(bj * 128 + n * 16 + r) * (NN * 2), AUX_SYS);
                                    if (CFG_LAST > 1) { WAIT_VM0(); __builtin_amdgcn_s_sleep(CFG_LAST > 1 ? CFG_LAST - 1 : 1); } }
                        else STORE16(o, rol, vo, so + (unsigned)(bj * 128 + n * 16 + r) * (NN * 2), CFG_RST);
                    }
            if (shared) {
                WAIT_VM0();
                unsigned old = 0;
                if (ln == 0) old = __hip_atomic_fetch_add(ctl + 1, 1u, __ATOMIC_RELAXED, __HIP_MEMORY_SCOPE_WORKGROUP);
                old = __builtin_amdgcn_readfirstlane(old);
                if ((old & 7u) == 7u && ln < NRANK && ln != rank)
                    st_sys((unsigned*)(P.ws[ln] + WS_FLAG + rank * FLAGB) + NIN * 4 + cur.tm * NS + cur.v, call);
            }
        }
        STAMP(P, 3 + (ui >= KX ? 1 : 0));
        if (!has_next) break;
        if (wr == 1) BAR;
        cur = nxt; ui += KX;
    }
    WAIT_V(0);
}

__global__ __launch_bounds__(512, 2) void fused_gemm_ar(Params P) {
    extern __shared__ __attribute__((aligned(16))) unsigned char shm[];
    LAS unsigned char* lds = (LAS unsigned char*)shm;
    const int wid = __builtin_amdgcn_readfirstlane(threadIdx.x >> 6), lane = threadIdx.x & 63;
    const int b = __builtin_amdgcn_readfirstlane(blockIdx.x);
    unsigned* tkp = (unsigned*)(P.ws[P.rank] + WS_TICKET + b * 128 + wid * 16);
    const unsigned tk = ld_agent(tkp);
    if (b < NPULL) {
        const unsigned call = __builtin_amdgcn_readfirstlane(tk) + 1u;
        st_agent(tkp, call);
        const int pi = b / PW, slot = b % PW;
        const int peer = (P.rank + 1 + pi) & (NRANK - 1);
        pull_role(P, call, peer, slot * 8 + wid, lane);
    } else {
        const int g = b - NPULL;
        ((LAS unsigned*)(lds + STAGE_BYTES))[lane & 3] = 0u;
        STAMP(P, 0);
        pack_role(P, tk, g, wid, lane);
        const unsigned call = __builtin_amdgcn_readfirstlane(tk) + 1u;
        st_agent(tkp, call);
        STAMP(P, 1);
        asm volatile("s_waitcnt lgkmcnt(0)" ::: "memory");
        __builtin_amdgcn_s_barrier();
        if (b >= NPS) gemm_role(P, call, b & 7, (b - NPS) >> 3, lds);
    }
}
}

extern "C" void kernel_launch(int rank, int nranks, ncclComm_t comm,
                              hipStream_t stream,
                              void* const* d_in, const int* in_sizes, int n_in,
                              void* d_out, int out_size,
                              const int* in_dtypes, int out_dtype,
                              void* const* d_out_all,
                              void* const* d_ws_all, size_t ws_bytes) {
    if (nranks != NRANK || (unsigned)rank >= (unsigned)NRANK || ws_bytes < (size_t)WS_END || n_in < 2 ||
        in_sizes[0] != MM * KSH || in_sizes[1] != KSH * NN || out_size != MM * NN) {
        fprintf(stderr, "unexpected problem geometry\n");
        return;
    }
    static bool attr_done[NRANK] = {};
    if (!attr_done[rank]) {
        HIP_CHECK(hipFuncSetAttribute((const void*)fused_gemm_ar, hipFuncAttributeMaxDynamicSharedMemorySize, LDS_BYTES));
        attr_done[rank] = true;
    }
    Params P;
    P.x = d_in[0]; P.w = d_in[1]; P.rank = rank;
    for (int p = 0; p < NRANK; ++p) { P.out[p] = (char*)d_out_all[p]; P.ws[p] = (char*)d_ws_all[p]; }
    hipLaunchKernelGGL(fused_gemm_ar, dim3(GRID), dim3(512), LDS_BYTES, stream, P);
    HIP_CHECK(hipGetLastError());
}
```

```cpp
#include <hip/hip_runtime.h>
#include <rccl/rccl.h>
#include <cstdio>
#include <type_traits>

#define HIP_CHECK(expr) do { hipError_t _e = (expr); if (_e != hipSuccess) { \
    fprintf(stderr, "HIP error %s at %s:%d\n", hipGetErrorString(_e), __FILE__, __LINE__); \
    return; } } while (0)

#define LAS __attribute__((address_space(3)))
typedef short bf16x8 __attribute__((ext_vector_type(8)));
typedef float f32x4 __attribute__((ext_vector_type(4)));
typedef unsigned u32x4 __attribute__((ext_vector_type(4)));
typedef __amdgpu_buffer_rsrc_t rsrc_t;

namespace {
constexpr int NRANK = 8;
constexpr int MM = 4096, KK = 4096, NN = 32768, KSH = KK / NRANK;
constexpr int KT = KK / 64, KTS = KSH / 64, NTM = MM / 256, NPAN = NN / 256;
#define CFG_NS 6
#define CFG_PW 8
constexpr int NS = CFG_NS;
constexpr int NRP = NPAN - NRANK * NS;
constexpr int NWP = NS + NRP;
constexpr int NIN = NTM + NWP;
constexpr int NOUT = NS * NTM;
constexpr int NFLAG = NIN * 4 + NOUT;
constexpr int NUNIT = NTM * 4 + NPAN * 4;
constexpr int GRID = 256;
constexpr int PW = CFG_PW;
constexpr int NPULL = PW * (NRANK - 1);
constexpr int NW = PW * 8;
constexpr int NPK = GRID - NPULL;
constexpr int NPS = (NPULL + 7) / 8 * 8;
constexpr int KX = (GRID - NPS) / 8;
#define CFG_GW (2 * (KX / 4))
constexpr int GW = CFG_GW;
constexpr int G0 = GW + NTM;
constexpr int NGF = (NWP - GW) / GW, GL = (NWP - GW) % GW;
static_assert(NS <= GW && NWP > GW, "shared panels sit in the first group");
#define CFG_PU 4
constexpr int PU = CFG_PU, PSH = PU == 1 ? 0 : PU == 2 ? 1 : PU == 4 ? 2 : 3;
#define CFG_PUSH 3
constexpr int NPUSH = CFG_PUSH;
constexpr int NPIN = (NIN * 4 - NPUSH) * 64 / PU;
static_assert(NPUSH <= NS, "pushed units have a single consumer");
constexpr int NPC = NPIN + NOUT * 128 / PU;
constexpr unsigned CNT_FULL = 64;

constexpr bool SPLIT = 2 * (4 * ((NWP + 1) / 2) % KX) <= KX && 4 * ((NWP + 1) / 2) % KX != 0;
constexpr unsigned KTB = 32768, HTB = 16384;
constexpr unsigned PANB = KT * KTB;
constexpr unsigned CHB = KTS * KTB;
constexpr unsigned UNB = 2 * KTB;
constexpr unsigned WS_TICKET = 0;
constexpr unsigned WS_TMO = WS_TICKET + GRID * 128;
constexpr unsigned WS_FLAG = WS_TMO + 128;
constexpr unsigned FLAGB = 4096;
constexpr unsigned WS_DONE = WS_FLAG + NRANK * FLAGB;
constexpr unsigned WS_HEAD = WS_DONE + NRANK * 512;
constexpr unsigned WS_CNT = WS_HEAD + NRANK * 128;
constexpr unsigned WS_CTRL_END = WS_CNT + NIN * NRANK * 128;
constexpr unsigned WS_DBG = 512u << 10;
constexpr unsigned WS_SND = 1u << 20;
constexpr unsigned ROWB = NS * 512;
constexpr unsigned WS_OSN = WS_SND + NUNIT * UNB;
constexpr unsigned WS_PAN = (WS_OSN + MM * ROWB + PANB - 1) / PANB * PANB;
constexpr unsigned WS_END = WS_PAN + NIN * PANB;
constexpr int U0 = G0 + (NRANK - 1) * NS;
constexpr unsigned OUTB = (unsigned)MM * NN * 2;
static_assert(WS_CTRL_END <= WS_DBG, "ctrl region");
static_assert(NFLAG <= 512 && 2048 <= (int)FLAGB, "flag region");
static_assert(NW <= 128, "done words");
static_assert(PU == 4, "block size");
#define CFG_DEEP 3
constexpr unsigned STAGE_BYTES = 8 * HTB;
constexpr unsigned LDS_BYTES = STAGE_BYTES + 64;
constexpr unsigned long long DEADLINE = 50000000ull;
constexpr int AUX_SYS = 17, AUX_SYS_NT = 19, AUX_AGENT = 16;
#define CFG_PACE 40
#define CFG_PACE0 40
#define CFG_HOLD 0
#define CFG_STAG 300
#define CFG_P3 320
#define CFG_P3N 2
#define CFG_STAG2 0
#define CFG_STAGN 4
#define CFG_LOOK 256
#define CFG_TAIL NW
#define CFG_LAST 1
#define CFG_RST 0
#define CFG_RATE 1720
#define CFG_CRED 40
constexpr unsigned PACE16 = CFG_RATE * PW * (8 + CFG_DEEP) / 64;

struct Params {
    const void* x; const void* w;
    char* out[NRANK]; char* ws[NRANK];
    int rank;
};

__device__ __forceinline__ rsrc_t mk_rsrc(const void* p, unsigned bytes) {
    const unsigned long long v = (unsigned long long)p;
    const unsigned lo = __builtin_amdgcn_readfirstlane((unsigned)v), hi = __builtin_amdgcn_readfirstlane((unsigned)(v >> 32));
    return __builtin_amdgcn_make_buffer_rsrc((void*)(((unsigned long long)hi << 32) | lo), 0, bytes, 0x00020000);
}
__device__ __forceinline__ unsigned ld_sys(const unsigned* p) { return __hip_atomic_load(p, __ATOMIC_RELAXED, __HIP_MEMORY_SCOPE_SYSTEM); }
__device__ __forceinline__ unsigned ld_agent(const unsigned* p) { return __hip_atomic_load(p, __ATOMIC_RELAXED, __HIP_MEMORY_SCOPE_AGENT); }
__device__ __forceinline__ void st_sys(unsigned* p, unsigned v) { __hip_atomic_store(p, v, __ATOMIC_RELAXED, __HIP_MEMORY_SCOPE_SYSTEM); }
__device__ __forceinline__ void st_agent(unsigned* p, unsigned v) { __hip_atomic_store(p, v, __ATOMIC_RELAXED, __HIP_MEMORY_SCOPE_AGENT); }
__device__ __forceinline__ void add_agent(unsigned* p, unsigned v) { (void)__hip_atomic_fetch_add(p, v, __ATOMIC_RELAXED, __HIP_MEMORY_SCOPE_AGENT); }
#define STORE16(v, rs, vo, so, aux) do { const u32x4 _d = (v); __builtin_amdgcn_raw_buffer_store_b128(_d, rs, vo, so, aux); asm volatile("s_nop 1" :: "v"(_d) : "memory"); } while (0)
#define WAIT_VM0() asm volatile("s_waitcnt vmcnt(0)" ::: "memory")
#define WAIT_VMN(n) do { asm volatile("" ::: "memory"); __builtin_amdgcn_s_waitcnt(0x0F70 | (n)); asm volatile("" ::: "memory"); } while (0)

__device__ __forceinline__ bool give_up(unsigned* tmo, unsigned call, unsigned long long t0) {
    const unsigned long long dt = __builtin_amdgcn_s_memrealtime() - t0;
    if (dt < 2000ull) return false;
    if ((unsigned)__builtin_amdgcn_readfirstlane(ld_agent(tmo)) == call) return true;
    if (dt > DEADLINE) { st_agent(tmo, call); return true; }
    return false;
}

__device__ __forceinline__ int gpan_of(int v, int consumer) { return v < NS ? consumer * NS + v : (NRANK - 1) * NS + v; }
__device__ __forceinline__ int unit_of(int o, int consumer, int& q, int& pos) {
    if (o < 4 * G0) { q = o / G0; const int m = o % G0; pos = q * U0 + gpan_of(m, consumer); return m < GW ? NTM + m : m - GW; }
    pos = o + 4 * (NRANK - 1) * NS;
    o -= 4 * G0;
    const int g = o / (4 * GW), r = o % (4 * GW);
    int i;
    if (GL == 0 || g < NGF) { q = r / GW; i = r % GW; } else { q = r / (GL ? GL : 1); i = r % (GL ? GL : 1); }
    return NTM + GW * (g + 1) + i;
}

__device__ __forceinline__ unsigned cvt_pk_bf16(float lo, float hi) { unsigned r; asm("v_cvt_pk_bf16_f32 %0, %1, %2" : "=v"(r) : "v"(lo), "v"(hi)); return r; }
__device__ __forceinline__ float silu(float y) {
    const float c = __builtin_amdgcn_fmed3f(y, -60.0f, 60.0f);
    const float e = __builtin_amdgcn_exp2f(c * -1.44269504088896f);
    return y * __builtin_amdgcn_rcpf(1.0f + e);
}

typedef float f32x2 __attribute__((ext_vector_type(2)));
__device__ __forceinline__ f32x2 silu2(f32x2 y) {
    const f32x2 t = __builtin_elementwise_fma(y, (f32x2){-1.44269504088896f, -1.44269504088896f}, (f32x2){-86.5617024533378f, -86.5617024533378f});
    const f32x2 e = {__builtin_amdgcn_fmed3f(__builtin_amdgcn_exp2f(t[0]), 0.0f, 1.0f), __builtin_amdgcn_fmed3f(__builtin_amdgcn_exp2f(t[1]), 0.0f, 1.0f)};
    const f32x2 d = __builtin_elementwise_fma(e, (f32x2){1.1420073898156842e26f, 1.1420073898156842e26f}, (f32x2){1.0f, 1.0f});
    return y * (f32x2){__builtin_amdgcn_rcpf(d[0]), __builtin_amdgcn_rcpf(d[1])};
}

#define STAMP1(P, slot) do {} while (0)
#define STAMP(P, slot) do {} while (0)

__device__ __forceinline__ void pull_role(const Params& P, unsigned call, int peer, int widx, int lane) {
    const int rank = P.rank;
    char* wsl = P.ws[rank];
    const rsrc_t rwr = mk_rsrc(P.ws[peer], WS_END), rwl = mk_rsrc(wsl, WS_END);
    const rsrc_t rol = mk_rsrc(P.out[rank], OUTB);
    unsigned* tmo = (unsigned*)(wsl + WS_TMO);
    constexpr unsigned OOB = 0xfffffff0u;
    const unsigned lane0 = lane == 0 ? 0u : OOB;
    const unsigned hbase = (call - 1u) * (unsigned)(NPC + NW);
    auto draw = [&]() { return (unsigned)__builtin_amdgcn_raw_ptr_buffer_atomic_add_i32(1, rwl, lane0, WS_HEAD + (unsigned)peer * 128u, 0); };
    auto arrive = [&](unsigned vo, unsigned cnt) { (void)__builtin_amdgcn_raw_ptr_buffer_atomic_add_i32(PU, rwl, vo, cnt, 0); };
    const bool deep0 = (widx & 7) < CFG_DEEP;
    bool bad = false; int stamped = 0; (void)stamped;
    STAMP(P, 0);
    auto index_of = [&](unsigned v) { return (int)((unsigned)__builtin_amdgcn_readfirstlane(v) - hbase); };
    const unsigned long long tb = __builtin_amdgcn_s_memrealtime();
    int cur = index_of(draw());
    while (CFG_HOLD > 0 && __builtin_amdgcn_s_memrealtime() - tb < (unsigned long long)CFG_HOLD) __builtin_amdgcn_s_sleep(2);
    constexpr int BPB = NS * 128 / PU;
    const unsigned fso = WS_FLAG + (unsigned)peer * FLAGB;
    auto first_of = [&](int i) { return i < NPIN ? (i >> (6 - PSH)) + NPUSH : NIN * 4 + (i - NPIN) / BPB * NS; };
    auto limit_of = [&](const u32x4& f0, const u32x4& f1, int i) {
        const int first = first_of(i);
        unsigned un = 0;
#pragma unroll
        for (int j = 0; j < 8; ++j) un |= ((int)((j < 4 ? f0[j & 3] : f1[j & 3]) - call) < 0 && lane * 8 + j >= first ? 1u : 0u) << j;
        const int l = __builtin_ctzll(__ballot(un != 0u) | (1ull << 63));
        const int u = l * 8 + __builtin_ctz(__builtin_amdgcn_readlane(un, l) | 256u);
        return u <= NIN * 4 ? (u - NPUSH) << (6 - PSH) : u >= NFLAG ? NPC : NPIN + (u - NIN * 4) / NS * BPB;
    };
    auto offs_of = [&](int i, unsigned (&o)[4], unsigned& cnt) -> unsigned {
        if (i < NPIN) {
            int q, pos; const int pan = unit_of((i >> (6 - PSH)) + NPUSH, rank, q, pos);
            cnt = WS_CNT + (unsigned)(pan * NRANK + peer) * 128u + (unsigned)q * 4u;
            const unsigned sub = (unsigned)((i << PSH) & 63) * 1024u;
            const unsigned b = WS_PAN + (unsigned)pan * PANB + (unsigned)peer * CHB + (unsigned)q * UNB + sub + (unsigned)lane * 16u;
#pragma unroll
            for (int k = 0; k < 4; ++k) o[k] = b + k * 1024u;
            return WS_SND + (unsigned)pos * UNB + sub;
        }
        cnt = 0;
#pragma unroll
        for (int k = 0; k < 4; ++k) {
            const unsigned sg = (unsigned)(((i - NPIN) << PSH) + k) * 2u + (unsigned)(lane >> 5);
            o[k] = (unsigned)peer * ROWB + (unsigned)(lane & 31) * 16u + sg / NS * (NN * 2u) + sg % NS * 512u;
        }
        return WS_OSN + (unsigned)((i - NPIN) << PSH) * 1024u;
    };
    int lim = 0;
    while ((unsigned)cur < (unsigned)NPC) {
        if (!bad) {
            const unsigned long long t0 = __builtin_amdgcn_s_memrealtime();
            const unsigned pvo = (unsigned)(lane - (first_of(cur) >> 3)) < 4u ? (unsigned)lane * 32u : OOB - 16u;
            for (;;) {
                const u32x4 f0 = __builtin_amdgcn_raw_buffer_load_b128(rwl, pvo, fso, AUX_SYS), f1 = __builtin_amdgcn_raw_buffer_load_b128(rwl, pvo + 16u, fso, AUX_SYS);
                lim = limit_of(f0, f1, cur);
                if (cur < lim) break;
                if (give_up(tmo, call, t0)) { bad = true; break; }
                __builtin_amdgcn_s_sleep(2);
            }
            asm volatile("" ::: "memory");
        }
        unsigned o[4], p2 = OOB, p3 = OOB, va = OOB, cprev = 0;
        if (bad) {
            (void)offs_of(cur, o, cprev);
            if (cur >= NPIN) {
#pragma unroll
                for (int k = 0; k < PU; ++k) __builtin_amdgcn_raw_buffer_store_b128((u32x4){0x7fc07fc0u, 0x7fc07fc0u, 0x7fc07fc0u, 0x7fc07fc0u}, rol, o[k], 0, AUX_AGENT);
            }
            cur = index_of(draw()); continue;
        }
        STAMP1(P, 1);
        rsrc_t rp = rwl;
        u32x4 v[4] = {};
        const unsigned per = deep0 ? PACE16 / 2 : PACE16;
        unsigned slot = 0;
        if (CFG_RATE) { const unsigned now = (unsigned)__builtin_amdgcn_s_memrealtime() << 4; slot = now - (now & 0xffffffu) % PACE16 + (unsigned)((widx & 7) * PW + (widx >> 3)) * (PACE16 / NW) - PACE16; }
        do {
            unsigned cb; const unsigned src = offs_of(cur, o, cb);
            const bool in = cur < NPIN, deep = deep0 || cur >= NPC - CFG_TAIL;
            const rsrc_t rd = in ? rwl : rol;
            const bool look = lim < NPC && cur + CFG_LOOK >= lim;
            const unsigned fvo = look ? (unsigned)lane * 32u : OOB - 16u;
            unsigned nv = 0; u32x4 f0, f1;
#pragma unroll
            for (int k = 0; k < 4; ++k) {
                if (CFG_RATE && cur < NPC - CFG_TAIL) {
                    slot += per;
                    for (;;) {
                        const int d = (int)(((unsigned)__builtin_amdgcn_s_memrealtime() << 4) - slot);
                        if (d > CFG_CRED * 16) slot += (unsigned)(d - CFG_CRED * 16);
                        if (d >= 0) break;
                        __builtin_amdgcn_s_sleep(1);
                    }
                }
                v[k] = __builtin_amdgcn_raw_buffer_load_b128(rwr, lane * 16 + k * 1024, src, AUX_SYS_NT);
                if (k == 0) {
                    nv = draw();
                    f0 = __builtin_amdgcn_raw_buffer_load_b128(rwl, fvo, fso, AUX_SYS); f1 = __builtin_amdgcn_raw_buffer_load_b128(rwl, fvo + 16u, fso, AUX_SYS);
                }
                __builtin_amdgcn_raw_buffer_store_b128(v[k ^ 2], k < 2 ? rp : rd, k == 0 ? p2 : k == 1 ? p3 : o[k ^ 2], 0, AUX_AGENT);
                if (k == 0) WAIT_VMN(7); else if (k == 1) WAIT_VMN(6); else WAIT_VMN(3);
                if (!deep) { if (k == 0) WAIT_VMN(4); else WAIT_VMN(1); }
            }
            arrive(va, cprev);
            cprev = cb; va = in ? lane0 : OOB; rp = rd; p2 = o[2]; p3 = o[3];
            cur = index_of(nv);
            if (look) { const int l2 = limit_of(f0, f1, cur); lim = l2 > lim ? l2 : lim; }
        } while ((unsigned)cur < (unsigned)lim);
        __builtin_amdgcn_raw_buffer_store_b128(v[2], rp, p2, 0, AUX_AGENT);
        __builtin_amdgcn_raw_buffer_store_b128(v[3], rp, p3, 0, AUX_AGENT);
        if (cprev) { WAIT_VM0(); arrive(va, cprev); }
    }
    STAMP(P, 4);
    asm volatile("" ::: "memory");
    if (lane == 0) st_sys((unsigned*)(P.ws[peer] + WS_DONE + rank * 512) + widx, call);
    if (widx == 0) {
        const unsigned* dn = (const unsigned*)(wsl + WS_DONE + peer * 512);
        const unsigned long long t0 = __builtin_amdgcn_s_memrealtime();
        for (;;) {
            bool ok = true;
            if (lane < NW) ok = (int)(ld_sys(dn + lane) - call) >= 0;
            if (NW > 64 && lane + 64 < NW) ok = ok && (int)(ld_sys(dn + 64 + lane) - call) >= 0;
            if (__all(ok)) break;
            if (give_up(tmo, call, t0)) {
                if (lane == 0) __builtin_amdgcn_raw_buffer_store_b32(0x7fc07fc0u, rol, (unsigned)(rank * NS) * 512u, 0, AUX_SYS);
                break;
            }
            __builtin_amdgcn_s_sleep(2);
        }
        STAMP(P, 5);
    }
}

__device__ __forceinline__ void pack_role(const Params& P, unsigned tk, int g, int wid, int lane) {
    const int rank = P.rank;
    char* wsl = P.ws[rank];
    const rsrc_t rx = mk_rsrc(P.x, (unsigned)MM * KSH * 2), rw = mk_rsrc(P.w, (unsigned)KSH * NN * 2);
    const int fr = lane & 15, fq = lane >> 4;
    const unsigned long long t0 = __builtin_amdgcn_s_memrealtime();
    for (int u = g; u < NUNIT; u += NPK) {
        int o, dest = -1;
        if (u < 4 * U0) {
            const int r = u % U0;
            if (r < NRANK * NS) { o = r >> 3; dest = (rank + 1 + r) & (NRANK - 1); } else o = r - (NRANK - 1) * NS;
            o += u / U0 * G0;
        } else o = u - 4 * (NRANK - 1) * NS;
        int q, pos; const int pan = unit_of(o, dest < 0 ? 0 : dest, q, pos);
        const bool push = o < NPUSH && dest != rank;
        const rsrc_t rwl = mk_rsrc(P.ws[push ? dest & (NRANK - 1) : rank], WS_END);
        while (CFG_PACE > 0 && (unsigned)(__builtin_amdgcn_s_memrealtime() - t0) < (unsigned)((u < CFG_PACE0 ? 0 : u - CFG_PACE0) * CFG_PACE / 10)) __builtin_amdgcn_s_sleep(1);
        const int ktl = q * 2 + (wid >> 2);
        const unsigned dst = push ? WS_PAN + (unsigned)pan * PANB + (unsigned)(rank * KTS + ktl) * KTB : WS_SND + (unsigned)pos * UNB + (unsigned)(wid >> 2) * KTB;
        u32x4 v[8];
        if (pan < NTM) {
            const int rb0 = (wid & 3) * 4;
            const unsigned vo = (unsigned)fr * (KSH * 2) + (unsigned)fq * 16u;
            const unsigned so = (unsigned)(pan * 256 + rb0 * 16) * (KSH * 2) + (unsigned)ktl * 128u;
#pragma unroll
            for (int i = 0; i < 8; ++i) v[i] = __builtin_amdgcn_raw_buffer_load_b128(rx, vo + (i & 1) * 64, so + (unsigned)(i >> 1) * (16 * KSH * 2), 0);
#pragma unroll
            for (int i = 0; i < 8; ++i) STORE16(v[i], rwl, lane * 16 + (i & 1) * 1024, dst + (unsigned)(rb0 + (i >> 1)) * 2048u, AUX_SYS);
        } else {
            const int ks = (wid >> 1) & 1, wr = wid & 1;
            const unsigned vo = (unsigned)(fq * 8) * (NN * 2) + (unsigned)fr * 16u;
            const unsigned so = (unsigned)(ktl * 64 + ks * 32) * (NN * 2) + (unsigned)(gpan_of(pan - NTM, dest < 0 ? 0 : dest) * 256 + wr * 128) * 2u;
#pragma unroll
            for (int j = 0; j < 8; ++j) v[j] = __builtin_amdgcn_raw_buffer_load_b128(rw, vo, so + (unsigned)j * (NN * 2), 2);
            while (CFG_P3 && push && o >= CFG_P3N && (unsigned)(__builtin_amdgcn_s_memrealtime() - t0) < (unsigned)(CFG_P3 + (o - CFG_P3N) * 110)) __builtin_amdgcn_s_sleep(1);
#pragma unroll
            for (int i = 0; i < 8; ++i) {
                u32x4 o;
#pragma unroll
                for (int e = 0; e < 4; ++e) o[e] = __builtin_amdgcn_perm(v[2 * e + 1][i >> 1], v[2 * e][i >> 1], (i & 1) ? 0x07060302u : 0x05040100u);
                STORE16(o, rwl, lane * 16, dst + (unsigned)(i >> 2) * HTB + (unsigned)wr * 8192u + (unsigned)(i & 3) * 2048u + (unsigned)ks * 1024u, AUX_SYS);
            }
        }
        WAIT_VM0();
        if (u == g) STAMP(P, 6);
        __builtin_amdgcn_s_barrier();
        if (wid == 0 && lane < NRANK && (dest < 0 || dest == lane)) {
            unsigned c = tk; asm volatile("" : "+v"(c));
            if (lane == rank || push) st_sys((unsigned*)(P.ws[lane] + WS_CNT) + (pan * NRANK + rank) * 32 + q, (c + 1u) * CNT_FULL);
            else st_sys((unsigned*)(P.ws[lane] + WS_FLAG + rank * FLAGB) + o, c + 1u);
        }
    }
}

struct Tile { int tm, v, half; unsigned sA, stA; };
__device__ __forceinline__ bool tile_of(int first, int slot, int col, int rank, Tile& t) {
    const int nt = 4 * ((NWP + 1 - (col >> 2)) >> 1);
    const bool split = SPLIT && 2 * (nt - first) <= KX;
    const int idx = first + (split ? slot >> 1 : slot);
    t.half = split ? slot & 1 : -1;
    const int v = 2 * (idx >> 2) + (col >> 2);
    if (idx >= nt) return false;
    t.tm = 4 * (col & 3) + (idx & 3); t.v = v;
    const int g = v / GW, i = v % GW;
    t.sA = WS_SND + (unsigned)(g == 0 ? gpan_of(v, rank) : 4 * U0 + (g - 1) * 4 * GW + i) * UNB;
    t.stA = (unsigned)(g == 0 ? U0 : g <= NGF ? GW : GL) * UNB;
    return true;
}

template <int wr>
__device__ __forceinline__ void gemm_role(const Params& P, unsigned call, int col, int slot, LAS unsigned char* lds) {
    const int tid = threadIdx.x, wid = __builtin_amdgcn_readfirstlane(tid >> 6), lane = tid & 63, wc = wid & 3;
    const int rank = P.rank;
    char* wsl = P.ws[rank];
    const rsrc_t rws = mk_rsrc(wsl, WS_END);
    LAS unsigned* ctl = (LAS unsigned*)(lds + STAGE_BYTES);
    const unsigned voff = (unsigned)tid * 16u;
    const unsigned ldsw = (unsigned)wid * 1024u;
    const int aoff = wr * 8192 + lane * 16, boff = wc * 4096 + lane * 16;

    auto ready = [&](const Tile& t, int want) -> int {
        if (wr == 0 && wid == 0) {
            const unsigned* c = (const unsigned*)(wsl + WS_CNT) + ((((lane & 32) ? NTM + t.v : t.tm) * NRANK + ((lane >> 2) & 7)) * 32 + (lane & 3));
            unsigned* tmo = (unsigned*)(wsl + WS_TMO);
            const unsigned need = call * CNT_FULL;
            const unsigned long long t0 = __builtin_amdgcn_s_memrealtime();
            int got;
            for (;;) {
                const unsigned long long miss = __ballot((int)(ld_sys(c) - need) < 0);
                unsigned m = (unsigned)miss | (unsigned)(miss >> 32);
                const bool own = ((m >> (rank * 4)) & 15u) == 0u;
                m |= m >> 16; m |= m >> 8; m |= m >> 4;
                got = own ? 1 + __builtin_ctz(m | 16u) : 0;
                if (got >= want) break;
                if (give_up(tmo, call, t0)) { ctl[0] = 1u; got = 5; break; }
                __builtin_amdgcn_s_sleep(4);
            }
            ctl[2] = (unsigned)got;
            __builtin_amdgcn_fence(__ATOMIC_ACQUIRE, "agent");
            asm volatile("s_waitcnt vmcnt(0) lgkmcnt(0)" ::: "memory");
        }
        asm volatile("" ::: "memory"); __builtin_amdgcn_s_barrier(); asm volatile("" ::: "memory");
        const int r = __builtin_amdgcn_readfirstlane(ctl[2]);
        asm volatile("s_waitcnt lgkmcnt(0)" ::: "memory");
        return r;
    };

#define SA(b, h) (((b) * 2 + (h)) * HTB)
#define SB(b, h) ((4 + (b) * 2 + (h)) * HTB)
#define STAGE(bufoff, soff) do { _Pragma("unroll") for (int _i = 0; _i < 2; ++_i) \
        __builtin_amdgcn_raw_ptr_buffer_load_lds(rws, (LAS unsigned*)(lds + (bufoff) + ldsw + _i * 8192), 16, voff, (soff) + _i * 8192u, 0, 0); } while (0)
#define LDA(dst, b, h) do { _Pragma("unroll") for (int m = 0; m < 4; ++m) _Pragma("unroll") for (int k = 0; k < 2; ++k) dst[m][k] = *(const LAS bf16x8*)(lds + SA(b, h) + aoff + m * 2048 + k * 1024); } while (0)
#define LDB(dst, b, h) do { _Pragma("unroll") for (int n = 0; n < 2; ++n) _Pragma("unroll") for (int k = 0; k < 2; ++k) dst[n][k] = *(const LAS bf16x8*)(lds + SB(b, h) + boff + n * 2048 + k * 1024); } while (0)
#define MMA(ai, first, prio) do { __builtin_amdgcn_s_setprio(prio); \
        _Pragma("unroll") for (int i = 0; i < 32; ++i) { const int bj = i >> 4, m = (i >> 2) & 3, n = (i >> 1) & 1, k = i & 1; \
            if (!SPLIT || cur.half != 1 - bj) \
            acc[ai][bj][m][n] = __builtin_amdgcn_mfma_f32_16x16x32_bf16(bj ? B1[n][k] : B0[n][k], At[m][k], first && k == 0 ? (f32x4){0.f, 0.f, 0.f, 0.f} : acc[ai][bj][m][n], 0, 0, 0); } \
        __builtin_amdgcn_s_setprio(0); } while (0)
#define WAIT_L0() do { asm volatile("" ::: "memory"); __builtin_amdgcn_s_waitcnt(0xC07F); asm volatile("" ::: "memory"); } while (0)
#define WAIT_V(n) asm volatile("s_waitcnt vmcnt(" #n ")" ::: "memory")
#define WAIT_L(n) asm volatile("s_waitcnt lgkmcnt(" #n ")" ::: "memory")
#define BAR __builtin_amdgcn_s_barrier()
#define SCHED __builtin_amdgcn_sched_barrier(0)

    unsigned long long outp = (unsigned long long)P.out[rank];
    asm volatile("" : "+v"(outp));
    Tile cur, nxt; int ui = 0;
    if (!tile_of(ui, slot, col, rank, cur)) return;
    bf16x8 At[4][2], B0[2][2], B1[2][2];
    constexpr unsigned SB0 = WS_SND + ((NRANK - 1) * NS + GW) * UNB, STB = U0 * UNB;
    const unsigned kov = (unsigned)((rank + 1 + (lane + 3) % 7) & (NRANK - 1)) * CHB + (unsigned)((lane - 4) / 7) * UNB;
    int lvl = ready(cur, 1);
    STAMP(P, 2);
    {
        const unsigned ua = cur.sA, ub = SB0 + (unsigned)cur.tm * UNB;
        STAGE(SB(0, 0), ub); STAGE(SA(0, 0), ua); STAGE(SB(0, 1), ub + HTB); STAGE(SA(0, 1), ua + HTB);
        STAGE(SB(1, 0), ub + KTB); STAGE(SA(1, 0), ua + KTB); STAGE(SB(1, 1), ub + HTB + KTB);
    }
    WAIT_V(0); BAR;
#define TICK(i, rnd) do {} while (0)
#pragma unroll 1
    for (;;) {
        f32x4 acc[2][2][4][2];
        const bool has_next = tile_of(ui + KX, slot, col, rank, nxt);
        if (!has_next) nxt = cur;
        unsigned tabA = lane < 4 ? cur.sA + (unsigned)lane * cur.stA : lane < 32 ? WS_PAN + (unsigned)(NTM + cur.v) * PANB + kov : nxt.sA;
        unsigned tabB = lane < 4 ? SB0 + (unsigned)cur.tm * UNB + (unsigned)lane * STB : lane < 32 ? WS_PAN + (unsigned)cur.tm * PANB + kov : SB0 + (unsigned)nxt.tm * UNB;
        int jchk = lvl < 5 ? 7 * lvl - 4 : has_next ? 31 : 32;
        TICK(0, 2); TICK(4, 3);
#define SYNC_MMA(ai, first) do { WAIT_L0(); if (wr == 1) BAR; MMA(ai, first, wr + 1); if (wr == 0) BAR; SCHED; } while (0)
#define PASS(first, check) do { \
            LDB(B0, 0, 0); LDA(At, 0, 0); LDB(B1, 0, 1); STAGE(SA(1, 1), a2 + KTB + HTB); SCHED; \
            check; \
            a2 = __builtin_amdgcn_readlane(tabA, j + 1); b2 = __builtin_amdgcn_readlane(tabB, j + 1); \
            if (!first) WAIT_V(8); \
            SYNC_MMA(0, first); \
            LDA(At, 0, 1); STAGE(SB(0, 0), b2); STAGE(SA(0, 0), a2); STAGE(SB(0, 1), b2 + HTB); \
            if (!first) WAIT_V(8); \
            SYNC_MMA(1, first); \
            LDB(B0, 1, 0); LDA(At, 1, 0); LDB(B1, 1, 1); STAGE(SA(0, 1), a2 + HTB); \
            WAIT_V(8); SYNC_MMA(0, false); \
            LDA(At, 1, 1); STAGE(SB(1, 0), b2 + KTB); STAGE(SA(1, 0), a2 + KTB); STAGE(SB(1, 1), b2 + KTB + HTB); \
            WAIT_V(8); SYNC_MMA(1, false); } while (0)
        unsigned a2 = cur.sA, b2;
        int j = 0;
        PASS(true, );
#pragma unroll 1
        for (j = 1; j < KT / 2; ++j)
            PASS(false, if (__builtin_expect(j == jchk, 0)) {
                    const bool last = j == KT / 2 - 1;
                    lvl = ready(last ? nxt : cur, last ? 1 : lvl + 1);
                    jchk = lvl < 5 ? 7 * lvl - 4 : has_next ? 31 : 32;
                });
        TICK(1, 2);
        TICK(2, 2);
        {
            int tz = threadIdx.x; asm volatile("" : "+v"(tz));
            const int ln = tz & 63, fr = ln & 15, fq = ln >> 4;
            const rsrc_t rol = mk_rsrc((const void*)outp, OUTB);
            const bool shared = cur.v < NS;
            if (__builtin_amdgcn_readfirstlane(ctl[0]) != 0u) {
#pragma unroll
                for (int i = 0; i < 32; ++i) acc[i >> 4][(i >> 3) & 1][(i >> 1) & 3][i & 1] = __builtin_bit_cast(f32x4, (u32x4){0x7fc00000u, 0x7fc00000u, 0x7fc00000u, 0x7fc00000u});
            }
            if (CFG_STAG && ui < KX && cur.v < GW) {
                const unsigned long long t0 = __builtin_amdgcn_s_memrealtime();
                while (__builtin_amdgcn_s_memrealtime() - t0 < (unsigned long long)(((col & 3) * 2 + (col >> 2)) * CFG_STAG + slot % CFG_STAGN * CFG_STAG2)) __builtin_amdgcn_s_sleep(8);
            }
            const bool thru = !has_next;
            const unsigned vo = (unsigned)(wc * 32 + fq * 4) * (NN * 2) + (unsigned)(wr * 256 + fr * 16);
            const unsigned so = (unsigned)(cur.tm * 256) * (NN * 2) + (unsigned)gpan_of(cur.v, rank) * 512u;
            const unsigned vs = (unsigned)(wc * 32 + fq * 4) * ROWB + (unsigned)(wr * 256 + fr * 16);
            const unsigned ss = WS_OSN + (unsigned)(cur.tm * 256) * ROWB + (unsigned)cur.v * 512u;
            auto finish = [&](auto mode) {
#pragma unroll
                for (int bj = 0; bj < 2; ++bj) if (!SPLIT || mode != 2 || cur.half != 1 - bj)
#pragma unroll
                    for (int n = 0; n < 2; ++n)
#pragma unroll
                        for (int r = 0; r < 4; ++r) {
                            u32x4 o;
                            if ((r & 1) == 0) {
#pragma unroll
                                for (int i = 0; i < 8; ++i) { f32x4& a = acc[i >> 2][bj][i & 3][n]; const f32x2 z = silu2((f32x2){a[r], a[r + 1]}); a[r] = z[0]; a[r + 1] = z[1]; }
                            }
#pragma unroll
                            for (int i = 0; i < 4; ++i) o[i] = cvt_pk_bf16(acc[i >> 1][bj][(i & 1) * 2][n][r], acc[i >> 1][bj][(i & 1) * 2 + 1][n][r]);
                            if (mode == 1) STORE16(o, rws, vs, ss + (unsigned)(bj * 128 + n * 16 + r) * ROWB, AUX_SYS);
                            STORE16(o, rol, vo, so + (unsigned)(bj * 128 + n * 16 + r) * (NN * 2), mode == 2 ? AUX_SYS : CFG_RST);
                            if (mode == 2 && CFG_LAST > 1) { WAIT_VM0(); __builtin_amdgcn_s_sleep(CFG_LAST > 1 ? CFG_LAST - 1 : 1); }
                        }
            };
            if (shared) finish(std::integral_constant<int, 1>());
            else if (thru) finish(std::integral_constant<int, 2>());
            else finish(std::integral_constant<int, 0>());
            WAIT_V(16);
            if (shared) {
                WAIT_VM0();
                unsigned old = 0;
                if (ln == 0) old = __hip_atomic_fetch_add(ctl + 1, 1u, __ATOMIC_RELAXED, __HIP_MEMORY_SCOPE_WORKGROUP);
                old = __builtin_amdgcn_readfirstlane(old);
                if ((old & 7u) == 7u && ln < NRANK && ln != rank)
                    st_sys((unsigned*)(P.ws[ln] + WS_FLAG + rank * FLAGB) + NIN * 4 + cur.tm * NS + cur.v, call);
            }
        }
        TICK(3, 2);
        STAMP(P, 3 + (ui >= KX ? 1 : 0));
        if (!has_next) break;
        cur = nxt; ui += KX;
    }
    WAIT_V(0);
}

__global__ __launch_bounds__(512, 2) void fused_gemm_ar(Params P) {
    extern __shared__ __attribute__((aligned(16))) unsigned char shm[];
    LAS unsigned char* lds = (LAS unsigned char*)shm;
    const int wid = __builtin_amdgcn_readfirstlane(threadIdx.x >> 6), lane = threadIdx.x & 63;
    const int b = __builtin_amdgcn_readfirstlane(blockIdx.x);
    unsigned* tkp = (unsigned*)(P.ws[P.rank] + WS_TICKET + b * 128 + wid * 16);
    const unsigned tk = ld_agent(tkp);
    if (b < NPULL) {
        const unsigned call = __builtin_amdgcn_readfirstlane(tk) + 1u;
        st_agent(tkp, call);
        const int pi = b / PW, slot = b % PW;
        const int peer = (P.rank + 1 + pi) & (NRANK - 1);
        pull_role(P, call, peer, slot * 8 + wid, lane);
    } else {
        const int g = b - NPULL;
        ((LAS unsigned*)(lds + STAGE_BYTES))[lane & 3] = 0u;
        STAMP(P, 0);
        pack_role(P, tk, g, wid, lane);
        const unsigned call = __builtin_amdgcn_readfirstlane(tk) + 1u;
        st_agent(tkp, call);
        STAMP(P, 1);
        asm volatile("s_waitcnt lgkmcnt(0)" ::: "memory");
        __builtin_amdgcn_s_barrier();
        if (b >= NPS) {
            if (wid < 4) gemm_role<0>(P, call, b & 7, (b - NPS) >> 3, lds);
            else gemm_role<1>(P, call, b & 7, (b - NPS) >> 3, lds);
        }
    }
}
}

extern "C" void kernel_launch(int rank, int nranks, ncclComm_t comm,
                              hipStream_t stream,
                              void* const* d_in, const int* in_sizes, int n_in,
                              void* d_out, int out_size,
                              const int* in_dtypes, int out_dtype,
                              void* const* d_out_all,
                              void* const* d_ws_all, size_t ws_bytes) {
    if (nranks != NRANK || (unsigned)rank >= (unsigned)NRANK || ws_bytes < (size_t)WS_END || n_in < 2 ||
        in_sizes[0] != MM * KSH || in_sizes[1] != KSH * NN || out_size != MM * NN) {
        fprintf(stderr, "unexpected problem geometry\n");
        return;
    }
    static bool attr_done[NRANK] = {};
    if (!attr_done[rank]) {
        HIP_CHECK(hipFuncSetAttribute((const void*)fused_gemm_ar, hipFuncAttributeMaxDynamicSharedMemorySize, LDS_BYTES));
        attr_done[rank] = true;
    }
    Params P;
    P.x = d_in[0]; P.w = d_in[1]; P.rank = rank;
    for (int p = 0; p < NRANK; ++p) { P.out[p] = (char*)d_out_all[p]; P.ws[p] = (char*)d_ws_all[p]; }
    hipLaunchKernelGGL(fused_gemm_ar, dim3(GRID), dim3(512), LDS_BYTES, stream, P);
    HIP_CHECK(hipGetLastError());
}
```

```cpp
#include <hip/hip_runtime.h>
#include <rccl/rccl.h>
#include <cstdio>
#include <type_traits>

#define HIP_CHECK(expr) do { hipError_t _e = (expr); if (_e != hipSuccess) { \
    fprintf(stderr, "HIP error %s at %s:%d\n", hipGetErrorString(_e), __FILE__, __LINE__); \
    return; } } while (0)

#define LAS __attribute__((address_space(3)))
typedef short bf16x8 __attribute__((ext_vector_type(8)));
typedef float f32x4 __attribute__((ext_vector_type(4)));
typedef unsigned u32x4 __attribute__((ext_vector_type(4)));
typedef __amdgpu_buffer_rsrc_t rsrc_t;

namespace {
constexpr int NRANK = 8;
constexpr int MM = 4096, KK = 4096, NN = 32768, KSH = KK / NRANK;
constexpr int KT = KK / 64, KTS = KSH / 64, NTM = MM / 256, NPAN = NN / 256;
#define CFG_NS 6
#define CFG_PW 8
constexpr int NS = CFG_NS;
constexpr int NRP = NPAN - NRANK * NS;
constexpr int NWP = NS + NRP;
constexpr int NIN = NTM + NWP;
constexpr int NOUT = NS * NTM;
constexpr int NFLAG = NIN * 4 + NOUT;
constexpr int NUNIT = NTM * 4 + NPAN * 4;
constexpr int GRID = 256;
constexpr int PW = CFG_PW;
constexpr int NPULL = PW * (NRANK - 1);
constexpr int NW = PW * 8;
constexpr int NPK = GRID - NPULL;
constexpr int NPS = (NPULL + 7) / 8 * 8;
constexpr int KX = (GRID - NPS) / 8;
#define CFG_GW (2 * (KX / 4))
constexpr int GW = CFG_GW;
constexpr int G0 = GW + NTM;
#define CFG_GM 2
constexpr int GM = CFG_GM;
constexpr int NGF = (NWP - GW - GM) / GW, GL = (NWP - GW - GM) % GW;
static_assert(NS <= GW && NWP > GW, "shared panels sit in the first group");
#define CFG_PU 4
constexpr int PU = CFG_PU, PSH = PU == 1 ? 0 : PU == 2 ? 1 : PU == 4 ? 2 : 3;
#define CFG_PUSH 3
constexpr int NPUSH = CFG_PUSH;
constexpr int NPIN = (NIN * 4 - NPUSH) * 64 / PU;
static_assert(NPUSH <= NS, "pushed units have a single consumer");
constexpr int NPC = NPIN + NOUT * 128 / PU;
constexpr unsigned CNT_FULL = 64;

constexpr int NT0 = 4 * ((NWP + 1) / 2), NT1 = 4 * (NWP / 2);
constexpr int RS = (NT1 - 1) / KX;
constexpr int PL0 = NT0 - RS * KX, PL1 = NT1 - RS * KX, PN = 4 * (PL0 + PL1);
#define CFG_MO 7
#define CFG_SWAP 1
#define CFG_LW 0
#define CFG_FD (-8)
constexpr int PFULL = 2 * PN - 8 * KX - CFG_FD < 0 ? 0 : 2 * PN - 8 * KX - CFG_FD > PN ? PN : 2 * PN - 8 * KX - CFG_FD;
constexpr int NITEM = PFULL + 2 * (PN - PFULL);
constexpr unsigned KTB = 32768, HTB = 16384;
constexpr unsigned PANB = KT * KTB;
constexpr unsigned CHB = KTS * KTB;
constexpr unsigned UNB = 2 * KTB;
constexpr unsigned WS_TICKET = 0;
constexpr unsigned WS_TMO = WS_TICKET + GRID * 128;
constexpr unsigned WS_FLAG = WS_TMO + 128;
constexpr unsigned FLAGB = 4096;
constexpr unsigned WS_DONE = WS_FLAG + NRANK * FLAGB;
constexpr unsigned WS_HEAD = WS_DONE + NRANK * 512;
constexpr unsigned WS_CNT = WS_HEAD + NRANK * 128;
constexpr unsigned WS_CTRL_END = WS_CNT + NIN * NRANK * 128;
constexpr unsigned WS_DBG = 512u << 10;
constexpr unsigned WS_SND = 1u << 20;
constexpr unsigned ROWB = NS * 512;
constexpr unsigned WS_OSN = WS_SND + NUNIT * UNB;
constexpr unsigned WS_PAN = (WS_OSN + MM * ROWB + PANB - 1) / PANB * PANB;
constexpr unsigned WS_END = WS_PAN + NIN * PANB;
constexpr int U0 = G0 + (NRANK - 1) * NS;
constexpr unsigned OUTB = (unsigned)MM * NN * 2;
static_assert(WS_CTRL_END <= WS_DBG, "ctrl region");
static_assert(NFLAG <= 512 && 2048 <= (int)FLAGB, "flag region");
static_assert(NW <= 128, "done words");
static_assert(PU == 4, "block size");
#define CFG_DEEP 3
constexpr unsigned STAGE_BYTES = 9 * HTB;
constexpr unsigned LDS_BYTES = STAGE_BYTES + 512;
constexpr unsigned long long DEADLINE = 50000000ull;
constexpr int AUX_SYS = 17, AUX_SYS_NT = 19, AUX_AGENT = 16;
#define CFG_PACE 40
#define CFG_PACE0 40
#define CFG_HOLD 0
#define CFG_STAG 600
#define CFG_P3 320
#define CFG_P3N 2
#define CFG_STAGB 300
#define CFG_LOOK 256
#define CFG_TAIL NW
#define CFG_LAST 1
#define CFG_RST 0
#define CFG_RATE 1720
#define CFG_CRED 40
constexpr unsigned PACE16 = CFG_RATE * PW * (8 + CFG_DEEP) / 64;

struct Params {
    const void* x; const void* w;
    char* out[NRANK]; char* ws[NRANK];
    int rank;
};

__device__ __forceinline__ rsrc_t mk_rsrc(const void* p, unsigned bytes) {
    const unsigned long long v = (unsigned long long)p;
    const unsigned lo = __builtin_amdgcn_readfirstlane((unsigned)v), hi = __builtin_amdgcn_readfirstlane((unsigned)(v >> 32));
    return __builtin_amdgcn_make_buffer_rsrc((void*)(((unsigned long long)hi << 32) | lo), 0, bytes, 0x00020000);
}
__device__ __forceinline__ unsigned ld_sys(const unsigned* p) { return __hip_atomic_load(p, __ATOMIC_RELAXED, __HIP_MEMORY_SCOPE_SYSTEM); }
__device__ __forceinline__ unsigned ld_agent(const unsigned* p) { return __hip_atomic_load(p, __ATOMIC_RELAXED, __HIP_MEMORY_SCOPE_AGENT); }
__device__ __forceinline__ void st_sys(unsigned* p, unsigned v) { __hip_atomic_store(p, v, __ATOMIC_RELAXED, __HIP_MEMORY_SCOPE_SYSTEM); }
__device__ __forceinline__ void st_agent(unsigned* p, unsigned v) { __hip_atomic_store(p, v, __ATOMIC_RELAXED, __HIP_MEMORY_SCOPE_AGENT); }
__device__ __forceinline__ void add_agent(unsigned* p, unsigned v) { (void)__hip_atomic_fetch_add(p, v, __ATOMIC_RELAXED, __HIP_MEMORY_SCOPE_AGENT); }
#define STORE16(v, rs, vo, so, aux) do { const u32x4 _d = (v); __builtin_amdgcn_raw_buffer_store_b128(_d, rs, vo, so, aux); asm volatile("s_nop 1" :: "v"(_d) : "memory"); } while (0)
#define WAIT_VM0() asm volatile("s_waitcnt vmcnt(0)" ::: "memory")
#define WAIT_VMN(n) do { asm volatile("" ::: "memory"); __builtin_amdgcn_s_waitcnt(0x0F70 | (n)); asm volatile("" ::: "memory"); } while (0)

__device__ __forceinline__ bool give_up(unsigned* tmo, unsigned call, unsigned long long t0) {
    const unsigned long long dt = __builtin_amdgcn_s_memrealtime() - t0;
    if (dt < 2000ull) return false;
    if ((unsigned)__builtin_amdgcn_readfirstlane(ld_agent(tmo)) == call) return true;
    if (dt > DEADLINE) { st_agent(tmo, call); return true; }
    return false;
}

__device__ __forceinline__ int gpan_of(int v, int consumer) { return v < NS ? consumer * NS + v : (NRANK - 1) * NS + v; }
__device__ __forceinline__ int unit_of(int o, int consumer, int& q, int& pos) {
    if (o < 4 * G0) { q = o / G0; const int m = o % G0; pos = q * U0 + gpan_of(m, consumer); return m < GW ? NTM + m : m - GW; }
    pos = o + 4 * (NRANK - 1) * NS;
    o -= 4 * G0;
    if (o < 4 * GM) { q = o / (GM ? GM : 1); return NTM + GW + o % (GM ? GM : 1); }
    o -= 4 * GM;
    const int g = o / (4 * GW), r = o % (4 * GW);
    int i;
    if (GL == 0 || g < NGF) { q = r / GW; i = r % GW; } else { q = r / (GL ? GL : 1); i = r % (GL ? GL : 1); }
    return NTM + GM + GW * (g + 1) + i;
}

__device__ __forceinline__ unsigned cvt_pk_bf16(float lo, float hi) { unsigned r; asm("v_cvt_pk_bf16_f32 %0, %1, %2" : "=v"(r) : "v"(lo), "v"(hi)); return r; }
__device__ __forceinline__ float silu(float y) {
    const float c = __builtin_amdgcn_fmed3f(y, -60.0f, 60.0f);
    const float e = __builtin_amdgcn_exp2f(c * -1.44269504088896f);
    return y * __builtin_amdgcn_rcpf(1.0f + e);
}

typedef float f32x2 __attribute__((ext_vector_type(2)));
__device__ __forceinline__ f32x2 silu2(f32x2 y) {
    const f32x2 t = __builtin_elementwise_fma(y, (f32x2){-1.44269504088896f, -1.44269504088896f}, (f32x2){-86.5617024533378f, -86.5617024533378f});
    const f32x2 e = {__builtin_amdgcn_fmed3f(__builtin_amdgcn_exp2f(t[0]), 0.0f, 1.0f), __builtin_amdgcn_fmed3f(__builtin_amdgcn_exp2f(t[1]), 0.0f, 1.0f)};
    const f32x2 d = __builtin_elementwise_fma(e, (f32x2){1.1420073898156842e26f, 1.1420073898156842e26f}, (f32x2){1.0f, 1.0f});
    return y * (f32x2){__builtin_amdgcn_rcpf(d[0]), __builtin_amdgcn_rcpf(d[1])};
}

#define STAMP1(P, slot) do {} while (0)
#define STAMP(P, slot) do {} while (0)

__device__ __forceinline__ void pull_role(const Params& P, unsigned call, int peer, int widx, int lane) {
    const int rank = P.rank;
    char* wsl = P.ws[rank];
    const rsrc_t rwr = mk_rsrc(P.ws[peer], WS_END), rwl = mk_rsrc(wsl, WS_END);
    const rsrc_t rol = mk_rsrc(P.out[rank], OUTB);
    unsigned* tmo = (unsigned*)(wsl + WS_TMO);
    constexpr unsigned OOB = 0xfffffff0u;
    const unsigned lane0 = lane == 0 ? 0u : OOB;
    const unsigned hbase = (call - 1u) * (unsigned)(NPC + NW);
    auto draw = [&]() { return (unsigned)__builtin_amdgcn_raw_ptr_buffer_atomic_add_i32(1, rwl, lane0, WS_HEAD + (unsigned)peer * 128u, 0); };
    auto arrive = [&](unsigned vo, unsigned cnt) { (void)__builtin_amdgcn_raw_ptr_buffer_atomic_add_i32(PU, rwl, vo, cnt, 0); };
    const bool deep0 = (widx & 7) < CFG_DEEP;
    bool bad = false; int stamped = 0; (void)stamped;
    STAMP(P, 0);
    auto index_of = [&](unsigned v) { return (int)((unsigned)__builtin_amdgcn_readfirstlane(v) - hbase); };
    const unsigned long long tb = __builtin_amdgcn_s_memrealtime();
    int cur = index_of(draw());
    while (CFG_HOLD > 0 && __builtin_amdgcn_s_memrealtime() - tb < (unsigned long long)CFG_HOLD) __builtin_amdgcn_s_sleep(2);
    constexpr int BPB = NS * 128 / PU;
    const unsigned fso = WS_FLAG + (unsigned)peer * FLAGB;
    auto first_of = [&](int i) { return i < NPIN ? (i >> (6 - PSH)) + NPUSH : NIN * 4 + (i - NPIN) / BPB * NS; };
    auto limit_of = [&](const u32x4& f0, const u32x4& f1, int i) {
        const int first = first_of(i);
        unsigned un = 0;
#pragma unroll
        for (int j = 0; j < 8; ++j) un |= ((int)((j < 4 ? f0[j & 3] : f1[j & 3]) - call) < 0 && lane * 8 + j >= first ? 1u : 0u) << j;
        const int l = __builtin_ctzll(__ballot(un != 0u) | (1ull << 63));
        const int u = l * 8 + __builtin_ctz(__builtin_amdgcn_readlane(un, l) | 256u);
        return u <= NIN * 4 ? (u - NPUSH) << (6 - PSH) : u >= NFLAG ? NPC : NPIN + (u - NIN * 4) / NS * BPB;
    };
    auto offs_of = [&](int i, unsigned (&o)[4], unsigned& cnt) -> unsigned {
        if (i < NPIN) {
            int q, pos; const int pan = unit_of((i >> (6 - PSH)) + NPUSH, rank, q, pos);
            cnt = WS_CNT + (unsigned)(pan * NRANK + peer) * 128u + (unsigned)q * 4u;
            const unsigned sub = (unsigned)((i << PSH) & 63) * 1024u;
            const unsigned b = WS_PAN + (unsigned)pan * PANB + (unsigned)peer * CHB + (unsigned)q * UNB + sub + (unsigned)lane * 16u;
#pragma unroll
            for (int k = 0; k < 4; ++k) o[k] = b + k * 1024u;
            return WS_SND + (unsigned)pos * UNB + sub;
        }
        cnt = 0;
#pragma unroll
        for (int k = 0; k < 4; ++k) {
            const unsigned sg = (unsigned)(((i - NPIN) << PSH) + k) * 2u + (unsigned)(lane >> 5);
            o[k] = (unsigned)peer * ROWB + (unsigned)(lane & 31) * 16u + sg / NS * (NN * 2u) + sg % NS * 512u;
        }
        return WS_OSN + (unsigned)((i - NPIN) << PSH) * 1024u;
    };
    int lim = 0;
    while ((unsigned)cur < (unsigned)NPC) {
        if (!bad) {
            const unsigned long long t0 = __builtin_amdgcn_s_memrealtime();
            const unsigned pvo = (unsigned)(lane - (first_of(cur) >> 3)) < 4u ? (unsigned)lane * 32u : OOB - 16u;
            for (;;) {
                const u32x4 f0 = __builtin_amdgcn_raw_buffer_load_b128(rwl, pvo, fso, AUX_SYS), f1 = __builtin_amdgcn_raw_buffer_load_b128(rwl, pvo + 16u, fso, AUX_SYS);
                lim = limit_of(f0, f1, cur);
                if (cur < lim) break;
                if (give_up(tmo, call, t0)) { bad = true; break; }
                __builtin_amdgcn_s_sleep(2);
            }
            asm volatile("" ::: "memory");
        }
        unsigned o[4], p2 = OOB, p3 = OOB, va = OOB, cprev = 0;
        if (bad) {
            (void)offs_of(cur, o, cprev);
            if (cur >= NPIN) {
#pragma unroll
                for (int k = 0; k < PU; ++k) __builtin_amdgcn_raw_buffer_store_b128((u32x4){0x7fc07fc0u, 0x7fc07fc0u, 0x7fc07fc0u, 0x7fc07fc0u}, rol, o[k], 0, AUX_AGENT);
            }
            cur = index_of(draw()); continue;
        }
        STAMP1(P, 1);
        rsrc_t rp = rwl;
        u32x4 v[4] = {};
        const unsigned per = deep0 ? PACE16 / 2 : PACE16;
        unsigned slot = 0;
        if (CFG_RATE) { const unsigned now = (unsigned)__builtin_amdgcn_s_memrealtime() << 4; slot = now - (now & 0xffffffu) % PACE16 + (unsigned)((widx & 7) * PW + (widx >> 3)) * (PACE16 / NW) - PACE16; }
        do {
            unsigned cb; const unsigned src = offs_of(cur, o, cb);
            const bool in = cur < NPIN, deep = deep0 || cur >= NPC - CFG_TAIL;
            const rsrc_t rd = in ? rwl : rol;
            const bool look = lim < NPC && cur + CFG_LOOK >= lim;
            const unsigned fvo = look ? (unsigned)lane * 32u : OOB - 16u;
            unsigned nv = 0; u32x4 f0, f1;
#pragma unroll
            for (int k = 0; k < 4; ++k) {
                if (CFG_RATE && cur < NPC - CFG_TAIL) {
                    slot += per;
                    for (;;) {
                        const int d = (int)(((unsigned)__builtin_amdgcn_s_memrealtime() << 4) - slot);
                        if (d > CFG_CRED * 16) slot += (unsigned)(d - CFG_CRED * 16);
                        if (d >= 0) break;
                        __builtin_amdgcn_s_sleep(1);
                    }
                }
                v[k] = __builtin_amdgcn_raw_buffer_load_b128(rwr, lane * 16 + k * 1024, src, AUX_SYS_NT);
                if (k == 0) {
                    nv = draw();
                    f0 = __builtin_amdgcn_raw_buffer_load_b128(rwl, fvo, fso, AUX_SYS); f1 = __builtin_amdgcn_raw_buffer_load_b128(rwl, fvo + 16u, fso, AUX_SYS);
                }
                __builtin_amdgcn_raw_buffer_store_b128(v[k ^ 2], k < 2 ? rp : rd, k == 0 ? p2 : k == 1 ? p3 : o[k ^ 2], 0, AUX_AGENT);
                if (k == 0) WAIT_VMN(7); else if (k == 1) WAIT_VMN(6); else WAIT_VMN(3);
                if (!deep) { if (k == 0) WAIT_VMN(4); else WAIT_VMN(1); }
            }
            arrive(va, cprev);
            cprev = cb; va = in ? lane0 : OOB; rp = rd; p2 = o[2]; p3 = o[3];
            cur = index_of(nv);
            if (look) { const int l2 = limit_of(f0, f1, cur); lim = l2 > lim ? l2 : lim; }
        } while ((unsigned)cur < (unsigned)lim);
        __builtin_amdgcn_raw_buffer_store_b128(v[2], rp, p2, 0, AUX_AGENT);
        __builtin_amdgcn_raw_buffer_store_b128(v[3], rp, p3, 0, AUX_AGENT);
        if (cprev) { WAIT_VM0(); arrive(va, cprev); }
    }
    STAMP(P, 4);
    asm volatile("" ::: "memory");
    if (lane == 0) st_sys((unsigned*)(P.ws[peer] + WS_DONE + rank * 512) + widx, call);
    if (widx == 0) {
        const unsigned* dn = (const unsigned*)(wsl + WS_DONE + peer * 512);
        const unsigned long long t0 = __builtin_amdgcn_s_memrealtime();
        for (;;) {
            bool ok = true;
            if (lane < NW) ok = (int)(ld_sys(dn + lane) - call) >= 0;
            if (NW > 64 && lane + 64 < NW) ok = ok && (int)(ld_sys(dn + 64 + lane) - call) >= 0;
            if (__all(ok)) break;
            if (give_up(tmo, call, t0)) {
                if (lane == 0) __builtin_amdgcn_raw_buffer_store_b32(0x7fc07fc0u, rol, (unsigned)(rank * NS) * 512u, 0, AUX_SYS);
                break;
            }
            __builtin_amdgcn_s_sleep(2);
        }
        STAMP(P, 5);
    }
}

__device__ __forceinline__ void pack_role(const Params& P, unsigned tk, int g, int wid, int lane) {
    const int rank = P.rank;
    char* wsl = P.ws[rank];
    const rsrc_t rx = mk_rsrc(P.x, (unsigned)MM * KSH * 2), rw = mk_rsrc(P.w, (unsigned)KSH * NN * 2);
    const int fr = lane & 15, fq = lane >> 4;
    const unsigned long long t0 = __builtin_amdgcn_s_memrealtime();
    for (int u = g; u < NUNIT; u += NPK) {
        int o, dest = -1;
        if (u < 4 * U0) {
            const int r = u % U0;
            if (r < NRANK * NS) { o = r >> 3; dest = (rank + 1 + r) & (NRANK - 1); } else o = r - (NRANK - 1) * NS;
            o += u / U0 * G0;
        } else o = u - 4 * (NRANK - 1) * NS;
        int q, pos; const int pan = unit_of(o, dest < 0 ? 0 : dest, q, pos);
        const bool push = o < NPUSH && dest != rank;
        const rsrc_t rwl = mk_rsrc(P.ws[push ? dest & (NRANK - 1) : rank], WS_END);
        while (CFG_PACE > 0 && (unsigned)(__builtin_amdgcn_s_memrealtime() - t0) < (unsigned)((u < CFG_PACE0 ? 0 : u - CFG_PACE0) * CFG_PACE / 10)) __builtin_amdgcn_s_sleep(1);
        const int ktl = q * 2 + (wid >> 2);
        const unsigned dst = push ? WS_PAN + (unsigned)pan * PANB + (unsigned)(rank * KTS + ktl) * KTB : WS_SND + (unsigned)pos * UNB + (unsigned)(wid >> 2) * KTB;
        u32x4 v[8];
        if (pan < NTM) {
            const int rb0 = (wid & 3) * 4;
            const unsigned vo = (unsigned)fr * (KSH * 2) + (unsigned)fq * 16u;
            const unsigned so = (unsigned)(pan * 256 + rb0 * 16) * (KSH * 2) + (unsigned)ktl * 128u;
#pragma unroll
            for (int i = 0; i < 8; ++i) v[i] = __builtin_amdgcn_raw_buffer_load_b128(rx, vo + (i & 1) * 64, so + (unsigned)(i >> 1) * (16 * KSH * 2), 0);
#pragma unroll
            for (int i = 0; i < 8; ++i) STORE16(v[i], rwl, lane * 16 + (i & 1) * 1024, dst + (unsigned)(rb0 + (i >> 1)) * 2048u, AUX_SYS);
        } else {
            const int ks = (wid >> 1) & 1, wr = wid & 1;
            const unsigned vo = (unsigned)(fq * 8) * (NN * 2) + (unsigned)fr * 16u;
            const unsigned so = (unsigned)(ktl * 64 + ks * 32) * (NN * 2) + (unsigned)(gpan_of(pan - NTM, dest < 0 ? 0 : dest) * 256 + wr * 128) * 2u;
#pragma unroll
            for (int j = 0; j < 8; ++j) v[j] = __builtin_amdgcn_raw_buffer_load_b128(rw, vo, so + (unsigned)j * (NN * 2), 2);
            while (CFG_P3 && push && o >= CFG_P3N && (unsigned)(__builtin_amdgcn_s_memrealtime() - t0) < (unsigned)(CFG_P3 + (o - CFG_P3N) * 110)) __builtin_amdgcn_s_sleep(1);
#pragma unroll
            for (int i = 0; i < 8; ++i) {
                u32x4 o;
#pragma unroll
                for (int e = 0; e < 4; ++e) o[e] = __builtin_amdgcn_perm(v[2 * e + 1][i >> 1], v[2 * e][i >> 1], (i & 1) ? 0x07060302u : 0x05040100u);
                STORE16(o, rwl, lane * 16, dst + (unsigned)(i >> 2) * HTB + (unsigned)wr * 8192u + (unsigned)(i & 3) * 2048u + (unsigned)ks * 1024u, AUX_SYS);
            }
        }
        WAIT_VM0();
        if (u == g) STAMP(P, 6);
        __builtin_amdgcn_s_barrier();
        if (wid == 0 && lane < NRANK && (dest < 0 || dest == lane)) {
            unsigned c = tk; asm volatile("" : "+v"(c));
            if (lane == rank || push) st_sys((unsigned*)(P.ws[lane] + WS_CNT) + (pan * NRANK + rank) * 32 + q, (c + 1u) * CNT_FULL);
            else st_sys((unsigned*)(P.ws[lane] + WS_FLAG + rank * FLAGB) + o, c + 1u);
        }
    }
}

struct Tile { int tm, v, half; unsigned sA, stA; };
__device__ __forceinline__ void tile_of(int idx, int col, int rank, Tile& t) {
    const int v = 2 * (idx >> 2) + (col >> 2);
    t.tm = 4 * (col & 3) + (idx & 3); t.v = v; t.half = -1;
    const int g = (v - GM) / GW, i = (v - GM) % GW;
    t.sA = WS_SND + (unsigned)(v < GW ? gpan_of(v, rank) : v < GW + GM ? 4 * U0 + v - GW : 4 * U0 + 4 * GM + (g - 1) * 4 * GW + i) * UNB;
    t.stA = (unsigned)(v < GW ? U0 : v < GW + GM ? GM : g <= NGF ? GW : GL) * UNB;
}
__device__ __forceinline__ void item_of(int q, int rank, Tile& t) {
    const int n = q < PFULL ? q : PFULL + ((q - PFULL) >> 1);
    if (n < 4 * PL0) tile_of(RS * KX + n % PL0, n / PL0, rank, t);
    else tile_of(RS * KX + (n - 4 * PL0) % PL1, 4 + (n - 4 * PL0) / PL1, rank, t);
    if (q >= PFULL) t.half = (q - PFULL) & 1;
}

template <int wr>
__device__ __forceinline__ void gemm_role(const Params& P, unsigned call, int col, int slot, LAS unsigned char* lds) {
    const int tid = threadIdx.x, wid = __builtin_amdgcn_readfirstlane(tid >> 6), lane = tid & 63, wc = wid & 3;
    const int rank = P.rank;
    char* wsl = P.ws[rank];
    const rsrc_t rws = mk_rsrc(wsl, WS_END);
    LAS unsigned* ctl = (LAS unsigned*)(lds + STAGE_BYTES);
    const unsigned voff = (unsigned)tid * 16u;
    const unsigned ldsw = (unsigned)wid * 1024u;
    const int aoff = wr * 8192 + lane * 16, boff = wc * 4096 + lane * 16;

    auto lane_now = [&]() { int tz = threadIdx.x; asm volatile("" : "+v"(tz)); return tz & 63; };
    auto cnt_of = [&](const Tile& t) { const int ln = lane_now(); return WS_CNT + (unsigned)((((ln & 32) ? NTM + t.v : t.tm) * NRANK + ((ln >> 2) & 7)) * 32 + (ln & 3)) * 4u; };
    auto level_of = [&](unsigned c) -> int {
        const unsigned long long miss = __ballot((int)(c - call * CNT_FULL) < 0);
        unsigned m = (unsigned)miss | (unsigned)(miss >> 32);
        const bool own = ((m >> (rank * 4)) & 15u) == 0u;
        m |= m >> 16; m |= m >> 8; m |= m >> 4;
        return own ? 1 + __builtin_ctz(m | 16u) : 0;
    };
    auto ready = [&](const Tile& t, int want) -> int {
        if (wr == 0 && wid == 0) {
            const unsigned* c = (const unsigned*)(wsl + cnt_of(t));
            unsigned* tmo = (unsigned*)(wsl + WS_TMO);
            const unsigned long long t0 = __builtin_amdgcn_s_memrealtime();
            int got;
            for (;;) {
                got = level_of(ld_sys(c));
                if (got >= want) break;
                if (give_up(tmo, call, t0)) { ctl[0] = 1u; got = 5; break; }
                __builtin_amdgcn_s_sleep(4);
            }
            ctl[2] = (unsigned)got;
            __builtin_amdgcn_fence(__ATOMIC_ACQUIRE, "agent");
            asm volatile("s_waitcnt vmcnt(0) lgkmcnt(0)" ::: "memory");
        }
        asm volatile("" ::: "memory"); __builtin_amdgcn_s_barrier(); asm volatile("" ::: "memory");
        const int r = __builtin_amdgcn_readfirstlane(ctl[2]);
        asm volatile("s_waitcnt lgkmcnt(0)" ::: "memory");
        return r;
    };

#define SA(b, h) (((b) * 2 + (h)) * HTB)
#define SB(b, h) ((4 + (b) * 2 + (h)) * HTB)
#define STAGE(bufoff, soff) do { _Pragma("unroll") for (int _i = 0; _i < 2; ++_i) \
        __builtin_amdgcn_raw_ptr_buffer_load_lds(rws, (LAS unsigned*)(lds + (bufoff) + ldsw + _i * 8192), 16, voff, (soff) + _i * 8192u, 0, 0); } while (0)
#define LDA(dst, b, h) do { _Pragma("unroll") for (int m = 0; m < 4; ++m) _Pragma("unroll") for (int k = 0; k < 2; ++k) dst[m][k] = *(const LAS bf16x8*)(lds + SA(b, h) + aoff + m * 2048 + k * 1024); } while (0)
#define LDB(dst, b, h) do { _Pragma("unroll") for (int n = 0; n < 2; ++n) _Pragma("unroll") for (int k = 0; k < 2; ++k) dst[n][k] = *(const LAS bf16x8*)(lds + SB(b, h) + boff + n * 2048 + k * 1024); } while (0)
#define MMA(ai, first, prio) do { __builtin_amdgcn_s_setprio(prio); \
        _Pragma("unroll") for (int i = 0; i < 32; ++i) { \
            const int sn = ((i >> 2) & 1) ? 3 - (i & 3) : i & 3; \
            const int bj = CFG_MO == 7 ? ((i >> 2) ^ (i >> 3)) & 1 : CFG_MO == 0 || CFG_MO == 4 ? i >> 4 : CFG_MO == 1 ? (i >> 1) & 1 : CFG_MO == 5 ? sn >> 1 : (i >> 3) & 1; \
            const int m = CFG_MO == 7 ? i >> 3 : CFG_MO == 2 ? i & 3 : CFG_MO == 6 ? sn : (i >> 2) & 3; \
            const int n = CFG_MO == 7 ? ((i >> 1) ^ (i >> 3)) & 1 : CFG_MO == 0 || CFG_MO == 4 ? (i >> 1) & 1 : CFG_MO == 1 ? i & 1 : CFG_MO == 5 ? sn & 1 : (i >> 2) & 1; \
            const int k = CFG_MO == 7 ? (i ^ (i >> 1)) & 1 : CFG_MO == 0 || CFG_MO == 4 ? i & 1 : i >> 4; \
            if (CFG_MO >= 4) SCHED; \
            acc[ai][bj][m][n] = __builtin_amdgcn_mfma_f32_16x16x32_bf16(bj ? B1[n][k] : B0[n][k], At[m][k], first && (CFG_MO == 7 ? (i & 1) == 0 : k == 0) ? (f32x4){0.f, 0.f, 0.f, 0.f} : acc[ai][bj][m][n], 0, 0, 0); } \
        __builtin_amdgcn_s_setprio(0); } while (0)
#define WAIT_L0() do { asm volatile("" ::: "memory"); __builtin_amdgcn_s_waitcnt(0xC07F); asm volatile("" ::: "memory"); } while (0)
#define WAIT_V(n) asm volatile("s_waitcnt vmcnt(" #n ")" ::: "memory")
#define WAIT_L(n) asm volatile("s_waitcnt lgkmcnt(" #n ")" ::: "memory")
#define BAR __builtin_amdgcn_s_barrier()
#define SCHED __builtin_amdgcn_sched_barrier(0)

    unsigned long long outp = (unsigned long long)P.out[rank];
    asm volatile("" : "+v"(outp));
    Tile cur, nxt; int rnd = 0;
    tile_of(slot, col, rank, cur);
    bf16x8 At[4][2], B0[2][2], B1[2][2];
    constexpr unsigned SB0 = WS_SND + ((NRANK - 1) * NS + GW) * UNB, STB = U0 * UNB;
    const unsigned kov = WS_PAN + (unsigned)((rank + 1 + (lane + 3) % 7) & (NRANK - 1)) * CHB + (unsigned)(((lane & 31) - 4) / 7) * UNB;
    unsigned tabA, tabB;
    auto tables = [&](const Tile& t) {
        const int ln = lane_now();
        const unsigned la = t.sA + (unsigned)ln * t.stA, ra = (unsigned)(NTM + t.v) * PANB + kov;
        const unsigned lb = SB0 + (unsigned)t.tm * UNB + (unsigned)ln * STB, rb = (unsigned)t.tm * PANB + kov;
        tabA = ln < 4 ? la : ra;
        tabB = (ln < 4 ? lb : rb) + (t.half == 1 ? HTB : 0u);
    };
    auto pick = [&]() {
        Tile t = cur; bool ok = true;
        if (rnd + 1 < RS) tile_of((rnd + 1) * KX + slot, col, rank, t);
        else {
            const unsigned d = (unsigned)__builtin_amdgcn_raw_ptr_buffer_atomic_add_i32(1, rws, lane_now() == 0 ? 0u : 0xfffffff0u, WS_HEAD + (unsigned)rank * 128u, 0);
            const unsigned q = (unsigned)__builtin_amdgcn_readfirstlane(d) - (call - 1u) * (unsigned)(NITEM + 8 * KX);
            ok = q < (unsigned)NITEM;
            if (ok) item_of((int)q, rank, t);
        }
        *(LAS u32x4*)(ctl + 4) = (u32x4){ok ? (unsigned)(t.tm | t.v << 8 | (t.half + 2) << 16) : 0u, t.sA, t.stA, 0u};
        __builtin_amdgcn_raw_ptr_buffer_load_lds(rws, ctl + 64, 4, cnt_of(t), 0, 0, AUX_SYS);
    };
    auto picked = [&](int& level) -> bool {
        const u32x4 f = *(const LAS u32x4*)(ctl + 4);
        level = level_of(ctl[64 + lane_now()]);
        const unsigned w = __builtin_amdgcn_readfirstlane(f[0]);
        nxt.tm = w & 255u; nxt.v = (w >> 8) & 255u; nxt.half = (int)(w >> 16) - 2;
        nxt.sA = __builtin_amdgcn_readfirstlane(f[1]); nxt.stA = __builtin_amdgcn_readfirstlane(f[2]);
        return w != 0u;
    };
    constexpr int JP = KT / 2 - 4, JI = KT / 2 - 2, JL = KT / 2 - 1;
    const int jend = wr == 0 && wid == 0 ? JP : JL;
    int lvl = ready(cur, 1);
    STAMP(P, 2);
    tables(cur);
    unsigned a2 = cur.sA, b2 = SB0 + (unsigned)cur.tm * UNB;
    STAGE(SB(0, 0), b2); STAGE(SA(0, 0), a2); STAGE(SB(0, 1), b2 + HTB); STAGE(SA(0, 1), a2 + HTB);
    STAGE(SB(1, 0), b2 + KTB); STAGE(SA(1, 0), a2 + KTB); STAGE(SB(1, 1), b2 + HTB + KTB);
    WAIT_V(0); BAR;
#define TICK(i, r) do {} while (0)
#define ROUND_DONE() do {} while (0)
    bool has_next = false;
#pragma unroll 1
    for (;;) {
        f32x4 acc[2][2][4][2];
        int jchk = lvl < 5 ? 7 * lvl - 4 : jend;
        TICK(0, 2); TICK(4, 3);
#define SYNC_MMA(ai, first, wv) do { if (wv && (wr == 1 || !CFG_LW)) WAIT_V(8); WAIT_L0(); if (wr == 1) BAR; MMA(ai, first, wr + 1); if (wv && wr == 0 && CFG_LW) WAIT_V(8); if (wr == 0) BAR; SCHED; } while (0)
#define PASS(first, check) do { \
            LDB(B0, 0, 0); LDA(At, 0, 0); LDB(B1, 0, 1); STAGE(SA(1, 1), a2 + KTB + HTB); SCHED; \
            check; \
            a2 = __builtin_amdgcn_readlane(tabA, (j + 1) & 31); b2 = __builtin_amdgcn_readlane(tabB, (j + 1) & 31); \
            SYNC_MMA(0, first, !first); \
            LDA(At, 0, 1); STAGE(SB(0, 0), b2); STAGE(SA(0, 0), a2); STAGE(SB(0, 1), b2 + HTB); \
            SYNC_MMA(1, first, !first); \
            LDB(B0, 1, 0); LDA(At, 1, 0); LDB(B1, 1, 1); STAGE(SA(0, 1), a2 + HTB); \
            SYNC_MMA(0, false, true); \
            LDA(At, 1, 1); STAGE(SB(1, 0), b2 + KTB); STAGE(SA(1, 0), a2 + KTB); STAGE(SB(1, 1), b2 + KTB + HTB); \
            SYNC_MMA(1, false, true); } while (0)
        int j = 0;
        PASS(true, );
#pragma unroll 1
        for (j = 1; j < KT / 2; ++j)
            PASS(false, if (__builtin_expect(j == jchk, 0)) {
                    if (j < JP) { lvl = ready(cur, lvl + 1); jchk = lvl < 5 ? 7 * lvl - 4 : jend; }
                    else if (wr == 0 && j == JP) { pick(); jchk = JI; }
                    else if (wr == 0 && j == JI) { asm volatile("buffer_inv sc1" ::: "memory"); jchk = JL; }
                    else {
                        has_next = picked(lvl);
                        if (has_next) {
                            const int want = nxt.half < 0 ? 1 : 5;
                            if (lvl < want) lvl = ready(nxt, want);
                            tables(nxt);
                        }
                    }
                });
        if (has_next && nxt.half >= 0) STAGE(SA(1, 1), a2 + KTB + HTB);
        TICK(1, 2);
        TICK(2, 2);
        {
            const int ln = lane_now(), fr = ln & 15, fq = ln >> 4;
            const rsrc_t rol = mk_rsrc((const void*)outp, OUTB);
            const bool shared = cur.v < NS;
            if (__builtin_amdgcn_readfirstlane(ctl[0]) != 0u) {
#pragma unroll
                for (int i = 0; i < 32; ++i) acc[i >> 4][(i >> 3) & 1][(i >> 1) & 3][i & 1] = __builtin_bit_cast(f32x4, (u32x4){0x7fc00000u, 0x7fc00000u, 0x7fc00000u, 0x7fc00000u});
            }
            if (rnd == 0 && cur.v < GW) {
                const unsigned long long t0 = __builtin_amdgcn_s_memrealtime();
                while (__builtin_amdgcn_s_memrealtime() - t0 < (unsigned long long)((col & 3) * CFG_STAG + (col >> 2) * CFG_STAGB)) __builtin_amdgcn_s_sleep(8);
            }
            const bool thru = rnd >= RS;
            const unsigned vo = (unsigned)(wc * 32 + fq * 4) * (NN * 2) + (unsigned)(wr * 256 + fr * 16);
            const unsigned so = (unsigned)(cur.tm * 256) * (NN * 2) + (unsigned)gpan_of(cur.v, rank) * 512u;
            const unsigned vs = (unsigned)(wc * 32 + fq * 4) * ROWB + (unsigned)(wr * 256 + fr * 16);
            const unsigned ss = WS_OSN + (unsigned)(cur.tm * 256) * ROWB + (unsigned)cur.v * 512u;
            auto finish = [&](auto mode) {
#pragma unroll
                for (int bj = 0; bj < 2; ++bj)
#pragma unroll
                    for (int n = 0; n < 2; ++n)
#pragma unroll
                        for (int r = 0; r < 4; ++r) {
                            u32x4 o;
                            if ((r & 1) == 0) {
#pragma unroll
                                for (int i = 0; i < 8; ++i) { f32x4& a = acc[i >> 2][bj][i & 3][n]; const f32x2 z = silu2((f32x2){a[r], a[r + 1]}); a[r] = z[0]; a[r + 1] = z[1]; }
                            }
#pragma unroll
                            for (int i = 0; i < 4; ++i) o[i] = cvt_pk_bf16(acc[i >> 1][bj][(i & 1) * 2][n][r], acc[i >> 1][bj][(i & 1) * 2 + 1][n][r]);
                            if (mode == 1) STORE16(o, rws, vs, ss + (unsigned)(bj * 128 + n * 16 + r) * ROWB, AUX_SYS);
                            STORE16(o, rol, vo, so + (unsigned)(bj * 128 + n * 16 + r) * (NN * 2), mode == 2 ? AUX_SYS : CFG_RST);
                            if (mode == 2 && CFG_LAST > 1) { WAIT_VM0(); __builtin_amdgcn_s_sleep(CFG_LAST > 1 ? CFG_LAST - 1 : 1); }
                        }
            };
            if (shared) finish(std::integral_constant<int, 1>());
            else if (thru) finish(std::integral_constant<int, 2>());
            else finish(std::integral_constant<int, 0>());
            WAIT_V(16);
            if (shared) {
                WAIT_VM0();
                unsigned old = 0;
                if (ln == 0) old = __hip_atomic_fetch_add(ctl + 1, 1u, __ATOMIC_RELAXED, __HIP_MEMORY_SCOPE_WORKGROUP);
                old = __builtin_amdgcn_readfirstlane(old);
                if ((old & 7u) == 7u && ln < NRANK && ln != rank)
                    st_sys((unsigned*)(P.ws[ln] + WS_FLAG + rank * FLAGB) + NIN * 4 + cur.tm * NS + cur.v, call);
            }
        }
        TICK(3, 2);
        ROUND_DONE();
        if (!has_next) break;
        cur = nxt; ++rnd;
        if (cur.half >= 0) break;
    }
#define HB(b, i) ((b) < 2 ? ((i) < 2 ? SA(b, i) : SB(b, 0)) : (i) < 2 ? SB(i, 1) : 8 * HTB)
#define HSTAGE(b, t) do { const unsigned _o = ((t) & 1) * KTB, _a = __builtin_amdgcn_readlane(tabA, ((t) >> 1) & 31) + _o, _b = __builtin_amdgcn_readlane(tabB, ((t) >> 1) & 31) + _o; \
        STAGE(HB(b, 0), _a); STAGE(HB(b, 1), _a + HTB); STAGE(HB(b, 2), _b); } while (0)
#define HLOAD(b) do { _Pragma("unroll") for (int i = 0; i < 4; ++i) B0[i >> 1][i & 1] = *(const LAS bf16x8*)(lds + HB(b, 2) + boff + i * 1024); \
        _Pragma("unroll") for (int i = 0; i < 8; ++i) At[i >> 1][i & 1] = *(const LAS bf16x8*)(lds + HB(b, 0) + aoff + i * 1024); \
        _Pragma("unroll") for (int i = 0; i < 8; ++i) Au[i >> 1][i & 1] = *(const LAS bf16x8*)(lds + HB(b, 1) + aoff + i * 1024); } while (0)
#define HPASS(b, t, first) do { \
        if (wr == 0) { HLOAD(b); HSTAGE((b + 2) % 3, (t) + 2); } else { HSTAGE((b + 2) % 3, (t) + 2); SCHED; HLOAD(b); if (!first) WAIT_V(6); } \
        WAIT_L0(); if (wr == 1) BAR; \
        __builtin_amdgcn_s_setprio(wr + 1); \
        _Pragma("unroll") for (int i = 0; i < 32; ++i) { const int ai = i >> 4, m = (i >> 2) & 3, n = (i >> 1) & 1, k = i & 1; \
            acc[ai][m][n] = __builtin_amdgcn_mfma_f32_16x16x32_bf16(B0[n][k], ai ? Au[m][k] : At[m][k], first && k == 0 ? (f32x4){0.f, 0.f, 0.f, 0.f} : acc[ai][m][n], 0, 0, 0); } \
        __builtin_amdgcn_s_setprio(0); \
        if (wr == 0) { if (!first) WAIT_V(6); BAR; } SCHED; } while (0)
#pragma unroll 1
    while (PFULL < PN && has_next) {
        f32x4 acc[2][4][2];
        bf16x8 Au[4][2];
        BAR;
        HPASS(0, 0, true);
#pragma unroll 1
        for (int t = 1; t < KT; t += 3) {
            if (wr == 0 && wid == 0 && t == KT - 9) pick();
            HPASS(1, t, false); HPASS(2, t + 1, false); HPASS(0, t + 2, false);
        }
        has_next = picked(lvl);
        if (has_next) {
            (void)ready(nxt, 5);
            tables(nxt);
            HSTAGE(0, 0); HSTAGE(1, 1);
        }
        const int ln = lane_now(), fr = ln & 15, fq = ln >> 4;
        const rsrc_t rol = mk_rsrc((const void*)outp, OUTB);
        if (__builtin_amdgcn_readfirstlane(ctl[0]) != 0u) {
#pragma unroll
            for (int i = 0; i < 16; ++i) acc[i >> 3][(i >> 1) & 3][i & 1] = __builtin_bit_cast(f32x4, (u32x4){0x7fc00000u, 0x7fc00000u, 0x7fc00000u, 0x7fc00000u});
        }
        const unsigned vo = (unsigned)(wc * 32 + fq * 4) * (NN * 2) + (unsigned)(wr * 256 + fr * 16);
        const unsigned so = (unsigned)(cur.tm * 256 + cur.half * 128) * (NN * 2) + (unsigned)gpan_of(cur.v, rank) * 512u;
#pragma unroll
        for (int n = 0; n < 2; ++n)
#pragma unroll
            for (int r = 0; r < 4; ++r) {
                u32x4 o;
                if ((r & 1) == 0) {
#pragma unroll
                    for (int i = 0; i < 8; ++i) { f32x4& a = acc[i >> 2][i & 3][n]; const f32x2 z = silu2((f32x2){a[r], a[r + 1]}); a[r] = z[0]; a[r + 1] = z[1]; }
                }
#pragma unroll
                for (int i = 0; i < 4; ++i) o[i] = cvt_pk_bf16(acc[i >> 1][(i & 1) * 2][n][r], acc[i >> 1][(i & 1) * 2 + 1][n][r]);
                STORE16(o, rol, vo, so + (unsigned)(n * 16 + r) * (NN * 2), AUX_SYS);
            }
        WAIT_V(8);
        ROUND_DONE();
        cur = nxt; ++rnd;
    }
    WAIT_V(0);
}

__global__ __launch_bounds__(512, 2) void fused_gemm_ar(Params P) {
    extern __shared__ __attribute__((aligned(16))) unsigned char shm[];
    LAS unsigned char* lds = (LAS unsigned char*)shm;
    const int wid = __builtin_amdgcn_readfirstlane(threadIdx.x >> 6), lane = threadIdx.x & 63;
    const int b = __builtin_amdgcn_readfirstlane(blockIdx.x);
    unsigned* tkp = (unsigned*)(P.ws[P.rank] + WS_TICKET + b * 128 + wid * 16);
    const unsigned tk = ld_agent(tkp);
    if (b < NPULL) {
        const unsigned call = __builtin_amdgcn_readfirstlane(tk) + 1u;
        st_agent(tkp, call);
        const int pi = b / PW, slot = b % PW;
        const int peer = (P.rank + 1 + pi) & (NRANK - 1);
        pull_role(P, call, peer, slot * 8 + wid, lane);
    } else {
        const int g = b - NPULL;
        ((LAS unsigned*)(lds + STAGE_BYTES))[lane & 3] = 0u;
        STAMP(P, 0);
        pack_role(P, tk, g, wid, lane);
        const unsigned call = __builtin_amdgcn_readfirstlane(tk) + 1u;
        st_agent(tkp, call);
        STAMP(P, 1);
        asm volatile("s_waitcnt lgkmcnt(0)" ::: "memory");
        __builtin_amdgcn_s_barrier();
        if (b >= NPS) {
            const int col = CFG_SWAP == 1 ? (b & 7) ^ (b >> 1 & 1) : CFG_SWAP == 2 ? (b & 4) | ((0x1320 >> (b & 3) * 4) & 3) : b & 7;
            if (wid < 4) gemm_role<0>(P, call, col, (b - NPS) >> 3, lds);
            else gemm_role<1>(P, call, col, (b - NPS) >> 3, lds);
        }
    }
}
}

extern "C" void kernel_launch(int rank, int nranks, ncclComm_t comm,
                              hipStream_t stream,
                              void* const* d_in, const int* in_sizes, int n_in,
                              void* d_out, int out_size,
                              const int* in_dtypes, int out_dtype,
                              void* const* d_out_all,
                              void* const* d_ws_all, size_t ws_bytes) {
    if (nranks != NRANK || (unsigned)rank >= (unsigned)NRANK || ws_bytes < (size_t)WS_END || n_in < 2 ||
        in_sizes[0] != MM * KSH || in_sizes[1] != KSH * NN || out_size != MM * NN) {
        fprintf(stderr, "unexpected problem geometry\n");
        return;
    }
    static bool attr_done[NRANK] = {};
    if (!attr_done[rank]) {
        HIP_CHECK(hipFuncSetAttribute((const void*)fused_gemm_ar, hipFuncAttributeMaxDynamicSharedMemorySize, LDS_BYTES));
        attr_done[rank] = true;
    }
    Params P;
    P.x = d_in[0]; P.w = d_in[1]; P.rank = rank;
    for (int p = 0; p < NRANK; ++p) { P.out[p] = (char*)d_out_all[p]; P.ws[p] = (char*)d_ws_all[p]; }
    hipLaunchKernelGGL(fused_gemm_ar, dim3(GRID), dim3(512), LDS_BYTES, stream, P);
    HIP_CHECK(hipGetLastError());
}
```

```cpp
#include <hip/hip_runtime.h>
#include <rccl/rccl.h>
#include <cstdio>
#include <type_traits>

#define HIP_CHECK(expr) do { hipError_t _e = (expr); if (_e != hipSuccess) { \
    fprintf(stderr, "HIP error %s at %s:%d\n", hipGetErrorString(_e), __FILE__, __LINE__); \
    return; } } while (0)

#define LAS __attribute__((address_space(3)))
typedef short bf16x8 __attribute__((ext_vector_type(8)));
typedef float f32x4 __attribute__((ext_vector_type(4)));
typedef unsigned u32x4 __attribute__((ext_vector_type(4)));
typedef __amdgpu_buffer_rsrc_t rsrc_t;

namespace {
constexpr int NRANK = 8;
constexpr int MM = 4096, KK = 4096, NN = 32768, KSH = KK / NRANK;
constexpr int KT = KK / 64, KTS = KSH / 64, NTM = MM / 256, NPAN = NN / 256;
#define CFG_NS 6
#define CFG_PW 8
constexpr int NS = CFG_NS;
constexpr int NRP = NPAN - NRANK * NS;
constexpr int NWP = NS + NRP;
constexpr int NIN = NTM + NWP;
constexpr int NOUT = NS * NTM;
constexpr int NFLAG = NIN * 4 + NOUT;
constexpr int NUNIT = NTM * 4 + NPAN * 4;
constexpr int GRID = 256;
constexpr int PW = CFG_PW;
constexpr int NPULL = PW * (NRANK - 1);
constexpr int NW = PW * 8;
constexpr int NPK = GRID - NPULL;
constexpr int NPS = (NPULL + 7) / 8 * 8;
constexpr int KX = (GRID - NPS) / 8;
#define CFG_GW (2 * (KX / 4))
constexpr int GW = CFG_GW;
constexpr int G0 = GW + NTM;
#define CFG_GM 2
constexpr int GM = CFG_GM;
constexpr int NGF = (NWP - GW - GM) / GW, GL = (NWP - GW - GM) % GW;
static_assert(NS <= GW && NWP > GW, "shared panels sit in the first group");
#define CFG_PU 4
constexpr int PU = CFG_PU, PSH = PU == 1 ? 0 : PU == 2 ? 1 : PU == 4 ? 2 : 3;
#define CFG_PUSH 3
constexpr int NPUSH = CFG_PUSH;
constexpr int NPIN = (NIN * 4 - NPUSH) * 64 / PU;
static_assert(NPUSH <= NS, "pushed units have a single consumer");
constexpr int NPC = NPIN + NOUT * 128 / PU;
constexpr unsigned CNT_FULL = 64;

constexpr int NT0 = 4 * ((NWP + 1) / 2), NT1 = 4 * (NWP / 2);
constexpr int RS = (NT1 - 1) / KX;
constexpr int PL0 = NT0 - RS * KX, PL1 = NT1 - RS * KX, PN = 4 * (PL0 + PL1);
#define CFG_MO 7
#define CFG_SWAP 1
#define CFG_LW 0
#define CFG_FD (-8)
constexpr int PFULL = 2 * PN - 8 * KX - CFG_FD < 0 ? 0 : 2 * PN - 8 * KX - CFG_FD > PN ? PN : 2 * PN - 8 * KX - CFG_FD;
constexpr int NITEM = PFULL + 2 * (PN - PFULL);
constexpr unsigned KTB = 32768, HTB = 16384;
constexpr unsigned PANB = KT * KTB;
constexpr unsigned CHB = KTS * KTB;
constexpr unsigned UNB = 2 * KTB;
constexpr unsigned WS_TICKET = 0;
constexpr unsigned WS_TMO = WS_TICKET + GRID * 128;
constexpr unsigned WS_FLAG = WS_TMO + 128;
constexpr unsigned FLAGB = 4096;
constexpr unsigned WS_DONE = WS_FLAG + NRANK * FLAGB;
constexpr unsigned WS_HEAD = WS_DONE + NRANK * 512;
constexpr unsigned WS_PH = WS_HEAD + NRANK * 128;
constexpr unsigned WS_ZC = WS_PH + 2 * NRANK * 128 + 128;
constexpr unsigned WS_CNT = WS_ZC + NRANK * NRANK * 128;
constexpr unsigned WS_CTRL_END = WS_CNT + NIN * NRANK * 128;
constexpr unsigned WS_DBG = 512u << 10;
constexpr unsigned WS_SND = 1u << 20;
constexpr unsigned ROWB = NS * 512;
constexpr unsigned WS_OSN = WS_SND + NUNIT * UNB;
constexpr unsigned WS_PAN = (WS_OSN + MM * ROWB + PANB - 1) / PANB * PANB;
constexpr unsigned WS_END = WS_PAN + NIN * PANB;
constexpr int U0 = G0 + (NRANK - 1) * NS;
constexpr unsigned OUTB = (unsigned)MM * NN * 2;
static_assert(WS_CTRL_END <= WS_DBG, "ctrl region");
static_assert(NFLAG <= 512 && 2048 <= (int)FLAGB, "flag region");
static_assert(NW <= 128, "done words");
static_assert(PU == 4, "block size");
#define CFG_DEEP 3
constexpr unsigned STAGE_BYTES = 9 * HTB;
constexpr unsigned LDS_BYTES = STAGE_BYTES + 512;
constexpr unsigned long long DEADLINE = 50000000ull;
constexpr int AUX_SYS = 17, AUX_SYS_NT = 19, AUX_AGENT = 16;
#define CFG_PACE 140
#define CFG_PACE0 40
#define CFG_STAG 600
#define CFG_P3 320
#define CFG_P3N 2
#define CFG_STAGB 300
#define CFG_LOOK 256
#define CFG_TAIL NW
#define CFG_ZB 120
#define CFG_LOOKP 100
#define CFG_PLAN (CFG_ZB + CFG_LOOKP + 50)
#define CFG_BAL 8
constexpr int ZB = CFG_ZB;
static_assert(ZB <= 127 && ZB <= NS * 128 / PU && NFLAG <= 504, "plan");
#define CFG_LAST 1
#define CFG_RST 0
#define CFG_RATE 1710
#define CFG_CRED 25
constexpr unsigned PACE16 = CFG_RATE * PW * (8 + CFG_DEEP) / 64;

struct Params {
    const void* x; const void* w;
    char* out[NRANK]; char* ws[NRANK];
    int rank;
};

__device__ __forceinline__ rsrc_t mk_rsrc(const void* p, unsigned bytes) {
    const unsigned long long v = (unsigned long long)p;
    const unsigned lo = __builtin_amdgcn_readfirstlane((unsigned)v), hi = __builtin_amdgcn_readfirstlane((unsigned)(v >> 32));
    return __builtin_amdgcn_make_buffer_rsrc((void*)(((unsigned long long)hi << 32) | lo), 0, bytes, 0x00020000);
}
__device__ __forceinline__ unsigned ld_sys(const unsigned* p) { return __hip_atomic_load(p, __ATOMIC_RELAXED, __HIP_MEMORY_SCOPE_SYSTEM); }
__device__ __forceinline__ unsigned ld_agent(const unsigned* p) { return __hip_atomic_load(p, __ATOMIC_RELAXED, __HIP_MEMORY_SCOPE_AGENT); }
__device__ __forceinline__ void st_sys(unsigned* p, unsigned v) { __hip_atomic_store(p, v, __ATOMIC_RELAXED, __HIP_MEMORY_SCOPE_SYSTEM); }
__device__ __forceinline__ void st_agent(unsigned* p, unsigned v) { __hip_atomic_store(p, v, __ATOMIC_RELAXED, __HIP_MEMORY_SCOPE_AGENT); }
__device__ __forceinline__ void add_agent(unsigned* p, unsigned v) { (void)__hip_atomic_fetch_add(p, v, __ATOMIC_RELAXED, __HIP_MEMORY_SCOPE_AGENT); }
#define STORE16(v, rs, vo, so, aux) do { const u32x4 _d = (v); __builtin_amdgcn_raw_buffer_store_b128(_d, rs, vo, so, aux); asm volatile("s_nop 1" :: "v"(_d) : "memory"); } while (0)
#define WAIT_VM0() asm volatile("s_waitcnt vmcnt(0)" ::: "memory")
#define WAIT_VMN(n) do { asm volatile("" ::: "memory"); __builtin_amdgcn_s_waitcnt(0x0F70 | (n)); asm volatile("" ::: "memory"); } while (0)

__device__ __forceinline__ bool give_up(unsigned* tmo, unsigned call, unsigned long long t0) {
    const unsigned long long dt = __builtin_amdgcn_s_memrealtime() - t0;
    if (dt < 2000ull) return false;
    if ((unsigned)__builtin_amdgcn_readfirstlane(ld_agent(tmo)) == call) return true;
    if (dt > DEADLINE) { st_agent(tmo, call); return true; }
    return false;
}

__device__ __forceinline__ int gpan_of(int v, int consumer) { return v < NS ? consumer * NS + v : (NRANK - 1) * NS + v; }
__device__ __forceinline__ int band_of(int e, int consumer) { return e < 7 ? (consumer + 1 + e) & 7 : e < 15 ? 8 + ((consumer + e) & 7) : consumer; }
__device__ __forceinline__ int turn_of(int band, int consumer) { return band >= 8 ? 7 + ((band - consumer + 1) & 7) : band == consumer ? 15 : (band - consumer - 1) & 7; }
__device__ __forceinline__ int unit_of(int o, int consumer, int& q, int& pos) {
    if (o < 4 * G0) { q = o / G0; const int m = o % G0; pos = q * U0 + gpan_of(m, consumer); return m < GW ? NTM + m : m - GW; }
    pos = o + 4 * (NRANK - 1) * NS;
    o -= 4 * G0;
    if (o < 4 * GM) { q = o / (GM ? GM : 1); return NTM + GW + o % (GM ? GM : 1); }
    o -= 4 * GM;
    const int g = o / (4 * GW), r = o % (4 * GW);
    int i;
    if (GL == 0 || g < NGF) { q = r / GW; i = r % GW; } else { q = r / (GL ? GL : 1); i = r % (GL ? GL : 1); }
    return NTM + GM + GW * (g + 1) + i;
}

__device__ __forceinline__ unsigned cvt_pk_bf16(float lo, float hi) { unsigned r; asm("v_cvt_pk_bf16_f32 %0, %1, %2" : "=v"(r) : "v"(lo), "v"(hi)); return r; }
__device__ __forceinline__ float silu(float y) {
    const float c = __builtin_amdgcn_fmed3f(y, -60.0f, 60.0f);
    const float e = __builtin_amdgcn_exp2f(c * -1.44269504088896f);
    return y * __builtin_amdgcn_rcpf(1.0f + e);
}

typedef float f32x2 __attribute__((ext_vector_type(2)));
__device__ __forceinline__ f32x2 silu2(f32x2 y) {
    const f32x2 t = __builtin_elementwise_fma(y, (f32x2){-1.44269504088896f, -1.44269504088896f}, (f32x2){-86.5617024533378f, -86.5617024533378f});
    const f32x2 e = {__builtin_amdgcn_fmed3f(__builtin_amdgcn_exp2f(t[0]), 0.0f, 1.0f), __builtin_amdgcn_fmed3f(__builtin_amdgcn_exp2f(t[1]), 0.0f, 1.0f)};
    const f32x2 d = __builtin_elementwise_fma(e, (f32x2){1.1420073898156842e26f, 1.1420073898156842e26f}, (f32x2){1.0f, 1.0f});
    return y * (f32x2){__builtin_amdgcn_rcpf(d[0]), __builtin_amdgcn_rcpf(d[1])};
}

#define STAMP1(P, slot) do {} while (0)
#define STAMP(P, slot) do {} while (0)

__device__ __forceinline__ void pull_role(const Params& P, unsigned call, int peer, int widx, int lane) {
    const int rank = P.rank;
    char* wsl = P.ws[rank];
    const rsrc_t rwr = mk_rsrc(P.ws[peer], WS_END), rwl = mk_rsrc(wsl, WS_END);
    const rsrc_t rol = mk_rsrc(P.out[rank], OUTB), ror = mk_rsrc(P.out[peer], OUTB);
    unsigned* tmo = (unsigned*)(wsl + WS_TMO);
    constexpr unsigned OOB = 0xfffffff0u;
    const unsigned lane0 = lane == 0 ? 0u : OOB;
    const unsigned hoff = WS_PH + (call & 1u) * (NRANK * 128u);
    auto draw = [&]() { return (unsigned)__builtin_amdgcn_raw_ptr_buffer_atomic_add_i32(1, rwl, lane0, hoff + (unsigned)peer * 128u, 0); };
    if (widx == 0) __builtin_amdgcn_raw_buffer_store_b32(0u, rwl, lane0, WS_PH + (~call & 1u) * (NRANK * 128u) + (unsigned)peer * 128u, AUX_SYS);
    auto arrive = [&](unsigned vo, unsigned cnt) { (void)__builtin_amdgcn_raw_ptr_buffer_atomic_add_i32(PU, rwl, vo, cnt, 0); };
    const bool deep0 = (widx & 7) < CFG_DEEP;
    bool bad = false; int stamped = 0; (void)stamped;
    STAMP(P, 0);
    auto index_of = [&](unsigned v) { return (int)__builtin_amdgcn_readfirstlane(v); };
    int cur = index_of(draw());
    constexpr int BPB = NS * 128 / PU;
    const unsigned fso = WS_FLAG + (unsigned)peer * FLAGB;
    auto first_of = [&](int i) { return i < NPIN ? (i >> (6 - PSH)) + NPUSH : NIN * 4 + (i - NPIN) / BPB * NS; };
    int lim = 0, flim = 0, fin = NPC;
    bool planned = ZB == 0;
    unsigned long long plan = 0;
    auto moved = [&](int j) { return (int)(signed char)(plan >> (j * 8)); };
    auto update = [&](const u32x4& f0, const u32x4& f1, int i) {
        const int first = first_of(i);
        unsigned un = 0;
#pragma unroll
        for (int j = 0; j < 8; ++j) un |= ((int)((j < 4 ? f0[j & 3] : f1[j & 3]) - call) < 0 && lane * 8 + j >= first ? 1u : 0u) << j;
        const int l = __builtin_ctzll(__ballot(un != 0u) | (1ull << 63));
        const int u = l * 8 + __builtin_ctz(__builtin_amdgcn_readlane(un, l) | 256u);
        const int fl = u <= NIN * 4 ? (u - NPUSH) << (6 - PSH) : u >= NFLAG ? NPC + 128 : NPIN + (u - NIN * 4) / NS * BPB;
        flim = fl > flim ? fl : flim;
        if (!planned && (unsigned)__builtin_amdgcn_readlane(f0[0], 63) == call && (unsigned)__builtin_amdgcn_readlane(f0[2], 63) == call) {
            plan = (unsigned long long)(unsigned)__builtin_amdgcn_readlane(f0[3], 63) << 32 | (unsigned)__builtin_amdgcn_readlane(f0[1], 63);
            planned = true; fin = NPC - moved(peer);
        }
        const int pl = planned ? fin : NPC - ZB;
        lim = flim < pl ? flim : pl;
    };
    auto make_plan = [&]() {
        unsigned old = 0;
        if (lane == 0) old = __hip_atomic_exchange((unsigned*)(wsl + WS_PH + 2 * NRANK * 128), call, __ATOMIC_RELAXED, __HIP_MEMORY_SCOPE_AGENT);
        if ((unsigned)__builtin_amdgcn_readfirstlane(old) == call) return;
        const unsigned hd = __builtin_amdgcn_raw_buffer_load_b32(rwl, lane < NRANK ? (unsigned)lane * 128u : OOB, hoff, AUX_SYS);
        const int t = lane >> 3, s = lane & 7;
        const unsigned zc = ld_sys((const unsigned*)(P.ws[t] + WS_ZC + (rank * NRANK + s) * 128));
        const bool ok = __all(t == rank || s == rank || s == t || (int)(zc - call * (unsigned)(ZB * PU)) >= 0);
        int h[NRANK], sum = 0, give = 0, take = 0;
#pragma unroll
        for (int j = 0; j < NRANK; ++j) { h[j] = __builtin_amdgcn_readlane(hd, j); sum += j == rank ? 0 : h[j]; }
#pragma unroll
        for (int j = 0; j < NRANK; ++j) {
            const int d = j == rank ? 0 : (sum - (NRANK - 1) * h[j]) * CFG_BAL / (8 * (NRANK - 1));
            h[j] = d > ZB ? ZB : d < -127 ? -127 : d;
            if (h[j] > 0) give += h[j]; else take -= h[j];
        }
        give = take = !ok ? 0 : give < take ? give : take;
        unsigned long long pl = 0;
#pragma unroll
        for (int j = 0; j < NRANK; ++j) {
            int d = h[j];
            if (d > 0) { d = d < give ? d : give; give -= d; } else { d = -d < take ? d : -take; take += d; }
            pl |= (unsigned long long)(d & 255) << (j * 8);
        }
        __builtin_amdgcn_raw_buffer_store_b128((u32x4){call, (unsigned)pl, call, (unsigned)(pl >> 32)}, rwl, lane < NRANK ? WS_FLAG + 63 * 32 + (unsigned)lane * FLAGB : OOB, 0, AUX_SYS);
    };
    auto offs_of = [&](int i, unsigned (&o)[4], unsigned& cnt) -> unsigned {
        if (i < NPIN) {
            int q, pos; const int pan = unit_of((i >> (6 - PSH)) + NPUSH, rank, q, pos);
            cnt = WS_CNT + (unsigned)(pan * NRANK + peer) * 128u + (unsigned)q * 4u;
            const unsigned sub = (unsigned)((i << PSH) & 63) * 1024u;
            const unsigned b = WS_PAN + (unsigned)pan * PANB + (unsigned)peer * CHB + (unsigned)q * UNB + sub + (unsigned)lane * 16u;
#pragma unroll
            for (int k = 0; k < 4; ++k) o[k] = b + k * 1024u;
            return WS_SND + (unsigned)pos * UNB + sub;
        }
        int src = peer;
        if (ZB && i >= NPC) {
            int x = i - NPC;
            for (int j = 0; j < peer; ++j) x -= moved(j) < 0 ? moved(j) : 0;
            for (int j = 0; j < NRANK; ++j) { const int g = moved(j) > 0 ? moved(j) : 0; if ((unsigned)x < (unsigned)g) { src = j; i = NPC - g + x; } x -= g; }
        }
        const int e = (i - NPIN) / BPB, r = (i - NPIN) % BPB, pb = band_of(e, rank) * BPB + r;
        cnt = e < NRANK - 1 && r >= BPB - ZB ? WS_ZC + (unsigned)(band_of(e, rank) * NRANK + peer) * 128u : 0u;
#pragma unroll
        for (int k = 0; k < 4; ++k) {
            const unsigned sg = (unsigned)((pb << PSH) + k) * 2u + (unsigned)(lane >> 5);
            o[k] = (unsigned)src * ROWB + (unsigned)(lane & 31) * 16u + sg / NS * (NN * 2u) + sg % NS * 512u;
        }
        return WS_OSN + (unsigned)(pb << PSH) * 1024u;
    };
    while (!planned || cur < fin) {
        if (!bad) {
            const unsigned long long t0 = __builtin_amdgcn_s_memrealtime();
            const unsigned pvo = (unsigned)(lane - (first_of(cur) >> 3)) < 4u || (lane == 63 && cur >= NPC - ZB) ? (unsigned)lane * 32u : OOB - 16u;
            for (;;) {
                const u32x4 f0 = __builtin_amdgcn_raw_buffer_load_b128(rwl, pvo, fso, AUX_SYS), f1 = __builtin_amdgcn_raw_buffer_load_b128(rwl, pvo + 16u, fso, AUX_SYS);
                update(f0, f1, cur);
                if (cur < lim || (planned && cur >= fin)) break;
                if (give_up(tmo, call, t0)) { bad = true; planned = true; plan = 0; fin = NPC; break; }
                __builtin_amdgcn_s_sleep(2);
            }
            asm volatile("" ::: "memory");
            if (cur >= fin && planned) break;
        }
        unsigned o[4], p2 = OOB, p3 = OOB, cprev = 0;
        if (bad) {
            (void)offs_of(cur, o, cprev);
            if (cur >= NPIN) {
#pragma unroll
                for (int k = 0; k < PU; ++k) __builtin_amdgcn_raw_buffer_store_b128((u32x4){0x7fc07fc0u, 0x7fc07fc0u, 0x7fc07fc0u, 0x7fc07fc0u}, rol, o[k], 0, AUX_AGENT);
            }
            cur = index_of(draw()); continue;
        }
        STAMP1(P, 1);
        rsrc_t rp = rwl;
        u32x4 v[4] = {};
        const unsigned per = deep0 ? PACE16 / 2 : PACE16;
        unsigned slot = 0;
        if (CFG_RATE) { const unsigned now = (unsigned)__builtin_amdgcn_s_memrealtime() << 4; slot = now - (now & 0xffffffu) % PACE16 + (unsigned)((widx & 7) * PW + (widx >> 3)) * (PACE16 / NW) - PACE16; }
        unsigned cb, src, in[4];
        auto prepare = [&]() {
            if (ZB && cur == NPC - CFG_PLAN) make_plan();
            src = __builtin_amdgcn_readfirstlane(offs_of(cur, o, cb)); cb = __builtin_amdgcn_readfirstlane(cb);
            if (ZB && cur >= NPC) src = 0;
#pragma unroll
            for (int k = 0; k < 4; ++k) { in[k] = ZB && cur >= NPC ? o[k] : lane * 16 + k * 1024; asm volatile("" : "+v"(o[k]), "+v"(in[k])); }
        };
        prepare();
        do {
            const bool tail = cur >= fin - CFG_TAIL, deep = deep0 || tail;
            const rsrc_t rd = cur < NPIN ? rwl : rol, rs = ZB && cur >= NPC ? ror : rwr;
            const bool look = flim < NPC && cur + CFG_LOOK >= flim, lookp = !planned && cur + CFG_LOOKP >= NPC - ZB;
            const unsigned fvo = look || (lookp && lane == 63) ? (unsigned)lane * 32u : OOB - 16u;
            const unsigned cdone = cprev;
            unsigned nv = 0; u32x4 f0, f1;
#pragma unroll
            for (int k = 0; k < 4; ++k) {
                if (CFG_RATE && !tail) {
                    slot += per;
                    for (;;) {
                        const int d = (int)(((unsigned)__builtin_amdgcn_s_memrealtime() << 4) - slot);
                        if (d > CFG_CRED * 16) slot += (unsigned)(d - CFG_CRED * 16);
                        if (d >= 0) break;
                        __builtin_amdgcn_s_sleep(1);
                    }
                }
                v[k] = __builtin_amdgcn_raw_buffer_load_b128(rs, in[k], src, AUX_SYS_NT);
                if (k == 0) {
                    nv = draw();
                    f0 = __builtin_amdgcn_raw_buffer_load_b128(rwl, fvo, fso, AUX_SYS); f1 = __builtin_amdgcn_raw_buffer_load_b128(rwl, fvo + 16u, fso, AUX_SYS);
                }
                __builtin_amdgcn_raw_buffer_store_b128(v[k ^ 2], k < 2 ? rp : rd, k == 0 ? p2 : k == 1 ? p3 : o[k ^ 2], 0, AUX_AGENT);
                if (k == 3) {
                    asm volatile("" ::: "memory");
                    cprev = cb; rp = rd; p2 = o[2]; p3 = o[3];
                    cur = index_of(nv);
                    if (look || lookp) update(f0, f1, cur);
                    if (cur < lim) prepare();
                }
                if (k == 0) WAIT_VMN(7); else if (k == 1) WAIT_VMN(6); else WAIT_VMN(3);
                if (!deep) { if (k == 0) WAIT_VMN(4); else WAIT_VMN(1); }
            }
            arrive(cdone ? lane0 : OOB, cdone);
        } while (cur < lim);
        __builtin_amdgcn_raw_buffer_store_b128(v[2], rp, p2, 0, AUX_AGENT);
        __builtin_amdgcn_raw_buffer_store_b128(v[3], rp, p3, 0, AUX_AGENT);
        if (cprev) { WAIT_VM0(); arrive(lane0, cprev); }
    }
    STAMP(P, 4);
    asm volatile("" ::: "memory");
    if (lane == 0) st_sys((unsigned*)(P.ws[peer] + WS_DONE + rank * 512) + widx, call);
    if (widx == 0) {
        const unsigned* dn = (const unsigned*)(wsl + WS_DONE + peer * 512);
        const unsigned long long t0 = __builtin_amdgcn_s_memrealtime();
        for (;;) {
            bool ok = true;
            if (lane < NW) ok = (int)(ld_sys(dn + lane) - call) >= 0;
            if (NW > 64 && lane + 64 < NW) ok = ok && (int)(ld_sys(dn + 64 + lane) - call) >= 0;
            if (__all(ok)) break;
            if (give_up(tmo, call, t0)) {
                if (lane == 0) __builtin_amdgcn_raw_buffer_store_b32(0x7fc07fc0u, rol, (unsigned)(rank * NS) * 512u, 0, AUX_SYS);
                break;
            }
            __builtin_amdgcn_s_sleep(2);
        }
        STAMP(P, 5);
    }
}

__device__ __forceinline__ void pack_role(const Params& P, unsigned tk, int g, int wid, int lane) {
    const int rank = P.rank;
    char* wsl = P.ws[rank];
    const rsrc_t rx = mk_rsrc(P.x, (unsigned)MM * KSH * 2), rw = mk_rsrc(P.w, (unsigned)KSH * NN * 2);
    const int fr = lane & 15, fq = lane >> 4;
    const unsigned long long t0 = __builtin_amdgcn_s_memrealtime();
    for (int u = g; u < NUNIT; u += NPK) {
        int o, dest = -1;
        if (u < 4 * U0) {
            const int r = u % U0;
            if (r < NRANK * NS) { o = r >> 3; dest = (rank + 1 + r) & (NRANK - 1); } else o = r - (NRANK - 1) * NS;
            o += u / U0 * G0;
        } else o = u - 4 * (NRANK - 1) * NS;
        int q, pos; const int pan = unit_of(o, dest < 0 ? 0 : dest, q, pos);
        const bool push = o < NPUSH && dest != rank;
        const rsrc_t rwl = mk_rsrc(P.ws[push ? dest & (NRANK - 1) : rank], WS_END);
        while (CFG_PACE > 0 && (unsigned)(__builtin_amdgcn_s_memrealtime() - t0) < (unsigned)((u < CFG_PACE0 ? 0 : u - CFG_PACE0) * CFG_PACE / 10)) __builtin_amdgcn_s_sleep(1);
        const int ktl = q * 2 + (wid >> 2);
        const unsigned dst = push ? WS_PAN + (unsigned)pan * PANB + (unsigned)(rank * KTS + ktl) * KTB : WS_SND + (unsigned)pos * UNB + (unsigned)(wid >> 2) * KTB;
        u32x4 v[8];
        if (pan < NTM) {
            const int rb0 = (wid & 3) * 4;
            const unsigned vo = (unsigned)fr * (KSH * 2) + (unsigned)fq * 16u;
            const unsigned so = (unsigned)(pan * 256 + rb0 * 16) * (KSH * 2) + (unsigned)ktl * 128u;
#pragma unroll
            for (int i = 0; i < 8; ++i) v[i] = __builtin_amdgcn_raw_buffer_load_b128(rx, vo + (i & 1) * 64, so + (unsigned)(i >> 1) * (16 * KSH * 2), 0);
#pragma unroll
            for (int i = 0; i < 8; ++i) STORE16(v[i], rwl, lane * 16 + (i & 1) * 1024, dst + (unsigned)(rb0 + (i >> 1)) * 2048u, AUX_SYS);
        } else {
            const int ks = (wid >> 1) & 1, wr = wid & 1;
            const unsigned vo = (unsigned)(fq * 8) * (NN * 2) + (unsigned)fr * 16u;
            const unsigned so = (unsigned)(ktl * 64 + ks * 32) * (NN * 2) + (unsigned)(gpan_of(pan - NTM, dest < 0 ? 0 : dest) * 256 + wr * 128) * 2u;
#pragma unroll
            for (int j = 0; j < 8; ++j) v[j] = __builtin_amdgcn_raw_buffer_load_b128(rw, vo, so + (unsigned)j * (NN * 2), 2);
            while (CFG_P3 && push && o >= CFG_P3N && (unsigned)(__builtin_amdgcn_s_memrealtime() - t0) < (unsigned)(CFG_P3 + (o - CFG_P3N) * 110)) __builtin_amdgcn_s_sleep(1);
#pragma unroll
            for (int i = 0; i < 8; ++i) {
                u32x4 o;
#pragma unroll
                for (int e = 0; e < 4; ++e) o[e] = __builtin_amdgcn_perm(v[2 * e + 1][i >> 1], v[2 * e][i >> 1], (i & 1) ? 0x07060302u : 0x05040100u);
                STORE16(o, rwl, lane * 16, dst + (unsigned)(i >> 2) * HTB + (unsigned)wr * 8192u + (unsigned)(i & 3) * 2048u + (unsigned)ks * 1024u, AUX_SYS);
            }
        }
        WAIT_VM0();
        if (u == g) STAMP(P, 6);
        __builtin_amdgcn_s_barrier();
        if (wid == 0 && lane < NRANK && (dest < 0 || dest == lane)) {
            unsigned c = tk; asm volatile("" : "+v"(c));
            if (lane == rank || push) st_sys((unsigned*)(P.ws[lane] + WS_CNT) + (pan * NRANK + rank) * 32 + q, (c + 1u) * CNT_FULL);
            else st_sys((unsigned*)(P.ws[lane] + WS_FLAG + rank * FLAGB) + o, c + 1u);
        }
    }
}

struct Tile { int tm, v, half; unsigned sA, stA; };
__device__ __forceinline__ void tile_of(int idx, int col, int rank, Tile& t) {
    const int v = 2 * (idx >> 2) + (col >> 2);
    t.tm = 4 * (col & 3) + (idx & 3); t.v = v; t.half = -1;
    const int g = (v - GM) / GW, i = (v - GM) % GW;
    t.sA = WS_SND + (unsigned)(v < GW ? gpan_of(v, rank) : v < GW + GM ? 4 * U0 + v - GW : 4 * U0 + 4 * GM + (g - 1) * 4 * GW + i) * UNB;
    t.stA = (unsigned)(v < GW ? U0 : v < GW + GM ? GM : g <= NGF ? GW : GL) * UNB;
}
__device__ __forceinline__ void item_of(int q, int rank, Tile& t) {
    const int n = q < PFULL ? q : PFULL + ((q - PFULL) >> 1);
    if (n < 4 * PL0) tile_of(RS * KX + n % PL0, n / PL0, rank, t);
    else tile_of(RS * KX + (n - 4 * PL0) % PL1, 4 + (n - 4 * PL0) / PL1, rank, t);
    if (q >= PFULL) t.half = (q - PFULL) & 1;
}

template <int wr>
__device__ __forceinline__ void gemm_role(const Params& P, unsigned call, int col, int slot, LAS unsigned char* lds) {
    const int tid = threadIdx.x, wid = __builtin_amdgcn_readfirstlane(tid >> 6), lane = tid & 63, wc = wid & 3;
    const int rank = P.rank;
    char* wsl = P.ws[rank];
    const rsrc_t rws = mk_rsrc(wsl, WS_END);
    LAS unsigned* ctl = (LAS unsigned*)(lds + STAGE_BYTES);
    const unsigned voff = (unsigned)tid * 16u;
    const unsigned ldsw = (unsigned)wid * 1024u;
    const int aoff = wr * 8192 + lane * 16, boff = wc * 4096 + lane * 16;

    auto lane_now = [&]() { int tz = threadIdx.x; asm volatile("" : "+v"(tz)); return tz & 63; };
    auto cnt_of = [&](const Tile& t) { const int ln = lane_now(); return WS_CNT + (unsigned)((((ln & 32) ? NTM + t.v : t.tm) * NRANK + ((ln >> 2) & 7)) * 32 + (ln & 3)) * 4u; };
    auto level_of = [&](unsigned c) -> int {
        const unsigned long long miss = __ballot((int)(c - call * CNT_FULL) < 0);
        unsigned m = (unsigned)miss | (unsigned)(miss >> 32);
        const bool own = ((m >> (rank * 4)) & 15u) == 0u;
        m |= m >> 16; m |= m >> 8; m |= m >> 4;
        return own ? 1 + __builtin_ctz(m | 16u) : 0;
    };
    auto ready = [&](const Tile& t, int want) -> int {
        if (wr == 0 && wid == 0) {
            const unsigned* c = (const unsigned*)(wsl + cnt_of(t));
            unsigned* tmo = (unsigned*)(wsl + WS_TMO);
            const unsigned long long t0 = __builtin_amdgcn_s_memrealtime();
            int got;
            for (;;) {
                got = level_of(ld_sys(c));
                if (got >= want) break;
                if (give_up(tmo, call, t0)) { ctl[0] = 1u; got = 5; break; }
                __builtin_amdgcn_s_sleep(4);
            }
            ctl[2] = (unsigned)got;
            __builtin_amdgcn_fence(__ATOMIC_ACQUIRE, "agent");
            asm volatile("s_waitcnt vmcnt(0) lgkmcnt(0)" ::: "memory");
        }
        asm volatile("" ::: "memory"); __builtin_amdgcn_s_barrier(); asm volatile("" ::: "memory");
        const int r = __builtin_amdgcn_readfirstlane(ctl[2]);
        asm volatile("s_waitcnt lgkmcnt(0)" ::: "memory");
        return r;
    };

#define SA(b, h) (((b) * 2 + (h)) * HTB)
#define SB(b, h) ((4 + (b) * 2 + (h)) * HTB)
#define STAGE(bufoff, soff) do { _Pragma("unroll") for (int _i = 0; _i < 2; ++_i) \
        __builtin_amdgcn_raw_ptr_buffer_load_lds(rws, (LAS unsigned*)(lds + (bufoff) + ldsw + _i * 8192), 16, voff, (soff) + _i * 8192u, 0, 0); } while (0)
#define LDA(dst, b, h) do { _Pragma("unroll") for (int m = 0; m < 4; ++m) _Pragma("unroll") for (int k = 0; k < 2; ++k) dst[m][k] = *(const LAS bf16x8*)(lds + SA(b, h) + aoff + m * 2048 + k * 1024); } while (0)
#define LDB(dst, b, h) do { _Pragma("unroll") for (int n = 0; n < 2; ++n) _Pragma("unroll") for (int k = 0; k < 2; ++k) dst[n][k] = *(const LAS bf16x8*)(lds + SB(b, h) + boff + n * 2048 + k * 1024); } while (0)
#define MMA(ai, first, prio) do { __builtin_amdgcn_s_setprio(prio); \
        _Pragma("unroll") for (int i = 0; i < 32; ++i) { \
            const int sn = ((i >> 2) & 1) ? 3 - (i & 3) : i & 3; \
            const int bj = CFG_MO == 7 ? ((i >> 2) ^ (i >> 3)) & 1 : CFG_MO == 0 || CFG_MO == 4 ? i >> 4 : CFG_MO == 1 ? (i >> 1) & 1 : CFG_MO == 5 ? sn >> 1 : (i >> 3) & 1; \
            const int m = CFG_MO == 7 ? i >> 3 : CFG_MO == 2 ? i & 3 : CFG_MO == 6 ? sn : (i >> 2) & 3; \
            const int n = CFG_MO == 7 ? ((i >> 1) ^ (i >> 3)) & 1 : CFG_MO == 0 || CFG_MO == 4 ? (i >> 1) & 1 : CFG_MO == 1 ? i & 1 : CFG_MO == 5 ? sn & 1 : (i >> 2) & 1; \
            const int k = CFG_MO == 7 ? (i ^ (i >> 1)) & 1 : CFG_MO == 0 || CFG_MO == 4 ? i & 1 : i >> 4; \
            if (CFG_MO >= 4) SCHED; \
            acc[ai][bj][m][n] = __builtin_amdgcn_mfma_f32_16x16x32_bf16(bj ? B1[n][k] : B0[n][k], At[m][k], first && (CFG_MO == 7 ? (i & 1) == 0 : k == 0) ? (f32x4){0.f, 0.f, 0.f, 0.f} : acc[ai][bj][m][n], 0, 0, 0); } \
        __builtin_amdgcn_s_setprio(0); } while (0)
#define WAIT_L0() do { asm volatile("" ::: "memory"); __builtin_amdgcn_s_waitcnt(0xC07F); asm volatile("" ::: "memory"); } while (0)
#define WAIT_V(n) asm volatile("s_waitcnt vmcnt(" #n ")" ::: "memory")
#define WAIT_L(n) asm volatile("s_waitcnt lgkmcnt(" #n ")" ::: "memory")
#define BAR __builtin_amdgcn_s_barrier()
#define SCHED __builtin_amdgcn_sched_barrier(0)

    unsigned long long outp = (unsigned long long)P.out[rank];
    asm volatile("" : "+v"(outp));
    Tile cur, nxt; int rnd = 0;
    tile_of(slot, col, rank, cur);
    bf16x8 At[4][2], B0[2][2], B1[2][2];
    constexpr unsigned SB0 = WS_SND + ((NRANK - 1) * NS + GW) * UNB, STB = U0 * UNB;
    const unsigned kov = WS_PAN + (unsigned)((rank + 1 + (lane + 3) % 7) & (NRANK - 1)) * CHB + (unsigned)(((lane & 31) - 4) / 7) * UNB;
    unsigned tabA, tabB;
    auto tables = [&](const Tile& t) {
        const int ln = lane_now();
        const unsigned la = t.sA + (unsigned)ln * t.stA, ra = (unsigned)(NTM + t.v) * PANB + kov;
        const unsigned lb = SB0 + (unsigned)t.tm * UNB + (unsigned)ln * STB, rb = (unsigned)t.tm * PANB + kov;
        tabA = ln < 4 ? la : ra;
        tabB = (ln < 4 ? lb : rb) + (t.half == 1 ? HTB : 0u);
    };
    auto pick = [&]() {
        Tile t = cur; bool ok = true;
        if (rnd + 1 < RS) tile_of((rnd + 1) * KX + slot, col, rank, t);
        else {
            const unsigned d = (unsigned)__builtin_amdgcn_raw_ptr_buffer_atomic_add_i32(1, rws, lane_now() == 0 ? 0u : 0xfffffff0u, WS_HEAD + (unsigned)rank * 128u, 0);
            const unsigned q = (unsigned)__builtin_amdgcn_readfirstlane(d) - (call - 1u) * (unsigned)(NITEM + 8 * KX);
            ok = q < (unsigned)NITEM;
            if (ok) item_of((int)q, rank, t);
        }
        *(LAS u32x4*)(ctl + 4) = (u32x4){ok ? (unsigned)(t.tm | t.v << 8 | (t.half + 2) << 16) : 0u, t.sA, t.stA, 0u};
        __builtin_amdgcn_raw_ptr_buffer_load_lds(rws, ctl + 64, 4, cnt_of(t), 0, 0, AUX_SYS);
    };
    auto picked = [&](int& level) -> bool {
        const u32x4 f = *(const LAS u32x4*)(ctl + 4);
        level = level_of(ctl[64 + lane_now()]);
        const unsigned w = __builtin_amdgcn_readfirstlane(f[0]);
        nxt.tm = w & 255u; nxt.v = (w >> 8) & 255u; nxt.half = (int)(w >> 16) - 2;
        nxt.sA = __builtin_amdgcn_readfirstlane(f[1]); nxt.stA = __builtin_amdgcn_readfirstlane(f[2]);
        return w != 0u;
    };
    constexpr int JP = KT / 2 - 4, JI = KT / 2 - 2, JL = KT / 2 - 1;
    const int jend = wr == 0 && wid == 0 ? JP : JL;
    int lvl = ready(cur, 1);
    STAMP(P, 2);
    tables(cur);
    unsigned a2 = cur.sA, b2 = SB0 + (unsigned)cur.tm * UNB;
    STAGE(SB(0, 0), b2); STAGE(SA(0, 0), a2); STAGE(SB(0, 1), b2 + HTB); STAGE(SA(0, 1), a2 + HTB);
    STAGE(SB(1, 0), b2 + KTB); STAGE(SA(1, 0), a2 + KTB); STAGE(SB(1, 1), b2 + HTB + KTB);
    WAIT_V(0); BAR;
#define TICK(i, r) do {} while (0)
#define ROUND_DONE() do {} while (0)
    bool has_next = false;
#pragma unroll 1
    for (;;) {
        f32x4 acc[2][2][4][2];
        int jchk = lvl < 5 ? 7 * lvl - 4 : jend;
        TICK(0, 2); TICK(4, 3);
#define SYNC_MMA(ai, first, wv) do { if (wv && (wr == 1 || !CFG_LW)) WAIT_V(8); WAIT_L0(); if (wr == 1) BAR; MMA(ai, first, wr + 1); if (wv && wr == 0 && CFG_LW) WAIT_V(8); if (wr == 0) BAR; SCHED; } while (0)
#define PASS(first, check) do { \
            LDB(B0, 0, 0); LDA(At, 0, 0); LDB(B1, 0, 1); STAGE(SA(1, 1), a2 + KTB + HTB); SCHED; \
            check; \
            a2 = __builtin_amdgcn_readlane(tabA, (j + 1) & 31); b2 = __builtin_amdgcn_readlane(tabB, (j + 1) & 31); \
            SYNC_MMA(0, first, !first); \
            LDA(At, 0, 1); STAGE(SB(0, 0), b2); STAGE(SA(0, 0), a2); STAGE(SB(0, 1), b2 + HTB); \
            SYNC_MMA(1, first, !first); \
            LDB(B0, 1, 0); LDA(At, 1, 0); LDB(B1, 1, 1); STAGE(SA(0, 1), a2 + HTB); \
            SYNC_MMA(0, false, true); \
            LDA(At, 1, 1); STAGE(SB(1, 0), b2 + KTB); STAGE(SA(1, 0), a2 + KTB); STAGE(SB(1, 1), b2 + KTB + HTB); \
            SYNC_MMA(1, false, true); } while (0)
        int j = 0;
        PASS(true, );
#pragma unroll 1
        for (j = 1; j < KT / 2; ++j)
            PASS(false, if (__builtin_expect(j == jchk, 0)) {
                    if (j < JP) { lvl = ready(cur, lvl + 1); jchk = lvl < 5 ? 7 * lvl - 4 : jend; }
                    else if (wr == 0 && j == JP) { pick(); jchk = JI; }
                    else if (wr == 0 && j == JI) { asm volatile("buffer_inv sc1" ::: "memory"); jchk = JL; }
                    else {
                        has_next = picked(lvl);
                        if (has_next) {
                            const int want = nxt.half < 0 ? 1 : 5;
                            if (lvl < want) lvl = ready(nxt, want);
                            tables(nxt);
                        }
                    }
                });
        if (has_next && nxt.half >= 0) STAGE(SA(1, 1), a2 + KTB + HTB);
        TICK(1, 2);
        TICK(2, 2);
        {
            const int ln = lane_now(), fr = ln & 15, fq = ln >> 4;
            const rsrc_t rol = mk_rsrc((const void*)outp, OUTB);
            const bool shared = cur.v < NS;
            if (__builtin_amdgcn_readfirstlane(ctl[0]) != 0u) {
#pragma unroll
                for (int i = 0; i < 32; ++i) acc[i >> 4][(i >> 3) & 1][(i >> 1) & 3][i & 1] = __builtin_bit_cast(f32x4, (u32x4){0x7fc00000u, 0x7fc00000u, 0x7fc00000u, 0x7fc00000u});
            }
            if (rnd == 0 && cur.v < GW) {
                const unsigned long long t0 = __builtin_amdgcn_s_memrealtime();
                while (__builtin_amdgcn_s_memrealtime() - t0 < (unsigned long long)((col & 3) * CFG_STAG + (col >> 2) * CFG_STAGB)) __builtin_amdgcn_s_sleep(8);
            }
            const bool thru = rnd >= RS;
            const unsigned vo = (unsigned)(wc * 32 + fq * 4) * (NN * 2) + (unsigned)(wr * 256 + fr * 16);
            const unsigned so = (unsigned)(cur.tm * 256) * (NN * 2) + (unsigned)gpan_of(cur.v, rank) * 512u;
            const unsigned vs = (unsigned)(wc * 32 + fq * 4) * ROWB + (unsigned)(wr * 256 + fr * 16);
            const unsigned ss = WS_OSN + (unsigned)(cur.tm * 256) * ROWB + (unsigned)cur.v * 512u;
            auto finish = [&](auto mode) {
#pragma unroll
                for (int bj = 0; bj < 2; ++bj)
#pragma unroll
                    for (int n = 0; n < 2; ++n)
#pragma unroll
                        for (int r = 0; r < 4; ++r) {
                            u32x4 o;
                            if ((r & 1) == 0) {
#pragma unroll
                                for (int i = 0; i < 8; ++i) { f32x4& a = acc[i >> 2][bj][i & 3][n]; const f32x2 z = silu2((f32x2){a[r], a[r + 1]}); a[r] = z[0]; a[r + 1] = z[1]; }
                            }
#pragma unroll
                            for (int i = 0; i < 4; ++i) o[i] = cvt_pk_bf16(acc[i >> 1][bj][(i & 1) * 2][n][r], acc[i >> 1][bj][(i & 1) * 2 + 1][n][r]);
                            if (mode == 1) STORE16(o, rws, vs, ss + (unsigned)(bj * 128 + n * 16 + r) * ROWB, AUX_SYS);
                            STORE16(o, rol, vo, so + (unsigned)(bj * 128 + n * 16 + r) * (NN * 2), mode == 2 ? AUX_SYS : CFG_RST);
                            if (mode == 2 && CFG_LAST > 1) { WAIT_VM0(); __builtin_amdgcn_s_sleep(CFG_LAST > 1 ? CFG_LAST - 1 : 1); }
                        }
            };
            if (shared) finish(std::integral_constant<int, 1>());
            else if (thru) finish(std::integral_constant<int, 2>());
            else finish(std::integral_constant<int, 0>());
            WAIT_V(16);
            if (shared) {
                WAIT_VM0();
                unsigned old = 0;
                if (ln == 0) old = __hip_atomic_fetch_add(ctl + 1, 1u, __ATOMIC_RELAXED, __HIP_MEMORY_SCOPE_WORKGROUP);
                old = __builtin_amdgcn_readfirstlane(old);
                if ((old & 7u) == 7u && ln < NRANK && ln != rank)
                    st_sys((unsigned*)(P.ws[ln] + WS_FLAG + rank * FLAGB) + NIN * 4 + turn_of(cur.tm, ln) * NS + cur.v, call);
            }
        }
        TICK(3, 2);
        ROUND_DONE();
        if (!has_next) break;
        cur = nxt; ++rnd;
        if (cur.half >= 0) break;
    }
#define HB(b, i) ((b) < 2 ? ((i) < 2 ? SA(b, i) : SB(b, 0)) : (i) < 2 ? SB(i, 1) : 8 * HTB)
#define HSTAGE(b, t) do { const unsigned _o = ((t) & 1) * KTB, _a = __builtin_amdgcn_readlane(tabA, ((t) >> 1) & 31) + _o, _b = __builtin_amdgcn_readlane(tabB, ((t) >> 1) & 31) + _o; \
        STAGE(HB(b, 0), _a); STAGE(HB(b, 1), _a + HTB); STAGE(HB(b, 2), _b); } while (0)
#define HLOAD(b) do { _Pragma("unroll") for (int i = 0; i < 4; ++i) B0[i >> 1][i & 1] = *(const LAS bf16x8*)(lds + HB(b, 2) + boff + i * 1024); \
        _Pragma("unroll") for (int i = 0; i < 8; ++i) At[i >> 1][i & 1] = *(const LAS bf16x8*)(lds + HB(b, 0) + aoff + i * 1024); \
        _Pragma("unroll") for (int i = 0; i < 8; ++i) Au[i >> 1][i & 1] = *(const LAS bf16x8*)(lds + HB(b, 1) + aoff + i * 1024); } while (0)
#define HPASS(b, t, first) do { \
        if (wr == 0) { HLOAD(b); HSTAGE((b + 2) % 3, (t) + 2); } else { HSTAGE((b + 2) % 3, (t) + 2); SCHED; HLOAD(b); if (!first) WAIT_V(6); } \
        WAIT_L0(); if (wr == 1) BAR; \
        __builtin_amdgcn_s_setprio(wr + 1); \
        _Pragma("unroll") for (int i = 0; i < 32; ++i) { const int ai = i >> 4, m = (i >> 2) & 3, n = (i >> 1) & 1, k = i & 1; \
            acc[ai][m][n] = __builtin_amdgcn_mfma_f32_16x16x32_bf16(B0[n][k], ai ? Au[m][k] : At[m][k], first && k == 0 ? (f32x4){0.f, 0.f, 0.f, 0.f} : acc[ai][m][n], 0, 0, 0); } \
        __builtin_amdgcn_s_setprio(0); \
        if (wr == 0) { if (!first) WAIT_V(6); BAR; } SCHED; } while (0)
#pragma unroll 1
    while (PFULL < PN && has_next) {
        f32x4 acc[2][4][2];
        bf16x8 Au[4][2];
        BAR;
        HPASS(0, 0, true);
#pragma unroll 1
        for (int t = 1; t < KT; t += 3) {
            if (wr == 0 && wid == 0 && t == KT - 9) pick();
            HPASS(1, t, false); HPASS(2, t + 1, false); HPASS(0, t + 2, false);
        }
        has_next = picked(lvl);
        if (has_next) {
            (void)ready(nxt, 5);
            tables(nxt);
            HSTAGE(0, 0); HSTAGE(1, 1);
        }
        const int ln = lane_now(), fr = ln & 15, fq = ln >> 4;
        const rsrc_t rol = mk_rsrc((const void*)outp, OUTB);
        if (__builtin_amdgcn_readfirstlane(ctl[0]) != 0u) {
#pragma unroll
            for (int i = 0; i < 16; ++i) acc[i >> 3][(i >> 1) & 3][i & 1] = __builtin_bit_cast(f32x4, (u32x4){0x7fc00000u, 0x7fc00000u, 0x7fc00000u, 0x7fc00000u});
        }
        const unsigned vo = (unsigned)(wc * 32 + fq * 4) * (NN * 2) + (unsigned)(wr * 256 + fr * 16);
        const unsigned so = (unsigned)(cur.tm * 256 + cur.half * 128) * (NN * 2) + (unsigned)gpan_of(cur.v, rank) * 512u;
#pragma unroll
        for (int n = 0; n < 2; ++n)
#pragma unroll
            for (int r = 0; r < 4; ++r) {
                u32x4 o;
                if ((r & 1) == 0) {
#pragma unroll
                    for (int i = 0; i < 8; ++i) { f32x4& a = acc[i >> 2][i & 3][n]; const f32x2 z = silu2((f32x2){a[r], a[r + 1]}); a[r] = z[0]; a[r + 1] = z[1]; }
                }
#pragma unroll
                for (int i = 0; i < 4; ++i) o[i] = cvt_pk_bf16(acc[i >> 1][(i & 1) * 2][n][r], acc[i >> 1][(i & 1) * 2 + 1][n][r]);
                STORE16(o, rol, vo, so + (unsigned)(n * 16 + r) * (NN * 2), AUX_SYS);
            }
        WAIT_V(8);
        ROUND_DONE();
        cur = nxt; ++rnd;
    }
    WAIT_V(0);
}

__global__ __launch_bounds__(512, 2) void fused_gemm_ar(Params P) {
    extern __shared__ __attribute__((aligned(16))) unsigned char shm[];
    LAS unsigned char* lds = (LAS unsigned char*)shm;
    const int wid = __builtin_amdgcn_readfirstlane(threadIdx.x >> 6), lane = threadIdx.x & 63;
    const int b = __builtin_amdgcn_readfirstlane(blockIdx.x);
    unsigned* tkp = (unsigned*)(P.ws[P.rank] + WS_TICKET + b * 128 + wid * 16);
    const unsigned tk = ld_agent(tkp);
    if (b < NPULL) {
        const unsigned call = __builtin_amdgcn_readfirstlane(tk) + 1u;
        st_agent(tkp, call);
        const int pi = b / PW, slot = b % PW;
        const int peer = (P.rank + 1 + pi) & (NRANK - 1);
        pull_role(P, call, peer, slot * 8 + wid, lane);
    } else {
        const int g = b - NPULL;
        ((LAS unsigned*)(lds + STAGE_BYTES))[lane & 3] = 0u;
        STAMP(P, 0);
        pack_role(P, tk, g, wid, lane);
        const unsigned call = __builtin_amdgcn_readfirstlane(tk) + 1u;
        st_agent(tkp, call);
        STAMP(P, 1);
        asm volatile("s_waitcnt lgkmcnt(0)" ::: "memory");
        __builtin_amdgcn_s_barrier();
        if (b >= NPS) {
            const int col = CFG_SWAP == 1 ? (b & 7) ^ (b >> 1 & 1) : CFG_SWAP == 2 ? (b & 4) | ((0x1320 >> (b & 3) * 4) & 3) : b & 7;
            if (wid < 4) gemm_role<0>(P, call, col, (b - NPS) >> 3, lds);
            else gemm_role<1>(P, call, col, (b - NPS) >> 3, lds);
        }
    }
}
}

extern "C" void kernel_launch(int rank, int nranks, ncclComm_t comm,
                              hipStream_t stream,
                              void* const* d_in, const int* in_sizes, int n_in,
                              void* d_out, int out_size,
                              const int* in_dtypes, int out_dtype,
                              void* const* d_out_all,
                              void* const* d_ws_all, size_t ws_bytes) {
    if (nranks != NRANK || (unsigned)rank >= (unsigned)NRANK || ws_bytes < (size_t)WS_END || n_in < 2 ||
        in_sizes[0] != MM * KSH || in_sizes[1] != KSH * NN || out_size != MM * NN) {
        fprintf(stderr, "unexpected problem geometry\n");
        return;
    }
    static bool attr_done[NRANK] = {};
    if (!attr_done[rank]) {
        HIP_CHECK(hipFuncSetAttribute((const void*)fused_gemm_ar, hipFuncAttributeMaxDynamicSharedMemorySize, LDS_BYTES));
        attr_done[rank] = true;
    }
    Params P;
    P.x = d_in[0]; P.w = d_in[1]; P.rank = rank;
    for (int p = 0; p < NRANK; ++p) { P.out[p] = (char*)d_out_all[p]; P.ws[p] = (char*)d_ws_all[p]; }
    hipLaunchKernelGGL(fused_gemm_ar, dim3(GRID), dim3(512), LDS_BYTES, stream, P);
    HIP_CHECK(hipGetLastError());
}
```

```cpp
#include <hip/hip_runtime.h>
#include <rccl/rccl.h>
#include <cstdio>
#include <type_traits>

#define HIP_CHECK(expr) do { hipError_t _e = (expr); if (_e != hipSuccess) { \
    fprintf(stderr, "HIP error %s at %s:%d\n", hipGetErrorString(_e), __FILE__, __LINE__); \
    return; } } while (0)

#define LAS __attribute__((address_space(3)))
typedef short bf16x8 __attribute__((ext_vector_type(8)));
typedef float f32x4 __attribute__((ext_vector_type(4)));
typedef unsigned u32x4 __attribute__((ext_vector_type(4)));
typedef __amdgpu_buffer_rsrc_t rsrc_t;

namespace {
constexpr int NRANK = 8;
constexpr int MM = 4096, KK = 4096, NN = 32768, KSH = KK / NRANK;
constexpr int KT = KK / 64, KTS = KSH / 64, NTM = MM / 256, NPAN = NN / 256;
#define CFG_NS 6
#define CFG_PW 8
constexpr int NS = CFG_NS;
constexpr int NRP = NPAN - NRANK * NS;
constexpr int NWP = NS + NRP;
constexpr int NIN = NTM + NWP;
constexpr int NOUT = NS * NTM;
constexpr int NFLAG = NIN * 4 + NOUT;
constexpr int NUNIT = NTM * 4 + NPAN * 4;
constexpr int GRID = 256;
constexpr int PW = CFG_PW;
constexpr int NPULL = PW * (NRANK - 1);
constexpr int NW = PW * 8;
constexpr int NPK = GRID - NPULL;
constexpr int NPS = (NPULL + 7) / 8 * 8;
constexpr int KX = (GRID - NPS) / 8;
#define CFG_GW (2 * (KX / 4))
constexpr int GW = CFG_GW;
constexpr int G0 = GW + NTM;
#define CFG_GM 2
constexpr int GM = CFG_GM;
constexpr int NGF = (NWP - GW - GM) / GW, GL = (NWP - GW - GM) % GW;
static_assert(NS <= GW && NWP > GW, "shared panels sit in the first group");
#define CFG_PU 4
constexpr int PU = CFG_PU, PSH = PU == 1 ? 0 : PU == 2 ? 1 : PU == 4 ? 2 : 3;
#define CFG_PUSH 3
constexpr int NPUSH = CFG_PUSH;
constexpr int NPIN = (NIN * 4 - NPUSH) * 64 / PU;
static_assert(NPUSH <= NS, "pushed units have a single consumer");
constexpr int NPC = NPIN + NOUT * 128 / PU;
constexpr unsigned CNT_FULL = 64;

constexpr int NT0 = 4 * ((NWP + 1) / 2), NT1 = 4 * (NWP / 2);
constexpr int RS = (NT1 - 1) / KX;
constexpr int PL0 = NT0 - RS * KX, PL1 = NT1 - RS * KX, PN = 4 * (PL0 + PL1);
#define CFG_MO 7
#define CFG_SWAP 1
#define CFG_LW 0
#define CFG_FD (-8)
constexpr int PFULL = 2 * PN - 8 * KX - CFG_FD < 0 ? 0 : 2 * PN - 8 * KX - CFG_FD > PN ? PN : 2 * PN - 8 * KX - CFG_FD;
constexpr int NITEM = PFULL + 2 * (PN - PFULL);
constexpr unsigned KTB = 32768, HTB = 16384;
constexpr unsigned PANB = KT * KTB;
constexpr unsigned CHB = KTS * KTB;
constexpr unsigned UNB = 2 * KTB;
constexpr unsigned WS_TICKET = 0;
constexpr unsigned WS_TMO = WS_TICKET + GRID * 128;
constexpr unsigned WS_FLAG = WS_TMO + 128;
constexpr unsigned FLAGB = 4096;
constexpr unsigned WS_DONE = WS_FLAG + NRANK * FLAGB;
constexpr unsigned WS_HEAD = WS_DONE + NRANK * 512;
constexpr unsigned WS_PH = WS_HEAD + NRANK * 128;
constexpr unsigned WS_ZC = WS_PH + 2 * NRANK * 128 + 128;
constexpr unsigned WS_CNT = WS_ZC + NRANK * NRANK * 128;
constexpr unsigned WS_CTRL_END = WS_CNT + NIN * NRANK * 128;
constexpr unsigned WS_DBG = 512u << 10;
#define CFG_HS 0
constexpr unsigned WS_SND = 1u << 20, SNDB = CFG_HS ? 0 : NUNIT * UNB;
constexpr unsigned ROWB = NS * 512;
constexpr unsigned WS_OSN = WS_SND + NUNIT * UNB + SNDB;
constexpr unsigned WS_PAN = (WS_OSN + MM * ROWB + PANB - 1) / PANB * PANB;
constexpr unsigned WS_END = WS_PAN + NIN * PANB;
constexpr int U0 = G0 + (NRANK - 1) * NS;
constexpr unsigned OUTB = (unsigned)MM * NN * 2;
static_assert(WS_CTRL_END <= WS_DBG, "ctrl region");
static_assert(NFLAG <= 512 && 2048 <= (int)FLAGB, "flag region");
static_assert(NW <= 128, "done words");
static_assert(PU == 4, "block size");
#define CFG_DEEP 3
constexpr unsigned STAGE_BYTES = 9 * HTB;
constexpr unsigned LDS_BYTES = STAGE_BYTES + 512;
constexpr unsigned long long DEADLINE = 50000000ull;
constexpr int AUX_SYS = 17, AUX_SYS_NT = 19, AUX_AGENT = 16;
#define CFG_PACE 140
#define CFG_PACE0 40
#define CFG_STAG 600
#define CFG_P3 320
#define CFG_P3N 2
#define CFG_STAGB 300
#define CFG_LOOK 256
#define CFG_TAIL NW
#define CFG_ZB 120
#define CFG_LOOKP 100
#define CFG_PLAN (CFG_ZB + CFG_LOOKP + 50)
#define CFG_BAL 8
constexpr int ZB = CFG_ZB;
static_assert(ZB <= 127 && ZB <= NS * 128 / PU && NFLAG <= 504, "plan");
#define CFG_PWIN 4u
#define CFG_PS 2
#define CFG_LAST 1
#define CFG_RST 0
#define CFG_RATE 1710
#define CFG_CRED 25
#define CFG_ALLD 0
#define CFG_EARLY 1
constexpr unsigned PACE16 = CFG_RATE * PW * (CFG_ALLD ? CFG_ALLD : 8 + CFG_DEEP) / 64;

struct Params {
    const void* x; const void* w;
    char* wsl; char* outl;
    int rank, pad[7];
    char* ws[NRANK]; char* out[NRANK];
};

__device__ __forceinline__ rsrc_t mk_rsrc(const void* p, unsigned bytes) {
    const unsigned long long v = (unsigned long long)p;
    const unsigned lo = __builtin_amdgcn_readfirstlane((unsigned)v), hi = __builtin_amdgcn_readfirstlane((unsigned)(v >> 32));
    return __builtin_amdgcn_make_buffer_rsrc((void*)(((unsigned long long)hi << 32) | lo), 0, bytes, 0x00020000);
}
__device__ __forceinline__ unsigned ld_sys(const unsigned* p) { return __hip_atomic_load(p, __ATOMIC_RELAXED, __HIP_MEMORY_SCOPE_SYSTEM); }
__device__ __forceinline__ unsigned ld_agent(const unsigned* p) { return __hip_atomic_load(p, __ATOMIC_RELAXED, __HIP_MEMORY_SCOPE_AGENT); }
__device__ __forceinline__ void st_sys(unsigned* p, unsigned v) { __hip_atomic_store(p, v, __ATOMIC_RELAXED, __HIP_MEMORY_SCOPE_SYSTEM); }
__device__ __forceinline__ void st_agent(unsigned* p, unsigned v) { __hip_atomic_store(p, v, __ATOMIC_RELAXED, __HIP_MEMORY_SCOPE_AGENT); }
__device__ __forceinline__ void add_agent(unsigned* p, unsigned v) { (void)__hip_atomic_fetch_add(p, v, __ATOMIC_RELAXED, __HIP_MEMORY_SCOPE_AGENT); }
#define STORE16(v, rs, vo, so, aux) do { const u32x4 _d = (v); __builtin_amdgcn_raw_buffer_store_b128(_d, rs, vo, so, aux); asm volatile("s_nop 1" :: "v"(_d) : "memory"); } while (0)
#define WAIT_VM0() asm volatile("s_waitcnt vmcnt(0)" ::: "memory")
#define WAIT_VMN(n) do { asm volatile("" ::: "memory"); __builtin_amdgcn_s_waitcnt(0x0F70 | (n)); asm volatile("" ::: "memory"); } while (0)

__device__ __forceinline__ bool give_up(unsigned* tmo, unsigned call, unsigned long long t0) {
    const unsigned long long dt = __builtin_amdgcn_s_memrealtime() - t0;
    if (dt < 2000ull) return false;
    if ((unsigned)__builtin_amdgcn_readfirstlane(ld_agent(tmo)) == call) return true;
    if (dt > DEADLINE) { st_agent(tmo, call); return true; }
    return false;
}

__device__ __forceinline__ int gpan_of(int v, int consumer) { return v < NS ? consumer * NS + v : (NRANK - 1) * NS + v; }
__device__ __forceinline__ int band_of(int e, int consumer) { return e < 7 ? (consumer + 1 + e) & 7 : e < 15 ? 8 + ((consumer + e) & 7) : consumer; }
__device__ __forceinline__ int turn_of(int band, int consumer) { return band >= 8 ? 7 + ((band - consumer + 1) & 7) : band == consumer ? 15 : (band - consumer - 1) & 7; }
__device__ __forceinline__ int unit_of(int o, int consumer, int& q, int& pos) {
    if (o < 4 * G0) { q = o / G0; const int m = o % G0; pos = q * U0 + gpan_of(m, consumer); return m < GW ? NTM + m : m - GW; }
    pos = o + 4 * (NRANK - 1) * NS;
    o -= 4 * G0;
    if (o < 4 * GM) { q = o / (GM ? GM : 1); return NTM + GW + o % (GM ? GM : 1); }
    o -= 4 * GM;
    const int g = o / (4 * GW), r = o % (4 * GW);
    int i;
    if (GL == 0 || g < NGF) { q = r / GW; i = r % GW; } else { q = r / (GL ? GL : 1); i = r % (GL ? GL : 1); }
    return NTM + GM + GW * (g + 1) + i;
}

__device__ __forceinline__ unsigned cvt_pk_bf16(float lo, float hi) { unsigned r; asm("v_cvt_pk_bf16_f32 %0, %1, %2" : "=v"(r) : "v"(lo), "v"(hi)); return r; }
__device__ __forceinline__ float silu(float y) {
    const float c = __builtin_amdgcn_fmed3f(y, -60.0f, 60.0f);
    const float e = __builtin_amdgcn_exp2f(c * -1.44269504088896f);
    return y * __builtin_amdgcn_rcpf(1.0f + e);
}

typedef float f32x2 __attribute__((ext_vector_type(2)));
__device__ __forceinline__ f32x2 silu2(f32x2 y) {
    const f32x2 t = __builtin_elementwise_fma(y, (f32x2){-1.44269504088896f, -1.44269504088896f}, (f32x2){-86.5617024533378f, -86.5617024533378f});
    const f32x2 e = {__builtin_amdgcn_fmed3f(__builtin_amdgcn_exp2f(t[0]), 0.0f, 1.0f), __builtin_amdgcn_fmed3f(__builtin_amdgcn_exp2f(t[1]), 0.0f, 1.0f)};
    const f32x2 d = __builtin_elementwise_fma(e, (f32x2){1.1420073898156842e26f, 1.1420073898156842e26f}, (f32x2){1.0f, 1.0f});
    return y * (f32x2){__builtin_amdgcn_rcpf(d[0]), __builtin_amdgcn_rcpf(d[1])};
}

#define STAMP1(P, slot) do {} while (0)
#define STAMP(P, slot) do {} while (0)

__device__ __forceinline__ void pull_role(const Params& P, unsigned call, int peer, int widx, int lane) {
    const int rank = P.rank;
    char* wsl = P.wsl;
    const rsrc_t rwr = mk_rsrc(P.ws[peer], WS_END), rwl = mk_rsrc(wsl, WS_END);
    const rsrc_t rol = mk_rsrc(P.outl, OUTB), ror = mk_rsrc(P.out[peer], OUTB);
    unsigned* tmo = (unsigned*)(wsl + WS_TMO);
    constexpr unsigned OOB = 0xfffffff0u;
    const unsigned lane0 = lane == 0 ? 0u : OOB;
    const unsigned hoff = WS_PH + (call & 1u) * (NRANK * 128u);
    auto draw = [&]() { return (unsigned)__builtin_amdgcn_raw_ptr_buffer_atomic_add_i32(1, rwl, lane0, hoff + (unsigned)peer * 128u, 0); };
    if (widx == 0) __builtin_amdgcn_raw_buffer_store_b32(0u, rwl, lane0, WS_PH + (~call & 1u) * (NRANK * 128u) + (unsigned)peer * 128u, AUX_SYS);
    auto arrive = [&](unsigned vo, unsigned cnt) { (void)__builtin_amdgcn_raw_ptr_buffer_atomic_add_i32(PU, rwl, vo, cnt, 0); };
    if (CFG_ALLD && (widx & 7) * 2 + (widx >> 3 & 1) >= CFG_ALLD) return;
    const bool deep0 = CFG_ALLD || (widx & 7) < CFG_DEEP;
    bool bad = false; int stamped = 0; (void)stamped;
    STAMP(P, 0);
    auto index_of = [&](unsigned v) { return (int)__builtin_amdgcn_readfirstlane(v); };
    int cur = index_of(draw());
    constexpr int BPB = NS * 128 / PU;
    const unsigned fso = WS_FLAG + (unsigned)peer * FLAGB;
    auto first_of = [&](int i) { return i < NPIN ? (i >> (6 - PSH)) + NPUSH : NIN * 4 + (i - NPIN) / BPB * NS; };
    int lim = 0, flim = 0, fin = NPC;
    bool planned = ZB == 0;
    unsigned long long plan = 0;
    auto moved = [&](int j) { return (int)(signed char)(plan >> (j * 8)); };
    auto update = [&](const u32x4& f0, const u32x4& f1, int i) {
        const int first = first_of(i);
        unsigned un = 0;
#pragma unroll
        for (int j = 0; j < 8; ++j) un |= ((int)((j < 4 ? f0[j & 3] : f1[j & 3]) - call) < 0 && lane * 8 + j >= first ? 1u : 0u) << j;
        const int l = __builtin_ctzll(__ballot(un != 0u) | (1ull << 63));
        const int u = l * 8 + __builtin_ctz(__builtin_amdgcn_readlane(un, l) | 256u);
        const int fl = u <= NIN * 4 ? (u - NPUSH) << (6 - PSH) : u >= NFLAG ? NPC + 128 : NPIN + (u - NIN * 4) / NS * BPB;
        flim = fl > flim ? fl : flim;
        if (!planned && (unsigned)__builtin_amdgcn_readlane(f0[0], 63) == call && (unsigned)__builtin_amdgcn_readlane(f0[2], 63) == call) {
            plan = (unsigned long long)(unsigned)__builtin_amdgcn_readlane(f0[3], 63) << 32 | (unsigned)__builtin_amdgcn_readlane(f0[1], 63);
            planned = true; fin = NPC - moved(peer);
        }
        const int pl = planned ? fin : NPC - ZB;
        lim = flim < pl ? flim : pl;
    };
    auto make_plan = [&]() {
        unsigned old = 0;
        if (lane == 0) old = __hip_atomic_exchange((unsigned*)(wsl + WS_PH + 2 * NRANK * 128), call, __ATOMIC_RELAXED, __HIP_MEMORY_SCOPE_AGENT);
        if ((unsigned)__builtin_amdgcn_readfirstlane(old) == call) return;
        const unsigned hd = __builtin_amdgcn_raw_buffer_load_b32(rwl, lane < NRANK ? (unsigned)lane * 128u : OOB, hoff, AUX_SYS);
        const int t = lane >> 3, s = lane & 7;
        const unsigned zc = ld_sys((const unsigned*)(P.ws[t] + WS_ZC + (rank * NRANK + s) * 128));
        const bool ok = __all(t == rank || s == rank || s == t || (int)(zc - call * (unsigned)(ZB * PU)) >= 0);
        int h[NRANK], sum = 0, give = 0, take = 0;
#pragma unroll
        for (int j = 0; j < NRANK; ++j) { h[j] = __builtin_amdgcn_readlane(hd, j); sum += j == rank ? 0 : h[j]; }
#pragma unroll
        for (int j = 0; j < NRANK; ++j) {
            const int d = j == rank ? 0 : (sum - (NRANK - 1) * h[j]) * CFG_BAL / (8 * (NRANK - 1));
            h[j] = d > ZB ? ZB : d < -127 ? -127 : d;
            if (h[j] > 0) give += h[j]; else take -= h[j];
        }
        give = take = !ok ? 0 : give < take ? give : take;
        unsigned long long pl = 0;
#pragma unroll
        for (int j = 0; j < NRANK; ++j) {
            int d = h[j];
            if (d > 0) { d = d < give ? d : give; give -= d; } else { d = -d < take ? d : -take; take += d; }
            pl |= (unsigned long long)(d & 255) << (j * 8);
        }
        __builtin_amdgcn_raw_buffer_store_b128((u32x4){call, (unsigned)pl, call, (unsigned)(pl >> 32)}, rwl, lane < NRANK ? WS_FLAG + 63 * 32 + (unsigned)lane * FLAGB : OOB, 0, AUX_SYS);
    };
    auto offs_of = [&](int i, unsigned (&o)[4], unsigned& cnt) -> unsigned {
        if (i < NPIN) {
            int q, pos; const int pan = unit_of((i >> (6 - PSH)) + NPUSH, rank, q, pos);
            cnt = WS_CNT + (unsigned)(pan * NRANK + peer) * 128u + (unsigned)q * 4u;
            const unsigned sub = (unsigned)((i << PSH) & 63) * 1024u;
            const unsigned b = WS_PAN + (unsigned)pan * PANB + (unsigned)peer * CHB + (unsigned)q * UNB + sub + (unsigned)lane * 16u;
#pragma unroll
            for (int k = 0; k < 4; ++k) o[k] = b + k * 1024u;
            return WS_SND + (call & 1u) * SNDB + (unsigned)pos * UNB + sub;
        }
        int src = peer;
        if (ZB && i >= NPC) {
            int x = i - NPC;
            for (int j = 0; j < peer; ++j) x -= moved(j) < 0 ? moved(j) : 0;
            for (int j = 0; j < NRANK; ++j) { const int g = moved(j) > 0 ? moved(j) : 0; if ((unsigned)x < (unsigned)g) { src = j; i = NPC - g + x; } x -= g; }
        }
        const int e = (i - NPIN) / BPB, r = (i - NPIN) % BPB, pb = band_of(e, rank) * BPB + r;
        cnt = e < NRANK - 1 && r >= BPB - ZB ? WS_ZC + (unsigned)(band_of(e, rank) * NRANK + peer) * 128u : 0u;
#pragma unroll
        for (int k = 0; k < 4; ++k) {
            const unsigned sg = (unsigned)((pb << PSH) + k) * 2u + (unsigned)(lane >> 5);
            o[k] = (unsigned)src * ROWB + (unsigned)(lane & 31) * 16u + sg / NS * (NN * 2u) + sg % NS * 512u;
        }
        return WS_OSN + (unsigned)(pb << PSH) * 1024u;
    };
    while (!planned || cur < fin) {
        if (!bad) {
            const unsigned long long t0 = __builtin_amdgcn_s_memrealtime();
            const unsigned pvo = (unsigned)(lane - (first_of(cur) >> 3)) < CFG_PWIN || (lane == 63 && cur >= NPC - ZB) ? (unsigned)lane * 32u : OOB - 16u;
            for (;;) {
                const u32x4 f0 = __builtin_amdgcn_raw_buffer_load_b128(rwl, pvo, fso, AUX_SYS), f1 = __builtin_amdgcn_raw_buffer_load_b128(rwl, pvo + 16u, fso, AUX_SYS);
                update(f0, f1, cur);
                if (cur < lim || (planned && cur >= fin)) break;
                if (give_up(tmo, call, t0)) { bad = true; planned = true; plan = 0; fin = NPC; break; }
                __builtin_amdgcn_s_sleep(CFG_PS);
            }
            asm volatile("" ::: "memory");
            if (cur >= fin && planned) break;
        }
        unsigned o[4], p2 = OOB, p3 = OOB, cprev = 0;
        if (bad) {
            (void)offs_of(cur, o, cprev);
            if (cur >= NPIN) {
#pragma unroll
                for (int k = 0; k < PU; ++k) __builtin_amdgcn_raw_buffer_store_b128((u32x4){0x7fc07fc0u, 0x7fc07fc0u, 0x7fc07fc0u, 0x7fc07fc0u}, rol, o[k], 0, AUX_AGENT);
            }
            cur = index_of(draw()); continue;
        }
        STAMP1(P, 1);
        rsrc_t rp = rwl;
        u32x4 v[4] = {};
        const unsigned per = deep0 ? PACE16 / 2 : PACE16;
        unsigned slot = 0;
        if (CFG_RATE) { const unsigned now = (unsigned)__builtin_amdgcn_s_memrealtime() << 4; slot = now - (now & 0xffffffu) % PACE16 + (CFG_ALLD ? (unsigned)(((widx & 7) * 2 + (widx >> 3 & 1)) * (PW / 2) + (widx >> 4)) * (PACE16 / (PW * (CFG_ALLD ? CFG_ALLD : 1))) : (unsigned)((widx & 7) * PW + (widx >> 3)) * (PACE16 / NW)) - PACE16; }
        unsigned cb, src, in[4];
        auto prepare = [&]() {
            if (ZB && cur == NPC - CFG_PLAN) make_plan();
            src = __builtin_amdgcn_readfirstlane(offs_of(cur, o, cb)); cb = __builtin_amdgcn_readfirstlane(cb);
            if (ZB && cur >= NPC) src = 0;
#pragma unroll
            for (int k = 0; k < 4; ++k) { in[k] = ZB && cur >= NPC ? o[k] : lane * 16 + k * 1024; asm volatile("" : "+v"(o[k]), "+v"(in[k])); }
        };
        prepare();
        do {
            const bool tail = cur >= fin - CFG_TAIL, deep = deep0 || tail;
            const rsrc_t rd = cur < NPIN ? rwl : rol, rs = ZB && cur >= NPC ? ror : rwr;
            const bool look = flim < NPC && cur + CFG_LOOK >= flim, lookp = !planned && cur + CFG_LOOKP >= NPC - ZB;
            const unsigned fvo = look || (lookp && lane == 63) ? (unsigned)lane * 32u : OOB - 16u;
            unsigned cdone = cprev;
            unsigned nv = 0; u32x4 f0, f1;
#pragma unroll
            for (int k = 0; k < 4; ++k) {
                if (CFG_RATE && !tail) {
                    slot += per;
                    for (;;) {
                        const int d = (int)(((unsigned)__builtin_amdgcn_s_memrealtime() << 4) - slot);
                        if (d > CFG_CRED * 16) slot += (unsigned)(d - CFG_CRED * 16);
                        if (d >= 0) break;
                        __builtin_amdgcn_s_sleep(1);
                    }
                }
                v[k] = __builtin_amdgcn_raw_buffer_load_b128(rs, in[k], src, AUX_SYS_NT);
                if (k == 0) {
                    nv = draw();
                    f0 = __builtin_amdgcn_raw_buffer_load_b128(rwl, fvo, fso, AUX_SYS); f1 = __builtin_amdgcn_raw_buffer_load_b128(rwl, fvo + 16u, fso, AUX_SYS);
                }
                __builtin_amdgcn_raw_buffer_store_b128(v[k ^ 2], k < 2 ? rp : rd, k == 0 ? p2 : k == 1 ? p3 : o[k ^ 2], 0, AUX_AGENT);
                if (CFG_EARLY && !deep) {
                    unsigned& a = k == 0 ? p3 : o[k - 1];
                    __builtin_amdgcn_raw_buffer_store_b128(v[(k + 3) & 3], k < 1 ? rp : rd, a, 0, AUX_AGENT);
                    a = OOB;
                }
                if (k == 3) {
                    asm volatile("" ::: "memory");
                    cprev = cb; rp = rd; p2 = o[2]; p3 = o[3];
                    cur = index_of(nv);
                    if (look || lookp) update(f0, f1, cur);
                    if (cur < lim) prepare();
                }
                if (k == 0) WAIT_VMN(7); else if (k == 1) WAIT_VMN(6); else WAIT_VMN(3);
                if (!deep) {
                    if (k == 0) WAIT_VMN(4 + CFG_EARLY); else WAIT_VMN(1 + CFG_EARLY);
                    if (CFG_EARLY && k == 1) { arrive(cdone ? lane0 : OOB, cdone); cdone = 0; }
                }
            }
            arrive(cdone ? lane0 : OOB, cdone);
        } while (cur < lim);
        __builtin_amdgcn_raw_buffer_store_b128(v[2], rp, p2, 0, AUX_AGENT);
        __builtin_amdgcn_raw_buffer_store_b128(v[3], rp, p3, 0, AUX_AGENT);
        if (cprev) { WAIT_VM0(); arrive(lane0, cprev); }
    }
    STAMP(P, 4);
    asm volatile("" ::: "memory");
    if (CFG_HS && lane == 0) st_sys((unsigned*)(P.ws[peer] + WS_DONE + rank * 512) + widx, call);
    if (CFG_HS && widx == 0) {
        const unsigned* dn = (const unsigned*)(wsl + WS_DONE + peer * 512);
        const unsigned long long t0 = __builtin_amdgcn_s_memrealtime();
        for (;;) {
            bool ok = true;
            if (lane < NW) ok = (int)(ld_sys(dn + lane) - call) >= 0;
            if (NW > 64 && lane + 64 < NW) ok = ok && (int)(ld_sys(dn + 64 + lane) - call) >= 0;
            if (__all(ok)) break;
            if (give_up(tmo, call, t0)) {
                if (lane == 0) __builtin_amdgcn_raw_buffer_store_b32(0x7fc07fc0u, rol, (unsigned)(rank * NS) * 512u, 0, AUX_SYS);
                break;
            }
            __builtin_amdgcn_s_sleep(2);
        }
        STAMP(P, 5);
    }
}

__device__ __forceinline__ void pack_role(const Params& P, unsigned tk, int g, int wid, int lane) {
    const int rank = P.rank;
    const rsrc_t rx = mk_rsrc(P.x, (unsigned)MM * KSH * 2), rw = mk_rsrc(P.w, (unsigned)KSH * NN * 2);
    const int fr = lane & 15, fq = lane >> 4;
    const unsigned t0 = (unsigned)__builtin_amdgcn_s_memrealtime();
    unsigned long long wsp = (unsigned long long)P.ws[lane & (NRANK - 1)];
    for (int u = g; u < NUNIT; u += NPK) {
        int o, dest = -1;
        if (u < 4 * U0) {
            const int r = u % U0;
            if (r < NRANK * NS) { o = r >> 3; dest = (rank + 1 + r) & (NRANK - 1); } else o = r - (NRANK - 1) * NS;
            o += u / U0 * G0;
        } else o = u - 4 * (NRANK - 1) * NS;
        int q, pos; const int pan = unit_of(o, dest < 0 ? 0 : dest, q, pos);
        const bool push = o < NPUSH && dest != rank;
        if (u > CFG_PACE0) while ((int)((unsigned)__builtin_amdgcn_s_memrealtime() - t0) < (u - CFG_PACE0) * CFG_PACE / 10) __builtin_amdgcn_s_sleep(1);
        const int ktl = q * 2 + (wid >> 2);
        const unsigned dst = push ? WS_PAN + (unsigned)pan * PANB + (unsigned)(rank * KTS + ktl) * KTB : WS_SND + (unsigned)pos * UNB + (unsigned)(wid >> 2) * KTB;
        rsrc_t rwl; unsigned sv, c;
        auto target = [&]() {
            c = tk; asm volatile("" : "+v"(c), "+v"(wsp) :: "memory");
            const int to = push ? dest & (NRANK - 1) : rank;
            rwl = mk_rsrc((const void*)((unsigned long long)(unsigned)__builtin_amdgcn_readlane((unsigned)(wsp >> 32), to) << 32 | (unsigned)__builtin_amdgcn_readlane((unsigned)wsp, to)), WS_END);
            sv = (unsigned)lane * 16u + (push ? 0u : (~c & 1u) * SNDB);
        };
        u32x4 v[8];
        if (pan < NTM) {
            const int rb0 = (wid & 3) * 4;
            const unsigned vo = (unsigned)fr * (KSH * 2) + (unsigned)fq * 16u;
            const unsigned so = (unsigned)(pan * 256 + rb0 * 16) * (KSH * 2) + (unsigned)ktl * 128u;
#pragma unroll
            for (int i = 0; i < 8; ++i) v[i] = __builtin_amdgcn_raw_buffer_load_b128(rx, vo + (i & 1) * 64, so + (unsigned)(i >> 1) * (16 * KSH * 2), 0);
            target();
#pragma unroll
            for (int i = 0; i < 8; ++i) STORE16(v[i], rwl, sv + (i & 1) * 1024, dst + (unsigned)(rb0 + (i >> 1)) * 2048u, AUX_SYS);
        } else {
            const int ks = (wid >> 1) & 1, wr = wid & 1;
            const unsigned vo = (unsigned)(fq * 8) * (NN * 2) + (unsigned)fr * 16u;
            const unsigned so = (unsigned)(ktl * 64 + ks * 32) * (NN * 2) + (unsigned)(gpan_of(pan - NTM, dest < 0 ? 0 : dest) * 256 + wr * 128) * 2u;
#pragma unroll
            for (int j = 0; j < 8; ++j) v[j] = __builtin_amdgcn_raw_buffer_load_b128(rw, vo, so + (unsigned)j * (NN * 2), 2);
            target();
            while (CFG_P3 && push && o >= CFG_P3N && (int)((unsigned)__builtin_amdgcn_s_memrealtime() - t0) < CFG_P3 + (o - CFG_P3N) * 110) __builtin_amdgcn_s_sleep(1);
#pragma unroll
            for (int i = 0; i < 8; ++i) {
                u32x4 o;
#pragma unroll
                for (int e = 0; e < 4; ++e) o[e] = __builtin_amdgcn_perm(v[2 * e + 1][i >> 1], v[2 * e][i >> 1], (i & 1) ? 0x07060302u : 0x05040100u);
                STORE16(o, rwl, sv, dst + (unsigned)(i >> 2) * HTB + (unsigned)wr * 8192u + (unsigned)(i & 3) * 2048u + (unsigned)ks * 1024u, AUX_SYS);
            }
        }
        const bool own = lane == rank || push;
        unsigned long long fp = wsp + (own ? WS_CNT + (unsigned)((pan * NRANK + rank) * 32 + q) * 4u : WS_FLAG + (unsigned)rank * FLAGB + (unsigned)o * 4u);
        unsigned fv = own ? (c + 1u) * CNT_FULL : c + 1u;
        asm volatile("" : "+v"(fp), "+v"(fv));
        WAIT_VMN(0);
        if (u == g) STAMP(P, 6);
        __builtin_amdgcn_s_barrier();
        if (wid == 0 && lane < NRANK && (dest < 0 || dest == lane)) __hip_atomic_store((__attribute__((address_space(1))) unsigned*)fp, fv, __ATOMIC_RELAXED, __HIP_MEMORY_SCOPE_SYSTEM);
    }
}

struct Tile { int tm, v, half; unsigned sA, stA; };
__device__ __forceinline__ void tile_of(int idx, int col, int rank, Tile& t) {
    const int v = 2 * (idx >> 2) + (col >> 2);
    t.tm = 4 * (col & 3) + (idx & 3); t.v = v; t.half = -1;
    const int g = (v - GM) / GW, i = (v - GM) % GW;
    t.sA = WS_SND + (unsigned)(v < GW ? gpan_of(v, rank) : v < GW + GM ? 4 * U0 + v - GW : 4 * U0 + 4 * GM + (g - 1) * 4 * GW + i) * UNB;
    t.stA = (unsigned)(v < GW ? U0 : v < GW + GM ? GM : g <= NGF ? GW : GL) * UNB;
}
__device__ __forceinline__ void item_of(int q, int rank, Tile& t) {
    const int n = q < PFULL ? q : PFULL + ((q - PFULL) >> 1);
    if (n < 4 * PL0) tile_of(RS * KX + n % PL0, n / PL0, rank, t);
    else tile_of(RS * KX + (n - 4 * PL0) % PL1, 4 + (n - 4 * PL0) / PL1, rank, t);
    if (q >= PFULL) t.half = (q - PFULL) & 1;
}

template <int wr>
__device__ __forceinline__ void gemm_role(const Params& P, unsigned call, int col, int slot, LAS unsigned char* lds) {
    const int tid = threadIdx.x, wid = __builtin_amdgcn_readfirstlane(tid >> 6), lane = tid & 63, wc = wid & 3;
    const int rank = P.rank;
    char* wsl = P.wsl;
    const rsrc_t rws = mk_rsrc(wsl, WS_END);
    LAS unsigned* ctl = (LAS unsigned*)(lds + STAGE_BYTES);
    const unsigned voff = (unsigned)tid * 16u;
    const unsigned ldsw = (unsigned)wid * 1024u;
    const int aoff = wr * 8192 + lane * 16, boff = wc * 4096 + lane * 16;

    auto lane_now = [&]() { int tz = threadIdx.x; asm volatile("" : "+v"(tz)); return tz & 63; };
    auto cnt_of = [&](const Tile& t) { const int ln = lane_now(); return WS_CNT + (unsigned)((((ln & 32) ? NTM + t.v : t.tm) * NRANK + ((ln >> 2) & 7)) * 32 + (ln & 3)) * 4u; };
    auto level_of = [&](unsigned c) -> int {
        const unsigned long long miss = __ballot((int)(c - call * CNT_FULL) < 0);
        unsigned m = (unsigned)miss | (unsigned)(miss >> 32);
        const bool own = ((m >> (rank * 4)) & 15u) == 0u;
        m |= m >> 16; m |= m >> 8; m |= m >> 4;
        return own ? 1 + __builtin_ctz(m | 16u) : 0;
    };
    auto ready = [&](const Tile& t, int want) -> int {
        if (wr == 0 && wid == 0) {
            const unsigned* c = (const unsigned*)(wsl + cnt_of(t));
            unsigned* tmo = (unsigned*)(wsl + WS_TMO);
            const unsigned long long t0 = __builtin_amdgcn_s_memrealtime();
            int got;
            for (;;) {
                got = level_of(ld_sys(c));
                if (got >= want) break;
                if (give_up(tmo, call, t0)) { ctl[0] = 1u; got = 5; break; }
                __builtin_amdgcn_s_sleep(4);
            }
            ctl[2] = (unsigned)got;
            __builtin_amdgcn_fence(__ATOMIC_ACQUIRE, "agent");
            asm volatile("s_waitcnt vmcnt(0) lgkmcnt(0)" ::: "memory");
        }
        asm volatile("" ::: "memory"); __builtin_amdgcn_s_barrier(); asm volatile("" ::: "memory");
        const int r = __builtin_amdgcn_readfirstlane(ctl[2]);
        asm volatile("s_waitcnt lgkmcnt(0)" ::: "memory");
        return r;
    };

#define SA(b, h) (((b) * 2 + (h)) * HTB)
#define SB(b, h) ((4 + (b) * 2 + (h)) * HTB)
#define STAGE(bufoff, soff) do { _Pragma("unroll") for (int _i = 0; _i < 2; ++_i) \
        __builtin_amdgcn_raw_ptr_buffer_load_lds(rws, (LAS unsigned*)(lds + (bufoff) + ldsw + _i * 8192), 16, voff, (soff) + _i * 8192u, 0, 0); } while (0)
#define LDA(dst, b, h) do { _Pragma("unroll") for (int m = 0; m < 4; ++m) _Pragma("unroll") for (int k = 0; k < 2; ++k) dst[m][k] = *(const LAS bf16x8*)(lds + SA(b, h) + aoff + m * 2048 + k * 1024); } while (0)
#define LDB(dst, b, h) do { _Pragma("unroll") for (int n = 0; n < 2; ++n) _Pragma("unroll") for (int k = 0; k < 2; ++k) dst[n][k] = *(const LAS bf16x8*)(lds + SB(b, h) + boff + n * 2048 + k * 1024); } while (0)
#define MMA(ai, first, prio) do { __builtin_amdgcn_s_setprio(prio); \
        _Pragma("unroll") for (int i = 0; i < 32; ++i) { \
            const int sn = ((i >> 2) & 1) ? 3 - (i & 3) : i & 3; \
            const int bj = CFG_MO == 7 ? ((i >> 2) ^ (i >> 3)) & 1 : CFG_MO == 0 || CFG_MO == 4 ? i >> 4 : CFG_MO == 1 ? (i >> 1) & 1 : CFG_MO == 5 ? sn >> 1 : (i >> 3) & 1; \
            const int m = CFG_MO == 7 ? i >> 3 : CFG_MO == 2 ? i & 3 : CFG_MO == 6 ? sn : (i >> 2) & 3; \
            const int n = CFG_MO == 7 ? ((i >> 1) ^ (i >> 3)) & 1 : CFG_MO == 0 || CFG_MO == 4 ? (i >> 1) & 1 : CFG_MO == 1 ? i & 1 : CFG_MO == 5 ? sn & 1 : (i >> 2) & 1; \
            const int k = CFG_MO == 7 ? (i ^ (i >> 1)) & 1 : CFG_MO == 0 || CFG_MO == 4 ? i & 1 : i >> 4; \
            if (CFG_MO >= 4) SCHED; \
            acc[ai][bj][m][n] = __builtin_amdgcn_mfma_f32_16x16x32_bf16(bj ? B1[n][k] : B0[n][k], At[m][k], first && (CFG_MO == 7 ? (i & 1) == 0 : k == 0) ? (f32x4){0.f, 0.f, 0.f, 0.f} : acc[ai][bj][m][n], 0, 0, 0); } \
        __builtin_amdgcn_s_setprio(0); } while (0)
#define WAIT_L0() do { asm volatile("" ::: "memory"); __builtin_amdgcn_s_waitcnt(0xC07F); asm volatile("" ::: "memory"); } while (0)
#define WAIT_V(n) asm volatile("s_waitcnt vmcnt(" #n ")" ::: "memory")
#define WAIT_L(n) asm volatile("s_waitcnt lgkmcnt(" #n ")" ::: "memory")
#define BAR __builtin_amdgcn_s_barrier()
#define SCHED __builtin_amdgcn_sched_barrier(0)

    unsigned long long outp = (unsigned long long)P.outl;
    asm volatile("" : "+v"(outp));
    Tile cur, nxt; int rnd = 0;
    tile_of(slot, col, rank, cur);
    bf16x8 At[4][2], B0[2][2], B1[2][2];
    constexpr unsigned SB0 = WS_SND + ((NRANK - 1) * NS + GW) * UNB, STB = U0 * UNB;
    const unsigned kov = WS_PAN + (unsigned)((rank + 1 + (lane + 3) % 7) & (NRANK - 1)) * CHB + (unsigned)(((lane & 31) - 4) / 7) * UNB;
    unsigned tabA, tabB;
    auto tables = [&](const Tile& t) {
        const int ln = lane_now();
        const unsigned la = t.sA + (call & 1u) * SNDB + (unsigned)ln * t.stA, ra = (unsigned)(NTM + t.v) * PANB + kov;
        const unsigned lb = SB0 + (call & 1u) * SNDB + (unsigned)t.tm * UNB + (unsigned)ln * STB, rb = (unsigned)t.tm * PANB + kov;
        tabA = ln < 4 ? la : ra;
        tabB = (ln < 4 ? lb : rb) + (t.half == 1 ? HTB : 0u);
    };
    auto pick = [&]() {
        Tile t = cur; bool ok = true;
        if (rnd + 1 < RS) tile_of((rnd + 1) * KX + slot, col, rank, t);
        else {
            const unsigned d = (unsigned)__builtin_amdgcn_raw_ptr_buffer_atomic_add_i32(1, rws, lane_now() == 0 ? 0u : 0xfffffff0u, WS_HEAD + (unsigned)rank * 128u, 0);
            const unsigned q = (unsigned)__builtin_amdgcn_readfirstlane(d) - (call - 1u) * (unsigned)(NITEM + 8 * KX);
            ok = q < (unsigned)NITEM;
            if (ok) item_of((int)q, rank, t);
        }
        *(LAS u32x4*)(ctl + 4) = (u32x4){ok ? (unsigned)(t.tm | t.v << 8 | (t.half + 2) << 16) : 0u, t.sA, t.stA, 0u};
        __builtin_amdgcn_raw_ptr_buffer_load_lds(rws, ctl + 64, 4, cnt_of(t), 0, 0, AUX_SYS);
    };
    auto picked = [&](int& level) -> bool {
        const u32x4 f = *(const LAS u32x4*)(ctl + 4);
        level = level_of(ctl[64 + lane_now()]);
        const unsigned w = __builtin_amdgcn_readfirstlane(f[0]);
        nxt.tm = w & 255u; nxt.v = (w >> 8) & 255u; nxt.half = (int)(w >> 16) - 2;
        nxt.sA = __builtin_amdgcn_readfirstlane(f[1]); nxt.stA = __builtin_amdgcn_readfirstlane(f[2]);
        return w != 0u;
    };
    constexpr int JP = KT / 2 - 4, JI = KT / 2 - 2, JL = KT / 2 - 1;
    const int jend = wr == 0 && wid == 0 ? JP : JL;
    int lvl = ready(cur, 1);
    STAMP(P, 2);
    tables(cur);
    unsigned a2 = cur.sA + (call & 1u) * SNDB, b2 = SB0 + (call & 1u) * SNDB + (unsigned)cur.tm * UNB;
    STAGE(SB(0, 0), b2); STAGE(SA(0, 0), a2); STAGE(SB(0, 1), b2 + HTB); STAGE(SA(0, 1), a2 + HTB);
    STAGE(SB(1, 0), b2 + KTB); STAGE(SA(1, 0), a2 + KTB); STAGE(SB(1, 1), b2 + HTB + KTB);
    WAIT_V(0); BAR;
#define TICK(i, r) do {} while (0)
#define ROUND_DONE() do {} while (0)
    bool has_next = false;
#pragma unroll 1
    for (;;) {
        f32x4 acc[2][2][4][2];
        int jchk = lvl < 5 ? 7 * lvl - 4 : jend;
        TICK(0, 2); TICK(4, 3);
#define SYNC_MMA(ai, first, wv) do { if (wv && (wr == 1 || !CFG_LW)) WAIT_V(8); WAIT_L0(); if (wr == 1) BAR; MMA(ai, first, wr + 1); if (wv && wr == 0 && CFG_LW) WAIT_V(8); if (wr == 0) BAR; SCHED; } while (0)
#define PASS(first, check) do { \
            LDB(B0, 0, 0); LDA(At, 0, 0); LDB(B1, 0, 1); STAGE(SA(1, 1), a2 + KTB + HTB); SCHED; \
            check; \
            a2 = __builtin_amdgcn_readlane(tabA, (j + 1) & 31); b2 = __builtin_amdgcn_readlane(tabB, (j + 1) & 31); \
            SYNC_MMA(0, first, !first); \
            LDA(At, 0, 1); STAGE(SB(0, 0), b2); STAGE(SA(0, 0), a2); STAGE(SB(0, 1), b2 + HTB); \
            SYNC_MMA(1, first, !first); \
            LDB(B0, 1, 0); LDA(At, 1, 0); LDB(B1, 1, 1); STAGE(SA(0, 1), a2 + HTB); \
            SYNC_MMA(0, false, true); \
            LDA(At, 1, 1); STAGE(SB(1, 0), b2 + KTB); STAGE(SA(1, 0), a2 + KTB); STAGE(SB(1, 1), b2 + KTB + HTB); \
            SYNC_MMA(1, false, true); } while (0)
        int j = 0;
        PASS(true, );
#pragma unroll 1
        for (j = 1; j < KT / 2; ++j)
            PASS(false, if (__builtin_expect(j == jchk, 0)) {
                    if (j < JP) { lvl = ready(cur, lvl + 1); jchk = lvl < 5 ? 7 * lvl - 4 : jend; }
                    else if (wr == 0 && j == JP) { pick(); jchk = JI; }
                    else if (wr == 0 && j == JI) { asm volatile("buffer_inv sc1" ::: "memory"); jchk = JL; }
                    else {
                        has_next = picked(lvl);
                        if (has_next) {
                            const int want = nxt.half < 0 ? 1 : 5;
                            if (lvl < want) lvl = ready(nxt, want);
                            tables(nxt);
                        }
                    }
                });
        if (has_next && nxt.half >= 0) STAGE(SA(1, 1), a2 + KTB + HTB);
        TICK(1, 2);
        TICK(2, 2);
        {
            const int ln = lane_now(), fr = ln & 15, fq = ln >> 4;
            const rsrc_t rol = mk_rsrc((const void*)outp, OUTB);
            const bool shared = cur.v < NS;
            if (__builtin_amdgcn_readfirstlane(ctl[0]) != 0u) {
#pragma unroll
                for (int i = 0; i < 32; ++i) acc[i >> 4][(i >> 3) & 1][(i >> 1) & 3][i & 1] = __builtin_bit_cast(f32x4, (u32x4){0x7fc00000u, 0x7fc00000u, 0x7fc00000u, 0x7fc00000u});
            }
            if (rnd == 0 && cur.v < GW) {
                const unsigned long long t0 = __builtin_amdgcn_s_memrealtime();
                while (__builtin_amdgcn_s_memrealtime() - t0 < (unsigned long long)((col & 3) * CFG_STAG + (col >> 2) * CFG_STAGB)) __builtin_amdgcn_s_sleep(8);
            }
            const bool thru = rnd >= RS;
            const unsigned vo = (unsigned)(wc * 32 + fq * 4) * (NN * 2) + (unsigned)(wr * 256 + fr * 16);
            const unsigned so = (unsigned)(cur.tm * 256) * (NN * 2) + (unsigned)gpan_of(cur.v, rank) * 512u;
            const unsigned vs = (unsigned)(wc * 32 + fq * 4) * ROWB + (unsigned)(wr * 256 + fr * 16);
            const unsigned ss = WS_OSN + (unsigned)(cur.tm * 256) * ROWB + (unsigned)cur.v * 512u;
            auto finish = [&](auto mode) {
#pragma unroll
                for (int bj = 0; bj < 2; ++bj)
#pragma unroll
                    for (int n = 0; n < 2; ++n)
#pragma unroll
                        for (int r = 0; r < 4; ++r) {
                            u32x4 o;
                            if ((r & 1) == 0) {
#pragma unroll
                                for (int i = 0; i < 8; ++i) { f32x4& a = acc[i >> 2][bj][i & 3][n]; const f32x2 z = silu2((f32x2){a[r], a[r + 1]}); a[r] = z[0]; a[r + 1] = z[1]; }
                            }
#pragma unroll
                            for (int i = 0; i < 4; ++i) o[i] = cvt_pk_bf16(acc[i >> 1][bj][(i & 1) * 2][n][r], acc[i >> 1][bj][(i & 1) * 2 + 1][n][r]);
                            if (mode == 1) STORE16(o, rws, vs, ss + (unsigned)(bj * 128 + n * 16 + r) * ROWB, AUX_SYS);
                            STORE16(o, rol, vo, so + (unsigned)(bj * 128 + n * 16 + r) * (NN * 2), mode == 2 ? AUX_SYS : CFG_RST);
                            if (mode == 2 && CFG_LAST > 1) { WAIT_VM0(); __builtin_amdgcn_s_sleep(CFG_LAST > 1 ? CFG_LAST - 1 : 1); }
                        }
            };
            if (shared) finish(std::integral_constant<int, 1>());
            else if (thru) finish(std::integral_constant<int, 2>());
            else finish(std::integral_constant<int, 0>());
            WAIT_V(16);
            if (shared) {
                WAIT_VM0();
                unsigned old = 0;
                if (ln == 0) old = __hip_atomic_fetch_add(ctl + 1, 1u, __ATOMIC_RELAXED, __HIP_MEMORY_SCOPE_WORKGROUP);
                old = __builtin_amdgcn_readfirstlane(old);
                if ((old & 7u) == 7u && ln < NRANK && ln != rank)
                    st_sys((unsigned*)(P.ws[ln] + WS_FLAG + rank * FLAGB) + NIN * 4 + turn_of(cur.tm, ln) * NS + cur.v, call);
            }
        }
        TICK(3, 2);
        ROUND_DONE();
        if (!has_next) break;
        cur = nxt; ++rnd;
        if (cur.half >= 0) break;
    }
#define HB(b, i) ((b) < 2 ? ((i) < 2 ? SA(b, i) : SB(b, 0)) : (i) < 2 ? SB(i, 1) : 8 * HTB)
#define HSTAGE(b, t) do { const unsigned _o = ((t) & 1) * KTB, _a = __builtin_amdgcn_readlane(tabA, ((t) >> 1) & 31) + _o, _b = __builtin_amdgcn_readlane(tabB, ((t) >> 1) & 31) + _o; \
        STAGE(HB(b, 0), _a); STAGE(HB(b, 1), _a + HTB); STAGE(HB(b, 2), _b); } while (0)
#define HLOAD(b) do { _Pragma("unroll") for (int i = 0; i < 4; ++i) B0[i >> 1][i & 1] = *(const LAS bf16x8*)(lds + HB(b, 2) + boff + i * 1024); \
        _Pragma("unroll") for (int i = 0; i < 8; ++i) At[i >> 1][i & 1] = *(const LAS bf16x8*)(lds + HB(b, 0) + aoff + i * 1024); \
        _Pragma("unroll") for (int i = 0; i < 8; ++i) Au[i >> 1][i & 1] = *(const LAS bf16x8*)(lds + HB(b, 1) + aoff + i * 1024); } while (0)
#define HPASS(b, t, first) do { \
        if (wr == 0) { HLOAD(b); HSTAGE((b + 2) % 3, (t) + 2); } else { HSTAGE((b + 2) % 3, (t) + 2); SCHED; HLOAD(b); if (!first) WAIT_V(6); } \
        WAIT_L0(); if (wr == 1) BAR; \
        __builtin_amdgcn_s_setprio(wr + 1); \
        _Pragma("unroll") for (int i = 0; i < 32; ++i) { const int ai = i >> 4, m = (i >> 2) & 3, n = (i >> 1) & 1, k = i & 1; \
            acc[ai][m][n] = __builtin_amdgcn_mfma_f32_16x16x32_bf16(B0[n][k], ai ? Au[m][k] : At[m][k], first && k == 0 ? (f32x4){0.f, 0.f, 0.f, 0.f} : acc[ai][m][n], 0, 0, 0); } \
        __builtin_amdgcn_s_setprio(0); \
        if (wr == 0) { if (!first) WAIT_V(6); BAR; } SCHED; } while (0)
#pragma unroll 1
    while (PFULL < PN && has_next) {
        f32x4 acc[2][4][2];
        bf16x8 Au[4][2];
        BAR;
        HPASS(0, 0, true);
#pragma unroll 1
        for (int t = 1; t < KT; t += 3) {
            if (wr == 0 && wid == 0 && t == KT - 9) pick();
            HPASS(1, t, false); HPASS(2, t + 1, false); HPASS(0, t + 2, false);
        }
        has_next = picked(lvl);
        if (has_next) {
            (void)ready(nxt, 5);
            tables(nxt);
            HSTAGE(0, 0); HSTAGE(1, 1);
        }
        const int ln = lane_now(), fr = ln & 15, fq = ln >> 4;
        const rsrc_t rol = mk_rsrc((const void*)outp, OUTB);
        if (__builtin_amdgcn_readfirstlane(ctl[0]) != 0u) {
#pragma unroll
            for (int i = 0; i < 16; ++i) acc[i >> 3][(i >> 1) & 3][i & 1] = __builtin_bit_cast(f32x4, (u32x4){0x7fc00000u, 0x7fc00000u, 0x7fc00000u, 0x7fc00000u});
        }
        const unsigned vo = (unsigned)(wc * 32 + fq * 4) * (NN * 2) + (unsigned)(wr * 256 + fr * 16);
        const unsigned so = (unsigned)(cur.tm * 256 + cur.half * 128) * (NN * 2) + (unsigned)gpan_of(cur.v, rank) * 512u;
#pragma unroll
        for (int n = 0; n < 2; ++n)
#pragma unroll
            for (int r = 0; r < 4; ++r) {
                u32x4 o;
                if ((r & 1) == 0) {
#pragma unroll
                    for (int i = 0; i < 8; ++i) { f32x4& a = acc[i >> 2][i & 3][n]; const f32x2 z = silu2((f32x2){a[r], a[r + 1]}); a[r] = z[0]; a[r + 1] = z[1]; }
                }
#pragma unroll
                for (int i = 0; i < 4; ++i) o[i] = cvt_pk_bf16(acc[i >> 1][(i & 1) * 2][n][r], acc[i >> 1][(i & 1) * 2 + 1][n][r]);
                STORE16(o, rol, vo, so + (unsigned)(n * 16 + r) * (NN * 2), AUX_SYS);
            }
        WAIT_V(8);
        ROUND_DONE();
        cur = nxt; ++rnd;
    }
    WAIT_V(0);
}

__global__ __launch_bounds__(512, 2) void fused_gemm_ar(Params P) {
    extern __shared__ __attribute__((aligned(16))) unsigned char shm[];
    LAS unsigned char* lds = (LAS unsigned char*)shm;
    const int wid = __builtin_amdgcn_readfirstlane(threadIdx.x >> 6), lane = threadIdx.x & 63;
    const int b = __builtin_amdgcn_readfirstlane(blockIdx.x);
    asm volatile("" :: "s"(P.x), "s"(P.w), "s"(P.wsl), "s"(P.rank));
    unsigned* tkp = (unsigned*)(P.wsl + WS_TICKET + b * 128 + wid * 16);
    const unsigned tk = ld_agent(tkp);
    if (b < NPULL) {
        const unsigned call = __builtin_amdgcn_readfirstlane(tk) + 1u;
        st_agent(tkp, call);
        const int pi = b / PW, slot = b % PW;
        const int peer = (P.rank + 1 + pi) & (NRANK - 1);
        pull_role(P, call, peer, slot * 8 + wid, lane);
    } else {
        const int g = b - NPULL;
        ((LAS unsigned*)(lds + STAGE_BYTES))[lane & 3] = 0u;
        STAMP(P, 0);
        pack_role(P, tk, g, wid, lane);
        const unsigned call = __builtin_amdgcn_readfirstlane(tk) + 1u;
        st_agent(tkp, call);
        STAMP(P, 1);
        asm volatile("s_waitcnt lgkmcnt(0)" ::: "memory");
        __builtin_amdgcn_s_barrier();
        if (b >= NPS) {
            const int col = CFG_SWAP == 1 ? (b & 7) ^ (b >> 1 & 1) : CFG_SWAP == 2 ? (b & 4) | ((0x1320 >> (b & 3) * 4) & 3) : b & 7;
            if (wid < 4) gemm_role<0>(P, call, col, (b - NPS) >> 3, lds);
            else gemm_role<1>(P, call, col, (b - NPS) >> 3, lds);
        }
    }
}
}

extern "C" void kernel_launch(int rank, int nranks, ncclComm_t comm,
                              hipStream_t stream,
                              void* const* d_in, const int* in_sizes, int n_in,
                              void* d_out, int out_size,
                              const int* in_dtypes, int out_dtype,
                              void* const* d_out_all,
                              void* const* d_ws_all, size_t ws_bytes) {
    if (nranks != NRANK || (unsigned)rank >= (unsigned)NRANK || ws_bytes < (size_t)WS_END || n_in < 2 ||
        in_sizes[0] != MM * KSH || in_sizes[1] != KSH * NN || out_size != MM * NN) {
        fprintf(stderr, "unexpected problem geometry\n");
        return;
    }
    static bool attr_done[NRANK] = {};
    if (!attr_done[rank]) {
        HIP_CHECK(hipFuncSetAttribute((const void*)fused_gemm_ar, hipFuncAttributeMaxDynamicSharedMemorySize, LDS_BYTES));
        attr_done[rank] = true;
    }
    Params P = {};
    P.x = d_in[0]; P.w = d_in[1]; P.rank = rank; P.wsl = (char*)d_ws_all[rank]; P.outl = (char*)d_out_all[rank];
    for (int p = 0; p < NRANK; ++p) { P.out[p] = (char*)d_out_all[p]; P.ws[p] = (char*)d_ws_all[p]; }
    hipLaunchKernelGGL(fused_gemm_ar, dim3(GRID), dim3(512), LDS_BYTES, stream, P);
    HIP_CHECK(hipGetLastError());
}
```

```cpp
#include <hip/hip_runtime.h>
#include <rccl/rccl.h>
#include <cstdio>
#include <type_traits>

#define HIP_CHECK(expr) do { hipError_t _e = (expr); if (_e != hipSuccess) { \
    fprintf(stderr, "HIP error %s at %s:%d\n", hipGetErrorString(_e), __FILE__, __LINE__); \
    return; } } while (0)

#define LAS __attribute__((address_space(3)))
typedef short bf16x8 __attribute__((ext_vector_type(8)));
typedef float f32x4 __attribute__((ext_vector_type(4)));
typedef unsigned u32x4 __attribute__((ext_vector_type(4)));
typedef __amdgpu_buffer_rsrc_t rsrc_t;

namespace {
constexpr int NRANK = 8;
constexpr int MM = 4096, KK = 4096, NN = 32768, KSH = KK / NRANK;
constexpr int KT = KK / 64, KTS = KSH / 64, NTM = MM / 256, NPAN = NN / 256;
#define CFG_NS 6
#define CFG_PW 8
constexpr int NS = CFG_NS;
constexpr int NRP = NPAN - NRANK * NS;
constexpr int NWP = NS + NRP;
constexpr int NIN = NTM + NWP;
constexpr int NOUT = NS * NTM;
constexpr int NFLAG = NIN * 4 + NOUT;
constexpr int NUNIT = NTM * 4 + NPAN * 4;
constexpr int GRID = 256;
constexpr int PW = CFG_PW;
constexpr int NPULL = PW * (NRANK - 1);
constexpr int NW = PW * 8;
constexpr int NPK = GRID - NPULL;
constexpr int NPS = (NPULL + 7) / 8 * 8;
constexpr int KX = (GRID - NPS) / 8;
#define CFG_GW (2 * (KX / 4))
constexpr int GW = CFG_GW;
constexpr int G0 = GW + NTM;
#define CFG_SKIP 1
#define CFG_SYNC 1
#define CFG_GM (CFG_SKIP ? GW + 2 : 2)
constexpr int GM = CFG_GM;
constexpr int NGF = (NWP - GW - GM) / GW, GL = (NWP - GW - GM) % GW;
static_assert(NS <= GW && NWP > GW, "shared panels sit in the first group");
#define CFG_PU 4
constexpr int PU = CFG_PU, PSH = PU == 1 ? 0 : PU == 2 ? 1 : PU == 4 ? 2 : 3;
#define CFG_PUSH 3
constexpr int NPUSH = CFG_PUSH;
constexpr int NPIN = (NIN * 4 - NPUSH) * 64 / PU;
static_assert(NPUSH <= NS, "pushed units have a single consumer");
constexpr int NPC = NPIN + NOUT * 128 / PU;
constexpr unsigned CNT_FULL = 64;

constexpr int NT0 = 4 * ((NWP + 1) / 2), NT1 = 4 * (NWP / 2);
constexpr int DK = CFG_SKIP ? KX - 2 * GW : 0;
constexpr int RS = (NT1 - 1 + DK) / KX;
constexpr int PL0 = NT0 - RS * KX + DK, PL1 = NT1 - RS * KX + DK, PN = 4 * (PL0 + PL1);
#define CFG_MO 7
#define CFG_SWAP 0x67542310u
#define CFG_LW 0
#define CFG_FD (CFG_SKIP ? 0 : -8)
constexpr int PFULL = 2 * PN - 8 * KX - CFG_FD < 0 ? 0 : 2 * PN - 8 * KX - CFG_FD > PN ? PN : 2 * PN - 8 * KX - CFG_FD;
constexpr int NITEM = PFULL + 2 * (PN - PFULL);
#define CFG_AFF 0
#define CFG_AFB 8
constexpr int HC = (PN - PFULL) / 8;
static_assert(!CFG_AFF || HC * 8 == PN - PFULL, "halves");
constexpr unsigned KTB = 32768, HTB = 16384;
constexpr unsigned PANB = KT * KTB;
constexpr unsigned CHB = KTS * KTB;
constexpr unsigned UNB = 2 * KTB;
constexpr unsigned WS_TICKET = 0;
constexpr unsigned WS_TMO = WS_TICKET + GRID * 128;
constexpr unsigned WS_FLAG = WS_TMO + 128;
constexpr unsigned FLAGB = 4096;
constexpr unsigned WS_DONE = WS_FLAG + NRANK * FLAGB;
constexpr unsigned WS_HEAD = WS_DONE + NRANK * 512;
constexpr unsigned WS_COL = WS_HEAD + NRANK * 128;
constexpr unsigned WS_TQ = WS_COL + 8 * 128;
constexpr unsigned WS_PH = WS_TQ + 2 * 2048;
constexpr unsigned WS_ZC = WS_PH + 2 * NRANK * 128 + 128;
constexpr unsigned WS_CNT = WS_ZC + NRANK * NRANK * 128;
constexpr unsigned WS_CTRL_END = WS_CNT + NIN * NRANK * 128;
constexpr unsigned WS_DBG = 512u << 10;
#define CFG_HS 0
constexpr unsigned WS_SND = 1u << 20, SNDB = CFG_HS ? 0 : NUNIT * UNB;
constexpr unsigned ROWB = NS * 512;
constexpr unsigned WS_OSN = WS_SND + NUNIT * UNB + SNDB;
constexpr unsigned WS_PAN = (WS_OSN + MM * ROWB + PANB - 1) / PANB * PANB;
constexpr unsigned WS_END = WS_PAN + NIN * PANB;
constexpr int U0 = G0 + (NRANK - 1) * NS;
constexpr unsigned OUTB = (unsigned)MM * NN * 2;
static_assert(WS_CTRL_END <= WS_DBG, "ctrl region");
static_assert(NFLAG <= 512 && 2048 <= (int)FLAGB, "flag region");
static_assert(NW <= 128, "done words");
static_assert(PU == 4, "block size");
#define CFG_DEEP 3
constexpr unsigned STAGE_BYTES = 9 * HTB;
constexpr unsigned LDS_BYTES = STAGE_BYTES + 512;
constexpr unsigned long long DEADLINE = 50000000ull;
constexpr int AUX_SYS = 17, AUX_SYS_NT = 19, AUX_AGENT = 16;
#define CFG_PACE 140
#define CFG_UH (2 * (NPK - 8 * DK))
#define CFG_TH 5000
#define CFG_PH 40
#define CFG_PACE0 40
#define CFG_STAG 600
#define CFG_P3 320
#define CFG_P3N 2
#define CFG_STAGB 300
#define CFG_LOOK 256
#define CFG_TAIL NW
#define CFG_ZB 120
#define CFG_LOOKP 100
#define CFG_PLAN (CFG_ZB + CFG_LOOKP + 50)
#define CFG_BAL 8
constexpr int ZB = CFG_ZB;
static_assert(ZB <= 127 && ZB <= NS * 128 / PU && NFLAG <= 504, "plan");
#define CFG_PWIN 4u
#define CFG_PS 2
#define CFG_LAST 1
#define CFG_RST 0
#define CFG_RATE 1710
#define CFG_CRED 25
#define CFG_ALLD 0
#define CFG_EARLY 1
constexpr unsigned PACE16 = CFG_RATE * PW * (CFG_ALLD ? CFG_ALLD : 8 + CFG_DEEP) / 64;

struct Params {
    const void* x; const void* w;
    char* wsl; char* outl;
    int rank, pad[7];
    char* ws[NRANK]; char* out[NRANK];
};

__device__ __forceinline__ rsrc_t mk_rsrc(const void* p, unsigned bytes) {
    const unsigned long long v = (unsigned long long)p;
    const unsigned lo = __builtin_amdgcn_readfirstlane((unsigned)v), hi = __builtin_amdgcn_readfirstlane((unsigned)(v >> 32));
    return __builtin_amdgcn_make_buffer_rsrc((void*)(((unsigned long long)hi << 32) | lo), 0, bytes, 0x00020000);
}
__device__ __forceinline__ unsigned ld_sys(const unsigned* p) { return __hip_atomic_load(p, __ATOMIC_RELAXED, __HIP_MEMORY_SCOPE_SYSTEM); }
__device__ __forceinline__ unsigned ld_agent(const unsigned* p) { return __hip_atomic_load(p, __ATOMIC_RELAXED, __HIP_MEMORY_SCOPE_AGENT); }
__device__ __forceinline__ void st_sys(unsigned* p, unsigned v) { __hip_atomic_store(p, v, __ATOMIC_RELAXED, __HIP_MEMORY_SCOPE_SYSTEM); }
__device__ __forceinline__ void st_agent(unsigned* p, unsigned v) { __hip_atomic_store(p, v, __ATOMIC_RELAXED, __HIP_MEMORY_SCOPE_AGENT); }
__device__ __forceinline__ void add_agent(unsigned* p, unsigned v) { (void)__hip_atomic_fetch_add(p, v, __ATOMIC_RELAXED, __HIP_MEMORY_SCOPE_AGENT); }
#define STORE16(v, rs, vo, so, aux) do { const u32x4 _d = (v); __builtin_amdgcn_raw_buffer_store_b128(_d, rs, vo, so, aux); asm volatile("s_nop 1" :: "v"(_d) : "memory"); } while (0)
#define WAIT_VM0() asm volatile("s_waitcnt vmcnt(0)" ::: "memory")
#define WAIT_VMN(n) do { asm volatile("" ::: "memory"); __builtin_amdgcn_s_waitcnt(0x0F70 | (n)); asm volatile("" ::: "memory"); } while (0)

__device__ __forceinline__ bool give_up(unsigned* tmo, unsigned call, unsigned long long t0) {
    const unsigned long long dt = __builtin_amdgcn_s_memrealtime() - t0;
    if (dt < 2000ull) return false;
    if ((unsigned)__builtin_amdgcn_readfirstlane(ld_agent(tmo)) == call) return true;
    if (dt > DEADLINE) { st_agent(tmo, call); return true; }
    return false;
}

__device__ __forceinline__ int gpan_of(int v, int consumer) { return v < NS ? consumer * NS + v : (NRANK - 1) * NS + v; }
__device__ __forceinline__ int band_of(int e, int consumer) { return e < 7 ? (consumer + 1 + e) & 7 : e < 15 ? 8 + ((consumer + e) & 7) : consumer; }
__device__ __forceinline__ int turn_of(int band, int consumer) { return band >= 8 ? 7 + ((band - consumer + 1) & 7) : band == consumer ? 15 : (band - consumer - 1) & 7; }
__device__ __forceinline__ int unit_of(int o, int consumer, int& q, int& pos) {
    if (o < 4 * G0) { q = o / G0; const int m = o % G0; pos = q * U0 + gpan_of(m, consumer); return m < GW ? NTM + m : m - GW; }
    pos = o + 4 * (NRANK - 1) * NS;
    o -= 4 * G0;
    if (o < 4 * GM) { q = o / (GM ? GM : 1); return NTM + GW + o % (GM ? GM : 1); }
    o -= 4 * GM;
    const int g = o / (4 * GW), r = o % (4 * GW);
    int i;
    if (GL == 0 || g < NGF) { q = r / GW; i = r % GW; } else { q = r / (GL ? GL : 1); i = r % (GL ? GL : 1); }
    return NTM + GM + GW * (g + 1) + i;
}

__device__ __forceinline__ unsigned cvt_pk_bf16(float lo, float hi) { unsigned r; asm("v_cvt_pk_bf16_f32 %0, %1, %2" : "=v"(r) : "v"(lo), "v"(hi)); return r; }
__device__ __forceinline__ float silu(float y) {
    const float c = __builtin_amdgcn_fmed3f(y, -60.0f, 60.0f);
    const float e = __builtin_amdgcn_exp2f(c * -1.44269504088896f);
    return y * __builtin_amdgcn_rcpf(1.0f + e);
}

typedef float f32x2 __attribute__((ext_vector_type(2)));
__device__ __forceinline__ f32x2 silu2(f32x2 y) {
    const f32x2 t = __builtin_elementwise_fma(y, (f32x2){-1.44269504088896f, -1.44269504088896f}, (f32x2){-86.5617024533378f, -86.5617024533378f});
    const f32x2 e = {__builtin_amdgcn_fmed3f(__builtin_amdgcn_exp2f(t[0]), 0.0f, 1.0f), __builtin_amdgcn_fmed3f(__builtin_amdgcn_exp2f(t[1]), 0.0f, 1.0f)};
    const f32x2 d = __builtin_elementwise_fma(e, (f32x2){1.1420073898156842e26f, 1.1420073898156842e26f}, (f32x2){1.0f, 1.0f});
    return y * (f32x2){__builtin_amdgcn_rcpf(d[0]), __builtin_amdgcn_rcpf(d[1])};
}

#define STAMP1(P, slot) do {} while (0)
#define STAMP(P, slot) do {} while (0)

__device__ __forceinline__ void pull_role(const Params& P, unsigned call, int peer, int widx, int lane) {
    const int rank = P.rank;
    char* wsl = P.wsl;
    const rsrc_t rwr = mk_rsrc(P.ws[peer], WS_END), rwl = mk_rsrc(wsl, WS_END);
    const rsrc_t rol = mk_rsrc(P.outl, OUTB), ror = mk_rsrc(P.out[peer], OUTB);
    unsigned* tmo = (unsigned*)(wsl + WS_TMO);
    constexpr unsigned OOB = 0xfffffff0u;
    const unsigned lane0 = lane == 0 ? 0u : OOB;
    const unsigned hoff = WS_PH + (call & 1u) * (NRANK * 128u);
    auto draw = [&]() { return (unsigned)__builtin_amdgcn_raw_ptr_buffer_atomic_add_i32(1, rwl, lane0, hoff + (unsigned)peer * 128u, 0); };
    if (widx == 0) __builtin_amdgcn_raw_buffer_store_b32(0u, rwl, lane0, WS_PH + (~call & 1u) * (NRANK * 128u) + (unsigned)peer * 128u, AUX_SYS);
    auto arrive = [&](unsigned vo, unsigned cnt) { (void)__builtin_amdgcn_raw_ptr_buffer_atomic_add_i32(PU, rwl, vo, cnt, 0); };
    if (CFG_ALLD && (widx & 7) * 2 + (widx >> 3 & 1) >= CFG_ALLD) return;
    const bool deep0 = CFG_ALLD || (widx & 7) < CFG_DEEP;
    bool bad = false; int stamped = 0; (void)stamped;
    STAMP(P, 0);
    auto index_of = [&](unsigned v) { return (int)__builtin_amdgcn_readfirstlane(v); };
    int cur = index_of(draw());
    constexpr int BPB = NS * 128 / PU;
    const unsigned fso = WS_FLAG + (unsigned)peer * FLAGB;
    auto first_of = [&](int i) { return i < NPIN ? (i >> (6 - PSH)) + NPUSH : NIN * 4 + (i - NPIN) / BPB * NS; };
    int lim = 0, flim = 0, fin = NPC;
    bool planned = ZB == 0;
    unsigned long long plan = 0;
    auto moved = [&](int j) { return (int)(signed char)(plan >> (j * 8)); };
    auto update = [&](const u32x4& f0, const u32x4& f1, int i) {
        const int first = first_of(i);
        unsigned un = 0;
#pragma unroll
        for (int j = 0; j < 8; ++j) un |= ((int)((j < 4 ? f0[j & 3] : f1[j & 3]) - call) < 0 && lane * 8 + j >= first ? 1u : 0u) << j;
        const int l = __builtin_ctzll(__ballot(un != 0u) | (1ull << 63));
        const int u = l * 8 + __builtin_ctz(__builtin_amdgcn_readlane(un, l) | 256u);
        const int fl = u <= NIN * 4 ? (u - NPUSH) << (6 - PSH) : u >= NFLAG ? NPC + 128 : NPIN + (u - NIN * 4) / NS * BPB;
        flim = fl > flim ? fl : flim;
        if (!planned && (unsigned)__builtin_amdgcn_readlane(f0[0], 63) == call && (unsigned)__builtin_amdgcn_readlane(f0[2], 63) == call) {
            plan = (unsigned long long)(unsigned)__builtin_amdgcn_readlane(f0[3], 63) << 32 | (unsigned)__builtin_amdgcn_readlane(f0[1], 63);
            planned = true; fin = NPC - moved(peer);
        }
        const int pl = planned ? fin : NPC - ZB;
        lim = flim < pl ? flim : pl;
    };
    auto make_plan = [&]() {
        unsigned old = 0;
        if (lane == 0) old = __hip_atomic_exchange((unsigned*)(wsl + WS_PH + 2 * NRANK * 128), call, __ATOMIC_RELAXED, __HIP_MEMORY_SCOPE_AGENT);
        if ((unsigned)__builtin_amdgcn_readfirstlane(old) == call) return;
        const unsigned hd = __builtin_amdgcn_raw_buffer_load_b32(rwl, lane < NRANK ? (unsigned)lane * 128u : OOB, hoff, AUX_SYS);
        const int t = lane >> 3, s = lane & 7;
        const unsigned zc = ld_sys((const unsigned*)(P.ws[t] + WS_ZC + (rank * NRANK + s) * 128));
        const bool ok = __all(t == rank || s == rank || s == t || (int)(zc - call * (unsigned)(ZB * PU)) >= 0);
        int h[NRANK], sum = 0, give = 0, take = 0;
#pragma unroll
        for (int j = 0; j < NRANK; ++j) { h[j] = __builtin_amdgcn_readlane(hd, j); sum += j == rank ? 0 : h[j]; }
#pragma unroll
        for (int j = 0; j < NRANK; ++j) {
            const int d = j == rank ? 0 : (sum - (NRANK - 1) * h[j]) * CFG_BAL / (8 * (NRANK - 1));
            h[j] = d > ZB ? ZB : d < -127 ? -127 : d;
            if (h[j] > 0) give += h[j]; else take -= h[j];
        }
        give = take = !ok ? 0 : give < take ? give : take;
        unsigned long long pl = 0;
#pragma unroll
        for (int j = 0; j < NRANK; ++j) {
            int d = h[j];
            if (d > 0) { d = d < give ? d : give; give -= d; } else { d = -d < take ? d : -take; take += d; }
            pl |= (unsigned long long)(d & 255) << (j * 8);
        }
        __builtin_amdgcn_raw_buffer_store_b128((u32x4){call, (unsigned)pl, call, (unsigned)(pl >> 32)}, rwl, lane < NRANK ? WS_FLAG + 63 * 32 + (unsigned)lane * FLAGB : OOB, 0, AUX_SYS);
    };
    auto offs_of = [&](int i, unsigned (&o)[4], unsigned& cnt) -> unsigned {
        if (i < NPIN) {
            int q, pos; const int pan = unit_of((i >> (6 - PSH)) + NPUSH, rank, q, pos);
            cnt = WS_CNT + (unsigned)(pan * NRANK + peer) * 128u + (unsigned)q * 4u;
            const unsigned sub = (unsigned)((i << PSH) & 63) * 1024u;
            const unsigned b = WS_PAN + (unsigned)pan * PANB + (unsigned)peer * CHB + (unsigned)q * UNB + sub + (unsigned)lane * 16u;
#pragma unroll
            for (int k = 0; k < 4; ++k) o[k] = b + k * 1024u;
            return WS_SND + (call & 1u) * SNDB + (unsigned)pos * UNB + sub;
        }
        int src = peer;
        if (ZB && i >= NPC) {
            int x = i - NPC;
            for (int j = 0; j < peer; ++j) x -= moved(j) < 0 ? moved(j) : 0;
            for (int j = 0; j < NRANK; ++j) { const int g = moved(j) > 0 ? moved(j) : 0; if ((unsigned)x < (unsigned)g) { src = j; i = NPC - g + x; } x -= g; }
        }
        const int e = (i - NPIN) / BPB, r = (i - NPIN) % BPB, pb = band_of(e, rank) * BPB + r;
        cnt = e < NRANK - 1 && r >= BPB - ZB ? WS_ZC + (unsigned)(band_of(e, rank) * NRANK + peer) * 128u : 0u;
#pragma unroll
        for (int k = 0; k < 4; ++k) {
            const unsigned sg = (unsigned)((pb << PSH) + k) * 2u + (unsigned)(lane >> 5);
            o[k] = (unsigned)src * ROWB + (unsigned)(lane & 31) * 16u + sg / NS * (NN * 2u) + sg % NS * 512u;
        }
        return WS_OSN + (unsigned)(pb << PSH) * 1024u;
    };
    while (!planned || cur < fin) {
        if (!bad) {
            const unsigned long long t0 = __builtin_amdgcn_s_memrealtime();
            const unsigned pvo = (unsigned)(lane - (first_of(cur) >> 3)) < CFG_PWIN || (lane == 63 && cur >= NPC - ZB) ? (unsigned)lane * 32u : OOB - 16u;
            for (;;) {
                const u32x4 f0 = __builtin_amdgcn_raw_buffer_load_b128(rwl, pvo, fso, AUX_SYS), f1 = __builtin_amdgcn_raw_buffer_load_b128(rwl, pvo + 16u, fso, AUX_SYS);
                update(f0, f1, cur);
                if (cur < lim || (planned && cur >= fin)) break;
                if (give_up(tmo, call, t0)) { bad = true; planned = true; plan = 0; fin = NPC; break; }
                __builtin_amdgcn_s_sleep(CFG_PS);
            }
            asm volatile("" ::: "memory");
            if (cur >= fin && planned) break;
        }
        unsigned o[4], p2 = OOB, p3 = OOB, cprev = 0;
        if (bad) {
            (void)offs_of(cur, o, cprev);
            if (cur >= NPIN) {
#pragma unroll
                for (int k = 0; k < PU; ++k) __builtin_amdgcn_raw_buffer_store_b128((u32x4){0x7fc07fc0u, 0x7fc07fc0u, 0x7fc07fc0u, 0x7fc07fc0u}, rol, o[k], 0, AUX_AGENT);
            }
            cur = index_of(draw()); continue;
        }
        STAMP1(P, 1);
        rsrc_t rp = rwl;
        u32x4 v[4] = {};
        const unsigned per = deep0 ? PACE16 / 2 : PACE16;
        unsigned slot = 0;
        if (CFG_RATE) { const unsigned now = (unsigned)__builtin_amdgcn_s_memrealtime() << 4; slot = now - (now & 0xffffffu) % PACE16 + (CFG_ALLD ? (unsigned)(((widx & 7) * 2 + (widx >> 3 & 1)) * (PW / 2) + (widx >> 4)) * (PACE16 / (PW * (CFG_ALLD ? CFG_ALLD : 1))) : (unsigned)((widx & 7) * PW + (widx >> 3)) * (PACE16 / NW)) - PACE16; }
        unsigned cb, src, in[4];
        auto prepare = [&]() {
            if (ZB && cur == NPC - CFG_PLAN) make_plan();
            src = __builtin_amdgcn_readfirstlane(offs_of(cur, o, cb)); cb = __builtin_amdgcn_readfirstlane(cb);
            if (ZB && cur >= NPC) src = 0;
#pragma unroll
            for (int k = 0; k < 4; ++k) { in[k] = ZB && cur >= NPC ? o[k] : lane * 16 + k * 1024; asm volatile("" : "+v"(o[k]), "+v"(in[k])); }
        };
        prepare();
        do {
            const bool tail = cur >= fin - CFG_TAIL, deep = deep0 || tail;
            const rsrc_t rd = cur < NPIN ? rwl : rol, rs = ZB && cur >= NPC ? ror : rwr;
            const bool look = flim < NPC && cur + CFG_LOOK >= flim, lookp = !planned && cur + CFG_LOOKP >= NPC - ZB;
            const unsigned fvo = look || (lookp && lane == 63) ? (unsigned)lane * 32u : OOB - 16u;
            unsigned cdone = cprev;
            unsigned nv = 0; u32x4 f0, f1;
#pragma unroll
            for (int k = 0; k < 4; ++k) {
                if (CFG_RATE && !tail) {
                    slot += per;
                    for (;;) {
                        const int d = (int)(((unsigned)__builtin_amdgcn_s_memrealtime() << 4) - slot);
                        if (d > CFG_CRED * 16) slot += (unsigned)(d - CFG_CRED * 16);
                        if (d >= 0) break;
                        __builtin_amdgcn_s_sleep(1);
                    }
                }
                v[k] = __builtin_amdgcn_raw_buffer_load_b128(rs, in[k], src, AUX_SYS_NT);
                if (k == 0) {
                    nv = draw();
                    f0 = __builtin_amdgcn_raw_buffer_load_b128(rwl, fvo, fso, AUX_SYS); f1 = __builtin_amdgcn_raw_buffer_load_b128(rwl, fvo + 16u, fso, AUX_SYS);
                }
                __builtin_amdgcn_raw_buffer_store_b128(v[k ^ 2], k < 2 ? rp : rd, k == 0 ? p2 : k == 1 ? p3 : o[k ^ 2], 0, AUX_AGENT);
                if (CFG_EARLY && !deep) {
                    unsigned& a = k == 0 ? p3 : o[k - 1];
                    __builtin_amdgcn_raw_buffer_store_b128(v[(k + 3) & 3], k < 1 ? rp : rd, a, 0, AUX_AGENT);
                    a = OOB;
                }
                if (k == 3) {
                    asm volatile("" ::: "memory");
                    cprev = cb; rp = rd; p2 = o[2]; p3 = o[3];
                    cur = index_of(nv);
                    if (look || lookp) update(f0, f1, cur);
                    if (cur < lim) prepare();
                }
                if (k == 0) WAIT_VMN(7); else if (k == 1) WAIT_VMN(6); else WAIT_VMN(3);
                if (!deep) {
                    if (k == 0) WAIT_VMN(4 + CFG_EARLY); else WAIT_VMN(1 + CFG_EARLY);
                    if (CFG_EARLY && k == 1) { arrive(cdone ? lane0 : OOB, cdone); cdone = 0; }
                }
            }
            arrive(cdone ? lane0 : OOB, cdone);
        } while (cur < lim);
        __builtin_amdgcn_raw_buffer_store_b128(v[2], rp, p2, 0, AUX_AGENT);
        __builtin_amdgcn_raw_buffer_store_b128(v[3], rp, p3, 0, AUX_AGENT);
        if (cprev) { WAIT_VM0(); arrive(lane0, cprev); }
    }
    STAMP(P, 4);
    asm volatile("" ::: "memory");
    if (CFG_HS && lane == 0) st_sys((unsigned*)(P.ws[peer] + WS_DONE + rank * 512) + widx, call);
    if (CFG_HS && widx == 0) {
        const unsigned* dn = (const unsigned*)(wsl + WS_DONE + peer * 512);
        const unsigned long long t0 = __builtin_amdgcn_s_memrealtime();
        for (;;) {
            bool ok = true;
            if (lane < NW) ok = (int)(ld_sys(dn + lane) - call) >= 0;
            if (NW > 64 && lane + 64 < NW) ok = ok && (int)(ld_sys(dn + 64 + lane) - call) >= 0;
            if (__all(ok)) break;
            if (give_up(tmo, call, t0)) {
                if (lane == 0) __builtin_amdgcn_raw_buffer_store_b32(0x7fc07fc0u, rol, (unsigned)(rank * NS) * 512u, 0, AUX_SYS);
                break;
            }
            __builtin_amdgcn_s_sleep(2);
        }
        STAMP(P, 5);
    }
}

__device__ __forceinline__ void pack_role(const Params& P, unsigned tk, int g, int wid, int lane) {
    const int rank = P.rank;
    const rsrc_t rx = mk_rsrc(P.x, (unsigned)MM * KSH * 2), rw = mk_rsrc(P.w, (unsigned)KSH * NN * 2);
    const int fr = lane & 15, fq = lane >> 4;
    const unsigned t0 = (unsigned)__builtin_amdgcn_s_memrealtime();
    unsigned long long wsp = (unsigned long long)P.ws[lane & (NRANK - 1)];
    constexpr int NH = CFG_UH < NUNIT ? 8 * DK : 0, UH = NH ? CFG_UH : NUNIT;
    const bool late = g >= NPK - NH;
    for (int u = late ? UH + g - (NPK - NH) : g; u < (late ? NUNIT : UH); u += late ? NH : NPK - NH) {
        int o, dest = -1;
        if (u < 4 * U0) {
            const int r = u % U0;
            if (r < NRANK * NS) { o = r >> 3; dest = (rank + 1 + r) & (NRANK - 1); } else o = r - (NRANK - 1) * NS;
            o += u / U0 * G0;
        } else o = u - 4 * (NRANK - 1) * NS;
        int q, pos; const int pan = unit_of(o, dest < 0 ? 0 : dest, q, pos);
        const bool push = o < NPUSH && dest != rank;
        if (u > CFG_PACE0) while ((int)((unsigned)__builtin_amdgcn_s_memrealtime() - t0) < (late ? CFG_TH + (u - UH) * CFG_PH : (u - CFG_PACE0) * CFG_PACE / 10)) __builtin_amdgcn_s_sleep(1);
        const int ktl = q * 2 + (wid >> 2);
        const unsigned dst = push ? WS_PAN + (unsigned)pan * PANB + (unsigned)(rank * KTS + ktl) * KTB : WS_SND + (unsigned)pos * UNB + (unsigned)(wid >> 2) * KTB;
        rsrc_t rwl; unsigned sv, c;
        auto target = [&]() {
            c = tk; asm volatile("" : "+v"(c), "+v"(wsp) :: "memory");
            const int to = push ? dest & (NRANK - 1) : rank;
            rwl = mk_rsrc((const void*)((unsigned long long)(unsigned)__builtin_amdgcn_readlane((unsigned)(wsp >> 32), to) << 32 | (unsigned)__builtin_amdgcn_readlane((unsigned)wsp, to)), WS_END);
            sv = (unsigned)lane * 16u + (push ? 0u : (~c & 1u) * SNDB);
        };
        u32x4 v[8];
        if (pan < NTM) {
            const int rb0 = (wid & 3) * 4;
            const unsigned vo = (unsigned)fr * (KSH * 2) + (unsigned)fq * 16u;
            const unsigned so = (unsigned)(pan * 256 + rb0 * 16) * (KSH * 2) + (unsigned)ktl * 128u;
#pragma unroll
            for (int i = 0; i < 8; ++i) v[i] = __builtin_amdgcn_raw_buffer_load_b128(rx, vo + (i & 1) * 64, so + (unsigned)(i >> 1) * (16 * KSH * 2), 0);
            target();
#pragma unroll
            for (int i = 0; i < 8; ++i) STORE16(v[i], rwl, sv + (i & 1) * 1024, dst + (unsigned)(rb0 + (i >> 1)) * 2048u, AUX_SYS);
        } else {
            const int ks = (wid >> 1) & 1, wr = wid & 1;
            const unsigned vo = (unsigned)(fq * 8) * (NN * 2) + (unsigned)fr * 16u;
            const unsigned so = (unsigned)(ktl * 64 + ks * 32) * (NN * 2) + (unsigned)(gpan_of(pan - NTM, dest < 0 ? 0 : dest) * 256 + wr * 128) * 2u;
#pragma unroll
            for (int j = 0; j < 8; ++j) v[j] = __builtin_amdgcn_raw_buffer_load_b128(rw, vo, so + (unsigned)j * (NN * 2), 2);
            target();
            while (CFG_P3 && push && o >= CFG_P3N && (int)((unsigned)__builtin_amdgcn_s_memrealtime() - t0) < CFG_P3 + (o - CFG_P3N) * 110) __builtin_amdgcn_s_sleep(1);
#pragma unroll
            for (int i = 0; i < 8; ++i) {
                u32x4 o;
#pragma unroll
                for (int e = 0; e < 4; ++e) o[e] = __builtin_amdgcn_perm(v[2 * e + 1][i >> 1], v[2 * e][i >> 1], (i & 1) ? 0x07060302u : 0x05040100u);
                STORE16(o, rwl, sv, dst + (unsigned)(i >> 2) * HTB + (unsigned)wr * 8192u + (unsigned)(i & 3) * 2048u + (unsigned)ks * 1024u, AUX_SYS);
            }
        }
        const bool own = lane == rank || push;
        unsigned long long fp = wsp + (own ? WS_CNT + (unsigned)((pan * NRANK + rank) * 32 + q) * 4u : WS_FLAG + (unsigned)rank * FLAGB + (unsigned)o * 4u);
        unsigned fv = own ? (c + 1u) * CNT_FULL : c + 1u;
        asm volatile("" : "+v"(fp), "+v"(fv));
        WAIT_VMN(0);
        if (u == g) STAMP(P, 6);
        __builtin_amdgcn_s_barrier();
        if (wid == 0 && lane < NRANK && (dest < 0 || dest == lane)) __hip_atomic_store((__attribute__((address_space(1))) unsigned*)fp, fv, __ATOMIC_RELAXED, __HIP_MEMORY_SCOPE_SYSTEM);
    }
}

struct Tile { int tm, v, half; unsigned sA, stA; };
__device__ __forceinline__ void tile_of(int idx, int col, int rank, Tile& t) {
    const int v = 2 * (idx >> 2) + (col >> 2);
    t.tm = 4 * (col & 3) + (idx & 3); t.v = v; t.half = -1;
    const int g = (v - GM) / GW, i = (v - GM) % GW;
    t.sA = WS_SND + (unsigned)(v < GW ? gpan_of(v, rank) : v < GW + GM ? 4 * U0 + v - GW : 4 * U0 + 4 * GM + (g - 1) * 4 * GW + i) * UNB;
    t.stA = (unsigned)(v < GW ? U0 : v < GW + GM ? GM : g <= NGF ? GW : GL) * UNB;
}
__device__ __forceinline__ void item_of(int q, int rank, Tile& t) {
    const int n = q < PFULL ? q : PFULL + ((q - PFULL) >> 1);
    if (n < 4 * PL0) tile_of(RS * KX - DK + n % PL0, n / PL0, rank, t);
    else tile_of(RS * KX - DK + (n - 4 * PL0) % PL1, 4 + (n - 4 * PL0) / PL1, rank, t);
    if (q >= PFULL) t.half = (q - PFULL) & 1;
}

template <int wr>
__device__ __forceinline__ void gemm_role(const Params& P, unsigned call, int col, int slot, LAS unsigned char* lds) {
    const int tid = threadIdx.x, wid = __builtin_amdgcn_readfirstlane(tid >> 6), lane = tid & 63, wc = wid & 3;
    const int rank = P.rank;
    char* wsl = P.wsl;
    const rsrc_t rws = mk_rsrc(wsl, WS_END);
    LAS unsigned* ctl = (LAS unsigned*)(lds + STAGE_BYTES);
    const unsigned voff = (unsigned)tid * 16u;
    const unsigned ldsw = (unsigned)wid * 1024u;
    const int aoff = wr * 8192 + lane * 16, boff = wc * 4096 + lane * 16;

    auto lane_now = [&]() { int tz = threadIdx.x; asm volatile("" : "+v"(tz)); return tz & 63; };
    auto cnt_of = [&](const Tile& t) { const int ln = lane_now(); return WS_CNT + (unsigned)((((ln & 32) ? NTM + t.v : t.tm) * NRANK + ((ln >> 2) & 7)) * 32 + (ln & 3)) * 4u; };
    auto level_of = [&](unsigned c) -> int {
        const unsigned long long miss = __ballot((int)(c - call * CNT_FULL) < 0);
        unsigned m = (unsigned)miss | (unsigned)(miss >> 32);
        const bool own = ((m >> (rank * 4)) & 15u) == 0u;
        m |= m >> 16; m |= m >> 8; m |= m >> 4;
        return own ? 1 + __builtin_ctz(m | 16u) : 0;
    };
    auto ready = [&](const Tile& t, int want) -> int {
        if (wr == 0 && wid == 0) {
            const unsigned* c = (const unsigned*)(wsl + cnt_of(t));
            unsigned* tmo = (unsigned*)(wsl + WS_TMO);
            const unsigned long long t0 = __builtin_amdgcn_s_memrealtime();
            int got;
            for (;;) {
                got = level_of(ld_sys(c));
                if (got >= want) break;
                if (give_up(tmo, call, t0)) { ctl[0] = 1u; got = 5; break; }
                __builtin_amdgcn_s_sleep(4);
            }
            ctl[2] = (unsigned)got;
            __builtin_amdgcn_fence(__ATOMIC_ACQUIRE, "agent");
            asm volatile("s_waitcnt vmcnt(0) lgkmcnt(0)" ::: "memory");
        }
        asm volatile("" ::: "memory"); __builtin_amdgcn_s_barrier(); asm volatile("" ::: "memory");
        const int r = __builtin_amdgcn_readfirstlane(ctl[2]);
        asm volatile("s_waitcnt lgkmcnt(0)" ::: "memory");
        return r;
    };

#define SA(b, h) (((b) * 2 + (h)) * HTB)
#define SB(b, h) ((4 + (b) * 2 + (h)) * HTB)
#define STAGE(bufoff, soff) do { _Pragma("unroll") for (int _i = 0; _i < 2; ++_i) \
        __builtin_amdgcn_raw_ptr_buffer_load_lds(rws, (LAS unsigned*)(lds + (bufoff) + ldsw + _i * 8192), 16, voff, (soff) + _i * 8192u, 0, 0); } while (0)
#define LDA(dst, b, h) do { _Pragma("unroll") for (int m = 0; m < 4; ++m) _Pragma("unroll") for (int k = 0; k < 2; ++k) dst[m][k] = *(const LAS bf16x8*)(lds + SA(b, h) + aoff + m * 2048 + k * 1024); } while (0)
#define LDB(dst, b, h) do { _Pragma("unroll") for (int n = 0; n < 2; ++n) _Pragma("unroll") for (int k = 0; k < 2; ++k) dst[n][k] = *(const LAS bf16x8*)(lds + SB(b, h) + boff + n * 2048 + k * 1024); } while (0)
#define MMA(ai, first, prio) do { __builtin_amdgcn_s_setprio(prio); \
        _Pragma("unroll") for (int i = 0; i < 32; ++i) { \
            const int sn = ((i >> 2) & 1) ? 3 - (i & 3) : i & 3; \
            const int bj = CFG_MO == 7 ? ((i >> 2) ^ (i >> 3)) & 1 : CFG_MO == 0 || CFG_MO == 4 ? i >> 4 : CFG_MO == 1 ? (i >> 1) & 1 : CFG_MO == 5 ? sn >> 1 : (i >> 3) & 1; \
            const int m = CFG_MO == 7 ? i >> 3 : CFG_MO == 2 ? i & 3 : CFG_MO == 6 ? sn : (i >> 2) & 3; \
            const int n = CFG_MO == 7 ? ((i >> 1) ^ (i >> 3)) & 1 : CFG_MO == 0 || CFG_MO == 4 ? (i >> 1) & 1 : CFG_MO == 1 ? i & 1 : CFG_MO == 5 ? sn & 1 : (i >> 2) & 1; \
            const int k = CFG_MO == 7 ? (i ^ (i >> 1)) & 1 : CFG_MO == 0 || CFG_MO == 4 ? i & 1 : i >> 4; \
            if (CFG_MO >= 4) SCHED; \
            acc[ai][bj][m][n] = __builtin_amdgcn_mfma_f32_16x16x32_bf16(bj ? B1[n][k] : B0[n][k], At[m][k], first && (CFG_MO == 7 ? (i & 1) == 0 : k == 0) ? (f32x4){0.f, 0.f, 0.f, 0.f} : acc[ai][bj][m][n], 0, 0, 0); } \
        __builtin_amdgcn_s_setprio(0); } while (0)
#define WAIT_L0() do { asm volatile("" ::: "memory"); __builtin_amdgcn_s_waitcnt(0xC07F); asm volatile("" ::: "memory"); } while (0)
#define WAIT_V(n) asm volatile("s_waitcnt vmcnt(" #n ")" ::: "memory")
#define WAIT_L(n) asm volatile("s_waitcnt lgkmcnt(" #n ")" ::: "memory")
#define BAR __builtin_amdgcn_s_barrier()
#define SCHED __builtin_amdgcn_sched_barrier(0)

    unsigned long long outp = (unsigned long long)P.outl;
    asm volatile("" : "+v"(outp));
    Tile cur, nxt; int rnd = slot < KX - DK ? 0 : 1;
    tile_of(rnd * (KX - DK) + slot, col, rank, cur);
    bf16x8 At[4][2], B0[2][2], B1[2][2];
    constexpr unsigned SB0 = WS_SND + ((NRANK - 1) * NS + GW) * UNB, STB = U0 * UNB;
    const unsigned kov = WS_PAN + (unsigned)((rank + 1 + (lane + 3) % 7) & (NRANK - 1)) * CHB + (unsigned)(((lane & 31) - 4) / 7) * UNB;
    unsigned tabA, tabB;
    auto tables = [&](const Tile& t) {
        const int ln = lane_now();
        const unsigned la = t.sA + (call & 1u) * SNDB + (unsigned)ln * t.stA, ra = (unsigned)(NTM + t.v) * PANB + kov;
        const unsigned lb = SB0 + (call & 1u) * SNDB + (unsigned)t.tm * UNB + (unsigned)ln * STB, rb = (unsigned)t.tm * PANB + kov;
        tabA = ln < 4 ? la : ra;
        tabB = (ln < 4 ? lb : rb) + (t.half == 1 ? HTB : 0u);
    };
    auto pick = [&]() {
        Tile t = cur; bool ok = true;
        if (rnd + 1 < RS) tile_of((rnd + 1) * KX - DK + slot, col, rank, t);
        else if (CFG_AFF) {
            const unsigned qb = WS_TQ + (call & 1u) * 2048u;
            int c = col;
            for (;;) {
                const int ln = lane_now();
                const unsigned d = (unsigned)__builtin_amdgcn_raw_ptr_buffer_atomic_add_i32(1, rws, ln == 0 ? 0u : 0xfffffff0u, qb + (unsigned)c * 128u, 0);
                const unsigned all = __builtin_amdgcn_raw_buffer_load_b32(rws, ln < 8 ? (unsigned)ln * 128u : 0xfffffff0u, qb, AUX_SYS);
                const int q = __builtin_amdgcn_readfirstlane(d);
                if (c == 8) {
                    ok = q < 16 * HC;
                    if (ok) { const int n = q >> 1, hc = n / HC; tile_of((hc < 4 ? NT0 : NT1) - HC + n % HC, hc, rank, t); t.half = q & 1; }
                    break;
                }
                if (q < (c < 4 ? PL0 : PL1) - HC) { tile_of(RS * KX - DK + q, c, rank, t); break; }
                int best = 0; c = 8;
                for (int i = 0; i < 8; ++i) {
                    const int left = (i < 4 ? PL0 : PL1) - HC - __builtin_amdgcn_readlane(all, i);
                    if (left > 0 && left + (i == (col ^ 4) ? CFG_AFB : 0) > best) { best = left + (i == (col ^ 4) ? CFG_AFB : 0); c = i; }
                }
            }
        } else {
            const unsigned d = (unsigned)__builtin_amdgcn_raw_ptr_buffer_atomic_add_i32(1, rws, lane_now() == 0 ? 0u : 0xfffffff0u, WS_HEAD + (unsigned)rank * 128u, 0);
            const unsigned q = (unsigned)__builtin_amdgcn_readfirstlane(d) - (call - 1u) * (unsigned)(NITEM + 8 * KX);
            ok = q < (unsigned)NITEM;
            if (ok) item_of((int)q, rank, t);
        }
        *(LAS u32x4*)(ctl + 4) = (u32x4){ok ? (unsigned)(t.tm | t.v << 8 | (t.half + 2) << 16) : 0u, t.sA, t.stA, 0u};
        __builtin_amdgcn_raw_ptr_buffer_load_lds(rws, ctl + 64, 4, cnt_of(t), 0, 0, AUX_SYS);
    };
    auto picked = [&](int& level) -> bool {
        const u32x4 f = *(const LAS u32x4*)(ctl + 4);
        level = level_of(ctl[64 + lane_now()]);
        const unsigned w = __builtin_amdgcn_readfirstlane(f[0]);
        nxt.tm = w & 255u; nxt.v = (w >> 8) & 255u; nxt.half = (int)(w >> 16) - 2;
        nxt.sA = __builtin_amdgcn_readfirstlane(f[1]); nxt.stA = __builtin_amdgcn_readfirstlane(f[2]);
        return w != 0u;
    };
    constexpr int JP = KT / 2 - 4, JI = KT / 2 - 2, JL = KT / 2 - 1;
    const int jend = wr == 0 && wid == 0 ? JP : JL;
    if (CFG_AFF && wr == 0 && wid == 0 && slot < 2) __builtin_amdgcn_raw_buffer_store_b32(0u, rws, lane == 0 ? 0u : 0xfffffff0u, WS_TQ + (~call & 1u) * 2048u + (unsigned)(slot * 8 + col) * 128u, AUX_SYS);
    if (CFG_SYNC && rnd && wr == 0 && wid == 0) {
        const unsigned long long t0 = __builtin_amdgcn_s_memrealtime();
        while (__builtin_amdgcn_readfirstlane(ld_agent((const unsigned*)(wsl + WS_COL + col * 128))) != call && !give_up((unsigned*)(wsl + WS_TMO), call, t0)) __builtin_amdgcn_s_sleep(16);
    }
    int lvl = ready(cur, 1);
    STAMP(P, 2);
    tables(cur);
    unsigned a2 = cur.sA + (call & 1u) * SNDB, b2 = SB0 + (call & 1u) * SNDB + (unsigned)cur.tm * UNB;
    STAGE(SB(0, 0), b2); STAGE(SA(0, 0), a2); STAGE(SB(0, 1), b2 + HTB); STAGE(SA(0, 1), a2 + HTB);
    STAGE(SB(1, 0), b2 + KTB); STAGE(SA(1, 0), a2 + KTB); STAGE(SB(1, 1), b2 + HTB + KTB);
    WAIT_V(0); BAR;
#define TICK(i, r) do {} while (0)
#define ROUND_DONE() do {} while (0)
    bool has_next = false;
#pragma unroll 1
    for (;;) {
        f32x4 acc[2][2][4][2];
        int jchk = lvl < 5 ? 7 * lvl - 4 : jend;
        TICK(0, 2); TICK(4, 3);
#define SYNC_MMA(ai, first, wv) do { if (wv && (wr == 1 || !CFG_LW)) WAIT_V(8); WAIT_L0(); if (wr == 1) BAR; MMA(ai, first, wr + 1); if (wv && wr == 0 && CFG_LW) WAIT_V(8); if (wr == 0) BAR; SCHED; } while (0)
#define PASS(first, check) do { \
            LDB(B0, 0, 0); LDA(At, 0, 0); LDB(B1, 0, 1); STAGE(SA(1, 1), a2 + KTB + HTB); SCHED; \
            check; \
            a2 = __builtin_amdgcn_readlane(tabA, (j + 1) & 31); b2 = __builtin_amdgcn_readlane(tabB, (j + 1) & 31); \
            SYNC_MMA(0, first, !first); \
            LDA(At, 0, 1); STAGE(SB(0, 0), b2); STAGE(SA(0, 0), a2); STAGE(SB(0, 1), b2 + HTB); \
            SYNC_MMA(1, first, !first); \
            LDB(B0, 1, 0); LDA(At, 1, 0); LDB(B1, 1, 1); STAGE(SA(0, 1), a2 + HTB); \
            SYNC_MMA(0, false, true); \
            LDA(At, 1, 1); STAGE(SB(1, 0), b2 + KTB); STAGE(SA(1, 0), a2 + KTB); STAGE(SB(1, 1), b2 + KTB + HTB); \
            SYNC_MMA(1, false, true); } while (0)
        int j = 0;
        PASS(true, );
#pragma unroll 1
        for (j = 1; j < KT / 2; ++j)
            PASS(false, if (__builtin_expect(j == jchk, 0)) {
                    if (j < JP) { lvl = ready(cur, lvl + 1); jchk = lvl < 5 ? 7 * lvl - 4 : jend; }
                    else if (wr == 0 && j == JP) { pick(); jchk = JI; }
                    else if (wr == 0 && j == JI) { asm volatile("buffer_inv sc1" ::: "memory"); jchk = JL; }
                    else {
                        has_next = picked(lvl);
                        if (has_next) {
                            const int want = nxt.half < 0 ? 1 : 5;
                            if (lvl < want) lvl = ready(nxt, want);
                            tables(nxt);
                        }
                    }
                });
        if (has_next && nxt.half >= 0) STAGE(SA(1, 1), a2 + KTB + HTB);
        TICK(1, 2);
        TICK(2, 2);
        {
            const int ln = lane_now(), fr = ln & 15, fq = ln >> 4;
            const rsrc_t rol = mk_rsrc((const void*)outp, OUTB);
            const bool shared = cur.v < NS;
            if (__builtin_amdgcn_readfirstlane(ctl[0]) != 0u) {
#pragma unroll
                for (int i = 0; i < 32; ++i) acc[i >> 4][(i >> 3) & 1][(i >> 1) & 3][i & 1] = __builtin_bit_cast(f32x4, (u32x4){0x7fc00000u, 0x7fc00000u, 0x7fc00000u, 0x7fc00000u});
            }
            if (rnd == 0 && cur.v < GW) {
                const unsigned long long t0 = __builtin_amdgcn_s_memrealtime();
                while (__builtin_amdgcn_s_memrealtime() - t0 < (unsigned long long)((col & 3) * CFG_STAG + (col >> 2) * CFG_STAGB)) __builtin_amdgcn_s_sleep(8);
                if (DK && CFG_SYNC && wr == 0 && slot == 0 && wid == 0) __builtin_amdgcn_raw_buffer_store_b32(call, rws, ln == 0 ? 0u : 0xfffffff0u, WS_COL + (unsigned)col * 128u, AUX_AGENT);
            }
            const bool thru = rnd >= RS;
            const unsigned vo = (unsigned)(wc * 32 + fq * 4) * (NN * 2) + (unsigned)(wr * 256 + fr * 16);
            const unsigned so = (unsigned)(cur.tm * 256) * (NN * 2) + (unsigned)gpan_of(cur.v, rank) * 512u;
            const unsigned vs = (unsigned)(wc * 32 + fq * 4) * ROWB + (unsigned)(wr * 256 + fr * 16);
            const unsigned ss = WS_OSN + (unsigned)(cur.tm * 256) * ROWB + (unsigned)cur.v * 512u;
            auto finish = [&](auto mode) {
#pragma unroll
                for (int bj = 0; bj < 2; ++bj)
#pragma unroll
                    for (int n = 0; n < 2; ++n)
#pragma unroll
                        for (int r = 0; r < 4; ++r) {
                            u32x4 o;
                            if ((r & 1) == 0) {
#pragma unroll
                                for (int i = 0; i < 8; ++i) { f32x4& a = acc[i >> 2][bj][i & 3][n]; const f32x2 z = silu2((f32x2){a[r], a[r + 1]}); a[r] = z[0]; a[r + 1] = z[1]; }
                            }
#pragma unroll
                            for (int i = 0; i < 4; ++i) o[i] = cvt_pk_bf16(acc[i >> 1][bj][(i & 1) * 2][n][r], acc[i >> 1][bj][(i & 1) * 2 + 1][n][r]);
                            if (mode == 1) STORE16(o, rws, vs, ss + (unsigned)(bj * 128 + n * 16 + r) * ROWB, AUX_SYS);
                            STORE16(o, rol, vo, so + (unsigned)(bj * 128 + n * 16 + r) * (NN * 2), mode == 2 ? AUX_SYS : CFG_RST);
                            if (mode == 2 && CFG_LAST > 1) { WAIT_VM0(); __builtin_amdgcn_s_sleep(CFG_LAST > 1 ? CFG_LAST - 1 : 1); }
                        }
            };
            if (shared) finish(std::integral_constant<int, 1>());
            else if (thru) finish(std::integral_constant<int, 2>());
            else finish(std::integral_constant<int, 0>());
            WAIT_V(16);
            if (shared) {
                WAIT_VM0();
                unsigned old = 0;
                if (ln == 0) old = __hip_atomic_fetch_add(ctl + 1, 1u, __ATOMIC_RELAXED, __HIP_MEMORY_SCOPE_WORKGROUP);
                old = __builtin_amdgcn_readfirstlane(old);
                if ((old & 7u) == 7u && ln < NRANK && ln != rank)
                    st_sys((unsigned*)(P.ws[ln] + WS_FLAG + rank * FLAGB) + NIN * 4 + turn_of(cur.tm, ln) * NS + cur.v, call);
            }
        }
        TICK(3, 2);
        ROUND_DONE();
        if (!has_next) break;
        cur = nxt; ++rnd;
        if (cur.half >= 0) break;
    }
#define HB(b, i) ((b) < 2 ? ((i) < 2 ? SA(b, i) : SB(b, 0)) : (i) < 2 ? SB(i, 1) : 8 * HTB)
#define HSTAGE(b, t) do { const unsigned _o = ((t) & 1) * KTB, _a = __builtin_amdgcn_readlane(tabA, ((t) >> 1) & 31) + _o, _b = __builtin_amdgcn_readlane(tabB, ((t) >> 1) & 31) + _o; \
        STAGE(HB(b, 0), _a); STAGE(HB(b, 1), _a + HTB); STAGE(HB(b, 2), _b); } while (0)
#define HLOAD(b) do { _Pragma("unroll") for (int i = 0; i < 4; ++i) B0[i >> 1][i & 1] = *(const LAS bf16x8*)(lds + HB(b, 2) + boff + i * 1024); \
        _Pragma("unroll") for (int i = 0; i < 8; ++i) At[i >> 1][i & 1] = *(const LAS bf16x8*)(lds + HB(b, 0) + aoff + i * 1024); \
        _Pragma("unroll") for (int i = 0; i < 8; ++i) Au[i >> 1][i & 1] = *(const LAS bf16x8*)(lds + HB(b, 1) + aoff + i * 1024); } while (0)
#define HPASS(b, t, first) do { \
        if (wr == 0) { HLOAD(b); HSTAGE((b + 2) % 3, (t) + 2); } else { HSTAGE((b + 2) % 3, (t) + 2); SCHED; HLOAD(b); if (!first) WAIT_V(6); } \
        WAIT_L0(); if (wr == 1) BAR; \
        __builtin_amdgcn_s_setprio(wr + 1); \
        _Pragma("unroll") for (int i = 0; i < 32; ++i) { const int ai = i >> 4, m = (i >> 2) & 3, n = (i >> 1) & 1, k = i & 1; \
            acc[ai][m][n] = __builtin_amdgcn_mfma_f32_16x16x32_bf16(B0[n][k], ai ? Au[m][k] : At[m][k], first && k == 0 ? (f32x4){0.f, 0.f, 0.f, 0.f} : acc[ai][m][n], 0, 0, 0); } \
        __builtin_amdgcn_s_setprio(0); \
        if (wr == 0) { if (!first) WAIT_V(6); BAR; } SCHED; } while (0)
#pragma unroll 1
    while (PFULL < PN && has_next) {
        f32x4 acc[2][4][2];
        bf16x8 Au[4][2];
        BAR;
        HPASS(0, 0, true);
#pragma unroll 1
        for (int t = 1; t < KT; t += 3) {
            if (wr == 0 && wid == 0 && t == KT - 9) pick();
            HPASS(1, t, false); HPASS(2, t + 1, false); HPASS(0, t + 2, false);
        }
        has_next = picked(lvl);
        if (has_next) {
            (void)ready(nxt, 5);
            tables(nxt);
            HSTAGE(0, 0); HSTAGE(1, 1);
        }
        const int ln = lane_now(), fr = ln & 15, fq = ln >> 4;
        const rsrc_t rol = mk_rsrc((const void*)outp, OUTB);
        if (__builtin_amdgcn_readfirstlane(ctl[0]) != 0u) {
#pragma unroll
            for (int i = 0; i < 16; ++i) acc[i >> 3][(i >> 1) & 3][i & 1] = __builtin_bit_cast(f32x4, (u32x4){0x7fc00000u, 0x7fc00000u, 0x7fc00000u, 0x7fc00000u});
        }
        const unsigned vo = (unsigned)(wc * 32 + fq * 4) * (NN * 2) + (unsigned)(wr * 256 + fr * 16);
        const unsigned so = (unsigned)(cur.tm * 256 + cur.half * 128) * (NN * 2) + (unsigned)gpan_of(cur.v, rank) * 512u;
#pragma unroll
        for (int n = 0; n < 2; ++n)
#pragma unroll
            for (int r = 0; r < 4; ++r) {
                u32x4 o;
                if ((r & 1) == 0) {
#pragma unroll
                    for (int i = 0; i < 8; ++i) { f32x4& a = acc[i >> 2][i & 3][n]; const f32x2 z = silu2((f32x2){a[r], a[r + 1]}); a[r] = z[0]; a[r + 1] = z[1]; }
                }
#pragma unroll
                for (int i = 0; i < 4; ++i) o[i] = cvt_pk_bf16(acc[i >> 1][(i & 1) * 2][n][r], acc[i >> 1][(i & 1) * 2 + 1][n][r]);
                STORE16(o, rol, vo, so + (unsigned)(n * 16 + r) * (NN * 2), AUX_SYS);
            }
        WAIT_V(8);
        ROUND_DONE();
        cur = nxt; ++rnd;
    }
    WAIT_V(0);
}

__global__ __launch_bounds__(512, 2) void fused_gemm_ar(Params P) {
    extern __shared__ __attribute__((aligned(16))) unsigned char shm[];
    LAS unsigned char* lds = (LAS unsigned char*)shm;
    const int wid = __builtin_amdgcn_readfirstlane(threadIdx.x >> 6), lane = threadIdx.x & 63;
    const int b = __builtin_amdgcn_readfirstlane(blockIdx.x);
    asm volatile("" :: "s"(P.x), "s"(P.w), "s"(P.wsl), "s"(P.rank));
    unsigned* tkp = (unsigned*)(P.wsl + WS_TICKET + b * 128 + wid * 16);
    const unsigned tk = ld_agent(tkp);
    if (b < NPULL) {
        const unsigned call = __builtin_amdgcn_readfirstlane(tk) + 1u;
        st_agent(tkp, call);
        const int pi = b / PW, slot = b % PW;
        const int peer = (P.rank + 1 + pi) & (NRANK - 1);
        pull_role(P, call, peer, slot * 8 + wid, lane);
    } else {
        const int g = b - NPULL;
        ((LAS unsigned*)(lds + STAGE_BYTES))[lane & 3] = 0u;
        STAMP(P, 0);
        pack_role(P, tk, g, wid, lane);
        const unsigned call = __builtin_amdgcn_readfirstlane(tk) + 1u;
        st_agent(tkp, call);
        STAMP(P, 1);
        asm volatile("s_waitcnt lgkmcnt(0)" ::: "memory");
        __builtin_amdgcn_s_barrier();
        if (b >= NPS) {
            const int col = (CFG_SWAP >> (b & 7) * 4) & 7;
            if (wid < 4) gemm_role<0>(P, call, col, (b - NPS) >> 3, lds);
            else gemm_role<1>(P, call, col, (b - NPS) >> 3, lds);
        }
    }
}
}

extern "C" void kernel_launch(int rank, int nranks, ncclComm_t comm,
                              hipStream_t stream,
                              void* const* d_in, const int* in_sizes, int n_in,
                              void* d_out, int out_size,
                              const int* in_dtypes, int out_dtype,
                              void* const* d_out_all,
                              void* const* d_ws_all, size_t ws_bytes) {
    if (nranks != NRANK || (unsigned)rank >= (unsigned)NRANK || ws_bytes < (size_t)WS_END || n_in < 2 ||
        in_sizes[0] != MM * KSH || in_sizes[1] != KSH * NN || out_size != MM * NN) {
        fprintf(stderr, "unexpected problem geometry\n");
        return;
    }
    static bool attr_done[NRANK] = {};
    if (!attr_done[rank]) {
        HIP_CHECK(hipFuncSetAttribute((const void*)fused_gemm_ar, hipFuncAttributeMaxDynamicSharedMemorySize, LDS_BYTES));
        attr_done[rank] = true;
    }
    Params P = {};
    P.x = d_in[0]; P.w = d_in[1]; P.rank = rank; P.wsl = (char*)d_ws_all[rank]; P.outl = (char*)d_out_all[rank];
    for (int p = 0; p < NRANK; ++p) { P.out[p] = (char*)d_out_all[p]; P.ws[p] = (char*)d_ws_all[p]; }
    hipLaunchKernelGGL(fused_gemm_ar, dim3(GRID), dim3(512), LDS_BYTES, stream, P);
    HIP_CHECK(hipGetLastError());
}
```

```cpp
#include <hip/hip_runtime.h>
#include <rccl/rccl.h>
#include <cstdio>
#include <type_traits>

#define HIP_CHECK(expr) do { hipError_t _e = (expr); if (_e != hipSuccess) { \
    fprintf(stderr, "HIP error %s at %s:%d\n", hipGetErrorString(_e), __FILE__, __LINE__); \
    return; } } while (0)

#define LAS __attribute__((address_space(3)))
typedef short bf16x8 __attribute__((ext_vector_type(8)));
typedef float f32x4 __attribute__((ext_vector_type(4)));
typedef unsigned u32x4 __attribute__((ext_vector_type(4)));
typedef __amdgpu_buffer_rsrc_t rsrc_t;

namespace {
constexpr int NRANK = 8;
constexpr int MM = 4096, KK = 4096, NN = 32768, KSH = KK / NRANK;
constexpr int KT = KK / 64, KTS = KSH / 64, NTM = MM / 256, NPAN = NN / 256;
#define CFG_NS 6
#define CFG_PW 8
constexpr int NS = CFG_NS;
constexpr int NRP = NPAN - NRANK * NS;
constexpr int NWP = NS + NRP;
constexpr int NIN = NTM + NWP;
constexpr int NOUT = NS * NTM;
constexpr int NFLAG = NIN * 4 + NOUT;
constexpr int NUNIT = NTM * 4 + NPAN * 4;
constexpr int GRID = 256;
constexpr int PW = CFG_PW;
constexpr int NPULL = PW * (NRANK - 1);
constexpr int NW = PW * 8;
constexpr int NPK = GRID - NPULL;
constexpr int NPS = (NPULL + 7) / 8 * 8;
constexpr int KX = (GRID - NPS) / 8;
#define CFG_GW (2 * (KX / 4))
constexpr int GW = CFG_GW;
constexpr int G0 = GW + NTM;
#define CFG_SKIP 1
#define CFG_SYNC 1
#define CFG_GM (CFG_SKIP ? GW + 2 : 2)
constexpr int GM = CFG_GM;
constexpr int NGF = (NWP - GW - GM) / GW, GL = (NWP - GW - GM) % GW;
static_assert(NS <= GW && NWP > GW, "shared panels sit in the first group");
constexpr int PU = 4, PSH = 2;
#define CFG_PUSH 2
constexpr int NPUSH = CFG_PUSH;
constexpr int NPIN = (NIN * 4 - NPUSH) * 64 / PU;
static_assert(NPUSH <= NS, "pushed units have a single consumer");
constexpr int NPC = NPIN + NOUT * 128 / PU;
constexpr unsigned CNT_FULL = 64;

constexpr int NT0 = 4 * ((NWP + 1) / 2), NT1 = 4 * (NWP / 2);
constexpr int DK = CFG_SKIP ? KX - 2 * GW : 0;
constexpr int RS = (NT1 - 1 + DK) / KX;
constexpr int PL0 = NT0 - RS * KX + DK, PL1 = NT1 - RS * KX + DK, PN = 4 * (PL0 + PL1);
#define CFG_SWAP 0x67542310u
#define CFG_FD (CFG_SKIP ? 0 : -8)
constexpr int PFULL = 2 * PN - 8 * KX - CFG_FD < 0 ? 0 : 2 * PN - 8 * KX - CFG_FD > PN ? PN : 2 * PN - 8 * KX - CFG_FD;
constexpr int NITEM = PFULL + 2 * (PN - PFULL);
constexpr unsigned KTB = 32768, HTB = 16384;
constexpr unsigned PANB = KT * KTB;
constexpr unsigned CHB = KTS * KTB;
constexpr unsigned UNB = 2 * KTB;
constexpr unsigned WS_TICKET = 0;
constexpr unsigned WS_TMO = WS_TICKET + GRID * 128;
constexpr unsigned WS_FLAG = WS_TMO + 128;
constexpr unsigned FLAGB = 4096;
constexpr unsigned WS_LEAD = WS_FLAG + NRANK * FLAGB;
constexpr unsigned WS_HEAD = WS_LEAD + NRANK * 512;
constexpr unsigned WS_COL = WS_HEAD + NRANK * 128;
#define CFG_PHOFF 0
constexpr unsigned WS_PH = WS_COL + 8 * 128 + CFG_PHOFF;
constexpr unsigned WS_ZC = WS_PH + 2 * NRANK * 128 + 128;
constexpr unsigned WS_CNT = WS_ZC + NRANK * NRANK * 128;
constexpr unsigned WS_CTRL_END = WS_CNT + NIN * NRANK * 128;
constexpr unsigned WS_DBG = 512u << 10;
constexpr unsigned WS_SND = 1u << 20, SNDB = NUNIT * UNB;
constexpr unsigned ROWB = NS * 512;
constexpr unsigned WS_OSN = WS_SND + NUNIT * UNB + SNDB;
constexpr unsigned WS_PAN = (WS_OSN + MM * ROWB + PANB - 1) / PANB * PANB;
constexpr unsigned WS_END = WS_PAN + NIN * PANB;
constexpr int U0 = G0 + (NRANK - 1) * NS;
constexpr unsigned OUTB = (unsigned)MM * NN * 2;
static_assert(WS_CTRL_END <= WS_DBG, "ctrl region");
static_assert(NFLAG <= 512 && 2048 <= (int)FLAGB, "flag region");
#define CFG_DEEP 5
constexpr unsigned STAGE_BYTES = 9 * HTB;
constexpr unsigned LDS_BYTES = STAGE_BYTES + 512;
constexpr unsigned long long DEADLINE = 50000000ull;
constexpr int AUX_SYS = 17, AUX_SYS_NT = 19, AUX_AGENT = 16;
#define CFG_PACE 140
#define CFG_UH (2 * (NPK - 8 * DK))
#define CFG_TH 5000
#define CFG_PH 40
#define CFG_PWND 3
#define CFG_EH 1
#define CFG_FK 2
#define CFG_MIX 1
static_assert(4 * NRANK * NS <= NPK && (NS - NPUSH) * 64 / PU <= NW && NS * 64 <= 512, "leading units");
#define CFG_PACE0 40
#define CFG_STAG 600
#define CFG_P3 320
#define CFG_P3N 2
#define CFG_STAGB 300
#define CFG_LOOK 256
#define CFG_TAIL NW
#define CFG_EVEN 200
#define CFG_EVEND 140
#define CFG_ERATE CFG_RATE
#define CFG_ZB 120
#define CFG_LOOKP 100
#define CFG_PLAN (CFG_ZB + CFG_LOOKP + 50)
#define CFG_BAL 8
constexpr int ZB = CFG_ZB;
static_assert(ZB <= 127 && ZB <= NS * 128 / PU && NFLAG <= 504, "plan");
#define CFG_PWIN 4u
#define CFG_PS 2
#define CFG_RATE 1680
#define CFG_CRED 25
#define CFG_GRID 36
#define CFG_SPIN 8
#define CFG_EARLY 1
constexpr unsigned PACE16 = CFG_RATE * PW * (8 + CFG_DEEP) / 64;

struct Params {
    const void* x; const void* w;
    char* wsl; char* outl;
    int rank, pad[7];
    char* ws[NRANK]; char* out[NRANK];
};

__device__ __forceinline__ rsrc_t mk_rsrc(const void* p, unsigned bytes) {
    const unsigned long long v = (unsigned long long)p;
    const unsigned lo = __builtin_amdgcn_readfirstlane((unsigned)v), hi = __builtin_amdgcn_readfirstlane((unsigned)(v >> 32));
    return __builtin_amdgcn_make_buffer_rsrc((void*)(((unsigned long long)hi << 32) | lo), 0, bytes, 0x00020000);
}
__device__ __forceinline__ unsigned ld_sys(const unsigned* p) { return __hip_atomic_load(p, __ATOMIC_RELAXED, __HIP_MEMORY_SCOPE_SYSTEM); }
__device__ __forceinline__ unsigned ld_agent(const unsigned* p) { return __hip_atomic_load(p, __ATOMIC_RELAXED, __HIP_MEMORY_SCOPE_AGENT); }
__device__ __forceinline__ void st_sys(unsigned* p, unsigned v) { __hip_atomic_store(p, v, __ATOMIC_RELAXED, __HIP_MEMORY_SCOPE_SYSTEM); }
__device__ __forceinline__ void st_agent(unsigned* p, unsigned v) { __hip_atomic_store(p, v, __ATOMIC_RELAXED, __HIP_MEMORY_SCOPE_AGENT); }
__device__ __forceinline__ void add_agent(unsigned* p, unsigned v) { (void)__hip_atomic_fetch_add(p, v, __ATOMIC_RELAXED, __HIP_MEMORY_SCOPE_AGENT); }
#define STORE16(v, rs, vo, so, aux) do { const u32x4 _d = (v); __builtin_amdgcn_raw_buffer_store_b128(_d, rs, vo, so, aux); asm volatile("s_nop 1" :: "v"(_d) : "memory"); } while (0)
#define WAIT_VM0() asm volatile("s_waitcnt vmcnt(0)" ::: "memory")
#define WAIT_VMN(n) do { asm volatile("" ::: "memory"); __builtin_amdgcn_s_waitcnt(0x0F70 | (n)); asm volatile("" ::: "memory"); } while (0)

__device__ __forceinline__ bool give_up(unsigned* tmo, unsigned call, unsigned long long t0) {
    const unsigned long long dt = __builtin_amdgcn_s_memrealtime() - t0;
    if (dt < 2000ull) return false;
    if ((unsigned)__builtin_amdgcn_readfirstlane(ld_agent(tmo)) == call) return true;
    if (dt > DEADLINE) { st_agent(tmo, call); return true; }
    return false;
}

__device__ __forceinline__ int gpan_of(int v, int consumer) { return v < NS ? consumer * NS + v : (NRANK - 1) * NS + v; }
__device__ __forceinline__ int band_of(int e, int consumer) { return e < 7 ? (consumer + 1 + e) & 7 : e < 15 ? 8 + ((consumer + e) & 7) : consumer; }
__device__ __forceinline__ int turn_of(int band, int consumer) { return band >= 8 ? 7 + ((band - consumer + 1) & 7) : band == consumer ? 15 : (band - consumer - 1) & 7; }
__device__ __forceinline__ int unit_of(int o, int consumer, int& q, int& pos) {
    if (o < 4 * G0) { q = o / G0; const int m = o % G0; pos = q * U0 + gpan_of(m, consumer); return m < GW ? NTM + m : m - GW; }
    pos = o + 4 * (NRANK - 1) * NS;
    o -= 4 * G0;
    if (o < 4 * GM) { q = o / (GM ? GM : 1); return NTM + GW + o % (GM ? GM : 1); }
    o -= 4 * GM;
    const int g = o / (4 * GW), r = o % (4 * GW);
    int i;
    if (GL == 0 || g < NGF) { q = r / GW; i = r % GW; } else { q = r / (GL ? GL : 1); i = r % (GL ? GL : 1); }
    return NTM + GM + GW * (g + 1) + i;
}

__device__ __forceinline__ unsigned cvt_pk_bf16(float lo, float hi) { unsigned r; asm("v_cvt_pk_bf16_f32 %0, %1, %2" : "=v"(r) : "v"(lo), "v"(hi)); return r; }
__device__ __forceinline__ float silu(float y) {
    const float c = __builtin_amdgcn_fmed3f(y, -60.0f, 60.0f);
    const float e = __builtin_amdgcn_exp2f(c * -1.44269504088896f);
    return y * __builtin_amdgcn_rcpf(1.0f + e);
}

typedef float f32x2 __attribute__((ext_vector_type(2)));
__device__ __forceinline__ f32x2 silu2(f32x2 y) {
    const f32x2 t = __builtin_elementwise_fma(y, (f32x2){-1.44269504088896f, -1.44269504088896f}, (f32x2){-86.5617024533378f, -86.5617024533378f});
    const f32x2 e = {__builtin_amdgcn_fmed3f(__builtin_amdgcn_exp2f(t[0]), 0.0f, 1.0f), __builtin_amdgcn_fmed3f(__builtin_amdgcn_exp2f(t[1]), 0.0f, 1.0f)};
    const f32x2 d = __builtin_elementwise_fma(e, (f32x2){1.1420073898156842e26f, 1.1420073898156842e26f}, (f32x2){1.0f, 1.0f});
    return y * (f32x2){__builtin_amdgcn_rcpf(d[0]), __builtin_amdgcn_rcpf(d[1])};
}

#define STAMP1(P, slot) do {} while (0)
#define STAMP(P, slot) do {} while (0)

__device__ __forceinline__ void pull_role(const Params& P, unsigned call, int peer, int widx, int lane) {
    const int rank = P.rank;
    char* wsl = P.wsl;
    const rsrc_t rwr = mk_rsrc(P.ws[peer], WS_END), rwl = mk_rsrc(wsl, WS_END);
    const rsrc_t rol = mk_rsrc(P.outl, OUTB), ror = mk_rsrc(P.out[peer], OUTB);
    unsigned* tmo = (unsigned*)(wsl + WS_TMO);
    constexpr unsigned OOB = 0xfffffff0u;
    const unsigned lane0 = lane == 0 ? 0u : OOB;
    const unsigned hoff = WS_PH + (call & 1u) * (NRANK * 128u);
    auto draw = [&]() { return (unsigned)__builtin_amdgcn_raw_ptr_buffer_atomic_add_i32(1, rwl, lane0, hoff + (unsigned)peer * 128u, 0); };
    if (widx == 0) __builtin_amdgcn_raw_buffer_store_b32(0u, rwl, lane0, WS_PH + (~call & 1u) * (NRANK * 128u) + (unsigned)peer * 128u, AUX_SYS);
    auto arrive = [&](unsigned vo, unsigned cnt) { (void)__builtin_amdgcn_raw_ptr_buffer_atomic_add_i32(PU, rwl, vo, cnt, 0); };
    const bool deep0 = (widx & 7) < CFG_DEEP;
    bool bad = false; int stamped = 0; (void)stamped;
    auto index_of = [&](unsigned v) { return (int)__builtin_amdgcn_readfirstlane(v) + NW; };
    int cur = index_of((unsigned)(widx - NW));
    constexpr int BPB = NS * 128 / PU;
    const unsigned fso = WS_FLAG + (unsigned)peer * FLAGB;
    auto first_of = [&](int i) { return i < NPIN ? (i >> (6 - PSH)) + NPUSH : NIN * 4 + (i - NPIN) / BPB * NS; };
    int lim = 0, flim = 0, fin = NPC;
    bool planned = ZB == 0;
    unsigned long long plan = 0;
    auto moved = [&](int j) { return (int)(signed char)(plan >> (j * 8)); };
    auto update = [&](const u32x4& f0, const u32x4& f1, int i) {
        const int first = first_of(i);
        unsigned un = 0;
#pragma unroll
        for (int j = 0; j < 8; ++j) un |= ((int)((j < 4 ? f0[j & 3] : f1[j & 3]) - call) < 0 && lane * 8 + j >= first ? 1u : 0u) << j;
        const int l = __builtin_ctzll(__ballot(un != 0u) | (1ull << 63));
        const int u = l * 8 + __builtin_ctz(__builtin_amdgcn_readlane(un, l) | 256u);
        const int fl = u <= NIN * 4 ? (u - NPUSH) << (6 - PSH) : u >= NFLAG ? NPC + 128 : NPIN + (u - NIN * 4) / NS * BPB;
        flim = fl > flim ? fl : flim;
        if (!planned && (unsigned)__builtin_amdgcn_readlane(f0[0], 63) == call && (unsigned)__builtin_amdgcn_readlane(f0[2], 63) == call) {
            plan = (unsigned long long)(unsigned)__builtin_amdgcn_readlane(f0[3], 63) << 32 | (unsigned)__builtin_amdgcn_readlane(f0[1], 63);
            planned = true; fin = NPC - moved(peer);
        }
        const int pl = planned ? fin : NPC - ZB;
        lim = flim < pl ? flim : pl;
    };
    auto make_plan = [&]() {
        unsigned old = 0;
        if (lane == 0) old = __hip_atomic_exchange((unsigned*)(wsl + WS_PH + 2 * NRANK * 128), call, __ATOMIC_RELAXED, __HIP_MEMORY_SCOPE_AGENT);
        if ((unsigned)__builtin_amdgcn_readfirstlane(old) == call) return;
        const unsigned hd = __builtin_amdgcn_raw_buffer_load_b32(rwl, lane < NRANK ? (unsigned)lane * 128u : OOB, hoff, AUX_SYS);
        const int t = lane >> 3, s = lane & 7;
        const unsigned zc = ld_sys((const unsigned*)(P.ws[t] + WS_ZC + (rank * NRANK + s) * 128));
        const bool ok = __all(t == rank || s == rank || s == t || (int)(zc - call * (unsigned)(ZB * PU)) >= 0);
        int h[NRANK], sum = 0, give = 0, take = 0;
#pragma unroll
        for (int j = 0; j < NRANK; ++j) { h[j] = __builtin_amdgcn_readlane(hd, j); sum += j == rank ? 0 : h[j]; }
#pragma unroll
        for (int j = 0; j < NRANK; ++j) {
            const int d = j == rank ? 0 : (sum - (NRANK - 1) * h[j]) * CFG_BAL / (8 * (NRANK - 1));
            h[j] = d > ZB ? ZB : d < -127 ? -127 : d;
            if (h[j] > 0) give += h[j]; else take -= h[j];
        }
        give = take = !ok ? 0 : give < take ? give : take;
        unsigned long long pl = 0;
#pragma unroll
        for (int j = 0; j < NRANK; ++j) {
            int d = h[j];
            if (d > 0) { d = d < give ? d : give; give -= d; } else { d = -d < take ? d : -take; take += d; }
            pl |= (unsigned long long)(d & 255) << (j * 8);
        }
        __builtin_amdgcn_raw_buffer_store_b128((u32x4){call, (unsigned)pl, call, (unsigned)(pl >> 32)}, rwl, lane < NRANK ? WS_FLAG + 63 * 32 + (unsigned)lane * FLAGB : OOB, 0, AUX_SYS);
    };
    auto offs_of = [&](int i, unsigned (&o)[4], unsigned& cnt) -> unsigned {
        if (i < NPIN) {
            int q, pos; const int pan = unit_of((i >> (6 - PSH)) + NPUSH, rank, q, pos);
            cnt = WS_CNT + (unsigned)(pan * NRANK + peer) * 128u + (unsigned)q * 4u;
            const unsigned sub = (unsigned)((i << PSH) & 63) * 1024u;
            const unsigned b = WS_PAN + (unsigned)pan * PANB + (unsigned)peer * CHB + (unsigned)q * UNB + sub + (unsigned)lane * 16u;
#pragma unroll
            for (int k = 0; k < 4; ++k) o[k] = b + k * 1024u;
            return WS_SND + (call & 1u) * SNDB + (unsigned)pos * UNB + sub;
        }
        int src = peer;
        if (ZB && i >= NPC) {
            int x = i - NPC;
            for (int j = 0; j < peer; ++j) x -= moved(j) < 0 ? moved(j) : 0;
            for (int j = 0; j < NRANK; ++j) { const int g = moved(j) > 0 ? moved(j) : 0; if ((unsigned)x < (unsigned)g) { src = j; i = NPC - g + x; } x -= g; }
        }
        const int e = (i - NPIN) / BPB, r = (i - NPIN) % BPB, pb = band_of(e, rank) * BPB + r;
        cnt = e < NRANK - 1 && r >= BPB - ZB ? WS_ZC + (unsigned)(band_of(e, rank) * NRANK + peer) * 128u : 0u;
#pragma unroll
        for (int k = 0; k < 4; ++k) {
            const unsigned sg = (unsigned)((pb << PSH) + k) * 2u + (unsigned)(lane >> 5);
            o[k] = (unsigned)src * ROWB + (unsigned)(lane & 31) * 16u + sg / NS * (NN * 2u) + sg % NS * 512u;
        }
        return WS_OSN + (unsigned)(pb << PSH) * 1024u;
    };
    if (cur < (NS - NPUSH) << (6 - PSH)) {
        const unsigned long long t0 = __builtin_amdgcn_s_memrealtime();
        const unsigned fo = fso + 2048u + (unsigned)(((cur >> 3) + 2 * NPUSH) * 2 + (cur >> 1 & 1)) * 8u;
        for (;;) {
            const unsigned f = __builtin_amdgcn_raw_buffer_load_b32(rwl, (unsigned)(lane & 1) * 4u, fo, AUX_SYS);
            if (__all((int)(f - call) >= 0)) { lim = flim = cur + 1; break; }
            if (give_up(tmo, call, t0)) { bad = true; planned = true; break; }
            __builtin_amdgcn_s_sleep(CFG_PS);
        }
        asm volatile("" ::: "memory");
    }
    while (!planned || cur < fin) {
        if (!bad && cur >= lim) {
            const unsigned long long t0 = __builtin_amdgcn_s_memrealtime();
            const unsigned pvo = (unsigned)(lane - (first_of(cur) >> 3)) < CFG_PWIN || (lane == 63 && cur >= NPC - ZB) ? (unsigned)lane * 32u : OOB - 16u;
            for (;;) {
                const u32x4 f0 = __builtin_amdgcn_raw_buffer_load_b128(rwl, pvo, fso, AUX_SYS), f1 = __builtin_amdgcn_raw_buffer_load_b128(rwl, pvo + 16u, fso, AUX_SYS);
                update(f0, f1, cur);
                if (cur < lim || (planned && cur >= fin)) break;
                if (give_up(tmo, call, t0)) { bad = true; planned = true; plan = 0; fin = NPC; break; }
                __builtin_amdgcn_s_sleep(CFG_PS);
            }
            asm volatile("" ::: "memory");
            if (cur >= fin && planned) break;
        }
        unsigned o[4], p2 = OOB, p3 = OOB, cprev = 0;
        if (bad) {
            (void)offs_of(cur, o, cprev);
            if (cur >= NPIN) {
#pragma unroll
                for (int k = 0; k < PU; ++k) __builtin_amdgcn_raw_buffer_store_b128((u32x4){0x7fc07fc0u, 0x7fc07fc0u, 0x7fc07fc0u, 0x7fc07fc0u}, rol, o[k], 0, AUX_AGENT);
            }
            cur = index_of(draw()); continue;
        }
        STAMP1(P, 1);
        rsrc_t rp = rwl;
        u32x4 v[4] = {};
        unsigned slot = 0;
        if (CFG_RATE) { const unsigned now = (unsigned)__builtin_amdgcn_s_memrealtime() << 4; slot = now - (now & 0xffffffu) % PACE16 + (unsigned)((widx & 7) * PW + (widx >> 3)) * (PACE16 / NW) - PACE16; }
        unsigned cb, src, in[4];
        auto prepare = [&]() {
            if (ZB && cur == NPC - CFG_PLAN) make_plan();
            src = __builtin_amdgcn_readfirstlane(offs_of(cur, o, cb)); cb = __builtin_amdgcn_readfirstlane(cb);
            if (ZB && cur >= NPC) src = 0;
#pragma unroll
            for (int k = 0; k < 4; ++k) { in[k] = ZB && cur >= NPC ? o[k] : lane * 16 + k * 1024; asm volatile("" : "+v"(o[k]), "+v"(in[k])); }
        };
        prepare();
        do {
            const bool tail = cur >= fin - CFG_TAIL, even = cur >= fin - (deep0 ? CFG_EVEND : CFG_EVEN), deep = deep0 || tail || even;
            const unsigned per = even ? CFG_ERATE * NW / 64 : deep0 ? PACE16 / 2 : PACE16;
            const rsrc_t rd = cur < NPIN ? rwl : rol, rs = ZB && cur >= NPC ? ror : rwr;
            const bool look = flim < NPC && cur + CFG_LOOK >= flim, lookp = !planned && cur + CFG_LOOKP >= NPC - ZB;
            const unsigned fvo = look || (lookp && lane == 63) ? (unsigned)lane * 32u : OOB - 16u;
            unsigned cdone = cprev;
            unsigned nv = 0; u32x4 f0, f1;
#pragma unroll
            for (int k = 0; k < 4; ++k) {
                if (CFG_RATE && !tail) {
                    slot += per;
                    if (CFG_GRID) {
                        unsigned go = (unsigned)__builtin_amdgcn_s_memrealtime();
                        const int d = (int)((go << 4) - slot);
                        if (d > CFG_CRED * 16) slot += (unsigned)(d - CFG_CRED * 16);
                        if (d < 0) go += (unsigned)(15 - d) >> 4;
                        go += CFG_GRID - (go & 0xffffffu) % (CFG_GRID ? CFG_GRID : 1);
                        for (;;) {
                            const int w = (int)(go - (unsigned)__builtin_amdgcn_s_memrealtime());
                            if (w <= 0) break;
                            if (w > CFG_SPIN) __builtin_amdgcn_s_sleep(1);
                        }
                    } else for (;;) {
                        const int d = (int)(((unsigned)__builtin_amdgcn_s_memrealtime() << 4) - slot);
                        if (d > CFG_CRED * 16) slot += (unsigned)(d - CFG_CRED * 16);
                        if (d >= 0) break;
                        __builtin_amdgcn_s_sleep(1);
                    }
                }
                if (CFG_FK && k == 2) { f0 = __builtin_amdgcn_raw_buffer_load_b128(rwl, fvo, fso, AUX_SYS); f1 = __builtin_amdgcn_raw_buffer_load_b128(rwl, fvo + 16u, fso, AUX_SYS); asm volatile("" ::: "memory"); }
                v[k] = __builtin_amdgcn_raw_buffer_load_b128(rs, in[k], src, AUX_SYS_NT);
                if (k == 0) {
                    nv = draw();
                    if (!CFG_FK) { f0 = __builtin_amdgcn_raw_buffer_load_b128(rwl, fvo, fso, AUX_SYS); f1 = __builtin_amdgcn_raw_buffer_load_b128(rwl, fvo + 16u, fso, AUX_SYS); }
                }
                __builtin_amdgcn_raw_buffer_store_b128(v[k ^ 2], k < 2 ? rp : rd, k == 0 ? p2 : k == 1 ? p3 : o[k ^ 2], 0, AUX_AGENT);
                if (CFG_EARLY && !deep) {
                    unsigned& a = k == 0 ? p3 : o[k - 1];
                    __builtin_amdgcn_raw_buffer_store_b128(v[(k + 3) & 3], k < 1 ? rp : rd, a, 0, AUX_AGENT);
                    a = OOB;
                }
                if (k == 3) {
                    asm volatile("" ::: "memory");
                    cprev = cb; rp = rd; p2 = o[2]; p3 = o[3];
                    cur = index_of(nv);
                    if (look || lookp) update(f0, f1, cur);
                    if (cur < lim) prepare();
                }
                if (k == 0) WAIT_VMN(CFG_FK ? 5 : 7); else if (k == 1) WAIT_VMN(CFG_FK ? 4 : 6); else if (k == 2) WAIT_VMN(CFG_FK ? 5 : 3); else WAIT_VMN(3);
                if (!deep) {
                    if (k == 0) WAIT_VMN((CFG_FK ? 2 : 4) + CFG_EARLY); else WAIT_VMN(1 + CFG_EARLY);
                    if (CFG_EARLY && k == 1) { arrive(cdone ? lane0 : OOB, cdone); cdone = 0; }
                }
            }
            arrive(cdone ? lane0 : OOB, cdone);
        } while (cur < lim);
        __builtin_amdgcn_raw_buffer_store_b128(v[2], rp, p2, 0, AUX_AGENT);
        __builtin_amdgcn_raw_buffer_store_b128(v[3], rp, p3, 0, AUX_AGENT);
        if (cprev) { WAIT_VM0(); arrive(lane0, cprev); }
    }
    STAMP(P, 4);
}

__device__ __forceinline__ void load_w(u32x4 (&v)[8], const rsrc_t rw, int gpan, int q, int job, int lane) {
    const unsigned vo = (unsigned)((lane >> 4) * 8) * (NN * 2) + (unsigned)(lane & 15) * 16u;
    const unsigned so = (unsigned)(q * 128 + (job >> 1) * 32) * (NN * 2) + (unsigned)(gpan * 256 + (job & 1) * 128) * 2u;
#pragma unroll
    for (int j = 0; j < 8; ++j) v[j] = __builtin_amdgcn_raw_buffer_load_b128(rw, vo, so + (unsigned)j * (NN * 2), 2);
}
__device__ __forceinline__ void store_w(const u32x4 (&v)[8], const rsrc_t rd, unsigned vo, unsigned unit, int job, bool slow) {
    unit += (unsigned)(job >> 2) * KTB + (unsigned)(job & 1) * 8192u + (unsigned)(job >> 1 & 1) * 1024u;
#pragma unroll
    for (int i = 0; i < 8; ++i) {
        u32x4 o;
#pragma unroll
        for (int e = 0; e < 4; ++e) o[e] = __builtin_amdgcn_perm(v[2 * e + 1][i >> 1], v[2 * e][i >> 1], (i & 1) ? 0x07060302u : 0x05040100u);
        STORE16(o, rd, vo, unit + (unsigned)(i >> 2) * HTB + (unsigned)(i & 3) * 2048u, AUX_SYS);
        if (CFG_PWND && i >= CFG_PWND - 1 && i < 7 && slow) WAIT_VMN(CFG_PWND ? CFG_PWND - 1 : 0);
    }
}
typedef __attribute__((address_space(1))) unsigned* gptr_t;

__device__ __forceinline__ void lead_role(const Params& P, int rank, unsigned long long wsp, unsigned tk, unsigned t0, int g, int wid, int lane) {
    const int o = NS - 1 - (g >> 5), c = (g >> 3) & 3, job = (c >> 1) * 4 + wid * 2 + (c & 1);
    const int dest = (rank + 1 + g + (o < NPUSH ? CFG_MIX * (2 * c + o) : 0)) & (NRANK - 1);
    u32x4 v[8];
    load_w(v, mk_rsrc(P.w, (unsigned)KSH * NN * 2), dest * NS + o, 0, job, lane);
    asm volatile("" : "+v"(tk), "+v"(wsp) :: "memory");
    const bool push = o < NPUSH && dest != rank, count = o < NPUSH || dest == rank;
    char* wsd = (char*)((unsigned long long)(unsigned)__builtin_amdgcn_readlane((unsigned)(wsp >> 32), dest) << 32 | (unsigned)__builtin_amdgcn_readlane((unsigned)wsp, dest));
    gptr_t fp = (gptr_t)(wsd + (count ? WS_CNT + (unsigned)((NTM + o) * NRANK + rank) * 128u : WS_FLAG + (unsigned)rank * FLAGB + 2048u + (unsigned)((o * 4 + c) * 2 + wid) * 4u));
    while (CFG_P3 && push && o >= CFG_P3N && (int)((unsigned)__builtin_amdgcn_s_memrealtime() - t0) < CFG_P3 + (o - CFG_P3N) * 110) __builtin_amdgcn_s_sleep(1);
    store_w(v, mk_rsrc(push ? wsd : P.wsl, WS_END), (unsigned)lane * 16u + (push ? 0u : (~tk & 1u) * SNDB),
            push ? WS_PAN + (unsigned)(NTM + o) * PANB + (unsigned)rank * CHB : WS_SND + (unsigned)(dest * NS + o) * UNB, job, push);
    WAIT_VMN(0);
    STAMP(P, 6);
    if (lane == 0) {
        if (!count) __hip_atomic_store(fp, tk + 1u, __ATOMIC_RELAXED, __HIP_MEMORY_SCOPE_SYSTEM);
        else if ((__hip_atomic_fetch_add((gptr_t)(P.wsl + WS_LEAD + (o * NRANK + dest) * 64), 1u, __ATOMIC_RELAXED, __HIP_MEMORY_SCOPE_AGENT) & 7u) == 7u)
            __hip_atomic_store(fp, (tk + 1u) * CNT_FULL, __ATOMIC_RELAXED, __HIP_MEMORY_SCOPE_SYSTEM);
    }
}

__device__ __forceinline__ void pack_role(const Params& P, unsigned long long wsp, unsigned tk, unsigned t0, int g, int wid, int lane) {
    const int rank = P.rank;
    const rsrc_t rx = mk_rsrc(P.x, (unsigned)MM * KSH * 2), rw = mk_rsrc(P.w, (unsigned)KSH * NN * 2), rwl = mk_rsrc(P.wsl, WS_END);
    const unsigned sv = (unsigned)lane * 16u + (~tk & 1u) * SNDB;
    constexpr int NH = CFG_UH < NUNIT ? 8 * DK : 0, UH = NH ? CFG_UH : NUNIT;
    const bool late = g >= NPK - NH;
    constexpr int U1 = NRANK * NS, NE = CFG_EH ? NH : 0;
    static_assert(CFG_TH < (UH - U1 - NH) * CFG_PH, "unpaced");
    for (int u = late ? (NE ? U1 : UH) + g - (NPK - NH) : U1 + NE + g; u < (late ? NUNIT : UH); u += !late ? NPK - NH : u < UH ? UH - U1 : NH) {
        int o, dest = -1;
        if (u < 4 * U0) {
            const int r = u % U0;
            if (r < NRANK * NS) { o = r >> 3; dest = (rank + 1 + r) & (NRANK - 1); } else o = r - (NRANK - 1) * NS;
            o += u / U0 * G0;
        } else o = u - 4 * (NRANK - 1) * NS;
        int q, pos; const int pan = unit_of(o, dest < 0 ? 0 : dest, q, pos);
        while ((int)((unsigned)__builtin_amdgcn_s_memrealtime() - t0) < (late ? CFG_TH + (u - UH) * CFG_PH : (u - CFG_PACE0) * CFG_PACE / 10)) __builtin_amdgcn_s_sleep(1);
        const unsigned dst = WS_SND + (unsigned)pos * UNB;
        u32x4 v[8];
        if (pan < NTM) {
            const int rb0 = (wid & 3) * 4;
            const unsigned vo = (unsigned)(lane & 15) * (KSH * 2) + (unsigned)(lane >> 4) * 16u;
            const unsigned so = (unsigned)(pan * 256 + rb0 * 16) * (KSH * 2) + (unsigned)(q * 2 + (wid >> 2)) * 128u;
#pragma unroll
            for (int i = 0; i < 8; ++i) v[i] = __builtin_amdgcn_raw_buffer_load_b128(rx, vo + (i & 1) * 64, so + (unsigned)(i >> 1) * (16 * KSH * 2), 0);
#pragma unroll
            for (int i = 0; i < 8; ++i) STORE16(v[i], rwl, sv + (i & 1) * 1024, dst + (unsigned)(wid >> 2) * KTB + (unsigned)(rb0 + (i >> 1)) * 2048u, AUX_SYS);
        } else {
            load_w(v, rw, gpan_of(pan - NTM, dest < 0 ? 0 : dest), q, wid, lane);
            store_w(v, rwl, sv, dst, wid, false);
        }
        const bool own = lane == rank;
        unsigned long long fp = wsp + (own ? WS_CNT + (unsigned)((pan * NRANK + rank) * 32 + q) * 4u : WS_FLAG + (unsigned)rank * FLAGB + (unsigned)o * 4u);
        unsigned fv = own ? (tk + 1u) * CNT_FULL : tk + 1u;
        asm volatile("" : "+v"(fp), "+v"(fv));
        WAIT_VMN(0);
        __builtin_amdgcn_s_barrier();
        if (wid == 0 && lane < NRANK && (dest < 0 || dest == lane)) __hip_atomic_store((gptr_t)fp, fv, __ATOMIC_RELAXED, __HIP_MEMORY_SCOPE_SYSTEM);
    }
}

struct Tile { int tm, v, half; unsigned sA, stA; };
__device__ __forceinline__ void tile_of(int idx, int col, int rank, Tile& t) {
    const int v = 2 * (idx >> 2) + (col >> 2);
    t.tm = 4 * (col & 3) + (idx & 3); t.v = v; t.half = -1;
    const int g = (v - GM) / GW, i = (v - GM) % GW;
    t.sA = WS_SND + (unsigned)(v < GW ? gpan_of(v, rank) : v < GW + GM ? 4 * U0 + v - GW : 4 * U0 + 4 * GM + (g - 1) * 4 * GW + i) * UNB;
    t.stA = (unsigned)(v < GW ? U0 : v < GW + GM ? GM : g <= NGF ? GW : GL) * UNB;
}
__device__ __forceinline__ void item_of(int q, int rank, Tile& t) {
    const int n = q < PFULL ? q : PFULL + ((q - PFULL) >> 1);
    if (n < 4 * PL0) tile_of(RS * KX - DK + n % PL0, n / PL0, rank, t);
    else tile_of(RS * KX - DK + (n - 4 * PL0) % PL1, 4 + (n - 4 * PL0) / PL1, rank, t);
    if (q >= PFULL) t.half = (q - PFULL) & 1;
}

template <int wr>
__device__ __forceinline__ void gemm_role(const Params& P, unsigned call, int col, int slot, LAS unsigned char* lds) {
    const int tid = threadIdx.x, wid = __builtin_amdgcn_readfirstlane(tid >> 6), lane = tid & 63, wc = wid & 3;
    const int rank = P.rank;
    char* wsl = P.wsl;
    const rsrc_t rws = mk_rsrc(wsl, WS_END);
    LAS unsigned* ctl = (LAS unsigned*)(lds + STAGE_BYTES);
    const unsigned voff = (unsigned)tid * 16u;
    const unsigned ldsw = (unsigned)wid * 1024u;
    const int aoff = wr * 8192 + lane * 16, boff = wc * 4096 + lane * 16;

    auto lane_now = [&]() { int tz = threadIdx.x; asm volatile("" : "+v"(tz)); return tz & 63; };
    auto cnt_of = [&](const Tile& t) { const int ln = lane_now(); return WS_CNT + (unsigned)((((ln & 32) ? NTM + t.v : t.tm) * NRANK + ((ln >> 2) & 7)) * 32 + (ln & 3)) * 4u; };
    auto level_of = [&](unsigned c) -> int {
        const unsigned long long miss = __ballot((int)(c - call * CNT_FULL) < 0);
        unsigned m = (unsigned)miss | (unsigned)(miss >> 32);
        const bool own = ((m >> (rank * 4)) & 15u) == 0u;
        m |= m >> 16; m |= m >> 8; m |= m >> 4;
        return own ? 1 + __builtin_ctz(m | 16u) : 0;
    };
    auto ready = [&](const Tile& t, int want) -> int {
        if (wr == 0 && wid == 0) {
            const unsigned* c = (const unsigned*)(wsl + cnt_of(t));
            unsigned* tmo = (unsigned*)(wsl + WS_TMO);
            const unsigned long long t0 = __builtin_amdgcn_s_memrealtime();
            int got;
            for (;;) {
                got = level_of(ld_sys(c));
                if (got >= want) break;
                if (give_up(tmo, call, t0)) { ctl[0] = 1u; got = 5; break; }
                __builtin_amdgcn_s_sleep(4);
            }
            ctl[2] = (unsigned)got;
            __builtin_amdgcn_fence(__ATOMIC_ACQUIRE, "agent");
            asm volatile("s_waitcnt vmcnt(0) lgkmcnt(0)" ::: "memory");
        }
        asm volatile("" ::: "memory"); __builtin_amdgcn_s_barrier(); asm volatile("" ::: "memory");
        const int r = __builtin_amdgcn_readfirstlane(ctl[2]);
        asm volatile("s_waitcnt lgkmcnt(0)" ::: "memory");
        return r;
    };

#define SA(b, h) (((b) * 2 + (h)) * HTB)
#define SB(b, h) ((4 + (b) * 2 + (h)) * HTB)
#define STAGE(bufoff, soff) do { _Pragma("unroll") for (int _i = 0; _i < 2; ++_i) \
        __builtin_amdgcn_raw_ptr_buffer_load_lds(rws, (LAS unsigned*)(lds + (bufoff) + ldsw + _i * 8192), 16, voff, (soff) + _i * 8192u, 0, 0); } while (0)
#define LDA(dst, b, h) do { _Pragma("unroll") for (int m = 0; m < 4; ++m) _Pragma("unroll") for (int k = 0; k < 2; ++k) dst[m][k] = *(const LAS bf16x8*)(lds + SA(b, h) + aoff + m * 2048 + k * 1024); } while (0)
#define LDB(dst, b, h) do { _Pragma("unroll") for (int n = 0; n < 2; ++n) _Pragma("unroll") for (int k = 0; k < 2; ++k) dst[n][k] = *(const LAS bf16x8*)(lds + SB(b, h) + boff + n * 2048 + k * 1024); } while (0)
#define MMA(ai, first, prio) do { __builtin_amdgcn_s_setprio(prio); \
        _Pragma("unroll") for (int i = 0; i < 32; ++i) { \
            const int bj = ((i >> 2) ^ (i >> 3)) & 1, m = i >> 3, n = ((i >> 1) ^ (i >> 3)) & 1, k = (i ^ (i >> 1)) & 1;     \
            SCHED; \
            acc[ai][bj][m][n] = __builtin_amdgcn_mfma_f32_16x16x32_bf16(bj ? B1[n][k] : B0[n][k], At[m][k], first && (i & 1) == 0 ? (f32x4){0.f, 0.f, 0.f, 0.f} : acc[ai][bj][m][n], 0, 0, 0); } \
        __builtin_amdgcn_s_setprio(0); } while (0)
#define WAIT_L0() do { asm volatile("" ::: "memory"); __builtin_amdgcn_s_waitcnt(0xC07F); asm volatile("" ::: "memory"); } while (0)
#define WAIT_V(n) asm volatile("s_waitcnt vmcnt(" #n ")" ::: "memory")
#define WAIT_L(n) asm volatile("s_waitcnt lgkmcnt(" #n ")" ::: "memory")
#define BAR __builtin_amdgcn_s_barrier()
#define SCHED __builtin_amdgcn_sched_barrier(0)

    unsigned long long outp = (unsigned long long)P.outl;
    asm volatile("" : "+v"(outp));
    Tile cur, nxt; int rnd = slot < KX - DK ? 0 : 1;
    tile_of(rnd * (KX - DK) + slot, col, rank, cur);
    bf16x8 At[4][2], B0[2][2], B1[2][2];
    constexpr unsigned SB0 = WS_SND + ((NRANK - 1) * NS + GW) * UNB, STB = U0 * UNB;
    const unsigned kov = WS_PAN + (unsigned)((rank + 1 + (lane + 3) % 7) & (NRANK - 1)) * CHB + (unsigned)(((lane & 31) - 4) / 7) * UNB;
    unsigned tabA, tabB;
    auto tables = [&](const Tile& t) {
        const int ln = lane_now();
        const unsigned la = t.sA + (call & 1u) * SNDB + (unsigned)ln * t.stA, ra = (unsigned)(NTM + t.v) * PANB + kov;
        const unsigned lb = SB0 + (call & 1u) * SNDB + (unsigned)t.tm * UNB + (unsigned)ln * STB, rb = (unsigned)t.tm * PANB + kov;
        tabA = ln < 4 ? la : ra;
        tabB = (ln < 4 ? lb : rb) + (t.half == 1 ? HTB : 0u);
    };
    auto pick = [&]() {
        Tile t = cur; bool ok = true;
        if (rnd + 1 < RS) tile_of((rnd + 1) * KX - DK + slot, col, rank, t);
        else {
            const unsigned d = (unsigned)__builtin_amdgcn_raw_ptr_buffer_atomic_add_i32(1, rws, lane_now() == 0 ? 0u : 0xfffffff0u, WS_HEAD + (unsigned)rank * 128u, 0);
            const unsigned q = (unsigned)__builtin_amdgcn_readfirstlane(d) - (call - 1u) * (unsigned)(NITEM + 8 * KX);
            ok = q < (unsigned)NITEM;
            if (ok) item_of((int)q, rank, t);
        }
        *(LAS u32x4*)(ctl + 4) = (u32x4){ok ? (unsigned)(t.tm | t.v << 8 | (t.half + 2) << 16) : 0u, t.sA, t.stA, 0u};
        __builtin_amdgcn_raw_ptr_buffer_load_lds(rws, ctl + 64, 4, cnt_of(t), 0, 0, AUX_SYS);
    };
    auto picked = [&](int& level) -> bool {
        const u32x4 f = *(const LAS u32x4*)(ctl + 4);
        level = level_of(ctl[64 + lane_now()]);
        const unsigned w = __builtin_amdgcn_readfirstlane(f[0]);
        nxt.tm = w & 255u; nxt.v = (w >> 8) & 255u; nxt.half = (int)(w >> 16) - 2;
        nxt.sA = __builtin_amdgcn_readfirstlane(f[1]); nxt.stA = __builtin_amdgcn_readfirstlane(f[2]);
        return w != 0u;
    };
    constexpr int JP = KT / 2 - 4, JI = KT / 2 - 2, JL = KT / 2 - 1;
    const int jend = wr == 0 && wid == 0 ? JP : JL;
    if (CFG_SYNC && rnd && wr == 0 && wid == 0) {
        const unsigned long long t0 = __builtin_amdgcn_s_memrealtime();
        while (__builtin_amdgcn_readfirstlane(ld_agent((const unsigned*)(wsl + WS_COL + col * 128))) != call && !give_up((unsigned*)(wsl + WS_TMO), call, t0)) __builtin_amdgcn_s_sleep(16);
    }
    int lvl = ready(cur, 1);
    STAMP(P, 2);
    tables(cur);
    unsigned a2 = cur.sA + (call & 1u) * SNDB, b2 = SB0 + (call & 1u) * SNDB + (unsigned)cur.tm * UNB;
    STAGE(SB(0, 0), b2); STAGE(SA(0, 0), a2); STAGE(SB(0, 1), b2 + HTB); STAGE(SA(0, 1), a2 + HTB);
    STAGE(SB(1, 0), b2 + KTB); STAGE(SA(1, 0), a2 + KTB); STAGE(SB(1, 1), b2 + HTB + KTB);
    WAIT_V(0); BAR;
#define TICK(i, r) do {} while (0)
#define ROUND_DONE() do {} while (0)
    bool has_next = false;
#pragma unroll 1
    for (;;) {
        f32x4 acc[2][2][4][2];
        int jchk = lvl < 5 ? 7 * lvl - 4 : jend;
        TICK(0, 2); TICK(4, 3);
#define SYNC_MMA(ai, first, wv) do { if (wv) WAIT_V(8); WAIT_L0(); if (wr == 1) BAR; MMA(ai, first, wr + 1); if (wr == 0) BAR; SCHED; } while (0)
#define PASS(first, check) do { \
            LDB(B0, 0, 0); LDA(At, 0, 0); LDB(B1, 0, 1); STAGE(SA(1, 1), a2 + KTB + HTB); SCHED; \
            check; \
            a2 = __builtin_amdgcn_readlane(tabA, (j + 1) & 31); b2 = __builtin_amdgcn_readlane(tabB, (j + 1) & 31); \
            SYNC_MMA(0, first, !first); \
            LDA(At, 0, 1); STAGE(SB(0, 0), b2); STAGE(SA(0, 0), a2); STAGE(SB(0, 1), b2 + HTB); \
            SYNC_MMA(1, first, !first); \
            LDB(B0, 1, 0); LDA(At, 1, 0); LDB(B1, 1, 1); STAGE(SA(0, 1), a2 + HTB); \
            SYNC_MMA(0, false, true); \
            LDA(At, 1, 1); STAGE(SB(1, 0), b2 + KTB); STAGE(SA(1, 0), a2 + KTB); STAGE(SB(1, 1), b2 + KTB + HTB); \
            SYNC_MMA(1, false, true); } while (0)
        int j = 0;
        PASS(true, );
#pragma unroll 1
        for (j = 1; j < KT / 2; ++j)
            PASS(false, if (__builtin_expect(j == jchk, 0)) {
                    if (j < JP) { lvl = ready(cur, lvl + 1); jchk = lvl < 5 ? 7 * lvl - 4 : jend; }
                    else if (wr == 0 && j == JP) { pick(); jchk = JI; }
                    else if (wr == 0 && j == JI) { asm volatile("buffer_inv sc1" ::: "memory"); jchk = JL; }
                    else {
                        has_next = picked(lvl);
                        if (has_next) {
                            const int want = nxt.half < 0 ? 1 : 5;
                            if (lvl < want) lvl = ready(nxt, want);
                            tables(nxt);
                        }
                    }
                });
        if (has_next && nxt.half >= 0) STAGE(SA(1, 1), a2 + KTB + HTB);
        TICK(1, 2);
        TICK(2, 2);
        {
            const int ln = lane_now(), fr = ln & 15, fq = ln >> 4;
            const rsrc_t rol = mk_rsrc((const void*)outp, OUTB);
            const bool shared = cur.v < NS;
            if (__builtin_amdgcn_readfirstlane(ctl[0]) != 0u) {
#pragma unroll
                for (int i = 0; i < 32; ++i) acc[i >> 4][(i >> 3) & 1][(i >> 1) & 3][i & 1] = __builtin_bit_cast(f32x4, (u32x4){0x7fc00000u, 0x7fc00000u, 0x7fc00000u, 0x7fc00000u});
            }
            if (rnd == 0 && cur.v < GW) {
                const unsigned long long t0 = __builtin_amdgcn_s_memrealtime();
                while (__builtin_amdgcn_s_memrealtime() - t0 < (unsigned long long)((col & 3) * CFG_STAG + (col >> 2) * CFG_STAGB)) __builtin_amdgcn_s_sleep(8);
                if (DK && CFG_SYNC && wr == 0 && slot == 0 && wid == 0) __builtin_amdgcn_raw_buffer_store_b32(call, rws, ln == 0 ? 0u : 0xfffffff0u, WS_COL + (unsigned)col * 128u, AUX_AGENT);
            }
            const bool thru = rnd >= RS;
            const unsigned vo = (unsigned)(wc * 32 + fq * 4) * (NN * 2) + (unsigned)(wr * 256 + fr * 16);
            const unsigned so = (unsigned)(cur.tm * 256) * (NN * 2) + (unsigned)gpan_of(cur.v, rank) * 512u;
            const unsigned vs = (unsigned)(wc * 32 + fq * 4) * ROWB + (unsigned)(wr * 256 + fr * 16);
            const unsigned ss = WS_OSN + (unsigned)(cur.tm * 256) * ROWB + (unsigned)cur.v * 512u;
            auto finish = [&](auto mode) {
#pragma unroll
                for (int bj = 0; bj < 2; ++bj)
#pragma unroll
                    for (int n = 0; n < 2; ++n)
#pragma unroll
                        for (int r = 0; r < 4; ++r) {
                            u32x4 o;
                            if ((r & 1) == 0) {
#pragma unroll
                                for (int i = 0; i < 8; ++i) { f32x4& a = acc[i >> 2][bj][i & 3][n]; const f32x2 z = silu2((f32x2){a[r], a[r + 1]}); a[r] = z[0]; a[r + 1] = z[1]; }
                            }
#pragma unroll
                            for (int i = 0; i < 4; ++i) o[i] = cvt_pk_bf16(acc[i >> 1][bj][(i & 1) * 2][n][r], acc[i >> 1][bj][(i & 1) * 2 + 1][n][r]);
                            if (mode == 1) STORE16(o, rws, vs, ss + (unsigned)(bj * 128 + n * 16 + r) * ROWB, AUX_SYS);
                            STORE16(o, rol, vo, so + (unsigned)(bj * 128 + n * 16 + r) * (NN * 2), mode == 2 ? AUX_SYS : 0);
                        }
            };
            if (shared) finish(std::integral_constant<int, 1>());
            else if (thru) finish(std::integral_constant<int, 2>());
            else finish(std::integral_constant<int, 0>());
            WAIT_V(16);
            if (shared) {
                WAIT_VM0();
                unsigned old = 0;
                if (ln == 0) old = __hip_atomic_fetch_add(ctl + 1, 1u, __ATOMIC_RELAXED, __HIP_MEMORY_SCOPE_WORKGROUP);
                old = __builtin_amdgcn_readfirstlane(old);
                if ((old & 7u) == 7u && ln < NRANK && ln != rank)
                    st_sys((unsigned*)(P.ws[ln] + WS_FLAG + rank * FLAGB) + NIN * 4 + turn_of(cur.tm, ln) * NS + cur.v, call);
            }
        }
        TICK(3, 2);
        ROUND_DONE();
        if (!has_next) break;
        cur = nxt; ++rnd;
        if (cur.half >= 0) break;
    }
#define HB(b, i) ((b) < 2 ? ((i) < 2 ? SA(b, i) : SB(b, 0)) : (i) < 2 ? SB(i, 1) : 8 * HTB)
#define HSTAGE(b, t) do { const unsigned _o = ((t) & 1) * KTB, _a = __builtin_amdgcn_readlane(tabA, ((t) >> 1) & 31) + _o, _b = __builtin_amdgcn_readlane(tabB, ((t) >> 1) & 31) + _o; \
        STAGE(HB(b, 0), _a); STAGE(HB(b, 1), _a + HTB); STAGE(HB(b, 2), _b); } while (0)
#define HLOAD(b) do { _Pragma("unroll") for (int i = 0; i < 4; ++i) B0[i >> 1][i & 1] = *(const LAS bf16x8*)(lds + HB(b, 2) + boff + i * 1024); \
        _Pragma("unroll") for (int i = 0; i < 8; ++i) At[i >> 1][i & 1] = *(const LAS bf16x8*)(lds + HB(b, 0) + aoff + i * 1024); \
        _Pragma("unroll") for (int i = 0; i < 8; ++i) Au[i >> 1][i & 1] = *(const LAS bf16x8*)(lds + HB(b, 1) + aoff + i * 1024); } while (0)
#define HPASS(b, t, first) do { \
        if (wr == 0) { HLOAD(b); HSTAGE((b + 2) % 3, (t) + 2); } else { HSTAGE((b + 2) % 3, (t) + 2); SCHED; HLOAD(b); if (!first) WAIT_V(6); } \
        WAIT_L0(); if (wr == 1) BAR; \
        __builtin_amdgcn_s_setprio(wr + 1); \
        _Pragma("unroll") for (int i = 0; i < 32; ++i) { const int ai = i >> 4, m = (i >> 2) & 3, n = (i >> 1) & 1, k = i & 1; \
            acc[ai][m][n] = __builtin_amdgcn_mfma_f32_16x16x32_bf16(B0[n][k], ai ? Au[m][k] : At[m][k], first && k == 0 ? (f32x4){0.f, 0.f, 0.f, 0.f} : acc[ai][m][n], 0, 0, 0); } \
        __builtin_amdgcn_s_setprio(0); \
        if (wr == 0) { if (!first) WAIT_V(6); BAR; } SCHED; } while (0)
#pragma unroll 1
    while (PFULL < PN && has_next) {
        f32x4 acc[2][4][2];
        bf16x8 Au[4][2];
        BAR;
        HPASS(0, 0, true);
#pragma unroll 1
        for (int t = 1; t < KT; t += 3) {
            if (wr == 0 && wid == 0 && t == KT - 9) pick();
            HPASS(1, t, false); HPASS(2, t + 1, false); HPASS(0, t + 2, false);
        }
        has_next = picked(lvl);
        if (has_next) {
            (void)ready(nxt, 5);
            tables(nxt);
            HSTAGE(0, 0); HSTAGE(1, 1);
        }
        const int ln = lane_now(), fr = ln & 15, fq = ln >> 4;
        const rsrc_t rol = mk_rsrc((const void*)outp, OUTB);
        if (__builtin_amdgcn_readfirstlane(ctl[0]) != 0u) {
#pragma unroll
            for (int i = 0; i < 16; ++i) acc[i >> 3][(i >> 1) & 3][i & 1] = __builtin_bit_cast(f32x4, (u32x4){0x7fc00000u, 0x7fc00000u, 0x7fc00000u, 0x7fc00000u});
        }
        const unsigned vo = (unsigned)(wc * 32 + fq * 4) * (NN * 2) + (unsigned)(wr * 256 + fr * 16);
        const unsigned so = (unsigned)(cur.tm * 256 + cur.half * 128) * (NN * 2) + (unsigned)gpan_of(cur.v, rank) * 512u;
#pragma unroll
        for (int n = 0; n < 2; ++n)
#pragma unroll
            for (int r = 0; r < 4; ++r) {
                u32x4 o;
                if ((r & 1) == 0) {
#pragma unroll
                    for (int i = 0; i < 8; ++i) { f32x4& a = acc[i >> 2][i & 3][n]; const f32x2 z = silu2((f32x2){a[r], a[r + 1]}); a[r] = z[0]; a[r + 1] = z[1]; }
                }
#pragma unroll
                for (int i = 0; i < 4; ++i) o[i] = cvt_pk_bf16(acc[i >> 1][(i & 1) * 2][n][r], acc[i >> 1][(i & 1) * 2 + 1][n][r]);
                STORE16(o, rol, vo, so + (unsigned)(n * 16 + r) * (NN * 2), AUX_SYS);
            }
        WAIT_V(8);
        ROUND_DONE();
        cur = nxt; ++rnd;
    }
    WAIT_V(0);
}

__global__ __launch_bounds__(512, 2) void fused_gemm_ar(Params P) {
    extern __shared__ __attribute__((aligned(16))) unsigned char shm[];
    LAS unsigned char* lds = (LAS unsigned char*)shm;
    const int wid = __builtin_amdgcn_readfirstlane(threadIdx.x >> 6), lane = threadIdx.x & 63;
    const int b = __builtin_amdgcn_readfirstlane(blockIdx.x);
    const unsigned long long wsp = (unsigned long long)P.ws[lane & (NRANK - 1)];
    const unsigned t0 = (unsigned)__builtin_amdgcn_s_memrealtime();
    int rk = P.rank;
    asm volatile("" : "+s"(rk) : "s"(P.x), "s"(P.w), "s"(P.wsl), "s"(P.rank) : "memory");
    unsigned* tkp = (unsigned*)(P.wsl + WS_TICKET + b * 128 + wid * 16);
    const unsigned tk = ld_agent(tkp);
    if (b < NPULL) {
        const unsigned call = __builtin_amdgcn_readfirstlane(tk) + 1u;
        st_agent(tkp, call);
        const int pi = b / PW, slot = b % PW;
        const int peer = (P.rank + 1 + pi) & (NRANK - 1);
        pull_role(P, call, peer, slot * 8 + wid, lane);
    } else {
        const int g = b - NPULL;
        if (g < 4 * NRANK * NS && wid < 2) lead_role(P, rk, wsp, tk, t0, g, wid, lane);
        ((LAS unsigned*)(lds + STAGE_BYTES))[lane & 3] = 0u;
        pack_role(P, wsp, tk, t0, g, wid, lane);
        const unsigned call = __builtin_amdgcn_readfirstlane(tk) + 1u;
        st_agent(tkp, call);
        STAMP(P, 1);
        asm volatile("s_waitcnt lgkmcnt(0)" ::: "memory");
        __builtin_amdgcn_s_barrier();
        if (b >= NPS) {
            const int col = (CFG_SWAP >> (b & 7) * 4) & 7;
            if (wid < 4) gemm_role<0>(P, call, col, (b - NPS) >> 3, lds);
            else gemm_role<1>(P, call, col, (b - NPS) >> 3, lds);
        }
    }
}
}

extern "C" void kernel_launch(int rank, int nranks, ncclComm_t comm,
                              hipStream_t stream,
                              void* const* d_in, const int* in_sizes, int n_in,
                              void* d_out, int out_size,
                              const int* in_dtypes, int out_dtype,
                              void* const* d_out_all,
                              void* const* d_ws_all, size_t ws_bytes) {
    if (nranks != NRANK || (unsigned)rank >= (unsigned)NRANK || ws_bytes < (size_t)WS_END || n_in < 2 ||
        in_sizes[0] != MM * KSH || in_sizes[1] != KSH * NN || out_size != MM * NN) {
        fprintf(stderr, "unexpected problem geometry\n");
        return;
    }
    static bool attr_done[NRANK] = {};
    if (!attr_done[rank]) {
        HIP_CHECK(hipFuncSetAttribute((const void*)fused_gemm_ar, hipFuncAttributeMaxDynamicSharedMemorySize, LDS_BYTES));
        attr_done[rank] = true;
    }
    Params P = {};
    P.x = d_in[0]; P.w = d_in[1]; P.rank = rank; P.wsl = (char*)d_ws_all[rank]; P.outl = (char*)d_out_all[rank];
    for (int p = 0; p < NRANK; ++p) { P.out[p] = (char*)d_out_all[p]; P.ws[p] = (char*)d_ws_all[p]; }
    hipLaunchKernelGGL(fused_gemm_ar, dim3(GRID), dim3(512), LDS_BYTES, stream, P);
    HIP_CHECK(hipGetLastError());
}
```

```cpp
#include <hip/hip_runtime.h>
#include <rccl/rccl.h>
#include <cstdio>
#include <type_traits>

#define HIP_CHECK(expr) do { hipError_t _e = (expr); if (_e != hipSuccess) { \
    fprintf(stderr, "HIP error %s at %s:%d\n", hipGetErrorString(_e), __FILE__, __LINE__); \
    return; } } while (0)

#define LAS __attribute__((address_space(3)))
typedef short bf16x8 __attribute__((ext_vector_type(8)));
typedef float f32x4 __attribute__((ext_vector_type(4)));
typedef unsigned u32x4 __attribute__((ext_vector_type(4)));
typedef __amdgpu_buffer_rsrc_t rsrc_t;

namespace {
constexpr int NRANK = 8;
constexpr int MM = 4096, KK = 4096, NN = 32768, KSH = KK / NRANK;
constexpr int KT = KK / 64, KTS = KSH / 64, NTM = MM / 256, NPAN = NN / 256;
#define CFG_NS 6
#define CFG_PW 8
constexpr int NS = CFG_NS;
constexpr int NRP = NPAN - NRANK * NS;
constexpr int NWP = NS + NRP;
constexpr int NIN = NTM + NWP;
constexpr int NOUT = NS * NTM;
constexpr int NFLAG = NIN * 4 + NOUT;
constexpr int NUNIT = NTM * 4 + NPAN * 4;
constexpr int GRID = 256;
constexpr int PW = CFG_PW;
constexpr int NPULL = PW * (NRANK - 1);
constexpr int NW = PW * 8;
constexpr int NPK = GRID - NPULL;
constexpr int NPS = (NPULL + 7) / 8 * 8;
constexpr int KX = (GRID - NPS) / 8;
#define CFG_GW (2 * (KX / 4))
constexpr int GW = CFG_GW;
constexpr int G0 = GW + NTM;
#define CFG_SKIP 1
#define CFG_SYNC 1
#define CFG_GM (CFG_SKIP ? GW + 2 : 2)
constexpr int GM = CFG_GM;
constexpr int NGF = (NWP - GW - GM) / GW, GL = (NWP - GW - GM) % GW;
static_assert(NS <= GW && NWP > GW, "shared panels sit in the first group");
constexpr int PU = 4, PSH = 2;
#define CFG_PUSH 2
constexpr int NPUSH = CFG_PUSH;
constexpr int NPIN = (NIN * 4 - NPUSH) * 64 / PU;
static_assert(NPUSH <= NS, "pushed units have a single consumer");
constexpr int NPC = NPIN + NOUT * 128 / PU;
constexpr unsigned CNT_FULL = 64;

constexpr int NT0 = 4 * ((NWP + 1) / 2), NT1 = 4 * (NWP / 2);
constexpr int DK = CFG_SKIP ? KX - 2 * GW : 0;
constexpr int RS = (NT1 - 1 + DK) / KX;
constexpr int PL0 = NT0 - RS * KX + DK, PL1 = NT1 - RS * KX + DK, PN = 4 * (PL0 + PL1);
#define CFG_SWAP 0x57461302u
#define CFG_FD (CFG_SKIP ? 0 : -8)
constexpr int PFULL = 2 * PN - 8 * KX - CFG_FD < 0 ? 0 : 2 * PN - 8 * KX - CFG_FD > PN ? PN : 2 * PN - 8 * KX - CFG_FD;
constexpr int NITEM = PFULL + 2 * (PN - PFULL);
constexpr unsigned KTB = 32768, HTB = 16384;
constexpr unsigned PANB = KT * KTB;
constexpr unsigned CHB = KTS * KTB;
constexpr unsigned UNB = 2 * KTB;
constexpr unsigned WS_TICKET = 0;
constexpr unsigned WS_TMO = WS_TICKET + GRID * 128;
constexpr unsigned WS_FLAG = WS_TMO + 128;
constexpr unsigned FLAGB = 4096;
constexpr unsigned WS_LEAD = WS_FLAG + NRANK * FLAGB;
constexpr unsigned WS_HEAD = WS_LEAD + NRANK * 512;
constexpr unsigned WS_COL = WS_HEAD + NRANK * 128;
#define CFG_PHOFF 0
constexpr unsigned WS_PH = WS_COL + 8 * 128 + CFG_PHOFF;
constexpr unsigned WS_ZC = WS_PH + 2 * NRANK * 128 + 128;
constexpr unsigned WS_GRP = WS_ZC + NRANK * NRANK * 128;
constexpr unsigned WS_BAND = WS_GRP + 64 * NRANK * 4;
constexpr unsigned WS_CNT = WS_BAND + NTM * 128;
constexpr unsigned WS_CTRL_END = WS_CNT + NIN * NRANK * 128;
constexpr unsigned WS_DBG = 512u << 10;
constexpr unsigned WS_SND = 1u << 20, SNDB = NUNIT * UNB;
constexpr unsigned ROWB = NS * 512;
constexpr unsigned WS_OSN = WS_SND + NUNIT * UNB + SNDB;
constexpr unsigned WS_PAN = (WS_OSN + MM * ROWB + PANB - 1) / PANB * PANB;
constexpr unsigned WS_END = WS_PAN + NIN * PANB;
constexpr int U0 = G0 + (NRANK - 1) * NS;
constexpr unsigned OUTB = (unsigned)MM * NN * 2;
static_assert(WS_CTRL_END <= WS_DBG, "ctrl region");
static_assert(NFLAG <= 512 && 2048 <= (int)FLAGB, "flag region");
#define CFG_DEEP 5
constexpr unsigned STAGE_BYTES = 9 * HTB;
constexpr unsigned LDS_BYTES = STAGE_BYTES + 768;
constexpr unsigned long long DEADLINE = 50000000ull;
constexpr int AUX_SYS = 17, AUX_SYS_NT = 19, AUX_AGENT = 16;
#define CFG_PACE 140
#define CFG_UH (2 * (NPK - 8 * DK))
#define CFG_TH 5000
#define CFG_PH 40
#define CFG_OST 0
#define CFG_OTH AUX_SYS
#define CFG_PST AUX_AGENT
#define CFG_LDA 0
#define CFG_RLD AUX_SYS_NT
#define CFG_PFA 0
#define CFG_VD 999
#define CFG_OWN AUX_AGENT
#define CFG_PFB 1
#define CFG_JP 4
#define CFG_AGG 24
#define CFG_AGB 1
static_assert(CFG_AGG % 8 == 0 && CFG_AGG >= NS && NS <= 8 && NIN * 4 <= 512, "groups");
#define CFG_PWND 3
#define CFG_EH 1
#define CFG_FK 2
#define CFG_MIX 1
static_assert(4 * NRANK * NS <= NPK && (NS - NPUSH) * 64 / PU <= NW && NS * 64 <= 512, "leading units");
#define CFG_PACE0 40
#define CFG_STAG 0
#define CFG_P3 320
#define CFG_P3N 2
#define CFG_STAGB 400
#define CFG_LOOK 104
#define CFG_TAIL NW
#define CFG_EVEN 280
#define CFG_EVEND 200
#define CFG_ERATE CFG_RATE
#define CFG_ZB 120
#define CFG_LOOKP 100
#define CFG_PLAN (CFG_ZB + CFG_LOOKP + 50)
#define CFG_BAL 8
constexpr int ZB = CFG_ZB;
static_assert(ZB <= 127 && ZB <= NS * 128 / PU && NFLAG <= 504, "plan");
#define CFG_PWIN 4u
#define CFG_PS 2
#define CFG_RATE 1680
#define CFG_CRED 25
#define CFG_GRID 36
#define CFG_SPIN 8
#define CFG_EARLY 1
constexpr unsigned PACE16 = CFG_RATE * PW * (8 + CFG_DEEP) / 64;

#define CFG_EA 12
#define CFG_EB 6
#define CFG_ER 78
#define CFG_ES 17
#define CFG_EMIN 16
constexpr int NTAB = 4 * (G0 + GM);
static_assert(NTAB <= 256, "byte tables");
struct Order {
    unsigned char from[256], to[256];
    static constexpr int due(int o) {
        const int d = o >= 4 * G0 ? CFG_ER + (o - 4 * G0) / GM * CFG_ES + ((o - 4 * G0) % GM & 1) * CFG_EB
                                  : o / G0 * CFG_ES + (o % G0 < GW ? o % G0 / (GW / 2) * CFG_EB : (o % G0 - GW) / 4 * CFG_EA);
        return d < CFG_EMIN ? CFG_EMIN : d;
    }
    constexpr Order() : from(), to() {
        for (int o = 0; o < NTAB; ++o) {
            int n = 0;
            for (int k = 0; k < NTAB; ++k) n += due(k) < due(o) || (due(k) == due(o) && k < o);
            to[o] = (unsigned char)n; from[n] = (unsigned char)o;
        }
    }
};
constexpr Order ORDER;
static_assert(ORDER.to[NS - 1] == NS - 1, "the leading units come first");

struct Params {
    const void* x; const void* w;
    char* wsl; char* outl;
    int rank, pad[7];
    char* ws[NRANK]; char* out[NRANK];
    Order order;
};
__device__ __forceinline__ int look_up(unsigned tab, int o) { return o < NTAB ? (unsigned)__builtin_amdgcn_readlane(tab, o >> 2) >> (o & 3) * 8 & 255u : o; }

__device__ __forceinline__ rsrc_t mk_rsrc(const void* p, unsigned bytes) {
    const unsigned long long v = (unsigned long long)p;
    const unsigned lo = __builtin_amdgcn_readfirstlane((unsigned)v), hi = __builtin_amdgcn_readfirstlane((unsigned)(v >> 32));
    return __builtin_amdgcn_make_buffer_rsrc((void*)(((unsigned long long)hi << 32) | lo), 0, bytes, 0x00020000);
}
__device__ __forceinline__ unsigned ld_sys(const unsigned* p) { return __hip_atomic_load(p, __ATOMIC_RELAXED, __HIP_MEMORY_SCOPE_SYSTEM); }
__device__ __forceinline__ unsigned ld_agent(const unsigned* p) { return __hip_atomic_load(p, __ATOMIC_RELAXED, __HIP_MEMORY_SCOPE_AGENT); }
__device__ __forceinline__ void st_sys(unsigned* p, unsigned v) { __hip_atomic_store(p, v, __ATOMIC_RELAXED, __HIP_MEMORY_SCOPE_SYSTEM); }
__device__ __forceinline__ void st_agent(unsigned* p, unsigned v) { __hip_atomic_store(p, v, __ATOMIC_RELAXED, __HIP_MEMORY_SCOPE_AGENT); }
__device__ __forceinline__ void add_agent(unsigned* p, unsigned v) { (void)__hip_atomic_fetch_add(p, v, __ATOMIC_RELAXED, __HIP_MEMORY_SCOPE_AGENT); }
#define STORE16(v, rs, vo, so, aux) do { const u32x4 _d = (v); __builtin_amdgcn_raw_buffer_store_b128(_d, rs, vo, so, aux); asm volatile("s_nop 1" :: "v"(_d) : "memory"); } while (0)
#define WAIT_VM0() asm volatile("s_waitcnt vmcnt(0)" ::: "memory")
#define WAIT_VMN(n) do { asm volatile("" ::: "memory"); __builtin_amdgcn_s_waitcnt(0x0F70 | (n)); asm volatile("" ::: "memory"); } while (0)

__device__ __forceinline__ bool give_up(unsigned* tmo, unsigned call, unsigned long long t0) {
    const unsigned long long dt = __builtin_amdgcn_s_memrealtime() - t0;
    if (dt < 2000ull) return false;
    if ((unsigned)__builtin_amdgcn_readfirstlane(ld_agent(tmo)) == call) return true;
    if (dt > DEADLINE) { st_agent(tmo, call); return true; }
    return false;
}

__device__ __forceinline__ int gpan_of(int v, int consumer) { return v < NS ? consumer * NS + v : (NRANK - 1) * NS + v; }
__device__ __forceinline__ int band_of(int e, int consumer) { return e < 7 ? (consumer + 1 + e) & 7 : e < 15 ? 8 + ((consumer + e) & 7) : consumer; }
__device__ __forceinline__ int turn_of(int band, int consumer) { return band >= 8 ? 7 + ((band - consumer + 1) & 7) : band == consumer ? 15 : (band - consumer - 1) & 7; }
__device__ __forceinline__ int unit_of(int o, int consumer, int& q, int& pos) {
    if (o < 4 * G0) { q = o / G0; const int m = o % G0; pos = q * U0 + gpan_of(m, consumer); return m < GW ? NTM + m : m - GW; }
    pos = o + 4 * (NRANK - 1) * NS;
    o -= 4 * G0;
    if (o < 4 * GM) { q = o / (GM ? GM : 1); return NTM + GW + o % (GM ? GM : 1); }
    o -= 4 * GM;
    const int g = o / (4 * GW), r = o % (4 * GW);
    int i;
    if (GL == 0 || g < NGF) { q = r / GW; i = r % GW; } else { q = r / (GL ? GL : 1); i = r % (GL ? GL : 1); }
    return NTM + GM + GW * (g + 1) + i;
}

__device__ __forceinline__ unsigned cvt_pk_bf16(float lo, float hi) { unsigned r; asm("v_cvt_pk_bf16_f32 %0, %1, %2" : "=v"(r) : "v"(lo), "v"(hi)); return r; }
__device__ __forceinline__ float silu(float y) {
    const float c = __builtin_amdgcn_fmed3f(y, -60.0f, 60.0f);
    const float e = __builtin_amdgcn_exp2f(c * -1.44269504088896f);
    return y * __builtin_amdgcn_rcpf(1.0f + e);
}

typedef float f32x2 __attribute__((ext_vector_type(2)));
__device__ __forceinline__ f32x2 silu2(f32x2 y) {
    const f32x2 t = __builtin_elementwise_fma(y, (f32x2){-1.44269504088896f, -1.44269504088896f}, (f32x2){-86.5617024533378f, -86.5617024533378f});
    const f32x2 e = {__builtin_amdgcn_fmed3f(__builtin_amdgcn_exp2f(t[0]), 0.0f, 1.0f), __builtin_amdgcn_fmed3f(__builtin_amdgcn_exp2f(t[1]), 0.0f, 1.0f)};
    const f32x2 d = __builtin_elementwise_fma(e, (f32x2){1.1420073898156842e26f, 1.1420073898156842e26f}, (f32x2){1.0f, 1.0f});
    return y * (f32x2){__builtin_amdgcn_rcpf(d[0]), __builtin_amdgcn_rcpf(d[1])};
}

#define STAMP1(P, slot) do {} while (0)
#define STAMP(P, slot) do {} while (0)

__device__ __forceinline__ void pull_role(const Params& P, unsigned call, int peer, int widx, int lane) {
    const int rank = P.rank;
    char* wsl = P.wsl;
    const rsrc_t rwr = mk_rsrc(P.ws[peer], WS_END), rwl = mk_rsrc(wsl, WS_END);
    const rsrc_t rol = mk_rsrc(P.outl, OUTB), ror = mk_rsrc(P.out[peer], OUTB);
    unsigned* tmo = (unsigned*)(wsl + WS_TMO);
    constexpr unsigned OOB = 0xfffffff0u;
    const unsigned lane0 = lane == 0 ? 0u : OOB;
    const unsigned hoff = WS_PH + (call & 1u) * (NRANK * 128u);
    auto draw = [&]() { return (unsigned)__builtin_amdgcn_raw_ptr_buffer_atomic_add_i32(1, rwl, lane0, hoff + (unsigned)peer * 128u, 0); };
    if (widx == 0) __builtin_amdgcn_raw_buffer_store_b32(0u, rwl, lane0, WS_PH + (~call & 1u) * (NRANK * 128u) + (unsigned)peer * 128u, AUX_SYS);
    auto arrive = [&](unsigned vo, unsigned cnt) { (void)__builtin_amdgcn_raw_ptr_buffer_atomic_add_i32(PU, rwl, vo, cnt, 0); };
    const bool deep0 = (widx & 7) < CFG_DEEP;
    const unsigned from = ((const unsigned*)P.order.from)[lane];
    bool bad = false; int stamped = 0; (void)stamped;
    auto index_of = [&](unsigned v) { return (int)__builtin_amdgcn_readfirstlane(v) + NW; };
    int cur = index_of((unsigned)(widx - NW));
    constexpr int BPB = NS * 128 / PU;
    const unsigned fso = WS_FLAG + (unsigned)peer * FLAGB;
    auto first_of = [&](int i) { return i < NPIN ? (i >> (6 - PSH)) + NPUSH : NIN * 4 + (i - NPIN) / BPB * NS; };
    int lim = 0, flim = 0, fin = NPC;
    bool planned = ZB == 0;
    unsigned long long plan = 0;
    auto moved = [&](int j) { return (int)(signed char)(plan >> (j * 8)); };
    auto update = [&](const u32x4& f0, const u32x4& f1, int i) {
        const int first = first_of(i);
        unsigned un = 0;
#pragma unroll
        for (int j = 0; j < 8; ++j) un |= ((int)((j < 4 ? f0[j & 3] : f1[j & 3]) - call) < 0 && lane * 8 + j >= first ? 1u : 0u) << j;
        const int l = __builtin_ctzll(__ballot(un != 0u) | (1ull << 63));
        const int u = l * 8 + __builtin_ctz(__builtin_amdgcn_readlane(un, l) | 256u);
        const int fl = u <= NIN * 4 ? (u - NPUSH) << (6 - PSH) : u >= NFLAG ? NPC + 128 : NPIN + (u - NIN * 4) / NS * BPB;
        flim = fl > flim ? fl : flim;
        if (!planned && (unsigned)__builtin_amdgcn_readlane(f0[0], 63) == call && (unsigned)__builtin_amdgcn_readlane(f0[2], 63) == call) {
            plan = (unsigned long long)(unsigned)__builtin_amdgcn_readlane(f0[3], 63) << 32 | (unsigned)__builtin_amdgcn_readlane(f0[1], 63);
            planned = true; fin = NPC - moved(peer);
        }
        const int pl = planned ? fin : NPC - ZB;
        lim = flim < pl ? flim : pl;
    };
    auto make_plan = [&]() {
        unsigned old = 0;
        if (lane == 0) old = __hip_atomic_exchange((unsigned*)(wsl + WS_PH + 2 * NRANK * 128), call, __ATOMIC_RELAXED, __HIP_MEMORY_SCOPE_AGENT);
        if ((unsigned)__builtin_amdgcn_readfirstlane(old) == call) return;
        const unsigned hd = __builtin_amdgcn_raw_buffer_load_b32(rwl, lane < NRANK ? (unsigned)lane * 128u : OOB, hoff, AUX_SYS);
        const int t = lane >> 3, s = lane & 7;
        const unsigned zc = ld_sys((const unsigned*)(P.ws[t] + WS_ZC + (rank * NRANK + s) * 128));
        const bool ok = __all(t == rank || s == rank || s == t || (int)(zc - call * (unsigned)(ZB * PU)) >= 0);
        int h[NRANK], sum = 0, give = 0, take = 0;
#pragma unroll
        for (int j = 0; j < NRANK; ++j) { h[j] = __builtin_amdgcn_readlane(hd, j); sum += j == rank ? 0 : h[j]; }
#pragma unroll
        for (int j = 0; j < NRANK; ++j) {
            const int d = j == rank ? 0 : (sum - (NRANK - 1) * h[j]) * CFG_BAL / (8 * (NRANK - 1));
            h[j] = d > ZB ? ZB : d < -127 ? -127 : d;
            if (h[j] > 0) give += h[j]; else take -= h[j];
        }
        give = take = !ok ? 0 : give < take ? give : take;
        unsigned long long pl = 0;
#pragma unroll
        for (int j = 0; j < NRANK; ++j) {
            int d = h[j];
            if (d > 0) { d = d < give ? d : give; give -= d; } else { d = -d < take ? d : -take; take += d; }
            pl |= (unsigned long long)(d & 255) << (j * 8);
        }
        __builtin_amdgcn_raw_buffer_store_b128((u32x4){call, (unsigned)pl, call, (unsigned)(pl >> 32)}, rwl, lane < NRANK ? WS_FLAG + 63 * 32 + (unsigned)lane * FLAGB : OOB, 0, AUX_SYS);
    };
    auto offs_of = [&](int i, unsigned (&o)[4], unsigned& cnt) -> unsigned {
        if (i < NPIN) {
            int q, pos; const int pan = unit_of(look_up(from, (i >> (6 - PSH)) + NPUSH), rank, q, pos);
            cnt = WS_CNT + (unsigned)(pan * NRANK + peer) * 128u + (unsigned)q * 4u;
            const unsigned sub = (unsigned)((i << PSH) & 63) * 1024u;
            const unsigned b = WS_PAN + (unsigned)pan * PANB + (unsigned)peer * CHB + (unsigned)q * UNB + sub + (unsigned)lane * 16u;
#pragma unroll
            for (int k = 0; k < 4; ++k) o[k] = b + k * 1024u;
            return WS_SND + (call & 1u) * SNDB + (unsigned)pos * UNB + sub;
        }
        int src = peer;
        if (ZB && i >= NPC) {
            int x = i - NPC;
            for (int j = 0; j < peer; ++j) x -= moved(j) < 0 ? moved(j) : 0;
            for (int j = 0; j < NRANK; ++j) { const int g = moved(j) > 0 ? moved(j) : 0; if ((unsigned)x < (unsigned)g) { src = j; i = NPC - g + x; } x -= g; }
        }
        const int e = (i - NPIN) / BPB, r = (i - NPIN) % BPB, pb = band_of(e, rank) * BPB + r;
        cnt = e < NRANK - 1 && r >= BPB - ZB ? WS_ZC + (unsigned)(band_of(e, rank) * NRANK + peer) * 128u : 0u;
#pragma unroll
        for (int k = 0; k < 4; ++k) {
            const unsigned sg = (unsigned)((pb << PSH) + k) * 2u + (unsigned)(lane >> 5);
            o[k] = (unsigned)src * ROWB + (unsigned)(lane & 31) * 16u + sg / NS * (NN * 2u) + sg % NS * 512u;
        }
        return WS_OSN + (unsigned)(pb << PSH) * 1024u;
    };
    if (cur < (NS - NPUSH) << (6 - PSH)) {
        const unsigned long long t0 = __builtin_amdgcn_s_memrealtime();
        const unsigned fo = fso + 2048u + (unsigned)(((cur >> 3) + 2 * NPUSH) * 2 + (cur >> 1 & 1)) * 8u;
        for (;;) {
            const unsigned f = __builtin_amdgcn_raw_buffer_load_b32(rwl, (unsigned)(lane & 1) * 4u, fo, AUX_SYS);
            if (__all((int)(f - call) >= 0)) { lim = flim = cur + 1; break; }
            if (give_up(tmo, call, t0)) { bad = true; planned = true; break; }
            __builtin_amdgcn_s_sleep(CFG_PS);
        }
        asm volatile("" ::: "memory");
    }
    asm volatile("" :: "v"(from));
    while (!planned || cur < fin) {
        if (!bad && cur >= lim) {
            const unsigned long long t0 = __builtin_amdgcn_s_memrealtime();
            const unsigned pvo = (unsigned)(lane - (first_of(cur) >> 3)) < CFG_PWIN || (lane == 63 && cur >= NPC - ZB) ? (unsigned)lane * 32u : OOB - 16u;
            for (;;) {
                const u32x4 f0 = __builtin_amdgcn_raw_buffer_load_b128(rwl, pvo, fso, AUX_SYS), f1 = __builtin_amdgcn_raw_buffer_load_b128(rwl, pvo + 16u, fso, AUX_SYS);
                update(f0, f1, cur);
                if (cur < lim || (planned && cur >= fin)) break;
                if (give_up(tmo, call, t0)) { bad = true; planned = true; plan = 0; fin = NPC; break; }
                __builtin_amdgcn_s_sleep(CFG_PS);
            }
            asm volatile("" ::: "memory");
            if (cur >= fin && planned) break;
        }
        unsigned o[4], p2 = OOB, p3 = OOB, cprev = 0;
        if (bad) {
            (void)offs_of(cur, o, cprev);
            if (cur >= NPIN) {
#pragma unroll
                for (int k = 0; k < PU; ++k) __builtin_amdgcn_raw_buffer_store_b128((u32x4){0x7fc07fc0u, 0x7fc07fc0u, 0x7fc07fc0u, 0x7fc07fc0u}, rol, o[k], 0, AUX_AGENT);
            }
            cur = index_of(draw()); continue;
        }
        STAMP1(P, 1);
        rsrc_t rp = rwl;
        auto put = [&](const u32x4& d, const rsrc_t r, unsigned a) { __builtin_amdgcn_raw_buffer_store_b128(d, r, a, 0, CFG_PST); };
        u32x4 v[4] = {};
        unsigned slot = 0;
        if (CFG_RATE) { const unsigned now = (unsigned)__builtin_amdgcn_s_memrealtime() << 4; slot = now - (now & 0xffffffu) % PACE16 + (unsigned)((widx & 7) * PW + (widx >> 3)) * (PACE16 / NW) - PACE16; }
        unsigned cb, src, in[4];
        auto prepare = [&]() {
            if (ZB && cur == NPC - CFG_PLAN) make_plan();
            src = __builtin_amdgcn_readfirstlane(offs_of(cur, o, cb)); cb = __builtin_amdgcn_readfirstlane(cb);
            if (ZB && cur >= NPC) src = 0;
#pragma unroll
            for (int k = 0; k < 4; ++k) { in[k] = ZB && cur >= NPC ? o[k] : lane * 16 + k * 1024; asm volatile("" : "+v"(o[k]), "+v"(in[k])); }
        };
        prepare();
        do {
            const bool tail = cur >= fin - CFG_TAIL, even = cur >= fin - (deep0 ? CFG_EVEND : CFG_EVEN), deep = deep0 || tail || even;
            const unsigned per = even ? CFG_ERATE * NW / 64 : deep0 ? PACE16 / 2 : PACE16;
            const rsrc_t rd = cur < NPIN ? rwl : rol, rs = ZB && cur >= NPC ? ror : rwr;
            const bool look = flim < NPC && cur + CFG_LOOK >= flim, lookp = !planned && cur + CFG_LOOKP >= NPC - ZB;
            const unsigned fvo = look || (lookp && lane == 63) ? (unsigned)lane * 32u : OOB - 16u;
            unsigned cdone = cprev;
            unsigned nv = 0; u32x4 f0, f1;
#pragma unroll
            for (int k = 0; k < 4; ++k) {
                if (CFG_RATE && !tail) {
                    slot += per;
                    if (CFG_GRID) {
                        unsigned go = (unsigned)__builtin_amdgcn_s_memrealtime();
                        const int d = (int)((go << 4) - slot);
                        if (d > CFG_CRED * 16) slot += (unsigned)(d - CFG_CRED * 16);
                        if (d < 0) go += (unsigned)(15 - d) >> 4;
                        go += CFG_GRID - (go & 0xffffffu) % (CFG_GRID ? CFG_GRID : 1);
                        for (;;) {
                            const int w = (int)(go - (unsigned)__builtin_amdgcn_s_memrealtime());
                            if (w <= 0) break;
                            if (w > CFG_SPIN) __builtin_amdgcn_s_sleep(1);
                        }
                    } else for (;;) {
                        const int d = (int)(((unsigned)__builtin_amdgcn_s_memrealtime() << 4) - slot);
                        if (d > CFG_CRED * 16) slot += (unsigned)(d - CFG_CRED * 16);
                        if (d >= 0) break;
                        __builtin_amdgcn_s_sleep(1);
                    }
                }
                if (CFG_FK && k == 2) { f0 = __builtin_amdgcn_raw_buffer_load_b128(rwl, fvo, fso, AUX_SYS); f1 = __builtin_amdgcn_raw_buffer_load_b128(rwl, fvo + 16u, fso, AUX_SYS); asm volatile("" ::: "memory"); }
                v[k] = __builtin_amdgcn_raw_buffer_load_b128(rs, in[k], src, CFG_RLD);
                if (k == 0) {
                    nv = draw();
                    if (!CFG_FK) { f0 = __builtin_amdgcn_raw_buffer_load_b128(rwl, fvo, fso, AUX_SYS); f1 = __builtin_amdgcn_raw_buffer_load_b128(rwl, fvo + 16u, fso, AUX_SYS); }
                }
                put(v[k ^ 2], k < 2 ? rp : rd, k == 0 ? p2 : k == 1 ? p3 : o[k ^ 2]);
                if (CFG_EARLY && !deep) {
                    unsigned& a = k == 0 ? p3 : o[k - 1];
                    put(v[(k + 3) & 3], k < 1 ? rp : rd, a);
                    a = OOB;
                }
                if (k == 3) {
                    asm volatile("" ::: "memory");
                    cprev = cb; rp = rd; p2 = o[2]; p3 = o[3];
                    cur = index_of(nv);
                    if (look || lookp) update(f0, f1, cur);
                    if (cur < lim) prepare();
                }
                if (k == 0) WAIT_VMN(CFG_FK ? 5 : 7); else if (k == 1) WAIT_VMN(CFG_FK ? 4 : 6); else if (k == 2) WAIT_VMN(CFG_FK ? 5 : 3); else WAIT_VMN(3);
                if (!deep) {
                    if (k == 0) WAIT_VMN((CFG_FK ? 2 : 4) + CFG_EARLY); else WAIT_VMN(1 + CFG_EARLY);
                    if (CFG_EARLY && k == 1) { arrive(cdone ? lane0 : OOB, cdone); cdone = 0; }
                }
            }
            arrive(cdone ? lane0 : OOB, cdone);
        } while (cur < lim);
        put(v[2], rp, p2);
        put(v[3], rp, p3);
        if (cprev) { WAIT_VM0(); arrive(lane0, cprev); }
    }
    STAMP(P, 4);
}

__device__ __forceinline__ void load_w(u32x4 (&v)[8], const rsrc_t rw, int gpan, int q, int job, int lane) {
    const unsigned vo = (unsigned)((lane >> 4) * 8) * (NN * 2) + (unsigned)(lane & 15) * 16u;
    const unsigned so = (unsigned)(q * 128 + (job >> 1) * 32) * (NN * 2) + (unsigned)(gpan * 256 + (job & 1) * 128) * 2u;
#pragma unroll
    for (int j = 0; j < 8; ++j) v[j] = __builtin_amdgcn_raw_buffer_load_b128(rw, vo, so + (unsigned)j * (NN * 2), 2);
}
template <bool twice>
__device__ __forceinline__ void store_w(const u32x4 (&v)[8], const rsrc_t rd, unsigned vo, unsigned vo2, unsigned unit, int job, bool slow) {
    unit += (unsigned)(job >> 2) * KTB + (unsigned)(job & 1) * 8192u + (unsigned)(job >> 1 & 1) * 1024u;
#pragma unroll
    for (int i = 0; i < 8; ++i) {
        u32x4 o;
#pragma unroll
        for (int e = 0; e < 4; ++e) o[e] = __builtin_amdgcn_perm(v[2 * e + 1][i >> 1], v[2 * e][i >> 1], (i & 1) ? 0x07060302u : 0x05040100u);
        STORE16(o, rd, vo, unit + (unsigned)(i >> 2) * HTB + (unsigned)(i & 3) * 2048u, AUX_SYS);
        if (twice) STORE16(o, rd, vo2, unit + (unsigned)(i >> 2) * HTB + (unsigned)(i & 3) * 2048u, CFG_OWN);
        if (CFG_PWND && i >= CFG_PWND - 1 && i < 7 && slow) WAIT_VMN(CFG_PWND ? CFG_PWND - 1 : 0);
    }
}
typedef __attribute__((address_space(1))) unsigned* gptr_t;

__device__ __forceinline__ void lead_role(const Params& P, int rank, unsigned long long wsp, unsigned tk, unsigned t0, int g, int wid, int lane) {
    const int o = NS - 1 - (g >> 5), c = (g >> 3) & 3, job = (c >> 1) * 4 + wid * 2 + (c & 1);
    const int dest = (rank + 1 + g + (o < NPUSH ? CFG_MIX * (2 * c + o) : 0)) & (NRANK - 1);
    u32x4 v[8];
    load_w(v, mk_rsrc(P.w, (unsigned)KSH * NN * 2), dest * NS + o, 0, job, lane);
    asm volatile("" : "+v"(tk), "+v"(wsp) :: "memory");
    const bool push = o < NPUSH && dest != rank, count = o < NPUSH || dest == rank;
    char* wsd = (char*)((unsigned long long)(unsigned)__builtin_amdgcn_readlane((unsigned)(wsp >> 32), dest) << 32 | (unsigned)__builtin_amdgcn_readlane((unsigned)wsp, dest));
    gptr_t fp = (gptr_t)(wsd + (count ? WS_CNT + (unsigned)((NTM + o) * NRANK + rank) * 128u : WS_FLAG + (unsigned)rank * FLAGB + 2048u + (unsigned)((o * 4 + c) * 2 + wid) * 4u));
    while (CFG_P3 && push && o >= CFG_P3N && (int)((unsigned)__builtin_amdgcn_s_memrealtime() - t0) < CFG_P3 + (o - CFG_P3N) * 110) __builtin_amdgcn_s_sleep(1);
    store_w<false>(v, mk_rsrc(push ? wsd : P.wsl, WS_END), (unsigned)lane * 16u + (push ? 0u : (~tk & 1u) * SNDB), 0u,
            push ? WS_PAN + (unsigned)(NTM + o) * PANB + (unsigned)rank * CHB : WS_SND + (unsigned)(dest * NS + o) * UNB, job, push);
    WAIT_VMN(0);
    STAMP(P, 6);
    if (lane == 0) {
        if (!count) __hip_atomic_store(fp, tk + 1u, __ATOMIC_RELAXED, __HIP_MEMORY_SCOPE_SYSTEM);
        else if ((__hip_atomic_fetch_add((gptr_t)(P.wsl + WS_LEAD + (o * NRANK + dest) * 64), 1u, __ATOMIC_RELAXED, __HIP_MEMORY_SCOPE_AGENT) & 7u) == 7u)
            __hip_atomic_store(fp, (tk + 1u) * CNT_FULL, __ATOMIC_RELAXED, __HIP_MEMORY_SCOPE_SYSTEM);
    }
}

__device__ __forceinline__ void pack_role(const Params& P, unsigned long long wsp, unsigned tk, unsigned t0, int g, int wid, int lane) {
    const int rank = P.rank;
    const rsrc_t rx = mk_rsrc(P.x, (unsigned)MM * KSH * 2), rw = mk_rsrc(P.w, (unsigned)KSH * NN * 2), rwl = mk_rsrc(P.wsl, WS_END);
    const unsigned sv = (unsigned)lane * 16u + (~tk & 1u) * SNDB;
    const unsigned to = ((const unsigned*)P.order.to)[lane];
    constexpr int NH = CFG_UH < NUNIT ? 8 * DK : 0, UH = NH ? CFG_UH : NUNIT;
    const bool late = g >= NPK - NH;
    constexpr int U1 = NRANK * NS, NE = CFG_EH ? NH : 0;
    static_assert(CFG_TH < (UH - U1 - NH) * CFG_PH, "unpaced");
    for (int u = late ? (NE ? U1 : UH) + g - (NPK - NH) : U1 + NE + g; u < (late ? NUNIT : UH); u += !late ? NPK - NH : u < UH ? UH - U1 : NH) {
        int o, dest = -1;
        if (u < 4 * U0) {
            const int r = u % U0;
            if (r < NRANK * NS) { o = r >> 3; dest = (rank + 1 + r) & (NRANK - 1); } else o = r - (NRANK - 1) * NS;
            o += u / U0 * G0;
        } else o = u - 4 * (NRANK - 1) * NS;
        int q, pos; const int pan = unit_of(o, dest < 0 ? 0 : dest, q, pos);
        while ((int)((unsigned)__builtin_amdgcn_s_memrealtime() - t0) < (late ? CFG_TH + (u - UH) * CFG_PH : (u - CFG_PACE0) * CFG_PACE / 10)) __builtin_amdgcn_s_sleep(1);
        const unsigned dst = WS_SND + (unsigned)pos * UNB;
        u32x4 v[8];
        if (pan < NTM) {
            const int rb0 = (wid & 3) * 4;
            const unsigned vo = (unsigned)(lane & 15) * (KSH * 2) + (unsigned)(lane >> 4) * 16u;
            const unsigned so = (unsigned)(pan * 256 + rb0 * 16) * (KSH * 2) + (unsigned)(q * 2 + (wid >> 2)) * 128u;
#pragma unroll
            for (int i = 0; i < 8; ++i) v[i] = __builtin_amdgcn_raw_buffer_load_b128(rx, vo + (i & 1) * 64, so + (unsigned)(i >> 1) * (16 * KSH * 2), 0);
#pragma unroll
            for (int i = 0; i < 8; ++i) STORE16(v[i], rwl, sv + (i & 1) * 1024, dst + (unsigned)(wid >> 2) * KTB + (unsigned)(rb0 + (i >> 1)) * 2048u, AUX_SYS);
        } else {
            load_w(v, rw, gpan_of(pan - NTM, dest < 0 ? 0 : dest), q, wid, lane);
            if (pan - NTM >= CFG_VD) store_w<true>(v, rwl, sv, WS_PAN + (unsigned)pan * PANB + (unsigned)rank * CHB + (unsigned)q * UNB - dst + (unsigned)lane * 16u, dst, wid, false);
            else store_w<false>(v, rwl, sv, 0u, dst, wid, false);
        }
        o = look_up(to, o);
        const bool own = lane == rank;
        unsigned long long fp = wsp + (own ? WS_CNT + (unsigned)((pan * NRANK + rank) * 32 + q) * 4u : WS_FLAG + (unsigned)rank * FLAGB + (unsigned)o * 4u);
        unsigned fv = own ? (tk + 1u) * CNT_FULL : tk + 1u;
        asm volatile("" : "+v"(fp), "+v"(fv));
        WAIT_VMN(0);
        __builtin_amdgcn_s_barrier();
        const bool mine = lane < NRANK && (dest < 0 || dest == lane);
        if (wid == 0 && mine && (own || o < CFG_AGG || o >= NIN * 4 / 8 * 8)) __hip_atomic_store((gptr_t)fp, fv, __ATOMIC_RELAXED, __HIP_MEMORY_SCOPE_SYSTEM);
        if (wid == 1 && o >= CFG_AGG && o < NIN * 4 / 8 * 8) {
            unsigned old = 0;
            if (mine && !own) old = __hip_atomic_fetch_add((gptr_t)(P.wsl + WS_GRP) + (o >> 3) * NRANK + lane, 1u, __ATOMIC_RELAXED, __HIP_MEMORY_SCOPE_AGENT);
            const unsigned full = (unsigned)__ballot((old & 7u) == 7u);
            const unsigned long long wsq = (unsigned long long)(unsigned)__builtin_amdgcn_ds_bpermute(lane >> 3 << 2, (int)(wsp >> 32)) << 32 | (unsigned)__builtin_amdgcn_ds_bpermute(lane >> 3 << 2, (int)wsp);
            if (full >> (lane >> 3) & 1) __hip_atomic_store((gptr_t)(wsq + WS_FLAG + (unsigned)rank * FLAGB) + (o & ~7) + (lane & 7), tk + 1u, __ATOMIC_RELAXED, __HIP_MEMORY_SCOPE_SYSTEM);
        }
    }
}

struct Tile { int tm, v, half; unsigned sA, stA; };
__device__ __forceinline__ void tile_of(int idx, int col, int rank, unsigned snd, Tile& t) {
    const int v = idx < 2 * GW ? (idx >> 2) + (col >> 2) * (GW / 2) : 2 * (idx >> 2) + (col >> 2);
    t.tm = 4 * (col & 3) + (idx & 3); t.v = v; t.half = -1;
    const int g = (v - GM) / GW, i = (v - GM) % GW;
    t.sA = snd + (unsigned)(v < GW ? gpan_of(v, rank) : v < GW + GM ? 4 * U0 + v - GW : 4 * U0 + 4 * GM + (g - 1) * 4 * GW + i) * UNB;
    t.stA = (unsigned)(v < GW ? U0 : v < GW + GM ? GM : g <= NGF ? GW : GL) * UNB;
    if (v >= CFG_VD) { t.sA = WS_PAN + (unsigned)(NTM + v) * PANB + (unsigned)rank * CHB; t.stA = UNB; }
}
__device__ __forceinline__ void item_of(int q, int rank, unsigned snd, Tile& t) {
    const int n = q < PFULL ? q : PFULL + ((q - PFULL) >> 1);
    if (n < 4 * PL0) tile_of(RS * KX - DK + n % PL0, n / PL0, rank, snd, t);
    else tile_of(RS * KX - DK + (n - 4 * PL0) % PL1, 4 + (n - 4 * PL0) / PL1, rank, snd, t);
    if (q >= PFULL) t.half = (q - PFULL) & 1;
}

template <int wr>
__device__ __forceinline__ void gemm_role(const Params& P, unsigned call, int col, int slot, LAS unsigned char* lds) {
    const int tid = threadIdx.x, wid = __builtin_amdgcn_readfirstlane(tid >> 6), lane = tid & 63, wc = wid & 3;
    const int rank = P.rank;
    char* wsl = P.wsl;
    const rsrc_t rws = mk_rsrc(wsl, WS_END);
    LAS unsigned* ctl = (LAS unsigned*)(lds + STAGE_BYTES);
    const unsigned voff = (unsigned)tid * 16u;
    const unsigned ldsw = (unsigned)wid * 1024u;
    const int aoff = wr * 8192 + lane * 16, boff = wc * 4096 + lane * 16;

    auto lane_now = [&]() { int tz = threadIdx.x; asm volatile("" : "+v"(tz)); return tz & 63; };
    auto cnt_of = [&](const Tile& t) { const int ln = lane_now(); return WS_CNT + (unsigned)((((ln & 32) ? NTM + t.v : t.tm) * NRANK + ((ln >> 2) & 7)) * 32 + (ln & 3)) * 4u; };
    auto level_of = [&](unsigned c) -> int {
        const unsigned long long miss = __ballot((int)(c - call * CNT_FULL) < 0);
        unsigned m = (unsigned)miss | (unsigned)(miss >> 32);
        const bool own = ((m >> (rank * 4)) & 15u) == 0u;
        m |= m >> 16; m |= m >> 8; m |= m >> 4;
        return own ? 1 + __builtin_ctz(m | 16u) : 0;
    };
    auto ready = [&](const Tile& t, int want) -> int {
        if (wr == 0 && wid == 0) {
            const unsigned* c = (const unsigned*)(wsl + cnt_of(t));
            unsigned* tmo = (unsigned*)(wsl + WS_TMO);
            const unsigned long long t0 = __builtin_amdgcn_s_memrealtime();
            int got;
            for (;;) {
                got = level_of(ld_sys(c));
                if (got >= want) break;
                if (give_up(tmo, call, t0)) { ctl[0] = 1u; got = 5; break; }
                __builtin_amdgcn_s_sleep(4);
            }
            ctl[2] = (unsigned)got;
            __builtin_amdgcn_fence(__ATOMIC_ACQUIRE, "agent");
            asm volatile("s_waitcnt vmcnt(0) lgkmcnt(0)" ::: "memory");
        }
        asm volatile("" ::: "memory"); __builtin_amdgcn_s_barrier(); asm volatile("" ::: "memory");
        const int r = __builtin_amdgcn_readfirstlane(ctl[2]);
        asm volatile("s_waitcnt lgkmcnt(0)" ::: "memory");
        return r;
    };

#define SA(b, h) (((b) * 2 + (h)) * HTB)
#define SB(b, h) ((4 + (b) * 2 + (h)) * HTB)
#define STAGE(bufoff, soff) do { _Pragma("unroll") for (int _i = 0; _i < 2; ++_i) \
        __builtin_amdgcn_raw_ptr_buffer_load_lds(rws, (LAS unsigned*)(lds + (bufoff) + ldsw + _i * 8192), 16, voff, (soff) + _i * 8192u, 0, CFG_LDA); } while (0)
#define LDA(dst, b, h) do { _Pragma("unroll") for (int m = 0; m < 4; ++m) _Pragma("unroll") for (int k = 0; k < 2; ++k) dst[m][k] = *(const LAS bf16x8*)(lds + SA(b, h) + aoff + m * 2048 + k * 1024); } while (0)
#define LDB(dst, b, h) do { _Pragma("unroll") for (int n = 0; n < 2; ++n) _Pragma("unroll") for (int k = 0; k < 2; ++k) dst[n][k] = *(const LAS bf16x8*)(lds + SB(b, h) + boff + n * 2048 + k * 1024); } while (0)
#define MMA(ai, first, prio) do { __builtin_amdgcn_s_setprio(prio); \
        _Pragma("unroll") for (int i = 0; i < 32; ++i) { \
            const int bj = ((i >> 2) ^ (i >> 3)) & 1, m = i >> 3, n = ((i >> 1) ^ (i >> 3)) & 1, k = (i ^ (i >> 1)) & 1;     \
            SCHED; \
            acc[ai][bj][m][n] = __builtin_amdgcn_mfma_f32_16x16x32_bf16(bj ? B1[n][k] : B0[n][k], At[m][k], first && (i & 1) == 0 ? (f32x4){0.f, 0.f, 0.f, 0.f} : acc[ai][bj][m][n], 0, 0, 0); } \
        __builtin_amdgcn_s_setprio(0); } while (0)
#define WAIT_L0() do { asm volatile("" ::: "memory"); __builtin_amdgcn_s_waitcnt(0xC07F); asm volatile("" ::: "memory"); } while (0)
#define WAIT_V(n) asm volatile("s_waitcnt vmcnt(" #n ")" ::: "memory")
#define WAIT_L(n) asm volatile("s_waitcnt lgkmcnt(" #n ")" ::: "memory")
#define BAR __builtin_amdgcn_s_barrier()
#define SCHED __builtin_amdgcn_sched_barrier(0)

    unsigned long long outp = (unsigned long long)P.outl;
    asm volatile("" : "+v"(outp));
    Tile cur, nxt; int rnd = slot < KX - DK ? 0 : 1;
    const unsigned snd = WS_SND + (call & 1u) * SNDB;
    tile_of(rnd * (KX - DK) + slot, col, rank, snd, cur);
    bf16x8 At[4][2], B0[2][2], B1[2][2];
    constexpr unsigned SB0 = ((NRANK - 1) * NS + GW) * UNB, STB = U0 * UNB;
    const unsigned kov = WS_PAN + (unsigned)((rank + 1 + (lane + 3) % 7) & (NRANK - 1)) * CHB + (unsigned)(((lane & 31) - 4) / 7) * UNB;
    unsigned tabA, tabB;
    auto tables = [&](const Tile& t) {
        const int ln = lane_now();
        const unsigned la = t.sA + (unsigned)ln * t.stA, ra = (unsigned)(NTM + t.v) * PANB + kov;
        const unsigned lb = SB0 + snd + (unsigned)t.tm * UNB + (unsigned)ln * STB, rb = (unsigned)t.tm * PANB + kov;
        tabA = ln < 4 ? la : ra;
        tabB = (ln < 4 ? lb : rb) + (t.half == 1 ? HTB : 0u);
    };
    auto pick = [&]() {
        Tile t = cur; bool ok = true;
        if (rnd + 1 < RS) tile_of((rnd + 1) * KX - DK + slot, col, rank, snd, t);
        else {
            const unsigned d = (unsigned)__builtin_amdgcn_raw_ptr_buffer_atomic_add_i32(1, rws, lane_now() == 0 ? 0u : 0xfffffff0u, WS_HEAD + (unsigned)rank * 128u, 0);
            const unsigned q = (unsigned)__builtin_amdgcn_readfirstlane(d) - (call - 1u) * (unsigned)(NITEM + 8 * KX);
            ok = q < (unsigned)NITEM;
            if (ok) item_of((int)q, rank, snd, t);
        }
        *(LAS u32x4*)(ctl + 4) = (u32x4){ok ? (unsigned)(t.tm | t.v << 8 | (t.half + 2) << 16) : 0u, t.sA, t.stA, 0u};
        __builtin_amdgcn_raw_ptr_buffer_load_lds(rws, ctl + 64, 4, cnt_of(t), 0, 0, AUX_SYS);
    };
    auto picked = [&](int& level) -> bool {
        const u32x4 f = *(const LAS u32x4*)(ctl + 4);
        level = level_of(ctl[64 + lane_now()]);
        const unsigned w = __builtin_amdgcn_readfirstlane(f[0]);
        nxt.tm = w & 255u; nxt.v = (w >> 8) & 255u; nxt.half = (int)(w >> 16) - 2;
        nxt.sA = __builtin_amdgcn_readfirstlane(f[1]); nxt.stA = __builtin_amdgcn_readfirstlane(f[2]);
        return w != 0u;
    };
    constexpr int JP = KT / 2 - CFG_JP, JV = JP + 1, JI = JP + 2, JL = KT / 2 - 1;
    static_assert(JP > 24 && JI < JL, "passes");
    const int jend = wr == 0 && wid == 0 ? JP : JI;
    if (CFG_SYNC && rnd && wr == 0 && wid == 0) {
        const unsigned long long t0 = __builtin_amdgcn_s_memrealtime();
        while (__builtin_amdgcn_readfirstlane(ld_agent((const unsigned*)(wsl + WS_COL + col * 128))) != call && !give_up((unsigned*)(wsl + WS_TMO), call, t0)) __builtin_amdgcn_s_sleep(16);
    }
    int lvl = ready(cur, 1);
    STAMP(P, 2);
    tables(cur);
    unsigned a2 = cur.sA, b2 = SB0 + snd + (unsigned)cur.tm * UNB;
    STAGE(SB(0, 0), b2); STAGE(SA(0, 0), a2); STAGE(SB(0, 1), b2 + HTB); STAGE(SA(0, 1), a2 + HTB);
    STAGE(SB(1, 0), b2 + KTB); STAGE(SA(1, 0), a2 + KTB); STAGE(SB(1, 1), b2 + HTB + KTB);
    WAIT_V(0); BAR;
#define TICK(i, r) do {} while (0)
#define ROUND_DONE() do {} while (0)
    bool has_next = false;
#pragma unroll 1
    for (;;) {
        f32x4 acc[2][2][4][2];
        int jchk = lvl < 5 ? 7 * lvl - 4 : jend;
        TICK(0, 2); TICK(4, 3);
#define SYNC_MMA(ai, first, wv) do { if (wv) WAIT_V(8); WAIT_L0(); if (wr == 1) BAR; MMA(ai, first, wr + 1); if (wr == 0) BAR; SCHED; } while (0)
#define PASS(first, check) do { \
            LDB(B0, 0, 0); LDA(At, 0, 0); LDB(B1, 0, 1); STAGE(SA(1, 1), a2 + KTB + HTB); SCHED; \
            check; \
            a2 = __builtin_amdgcn_readlane(tabA, (j + 1) & 31); b2 = __builtin_amdgcn_readlane(tabB, (j + 1) & 31); \
            SYNC_MMA(0, first, !first); \
            LDA(At, 0, 1); STAGE(SB(0, 0), b2); STAGE(SA(0, 0), a2); STAGE(SB(0, 1), b2 + HTB); \
            SYNC_MMA(1, first, !first); \
            LDB(B0, 1, 0); LDA(At, 1, 0); LDB(B1, 1, 1); STAGE(SA(0, 1), a2 + HTB); \
            SYNC_MMA(0, false, true); \
            LDA(At, 1, 1); STAGE(SB(1, 0), b2 + KTB); STAGE(SA(1, 0), a2 + KTB); STAGE(SB(1, 1), b2 + KTB + HTB); \
            SYNC_MMA(1, false, true); } while (0)
        int j = 0;
        PASS(true, );
#pragma unroll 1
        for (j = 1; j < KT / 2; ++j)
            PASS(false, if (__builtin_expect(j == jchk, 0)) {
                    if (j < JP) { lvl = ready(cur, lvl + 1); jchk = lvl < 5 ? 7 * lvl - 4 : jend; }
                    else if (wr == 0 && j == JP) { pick(); jchk = JV; }
                    else if (wr == 0 && j == JV) { asm volatile("buffer_inv sc1" ::: "memory"); jchk = JI; }
                    else if (j == JI) {
                        has_next = picked(lvl); jchk = JL;
                        if (has_next && lvl >= 1) {
                            unsigned lines = voff; asm volatile("" : "+v"(lines)); lines <<= 3;
                            if (CFG_PFA && nxt.v < CFG_VD) __builtin_amdgcn_raw_ptr_buffer_load_lds(rws, ctl + 128, 4, lines, nxt.sA + (unsigned)(nxt.tm & 3) * nxt.stA, 0, 0);
                            if (CFG_PFB == 2 || (CFG_PFB && rnd == 0)) __builtin_amdgcn_raw_ptr_buffer_load_lds(rws, ctl + 128, 4, lines, SB0 + snd + (unsigned)nxt.tm * UNB + (unsigned)(nxt.v >> 1 & 3) * STB, 0, 0);
                        }
                    } else if (has_next) {
                        const int want = nxt.half < 0 ? 1 : 5;
                        if (lvl < want) lvl = ready(nxt, want);
                        tables(nxt);
                    }
                });
        if (has_next && nxt.half >= 0) STAGE(SA(1, 1), a2 + KTB + HTB);
        TICK(1, 2);
        TICK(2, 2);
        {
            const int ln = lane_now(), fr = ln & 15, fq = ln >> 4;
            const rsrc_t rol = mk_rsrc((const void*)outp, OUTB);
            const bool shared = cur.v < NS;
            if (__builtin_amdgcn_readfirstlane(ctl[0]) != 0u) {
#pragma unroll
                for (int i = 0; i < 32; ++i) acc[i >> 4][(i >> 3) & 1][(i >> 1) & 3][i & 1] = __builtin_bit_cast(f32x4, (u32x4){0x7fc00000u, 0x7fc00000u, 0x7fc00000u, 0x7fc00000u});
            }
            if (rnd == 0 && cur.v < GW) {
                const unsigned long long t0 = __builtin_amdgcn_s_memrealtime();
                while (__builtin_amdgcn_s_memrealtime() - t0 < (unsigned long long)((col & 3) * CFG_STAG + (col >> 2) * CFG_STAGB)) __builtin_amdgcn_s_sleep(8);
                if (DK && CFG_SYNC && wr == 0 && slot == 0 && wid == 0) __builtin_amdgcn_raw_buffer_store_b32(call, rws, ln == 0 ? 0u : 0xfffffff0u, WS_COL + (unsigned)col * 128u, AUX_AGENT);
            }
            const bool thru = rnd >= RS;
            const unsigned vo = (unsigned)(wc * 32 + fq * 4) * (NN * 2) + (unsigned)(wr * 256 + fr * 16);
            const unsigned so = (unsigned)(cur.tm * 256) * (NN * 2) + (unsigned)gpan_of(cur.v, rank) * 512u;
            const unsigned vs = (unsigned)(wc * 32 + fq * 4) * ROWB + (unsigned)(wr * 256 + fr * 16);
            const unsigned ss = WS_OSN + (unsigned)(cur.tm * 256) * ROWB + (unsigned)cur.v * 512u;
            auto finish = [&](auto mode) {
#pragma unroll
                for (int bj = 0; bj < 2; ++bj)
#pragma unroll
                    for (int n = 0; n < 2; ++n)
#pragma unroll
                        for (int r = 0; r < 4; ++r) {
                            u32x4 o;
                            if ((r & 1) == 0) {
#pragma unroll
                                for (int i = 0; i < 8; ++i) { f32x4& a = acc[i >> 2][bj][i & 3][n]; const f32x2 z = silu2((f32x2){a[r], a[r + 1]}); a[r] = z[0]; a[r + 1] = z[1]; }
                            }
#pragma unroll
                            for (int i = 0; i < 4; ++i) o[i] = cvt_pk_bf16(acc[i >> 1][bj][(i & 1) * 2][n][r], acc[i >> 1][bj][(i & 1) * 2 + 1][n][r]);
                            if (mode == 1) STORE16(o, rws, vs, ss + (unsigned)(bj * 128 + n * 16 + r) * ROWB, AUX_SYS);
                            STORE16(o, rol, vo, so + (unsigned)(bj * 128 + n * 16 + r) * (NN * 2), mode == 2 ? CFG_OTH : CFG_OST);
                        }
            };
            if (shared) finish(std::integral_constant<int, 1>());
            else if (thru) finish(std::integral_constant<int, 2>());
            else finish(std::integral_constant<int, 0>());
            WAIT_V(16);
            if (shared) {
                WAIT_VM0();
                unsigned old = 0;
                if (ln == 0) old = __hip_atomic_fetch_add(ctl + 1, 1u, __ATOMIC_RELAXED, __HIP_MEMORY_SCOPE_WORKGROUP);
                old = __builtin_amdgcn_readfirstlane(old);
                if (!CFG_AGB) {
                    if ((old & 7u) == 7u && ln < NRANK && ln != rank)
                        st_sys((unsigned*)(P.ws[ln] + WS_FLAG + rank * FLAGB) + NIN * 4 + turn_of(cur.tm, ln) * NS + cur.v, call);
                } else if ((old & 7u) == 7u) {
                    const unsigned n = (unsigned)__builtin_amdgcn_raw_ptr_buffer_atomic_add_i32(1, rws, ln == 0 ? 0u : 0xfffffff0u, WS_BAND + (unsigned)cur.tm * 128u, 0);
                    if ((unsigned)__builtin_amdgcn_readfirstlane(n) + 1u == call * NS && (ln >> 3) != rank && (ln & 7) < NS)
                        st_sys((unsigned*)(P.ws[ln >> 3] + WS_FLAG + rank * FLAGB) + NIN * 4 + turn_of(cur.tm, ln >> 3) * NS + (ln & 7), call);
                }
            }
        }
        TICK(3, 2);
        ROUND_DONE();
        if (!has_next) break;
        cur = nxt; ++rnd;
        if (cur.half >= 0) break;
    }
#define HB(b, i) ((b) < 2 ? ((i) < 2 ? SA(b, i) : SB(b, 0)) : (i) < 2 ? SB(i, 1) : 8 * HTB)
#define HSTAGE(b, t) do { const unsigned _o = ((t) & 1) * KTB, _a = __builtin_amdgcn_readlane(tabA, ((t) >> 1) & 31) + _o, _b = __builtin_amdgcn_readlane(tabB, ((t) >> 1) & 31) + _o; \
        STAGE(HB(b, 0), _a); STAGE(HB(b, 1), _a + HTB); STAGE(HB(b, 2), _b); } while (0)
#define HLOAD(b) do { _Pragma("unroll") for (int i = 0; i < 4; ++i) B0[i >> 1][i & 1] = *(const LAS bf16x8*)(lds + HB(b, 2) + boff + i * 1024); \
        _Pragma("unroll") for (int i = 0; i < 8; ++i) At[i >> 1][i & 1] = *(const LAS bf16x8*)(lds + HB(b, 0) + aoff + i * 1024); \
        _Pragma("unroll") for (int i = 0; i < 8; ++i) Au[i >> 1][i & 1] = *(const LAS bf16x8*)(lds + HB(b, 1) + aoff + i * 1024); } while (0)
#define HPASS(b, t, first) do { \
        if (wr == 0) { HLOAD(b); HSTAGE((b + 2) % 3, (t) + 2); } else { HSTAGE((b + 2) % 3, (t) + 2); SCHED; HLOAD(b); if (!first) WAIT_V(6); } \
        WAIT_L0(); if (wr == 1) BAR; \
        __builtin_amdgcn_s_setprio(wr + 1); \
        _Pragma("unroll") for (int i = 0; i < 32; ++i) { const int ai = i >> 4, m = (i >> 2) & 3, n = (i >> 1) & 1, k = i & 1; \
            acc[ai][m][n] = __builtin_amdgcn_mfma_f32_16x16x32_bf16(B0[n][k], ai ? Au[m][k] : At[m][k], first && k == 0 ? (f32x4){0.f, 0.f, 0.f, 0.f} : acc[ai][m][n], 0, 0, 0); } \
        __builtin_amdgcn_s_setprio(0); \
        if (wr == 0) { if (!first) WAIT_V(6); BAR; } SCHED; } while (0)
#pragma unroll 1
    while (PFULL < PN && has_next) {
        f32x4 acc[2][4][2];
        bf16x8 Au[4][2];
        BAR;
        HPASS(0, 0, true);
#pragma unroll 1
        for (int t = 1; t < KT; t += 3) {
            if (wr == 0 && wid == 0 && t == KT - 9) pick();
            HPASS(1, t, false); HPASS(2, t + 1, false); HPASS(0, t + 2, false);
        }
        has_next = picked(lvl);
        if (has_next) {
            (void)ready(nxt, 5);
            tables(nxt);
            HSTAGE(0, 0); HSTAGE(1, 1);
        }
        const int ln = lane_now(), fr = ln & 15, fq = ln >> 4;
        const rsrc_t rol = mk_rsrc((const void*)outp, OUTB);
        if (__builtin_amdgcn_readfirstlane(ctl[0]) != 0u) {
#pragma unroll
            for (int i = 0; i < 16; ++i) acc[i >> 3][(i >> 1) & 3][i & 1] = __builtin_bit_cast(f32x4, (u32x4){0x7fc00000u, 0x7fc00000u, 0x7fc00000u, 0x7fc00000u});
        }
        const unsigned vo = (unsigned)(wc * 32 + fq * 4) * (NN * 2) + (unsigned)(wr * 256 + fr * 16);
        const unsigned so = (unsigned)(cur.tm * 256 + cur.half * 128) * (NN * 2) + (unsigned)gpan_of(cur.v, rank) * 512u;
#pragma unroll
        for (int n = 0; n < 2; ++n)
#pragma unroll
            for (int r = 0; r < 4; ++r) {
                u32x4 o;
                if ((r & 1) == 0) {
#pragma unroll
                    for (int i = 0; i < 8; ++i) { f32x4& a = acc[i >> 2][i & 3][n]; const f32x2 z = silu2((f32x2){a[r], a[r + 1]}); a[r] = z[0]; a[r + 1] = z[1]; }
                }
#pragma unroll
                for (int i = 0; i < 4; ++i) o[i] = cvt_pk_bf16(acc[i >> 1][(i & 1) * 2][n][r], acc[i >> 1][(i & 1) * 2 + 1][n][r]);
                STORE16(o, rol, vo, so + (unsigned)(n * 16 + r) * (NN * 2), CFG_OTH);
            }
        WAIT_V(8);
        ROUND_DONE();
        cur = nxt; ++rnd;
    }
    WAIT_V(0);
}

__global__ __launch_bounds__(512, 2) void fused_gemm_ar(Params P) {
    extern __shared__ __attribute__((aligned(16))) unsigned char shm[];
    LAS unsigned char* lds = (LAS unsigned char*)shm;
    const int wid = __builtin_amdgcn_readfirstlane(threadIdx.x >> 6), lane = threadIdx.x & 63;
    const int b = __builtin_amdgcn_readfirstlane(blockIdx.x);
    const unsigned long long wsp = (unsigned long long)P.ws[lane & (NRANK - 1)];
    const unsigned t0 = (unsigned)__builtin_amdgcn_s_memrealtime();
    int rk = P.rank;
    asm volatile("" : "+s"(rk) : "s"(P.x), "s"(P.w), "s"(P.wsl), "s"(P.rank), "s"(t0) : "memory");
    unsigned* tkp = (unsigned*)(P.wsl + WS_TICKET + b * 128 + wid * 16);
    const unsigned tk = ld_agent(tkp);
    if (b < NPULL) {
        const unsigned call = __builtin_amdgcn_readfirstlane(tk) + 1u;
        st_agent(tkp, call);
        const int pi = b / PW, slot = b % PW;
        const int peer = (P.rank + 1 + pi) & (NRANK - 1);
        pull_role(P, call, peer, slot * 8 + wid, lane);
    } else {
        const int g = b - NPULL;
        if (g < 4 * NRANK * NS && wid < 2) lead_role(P, rk, wsp, tk, t0, g, wid, lane);
        ((LAS unsigned*)(lds + STAGE_BYTES))[lane & 3] = 0u;
        pack_role(P, wsp, tk, t0, g, wid, lane);
        const unsigned call = __builtin_amdgcn_readfirstlane(tk) + 1u;
        st_agent(tkp, call);
        STAMP(P, 1);
        asm volatile("s_waitcnt lgkmcnt(0)" ::: "memory");
        __builtin_amdgcn_s_barrier();
        if (b >= NPS) {
            const int col = (CFG_SWAP >> (b & 7) * 4) & 7;
            if (wid < 4) gemm_role<0>(P, call, col, (b - NPS) >> 3, lds);
            else gemm_role<1>(P, call, col, (b - NPS) >> 3, lds);
        }
    }
}
}

extern "C" void kernel_launch(int rank, int nranks, ncclComm_t comm,
                              hipStream_t stream,
                              void* const* d_in, const int* in_sizes, int n_in,
                              void* d_out, int out_size,
                              const int* in_dtypes, int out_dtype,
                              void* const* d_out_all,
                              void* const* d_ws_all, size_t ws_bytes) {
    if (nranks != NRANK || (unsigned)rank >= (unsigned)NRANK || ws_bytes < (size_t)WS_END || n_in < 2 ||
        in_sizes[0] != MM * KSH || in_sizes[1] != KSH * NN || out_size != MM * NN) {
        fprintf(stderr, "unexpected problem geometry\n");
        return;
    }
    static bool attr_done[NRANK] = {};
    if (!attr_done[rank]) {
        HIP_CHECK(hipFuncSetAttribute((const void*)fused_gemm_ar, hipFuncAttributeMaxDynamicSharedMemorySize, LDS_BYTES));
        attr_done[rank] = true;
    }
    Params P = {};
    P.order = ORDER;
    P.x = d_in[0]; P.w = d_in[1]; P.rank = rank; P.wsl = (char*)d_ws_all[rank]; P.outl = (char*)d_out_all[rank];
    for (int p = 0; p < NRANK; ++p) { P.out[p] = (char*)d_out_all[p]; P.ws[p] = (char*)d_ws_all[p]; }
    hipLaunchKernelGGL(fused_gemm_ar, dim3(GRID), dim3(512), LDS_BYTES, stream, P);
    HIP_CHECK(hipGetLastError());
}
```

```cpp
#include <hip/hip_runtime.h>
#include <rccl/rccl.h>
#include <cstdio>
#include <type_traits>

#define HIP_CHECK(expr) do { hipError_t _e = (expr); if (_e != hipSuccess) { \
    fprintf(stderr, "HIP error %s at %s:%d\n", hipGetErrorString(_e), __FILE__, __LINE__); \
    return; } } while (0)

#define LAS __attribute__((address_space(3)))
typedef short bf16x8 __attribute__((ext_vector_type(8)));
typedef float f32x4 __attribute__((ext_vector_type(4)));
typedef unsigned u32x4 __attribute__((ext_vector_type(4)));
typedef __amdgpu_buffer_rsrc_t rsrc_t;

namespace {
constexpr int NRANK = 8;
constexpr int MM = 4096, KK = 4096, NN = 32768, KSH = KK / NRANK;
constexpr int KT = KK / 64, KTS = KSH / 64, NTM = MM / 256, NPAN = NN / 256;
#define CFG_NS 6
#define CFG_PW 8
constexpr int NS = CFG_NS;
constexpr int NRP = NPAN - NRANK * NS;
constexpr int NWP = NS + NRP;
constexpr int NIN = NTM + NWP;
constexpr int NOUT = NS * NTM;
constexpr int NFLAG = NIN * 4 + NTM;
constexpr int NUNIT = NTM * 4 + NPAN * 4;
constexpr int GRID = 256;
constexpr int PW = CFG_PW;
constexpr int NPULL = PW * (NRANK - 1);
constexpr int NW = PW * 8;
constexpr int NPK = GRID - NPULL;
constexpr int NPS = (NPULL + 7) / 8 * 8;
constexpr int KX = (GRID - NPS) / 8;
#define CFG_GW (2 * (KX / 4))
constexpr int GW = CFG_GW;
constexpr int G0 = GW + NTM;
#define CFG_SKIP 1
#define CFG_SYNC 1
#define CFG_GM (CFG_SKIP ? GW + 2 : 2)
constexpr int GM = CFG_GM;
constexpr int NGF = (NWP - GW - GM) / GW, GL = (NWP - GW - GM) % GW;
static_assert(NS <= GW && NWP > GW, "shared panels sit in the first group");
constexpr int PU = 4, PSH = 2;
#define CFG_PUSH 2
constexpr int NPUSH = CFG_PUSH;
constexpr int NPIN = (NIN * 4 - NPUSH) * 64 / PU;
static_assert(NPUSH <= NS, "pushed units have a single consumer");
constexpr int NPC = NPIN + NOUT * 128 / PU;
constexpr unsigned CNT_FULL = 64;

constexpr int NT0 = 4 * ((NWP + 1) / 2), NT1 = 4 * (NWP / 2);
constexpr int DK = CFG_SKIP ? KX - 2 * GW : 0;
constexpr int RS = (NT1 - 1 + DK) / KX;
constexpr int PL0 = NT0 - RS * KX + DK, PL1 = NT1 - RS * KX + DK, PN = 4 * (PL0 + PL1);
#define CFG_SWAP 0x57461302u
#define CFG_FD (CFG_SKIP ? 0 : -8)
constexpr int PFULL = 2 * PN - 8 * KX - CFG_FD < 0 ? 0 : 2 * PN - 8 * KX - CFG_FD > PN ? PN : 2 * PN - 8 * KX - CFG_FD;
constexpr int NITEM = PFULL + 2 * (PN - PFULL);
constexpr unsigned KTB = 32768, HTB = 16384;
constexpr unsigned PANB = KT * KTB;
constexpr unsigned CHB = KTS * KTB;
constexpr unsigned UNB = 2 * KTB;
constexpr unsigned WS_TICKET = 0;
constexpr unsigned WS_TMO = WS_TICKET + GRID * 128;
constexpr unsigned WS_FLAG = WS_TMO + 128;
constexpr unsigned FLAGB = 4096;
constexpr unsigned WS_LEAD = WS_FLAG + NRANK * FLAGB;
constexpr unsigned WS_HEAD = WS_LEAD + NRANK * 512;
constexpr unsigned WS_COL = WS_HEAD + NRANK * 128;
#define CFG_PHOFF 0
constexpr unsigned WS_PH = WS_COL + 8 * 128 + CFG_PHOFF;
constexpr unsigned WS_ZC = WS_PH + 2 * NRANK * 128 + 128;
constexpr unsigned WS_GRP = WS_ZC + NRANK * NRANK * 128;
constexpr unsigned WS_BAND = WS_GRP + 64 * NRANK * 4;
constexpr unsigned WS_CNT = WS_BAND + NTM * 128;
constexpr unsigned WS_CTRL_END = WS_CNT + NIN * NRANK * 128;
constexpr unsigned WS_DBG = 512u << 10;
constexpr unsigned WS_SND = 1u << 20, SNDB = NUNIT * UNB;
constexpr unsigned ROWB = NS * 512;
constexpr unsigned WS_OSN = WS_SND + NUNIT * UNB + SNDB;
constexpr unsigned WS_PAN = (WS_OSN + MM * ROWB + PANB - 1) / PANB * PANB;
constexpr unsigned WS_END = WS_PAN + NIN * PANB;
constexpr int U0 = G0 + (NRANK - 1) * NS;
constexpr unsigned OUTB = (unsigned)MM * NN * 2;
static_assert(WS_CTRL_END <= WS_DBG, "ctrl region");
static_assert(NFLAG <= 512 && 2048 <= (int)FLAGB, "flag region");
#define CFG_DEEP 24
#define CFG_DEEPO 32
constexpr unsigned STAGE_BYTES = 9 * HTB;
constexpr unsigned LDS_BYTES = STAGE_BYTES + 768;
constexpr unsigned long long DEADLINE = 50000000ull;
constexpr int AUX_SYS = 17, AUX_SYS_NT = 19, AUX_AGENT = 16;
#define CFG_PACE 140
#define CFG_UH (2 * (NPK - 8 * DK))
#define CFG_TH 5000
#define CFG_PH 40
#define CFG_OST 2
#define CFG_OTH AUX_SYS_NT
#define CFG_PST AUX_AGENT
#define CFG_LDA 0
#define CFG_RLD AUX_SYS
#define CFG_SW 0
constexpr unsigned OSTR = NN * 2;
#define CFG_GS 16
#define CFG_PFA 0
#define CFG_VD 999
#define CFG_OWN AUX_AGENT
#define CFG_PFB 1
#define CFG_JP 4
#define CFG_AGG 24
static_assert(CFG_AGG % 8 == 0 && CFG_AGG >= NS && NS <= 8 && NIN * 4 <= 512, "groups");
#define CFG_PWND 3
#define CFG_EH 1
#define CFG_FK 2
#define CFG_MIX 1
static_assert(4 * NRANK * NS <= NPK && (NS - NPUSH) * 64 / PU <= NW && NS * 64 <= 512, "leading units");
#define CFG_PACE0 40
#define CFG_STAG 0
#define CFG_P3 320
#define CFG_P3N 2
#define CFG_STAGB 400
#define CFG_LOOK 104
#define CFG_TAIL NW
#define CFG_EVEN 280
#define CFG_EVEND 200
#define CFG_ERATE CFG_RATE
#define CFG_ZB 120
#define CFG_LOOKP 100
#define CFG_PLAN (CFG_ZB + CFG_LOOKP + 50)
#define CFG_BAL 8
constexpr int ZB = CFG_ZB;
static_assert(ZB <= 127 && ZB <= NS * 128 / PU && NFLAG <= 504, "plan");
#define CFG_PWIN 4u
#define CFG_PS 2
#define CFG_RATE 1680
#define CFG_CRED 25
#define CFG_GRID 36
#define CFG_SPIN 8
#define CFG_EARLY 1
#define CFG_RATEO CFG_RATE
constexpr unsigned PACE16 = CFG_RATE * (NW + CFG_DEEP) / 64, PACEO16 = CFG_RATEO * (NW + CFG_DEEPO) / 64;

#define CFG_EA 12
#define CFG_EB 6
#define CFG_ER 78
#define CFG_ES 17
#define CFG_EMIN 16
constexpr int NTAB = 4 * (G0 + GM);
static_assert(NTAB <= 256, "byte tables");
struct Order {
    unsigned char from[256], to[256];
    static constexpr int due(int o) {
        const int d = o >= 4 * G0 ? CFG_ER + (o - 4 * G0) / GM * CFG_ES + ((o - 4 * G0) % GM & 1) * CFG_EB
                                  : o / G0 * CFG_ES + (o % G0 < GW ? o % G0 / (GW / 2) * CFG_EB : (o % G0 - GW) / 4 * CFG_EA);
        return d < CFG_EMIN ? CFG_EMIN : d;
    }
    constexpr Order() : from(), to() {
        for (int o = 0; o < NTAB; ++o) {
            int n = 0;
            for (int k = 0; k < NTAB; ++k) n += due(k) < due(o) || (due(k) == due(o) && k < o);
            to[o] = (unsigned char)n; from[n] = (unsigned char)o;
        }
    }
};
constexpr Order ORDER;
static_assert(ORDER.to[NS - 1] == NS - 1, "the leading units come first");

struct Params {
    const void* x; const void* w;
    char* wsl; char* outl;
    int rank, pad[7];
    char* ws[NRANK]; char* out[NRANK];
    Order order;
};
__device__ __forceinline__ int look_up(unsigned tab, int o) { return o < NTAB ? (unsigned)__builtin_amdgcn_readlane(tab, o >> 2) >> (o & 3) * 8 & 255u : o; }

__device__ __forceinline__ rsrc_t mk_rsrc(const void* p, unsigned bytes) {
    const unsigned long long v = (unsigned long long)p;
    const unsigned lo = __builtin_amdgcn_readfirstlane((unsigned)v), hi = __builtin_amdgcn_readfirstlane((unsigned)(v >> 32));
    return __builtin_amdgcn_make_buffer_rsrc((void*)(((unsigned long long)hi << 32) | lo), 0, bytes, 0x00020000);
}
__device__ __forceinline__ unsigned ld_sys(const unsigned* p) { return __hip_atomic_load(p, __ATOMIC_RELAXED, __HIP_MEMORY_SCOPE_SYSTEM); }
__device__ __forceinline__ unsigned ld_agent(const unsigned* p) { return __hip_atomic_load(p, __ATOMIC_RELAXED, __HIP_MEMORY_SCOPE_AGENT); }
__device__ __forceinline__ void st_sys(unsigned* p, unsigned v) { __hip_atomic_store(p, v, __ATOMIC_RELAXED, __HIP_MEMORY_SCOPE_SYSTEM); }
__device__ __forceinline__ void st_agent(unsigned* p, unsigned v) { __hip_atomic_store(p, v, __ATOMIC_RELAXED, __HIP_MEMORY_SCOPE_AGENT); }
__device__ __forceinline__ void add_agent(unsigned* p, unsigned v) { (void)__hip_atomic_fetch_add(p, v, __ATOMIC_RELAXED, __HIP_MEMORY_SCOPE_AGENT); }
#define STORE16(v, rs, vo, so, aux) do { const u32x4 _d = (v); __builtin_amdgcn_raw_buffer_store_b128(_d, rs, vo, so, aux); asm volatile("s_nop 1" :: "v"(_d) : "memory"); } while (0)
#define WAIT_VM0() asm volatile("s_waitcnt vmcnt(0)" ::: "memory")
#define WAIT_VMN(n) do { asm volatile("" ::: "memory"); __builtin_amdgcn_s_waitcnt(0x0F70 | (n)); asm volatile("" ::: "memory"); } while (0)

__device__ __forceinline__ bool give_up(unsigned* tmo, unsigned call, unsigned long long t0) {
    const unsigned long long dt = __builtin_amdgcn_s_memrealtime() - t0;
    if (dt < 2000ull) return false;
    if ((unsigned)__builtin_amdgcn_readfirstlane(ld_agent(tmo)) == call) return true;
    if (dt > DEADLINE) { st_agent(tmo, call); return true; }
    return false;
}

__device__ __forceinline__ int gpan_of(int v, int consumer) { return v < NS ? consumer * NS + v : (NRANK - 1) * NS + v; }
__device__ __forceinline__ int band_of(int e, int consumer) { return e < 7 ? (consumer + 1 + e) & 7 : e < 15 ? 8 + ((consumer + e) & 7) : consumer; }
__device__ __forceinline__ int turn_of(int band, int consumer) { return band >= 8 ? 7 + ((band - consumer + 1) & 7) : band == consumer ? 15 : (band - consumer - 1) & 7; }
__device__ __forceinline__ int unit_of(int o, int consumer, int& q, int& pos) {
    if (o < 4 * G0) { q = o / G0; const int m = o % G0; pos = q * U0 + gpan_of(m, consumer); return m < GW ? NTM + m : m - GW; }
    pos = o + 4 * (NRANK - 1) * NS;
    o -= 4 * G0;
    if (o < 4 * GM) { q = o / (GM ? GM : 1); return NTM + GW + o % (GM ? GM : 1); }
    o -= 4 * GM;
    const int g = o / (4 * GW), r = o % (4 * GW);
    int i;
    if (GL == 0 || g < NGF) { q = r / GW; i = r % GW; } else { q = r / (GL ? GL : 1); i = r % (GL ? GL : 1); }
    return NTM + GM + GW * (g + 1) + i;
}

__device__ __forceinline__ unsigned cvt_pk_bf16(float lo, float hi) { unsigned r; asm("v_cvt_pk_bf16_f32 %0, %1, %2" : "=v"(r) : "v"(lo), "v"(hi)); return r; }
__device__ __forceinline__ float silu(float y) {
    const float c = __builtin_amdgcn_fmed3f(y, -60.0f, 60.0f);
    const float e = __builtin_amdgcn_exp2f(c * -1.44269504088896f);
    return y * __builtin_amdgcn_rcpf(1.0f + e);
}

typedef float f32x2 __attribute__((ext_vector_type(2)));
__device__ __forceinline__ f32x2 silu2(f32x2 y) {
    const f32x2 t = __builtin_elementwise_fma(y, (f32x2){-1.44269504088896f, -1.44269504088896f}, (f32x2){-86.5617024533378f, -86.5617024533378f});
    const f32x2 e = {__builtin_amdgcn_fmed3f(__builtin_amdgcn_exp2f(t[0]), 0.0f, 1.0f), __builtin_amdgcn_fmed3f(__builtin_amdgcn_exp2f(t[1]), 0.0f, 1.0f)};
    const f32x2 d = __builtin_elementwise_fma(e, (f32x2){1.1420073898156842e26f, 1.1420073898156842e26f}, (f32x2){1.0f, 1.0f});
    return y * (f32x2){__builtin_amdgcn_rcpf(d[0]), __builtin_amdgcn_rcpf(d[1])};
}

#define STAMP1(P, slot) do {} while (0)
#define STAMP(P, slot) do {} while (0)

__device__ __forceinline__ void pull_role(const Params& P, unsigned call, int peer, int widx, int lane) {
    const int rank = P.rank;
    char* wsl = P.wsl;
    const rsrc_t rwr = mk_rsrc(P.ws[peer], WS_END), rwl = mk_rsrc(wsl, WS_END);
    const rsrc_t rol = mk_rsrc(P.outl, OUTB), ror = mk_rsrc(P.out[peer], OUTB);
    unsigned* tmo = (unsigned*)(wsl + WS_TMO);
    constexpr unsigned OOB = 0xfffffff0u;
    const unsigned lane0 = lane == 0 ? 0u : OOB;
    const unsigned hoff = WS_PH + (call & 1u) * (NRANK * 128u);
    auto draw = [&]() { return (unsigned)__builtin_amdgcn_raw_ptr_buffer_atomic_add_i32(1, rwl, lane0, hoff + (unsigned)peer * 128u, 0); };
    if (widx == 0) __builtin_amdgcn_raw_buffer_store_b32(0u, rwl, lane0, WS_PH + (~call & 1u) * (NRANK * 128u) + (unsigned)peer * 128u, AUX_SYS);
    auto arrive = [&](unsigned vo, unsigned cnt) { (void)__builtin_amdgcn_raw_ptr_buffer_atomic_add_i32(PU, rwl, vo, cnt, 0); };
    const int wrk = (widx & 7) * PW + (widx >> 3);
    const unsigned perI = wrk < CFG_DEEP ? PACE16 / 2 : PACE16, perO = wrk < CFG_DEEPO ? PACEO16 / 2 : PACEO16;
    const unsigned from = ((const unsigned*)P.order.from)[lane];
    bool bad = false; int stamped = 0; (void)stamped;
    auto index_of = [&](unsigned v) { return (int)__builtin_amdgcn_readfirstlane(v) + NW; };
    int cur = index_of((unsigned)(widx - NW));
    constexpr int BPB = NS * 128 / PU;
    const unsigned fso = WS_FLAG + (unsigned)peer * FLAGB;
    auto first_of = [&](int i) { return i < NPIN ? (i >> (6 - PSH)) + NPUSH : NIN * 4 + (i - NPIN) / BPB; };
    int lim = 0, flim = 0, fin = NPC;
    bool planned = ZB == 0;
    unsigned long long plan = 0;
    auto moved = [&](int j) { return (int)(signed char)(plan >> (j * 8)); };
    auto update = [&](const u32x4& f0, const u32x4& f1, int i) {
        const int first = first_of(i);
        unsigned un = 0;
#pragma unroll
        for (int j = 0; j < 8; ++j) un |= ((int)((j < 4 ? f0[j & 3] : f1[j & 3]) - call) < 0 && lane * 8 + j >= first ? 1u : 0u) << j;
        const int l = __builtin_ctzll(__ballot(un != 0u) | (1ull << 63));
        const int u = l * 8 + __builtin_ctz(__builtin_amdgcn_readlane(un, l) | 256u);
        const int fl = u <= NIN * 4 ? (u - NPUSH) << (6 - PSH) : u >= NFLAG ? NPC + 128 : NPIN + (u - NIN * 4) * BPB;
        flim = fl > flim ? fl : flim;
        if (!planned && (unsigned)__builtin_amdgcn_readlane(f0[0], 63) == call && (unsigned)__builtin_amdgcn_readlane(f0[2], 63) == call) {
            plan = (unsigned long long)(unsigned)__builtin_amdgcn_readlane(f0[3], 63) << 32 | (unsigned)__builtin_amdgcn_readlane(f0[1], 63);
            planned = true; fin = NPC - moved(peer);
        }
        const int pl = planned ? fin : NPC - ZB;
        lim = flim < pl ? flim : pl;
    };
    auto make_plan = [&]() {
        unsigned old = 0;
        if (lane == 0) old = __hip_atomic_exchange((unsigned*)(wsl + WS_PH + 2 * NRANK * 128), call, __ATOMIC_RELAXED, __HIP_MEMORY_SCOPE_AGENT);
        if ((unsigned)__builtin_amdgcn_readfirstlane(old) == call) return;
        const unsigned hd = __builtin_amdgcn_raw_buffer_load_b32(rwl, lane < NRANK ? (unsigned)lane * 128u : OOB, hoff, AUX_SYS);
        const int t = lane >> 3, s = lane & 7;
        const unsigned zc = ld_sys((const unsigned*)(P.ws[t] + WS_ZC + (rank * NRANK + s) * 128));
        const bool ok = __all(t == rank || s == rank || s == t || (int)(zc - call * (unsigned)(ZB * PU)) >= 0);
        int h[NRANK], sum = 0, give = 0, take = 0;
#pragma unroll
        for (int j = 0; j < NRANK; ++j) { h[j] = __builtin_amdgcn_readlane(hd, j); sum += j == rank ? 0 : h[j]; }
#pragma unroll
        for (int j = 0; j < NRANK; ++j) {
            const int d = j == rank ? 0 : (sum - (NRANK - 1) * h[j]) * CFG_BAL / (8 * (NRANK - 1));
            h[j] = d > ZB ? ZB : d < -127 ? -127 : d;
            if (h[j] > 0) give += h[j]; else take -= h[j];
        }
        give = take = !ok ? 0 : give < take ? give : take;
        unsigned long long pl = 0;
#pragma unroll
        for (int j = 0; j < NRANK; ++j) {
            int d = h[j];
            if (d > 0) { d = d < give ? d : give; give -= d; } else { d = -d < take ? d : -take; take += d; }
            pl |= (unsigned long long)(d & 255) << (j * 8);
        }
        __builtin_amdgcn_raw_buffer_store_b128((u32x4){call, (unsigned)pl, call, (unsigned)(pl >> 32)}, rwl, lane < NRANK ? WS_FLAG + 63 * 32 + (unsigned)lane * FLAGB : OOB, 0, AUX_SYS);
    };
    auto offs_of = [&](int i, unsigned (&o)[4], unsigned& cnt) -> unsigned {
        if (i < NPIN) {
            int q, pos; const int pan = unit_of(look_up(from, (i >> (6 - PSH)) + NPUSH), rank, q, pos);
            cnt = WS_CNT + (unsigned)(pan * NRANK + peer) * 128u + (unsigned)q * 4u;
            const unsigned sub = (unsigned)((i << PSH) & 63) * 1024u;
            const unsigned b = WS_PAN + (unsigned)pan * PANB + (unsigned)peer * CHB + (unsigned)q * UNB + sub + (unsigned)lane * 16u;
#pragma unroll
            for (int k = 0; k < 4; ++k) o[k] = b + k * 1024u;
            return WS_SND + (call & 1u) * SNDB + (unsigned)pos * UNB + sub;
        }
        int src = peer;
        if (ZB && i >= NPC) {
            int x = i - NPC;
            for (int j = 0; j < peer; ++j) x -= moved(j) < 0 ? moved(j) : 0;
            for (int j = 0; j < NRANK; ++j) { const int g = moved(j) > 0 ? moved(j) : 0; if ((unsigned)x < (unsigned)g) { src = j; i = NPC - g + x; } x -= g; }
        }
        const int e = (i - NPIN) / BPB, r = (i - NPIN) % BPB, pb = band_of(e, rank) * BPB + r;
        cnt = e < NRANK - 1 && r >= BPB - ZB ? WS_ZC + (unsigned)(band_of(e, rank) * NRANK + peer) * 128u : 0u;
#pragma unroll
        for (int k = 0; k < 4; ++k) {
            const unsigned sg = (unsigned)((pb << PSH) + k) * 2u + (unsigned)(lane >> 5);
            o[k] = (unsigned)src * ROWB + (unsigned)(lane & 31) * 16u + sg / NS * (NN * 2u) + sg % NS * 512u;
        }
        return WS_OSN + (unsigned)(pb << PSH) * 1024u;
    };
    if (cur < (NS - NPUSH) << (6 - PSH)) {
        const unsigned long long t0 = __builtin_amdgcn_s_memrealtime();
        const unsigned fo = fso + 2048u + (unsigned)(((cur >> 3) + 2 * NPUSH) * 2 + (cur >> 1 & 1)) * 8u;
        for (;;) {
            const unsigned f = __builtin_amdgcn_raw_buffer_load_b32(rwl, (unsigned)(lane & 1) * 4u, fo, AUX_SYS);
            if (__all((int)(f - call) >= 0)) { lim = flim = cur + 1; break; }
            if (give_up(tmo, call, t0)) { bad = true; planned = true; break; }
            __builtin_amdgcn_s_sleep(CFG_PS);
        }
        asm volatile("" ::: "memory");
    }
    asm volatile("" :: "v"(from));
    while (!planned || cur < fin) {
        if (!bad && cur >= lim) {
            const unsigned long long t0 = __builtin_amdgcn_s_memrealtime();
            const unsigned pvo = (unsigned)(lane - (first_of(cur) >> 3)) < CFG_PWIN || (lane == 63 && cur >= NPC - ZB) ? (unsigned)lane * 32u : OOB - 16u;
            for (;;) {
                const u32x4 f0 = __builtin_amdgcn_raw_buffer_load_b128(rwl, pvo, fso, AUX_SYS), f1 = __builtin_amdgcn_raw_buffer_load_b128(rwl, pvo + 16u, fso, AUX_SYS);
                update(f0, f1, cur);
                if (cur < lim || (planned && cur >= fin)) break;
                if (give_up(tmo, call, t0)) { bad = true; planned = true; plan = 0; fin = NPC; break; }
                __builtin_amdgcn_s_sleep(CFG_PS);
            }
            asm volatile("" ::: "memory");
            if (cur >= fin && planned) break;
        }
        unsigned o[4], p2 = OOB, p3 = OOB, cprev = 0;
        if (bad) {
            (void)offs_of(cur, o, cprev);
            if (cur >= NPIN) {
#pragma unroll
                for (int k = 0; k < PU; ++k) __builtin_amdgcn_raw_buffer_store_b128((u32x4){0x7fc07fc0u, 0x7fc07fc0u, 0x7fc07fc0u, 0x7fc07fc0u}, rol, o[k], 0, AUX_AGENT);
            }
            cur = index_of(draw()); continue;
        }
        STAMP1(P, 1);
        rsrc_t rp = rwl;
        auto put = [&](const u32x4& d, const rsrc_t r, unsigned a) { __builtin_amdgcn_raw_buffer_store_b128(d, r, a, 0, CFG_PST); };
        u32x4 v[4] = {};
        unsigned slot = 0;
        if (CFG_RATE) { const unsigned now = (unsigned)__builtin_amdgcn_s_memrealtime() << 4; slot = now - (now & 0xffffffu) % PACE16 + (unsigned)wrk * (PACE16 / NW) - PACE16; }
        unsigned cb, src, in[4];
        auto prepare = [&]() {
            if (ZB && cur == NPC - CFG_PLAN) make_plan();
            src = __builtin_amdgcn_readfirstlane(offs_of(cur, o, cb)); cb = __builtin_amdgcn_readfirstlane(cb);
            if (ZB && cur >= NPC) src = 0;
#pragma unroll
            for (int k = 0; k < 4; ++k) { in[k] = ZB && cur >= NPC ? o[k] : lane * 16 + k * 1024; asm volatile("" : "+v"(o[k]), "+v"(in[k])); }
        };
        prepare();
        do {
            const bool deep0 = wrk < (cur < NPIN ? CFG_DEEP : CFG_DEEPO);
            const bool tail = cur >= fin - CFG_TAIL, even = cur >= fin - (deep0 ? CFG_EVEND : CFG_EVEN), deep = deep0 || tail || even;
            const unsigned per = even ? CFG_ERATE * NW / 64 : cur < NPIN ? perI : perO;
            const rsrc_t rd = cur < NPIN ? rwl : rol, rs = ZB && cur >= NPC ? ror : rwr;
            const bool look = flim < NPC && cur + CFG_LOOK >= flim, lookp = !planned && cur + CFG_LOOKP >= NPC - ZB;
            const unsigned fvo = look || (lookp && lane == 63) ? (unsigned)lane * 32u : OOB - 16u;
            unsigned cdone = cprev;
            unsigned nv = 0; u32x4 f0, f1;
#pragma unroll
            for (int k = 0; k < 4; ++k) {
                if (CFG_RATE && !tail) {
                    slot += per;
                    if (CFG_GRID) {
                        unsigned go = (unsigned)__builtin_amdgcn_s_memrealtime();
                        const int d = (int)((go << 4) - slot);
                        if (d > CFG_CRED * 16) slot += (unsigned)(d - CFG_CRED * 16);
                        if (d < 0) go += (unsigned)(15 - d) >> 4;
                        go += CFG_GRID - (go & 0xffffffu) % (CFG_GRID ? CFG_GRID : 1);
                        for (;;) {
                            const int w = (int)(go - (unsigned)__builtin_amdgcn_s_memrealtime());
                            if (w <= 0) break;
                            if (w > CFG_SPIN) __builtin_amdgcn_s_sleep(1);
                        }
                    } else for (;;) {
                        const int d = (int)(((unsigned)__builtin_amdgcn_s_memrealtime() << 4) - slot);
                        if (d > CFG_CRED * 16) slot += (unsigned)(d - CFG_CRED * 16);
                        if (d >= 0) break;
                        __builtin_amdgcn_s_sleep(1);
                    }
                }
                if (CFG_FK && k == 2) { f0 = __builtin_amdgcn_raw_buffer_load_b128(rwl, fvo, fso, AUX_SYS); f1 = __builtin_amdgcn_raw_buffer_load_b128(rwl, fvo + 16u, fso, AUX_SYS); asm volatile("" ::: "memory"); }
                v[k] = __builtin_amdgcn_raw_buffer_load_b128(rs, in[k], src, CFG_RLD);
                if (k == 0) {
                    nv = draw();
                    if (!CFG_FK) { f0 = __builtin_amdgcn_raw_buffer_load_b128(rwl, fvo, fso, AUX_SYS); f1 = __builtin_amdgcn_raw_buffer_load_b128(rwl, fvo + 16u, fso, AUX_SYS); }
                }
                put(v[k ^ 2], k < 2 ? rp : rd, k == 0 ? p2 : k == 1 ? p3 : o[k ^ 2]);
                if (CFG_EARLY && !deep) {
                    unsigned& a = k == 0 ? p3 : o[k - 1];
                    put(v[(k + 3) & 3], k < 1 ? rp : rd, a);
                    a = OOB;
                }
                if (k == 3) {
                    asm volatile("" ::: "memory");
                    cprev = cb; rp = rd; p2 = o[2]; p3 = o[3];
                    cur = index_of(nv);
                    if (look || lookp) update(f0, f1, cur);
                    if (cur < lim) prepare();
                }
                if (k == 0) WAIT_VMN(CFG_FK ? 5 : 7); else if (k == 1) WAIT_VMN(CFG_FK ? 4 : 6); else if (k == 2) WAIT_VMN(CFG_FK ? 5 : 3); else WAIT_VMN(3);
                if (!deep) {
                    if (k == 0) WAIT_VMN((CFG_FK ? 2 : 4) + CFG_EARLY); else WAIT_VMN(1 + CFG_EARLY);
                    if (CFG_EARLY && k == 1) { arrive(cdone ? lane0 : OOB, cdone); cdone = 0; }
                }
            }
            arrive(cdone ? lane0 : OOB, cdone);
        } while (cur < lim);
        put(v[2], rp, p2);
        put(v[3], rp, p3);
        if (cprev) { WAIT_VM0(); arrive(lane0, cprev); }
    }
    STAMP(P, 4);
}

__device__ __forceinline__ void load_w(u32x4 (&v)[8], const rsrc_t rw, int gpan, int q, int job, int lane) {
    const unsigned vo = (unsigned)((lane >> 4) * 8) * (NN * 2) + (unsigned)(lane & 15) * 16u;
    const unsigned so = (unsigned)(q * 128 + (job >> 1) * 32) * (NN * 2) + (unsigned)(gpan * 256 + (job & 1) * 128) * 2u;
#pragma unroll
    for (int j = 0; j < 8; ++j) v[j] = __builtin_amdgcn_raw_buffer_load_b128(rw, vo, so + (unsigned)j * (NN * 2), 2);
}
template <bool twice>
__device__ __forceinline__ void store_w(const u32x4 (&v)[8], const rsrc_t rd, unsigned vo, unsigned vo2, unsigned unit, int job, bool slow) {
    unit += (unsigned)(job >> 2) * KTB + (unsigned)(job & 1) * 8192u + (unsigned)(job >> 1 & 1) * 1024u;
#pragma unroll
    for (int i = 0; i < 8; ++i) {
        u32x4 o;
#pragma unroll
        for (int e = 0; e < 4; ++e) o[e] = __builtin_amdgcn_perm(v[2 * e + 1][i >> 1], v[2 * e][i >> 1], (i & 1) ? 0x07060302u : 0x05040100u);
        STORE16(o, rd, vo, unit + (unsigned)(i >> 2) * HTB + (unsigned)(i & 3) * 2048u, AUX_SYS);
        if (twice) STORE16(o, rd, vo2, unit + (unsigned)(i >> 2) * HTB + (unsigned)(i & 3) * 2048u, CFG_OWN);
        if (CFG_PWND && i >= CFG_PWND - 1 && i < 7 && slow) WAIT_VMN(CFG_PWND ? CFG_PWND - 1 : 0);
    }
}
typedef __attribute__((address_space(1))) unsigned* gptr_t;

__device__ __forceinline__ void lead_role(const Params& P, int rank, unsigned long long wsp, unsigned tk, unsigned t0, int g, int wid, int lane) {
    const int o = NS - 1 - (g >> 5), c = (g >> 3) & 3, job = (c >> 1) * 4 + wid * 2 + (c & 1);
    const int dest = (rank + 1 + g + (o < NPUSH ? CFG_MIX * (2 * c + o) : 0)) & (NRANK - 1);
    u32x4 v[8];
    load_w(v, mk_rsrc(P.w, (unsigned)KSH * NN * 2), dest * NS + o, 0, job, lane);
    asm volatile("" : "+v"(tk), "+v"(wsp) :: "memory");
    const bool push = o < NPUSH && dest != rank, count = o < NPUSH || dest == rank;
    char* wsd = (char*)((unsigned long long)(unsigned)__builtin_amdgcn_readlane((unsigned)(wsp >> 32), dest) << 32 | (unsigned)__builtin_amdgcn_readlane((unsigned)wsp, dest));
    gptr_t fp = (gptr_t)(wsd + (count ? WS_CNT + (unsigned)((NTM + o) * NRANK + rank) * 128u : WS_FLAG + (unsigned)rank * FLAGB + 2048u + (unsigned)((o * 4 + c) * 2 + wid) * 4u));
    while (CFG_P3 && push && o >= CFG_P3N && (int)((unsigned)__builtin_amdgcn_s_memrealtime() - t0) < CFG_P3 + (o - CFG_P3N) * 110) __builtin_amdgcn_s_sleep(1);
    store_w<false>(v, mk_rsrc(push ? wsd : P.wsl, WS_END), (unsigned)lane * 16u + (push ? 0u : (~tk & 1u) * SNDB), 0u,
            push ? WS_PAN + (unsigned)(NTM + o) * PANB + (unsigned)rank * CHB : WS_SND + (unsigned)(dest * NS + o) * UNB, job, push);
    WAIT_VMN(0);
    STAMP(P, 6);
    if (lane == 0) {
        if (!count) __hip_atomic_store(fp, tk + 1u, __ATOMIC_RELAXED, __HIP_MEMORY_SCOPE_SYSTEM);
        else if ((__hip_atomic_fetch_add((gptr_t)(P.wsl + WS_LEAD + (o * NRANK + dest) * 64), 1u, __ATOMIC_RELAXED, __HIP_MEMORY_SCOPE_AGENT) & 7u) == 7u)
            __hip_atomic_store(fp, (tk + 1u) * CNT_FULL, __ATOMIC_RELAXED, __HIP_MEMORY_SCOPE_SYSTEM);
    }
}

__device__ __forceinline__ void pack_role(const Params& P, unsigned long long wsp, unsigned tk, unsigned t0, int g, int wid, int lane) {
    const int rank = P.rank;
    const rsrc_t rx = mk_rsrc(P.x, (unsigned)MM * KSH * 2), rw = mk_rsrc(P.w, (unsigned)KSH * NN * 2), rwl = mk_rsrc(P.wsl, WS_END);
    const unsigned sv = (unsigned)lane * 16u + (~tk & 1u) * SNDB;
    const unsigned to = ((const unsigned*)P.order.to)[lane];
    constexpr int NH = CFG_UH < NUNIT ? 8 * DK : 0, UH = NH ? CFG_UH : NUNIT;
    const bool late = g >= NPK - NH;
    constexpr int U1 = NRANK * NS, NE = CFG_EH ? NH : 0;
    static_assert(CFG_TH < (UH - U1 - NH) * CFG_PH, "unpaced");
    for (int u = late ? (NE ? U1 : UH) + g - (NPK - NH) : U1 + NE + g; u < (late ? NUNIT : UH); u += !late ? NPK - NH : u < UH ? UH - U1 : NH) {
        int o, dest = -1;
        if (u < 4 * U0) {
            const int r = u % U0;
            if (r < NRANK * NS) { o = r >> 3; dest = (rank + 1 + r) & (NRANK - 1); } else o = r - (NRANK - 1) * NS;
            o += u / U0 * G0;
        } else o = u - 4 * (NRANK - 1) * NS;
        int q, pos; const int pan = unit_of(o, dest < 0 ? 0 : dest, q, pos);
        while ((int)((unsigned)__builtin_amdgcn_s_memrealtime() - t0) < (late ? CFG_TH + (u - UH) * CFG_PH : (u - CFG_PACE0) * CFG_PACE / 10)) __builtin_amdgcn_s_sleep(1);
        const unsigned dst = WS_SND + (unsigned)pos * UNB;
        u32x4 v[8];
        if (pan < NTM) {
            const int rb0 = (wid & 3) * 4;
            const unsigned vo = (unsigned)(lane & 15) * (KSH * 2) + (unsigned)(lane >> 4) * 16u;
            const unsigned so = (unsigned)(pan * 256 + rb0 * 16) * (KSH * 2) + (unsigned)(q * 2 + (wid >> 2)) * 128u;
#pragma unroll
            for (int i = 0; i < 8; ++i) v[i] = __builtin_amdgcn_raw_buffer_load_b128(rx, vo + (i & 1) * 64, so + (unsigned)(i >> 1) * (16 * KSH * 2), 0);
#pragma unroll
            for (int i = 0; i < 8; ++i) STORE16(v[i], rwl, sv + (i & 1) * 1024, dst + (unsigned)(wid >> 2) * KTB + (unsigned)(rb0 + (i >> 1)) * 2048u, AUX_SYS);
        } else {
            load_w(v, rw, gpan_of(pan - NTM, dest < 0 ? 0 : dest), q, wid, lane);
            if (pan - NTM >= CFG_VD) store_w<true>(v, rwl, sv, WS_PAN + (unsigned)pan * PANB + (unsigned)rank * CHB + (unsigned)q * UNB - dst + (unsigned)lane * 16u, dst, wid, false);
            else store_w<false>(v, rwl, sv, 0u, dst, wid, false);
        }
        o = look_up(to, o);
        const bool own = lane == rank;
        unsigned long long fp = wsp + (own ? WS_CNT + (unsigned)((pan * NRANK + rank) * 32 + q) * 4u : WS_FLAG + (unsigned)rank * FLAGB + (unsigned)o * 4u);
        unsigned fv = own ? (tk + 1u) * CNT_FULL : tk + 1u;
        asm volatile("" : "+v"(fp), "+v"(fv));
        WAIT_VMN(0);
        __builtin_amdgcn_s_barrier();
        const bool mine = lane < NRANK && (dest < 0 || dest == lane);
        if (wid == 0 && mine && (own || o < CFG_AGG || o >= NIN * 4 / 8 * 8)) __hip_atomic_store((gptr_t)fp, fv, __ATOMIC_RELAXED, __HIP_MEMORY_SCOPE_SYSTEM);
        if (wid == 1 && o >= CFG_AGG && o < NIN * 4 / 8 * 8) {
            unsigned old = 0;
            if (mine && !own) old = __hip_atomic_fetch_add((gptr_t)(P.wsl + WS_GRP) + (o >> 3) * NRANK + lane, 1u, __ATOMIC_RELAXED, __HIP_MEMORY_SCOPE_AGENT);
            const unsigned full = (unsigned)__ballot((old & 7u) == 7u);
            const unsigned long long wsq = (unsigned long long)(unsigned)__builtin_amdgcn_ds_bpermute(lane >> 3 << 2, (int)(wsp >> 32)) << 32 | (unsigned)__builtin_amdgcn_ds_bpermute(lane >> 3 << 2, (int)wsp);
            if (full >> (lane >> 3) & 1) __hip_atomic_store((gptr_t)(wsq + WS_FLAG + (unsigned)rank * FLAGB) + (o & ~7) + (lane & 7), tk + 1u, __ATOMIC_RELAXED, __HIP_MEMORY_SCOPE_SYSTEM);
        }
    }
}

struct Tile { int tm, v, half; unsigned sA, stA; };
__device__ __forceinline__ void tile_of(int idx, int col, int rank, unsigned snd, Tile& t) {
    const int v = idx < 2 * GW ? (idx >> 2) + (col >> 2) * (GW / 2) : 2 * (idx >> 2) + (col >> 2);
    t.tm = 4 * (col & 3) + (idx & 3); t.v = v; t.half = -1;
    const int g = (v - GM) / GW, i = (v - GM) % GW;
    t.sA = snd + (unsigned)(v < GW ? gpan_of(v, rank) : v < GW + GM ? 4 * U0 + v - GW : 4 * U0 + 4 * GM + (g - 1) * 4 * GW + i) * UNB;
    t.stA = (unsigned)(v < GW ? U0 : v < GW + GM ? GM : g <= NGF ? GW : GL) * UNB;
    if (v >= CFG_VD) { t.sA = WS_PAN + (unsigned)(NTM + v) * PANB + (unsigned)rank * CHB; t.stA = UNB; }
}
__device__ __forceinline__ void item_of(int q, int rank, unsigned snd, Tile& t) {
    const int n = q < PFULL ? q : PFULL + ((q - PFULL) >> 1);
    if (n < 4 * PL0) tile_of(RS * KX - DK + n % PL0, n / PL0, rank, snd, t);
    else tile_of(RS * KX - DK + (n - 4 * PL0) % PL1, 4 + (n - 4 * PL0) / PL1, rank, snd, t);
    if (q >= PFULL) t.half = (q - PFULL) & 1;
}

template <int wr>
__device__ __forceinline__ void gemm_role(const Params& P, unsigned call, int col, int slot, LAS unsigned char* lds) {
    const int tid = threadIdx.x, wid = __builtin_amdgcn_readfirstlane(tid >> 6), lane = tid & 63, wc = wid & 3;
    const int rank = P.rank;
    char* wsl = P.wsl;
    const rsrc_t rws = mk_rsrc(wsl, WS_END);
    LAS unsigned* ctl = (LAS unsigned*)(lds + STAGE_BYTES);
    const unsigned voff = (unsigned)tid * 16u;
    const unsigned ldsw = (unsigned)wid * 1024u;
    const int aoff = wr * 8192 + lane * 16, boff = wc * 4096 + lane * 16;

    auto lane_now = [&]() { int tz = threadIdx.x; asm volatile("" : "+v"(tz)); return tz & 63; };
    auto cnt_of = [&](const Tile& t) { const int ln = lane_now(); return WS_CNT + (unsigned)((((ln & 32) ? NTM + t.v : t.tm) * NRANK + ((ln >> 2) & 7)) * 32 + (ln & 3)) * 4u; };
    auto level_of = [&](unsigned c) -> int {
        const unsigned long long miss = __ballot((int)(c - call * CNT_FULL) < 0);
        unsigned m = (unsigned)miss | (unsigned)(miss >> 32);
        const bool own = ((m >> (rank * 4)) & 15u) == 0u;
        m |= m >> 16; m |= m >> 8; m |= m >> 4;
        return own ? 1 + __builtin_ctz(m | 16u) : 0;
    };
    auto ready = [&](const Tile& t, int want) -> int {
        if (wr == 0 && wid == 0) {
            const unsigned* c = (const unsigned*)(wsl + cnt_of(t));
            unsigned* tmo = (unsigned*)(wsl + WS_TMO);
            const unsigned long long t0 = __builtin_amdgcn_s_memrealtime();
            int got;
            for (;;) {
                got = level_of(ld_sys(c));
                if (got >= want) break;
                if (give_up(tmo, call, t0)) { ctl[0] = 1u; got = 5; break; }
                __builtin_amdgcn_s_sleep(CFG_GS);
            }
            ctl[2] = (unsigned)got;
            __builtin_amdgcn_fence(__ATOMIC_ACQUIRE, "agent");
            asm volatile("s_waitcnt vmcnt(0) lgkmcnt(0)" ::: "memory");
        }
        asm volatile("" ::: "memory"); __builtin_amdgcn_s_barrier(); asm volatile("" ::: "memory");
        const int r = __builtin_amdgcn_readfirstlane(ctl[2]);
        asm volatile("s_waitcnt lgkmcnt(0)" ::: "memory");
        return r;
    };

#define SA(b, h) (((b) * 2 + (h)) * HTB)
#define SB(b, h) ((4 + (b) * 2 + (h)) * HTB)
#define STAGE(bufoff, soff) do { _Pragma("unroll") for (int _i = 0; _i < 2; ++_i) \
        __builtin_amdgcn_raw_ptr_buffer_load_lds(rws, (LAS unsigned*)(lds + (bufoff) + ldsw + _i * 8192), 16, voff, (soff) + _i * 8192u, 0, CFG_LDA); } while (0)
#define LDA(dst, b, h) do { _Pragma("unroll") for (int m = 0; m < 4; ++m) _Pragma("unroll") for (int k = 0; k < 2; ++k) dst[m][k] = *(const LAS bf16x8*)(lds + SA(b, h) + aoff + m * 2048 + k * 1024); } while (0)
#define LDB(dst, b, h) do { _Pragma("unroll") for (int n = 0; n < 2; ++n) _Pragma("unroll") for (int k = 0; k < 2; ++k) dst[n][k] = *(const LAS bf16x8*)(lds + SB(b, h) + boff + n * 2048 + k * 1024); } while (0)
#define MMA(ai, first, prio) do { __builtin_amdgcn_s_setprio(prio); \
        _Pragma("unroll") for (int i = 0; i < 32; ++i) { \
            const int bj = ((i >> 2) ^ (i >> 3)) & 1, m = i >> 3, n = ((i >> 1) ^ (i >> 3)) & 1, k = (i ^ (i >> 1)) & 1;     \
            SCHED; \
            acc[ai][bj][m][n] = __builtin_amdgcn_mfma_f32_16x16x32_bf16(bj ? B1[n][k] : B0[n][k], At[m][k], first && (i & 1) == 0 ? (f32x4){0.f, 0.f, 0.f, 0.f} : acc[ai][bj][m][n], 0, 0, 0); } \
        __builtin_amdgcn_s_setprio(0); } while (0)
#define WAIT_L0() do { asm volatile("" ::: "memory"); __builtin_amdgcn_s_waitcnt(0xC07F); asm volatile("" ::: "memory"); } while (0)
#define WAIT_V(n) asm volatile("s_waitcnt vmcnt(" #n ")" ::: "memory")
#define WAIT_L(n) asm volatile("s_waitcnt lgkmcnt(" #n ")" ::: "memory")
#define BAR __builtin_amdgcn_s_barrier()
#define SCHED __builtin_amdgcn_sched_barrier(0)

    unsigned long long outp = (unsigned long long)P.outl;
    asm volatile("" : "+v"(outp));
    Tile cur, nxt; int rnd = slot < KX - DK ? 0 : 1;
    const unsigned snd = WS_SND + (call & 1u) * SNDB;
    tile_of(rnd * (KX - DK) + slot, col, rank, snd, cur);
    bf16x8 At[4][2], B0[2][2], B1[2][2];
    constexpr unsigned SB0 = ((NRANK - 1) * NS + GW) * UNB, STB = U0 * UNB;
    const unsigned kov = WS_PAN + (unsigned)((rank + 1 + (lane + 3) % 7) & (NRANK - 1)) * CHB + (unsigned)(((lane & 31) - 4) / 7) * UNB;
    unsigned tabA, tabB;
    auto tables = [&](const Tile& t) {
        const int ln = lane_now();
        const unsigned la = t.sA + (unsigned)ln * t.stA, ra = (unsigned)(NTM + t.v) * PANB + kov;
        const unsigned lb = SB0 + snd + (unsigned)t.tm * UNB + (unsigned)ln * STB, rb = (unsigned)t.tm * PANB + kov;
        tabA = ln < 4 ? la : ra;
        tabB = (ln < 4 ? lb : rb) + (t.half == 1 ? HTB : 0u);
    };
    auto pick = [&]() {
        Tile t = cur; bool ok = true;
        if (rnd + 1 < RS) tile_of((rnd + 1) * KX - DK + slot, col, rank, snd, t);
        else {
            const unsigned d = (unsigned)__builtin_amdgcn_raw_ptr_buffer_atomic_add_i32(1, rws, lane_now() == 0 ? 0u : 0xfffffff0u, WS_HEAD + (unsigned)rank * 128u, 0);
            const unsigned q = (unsigned)__builtin_amdgcn_readfirstlane(d) - (call - 1u) * (unsigned)(NITEM + 8 * KX);
            ok = q < (unsigned)NITEM;
            if (ok) item_of((int)q, rank, snd, t);
        }
        *(LAS u32x4*)(ctl + 4) = (u32x4){ok ? (unsigned)(t.tm | t.v << 8 | (t.half + 2) << 16) : 0u, t.sA, t.stA, 0u};
        __builtin_amdgcn_raw_ptr_buffer_load_lds(rws, ctl + 64, 4, cnt_of(t), 0, 0, AUX_SYS);
    };
    auto picked = [&](int& level) -> bool {
        const u32x4 f = *(const LAS u32x4*)(ctl + 4);
        level = level_of(ctl[64 + lane_now()]);
        const unsigned w = __builtin_amdgcn_readfirstlane(f[0]);
        nxt.tm = w & 255u; nxt.v = (w >> 8) & 255u; nxt.half = (int)(w >> 16) - 2;
        nxt.sA = __builtin_amdgcn_readfirstlane(f[1]); nxt.stA = __builtin_amdgcn_readfirstlane(f[2]);
        return w != 0u;
    };
    constexpr int JP = KT / 2 - CFG_JP, JV = JP + 1, JI = JP + 2, JL = KT / 2 - 1;
    static_assert(JP > 24 && JI < JL, "passes");
    const int jend = wr == 0 && wid == 0 ? JP : JI;
    if (CFG_SYNC && rnd && wr == 0 && wid == 0) {
        const unsigned long long t0 = __builtin_amdgcn_s_memrealtime();
        while (__builtin_amdgcn_readfirstlane(ld_agent((const unsigned*)(wsl + WS_COL + col * 128))) != call && !give_up((unsigned*)(wsl + WS_TMO), call, t0)) __builtin_amdgcn_s_sleep(16);
    }
    int lvl = ready(cur, 1);
    STAMP(P, 2);
    tables(cur);
    unsigned a2 = cur.sA, b2 = SB0 + snd + (unsigned)cur.tm * UNB;
    STAGE(SB(0, 0), b2); STAGE(SA(0, 0), a2); STAGE(SB(0, 1), b2 + HTB); STAGE(SA(0, 1), a2 + HTB);
    STAGE(SB(1, 0), b2 + KTB); STAGE(SA(1, 0), a2 + KTB); STAGE(SB(1, 1), b2 + HTB + KTB);
    WAIT_V(0); BAR;
#define TICK(i, r) do {} while (0)
#define ROUND_DONE() do {} while (0)
    bool has_next = false;
#pragma unroll 1
    for (;;) {
        f32x4 acc[2][2][4][2];
        int jchk = lvl < 5 ? 7 * lvl - 4 : jend;
        TICK(0, 2); TICK(4, 3);
#define SYNC_MMA(ai, first, wv) do { if (wv) WAIT_V(8); WAIT_L0(); if (wr == 1) BAR; MMA(ai, first, wr + 1); if (wr == 0) BAR; SCHED; } while (0)
#define PASS(first, check) do { \
            LDB(B0, 0, 0); LDA(At, 0, 0); LDB(B1, 0, 1); STAGE(SA(1, 1), a2 + KTB + HTB); SCHED; \
            check; \
            a2 = __builtin_amdgcn_readlane(tabA, (j + 1) & 31); b2 = __builtin_amdgcn_readlane(tabB, (j + 1) & 31); \
            SYNC_MMA(0, first, !first); \
            LDA(At, 0, 1); STAGE(SB(0, 0), b2); STAGE(SA(0, 0), a2); STAGE(SB(0, 1), b2 + HTB); \
            SYNC_MMA(1, first, !first); \
            LDB(B0, 1, 0); LDA(At, 1, 0); LDB(B1, 1, 1); STAGE(SA(0, 1), a2 + HTB); \
            SYNC_MMA(0, false, true); \
            LDA(At, 1, 1); STAGE(SB(1, 0), b2 + KTB); STAGE(SA(1, 0), a2 + KTB); STAGE(SB(1, 1), b2 + KTB + HTB); \
            SYNC_MMA(1, false, true); } while (0)
        int j = 0;
        PASS(true, );
#pragma unroll 1
        for (j = 1; j < KT / 2; ++j)
            PASS(false, if (__builtin_expect(j == jchk, 0)) {
                    if (j < JP) { lvl = ready(cur, lvl + 1); jchk = lvl < 5 ? 7 * lvl - 4 : jend; }
                    else if (wr == 0 && j == JP) { pick(); jchk = JV; }
                    else if (wr == 0 && j == JV) { asm volatile("buffer_inv sc1" ::: "memory"); jchk = JI; }
                    else if (j == JI) {
                        has_next = picked(lvl); jchk = JL;
                        if (has_next && lvl >= 1) {
                            unsigned lines = voff; asm volatile("" : "+v"(lines)); lines <<= 3;
                            if (CFG_PFA && nxt.v < CFG_VD) __builtin_amdgcn_raw_ptr_buffer_load_lds(rws, ctl + 128, 4, lines, nxt.sA + (unsigned)(nxt.tm & 3) * nxt.stA, 0, 0);
                            if (CFG_PFB == 2 || (CFG_PFB && rnd == 0)) __builtin_amdgcn_raw_ptr_buffer_load_lds(rws, ctl + 128, 4, lines, SB0 + snd + (unsigned)nxt.tm * UNB + (unsigned)(nxt.v >> 1 & 3) * STB, 0, 0);
                        }
                    } else if (has_next) {
                        const int want = nxt.half < 0 ? 1 : 5;
                        if (lvl < want) lvl = ready(nxt, want);
                        tables(nxt);
                    }
                });
        if (has_next && nxt.half >= 0) STAGE(SA(1, 1), a2 + KTB + HTB);
        TICK(1, 2);
        TICK(2, 2);
        {
            const int ln = lane_now(), fr = ln & 15, fq = ln >> 4;
            const rsrc_t rol = mk_rsrc((const void*)outp, OUTB);
            const bool shared = cur.v < NS;
            if (__builtin_amdgcn_readfirstlane(ctl[0]) != 0u) {
#pragma unroll
                for (int i = 0; i < 32; ++i) acc[i >> 4][(i >> 3) & 1][(i >> 1) & 3][i & 1] = __builtin_bit_cast(f32x4, (u32x4){0x7fc00000u, 0x7fc00000u, 0x7fc00000u, 0x7fc00000u});
            }
            if (rnd == 0 && cur.v < GW) {
                const unsigned long long t0 = __builtin_amdgcn_s_memrealtime();
                while (__builtin_amdgcn_s_memrealtime() - t0 < (unsigned long long)((col & 3) * CFG_STAG + (col >> 2) * CFG_STAGB + slot * CFG_SW)) __builtin_amdgcn_s_sleep(8);
                if (DK && CFG_SYNC && wr == 0 && slot == (CFG_SW ? KX - DK - 1 : 0) && wid == 0) __builtin_amdgcn_raw_buffer_store_b32(call, rws, ln == 0 ? 0u : 0xfffffff0u, WS_COL + (unsigned)col * 128u, AUX_AGENT);
            }
            const bool thru = rnd >= RS;
            const unsigned vo = (unsigned)(wc * 32 + fq * 4) * OSTR + (unsigned)(wr * 256 + fr * 16);
            const unsigned so = (unsigned)(cur.tm * 256) * (NN * 2) + (unsigned)gpan_of(cur.v, rank) * 512u;
            const unsigned vs = (unsigned)(wc * 32 + fq * 4) * ROWB + (unsigned)(wr * 256 + fr * 16);
            const unsigned ss = WS_OSN + (unsigned)(cur.tm * 256) * ROWB + (unsigned)cur.v * 512u;
            auto finish = [&](auto mode) {
#pragma unroll
                for (int bj = 0; bj < 2; ++bj)
#pragma unroll
                    for (int n = 0; n < 2; ++n)
#pragma unroll
                        for (int r = 0; r < 4; ++r) {
                            u32x4 o;
                            if ((r & 1) == 0) {
#pragma unroll
                                for (int i = 0; i < 8; ++i) { f32x4& a = acc[i >> 2][bj][i & 3][n]; const f32x2 z = silu2((f32x2){a[r], a[r + 1]}); a[r] = z[0]; a[r + 1] = z[1]; }
                            }
#pragma unroll
                            for (int i = 0; i < 4; ++i) o[i] = cvt_pk_bf16(acc[i >> 1][bj][(i & 1) * 2][n][r], acc[i >> 1][bj][(i & 1) * 2 + 1][n][r]);
                            if (mode == 1) STORE16(o, rws, vs, ss + (unsigned)(bj * 128 + n * 16 + r) * ROWB, AUX_SYS);
                            STORE16(o, rol, vo, so + (unsigned)(bj * 128 + n * 16 + r) * OSTR, mode == 2 ? CFG_OTH : CFG_OST);
                        }
            };
            if (shared) finish(std::integral_constant<int, 1>());
            else if (thru) finish(std::integral_constant<int, 2>());
            else finish(std::integral_constant<int, 0>());
            WAIT_V(16);
            if (shared) {
                WAIT_VM0();
                unsigned old = 0;
                if (ln == 0) old = __hip_atomic_fetch_add(ctl + 1, 1u, __ATOMIC_RELAXED, __HIP_MEMORY_SCOPE_WORKGROUP);
                old = __builtin_amdgcn_readfirstlane(old);
                if ((old & 7u) == 7u) {
                    const unsigned n = (unsigned)__builtin_amdgcn_raw_ptr_buffer_atomic_add_i32(1, rws, ln == 0 ? 0u : 0xfffffff0u, WS_BAND + (unsigned)cur.tm * 128u, 0);
                    if ((unsigned)__builtin_amdgcn_readfirstlane(n) + 1u == call * NS && ln < NRANK && ln != rank)
                        st_sys((unsigned*)(P.ws[ln] + WS_FLAG + rank * FLAGB) + NIN * 4 + turn_of(cur.tm, ln), call);
                }
            }
        }
        TICK(3, 2);
        ROUND_DONE();
        if (!has_next) break;
        cur = nxt; ++rnd;
        if (cur.half >= 0) break;
    }
#define HB(b, i) ((b) < 2 ? ((i) < 2 ? SA(b, i) : SB(b, 0)) : (i) < 2 ? SB(i, 1) : 8 * HTB)
#define HSTAGE(b, t) do { const unsigned _o = ((t) & 1) * KTB, _a = __builtin_amdgcn_readlane(tabA, ((t) >> 1) & 31) + _o, _b = __builtin_amdgcn_readlane(tabB, ((t) >> 1) & 31) + _o; \
        STAGE(HB(b, 0), _a); STAGE(HB(b, 1), _a + HTB); STAGE(HB(b, 2), _b); } while (0)
#define HLOAD(b) do { _Pragma("unroll") for (int i = 0; i < 4; ++i) B0[i >> 1][i & 1] = *(const LAS bf16x8*)(lds + HB(b, 2) + boff + i * 1024); \
        _Pragma("unroll") for (int i = 0; i < 8; ++i) At[i >> 1][i & 1] = *(const LAS bf16x8*)(lds + HB(b, 0) + aoff + i * 1024); \
        _Pragma("unroll") for (int i = 0; i < 8; ++i) Au[i >> 1][i & 1] = *(const LAS bf16x8*)(lds + HB(b, 1) + aoff + i * 1024); } while (0)
#define HPASS(b, t, first) do { \
        if (wr == 0) { HLOAD(b); HSTAGE((b + 2) % 3, (t) + 2); } else { HSTAGE((b + 2) % 3, (t) + 2); SCHED; HLOAD(b); if (!first) WAIT_V(6); } \
        WAIT_L0(); if (wr == 1) BAR; \
        __builtin_amdgcn_s_setprio(wr + 1); \
        _Pragma("unroll") for (int i = 0; i < 32; ++i) { const int ai = i >> 4, m = (i >> 2) & 3, n = (i >> 1) & 1, k = i & 1; \
            acc[ai][m][n] = __builtin_amdgcn_mfma_f32_16x16x32_bf16(B0[n][k], ai ? Au[m][k] : At[m][k], first && k == 0 ? (f32x4){0.f, 0.f, 0.f, 0.f} : acc[ai][m][n], 0, 0, 0); } \
        __builtin_amdgcn_s_setprio(0); \
        if (wr == 0) { if (!first) WAIT_V(6); BAR; } SCHED; } while (0)
#pragma unroll 1
    while (PFULL < PN && has_next) {
        f32x4 acc[2][4][2];
        bf16x8 Au[4][2];
        BAR;
        HPASS(0, 0, true);
#pragma unroll 1
        for (int t = 1; t < KT; t += 3) {
            if (wr == 0 && wid == 0 && t == KT - 9) pick();
            HPASS(1, t, false); HPASS(2, t + 1, false); HPASS(0, t + 2, false);
        }
        has_next = picked(lvl);
        if (has_next) {
            (void)ready(nxt, 5);
            tables(nxt);
            HSTAGE(0, 0); HSTAGE(1, 1);
        }
        const int ln = lane_now(), fr = ln & 15, fq = ln >> 4;
        const rsrc_t rol = mk_rsrc((const void*)outp, OUTB);
        if (__builtin_amdgcn_readfirstlane(ctl[0]) != 0u) {
#pragma unroll
            for (int i = 0; i < 16; ++i) acc[i >> 3][(i >> 1) & 3][i & 1] = __builtin_bit_cast(f32x4, (u32x4){0x7fc00000u, 0x7fc00000u, 0x7fc00000u, 0x7fc00000u});
        }
        const unsigned vo = (unsigned)(wc * 32 + fq * 4) * (NN * 2) + (unsigned)(wr * 256 + fr * 16);
        const unsigned so = (unsigned)(cur.tm * 256 + cur.half * 128) * (NN * 2) + (unsigned)gpan_of(cur.v, rank) * 512u;
#pragma unroll
        for (int n = 0; n < 2; ++n)
#pragma unroll
            for (int r = 0; r < 4; ++r) {
                u32x4 o;
                if ((r & 1) == 0) {
#pragma unroll
                    for (int i = 0; i < 8; ++i) { f32x4& a = acc[i >> 2][i & 3][n]; const f32x2 z = silu2((f32x2){a[r], a[r + 1]}); a[r] = z[0]; a[r + 1] = z[1]; }
                }
#pragma unroll
                for (int i = 0; i < 4; ++i) o[i] = cvt_pk_bf16(acc[i >> 1][(i & 1) * 2][n][r], acc[i >> 1][(i & 1) * 2 + 1][n][r]);
                STORE16(o, rol, vo, so + (unsigned)(n * 16 + r) * (NN * 2), CFG_OTH);
            }
        WAIT_V(8);
        ROUND_DONE();
        cur = nxt; ++rnd;
    }
    WAIT_V(0);
}

__global__ __launch_bounds__(512, 2) void fused_gemm_ar(Params P) {
    extern __shared__ __attribute__((aligned(16))) unsigned char shm[];
    LAS unsigned char* lds = (LAS unsigned char*)shm;
    const int wid = __builtin_amdgcn_readfirstlane(threadIdx.x >> 6), lane = threadIdx.x & 63;
    const int b = __builtin_amdgcn_readfirstlane(blockIdx.x);
    const unsigned long long wsp = (unsigned long long)P.ws[lane & (NRANK - 1)];
    const unsigned t0 = (unsigned)__builtin_amdgcn_s_memrealtime();
    int rk = P.rank;
    asm volatile("" : "+s"(rk) : "s"(P.x), "s"(P.w), "s"(P.wsl), "s"(P.rank), "s"(t0) : "memory");
    unsigned* tkp = (unsigned*)(P.wsl + WS_TICKET + b * 128 + wid * 16);
    const unsigned tk = ld_agent(tkp);
    if (b < NPULL) {
        const unsigned call = __builtin_amdgcn_readfirstlane(tk) + 1u;
        st_agent(tkp, call);
        const int pi = b / PW, slot = b % PW;
        const int peer = (P.rank + 1 + pi) & (NRANK - 1);
        pull_role(P, call, peer, slot * 8 + wid, lane);
    } else {
        const int g = b - NPULL;
        if (g < 4 * NRANK * NS && wid < 2) lead_role(P, rk, wsp, tk, t0, g, wid, lane);
        ((LAS unsigned*)(lds + STAGE_BYTES))[lane & 3] = 0u;
        pack_role(P, wsp, tk, t0, g, wid, lane);
        const unsigned call = __builtin_amdgcn_readfirstlane(tk) + 1u;
        st_agent(tkp, call);
        STAMP(P, 1);
        asm volatile("s_waitcnt lgkmcnt(0)" ::: "memory");
        __builtin_amdgcn_s_barrier();
        if (b >= NPS) {
            const int col = (CFG_SWAP >> (b & 7) * 4) & 7;
            if (wid < 4) gemm_role<0>(P, call, col, (b - NPS) >> 3, lds);
            else gemm_role<1>(P, call, col, (b - NPS) >> 3, lds);
        }
    }
}
}

extern "C" void kernel_launch(int rank, int nranks, ncclComm_t comm,
                              hipStream_t stream,
                              void* const* d_in, const int* in_sizes, int n_in,
                              void* d_out, int out_size,
                              const int* in_dtypes, int out_dtype,
                              void* const* d_out_all,
                              void* const* d_ws_all, size_t ws_bytes) {
    if (nranks != NRANK || (unsigned)rank >= (unsigned)NRANK || ws_bytes < (size_t)WS_END || n_in < 2 ||
        in_sizes[0] != MM * KSH || in_sizes[1] != KSH * NN || out_size != MM * NN) {
        fprintf(stderr, "unexpected problem geometry\n");
        return;
    }
    static bool attr_done[NRANK] = {};
    if (!attr_done[rank]) {
        HIP_CHECK(hipFuncSetAttribute((const void*)fused_gemm_ar, hipFuncAttributeMaxDynamicSharedMemorySize, LDS_BYTES));
        attr_done[rank] = true;
    }
    Params P = {};
    P.order = ORDER;
    P.x = d_in[0]; P.w = d_in[1]; P.rank = rank; P.wsl = (char*)d_ws_all[rank]; P.outl = (char*)d_out_all[rank];
    for (int p = 0; p < NRANK; ++p) { P.out[p] = (char*)d_out_all[p]; P.ws[p] = (char*)d_ws_all[p]; }
    hipLaunchKernelGGL(fused_gemm_ar, dim3(GRID), dim3(512), LDS_BYTES, stream, P);
    HIP_CHECK(hipGetLastError());
}
```

```cpp
#include <hip/hip_runtime.h>
#include <rccl/rccl.h>
#include <cstdio>
#include <type_traits>

#define HIP_CHECK(expr) do { hipError_t _e = (expr); if (_e != hipSuccess) { \
    fprintf(stderr, "HIP error %s at %s:%d\n", hipGetErrorString(_e), __FILE__, __LINE__); \
    return; } } while (0)

#define LAS __attribute__((address_space(3)))
typedef short bf16x8 __attribute__((ext_vector_type(8)));
typedef float f32x4 __attribute__((ext_vector_type(4)));
typedef unsigned u32x4 __attribute__((ext_vector_type(4)));
typedef __amdgpu_buffer_rsrc_t rsrc_t;

namespace {
constexpr int NRANK = 8;
constexpr int MM = 4096, KK = 4096, NN = 32768, KSH = KK / NRANK;
constexpr int KT = KK / 64, KTS = KSH / 64, NTM = MM / 256, NPAN = NN / 256;
#define CFG_NS 6
#define CFG_PW 6
constexpr int NS = CFG_NS;
#define CFG_HALF 1
constexpr int HALF = CFG_HALF;
constexpr int SCOL = NS * 256 - HALF * 128;
constexpr int NRP = NPAN - NRANK * SCOL / 256;
constexpr int NWP = NS + NRP;
constexpr int NIN = NTM + NWP;
constexpr int NTURN = NIN * 4 - 2 * HALF;
constexpr int NFLAG = NTURN + NTM;
constexpr int NUNIT = (NTM + NRANK * NS + NRP) * 4;
constexpr int GRID = 256;
constexpr int PW = CFG_PW;
constexpr int NPULL = PW * (NRANK - 1);
constexpr int NW = PW * 8;
constexpr int NPK = GRID - NPULL;
constexpr int NPS = (NPULL + 7) / 8 * 8;
constexpr int KX = (GRID - NPS) / 8;
#define CFG_GW (2 * (KX / 4))
constexpr int GW = CFG_GW;
constexpr int G0 = GW + NTM;
#define CFG_SKIP 1
#define CFG_SYNC 1
#define CFG_GM (GW + 2)
constexpr int GM = CFG_GM;
constexpr int NGF = (NWP - GW) / GM, GL = (NWP - GW) % GM;
static_assert(NS <= GW && NWP > GW, "shared panels sit in the first group");
#define CFG_LSH 0
constexpr int LW = 1 << CFG_LSH;
constexpr int PU = 4 * LW, PSH = 2 + CFG_LSH;
#define CFG_PUSH 2
constexpr int NPUSH = CFG_PUSH;
constexpr int NPIN = (NTURN - NPUSH) * 64 / PU;
#define CFG_NL (NPUSH + NW * PU / 64 < NS - HALF ? NPUSH + NW * PU / 64 : NS - HALF)
constexpr int NL = CFG_NL;
static_assert(NPUSH <= NL && NL <= NS - HALF, "leading units have a single consumer");
constexpr int BPB = SCOL / 2 / PU;
constexpr int NPC = NPIN + NTM * BPB;
constexpr unsigned CNT_FULL = 64;

#define CFG_SWAP 0x57461302u
#define CFG_XTRA 1
constexpr int NG = CFG_XTRA ? NPK : 8 * KX;
constexpr int DK = CFG_SKIP ? KX - 2 * GW : 0, R0 = KX - DK;
constexpr int nt_of(int col) { return 4 * ((NWP + 1 - (col >> 2)) / 2); }
constexpr int kx_of(int col) { int n = 0; for (int b = GRID - NG; b < GRID; ++b) n += (int)(CFG_SWAP >> (b & 7) * 4 & 7) == col; return n; }
#define CFG_RS ((nt_of(7) - 1 - R0) / (KX + (NG > 8 * KX)) + 1)
constexpr int RS = CFG_RS;
constexpr int left_of(int col) { int n = 0; for (int c = 0; c < col; ++c) n += nt_of(c) - R0 - (RS - 1) * kx_of(c); return n; }
constexpr int PN = left_of(8);
#define CFG_FD 0
constexpr int PFULL = 2 * PN - NG - CFG_FD < 0 ? 0 : 2 * PN - NG - CFG_FD > PN ? PN : 2 * PN - NG - CFG_FD;
constexpr int NITEM = PFULL + 2 * (PN - PFULL);
constexpr unsigned KTB = 32768, HTB = 16384;
constexpr unsigned PANB = KT * KTB;
constexpr unsigned CHB = KTS * KTB;
constexpr unsigned UNB = 2 * KTB;
constexpr unsigned WS_TICKET = 0;
constexpr unsigned WS_TMO = WS_TICKET + GRID * 128;
constexpr unsigned WS_FLAG = WS_TMO + 128;
constexpr unsigned FLAGB = 4096;
constexpr unsigned WS_LEAD = WS_FLAG + NRANK * FLAGB;
constexpr unsigned WS_HEAD = WS_LEAD + NRANK * 512;
constexpr unsigned WS_COL = WS_HEAD + NRANK * 128;
#define CFG_PHOFF 0
constexpr unsigned WS_PH = WS_COL + 8 * 128 + CFG_PHOFF;
constexpr unsigned WS_ZC = WS_PH + 2 * NRANK * 128 + 128;
constexpr unsigned WS_GRP = WS_ZC + NRANK * NRANK * 128;
constexpr unsigned WS_BAND = WS_GRP + 64 * NRANK * 4;
constexpr unsigned WS_CNT = WS_BAND + NTM * 128;
constexpr unsigned WS_CTRL_END = WS_CNT + NIN * NRANK * 128;
constexpr unsigned WS_DBG = 512u << 10;
constexpr unsigned WS_SND = 1u << 20, SNDB = NUNIT * UNB;
constexpr unsigned ROWB = SCOL * 2;
constexpr unsigned WS_OSN = WS_SND + NUNIT * UNB + SNDB;
constexpr unsigned WS_PAN = (WS_OSN + MM * ROWB + PANB - 1) / PANB * PANB;
constexpr unsigned WS_END = WS_PAN + NIN * PANB;
constexpr int U0 = G0 + (NRANK - 1) * NS;
constexpr unsigned OUTB = (unsigned)MM * NN * 2;
static_assert(WS_CTRL_END <= WS_DBG, "ctrl region");
static_assert(NFLAG <= 512 && 2048 <= (int)FLAGB, "flag region");
#define CFG_DEEP (88 / LW - NW)
#define CFG_DEEPO CFG_DEEP
constexpr unsigned STAGE_BYTES = 9 * HTB;
constexpr unsigned LDS_BYTES = STAGE_BYTES + 768;
constexpr unsigned long long DEADLINE = 50000000ull;
constexpr int AUX_SYS = 17, AUX_SYS_NT = 19, AUX_AGENT = 16;
#define CFG_PACE 140
#define CFG_UH (2 * (NPK - 8 * DK))
#define CFG_TH 5000
#define CFG_PH 40
#define CFG_OST 2
#define CFG_OTH AUX_SYS_NT
#define CFG_PST AUX_AGENT
#define CFG_LDA 0
#define CFG_RLD AUX_SYS
#define CFG_TCH 3
#define CFG_GS 16
#define CFG_AGG 24
static_assert(CFG_AGG % 8 == 0 && CFG_AGG >= NS && NS <= 8 && NIN * 4 <= 512, "groups");
#define CFG_PWND 3
#define CFG_EH 1
#define CFG_MIX 1
static_assert(4 * NRANK * NL <= NPK && (NL - NPUSH) * 64 / PU <= NW && NL * 64 <= 512, "leading units");
#define CFG_PACE0 40
#define CFG_STAG 0
#define CFG_P3 320
#define CFG_P3N 2
#define CFG_STAGB 0
#define CFG_LOOK (104 / LW)
#define CFG_TAIL NW
#define CFG_EVEN (280 / LW)
#define CFG_EVEND (200 / LW)
#define CFG_ERATE CFG_RATE
#define CFG_ZB (120 / LW)
#define CFG_LOOKP (100 / LW)
#define CFG_PLAN (CFG_ZB + CFG_LOOKP + 50 / LW)
#define CFG_BAL 8
constexpr int ZB = CFG_ZB;
static_assert(ZB <= 127 && ZB <= BPB && NFLAG <= 504, "plan");
#define CFG_PWIN 4u
#define CFG_PS 2
#define CFG_RATE 1680
#define CFG_CRED 25
#define CFG_GRID 36
#define CFG_SPIN 8
#define CFG_EARLY 1
#define CFG_RATEO CFG_RATE
static_assert(CFG_DEEP >= 0 && CFG_DEEP <= NW, "window");
constexpr unsigned PACE16 = CFG_RATE * LW * (NW + CFG_DEEP) / 64, PACEO16 = CFG_RATEO * LW * (NW + CFG_DEEPO) / 64;

#define CFG_EA 8
#define CFG_EB 4
#define CFG_ER 80
#define CFG_ES 17
#define CFG_EMIN 16
constexpr int NTAB = 4 * (G0 + GM);
static_assert(NTAB <= 256, "byte tables");
struct Order {
    unsigned char from[256], to[256];
    static constexpr int due(int o) {
        const int d = o >= 4 * G0 ? CFG_ER + (o - 4 * G0) / GM * CFG_ES + ((o - 4 * G0) % GM & 1) * CFG_EB
                                  : o / G0 * CFG_ES + (o % G0 < GW ? o % G0 / (GW / 2) * CFG_EB : (o % G0 - GW) / 4 * CFG_EA);
        return HALF && o < 4 * G0 && o % G0 == NS - 1 && (o / G0 & 1) ? 1 << 20 : d < CFG_EMIN ? CFG_EMIN : d;
    }
    constexpr Order() : from(), to() {
        for (int o = 0; o < NTAB; ++o) {
            int n = 0;
            for (int k = 0; k < NTAB; ++k) n += due(k) < due(o) || (due(k) == due(o) && k < o);
            to[o] = (unsigned char)n; from[n] = (unsigned char)o;
        }
    }
};
constexpr Order ORDER;
static_assert(ORDER.to[NS - 1] == NS - 1, "the leading units come first");

struct Params {
    const void* x; const void* w;
    char* wsl; char* outl;
    int rank, pad[7];
    char* ws[NRANK]; char* out[NRANK];
    Order order;
};
__device__ __forceinline__ int look_up(unsigned tab, int o, int n, int beyond) { return o < n ? (unsigned)__builtin_amdgcn_readlane(tab, o >> 2) >> (o & 3) * 8 & 255u : o + beyond; }

__device__ __forceinline__ rsrc_t mk_rsrc(const void* p, unsigned bytes) {
    const unsigned long long v = (unsigned long long)p;
    const unsigned lo = __builtin_amdgcn_readfirstlane((unsigned)v), hi = __builtin_amdgcn_readfirstlane((unsigned)(v >> 32));
    return __builtin_amdgcn_make_buffer_rsrc((void*)(((unsigned long long)hi << 32) | lo), 0, bytes, 0x00020000);
}
__device__ __forceinline__ unsigned ld_sys(const unsigned* p) { return __hip_atomic_load(p, __ATOMIC_RELAXED, __HIP_MEMORY_SCOPE_SYSTEM); }
__device__ __forceinline__ unsigned ld_agent(const unsigned* p) { return __hip_atomic_load(p, __ATOMIC_RELAXED, __HIP_MEMORY_SCOPE_AGENT); }
__device__ __forceinline__ void st_sys(unsigned* p, unsigned v) { __hip_atomic_store(p, v, __ATOMIC_RELAXED, __HIP_MEMORY_SCOPE_SYSTEM); }
__device__ __forceinline__ void st_agent(unsigned* p, unsigned v) { __hip_atomic_store(p, v, __ATOMIC_RELAXED, __HIP_MEMORY_SCOPE_AGENT); }
__device__ __forceinline__ void add_agent(unsigned* p, unsigned v) { (void)__hip_atomic_fetch_add(p, v, __ATOMIC_RELAXED, __HIP_MEMORY_SCOPE_AGENT); }
#define STORE16(v, rs, vo, so, aux) do { const u32x4 _d = (v); __builtin_amdgcn_raw_buffer_store_b128(_d, rs, vo, so, aux); asm volatile("s_nop 1" :: "v"(_d) : "memory"); } while (0)
#define WAIT_VM0() asm volatile("s_waitcnt vmcnt(0)" ::: "memory")
#define WAIT_VMN(n) do { asm volatile("" ::: "memory"); __builtin_amdgcn_s_waitcnt(0x0F70 | (n)); asm volatile("" ::: "memory"); } while (0)

__device__ __forceinline__ bool give_up(unsigned* tmo, unsigned call, unsigned long long t0) {
    const unsigned long long dt = __builtin_amdgcn_s_memrealtime() - t0;
    if (dt < 2000ull) return false;
    if ((unsigned)__builtin_amdgcn_readfirstlane(ld_agent(tmo)) == call) return true;
    if (dt > DEADLINE) { st_agent(tmo, call); return true; }
    return false;
}

__device__ __forceinline__ int gpan_of(int v, int consumer) { return v < NS ? consumer * NS + v : (NRANK - 1) * NS + v; }
__device__ __forceinline__ int col_of(int v, int consumer) { return (v < NS ? consumer * SCOL : NRANK * SCOL - NS * 256) + v * 256; }
__device__ __forceinline__ int band_of(int e, int consumer) { return e < 7 ? (consumer + 1 + e) & 7 : e < 15 ? 8 + ((consumer + e) & 7) : consumer; }
__device__ __forceinline__ int turn_of(int band, int consumer) { return band >= 8 ? 7 + ((band - consumer + 1) & 7) : band == consumer ? 15 : (band - consumer - 1) & 7; }
__device__ __forceinline__ int unit_of(int o, int consumer, int& q, int& pos) {
    if (o < 4 * G0) { q = o / G0; const int m = o % G0; pos = q * U0 + gpan_of(m, consumer); return m < GW ? NTM + m : m - GW; }
    pos = o + 4 * (NRANK - 1) * NS;
    o -= 4 * G0;
    const int g = o / (4 * GM), r = o % (4 * GM);
    int i;
    if (GL == 0 || g < NGF) { q = r / GM; i = r % GM; } else { q = r / (GL ? GL : 1); i = r % (GL ? GL : 1); }
    return NTM + GW + GM * g + i;
}

__device__ __forceinline__ unsigned cvt_pk_bf16(float lo, float hi) { unsigned r; asm("v_cvt_pk_bf16_f32 %0, %1, %2" : "=v"(r) : "v"(lo), "v"(hi)); return r; }
__device__ __forceinline__ float silu(float y) {
    const float c = __builtin_amdgcn_fmed3f(y, -60.0f, 60.0f);
    const float e = __builtin_amdgcn_exp2f(c * -1.44269504088896f);
    return y * __builtin_amdgcn_rcpf(1.0f + e);
}

typedef float f32x2 __attribute__((ext_vector_type(2)));
__device__ __forceinline__ f32x2 silu2(f32x2 y) {
    const f32x2 t = __builtin_elementwise_fma(y, (f32x2){-1.44269504088896f, -1.44269504088896f}, (f32x2){-86.5617024533378f, -86.5617024533378f});
    const f32x2 e = {__builtin_amdgcn_fmed3f(__builtin_amdgcn_exp2f(t[0]), 0.0f, 1.0f), __builtin_amdgcn_fmed3f(__builtin_amdgcn_exp2f(t[1]), 0.0f, 1.0f)};
    const f32x2 d = __builtin_elementwise_fma(e, (f32x2){1.1420073898156842e26f, 1.1420073898156842e26f}, (f32x2){1.0f, 1.0f});
    return y * (f32x2){__builtin_amdgcn_rcpf(d[0]), __builtin_amdgcn_rcpf(d[1])};
}

#define STAMP1(P, slot) do {} while (0)
#define STAMP(P, slot) do {} while (0)

__device__ __forceinline__ void pull_role(const Params& P, unsigned call, int peer, int widx, int lane) {
    const int rank = P.rank;
    char* wsl = P.wsl;
    const rsrc_t rwr = mk_rsrc(P.ws[peer], WS_END), rwl = mk_rsrc(wsl, WS_END);
    const rsrc_t rol = mk_rsrc(P.outl, OUTB), ror = mk_rsrc(P.out[peer], OUTB);
    unsigned* tmo = (unsigned*)(wsl + WS_TMO);
    constexpr unsigned OOB = 0xfffffff0u;
    const unsigned lane0 = lane == 0 ? 0u : OOB;
    const unsigned hoff = WS_PH + (call & 1u) * (NRANK * 128u);
    auto draw = [&]() { return (unsigned)__builtin_amdgcn_raw_ptr_buffer_atomic_add_i32(1, rwl, lane0, hoff + (unsigned)peer * 128u, 0); };
    if (widx == 0) __builtin_amdgcn_raw_buffer_store_b32(0u, rwl, lane0, WS_PH + (~call & 1u) * (NRANK * 128u) + (unsigned)peer * 128u, AUX_SYS);
    auto arrive = [&](unsigned vo, unsigned cnt) { (void)__builtin_amdgcn_raw_ptr_buffer_atomic_add_i32(PU, rwl, vo, cnt, 0); };
    const int wrk = (widx & 7) * PW + (widx >> 3);
    const unsigned perI = wrk < CFG_DEEP ? PACE16 / 2 : PACE16, perO = wrk < CFG_DEEPO ? PACEO16 / 2 : PACEO16;
    const unsigned from = ((const unsigned*)P.order.from)[lane];
    bool bad = false; int stamped = 0; (void)stamped;
    auto index_of = [&](unsigned v) { return (int)__builtin_amdgcn_readfirstlane(v) + NW; };
    int cur = index_of((unsigned)(widx - NW));
    const unsigned fso = WS_FLAG + (unsigned)peer * FLAGB;
    auto first_of = [&](int i) { return i < NPIN ? (i >> (6 - PSH)) + NPUSH : NTURN + (i - NPIN) / BPB; };
    int lim = 0, flim = 0, fin = NPC;
    bool planned = ZB == 0;
    unsigned long long plan = 0;
    auto moved = [&](int j) { return (int)(signed char)(plan >> (j * 8)); };
    auto update = [&](const u32x4& f0, const u32x4& f1, int i) {
        const int first = first_of(i);
        unsigned un = 0;
#pragma unroll
        for (int j = 0; j < 8; ++j) un |= ((int)((j < 4 ? f0[j & 3] : f1[j & 3]) - call) < 0 && lane * 8 + j >= first ? 1u : 0u) << j;
        const int l = __builtin_ctzll(__ballot(un != 0u) | (1ull << 63));
        const int u = l * 8 + __builtin_ctz(__builtin_amdgcn_readlane(un, l) | 256u);
        const int fl = u <= NTURN ? (u - NPUSH) << (6 - PSH) : u >= NFLAG ? NPC + 128 : NPIN + (u - NTURN) * BPB;
        flim = fl > flim ? fl : flim;
        if (!planned && (unsigned)__builtin_amdgcn_readlane(f0[0], 63) == call && (unsigned)__builtin_amdgcn_readlane(f0[2], 63) == call) {
            plan = (unsigned long long)(unsigned)__builtin_amdgcn_readlane(f0[3], 63) << 32 | (unsigned)__builtin_amdgcn_readlane(f0[1], 63);
            planned = true; fin = NPC - moved(peer);
        }
        const int pl = planned ? fin : NPC - ZB;
        lim = flim < pl ? flim : pl;
    };
    auto make_plan = [&]() {
        unsigned old = 0;
        if (lane == 0) old = __hip_atomic_exchange((unsigned*)(wsl + WS_PH + 2 * NRANK * 128), call, __ATOMIC_RELAXED, __HIP_MEMORY_SCOPE_AGENT);
        if ((unsigned)__builtin_amdgcn_readfirstlane(old) == call) return;
        const unsigned hd = __builtin_amdgcn_raw_buffer_load_b32(rwl, lane < NRANK ? (unsigned)lane * 128u : OOB, hoff, AUX_SYS);
        const int t = lane >> 3, s = lane & 7;
        const unsigned zc = ld_sys((const unsigned*)(P.ws[t] + WS_ZC + (rank * NRANK + s) * 128));
        const bool ok = __all(t == rank || s == rank || s == t || (int)(zc - call * (unsigned)(ZB * PU)) >= 0);
        int h[NRANK], sum = 0, give = 0, take = 0;
#pragma unroll
        for (int j = 0; j < NRANK; ++j) { h[j] = __builtin_amdgcn_readlane(hd, j); sum += j == rank ? 0 : h[j]; }
#pragma unroll
        for (int j = 0; j < NRANK; ++j) {
            const int d = j == rank ? 0 : (sum - (NRANK - 1) * h[j]) * CFG_BAL / (8 * (NRANK - 1));
            h[j] = d > ZB ? ZB : d < -127 ? -127 : d;
            if (h[j] > 0) give += h[j]; else take -= h[j];
        }
        give = take = !ok ? 0 : give < take ? give : take;
        unsigned long long pl = 0;
#pragma unroll
        for (int j = 0; j < NRANK; ++j) {
            int d = h[j];
            if (d > 0) { d = d < give ? d : give; give -= d; } else { d = -d < take ? d : -take; take += d; }
            pl |= (unsigned long long)(d & 255) << (j * 8);
        }
        __builtin_amdgcn_raw_buffer_store_b128((u32x4){call, (unsigned)pl, call, (unsigned)(pl >> 32)}, rwl, lane < NRANK ? WS_FLAG + 63 * 32 + (unsigned)lane * FLAGB : OOB, 0, AUX_SYS);
    };
    auto offs_of = [&](int i, unsigned (&o)[PU], unsigned& cnt) -> unsigned {
        if (i < NPIN) {
            int q, pos; const int pan = unit_of(look_up(from, (i >> (6 - PSH)) + NPUSH, NTAB - 2 * HALF, 2 * HALF), rank, q, pos);
            cnt = WS_CNT + (unsigned)(pan * NRANK + peer) * 128u + (unsigned)q * 4u;
            unsigned sub = (unsigned)((i << PSH) & 63) * 1024u;
            if (HALF && pan == NTM + NS - 1) { sub += sub & ~8191u; pos += sub >> 16 ? U0 - 1 : 0; }
            const unsigned b = WS_PAN + (unsigned)pan * PANB + (unsigned)peer * CHB + (unsigned)q * UNB + sub + (unsigned)lane * 16u;
#pragma unroll
            for (int k = 0; k < PU; ++k) o[k] = b + k * 1024u;
            return WS_SND + (call & 1u) * SNDB + (unsigned)pos * UNB + sub;
        }
        int src = peer;
        if (ZB && i >= NPC) {
            int x = i - NPC;
            for (int j = 0; j < peer; ++j) x -= moved(j) < 0 ? moved(j) : 0;
            for (int j = 0; j < NRANK; ++j) { const int g = moved(j) > 0 ? moved(j) : 0; if ((unsigned)x < (unsigned)g) { src = j; i = NPC - g + x; } x -= g; }
        }
        const int e = (i - NPIN) / BPB, r = (i - NPIN) % BPB, pb = band_of(e, rank) * BPB + r;
        cnt = e < NRANK - 1 && r >= BPB - ZB ? WS_ZC + (unsigned)(band_of(e, rank) * NRANK + peer) * 128u : 0u;
#pragma unroll
        for (int k = 0; k < PU; ++k) {
            const unsigned sg = (unsigned)((pb << PSH) + k) * 4u + (unsigned)(lane >> 4);
            o[k] = (unsigned)src * ROWB + (unsigned)(lane & 15) * 16u + sg / (ROWB / 256) * (NN * 2u - ROWB) + sg * 256u;
        }
        return WS_OSN + (unsigned)(pb << PSH) * 1024u;
    };
    if (cur < (NL - NPUSH) << (6 - PSH)) {
        const unsigned long long t0 = __builtin_amdgcn_s_memrealtime();
        const unsigned fo = fso + 2048u + (unsigned)(((cur * LW >> 3) + 2 * NPUSH) * 2 + (cur * LW >> 1 & 1)) * 8u;
        for (;;) {
            const unsigned f = __builtin_amdgcn_raw_buffer_load_b32(rwl, (unsigned)(lane & (LW < 4 ? 1 : 3)) * 4u, fo, AUX_SYS);
            if (__all((int)(f - call) >= 0)) { lim = flim = cur + 1; break; }
            if (give_up(tmo, call, t0)) { bad = true; planned = true; break; }
            __builtin_amdgcn_s_sleep(CFG_PS);
        }
        asm volatile("" ::: "memory");
    }
    asm volatile("" :: "v"(from));
    while (!planned || cur < fin) {
        if (!bad && cur >= lim) {
            const unsigned long long t0 = __builtin_amdgcn_s_memrealtime();
            const unsigned pvo = (unsigned)(lane - (first_of(cur) >> 3)) < CFG_PWIN || (lane == 63 && cur >= NPC - ZB) ? (unsigned)lane * 32u : OOB - 16u;
            for (;;) {
                const u32x4 f0 = __builtin_amdgcn_raw_buffer_load_b128(rwl, pvo, fso, AUX_SYS), f1 = __builtin_amdgcn_raw_buffer_load_b128(rwl, pvo + 16u, fso, AUX_SYS);
                update(f0, f1, cur);
                if (cur < lim || (planned && cur >= fin)) break;
                if (give_up(tmo, call, t0)) { bad = true; planned = true; plan = 0; fin = NPC; break; }
                __builtin_amdgcn_s_sleep(CFG_PS);
            }
            asm volatile("" ::: "memory");
            if (cur >= fin && planned) break;
        }
        unsigned o[PU], p2[LW], p3[LW], cprev = 0;
#pragma unroll
        for (int j = 0; j < LW; ++j) p2[j] = p3[j] = OOB;
        if (bad) {
            (void)offs_of(cur, o, cprev);
            if (cur >= NPIN) {
#pragma unroll
                for (int k = 0; k < PU; ++k) __builtin_amdgcn_raw_buffer_store_b128((u32x4){0x7fc07fc0u, 0x7fc07fc0u, 0x7fc07fc0u, 0x7fc07fc0u}, rol, o[k], 0, AUX_AGENT);
            }
            cur = index_of(draw()); continue;
        }
        STAMP1(P, 1);
        rsrc_t rp = rwl;
        auto put = [&](const u32x4* d, const rsrc_t r, const unsigned* a) {
#pragma unroll
            for (int j = 0; j < LW; ++j) __builtin_amdgcn_raw_buffer_store_b128(d[j], r, a[j], 0, CFG_PST);
        };
        u32x4 v[PU] = {};
        unsigned slot = 0;
        if (CFG_RATE) { const unsigned now = (unsigned)__builtin_amdgcn_s_memrealtime() << 4; slot = now - (now & 0xffffffu) % PACE16 + (unsigned)wrk * (PACE16 / NW) - PACE16; }
        unsigned cb, src, in[PU];
        auto prepare = [&]() {
            if (ZB && cur == NPC - CFG_PLAN) make_plan();
            src = __builtin_amdgcn_readfirstlane(offs_of(cur, o, cb)); cb = __builtin_amdgcn_readfirstlane(cb);
            if (ZB && cur >= NPC) src = 0;
#pragma unroll
            for (int k = 0; k < PU; ++k) { in[k] = ZB && cur >= NPC ? o[k] : lane * 16 + k * 1024; asm volatile("" : "+v"(o[k]), "+v"(in[k])); }
        };
        prepare();
        do {
            const bool deep0 = wrk < (cur < NPIN ? CFG_DEEP : CFG_DEEPO);
            const bool tail = cur >= fin - CFG_TAIL, even = cur >= fin - (deep0 ? CFG_EVEND : CFG_EVEN), deep = deep0 || tail || even;
            const unsigned per = even ? CFG_ERATE * LW * NW / 64 : cur < NPIN ? perI : perO;
            const rsrc_t rd = cur < NPIN ? rwl : rol, rs = ZB && cur >= NPC ? ror : rwr;
            const bool look = flim < NPC && cur + CFG_LOOK >= flim, lookp = !planned && cur + CFG_LOOKP >= NPC - ZB;
            const unsigned fvo = look || (lookp && lane == 63) ? (unsigned)lane * 32u : OOB - 16u;
            unsigned cdone = cprev;
            unsigned nv = 0; u32x4 f0, f1;
#pragma unroll
            for (int k = 0; k < 4; ++k) {
                if (CFG_RATE && !tail) {
                    slot += per;
                    if (CFG_GRID) {
                        unsigned go = (unsigned)__builtin_amdgcn_s_memrealtime();
                        const int d = (int)((go << 4) - slot);
                        if (d > CFG_CRED * 16) slot += (unsigned)(d - CFG_CRED * 16);
                        if (d < 0) go += (unsigned)(15 - d) >> 4;
                        go += CFG_GRID - (go & 0xffffffu) % (CFG_GRID ? CFG_GRID : 1);
                        for (;;) {
                            const int w = (int)(go - (unsigned)__builtin_amdgcn_s_memrealtime());
                            if (w <= 0) break;
                            if (w > CFG_SPIN) __builtin_amdgcn_s_sleep(1);
                        }
                    } else for (;;) {
                        const int d = (int)(((unsigned)__builtin_amdgcn_s_memrealtime() << 4) - slot);
                        if (d > CFG_CRED * 16) slot += (unsigned)(d - CFG_CRED * 16);
                        if (d >= 0) break;
                        __builtin_amdgcn_s_sleep(1);
                    }
                }
                if (k == 2) { f0 = __builtin_amdgcn_raw_buffer_load_b128(rwl, fvo, fso, AUX_SYS); f1 = __builtin_amdgcn_raw_buffer_load_b128(rwl, fvo + 16u, fso, AUX_SYS); asm volatile("" ::: "memory"); }
#pragma unroll
                for (int j = 0; j < LW; ++j) v[k * LW + j] = __builtin_amdgcn_raw_buffer_load_b128(rs, in[k * LW + j], src, CFG_RLD);
                if (k == 0) nv = draw();
                put(v + (k ^ 2) * LW, k < 2 ? rp : rd, k == 0 ? p2 : k == 1 ? p3 : o + (k ^ 2) * LW);
                if (CFG_EARLY && !deep) {
                    unsigned* a = k == 0 ? p3 : o + (k - 1) * LW;
                    put(v + ((k + 3) & 3) * LW, k < 1 ? rp : rd, a);
#pragma unroll
                    for (int j = 0; j < LW; ++j) a[j] = OOB;
                }
                if (k == 3) {
                    asm volatile("" ::: "memory");
                    cprev = cb; rp = rd;
#pragma unroll
                    for (int j = 0; j < LW; ++j) { p2[j] = o[2 * LW + j]; p3[j] = o[3 * LW + j]; }
                    cur = index_of(nv);
                    if (look || lookp) update(f0, f1, cur);
                    if (cur < lim) prepare();
                }
                if (k == 0) WAIT_VMN(3 * LW + 2); else if (k == 1) WAIT_VMN(3 * LW + 1); else if (k == 2) WAIT_VMN(3 * LW + 2); else WAIT_VMN(3 * LW);
                if (!deep) {
                    if (k == 0) WAIT_VMN(1 + (1 + CFG_EARLY) * LW); else WAIT_VMN((1 + CFG_EARLY) * LW);
                    if (CFG_EARLY && k == 1) { arrive(cdone ? lane0 : OOB, cdone); cdone = 0; }
                }
            }
            arrive(cdone ? lane0 : OOB, cdone);
        } while (cur < lim);
        put(v + 2 * LW, rp, p2);
        put(v + 3 * LW, rp, p3);
        if (cprev) { WAIT_VM0(); arrive(lane0, cprev); }
    }
    STAMP(P, 4);
}

__device__ __forceinline__ void load_w(u32x4 (&v)[8], const rsrc_t rw, int col, int q, int job, int lane) {
    const unsigned vo = (unsigned)((lane >> 4) * 8) * (NN * 2) + (unsigned)(lane & 15) * 16u;
    const unsigned so = (unsigned)(q * 128 + (job >> 1) * 32) * (NN * 2) + (unsigned)(col + (job & 1) * 128) * 2u;
#pragma unroll
    for (int j = 0; j < 8; ++j) v[j] = __builtin_amdgcn_raw_buffer_load_b128(rw, vo, so + (unsigned)j * (NN * 2), 2);
}
__device__ __forceinline__ void store_w(const u32x4 (&v)[8], const rsrc_t rd, unsigned vo, unsigned unit, int job, bool slow) {
    unit += (unsigned)(job >> 2) * KTB + (unsigned)(job & 1) * 8192u + (unsigned)(job >> 1 & 1) * 1024u;
#pragma unroll
    for (int i = 0; i < 8; ++i) {
        u32x4 o;
#pragma unroll
        for (int e = 0; e < 4; ++e) o[e] = __builtin_amdgcn_perm(v[2 * e + 1][i >> 1], v[2 * e][i >> 1], (i & 1) ? 0x07060302u : 0x05040100u);
        STORE16(o, rd, vo, unit + (unsigned)(i >> 2) * HTB + (unsigned)(i & 3) * 2048u, AUX_SYS);
        if (CFG_PWND && i >= CFG_PWND - 1 && i < 7 && slow) WAIT_VMN(CFG_PWND ? CFG_PWND - 1 : 0);
    }
}
typedef __attribute__((address_space(1))) unsigned* gptr_t;

__device__ __forceinline__ void lead_role(const Params& P, int rank, unsigned long long wsp, unsigned tk, unsigned t0, int g, int wid, int lane) {
    const int o = NL - 1 - (g >> 5), c = (g >> 3) & 3, job = (c >> 1) * 4 + wid * 2 + (c & 1);
    const int dest = (rank + 1 + g + (o < NPUSH ? CFG_MIX * (2 * c + o) : 0)) & (NRANK - 1);
    u32x4 v[8];
    load_w(v, mk_rsrc(P.w, (unsigned)KSH * NN * 2), dest * SCOL + o * 256, 0, job, lane);
    asm volatile("" : "+v"(tk), "+v"(wsp) :: "memory");
    const bool push = o < NPUSH && dest != rank, count = o < NPUSH || dest == rank;
    char* wsd = (char*)((unsigned long long)(unsigned)__builtin_amdgcn_readlane((unsigned)(wsp >> 32), dest) << 32 | (unsigned)__builtin_amdgcn_readlane((unsigned)wsp, dest));
    gptr_t fp = (gptr_t)(wsd + (count ? WS_CNT + (unsigned)((NTM + o) * NRANK + rank) * 128u : WS_FLAG + (unsigned)rank * FLAGB + 2048u + (unsigned)((o * 4 + c) * 2 + wid) * 4u));
    while (CFG_P3 && push && o >= CFG_P3N && (int)((unsigned)__builtin_amdgcn_s_memrealtime() - t0) < CFG_P3 + (o - CFG_P3N) * 110) __builtin_amdgcn_s_sleep(1);
    store_w(v, mk_rsrc(push ? wsd : P.wsl, WS_END), (unsigned)lane * 16u + (push ? 0u : (~tk & 1u) * SNDB),
            push ? WS_PAN + (unsigned)(NTM + o) * PANB + (unsigned)rank * CHB : WS_SND + (unsigned)(dest * NS + o) * UNB, job, push);
    WAIT_VMN(0);
    STAMP(P, 6);
    if (lane == 0) {
        if (!count) __hip_atomic_store(fp, tk + 1u, __ATOMIC_RELAXED, __HIP_MEMORY_SCOPE_SYSTEM);
        else if ((__hip_atomic_fetch_add((gptr_t)(P.wsl + WS_LEAD + (o * NRANK + dest) * 64), 1u, __ATOMIC_RELAXED, __HIP_MEMORY_SCOPE_AGENT) & 7u) == 7u)
            __hip_atomic_store(fp, (tk + 1u) * CNT_FULL, __ATOMIC_RELAXED, __HIP_MEMORY_SCOPE_SYSTEM);
    }
}

__device__ __forceinline__ void pack_role(const Params& P, unsigned long long wsp, unsigned tk, unsigned t0, int g, int wid, int lane) {
    const int rank = P.rank;
    const rsrc_t rx = mk_rsrc(P.x, (unsigned)MM * KSH * 2), rw = mk_rsrc(P.w, (unsigned)KSH * NN * 2), rwl = mk_rsrc(P.wsl, WS_END);
    const unsigned sv = (unsigned)lane * 16u + (~tk & 1u) * SNDB;
    const unsigned to = ((const unsigned*)P.order.to)[lane];
    constexpr int NH = CFG_UH < NUNIT ? 8 * DK : 0, UH = NH ? CFG_UH : NUNIT;
    const bool late = g >= NPK - NH;
    constexpr int U1 = NRANK * NL, NE = CFG_EH ? NH : 0;
    static_assert(CFG_TH < (UH - U1 - NH) * CFG_PH, "unpaced");
    for (int u = late ? (NE ? U1 : UH) + g - (NPK - NH) : U1 + NE + g; u < (late ? NUNIT : UH); u += !late ? NPK - NH : u < UH ? UH - U1 : NH) {
        int o, dest = -1;
        if (u < 4 * U0) {
            const int r = u % U0;
            if (r < NRANK * NS) { o = r >> 3; dest = (rank + 1 + r) & (NRANK - 1); } else o = r - (NRANK - 1) * NS;
            o += u / U0 * G0;
        } else o = u - 4 * (NRANK - 1) * NS;
        int q, pos; const int pan = unit_of(o, dest < 0 ? 0 : dest, q, pos);
        const bool narrow = HALF && pan == NTM + NS - 1;
        if (narrow && (q & 1)) continue;
        while ((int)((unsigned)__builtin_amdgcn_s_memrealtime() - t0) < (late ? CFG_TH + (u - UH) * CFG_PH : (u - CFG_PACE0) * CFG_PACE / 10)) __builtin_amdgcn_s_sleep(1);
        const unsigned dst = WS_SND + (unsigned)pos * UNB;
        u32x4 v[8];
        if (pan < NTM) {
            const int rb0 = (wid & 3) * 4;
            const unsigned vo = (unsigned)(lane & 15) * (KSH * 2) + (unsigned)(lane >> 4) * 16u;
            const unsigned so = (unsigned)(pan * 256 + rb0 * 16) * (KSH * 2) + (unsigned)(q * 2 + (wid >> 2)) * 128u;
#pragma unroll
            for (int i = 0; i < 8; ++i) v[i] = __builtin_amdgcn_raw_buffer_load_b128(rx, vo + (i & 1) * 64, so + (unsigned)(i >> 1) * (16 * KSH * 2), 0);
#pragma unroll
            for (int i = 0; i < 8; ++i) STORE16(v[i], rwl, sv + (i & 1) * 1024, dst + (unsigned)(wid >> 2) * KTB + (unsigned)(rb0 + (i >> 1)) * 2048u, AUX_SYS);
        } else {
            const int job = narrow ? (wid & 3) * 2 : wid, q1 = narrow ? wid >> 2 : 0;
            load_w(v, rw, col_of(pan - NTM, dest < 0 ? 0 : dest), q + q1, job, lane);
            store_w(v, rwl, sv, dst + (unsigned)(q1 * U0) * UNB, job, false);
        }
        o = look_up(to, o, NTAB, -2 * HALF);
        const bool own = lane == rank;
        unsigned long long fp = wsp + (own ? WS_CNT + (unsigned)((pan * NRANK + rank) * 32 + q) * 4u : WS_FLAG + (unsigned)rank * FLAGB + (unsigned)o * 4u);
        unsigned fv = own ? (tk + 1u) * CNT_FULL : tk + 1u;
        asm volatile("" : "+v"(fp), "+v"(fv));
        WAIT_VMN(0);
        __builtin_amdgcn_s_barrier();
        const bool mine = lane < NRANK && (dest < 0 || dest == lane);
        if (wid == 0 && mine && (own || o < CFG_AGG || o >= NTURN / 8 * 8)) __hip_atomic_store((gptr_t)fp, fv, __ATOMIC_RELAXED, __HIP_MEMORY_SCOPE_SYSTEM);
        if (wid == 1 && o >= CFG_AGG && o < NTURN / 8 * 8) {
            unsigned old = 0;
            if (mine && !own) old = __hip_atomic_fetch_add((gptr_t)(P.wsl + WS_GRP) + (o >> 3) * NRANK + lane, 1u, __ATOMIC_RELAXED, __HIP_MEMORY_SCOPE_AGENT);
            const unsigned full = (unsigned)__ballot((old & 7u) == 7u);
            const unsigned long long wsq = (unsigned long long)(unsigned)__builtin_amdgcn_ds_bpermute(lane >> 3 << 2, (int)(wsp >> 32)) << 32 | (unsigned)__builtin_amdgcn_ds_bpermute(lane >> 3 << 2, (int)wsp);
            if (full >> (lane >> 3) & 1) __hip_atomic_store((gptr_t)(wsq + WS_FLAG + (unsigned)rank * FLAGB) + (o & ~7) + (lane & 7), tk + 1u, __ATOMIC_RELAXED, __HIP_MEMORY_SCOPE_SYSTEM);
        }
    }
}

struct Tile { int tm, v, half; unsigned sA, stA; };
__device__ __forceinline__ void tile_of(int idx, int col, int rank, unsigned snd, Tile& t) {
    const int v = idx < 2 * GW ? (idx >> 2) + (col >> 2) * (GW / 2) : 2 * (idx >> 2) + (col >> 2);
    t.tm = 4 * (col & 3) + (idx & 3); t.v = v; t.half = -1;
    const int g = (v - GW) / GM;
    t.sA = snd + (unsigned)(v < GW ? gpan_of(v, rank) : 4 * U0 + g * 3 * GM + v - GW) * UNB;
    t.stA = (unsigned)(v < GW ? U0 : g < NGF ? GM : GL) * UNB;
}
__device__ __forceinline__ void item_of(int q, int rank, unsigned snd, Tile& t) {
    const int n = q < PFULL ? q : PFULL + ((q - PFULL) >> 1);
    int at = 0;
    auto from = [&](auto c) { constexpr int l = left_of(c), a = (nt_of(c) - left_of(c + 1)) * 8 + c; at = n >= l ? a : at; };
#define C(i) std::integral_constant<int, i>()
    from(C(0)); from(C(1)); from(C(2)); from(C(3)); from(C(4)); from(C(5)); from(C(6)); from(C(7));
#undef C
    tile_of((at >> 3) + n, at & 7, rank, snd, t);
    if (q >= PFULL) t.half = (q - PFULL) & 1;
}

template <int wr>
__device__ __forceinline__ void gemm_role(const Params& P, unsigned call, int col, int slot, int kx, LAS unsigned char* lds) {
    const int tid = threadIdx.x, wid = __builtin_amdgcn_readfirstlane(tid >> 6), lane = tid & 63, wc = wid & 3;
    const int rank = P.rank;
    char* wsl = P.wsl;
    const rsrc_t rws = mk_rsrc(wsl, WS_END);
    LAS unsigned* ctl = (LAS unsigned*)(lds + STAGE_BYTES);
    const unsigned voff = (unsigned)tid * 16u;
    const unsigned ldsw = (unsigned)wid * 1024u;
    const int aoff = wr * 8192 + lane * 16, boff = wc * 4096 + lane * 16;

    auto lane_now = [&]() { int tz = threadIdx.x; asm volatile("" : "+v"(tz)); return tz & 63; };
    auto cnt_of = [&](const Tile& t) { const int ln = lane_now(); return WS_CNT + (unsigned)((((ln & 32) ? NTM + t.v : t.tm) * NRANK + ((ln >> 2) & 7)) * 32 + (ln & (HALF && t.v == NS - 1 && ln >= 32 ? 2 : 3))) * 4u; };
    auto level_of = [&](unsigned c) -> int {
        const unsigned long long miss = __ballot((int)(c - call * CNT_FULL) < 0);
        unsigned m = (unsigned)miss | (unsigned)(miss >> 32);
        const bool own = ((m >> (rank * 4)) & 15u) == 0u;
        m |= m >> 16; m |= m >> 8; m |= m >> 4;
        return own ? 1 + __builtin_ctz(m | 16u) : 0;
    };
    auto ready = [&](const Tile& t, int want, bool first = false) -> int {
        if (wr == 0 && wid == 0) {
            const unsigned* c = (const unsigned*)(wsl + cnt_of(t));
            unsigned* tmo = (unsigned*)(wsl + WS_TMO);
            const unsigned long long t0 = __builtin_amdgcn_s_memrealtime();
            int got;
            for (;;) {
                got = level_of(ld_sys(c));
                if (got >= want) break;
                if (give_up(tmo, call, t0)) { ctl[0] = 1u; got = 5; break; }
                __builtin_amdgcn_s_sleep(CFG_GS);
            }
            ctl[2] = (unsigned)got;
            if (first) __builtin_amdgcn_fence(__ATOMIC_ACQUIRE, "agent");
            asm volatile("s_waitcnt vmcnt(0) lgkmcnt(0)" ::: "memory");
        }
        asm volatile("" ::: "memory"); __builtin_amdgcn_s_barrier(); asm volatile("" ::: "memory");
        const int r = __builtin_amdgcn_readfirstlane(ctl[2]);
        asm volatile("s_waitcnt lgkmcnt(0)" ::: "memory");
        return r;
    };

#define SA(b, h) (((b) * 2 + (h)) * HTB)
#define SB(b, h) ((4 + (b) * 2 + (h)) * HTB)
#define STAGE(bufoff, soff) do { _Pragma("unroll") for (int _i = 0; _i < 2; ++_i) \
        __builtin_amdgcn_raw_ptr_buffer_load_lds(rws, (LAS unsigned*)(lds + (bufoff) + ldsw + _i * 8192), 16, voff, (soff) + _i * 8192u, 0, CFG_LDA); } while (0)
#define LDA(dst, b, h) do { _Pragma("unroll") for (int m = 0; m < 4; ++m) _Pragma("unroll") for (int k = 0; k < 2; ++k) dst[m][k] = *(const LAS bf16x8*)(lds + SA(b, h) + aoff + m * 2048 + k * 1024); } while (0)
#define LDB(dst, b, h) do { _Pragma("unroll") for (int n = 0; n < 2; ++n) _Pragma("unroll") for (int k = 0; k < 2; ++k) dst[n][k] = *(const LAS bf16x8*)(lds + SB(b, h) + boff + n * 2048 + k * 1024); } while (0)
#define MMA(ai, first, prio) do { __builtin_amdgcn_s_setprio(prio); \
        _Pragma("unroll") for (int i = 0; i < 32; ++i) { \
            const int bj = ((i >> 2) ^ (i >> 3)) & 1, m = i >> 3, n = ((i >> 1) ^ (i >> 3)) & 1, k = (i ^ (i >> 1)) & 1;     \
            SCHED; \
            acc[ai][bj][m][n] = __builtin_amdgcn_mfma_f32_16x16x32_bf16(bj ? B1[n][k] : B0[n][k], At[m][k], first && (i & 1) == 0 ? (f32x4){0.f, 0.f, 0.f, 0.f} : acc[ai][bj][m][n], 0, 0, 0); } \
        __builtin_amdgcn_s_setprio(0); } while (0)
#define WAIT_L0() do { asm volatile("" ::: "memory"); __builtin_amdgcn_s_waitcnt(0xC07F); asm volatile("" ::: "memory"); } while (0)
#define WAIT_V(n) asm volatile("s_waitcnt vmcnt(" #n ")" ::: "memory")
#define WAIT_L(n) asm volatile("s_waitcnt lgkmcnt(" #n ")" ::: "memory")
#define BAR __builtin_amdgcn_s_barrier()
#define SCHED __builtin_amdgcn_sched_barrier(0)

    unsigned long long outp = (unsigned long long)P.outl;
    asm volatile("" : "+v"(outp));
    Tile cur, nxt; int rnd = slot < R0 ? 0 : 1;
    const unsigned snd = WS_SND + (call & 1u) * SNDB;
    tile_of(rnd * R0 + slot, col, rank, snd, cur);
    bf16x8 At[4][2], B0[2][2], B1[2][2];
    constexpr unsigned SB0 = ((NRANK - 1) * NS + GW) * UNB, STB = U0 * UNB;
    const unsigned kov = WS_PAN + (unsigned)((rank + 1 + (lane + 3) % 7) & (NRANK - 1)) * CHB + (unsigned)(((lane & 31) - 4) / 7) * UNB;
    unsigned tabA, tabB;
    auto tables = [&](const Tile& t) {
        const int ln = lane_now();
        const unsigned la = t.sA + (unsigned)ln * t.stA, ra = (unsigned)(NTM + t.v) * PANB + kov;
        const unsigned lb = SB0 + snd + (unsigned)t.tm * UNB + (unsigned)ln * STB, rb = (unsigned)t.tm * PANB + kov;
        tabA = ln < 4 ? la : ra;
        tabB = (ln < 4 ? lb : rb) + (t.half == 1 ? HTB : 0u);
    };
    auto pick = [&]() {
        Tile t = cur; bool ok = true;
        if (rnd + 1 < RS) tile_of(R0 + rnd * kx + slot, col, rank, snd, t);
        else {
            const unsigned d = (unsigned)__builtin_amdgcn_raw_ptr_buffer_atomic_add_i32(1, rws, lane_now() == 0 ? 0u : 0xfffffff0u, WS_HEAD + (unsigned)rank * 128u, 0);
            const unsigned q = (unsigned)__builtin_amdgcn_readfirstlane(d) - (call - 1u) * (unsigned)(NITEM + NG);
            ok = q < (unsigned)NITEM;
            if (ok) item_of((int)q, rank, snd, t);
        }
        *(LAS u32x4*)(ctl + 4) = (u32x4){ok ? (unsigned)(t.tm | t.v << 8 | (t.half + 2) << 16) : 0u, t.sA, t.stA, 0u};
        __builtin_amdgcn_raw_ptr_buffer_load_lds(rws, ctl + 64, 4, cnt_of(t), 0, 0, AUX_SYS);
    };
    auto picked = [&](int& level) -> bool {
        const u32x4 f = *(const LAS u32x4*)(ctl + 4);
        level = level_of(ctl[64 + lane_now()]);
        const unsigned w = __builtin_amdgcn_readfirstlane(f[0]);
        nxt.tm = w & 255u; nxt.v = (w >> 8) & 255u; nxt.half = (int)(w >> 16) - 2;
        nxt.sA = __builtin_amdgcn_readfirstlane(f[1]); nxt.stA = __builtin_amdgcn_readfirstlane(f[2]);
        return w != 0u;
    };
    constexpr int JP = KT / 2 - 3, JL = KT / 2 - 1;
    const int jend = wr == 0 && wid == 0 ? JP : JL;
    if (CFG_SYNC && rnd && wr == 0 && wid == 0) {
        const unsigned long long t0 = __builtin_amdgcn_s_memrealtime();
        while (__builtin_amdgcn_readfirstlane(ld_agent((const unsigned*)(wsl + WS_COL + col * 128))) != call && !give_up((unsigned*)(wsl + WS_TMO), call, t0)) __builtin_amdgcn_s_sleep(16);
    }
    int lvl = ready(cur, 1, true);
    STAMP(P, 2);
    tables(cur);
    unsigned a2 = cur.sA, b2 = SB0 + snd + (unsigned)cur.tm * UNB;
    STAGE(SB(0, 0), b2); STAGE(SA(0, 0), a2); STAGE(SB(0, 1), b2 + HTB); STAGE(SA(0, 1), a2 + HTB);
    STAGE(SB(1, 0), b2 + KTB); STAGE(SA(1, 0), a2 + KTB); STAGE(SB(1, 1), b2 + HTB + KTB);
    WAIT_V(0); BAR;
#define ROUND_DONE() do {} while (0)
    bool has_next = false;
#pragma unroll 1
    for (;;) {
        f32x4 acc[2][2][4][2];
        int jchk = lvl < 5 ? 7 * lvl - 4 : jend;
#define SYNC_MMA(ai, first, wv) do { if (wv) WAIT_V(8); WAIT_L0(); if (wr == 1) BAR; MMA(ai, first, wr + 1); if (wr == 0) BAR; SCHED; } while (0)
#define PASS(first, check) do { \
            LDB(B0, 0, 0); LDA(At, 0, 0); LDB(B1, 0, 1); STAGE(SA(1, 1), a2 + KTB + HTB); SCHED; \
            check; \
            a2 = __builtin_amdgcn_readlane(tabA, (j + 1) & 31); b2 = __builtin_amdgcn_readlane(tabB, (j + 1) & 31); \
            SYNC_MMA(0, first, !first); \
            LDA(At, 0, 1); STAGE(SB(0, 0), b2); STAGE(SA(0, 0), a2); STAGE(SB(0, 1), b2 + HTB); \
            SYNC_MMA(1, first, !first); \
            LDB(B0, 1, 0); LDA(At, 1, 0); LDB(B1, 1, 1); STAGE(SA(0, 1), a2 + HTB); \
            SYNC_MMA(0, false, true); \
            LDA(At, 1, 1); STAGE(SB(1, 0), b2 + KTB); STAGE(SA(1, 0), a2 + KTB); STAGE(SB(1, 1), b2 + KTB + HTB); \
            SYNC_MMA(1, false, true); } while (0)
        int j = 0;
        PASS(true, );
#pragma unroll 1
        for (j = 1; j < KT / 2; ++j)
            PASS(false, if (__builtin_expect(j == jchk, 0)) {
                    if (j < JP) { lvl = ready(cur, lvl + 1); jchk = lvl < 5 ? 7 * lvl - 4 : jend; }
                    else if (wr == 0 && j == JP) { pick(); jchk = JL; }
                    else {
                        has_next = picked(lvl);
                        if (has_next) {
                            const int want = nxt.half < 0 ? 1 : 5;
                            if (lvl < want) lvl = ready(nxt, want);
                            tables(nxt);
                        }
                    }
                });
        if (has_next && nxt.half >= 0) STAGE(SA(1, 1), a2 + KTB + HTB);
        {
            const int ln = lane_now(), fr = ln & 15, fq = ln >> 4;
            const rsrc_t rol = mk_rsrc((const void*)outp, OUTB);
            const bool shared = cur.v < NS;
            if (__builtin_amdgcn_readfirstlane(ctl[0]) != 0u) {
#pragma unroll
                for (int i = 0; i < 32; ++i) acc[i >> 4][(i >> 3) & 1][(i >> 1) & 3][i & 1] = __builtin_bit_cast(f32x4, (u32x4){0x7fc00000u, 0x7fc00000u, 0x7fc00000u, 0x7fc00000u});
            }
            if (rnd == 0 && cur.v < GW) {
                const unsigned long long t0 = __builtin_amdgcn_s_memrealtime();
                while (__builtin_amdgcn_s_memrealtime() - t0 < (unsigned long long)((col & 3) * CFG_STAG + (col >> 2) * CFG_STAGB)) __builtin_amdgcn_s_sleep(8);
                if (CFG_SYNC && wr == 0 && slot == 0 && wid == 0) __builtin_amdgcn_raw_buffer_store_b32(call, rws, ln == 0 ? 0u : 0xfffffff0u, WS_COL + (unsigned)col * 128u, AUX_AGENT);
            }
            if (CFG_TCH && (unsigned)(rnd - CFG_TCH) < 2u && slot * 8 + col < SCOL / 8) {
                unsigned lines = voff; asm volatile("" : "+v"(lines)); lines <<= 2;
                __builtin_amdgcn_raw_ptr_buffer_load_lds(rws, ctl + 128, 4, lines, WS_OSN + (unsigned)((rnd - CFG_TCH) * (SCOL / 8) + slot * 8 + col) * 32768u, 0, 0);
            }
            const bool thru = rnd >= RS;
            const unsigned none = HALF && wr == 1 && cur.v == NS - 1 ? 0xf0000000u : 0u;
            const unsigned vo = (unsigned)(wc * 32 + fq * 4) * (NN * 2) + (unsigned)(wr * 256 + fr * 16) + none;
            const unsigned so = (unsigned)(cur.tm * 256) * (NN * 2) + (unsigned)col_of(cur.v, rank) * 2u;
            const unsigned vs = (unsigned)(wc * 32 + fq * 4) * ROWB + (unsigned)(wr * 256 + fr * 16) + none;
            const unsigned ss = WS_OSN + (unsigned)(cur.tm * 256) * ROWB + (unsigned)cur.v * 512u;
            auto finish = [&](auto mode) {
#pragma unroll
                for (int bj = 0; bj < 2; ++bj)
#pragma unroll
                    for (int n = 0; n < 2; ++n)
#pragma unroll
                        for (int r = 0; r < 4; ++r) {
                            u32x4 o;
                            if ((r & 1) == 0) {
#pragma unroll
                                for (int i = 0; i < 8; ++i) { f32x4& a = acc[i >> 2][bj][i & 3][n]; const f32x2 z = silu2((f32x2){a[r], a[r + 1]}); a[r] = z[0]; a[r + 1] = z[1]; }
                            }
#pragma unroll
                            for (int i = 0; i < 4; ++i) o[i] = cvt_pk_bf16(acc[i >> 1][bj][(i & 1) * 2][n][r], acc[i >> 1][bj][(i & 1) * 2 + 1][n][r]);
                            if (mode == 1) STORE16(o, rws, vs, ss + (unsigned)(bj * 128 + n * 16 + r) * ROWB, AUX_SYS);
                            STORE16(o, rol, vo, so + (unsigned)(bj * 128 + n * 16 + r) * (NN * 2), mode == 2 ? CFG_OTH : CFG_OST);
                        }
            };
            if (shared) finish(std::integral_constant<int, 1>());
            else if (thru) finish(std::integral_constant<int, 2>());
            else finish(std::integral_constant<int, 0>());
            WAIT_V(16);
            if (shared) {
                WAIT_VM0();
                unsigned old = 0;
                if (ln == 0) old = __hip_atomic_fetch_add(ctl + 1, 1u, __ATOMIC_RELAXED, __HIP_MEMORY_SCOPE_WORKGROUP);
                old = __builtin_amdgcn_readfirstlane(old);
                if ((old & 7u) == 7u) {
                    const unsigned n = (unsigned)__builtin_amdgcn_raw_ptr_buffer_atomic_add_i32(1, rws, ln == 0 ? 0u : 0xfffffff0u, WS_BAND + (unsigned)cur.tm * 128u, 0);
                    if ((unsigned)__builtin_amdgcn_readfirstlane(n) + 1u == call * NS && ln < NRANK && ln != rank)
                        st_sys((unsigned*)(P.ws[ln] + WS_FLAG + rank * FLAGB) + NTURN + turn_of(cur.tm, ln), call);
                }
            }
        }
        ROUND_DONE();
        if (!has_next) break;
        cur = nxt; ++rnd;
        if (cur.half >= 0) break;
    }
#define HB(b, i) ((b) < 2 ? ((i) < 2 ? SA(b, i) : SB(b, 0)) : (i) < 2 ? SB(i, 1) : 8 * HTB)
#define HSTAGE(b, t) do { const unsigned _o = ((t) & 1) * KTB, _a = __builtin_amdgcn_readlane(tabA, ((t) >> 1) & 31) + _o, _b = __builtin_amdgcn_readlane(tabB, ((t) >> 1) & 31) + _o; \
        STAGE(HB(b, 0), _a); STAGE(HB(b, 1), _a + HTB); STAGE(HB(b, 2), _b); } while (0)
#define HLOAD(b) do { _Pragma("unroll") for (int i = 0; i < 4; ++i) B0[i >> 1][i & 1] = *(const LAS bf16x8*)(lds + HB(b, 2) + boff + i * 1024); \
        _Pragma("unroll") for (int i = 0; i < 8; ++i) At[i >> 1][i & 1] = *(const LAS bf16x8*)(lds + HB(b, 0) + aoff + i * 1024); \
        _Pragma("unroll") for (int i = 0; i < 8; ++i) Au[i >> 1][i & 1] = *(const LAS bf16x8*)(lds + HB(b, 1) + aoff + i * 1024); } while (0)
#define HPASS(b, t, first) do { \
        if (wr == 0) { HLOAD(b); HSTAGE((b + 2) % 3, (t) + 2); } else { HSTAGE((b + 2) % 3, (t) + 2); SCHED; HLOAD(b); if (!first) WAIT_V(6); } \
        WAIT_L0(); if (wr == 1) BAR; \
        __builtin_amdgcn_s_setprio(wr + 1); \
        _Pragma("unroll") for (int i = 0; i < 32; ++i) { const int ai = i >> 4, m = (i >> 2) & 3, n = (i >> 1) & 1, k = i & 1; \
            acc[ai][m][n] = __builtin_amdgcn_mfma_f32_16x16x32_bf16(B0[n][k], ai ? Au[m][k] : At[m][k], first && k == 0 ? (f32x4){0.f, 0.f, 0.f, 0.f} : acc[ai][m][n], 0, 0, 0); } \
        __builtin_amdgcn_s_setprio(0); \
        if (wr == 0) { if (!first) WAIT_V(6); BAR; } SCHED; } while (0)
#pragma unroll 1
    while (PFULL < PN && has_next) {
        f32x4 acc[2][4][2];
        bf16x8 Au[4][2];
        BAR;
        HPASS(0, 0, true);
#pragma unroll 1
        for (int t = 1; t < KT; t += 3) {
            if (wr == 0 && wid == 0 && t == KT - 9) pick();
            HPASS(1, t, false); HPASS(2, t + 1, false); HPASS(0, t + 2, false);
        }
        has_next = picked(lvl);
        if (has_next) {
            (void)ready(nxt, 5);
            tables(nxt);
            HSTAGE(0, 0); HSTAGE(1, 1);
        }
        const int ln = lane_now(), fr = ln & 15, fq = ln >> 4;
        const rsrc_t rol = mk_rsrc((const void*)outp, OUTB);
        if (__builtin_amdgcn_readfirstlane(ctl[0]) != 0u) {
#pragma unroll
            for (int i = 0; i < 16; ++i) acc[i >> 3][(i >> 1) & 3][i & 1] = __builtin_bit_cast(f32x4, (u32x4){0x7fc00000u, 0x7fc00000u, 0x7fc00000u, 0x7fc00000u});
        }
        const unsigned vo = (unsigned)(wc * 32 + fq * 4) * (NN * 2) + (unsigned)(wr * 256 + fr * 16);
        const unsigned so = (unsigned)(cur.tm * 256 + cur.half * 128) * (NN * 2) + (unsigned)col_of(cur.v, rank) * 2u;
#pragma unroll
        for (int n = 0; n < 2; ++n)
#pragma unroll
            for (int r = 0; r < 4; ++r) {
                u32x4 o;
                if ((r & 1) == 0) {
#pragma unroll
                    for (int i = 0; i < 8; ++i) { f32x4& a = acc[i >> 2][i & 3][n]; const f32x2 z = silu2((f32x2){a[r], a[r + 1]}); a[r] = z[0]; a[r + 1] = z[1]; }
                }
#pragma unroll
                for (int i = 0; i < 4; ++i) o[i] = cvt_pk_bf16(acc[i >> 1][(i & 1) * 2][n][r], acc[i >> 1][(i & 1) * 2 + 1][n][r]);
                STORE16(o, rol, vo, so + (unsigned)(n * 16 + r) * (NN * 2), CFG_OTH);
            }
        WAIT_V(8);
        ROUND_DONE();
        cur = nxt; ++rnd;
    }
    WAIT_V(0);
}

__global__ __launch_bounds__(512, 2) void fused_gemm_ar(Params P) {
    extern __shared__ __attribute__((aligned(16))) unsigned char shm[];
    LAS unsigned char* lds = (LAS unsigned char*)shm;
    const int wid = __builtin_amdgcn_readfirstlane(threadIdx.x >> 6), lane = threadIdx.x & 63;
    const int b = __builtin_amdgcn_readfirstlane(blockIdx.x);
    const unsigned long long wsp = (unsigned long long)P.ws[lane & (NRANK - 1)];
    const unsigned t0 = (unsigned)__builtin_amdgcn_s_memrealtime();
    int rk = P.rank;
    asm volatile("" : "+s"(rk) : "s"(P.x), "s"(P.w), "s"(P.wsl), "s"(P.rank), "s"(t0) : "memory");
    unsigned* tkp = (unsigned*)(P.wsl + WS_TICKET + b * 128 + wid * 16);
    const unsigned tk = ld_agent(tkp);
    if (b < NPULL) {
        const unsigned call = __builtin_amdgcn_readfirstlane(tk) + 1u;
        st_agent(tkp, call);
        const int pi = b / PW, slot = b % PW;
        const int peer = (P.rank + 1 + pi) & (NRANK - 1);
        pull_role(P, call, peer, slot * 8 + wid, lane);
    } else {
        const int g = b - NPULL;
        if (g < 4 * NRANK * NL && wid < 2) lead_role(P, rk, wsp, tk, t0, g, wid, lane);
        ((LAS unsigned*)(lds + STAGE_BYTES))[lane & 3] = 0u;
        pack_role(P, wsp, tk, t0, g, wid, lane);
        const unsigned call = __builtin_amdgcn_readfirstlane(tk) + 1u;
        st_agent(tkp, call);
        STAMP(P, 1);
        asm volatile("s_waitcnt lgkmcnt(0)" ::: "memory");
        __builtin_amdgcn_s_barrier();
        if (b >= GRID - NG) {
            const int col = (CFG_SWAP >> (b & 7) * 4) & 7;
            const int slot = b < NPS ? KX : (b - NPS) >> 3, kx = KX + ((b & 7) >= 8 - (NG - 8 * KX));
            if (wid < 4) gemm_role<0>(P, call, col, slot, kx, lds);
            else gemm_role<1>(P, call, col, slot, kx, lds);
        }
    }
}
}

extern "C" void kernel_launch(int rank, int nranks, ncclComm_t comm,
                              hipStream_t stream,
                              void* const* d_in, const int* in_sizes, int n_in,
                              void* d_out, int out_size,
                              const int* in_dtypes, int out_dtype,
                              void* const* d_out_all,
                              void* const* d_ws_all, size_t ws_bytes) {
    if (nranks != NRANK || (unsigned)rank >= (unsigned)NRANK || ws_bytes < (size_t)WS_END || n_in < 2 ||
        in_sizes[0] != MM * KSH || in_sizes[1] != KSH * NN || out_size != MM * NN) {
        fprintf(stderr, "unexpected problem geometry\n");
        return;
    }
    static bool attr_done[NRANK] = {};
    if (!attr_done[rank]) {
        HIP_CHECK(hipFuncSetAttribute((const void*)fused_gemm_ar, hipFuncAttributeMaxDynamicSharedMemorySize, LDS_BYTES));
        attr_done[rank] = true;
    }
    Params P = {};
    P.order = ORDER;
    P.x = d_in[0]; P.w = d_in[1]; P.rank = rank; P.wsl = (char*)d_ws_all[rank]; P.outl = (char*)d_out_all[rank];
    for (int p = 0; p < NRANK; ++p) { P.out[p] = (char*)d_out_all[p]; P.ws[p] = (char*)d_ws_all[p]; }
    hipLaunchKernelGGL(fused_gemm_ar, dim3(GRID), dim3(512), LDS_BYTES, stream, P);
    HIP_CHECK(hipGetLastError());
}
```

```cpp
#include <hip/hip_runtime.h>
#include <rccl/rccl.h>
#include <cstdio>
#include <type_traits>

#define HIP_CHECK(expr) do { hipError_t _e = (expr); if (_e != hipSuccess) { \
    fprintf(stderr, "HIP error %s at %s:%d\n", hipGetErrorString(_e), __FILE__, __LINE__); \
    return; } } while (0)

#define LAS __attribute__((address_space(3)))
typedef short bf16x8 __attribute__((ext_vector_type(8)));
typedef float f32x4 __attribute__((ext_vector_type(4)));
typedef unsigned u32x4 __attribute__((ext_vector_type(4)));
typedef __amdgpu_buffer_rsrc_t rsrc_t;

namespace {
constexpr int NRANK = 8;
constexpr int MM = 4096, KK = 4096, NN = 32768, KSH = KK / NRANK;
constexpr int KT = KK / 64, KTS = KSH / 64, NTM = MM / 256, NPAN = NN / 256;
#define CFG_NS 5
#define CFG_PW 5
constexpr int NS = CFG_NS;
#define CFG_HALF 0
constexpr int HALF = CFG_HALF;
constexpr int SCOL = NS * 256 - HALF * 128;
constexpr int NRP = NPAN - NRANK * SCOL / 256;
constexpr int NWP = NS + NRP;
constexpr int NIN = NTM + NWP;
constexpr int NTURN = NIN * 4 - 2 * HALF;
constexpr int NFLAG = NTURN + NTM;
constexpr int NUNIT = (NTM + NRANK * NS + NRP) * 4;
constexpr int GRID = 256;
constexpr int PW = CFG_PW;
#define CFG_NPULL 0
#define CFG_NW (PW * 8)
constexpr int NPULL = CFG_NPULL ? CFG_NPULL : (CFG_NW * (NRANK - 1) + 7) / 8;
constexpr int NW = CFG_NPULL ? NPULL * 8 / (NRANK - 1) : CFG_NW;
constexpr int NPK = GRID - NPULL;
constexpr int NPS = (NPULL + 7) / 8 * 8;
constexpr int KX = (GRID - NPS) / 8;
#define CFG_GW (2 * (KX / 4))
constexpr int GW = CFG_GW;
constexpr int G0 = GW + NTM;
#define CFG_SKIP 1
#define CFG_SYNC 1
#define CFG_GM (GW + 2)
constexpr int GM = CFG_GM;
constexpr int NGF = (NWP - GW) / GM, GL = (NWP - GW) % GM;
static_assert(NS <= GW && NWP > GW, "shared panels sit in the first group");
constexpr int PU = 4, PSH = 2;
#define CFG_PUSH 2
constexpr int NPUSH = CFG_PUSH;
constexpr int NPIN = (NTURN - NPUSH) * 64 / PU;
#define CFG_NL (NPUSH + NW * PU / 64 < NS - HALF ? NPUSH + NW * PU / 64 : NS - HALF)
constexpr int NL = CFG_NL;
static_assert(NPUSH <= NL && NL <= NS - HALF, "leading units have a single consumer");
constexpr int BPB = SCOL / 2 / PU;
constexpr int NPC = NPIN + NTM * BPB;
constexpr unsigned CNT_FULL = 64;

#define CFG_SWAP 0x57461302u
#define CFG_XTRA 1
constexpr int NG = CFG_XTRA ? NPK : 8 * KX;
constexpr int DK = CFG_SKIP ? KX - 2 * GW : 0, R0 = KX - DK;
constexpr int nt_of(int col) { return 4 * ((NWP + 1 - (col >> 2)) / 2); }
constexpr int kx_of(int col) { int n = 0; for (int b = GRID - NG; b < GRID; ++b) n += (int)(CFG_SWAP >> (b & 7) * 4 & 7) == col; return n; }
#define CFG_RS ((nt_of(7) - 1 - R0) / (KX + (NG > 8 * KX)) + 1)
constexpr int RS = CFG_RS;
constexpr int left_of(int col) { int n = 0; for (int c = 0; c < col; ++c) n += nt_of(c) - R0 - (RS - 1) * kx_of(c); return n; }
constexpr int PN = left_of(8);
#define CFG_FD 0
constexpr int PFULL = 2 * PN - NG - CFG_FD < 0 ? 0 : 2 * PN - NG - CFG_FD > PN ? PN : 2 * PN - NG - CFG_FD;
constexpr int NITEM = PFULL + 2 * (PN - PFULL);
constexpr unsigned KTB = 32768, HTB = 16384;
constexpr unsigned PANB = KT * KTB;
constexpr unsigned CHB = KTS * KTB;
constexpr unsigned UNB = 2 * KTB;
constexpr unsigned WS_TICKET = 0;
constexpr unsigned WS_TMO = WS_TICKET + GRID * 128;
constexpr unsigned WS_FLAG = WS_TMO + 128;
constexpr unsigned FLAGB = 4096;
constexpr unsigned WS_LEAD = WS_FLAG + NRANK * FLAGB;
constexpr unsigned WS_HEAD = WS_LEAD + NRANK * 512;
constexpr unsigned WS_COL = WS_HEAD + NRANK * 128;
#define CFG_PHOFF 0
constexpr unsigned WS_PH = WS_COL + 8 * 128 + CFG_PHOFF;
constexpr unsigned WS_ZC = WS_PH + 2 * NRANK * 128 + 128;
constexpr unsigned WS_GRP = WS_ZC + NRANK * NRANK * 128;
constexpr unsigned WS_BAND = WS_GRP + 64 * NRANK * 4;
constexpr unsigned WS_CNT = WS_BAND + NTM * 128;
constexpr unsigned WS_CTRL_END = WS_CNT + NIN * NRANK * 128;
constexpr unsigned WS_DBG = 512u << 10;
constexpr unsigned WS_SND = 1u << 20, SNDB = NUNIT * UNB;
constexpr unsigned ROWB = SCOL * 2;
constexpr unsigned WS_OSN = WS_SND + NUNIT * UNB + SNDB;
constexpr unsigned WS_PAN = (WS_OSN + MM * ROWB + PANB - 1) / PANB * PANB;
constexpr unsigned WS_END = WS_PAN + NIN * PANB;
constexpr int U0 = G0 + (NRANK - 1) * NS;
constexpr unsigned OUTB = (unsigned)MM * NN * 2;
static_assert(WS_CTRL_END <= WS_DBG, "ctrl region");
static_assert(NFLAG <= 512 && 2048 <= (int)FLAGB, "flag region");
#define CFG_WIN 88
constexpr int BASE = (CFG_WIN - 1) / NW, DEEP = CFG_WIN - BASE * NW;
static_assert(BASE == 1 || BASE == 2, "window");
constexpr unsigned STAGE_BYTES = 9 * HTB;
constexpr unsigned LDS_BYTES = STAGE_BYTES + 768;
constexpr unsigned long long DEADLINE = 50000000ull;
constexpr int AUX_SYS = 17, AUX_SYS_NT = 19, AUX_AGENT = 16;
#define CFG_PACE 140
#define CFG_UH (2 * (NPK - 8 * DK))
#define CFG_TH 5000
#define CFG_PH 40
#define CFG_OST 2
#define CFG_OTH AUX_SYS_NT
#define CFG_PST AUX_AGENT
#define CFG_LDA 0
#define CFG_RLD AUX_SYS
#define CFG_TCH 3
#define CFG_GS 16
#define CFG_AGG 24
static_assert(CFG_AGG % 8 == 0 && CFG_AGG >= NS && NS <= 8 && NIN * 4 <= 512, "groups");
#define CFG_PWND 3
#define CFG_EH 1
#define CFG_MIX 1
static_assert(4 * NRANK * NL <= NPK && (NL - NPUSH) * 64 / PU <= NW && NL * 64 <= 512, "leading units");
#define CFG_PACE0 40
#define CFG_STAG 0
#define CFG_P3 320
#define CFG_P3N 2
#define CFG_STAGB 0
#define CFG_LOOK 104
#define CFG_TAIL NW
#define CFG_EVEN 280
#define CFG_EVEND 200
#define CFG_ERATE CFG_RATE
#define CFG_ZB 120
#define CFG_LOOKP 100
#define CFG_PLAN (CFG_ZB + CFG_LOOKP + 50)
#define CFG_BAL 8
constexpr int ZB = CFG_ZB;
static_assert(ZB <= 127 && ZB <= BPB && NFLAG <= 504, "plan");
#define CFG_PWIN 4u
#define CFG_PS 2
#define CFG_RATE 1680
#define CFG_CRED 25
#define CFG_GRID 36
#define CFG_EPS 0
#define CFG_SPIN 8
constexpr unsigned PACE16 = CFG_RATE * CFG_WIN / 64;
constexpr int KF = 3 - BASE, KA = (2 * BASE + 1) & 3, KE = 2 * BASE - 1;
constexpr int ops_since(int k, int back, int early) {
    int n = (k == 0) + 1 + early;
    for (int i = 1; i <= back; ++i) { const int c = (k - i + 1) & 3, b = (k - i) & 3; n += 1 + 2 * (c == KF) + (b == KA) + early * (b == KE) + (b == 0) + 1 + early; }
    return n;
}

#define CFG_EA 8
#define CFG_EB 4
#define CFG_ER 80
#define CFG_ES 17
#define CFG_EMIN 16
constexpr int NTAB = 4 * (G0 + GM);
static_assert(NTAB <= 256, "byte tables");
struct Order {
    unsigned char from[256], to[256];
    static constexpr int due(int o) {
        const int d = o >= 4 * G0 ? CFG_ER + (o - 4 * G0) / GM * CFG_ES + ((o - 4 * G0) % GM & 1) * CFG_EB
                                  : o / G0 * CFG_ES + (o % G0 < GW ? o % G0 / (GW / 2) * CFG_EB : (o % G0 - GW) / 4 * CFG_EA);
        return HALF && o < 4 * G0 && o % G0 == NS - 1 && (o / G0 & 1) ? 1 << 20 : d < CFG_EMIN ? CFG_EMIN : d;
    }
    constexpr Order() : from(), to() {
        for (int o = 0; o < NTAB; ++o) {
            int n = 0;
            for (int k = 0; k < NTAB; ++k) n += due(k) < due(o) || (due(k) == due(o) && k < o);
            to[o] = (unsigned char)n; from[n] = (unsigned char)o;
        }
    }
};
constexpr Order ORDER;
static_assert(ORDER.to[NS - 1] == NS - 1, "the leading units come first");

struct Params {
    const void* x; const void* w;
    char* wsl; char* outl;
    int rank, pad[7];
    char* ws[NRANK]; char* out[NRANK];
    Order order;
};
__device__ __forceinline__ int look_up(unsigned tab, int o, int n, int beyond) { return o < n ? (unsigned)__builtin_amdgcn_readlane(tab, o >> 2) >> (o & 3) * 8 & 255u : o + beyond; }

__device__ __forceinline__ rsrc_t mk_rsrc(const void* p, unsigned bytes) {
    const unsigned long long v = (unsigned long long)p;
    const unsigned lo = __builtin_amdgcn_readfirstlane((unsigned)v), hi = __builtin_amdgcn_readfirstlane((unsigned)(v >> 32));
    return __builtin_amdgcn_make_buffer_rsrc((void*)(((unsigned long long)hi << 32) | lo), 0, bytes, 0x00020000);
}
__device__ __forceinline__ unsigned ld_sys(const unsigned* p) { return __hip_atomic_load(p, __ATOMIC_RELAXED, __HIP_MEMORY_SCOPE_SYSTEM); }
__device__ __forceinline__ unsigned ld_agent(const unsigned* p) { return __hip_atomic_load(p, __ATOMIC_RELAXED, __HIP_MEMORY_SCOPE_AGENT); }
__device__ __forceinline__ void st_sys(unsigned* p, unsigned v) { __hip_atomic_store(p, v, __ATOMIC_RELAXED, __HIP_MEMORY_SCOPE_SYSTEM); }
__device__ __forceinline__ void st_agent(unsigned* p, unsigned v) { __hip_atomic_store(p, v, __ATOMIC_RELAXED, __HIP_MEMORY_SCOPE_AGENT); }
__device__ __forceinline__ void add_agent(unsigned* p, unsigned v) { (void)__hip_atomic_fetch_add(p, v, __ATOMIC_RELAXED, __HIP_MEMORY_SCOPE_AGENT); }
#define STORE16(v, rs, vo, so, aux) do { const u32x4 _d = (v); __builtin_amdgcn_raw_buffer_store_b128(_d, rs, vo, so, aux); asm volatile("s_nop 1" :: "v"(_d) : "memory"); } while (0)
#define WAIT_VM0() asm volatile("s_waitcnt vmcnt(0)" ::: "memory")
#define WAIT_VMN(n) do { asm volatile("" ::: "memory"); __builtin_amdgcn_s_waitcnt(0x0F70 | (n)); asm volatile("" ::: "memory"); } while (0)

__device__ __forceinline__ bool give_up(unsigned* tmo, unsigned call, unsigned long long t0) {
    const unsigned long long dt = __builtin_amdgcn_s_memrealtime() - t0;
    if (dt < 2000ull) return false;
    if ((unsigned)__builtin_amdgcn_readfirstlane(ld_agent(tmo)) == call) return true;
    if (dt > DEADLINE) { st_agent(tmo, call); return true; }
    return false;
}

__device__ __forceinline__ int gpan_of(int v, int consumer) { return v < NS ? consumer * NS + v : (NRANK - 1) * NS + v; }
__device__ __forceinline__ int col_of(int v, int consumer) { return (v < NS ? consumer * SCOL : NRANK * SCOL - NS * 256) + v * 256; }
__device__ __forceinline__ int band_of(int e, int consumer) { return e < 7 ? (consumer + 1 + e) & 7 : e < 15 ? 8 + ((consumer + e) & 7) : consumer; }
__device__ __forceinline__ int turn_of(int band, int consumer) { return band >= 8 ? 7 + ((band - consumer + 1) & 7) : band == consumer ? 15 : (band - consumer - 1) & 7; }
__device__ __forceinline__ int unit_of(int o, int consumer, int& q, int& pos) {
    if (o < 4 * G0) { q = o / G0; const int m = o % G0; pos = q * U0 + gpan_of(m, consumer); return m < GW ? NTM + m : m - GW; }
    pos = o + 4 * (NRANK - 1) * NS;
    o -= 4 * G0;
    const int g = o / (4 * GM), r = o % (4 * GM);
    int i;
    if (GL == 0 || g < NGF) { q = r / GM; i = r % GM; } else { q = r / (GL ? GL : 1); i = r % (GL ? GL : 1); }
    return NTM + GW + GM * g + i;
}

__device__ __forceinline__ unsigned cvt_pk_bf16(float lo, float hi) { unsigned r; asm("v_cvt_pk_bf16_f32 %0, %1, %2" : "=v"(r) : "v"(lo), "v"(hi)); return r; }
__device__ __forceinline__ float silu(float y) {
    const float c = __builtin_amdgcn_fmed3f(y, -60.0f, 60.0f);
    const float e = __builtin_amdgcn_exp2f(c * -1.44269504088896f);
    return y * __builtin_amdgcn_rcpf(1.0f + e);
}

typedef float f32x2 __attribute__((ext_vector_type(2)));
__device__ __forceinline__ f32x2 silu2(f32x2 y) {
    const f32x2 t = __builtin_elementwise_fma(y, (f32x2){-1.44269504088896f, -1.44269504088896f}, (f32x2){-86.5617024533378f, -86.5617024533378f});
    const f32x2 e = {__builtin_amdgcn_fmed3f(__builtin_amdgcn_exp2f(t[0]), 0.0f, 1.0f), __builtin_amdgcn_fmed3f(__builtin_amdgcn_exp2f(t[1]), 0.0f, 1.0f)};
    const f32x2 d = __builtin_elementwise_fma(e, (f32x2){1.1420073898156842e26f, 1.1420073898156842e26f}, (f32x2){1.0f, 1.0f});
    return y * (f32x2){__builtin_amdgcn_rcpf(d[0]), __builtin_amdgcn_rcpf(d[1])};
}

#define STAMP1(P, slot) do {} while (0)
#define STAMP(P, slot) do {} while (0)

__device__ __forceinline__ void pull_role(const Params& P, unsigned call, int peer, int widx, int lane) {
    const int rank = P.rank;
    char* wsl = P.wsl;
    const rsrc_t rwr = mk_rsrc(P.ws[peer], WS_END), rwl = mk_rsrc(wsl, WS_END);
    const rsrc_t rol = mk_rsrc(P.outl, OUTB), ror = mk_rsrc(P.out[peer], OUTB);
    unsigned* tmo = (unsigned*)(wsl + WS_TMO);
    constexpr unsigned OOB = 0xfffffff0u;
    const unsigned lane0 = lane == 0 ? 0u : OOB;
    const unsigned hoff = WS_PH + (call & 1u) * (NRANK * 128u);
    auto draw = [&]() { return (unsigned)__builtin_amdgcn_raw_ptr_buffer_atomic_add_i32(1, rwl, lane0, hoff + (unsigned)peer * 128u, 0); };
    if (widx == 0) __builtin_amdgcn_raw_buffer_store_b32(0u, rwl, lane0, WS_PH + (~call & 1u) * (NRANK * 128u) + (unsigned)peer * 128u, AUX_SYS);
    auto arrive = [&](unsigned vo, unsigned cnt) { (void)__builtin_amdgcn_raw_ptr_buffer_atomic_add_i32(PU, rwl, vo, cnt, 0); };
    const int wrk = CFG_NPULL ? (widx + ((peer - rank) & 7) * (NW / 7)) % NW : NW % 8 ? widx * 5 % NW : (widx & 7) * (NW / 8) + (widx >> 3);
    const bool deep0 = wrk < DEEP;
    const unsigned per0 = deep0 ? PACE16 / (BASE + 1) : PACE16 / BASE;
    const unsigned from = ((const unsigned*)P.order.from)[lane];
    bool bad = false; int stamped = 0; (void)stamped;
    auto index_of = [&](unsigned v) { return (int)__builtin_amdgcn_readfirstlane(v) + NW; };
    int cur = index_of((unsigned)(widx - NW));
    const unsigned fso = WS_FLAG + (unsigned)peer * FLAGB;
    auto first_of = [&](int i) { return i < NPIN ? (i >> (6 - PSH)) + NPUSH : NTURN + (i - NPIN) / BPB; };
    int lim = 0, flim = 0, fin = NPC;
    bool planned = ZB == 0;
    unsigned long long plan = 0;
    auto moved = [&](int j) { return (int)(signed char)(plan >> (j * 8)); };
    auto update = [&](const u32x4& f0, const u32x4& f1, int i) {
        const int first = first_of(i);
        unsigned un = 0;
#pragma unroll
        for (int j = 0; j < 8; ++j) un |= ((int)((j < 4 ? f0[j & 3] : f1[j & 3]) - call) < 0 && lane * 8 + j >= first ? 1u : 0u) << j;
        const int l = __builtin_ctzll(__ballot(un != 0u) | (1ull << 63));
        const int u = l * 8 + __builtin_ctz(__builtin_amdgcn_readlane(un, l) | 256u);
        const int fl = u <= NTURN ? (u - NPUSH) << (6 - PSH) : u >= NFLAG ? NPC + 128 : NPIN + (u - NTURN) * BPB;
        flim = fl > flim ? fl : flim;
        if (!planned && (unsigned)__builtin_amdgcn_readlane(f0[0], 63) == call && (unsigned)__builtin_amdgcn_readlane(f0[2], 63) == call) {
            plan = (unsigned long long)(unsigned)__builtin_amdgcn_readlane(f0[3], 63) << 32 | (unsigned)__builtin_amdgcn_readlane(f0[1], 63);
            planned = true; fin = NPC - moved(peer);
        }
        const int pl = planned ? fin : NPC - ZB;
        lim = flim < pl ? flim : pl;
    };
    auto make_plan = [&]() {
        unsigned old = 0;
        if (lane == 0) old = __hip_atomic_exchange((unsigned*)(wsl + WS_PH + 2 * NRANK * 128), call, __ATOMIC_RELAXED, __HIP_MEMORY_SCOPE_AGENT);
        if ((unsigned)__builtin_amdgcn_readfirstlane(old) == call) return;
        const unsigned hd = __builtin_amdgcn_raw_buffer_load_b32(rwl, lane < NRANK ? (unsigned)lane * 128u : OOB, hoff, AUX_SYS);
        const int t = lane >> 3, s = lane & 7;
        const unsigned zc = ld_sys((const unsigned*)(P.ws[t] + WS_ZC + (rank * NRANK + s) * 128));
        const bool ok = __all(t == rank || s == rank || s == t || (int)(zc - call * (unsigned)(ZB * PU)) >= 0);
        int h[NRANK], sum = 0, give = 0, take = 0;
#pragma unroll
        for (int j = 0; j < NRANK; ++j) { h[j] = __builtin_amdgcn_readlane(hd, j); sum += j == rank ? 0 : h[j]; }
#pragma unroll
        for (int j = 0; j < NRANK; ++j) {
            const int d = j == rank ? 0 : (sum - (NRANK - 1) * h[j]) * CFG_BAL / (8 * (NRANK - 1));
            h[j] = d > ZB ? ZB : d < -127 ? -127 : d;
            if (h[j] > 0) give += h[j]; else take -= h[j];
        }
        give = take = !ok ? 0 : give < take ? give : take;
        unsigned long long pl = 0;
#pragma unroll
        for (int j = 0; j < NRANK; ++j) {
            int d = h[j];
            if (d > 0) { d = d < give ? d : give; give -= d; } else { d = -d < take ? d : -take; take += d; }
            pl |= (unsigned long long)(d & 255) << (j * 8);
        }
        __builtin_amdgcn_raw_buffer_store_b128((u32x4){call, (unsigned)pl, call, (unsigned)(pl >> 32)}, rwl, lane < NRANK ? WS_FLAG + 63 * 32 + (unsigned)lane * FLAGB : OOB, 0, AUX_SYS);
    };
    auto offs_of = [&](int i, unsigned (&o)[PU], unsigned& cnt) -> unsigned {
        if (i < NPIN) {
            int q, pos; const int pan = unit_of(look_up(from, (i >> (6 - PSH)) + NPUSH, NTAB - 2 * HALF, 2 * HALF), rank, q, pos);
            cnt = WS_CNT + (unsigned)(pan * NRANK + peer) * 128u + (unsigned)q * 4u;
            unsigned sub = (unsigned)((i << PSH) & 63) * 1024u;
            if (HALF && pan == NTM + NS - 1) { sub += sub & ~8191u; pos += sub >> 16 ? U0 - 1 : 0; }
            const unsigned b = WS_PAN + (unsigned)pan * PANB + (unsigned)peer * CHB + (unsigned)q * UNB + sub + (unsigned)lane * 16u;
#pragma unroll
            for (int k = 0; k < PU; ++k) o[k] = b + k * 1024u;
            return WS_SND + (call & 1u) * SNDB + (unsigned)pos * UNB + sub;
        }
        int src = peer;
        if (ZB && i >= NPC) {
            int x = i - NPC;
            for (int j = 0; j < peer; ++j) x -= moved(j) < 0 ? moved(j) : 0;
            for (int j = 0; j < NRANK; ++j) { const int g = moved(j) > 0 ? moved(j) : 0; if ((unsigned)x < (unsigned)g) { src = j; i = NPC - g + x; } x -= g; }
        }
        const int e = (i - NPIN) / BPB, r = (i - NPIN) % BPB, pb = band_of(e, rank) * BPB + r;
        cnt = e < NRANK - 1 && r >= BPB - ZB ? WS_ZC + (unsigned)(band_of(e, rank) * NRANK + peer) * 128u : 0u;
#pragma unroll
        for (int k = 0; k < PU; ++k) {
            const unsigned sg = (unsigned)((pb << PSH) + k) * 4u + (unsigned)(lane >> 4);
            o[k] = (unsigned)src * ROWB + (unsigned)(lane & 15) * 16u + sg / (ROWB / 256) * (NN * 2u - ROWB) + sg * 256u;
        }
        return WS_OSN + (unsigned)(pb << PSH) * 1024u;
    };
    if (cur < (NL - NPUSH) << (6 - PSH)) {
        const unsigned long long t0 = __builtin_amdgcn_s_memrealtime();
        const unsigned fo = fso + 2048u + (unsigned)(((cur >> 3) + 2 * NPUSH) * 2 + (cur >> 1 & 1)) * 8u;
        for (;;) {
            const unsigned f = __builtin_amdgcn_raw_buffer_load_b32(rwl, (unsigned)(lane & 1) * 4u, fo, AUX_SYS);
            if (__all((int)(f - call) >= 0)) { lim = flim = cur + 1; break; }
            if (give_up(tmo, call, t0)) { bad = true; planned = true; break; }
            __builtin_amdgcn_s_sleep(CFG_PS);
        }
        asm volatile("" ::: "memory");
    }
    asm volatile("" :: "v"(from));
    while (!planned || cur < fin) {
        if (!bad && cur >= lim) {
            const unsigned long long t0 = __builtin_amdgcn_s_memrealtime();
            const unsigned pvo = (unsigned)(lane - (first_of(cur) >> 3)) < CFG_PWIN || (lane == 63 && cur >= NPC - ZB) ? (unsigned)lane * 32u : OOB - 16u;
            for (;;) {
                const u32x4 f0 = __builtin_amdgcn_raw_buffer_load_b128(rwl, pvo, fso, AUX_SYS), f1 = __builtin_amdgcn_raw_buffer_load_b128(rwl, pvo + 16u, fso, AUX_SYS);
                update(f0, f1, cur);
                if (cur < lim || (planned && cur >= fin)) break;
                if (give_up(tmo, call, t0)) { bad = true; planned = true; plan = 0; fin = NPC; break; }
                __builtin_amdgcn_s_sleep(CFG_PS);
            }
            asm volatile("" ::: "memory");
            if (cur >= fin && planned) break;
        }
        unsigned o[PU], a[PU] = {OOB, OOB, OOB, OOB}, cprev = 0, cold = 0;
        if (bad) {
            (void)offs_of(cur, o, cprev);
            if (cur >= NPIN) {
#pragma unroll
                for (int k = 0; k < PU; ++k) __builtin_amdgcn_raw_buffer_store_b128((u32x4){0x7fc07fc0u, 0x7fc07fc0u, 0x7fc07fc0u, 0x7fc07fc0u}, rol, o[k], 0, AUX_AGENT);
            }
            cur = index_of(draw()); continue;
        }
        STAMP1(P, 1);
        rsrc_t rp = rwl;
        u32x4 v[PU] = {};
        auto put = [&](int k, const rsrc_t r) { __builtin_amdgcn_raw_buffer_store_b128(v[k & 3], r, a[k & 3], 0, CFG_PST); };
        const unsigned now = (unsigned)__builtin_amdgcn_s_memrealtime() << 4;
        unsigned slot = now - (now & 0xffffffu) % PACE16 + (unsigned)wrk * (PACE16 / NW) - PACE16;
        unsigned cb, src, in[PU];
        auto prepare = [&]() {
            if (ZB && cur == NPC - CFG_PLAN) make_plan();
            src = __builtin_amdgcn_readfirstlane(offs_of(cur, o, cb)); cb = __builtin_amdgcn_readfirstlane(cb);
            if (ZB && cur >= NPC) src = 0;
#pragma unroll
            for (int k = 0; k < PU; ++k) { in[k] = ZB && cur >= NPC ? o[k] : lane * 16 + k * 1024; asm volatile("" : "+v"(o[k]), "+v"(in[k])); }
        };
        prepare();
        do {
            const bool tail = cur >= fin - CFG_TAIL, even = cur >= fin - (deep0 ? CFG_EVEND : CFG_EVEN), deep = deep0 || tail || even;
            const unsigned per = even ? CFG_ERATE * NW / 64 : per0;
            const rsrc_t rd = cur < NPIN ? rwl : rol, rs = ZB && cur >= NPC ? ror : rwr;
            const bool look = flim < NPC && cur + CFG_LOOK >= flim, lookp = !planned && cur + CFG_LOOKP >= NPC - ZB;
            const unsigned fvo = look || (lookp && lane == 63) ? (unsigned)lane * 32u : OOB - 16u;
            unsigned cdone = cprev;
            unsigned nv = 0; u32x4 f0, f1;
            auto step = [&](auto kc) {
                constexpr int k = kc;
                if (!tail) {
                    slot += per;
                    unsigned go = (unsigned)__builtin_amdgcn_s_memrealtime();
                    const int d = (int)((go << 4) - slot);
                    if (d > CFG_CRED * 16) slot += (unsigned)(d - CFG_CRED * 16);
                    if (d < 0) go += (unsigned)(15 - d) >> 4;
                    const unsigned over = (go & 0xffffffu) % CFG_GRID;
                    if (over >= CFG_EPS) go += CFG_GRID - over;
                    for (;;) {
                        const int w = (int)(go - (unsigned)__builtin_amdgcn_s_memrealtime());
                        if (w <= 0) break;
                        if (w > CFG_SPIN) __builtin_amdgcn_s_sleep(1);
                    }
                }
                if (k == KF) { f0 = __builtin_amdgcn_raw_buffer_load_b128(rwl, fvo, fso, AUX_SYS); f1 = __builtin_amdgcn_raw_buffer_load_b128(rwl, fvo + 16u, fso, AUX_SYS); asm volatile("" ::: "memory"); }
                v[k] = __builtin_amdgcn_raw_buffer_load_b128(rs, in[k], src, CFG_RLD);
                a[k] = o[k];
                if (k == 0) nv = draw();
                put(k - BASE - 1, k < BASE + 1 ? rp : rd);
                if (!deep) { put(k - BASE, k < BASE ? rp : rd); a[(k - BASE) & 3] = OOB; }
                if (k == 3) {
                    asm volatile("" ::: "memory");
                    cprev = cb; rp = rd;
                    cur = index_of(nv);
                    if (look || lookp) update(f0, f1, cur);
                    if (cur < lim) prepare();
                }
                WAIT_VMN(ops_since(k, BASE, 0));
                if (!deep) {
                    WAIT_VMN(ops_since(k, BASE - 1, 1));
                    if (k == KE) { arrive(cdone ? lane0 : OOB, cdone); cdone = 0; }
                }
                if (k == KA) { const unsigned c = BASE == 1 ? cdone : cold; arrive(c ? lane0 : OOB, c); }
            };
            step(std::integral_constant<int, 0>()); step(std::integral_constant<int, 1>()); step(std::integral_constant<int, 2>()); step(std::integral_constant<int, 3>());
            if (BASE > 1) cold = cdone;
        } while (cur < lim);
#pragma unroll
        for (int k = 3 - BASE; k < 4; ++k) put(k, rp);
        if (cprev | cold) { WAIT_VM0(); arrive(cprev ? lane0 : OOB, cprev); if (BASE > 1) arrive(cold ? lane0 : OOB, cold); }
    }
    STAMP(P, 4);
}

__device__ __forceinline__ void load_w(u32x4 (&v)[8], const rsrc_t rw, int col, int q, int job, int lane) {
    const unsigned vo = (unsigned)((lane >> 4) * 8) * (NN * 2) + (unsigned)(lane & 15) * 16u;
    const unsigned so = (unsigned)(q * 128 + (job >> 1) * 32) * (NN * 2) + (unsigned)(col + (job & 1) * 128) * 2u;
#pragma unroll
    for (int j = 0; j < 8; ++j) v[j] = __builtin_amdgcn_raw_buffer_load_b128(rw, vo, so + (unsigned)j * (NN * 2), 2);
}
__device__ __forceinline__ void store_w(const u32x4 (&v)[8], const rsrc_t rd, unsigned vo, unsigned unit, int job, bool slow) {
    unit += (unsigned)(job >> 2) * KTB + (unsigned)(job & 1) * 8192u + (unsigned)(job >> 1 & 1) * 1024u;
#pragma unroll
    for (int i = 0; i < 8; ++i) {
        u32x4 o;
#pragma unroll
        for (int e = 0; e < 4; ++e) o[e] = __builtin_amdgcn_perm(v[2 * e + 1][i >> 1], v[2 * e][i >> 1], (i & 1) ? 0x07060302u : 0x05040100u);
        STORE16(o, rd, vo, unit + (unsigned)(i >> 2) * HTB + (unsigned)(i & 3) * 2048u, AUX_SYS);
        if (CFG_PWND && i >= CFG_PWND - 1 && i < 7 && slow) WAIT_VMN(CFG_PWND ? CFG_PWND - 1 : 0);
    }
}
typedef __attribute__((address_space(1))) unsigned* gptr_t;

__device__ __forceinline__ void lead_role(const Params& P, int rank, unsigned long long wsp, unsigned tk, unsigned t0, int g, int wid, int lane) {
    const int o = NL - 1 - (g >> 5), c = (g >> 3) & 3, job = (c >> 1) * 4 + wid * 2 + (c & 1);
    const int dest = (rank + 1 + g + (o < NPUSH ? CFG_MIX * (2 * c + o) : 0)) & (NRANK - 1);
    u32x4 v[8];
    load_w(v, mk_rsrc(P.w, (unsigned)KSH * NN * 2), dest * SCOL + o * 256, 0, job, lane);
    asm volatile("" : "+v"(tk), "+v"(wsp) :: "memory");
    const bool push = o < NPUSH && dest != rank, count = o < NPUSH || dest == rank;
    char* wsd = (char*)((unsigned long long)(unsigned)__builtin_amdgcn_readlane((unsigned)(wsp >> 32), dest) << 32 | (unsigned)__builtin_amdgcn_readlane((unsigned)wsp, dest));
    gptr_t fp = (gptr_t)(wsd + (count ? WS_CNT + (unsigned)((NTM + o) * NRANK + rank) * 128u : WS_FLAG + (unsigned)rank * FLAGB + 2048u + (unsigned)((o * 4 + c) * 2 + wid) * 4u));
    while (CFG_P3 && push && o >= CFG_P3N && (int)((unsigned)__builtin_amdgcn_s_memrealtime() - t0) < CFG_P3 + (o - CFG_P3N) * 110) __builtin_amdgcn_s_sleep(1);
    store_w(v, mk_rsrc(push ? wsd : P.wsl, WS_END), (unsigned)lane * 16u + (push ? 0u : (~tk & 1u) * SNDB),
            push ? WS_PAN + (unsigned)(NTM + o) * PANB + (unsigned)rank * CHB : WS_SND + (unsigned)(dest * NS + o) * UNB, job, push);
    WAIT_VMN(0);
    STAMP(P, 6);
    if (lane == 0) {
        if (!count) __hip_atomic_store(fp, tk + 1u, __ATOMIC_RELAXED, __HIP_MEMORY_SCOPE_SYSTEM);
        else if ((__hip_atomic_fetch_add((gptr_t)(P.wsl + WS_LEAD + (o * NRANK + dest) * 64), 1u, __ATOMIC_RELAXED, __HIP_MEMORY_SCOPE_AGENT) & 7u) == 7u)
            __hip_atomic_store(fp, (tk + 1u) * CNT_FULL, __ATOMIC_RELAXED, __HIP_MEMORY_SCOPE_SYSTEM);
    }
}

__device__ __forceinline__ void pack_role(const Params& P, unsigned long long wsp, unsigned tk, unsigned t0, int g, int wid, int lane) {
    const int rank = P.rank;
    const rsrc_t rx = mk_rsrc(P.x, (unsigned)MM * KSH * 2), rw = mk_rsrc(P.w, (unsigned)KSH * NN * 2), rwl = mk_rsrc(P.wsl, WS_END);
    const unsigned sv = (unsigned)lane * 16u + (~tk & 1u) * SNDB;
    const unsigned to = ((const unsigned*)P.order.to)[lane];
    constexpr int NH = CFG_UH < NUNIT ? 8 * DK : 0, UH = NH ? CFG_UH : NUNIT;
    const bool late = g >= NPK - NH;
    constexpr int U1 = NRANK * NL, NE = CFG_EH ? NH : 0;
    static_assert(CFG_TH < (UH - U1 - NH) * CFG_PH, "unpaced");
    for (int u = late ? (NE ? U1 : UH) + g - (NPK - NH) : U1 + NE + g; u < (late ? NUNIT : UH); u += !late ? NPK - NH : u < UH ? UH - U1 : NH) {
        int o, dest = -1;
        if (u < 4 * U0) {
            const int r = u % U0;
            if (r < NRANK * NS) { o = r >> 3; dest = (rank + 1 + r) & (NRANK - 1); } else o = r - (NRANK - 1) * NS;
            o += u / U0 * G0;
        } else o = u - 4 * (NRANK - 1) * NS;
        int q, pos; const int pan = unit_of(o, dest < 0 ? 0 : dest, q, pos);
        const bool narrow = HALF && pan == NTM + NS - 1;
        if (narrow && (q & 1)) continue;
        while ((int)((unsigned)__builtin_amdgcn_s_memrealtime() - t0) < (late ? CFG_TH + (u - UH) * CFG_PH : (u - CFG_PACE0) * CFG_PACE / 10)) __builtin_amdgcn_s_sleep(1);
        const unsigned dst = WS_SND + (unsigned)pos * UNB;
        u32x4 v[8];
        if (pan < NTM) {
            const int rb0 = (wid & 3) * 4;
            const unsigned vo = (unsigned)(lane & 15) * (KSH * 2) + (unsigned)(lane >> 4) * 16u;
            const unsigned so = (unsigned)(pan * 256 + rb0 * 16) * (KSH * 2) + (unsigned)(q * 2 + (wid >> 2)) * 128u;
#pragma unroll
            for (int i = 0; i < 8; ++i) v[i] = __builtin_amdgcn_raw_buffer_load_b128(rx, vo + (i & 1) * 64, so + (unsigned)(i >> 1) * (16 * KSH * 2), 0);
#pragma unroll
            for (int i = 0; i < 8; ++i) STORE16(v[i], rwl, sv + (i & 1) * 1024, dst + (unsigned)(wid >> 2) * KTB + (unsigned)(rb0 + (i >> 1)) * 2048u, AUX_SYS);
        } else {
            const int job = narrow ? (wid & 3) * 2 : wid, q1 = narrow ? wid >> 2 : 0;
            load_w(v, rw, col_of(pan - NTM, dest < 0 ? 0 : dest), q + q1, job, lane);
            store_w(v, rwl, sv, dst + (unsigned)(q1 * U0) * UNB, job, false);
        }
        o = look_up(to, o, NTAB, -2 * HALF);
        const bool own = lane == rank;
        unsigned long long fp = wsp + (own ? WS_CNT + (unsigned)((pan * NRANK + rank) * 32 + q) * 4u : WS_FLAG + (unsigned)rank * FLAGB + (unsigned)o * 4u);
        unsigned fv = own ? (tk + 1u) * CNT_FULL : tk + 1u;
        asm volatile("" : "+v"(fp), "+v"(fv));
        WAIT_VMN(0);
        __builtin_amdgcn_s_barrier();
        const bool mine = lane < NRANK && (dest < 0 || dest == lane);
        if (wid == 0 && mine && (own || o < CFG_AGG || o >= NTURN / 8 * 8)) __hip_atomic_store((gptr_t)fp, fv, __ATOMIC_RELAXED, __HIP_MEMORY_SCOPE_SYSTEM);
        if (wid == 1 && o >= CFG_AGG && o < NTURN / 8 * 8) {
            unsigned old = 0;
            if (mine && !own) old = __hip_atomic_fetch_add((gptr_t)(P.wsl + WS_GRP) + (o >> 3) * NRANK + lane, 1u, __ATOMIC_RELAXED, __HIP_MEMORY_SCOPE_AGENT);
            const unsigned full = (unsigned)__ballot((old & 7u) == 7u);
            const unsigned long long wsq = (unsigned long long)(unsigned)__builtin_amdgcn_ds_bpermute(lane >> 3 << 2, (int)(wsp >> 32)) << 32 | (unsigned)__builtin_amdgcn_ds_bpermute(lane >> 3 << 2, (int)wsp);
            if (full >> (lane >> 3) & 1) __hip_atomic_store((gptr_t)(wsq + WS_FLAG + (unsigned)rank * FLAGB) + (o & ~7) + (lane & 7), tk + 1u, __ATOMIC_RELAXED, __HIP_MEMORY_SCOPE_SYSTEM);
        }
    }
}

struct Tile { int tm, v, half; unsigned sA, stA; };
__device__ __forceinline__ void tile_of(int idx, int col, int rank, unsigned snd, Tile& t) {
    const int v = idx < 2 * GW ? (idx >> 2) + (col >> 2) * (GW / 2) : 2 * (idx >> 2) + (col >> 2);
    t.tm = 4 * (col & 3) + (idx & 3); t.v = v; t.half = -1;
    const int g = (v - GW) / GM;
    t.sA = snd + (unsigned)(v < GW ? gpan_of(v, rank) : 4 * U0 + g * 3 * GM + v - GW) * UNB;
    t.stA = (unsigned)(v < GW ? U0 : g < NGF ? GM : GL) * UNB;
}
__device__ __forceinline__ void item_of(int q, int rank, unsigned snd, Tile& t) {
    const int n = q < PFULL ? q : PFULL + ((q - PFULL) >> 1);
    int at = 0;
    auto from = [&](auto c) { constexpr int l = left_of(c), a = (nt_of(c) - left_of(c + 1)) * 8 + c; at = n >= l ? a : at; };
#define C(i) std::integral_constant<int, i>()
    from(C(0)); from(C(1)); from(C(2)); from(C(3)); from(C(4)); from(C(5)); from(C(6)); from(C(7));
#undef C
    tile_of((at >> 3) + n, at & 7, rank, snd, t);
    if (q >= PFULL) t.half = (q - PFULL) & 1;
}

template <int wr>
__device__ __forceinline__ void gemm_role(const Params& P, unsigned call, int col, int slot, int kx, LAS unsigned char* lds) {
    const int tid = threadIdx.x, wid = __builtin_amdgcn_readfirstlane(tid >> 6), lane = tid & 63, wc = wid & 3;
    const int rank = P.rank;
    char* wsl = P.wsl;
    const rsrc_t rws = mk_rsrc(wsl, WS_END);
    LAS unsigned* ctl = (LAS unsigned*)(lds + STAGE_BYTES);
    const unsigned voff = (unsigned)tid * 16u;
    const unsigned ldsw = (unsigned)wid * 1024u;
    const int aoff = wr * 8192 + lane * 16, boff = wc * 4096 + lane * 16;

    auto lane_now = [&]() { int tz = threadIdx.x; asm volatile("" : "+v"(tz)); return tz & 63; };
    auto cnt_of = [&](const Tile& t) { const int ln = lane_now(); return WS_CNT + (unsigned)((((ln & 32) ? NTM + t.v : t.tm) * NRANK + ((ln >> 2) & 7)) * 32 + (ln & (HALF && t.v == NS - 1 && ln >= 32 ? 2 : 3))) * 4u; };
    auto level_of = [&](unsigned c) -> int {
        const unsigned long long miss = __ballot((int)(c - call * CNT_FULL) < 0);
        unsigned m = (unsigned)miss | (unsigned)(miss >> 32);
        const bool own = ((m >> (rank * 4)) & 15u) == 0u;
        m |= m >> 16; m |= m >> 8; m |= m >> 4;
        return own ? 1 + __builtin_ctz(m | 16u) : 0;
    };
    auto ready = [&](const Tile& t, int want, bool first = false) -> int {
        if (wr == 0 && wid == 0) {
            const unsigned* c = (const unsigned*)(wsl + cnt_of(t));
            unsigned* tmo = (unsigned*)(wsl + WS_TMO);
            const unsigned long long t0 = __builtin_amdgcn_s_memrealtime();
            int got;
            for (;;) {
                got = level_of(ld_sys(c));
                if (got >= want) break;
                if (give_up(tmo, call, t0)) { ctl[0] = 1u; got = 5; break; }
                __builtin_amdgcn_s_sleep(CFG_GS);
            }
            ctl[2] = (unsigned)got;
            if (first) __builtin_amdgcn_fence(__ATOMIC_ACQUIRE, "agent");
            asm volatile("s_waitcnt vmcnt(0) lgkmcnt(0)" ::: "memory");
        }
        asm volatile("" ::: "memory"); __builtin_amdgcn_s_barrier(); asm volatile("" ::: "memory");
        const int r = __builtin_amdgcn_readfirstlane(ctl[2]);
        asm volatile("s_waitcnt lgkmcnt(0)" ::: "memory");
        return r;
    };

#define SA(b, h) (((b) * 2 + (h)) * HTB)
#define SB(b, h) ((4 + (b) * 2 + (h)) * HTB)
#define STAGE(bufoff, soff) do { _Pragma("unroll") for (int _i = 0; _i < 2; ++_i) \
        __builtin_amdgcn_raw_ptr_buffer_load_lds(rws, (LAS unsigned*)(lds + (bufoff) + ldsw + _i * 8192), 16, voff, (soff) + _i * 8192u, 0, CFG_LDA); } while (0)
#define LDA(dst, b, h) do { _Pragma("unroll") for (int m = 0; m < 4; ++m) _Pragma("unroll") for (int k = 0; k < 2; ++k) dst[m][k] = *(const LAS bf16x8*)(lds + SA(b, h) + aoff + m * 2048 + k * 1024); } while (0)
#define LDB(dst, b, h) do { _Pragma("unroll") for (int n = 0; n < 2; ++n) _Pragma("unroll") for (int k = 0; k < 2; ++k) dst[n][k] = *(const LAS bf16x8*)(lds + SB(b, h) + boff + n * 2048 + k * 1024); } while (0)
#define MMA(ai, first, prio) do { __builtin_amdgcn_s_setprio(prio); \
        _Pragma("unroll") for (int i = 0; i < 32; ++i) { \
            const int bj = ((i >> 2) ^ (i >> 3)) & 1, m = i >> 3, n = ((i >> 1) ^ (i >> 3)) & 1, k = (i ^ (i >> 1)) & 1;     \
            SCHED; \
            acc[ai][bj][m][n] = __builtin_amdgcn_mfma_f32_16x16x32_bf16(bj ? B1[n][k] : B0[n][k], At[m][k], first && (i & 1) == 0 ? (f32x4){0.f, 0.f, 0.f, 0.f} : acc[ai][bj][m][n], 0, 0, 0); } \
        __builtin_amdgcn_s_setprio(0); } while (0)
#define WAIT_L0() do { asm volatile("" ::: "memory"); __builtin_amdgcn_s_waitcnt(0xC07F); asm volatile("" ::: "memory"); } while (0)
#define WAIT_V(n) asm volatile("s_waitcnt vmcnt(" #n ")" ::: "memory")
#define WAIT_L(n) asm volatile("s_waitcnt lgkmcnt(" #n ")" ::: "memory")
#define BAR __builtin_amdgcn_s_barrier()
#define SCHED __builtin_amdgcn_sched_barrier(0)

    unsigned long long outp = (unsigned long long)P.outl;
    asm volatile("" : "+v"(outp));
    Tile cur, nxt; int rnd = slot < R0 ? 0 : 1;
    const unsigned snd = WS_SND + (call & 1u) * SNDB;
    tile_of(rnd * R0 + slot, col, rank, snd, cur);
    bf16x8 At[4][2], B0[2][2], B1[2][2];
    constexpr unsigned SB0 = ((NRANK - 1) * NS + GW) * UNB, STB = U0 * UNB;
    const unsigned kov = WS_PAN + (unsigned)((rank + 1 + (lane + 3) % 7) & (NRANK - 1)) * CHB + (unsigned)(((lane & 31) - 4) / 7) * UNB;
    unsigned tabA, tabB;
    auto tables = [&](const Tile& t) {
        const int ln = lane_now();
        const unsigned la = t.sA + (unsigned)ln * t.stA, ra = (unsigned)(NTM + t.v) * PANB + kov;
        const unsigned lb = SB0 + snd + (unsigned)t.tm * UNB + (unsigned)ln * STB, rb = (unsigned)t.tm * PANB + kov;
        tabA = ln < 4 ? la : ra;
        tabB = (ln < 4 ? lb : rb) + (t.half == 1 ? HTB : 0u);
    };
    auto pick = [&]() {
        Tile t = cur; bool ok = true;
        if (rnd + 1 < RS) tile_of(R0 + rnd * kx + slot, col, rank, snd, t);
        else {
            const unsigned d = (unsigned)__builtin_amdgcn_raw_ptr_buffer_atomic_add_i32(1, rws, lane_now() == 0 ? 0u : 0xfffffff0u, WS_HEAD + (unsigned)rank * 128u, 0);
            const unsigned q = (unsigned)__builtin_amdgcn_readfirstlane(d) - (call - 1u) * (unsigned)(NITEM + NG);
            ok = q < (unsigned)NITEM;
            if (ok) item_of((int)q, rank, snd, t);
        }
        *(LAS u32x4*)(ctl + 4) = (u32x4){ok ? (unsigned)(t.tm | t.v << 8 | (t.half + 2) << 16) : 0u, t.sA, t.stA, 0u};
        __builtin_amdgcn_raw_ptr_buffer_load_lds(rws, ctl + 64, 4, cnt_of(t), 0, 0, AUX_SYS);
    };
    auto picked = [&](int& level) -> bool {
        const u32x4 f = *(const LAS u32x4*)(ctl + 4);
        level = level_of(ctl[64 + lane_now()]);
        const unsigned w = __builtin_amdgcn_readfirstlane(f[0]);
        nxt.tm = w & 255u; nxt.v = (w >> 8) & 255u; nxt.half = (int)(w >> 16) - 2;
        nxt.sA = __builtin_amdgcn_readfirstlane(f[1]); nxt.stA = __builtin_amdgcn_readfirstlane(f[2]);
        return w != 0u;
    };
    constexpr int JP = KT / 2 - 3, JL = KT / 2 - 1;
    const int jend = wr == 0 && wid == 0 ? JP : JL;
    if (CFG_SYNC && rnd && wr == 0 && wid == 0) {
        const unsigned long long t0 = __builtin_amdgcn_s_memrealtime();
        while (__builtin_amdgcn_readfirstlane(ld_agent((const unsigned*)(wsl + WS_COL + col * 128))) != call && !give_up((unsigned*)(wsl + WS_TMO), call, t0)) __builtin_amdgcn_s_sleep(16);
    }
    int lvl = ready(cur, 1, true);
    STAMP(P, 2);
    tables(cur);
    unsigned a2 = cur.sA, b2 = SB0 + snd + (unsigned)cur.tm * UNB;
    STAGE(SB(0, 0), b2); STAGE(SA(0, 0), a2); STAGE(SB(0, 1), b2 + HTB); STAGE(SA(0, 1), a2 + HTB);
    STAGE(SB(1, 0), b2 + KTB); STAGE(SA(1, 0), a2 + KTB); STAGE(SB(1, 1), b2 + HTB + KTB);
    WAIT_V(0); BAR;
#define ROUND_DONE() do {} while (0)
    bool has_next = false;
#pragma unroll 1
    for (;;) {
        f32x4 acc[2][2][4][2];
        int jchk = lvl < 5 ? 7 * lvl - 4 : jend;
#define SYNC_MMA(ai, first, wv) do { if (wv) WAIT_V(8); WAIT_L0(); if (wr == 1) BAR; MMA(ai, first, wr + 1); if (wr == 0) BAR; SCHED; } while (0)
#define PASS(first, check) do { \
            LDB(B0, 0, 0); LDA(At, 0, 0); LDB(B1, 0, 1); STAGE(SA(1, 1), a2 + KTB + HTB); SCHED; \
            check; \
            a2 = __builtin_amdgcn_readlane(tabA, (j + 1) & 31); b2 = __builtin_amdgcn_readlane(tabB, (j + 1) & 31); \
            SYNC_MMA(0, first, !first); \
            LDA(At, 0, 1); STAGE(SB(0, 0), b2); STAGE(SA(0, 0), a2); STAGE(SB(0, 1), b2 + HTB); \
            SYNC_MMA(1, first, !first); \
            LDB(B0, 1, 0); LDA(At, 1, 0); LDB(B1, 1, 1); STAGE(SA(0, 1), a2 + HTB); \
            SYNC_MMA(0, false, true); \
            LDA(At, 1, 1); STAGE(SB(1, 0), b2 + KTB); STAGE(SA(1, 0), a2 + KTB); STAGE(SB(1, 1), b2 + KTB + HTB); \
            SYNC_MMA(1, false, true); } while (0)
        int j = 0;
        PASS(true, );
#pragma unroll 1
        for (j = 1; j < KT / 2; ++j)
            PASS(false, if (__builtin_expect(j == jchk, 0)) {
                    if (j < JP) { lvl = ready(cur, lvl + 1); jchk = lvl < 5 ? 7 * lvl - 4 : jend; }
                    else if (wr == 0 && j == JP) { pick(); jchk = JL; }
                    else {
                        has_next = picked(lvl);
                        if (has_next) {
                            const int want = nxt.half < 0 ? 1 : 5;
                            if (lvl < want) lvl = ready(nxt, want);
                            tables(nxt);
                        }
                    }
                });
        if (has_next && nxt.half >= 0) STAGE(SA(1, 1), a2 + KTB + HTB);
        {
            const int ln = lane_now(), fr = ln & 15, fq = ln >> 4;
            const rsrc_t rol = mk_rsrc((const void*)outp, OUTB);
            const bool shared = cur.v < NS;
            if (__builtin_amdgcn_readfirstlane(ctl[0]) != 0u) {
#pragma unroll
                for (int i = 0; i < 32; ++i) acc[i >> 4][(i >> 3) & 1][(i >> 1) & 3][i & 1] = __builtin_bit_cast(f32x4, (u32x4){0x7fc00000u, 0x7fc00000u, 0x7fc00000u, 0x7fc00000u});
            }
            if (rnd == 0 && cur.v < GW) {
                const unsigned long long t0 = __builtin_amdgcn_s_memrealtime();
                while (__builtin_amdgcn_s_memrealtime() - t0 < (unsigned long long)((col & 3) * CFG_STAG + (col >> 2) * CFG_STAGB)) __builtin_amdgcn_s_sleep(8);
                if (CFG_SYNC && wr == 0 && slot == 0 && wid == 0) __builtin_amdgcn_raw_buffer_store_b32(call, rws, ln == 0 ? 0u : 0xfffffff0u, WS_COL + (unsigned)col * 128u, AUX_AGENT);
            }
            if (CFG_TCH && (unsigned)(rnd - CFG_TCH) < 2u && slot * 8 + col < SCOL / 8) {
                unsigned lines = voff; asm volatile("" : "+v"(lines)); lines <<= 2;
                __builtin_amdgcn_raw_ptr_buffer_load_lds(rws, ctl + 128, 4, lines, WS_OSN + (unsigned)((rnd - CFG_TCH) * (SCOL / 8) + slot * 8 + col) * 32768u, 0, 0);
            }
            const bool thru = rnd >= RS;
            const unsigned none = HALF && wr == 1 && cur.v == NS - 1 ? 0xf0000000u : 0u;
            const unsigned vo = (unsigned)(wc * 32 + fq * 4) * (NN * 2) + (unsigned)(wr * 256 + fr * 16) + none;
            const unsigned so = (unsigned)(cur.tm * 256) * (NN * 2) + (unsigned)col_of(cur.v, rank) * 2u;
            const unsigned vs = (unsigned)(wc * 32 + fq * 4) * ROWB + (unsigned)(wr * 256 + fr * 16) + none;
            const unsigned ss = WS_OSN + (unsigned)(cur.tm * 256) * ROWB + (unsigned)cur.v * 512u;
            auto finish = [&](auto mode) {
#pragma unroll
                for (int bj = 0; bj < 2; ++bj)
#pragma unroll
                    for (int n = 0; n < 2; ++n)
#pragma unroll
                        for (int r = 0; r < 4; ++r) {
                            u32x4 o;
                            if ((r & 1) == 0) {
#pragma unroll
                                for (int i = 0; i < 8; ++i) { f32x4& a = acc[i >> 2][bj][i & 3][n]; const f32x2 z = silu2((f32x2){a[r], a[r + 1]}); a[r] = z[0]; a[r + 1] = z[1]; }
                            }
#pragma unroll
                            for (int i = 0; i < 4; ++i) o[i] = cvt_pk_bf16(acc[i >> 1][bj][(i & 1) * 2][n][r], acc[i >> 1][bj][(i & 1) * 2 + 1][n][r]);
                            if (mode == 1) STORE16(o, rws, vs, ss + (unsigned)(bj * 128 + n * 16 + r) * ROWB, AUX_SYS);
                            STORE16(o, rol, vo, so + (unsigned)(bj * 128 + n * 16 + r) * (NN * 2), mode == 2 ? CFG_OTH : CFG_OST);
                        }
            };
            if (shared) finish(std::integral_constant<int, 1>());
            else if (thru) finish(std::integral_constant<int, 2>());
            else finish(std::integral_constant<int, 0>());
            WAIT_V(16);
            if (shared) {
                WAIT_VM0();
                unsigned old = 0;
                if (ln == 0) old = __hip_atomic_fetch_add(ctl + 1, 1u, __ATOMIC_RELAXED, __HIP_MEMORY_SCOPE_WORKGROUP);
                old = __builtin_amdgcn_readfirstlane(old);
                if ((old & 7u) == 7u) {
                    const unsigned n = (unsigned)__builtin_amdgcn_raw_ptr_buffer_atomic_add_i32(1, rws, ln == 0 ? 0u : 0xfffffff0u, WS_BAND + (unsigned)cur.tm * 128u, 0);
                    if ((unsigned)__builtin_amdgcn_readfirstlane(n) + 1u == call * NS && ln < NRANK && ln != rank)
                        st_sys((unsigned*)(P.ws[ln] + WS_FLAG + rank * FLAGB) + NTURN + turn_of(cur.tm, ln), call);
                }
            }
        }
        ROUND_DONE();
        if (!has_next) break;
        cur = nxt; ++rnd;
        if (cur.half >= 0) break;
    }
#define HB(b, i) ((b) < 2 ? ((i) < 2 ? SA(b, i) : SB(b, 0)) : (i) < 2 ? SB(i, 1) : 8 * HTB)
#define HSTAGE(b, t) do { const unsigned _o = ((t) & 1) * KTB, _a = __builtin_amdgcn_readlane(tabA, ((t) >> 1) & 31) + _o, _b = __builtin_amdgcn_readlane(tabB, ((t) >> 1) & 31) + _o; \
        STAGE(HB(b, 0), _a); STAGE(HB(b, 1), _a + HTB); STAGE(HB(b, 2), _b); } while (0)
#define HLOAD(b) do { _Pragma("unroll") for (int i = 0; i < 4; ++i) B0[i >> 1][i & 1] = *(const LAS bf16x8*)(lds + HB(b, 2) + boff + i * 1024); \
        _Pragma("unroll") for (int i = 0; i < 8; ++i) At[i >> 1][i & 1] = *(const LAS bf16x8*)(lds + HB(b, 0) + aoff + i * 1024); \
        _Pragma("unroll") for (int i = 0; i < 8; ++i) Au[i >> 1][i & 1] = *(const LAS bf16x8*)(lds + HB(b, 1) + aoff + i * 1024); } while (0)
#define HPASS(b, t, first) do { \
        if (wr == 0) { HLOAD(b); HSTAGE((b + 2) % 3, (t) + 2); } else { HSTAGE((b + 2) % 3, (t) + 2); SCHED; HLOAD(b); if (!first) WAIT_V(6); } \
        WAIT_L0(); if (wr == 1) BAR; \
        __builtin_amdgcn_s_setprio(wr + 1); \
        _Pragma("unroll") for (int i = 0; i < 32; ++i) { const int ai = i >> 4, m = (i >> 2) & 3, n = (i >> 1) & 1, k = i & 1; \
            acc[ai][m][n] = __builtin_amdgcn_mfma_f32_16x16x32_bf16(B0[n][k], ai ? Au[m][k] : At[m][k], first && k == 0 ? (f32x4){0.f, 0.f, 0.f, 0.f} : acc[ai][m][n], 0, 0, 0); } \
        __builtin_amdgcn_s_setprio(0); \
        if (wr == 0) { if (!first) WAIT_V(6); BAR; } SCHED; } while (0)
#pragma unroll 1
    while (PFULL < PN && has_next) {
        f32x4 acc[2][4][2];
        bf16x8 Au[4][2];
        BAR;
        HPASS(0, 0, true);
#pragma unroll 1
        for (int t = 1; t < KT; t += 3) {
            if (wr == 0 && wid == 0 && t == KT - 9) pick();
            HPASS(1, t, false); HPASS(2, t + 1, false); HPASS(0, t + 2, false);
        }
        has_next = picked(lvl);
        if (has_next) {
            (void)ready(nxt, 5);
            tables(nxt);
            HSTAGE(0, 0); HSTAGE(1, 1);
        }
        const int ln = lane_now(), fr = ln & 15, fq = ln >> 4;
        const rsrc_t rol = mk_rsrc((const void*)outp, OUTB);
        if (__builtin_amdgcn_readfirstlane(ctl[0]) != 0u) {
#pragma unroll
            for (int i = 0; i < 16; ++i) acc[i >> 3][(i >> 1) & 3][i & 1] = __builtin_bit_cast(f32x4, (u32x4){0x7fc00000u, 0x7fc00000u, 0x7fc00000u, 0x7fc00000u});
        }
        const unsigned vo = (unsigned)(wc * 32 + fq * 4) * (NN * 2) + (unsigned)(wr * 256 + fr * 16);
        const unsigned so = (unsigned)(cur.tm * 256 + cur.half * 128) * (NN * 2) + (unsigned)col_of(cur.v, rank) * 2u;
#pragma unroll
        for (int n = 0; n < 2; ++n)
#pragma unroll
            for (int r = 0; r < 4; ++r) {
                u32x4 o;
                if ((r & 1) == 0) {
#pragma unroll
                    for (int i = 0; i < 8; ++i) { f32x4& a = acc[i >> 2][i & 3][n]; const f32x2 z = silu2((f32x2){a[r], a[r + 1]}); a[r] = z[0]; a[r + 1] = z[1]; }
                }
#pragma unroll
                for (int i = 0; i < 4; ++i) o[i] = cvt_pk_bf16(acc[i >> 1][(i & 1) * 2][n][r], acc[i >> 1][(i & 1) * 2 + 1][n][r]);
                STORE16(o, rol, vo, so + (unsigned)(n * 16 + r) * (NN * 2), CFG_OTH);
            }
        WAIT_V(8);
        ROUND_DONE();
        cur = nxt; ++rnd;
    }
    WAIT_V(0);
}

__global__ __launch_bounds__(512, 2) void fused_gemm_ar(Params P) {
    extern __shared__ __attribute__((aligned(16))) unsigned char shm[];
    LAS unsigned char* lds = (LAS unsigned char*)shm;
    const int wid = __builtin_amdgcn_readfirstlane(threadIdx.x >> 6), lane = threadIdx.x & 63;
    const int b = __builtin_amdgcn_readfirstlane(blockIdx.x);
    const unsigned long long wsp = (unsigned long long)P.ws[lane & (NRANK - 1)];
    const unsigned t0 = (unsigned)__builtin_amdgcn_s_memrealtime();
    int rk = P.rank;
    asm volatile("" : "+s"(rk) : "s"(P.x), "s"(P.w), "s"(P.wsl), "s"(P.rank), "s"(t0) : "memory");
    unsigned* tkp = (unsigned*)(P.wsl + WS_TICKET + b * 128 + wid * 16);
    const unsigned tk = ld_agent(tkp);
    if (b < NPULL) {
        const unsigned call = __builtin_amdgcn_readfirstlane(tk) + 1u;
        st_agent(tkp, call);
        const int pi = CFG_NPULL ? (b * 8 + wid) % 7 : (b * 8 + wid) / NW, widx = CFG_NPULL ? (b * 8 + wid) / 7 : (b * 8 + wid) % NW;
        const int peer = (P.rank + 1 + pi) & (NRANK - 1);
        if (widx < NW && pi < NRANK - 1) pull_role(P, call, peer, widx, lane);
    } else {
        const int g = b - NPULL;
        if (g < 4 * NRANK * NL && wid < 2) lead_role(P, rk, wsp, tk, t0, g, wid, lane);
        ((LAS unsigned*)(lds + STAGE_BYTES))[lane & 3] = 0u;
        pack_role(P, wsp, tk, t0, g, wid, lane);
        const unsigned call = __builtin_amdgcn_readfirstlane(tk) + 1u;
        st_agent(tkp, call);
        STAMP(P, 1);
        asm volatile("s_waitcnt lgkmcnt(0)" ::: "memory");
        __builtin_amdgcn_s_barrier();
        if (b >= GRID - NG) {
            const int col = (CFG_SWAP >> (b & 7) * 4) & 7;
            const int slot = b < NPS ? KX : (b - NPS) >> 3, kx = KX + ((b & 7) >= 8 - (NG - 8 * KX));
            if (wid < 4) gemm_role<0>(P, call, col, slot, kx, lds);
            else gemm_role<1>(P, call, col, slot, kx, lds);
        }
    }
}
}

extern "C" void kernel_launch(int rank, int nranks, ncclComm_t comm,
                              hipStream_t stream,
                              void* const* d_in, const int* in_sizes, int n_in,
                              void* d_out, int out_size,
                              const int* in_dtypes, int out_dtype,
                              void* const* d_out_all,
                              void* const* d_ws_all, size_t ws_bytes) {
    if (nranks != NRANK || (unsigned)rank >= (unsigned)NRANK || ws_bytes < (size_t)WS_END || n_in < 2 ||
        in_sizes[0] != MM * KSH || in_sizes[1] != KSH * NN || out_size != MM * NN) {
        fprintf(stderr, "unexpected problem geometry\n");
        return;
    }
    static bool attr_done[NRANK] = {};
    if (!attr_done[rank]) {
        HIP_CHECK(hipFuncSetAttribute((const void*)fused_gemm_ar, hipFuncAttributeMaxDynamicSharedMemorySize, LDS_BYTES));
        attr_done[rank] = true;
    }
    Params P = {};
    P.order = ORDER;
    P.x = d_in[0]; P.w = d_in[1]; P.rank = rank; P.wsl = (char*)d_ws_all[rank]; P.outl = (char*)d_out_all[rank];
    for (int p = 0; p < NRANK; ++p) { P.out[p] = (char*)d_out_all[p]; P.ws[p] = (char*)d_ws_all[p]; }
    hipLaunchKernelGGL(fused_gemm_ar, dim3(GRID), dim3(512), LDS_BYTES, stream, P);
    HIP_CHECK(hipGetLastError());
}
```

```cpp
#include <hip/hip_runtime.h>
#include <rccl/rccl.h>
#include <cstdio>
#include <type_traits>

#define HIP_CHECK(expr) do { hipError_t _e = (expr); if (_e != hipSuccess) { \
    fprintf(stderr, "HIP error %s at %s:%d\n", hipGetErrorString(_e), __FILE__, __LINE__); \
    return; } } while (0)

#define LAS __attribute__((address_space(3)))
typedef short bf16x8 __attribute__((ext_vector_type(8)));
typedef float f32x4 __attribute__((ext_vector_type(4)));
typedef unsigned u32x4 __attribute__((ext_vector_type(4)));
typedef unsigned u32x2 __attribute__((ext_vector_type(2)));
typedef __amdgpu_buffer_rsrc_t rsrc_t;
typedef __attribute__((address_space(1))) unsigned* gptr_t;

namespace {
constexpr int NRANK = 8;
constexpr int MM = 4096, KK = 4096, NN = 32768, KSH = KK / NRANK;
constexpr int KT = KK / 64, KTS = KSH / 64, NTM = MM / 256, NPAN = NN / 256;
#define CFG_NS 6
#define CFG_PW 5
constexpr int NS = CFG_NS;
#define CFG_NO 1
constexpr int NO = CFG_NO, NB = NS - NO;
constexpr int SCOL = NS * 256;
constexpr int NRP = NPAN - NRANK * NS;
constexpr int NWP = NS + NRP;
constexpr int NIN = NTM + NWP;
constexpr int NPL = NIN + (NRANK - 1) * NO;
constexpr int NTURN = NIN * 4;
constexpr int NUNIT = (NTM + NRANK * NS + NRP) * 4;
constexpr int GRID = 256;
constexpr int PW = CFG_PW;
#define CFG_NPULL 0
#define CFG_NW (PW * 8)
constexpr int NPULL = CFG_NPULL ? CFG_NPULL : (CFG_NW * (NRANK - 1) + 7) / 8;
constexpr int NW = CFG_NPULL ? NPULL * 8 / (NRANK - 1) : CFG_NW;
constexpr int NPK = GRID - NPULL;
constexpr int NPS = (NPULL + 7) / 8 * 8;
constexpr int KX = (GRID - NPS) / 8;
#define CFG_GW (2 * (KX / 4))
constexpr int GW = CFG_GW;
constexpr int G0 = GW + NTM;
#define CFG_SKIP 1
#define CFG_GM (GW + 2)
constexpr int GM = CFG_GM;
constexpr int NGF = (NWP - GW) / GM, GL = (NWP - GW) % GM;
static_assert(NS <= GW && NWP > GW, "shared panels sit in the first group");
constexpr int PU = 4, PSH = 2;
#define CFG_PUSH 2
constexpr int NPUSH = CFG_PUSH;
constexpr int NPIN = (NTURN - NPUSH) * 64 / PU;
#define CFG_NL (NPUSH + NW * PU / 64 < NB ? NPUSH + NW * PU / 64 : NB)
constexpr int NL = CFG_NL;
static_assert(NPUSH <= NL && NL <= NB, "leading units have a single consumer");
constexpr int OPB = MM / 2 / PU, OSK = OPB / NRANK;
constexpr int NPO = NPIN + NO * OPB;
constexpr int BPB = NB * 128 / PU;
constexpr int NPC = NPO + NTM * BPB;
constexpr unsigned CNT_FULL = 64;

#define CFG_SWAP 0x57461302u
#define CFG_XTRA 1
constexpr int NG = CFG_XTRA ? NPK : 8 * KX;
constexpr int DK = CFG_SKIP ? KX - 2 * GW : 0, R0 = KX - DK;
constexpr int nt_of(int col) { return 4 * ((NWP + 1 - (col >> 2)) / 2); }
constexpr int kx_of(int col) { int n = 0; for (int b = GRID - NG; b < GRID; ++b) n += (int)(CFG_SWAP >> (b & 7) * 4 & 7) == col; return n; }
#define CFG_RS ((nt_of(7) - 1 - R0) / (KX + (NG > 8 * KX)) + 1)
constexpr int RS = CFG_RS;
#define CFG_NP 0u
#define CFG_PR0 2
constexpr int PR0 = CFG_PR0;
constexpr bool np_ok() { for (int c = 0; c < 8; ++c) if (PR0 + (int)(CFG_NP >> c * 4 & 15) > RS) return false; return true; }
static_assert(PR0 >= 1 && np_ok(), "only statically assigned tiles are started early");
constexpr int left_of(int col) { int n = 0; for (int c = 0; c < col; ++c) n += nt_of(c) - R0 - (RS - 1) * kx_of(c); return n; }
constexpr int PN = left_of(8);
#define CFG_FD 0
constexpr int OT = (NRANK - 1) * NTM;
__device__ constexpr int full_of(int m) { const int f = 2 * (PN + m * OT) - NG - CFG_FD; return f < 0 ? 0 : f > NG ? NG : f; }
__device__ constexpr int items_of(int m) { return 2 * (PN + m * OT) - full_of(m); }
constexpr unsigned KTB = 32768, HTB = 16384;
constexpr unsigned PANB = KT * KTB;
constexpr unsigned CHB = KTS * KTB;
constexpr unsigned UNB = 2 * KTB;
constexpr unsigned WS_TICKET = 0;
constexpr unsigned WS_TMO = WS_TICKET + GRID * 128;
constexpr unsigned WS_FLAG = WS_TMO + 128;
constexpr unsigned FLAGB = 4096;
constexpr unsigned WS_LEAD = WS_FLAG + NRANK * FLAGB;
constexpr unsigned WS_HEAD = WS_LEAD + NRANK * 512;
constexpr unsigned WS_COL = WS_HEAD + NRANK * 128;
#define CFG_PHOFF 0
constexpr unsigned WS_PH = WS_COL + 8 * 128 + CFG_PHOFF;
constexpr unsigned WS_ZC = WS_PH + 2 * NRANK * 128 + 128;
constexpr unsigned WS_GRP = WS_ZC + NRANK * NRANK * 128;
constexpr unsigned WS_BAND = WS_GRP + 64 * NRANK * 4;
constexpr unsigned WS_CNT = WS_BAND + 128;
constexpr unsigned WS_CTRL_END = WS_CNT + NPL * NRANK * 128;
constexpr unsigned WS_DBG = 512u << 10;
constexpr unsigned WS_SND = 1u << 20, SNDB = NUNIT * UNB;
constexpr unsigned WS_OSN = WS_SND + NUNIT * UNB + SNDB;
constexpr unsigned WS_PAN = (WS_OSN + MM * SCOL * 2 + PANB - 1) / PANB * PANB;
constexpr unsigned WS_PRK = WS_PAN + NPL * PANB;
constexpr unsigned WS_END = WS_PRK + (CFG_NP ? (RS - PR0) * GRID * 2 * UNB : 0);
constexpr int U0 = G0 + (NRANK - 1) * NS;
constexpr unsigned OUTB = (unsigned)MM * NN * 2;
static_assert(WS_CTRL_END <= WS_DBG, "ctrl region");
constexpr unsigned NLAST = NS * NTM + NO * NRANK * 4;
#define CFG_WIN 88
constexpr int BASE = (CFG_WIN - 1) / NW, DEEP = CFG_WIN - BASE * NW;
static_assert(BASE == 2, "window");
constexpr unsigned STAGE_BYTES = 9 * HTB;
constexpr unsigned LDS_BYTES = STAGE_BYTES + 768;
constexpr unsigned long long DEADLINE = 50000000ull;
constexpr int AUX_SYS = 17, AUX_SYS_NT = 19, AUX_AGENT = 16;
#define CFG_PACE 140
#define CFG_UH (2 * (NPK - 8 * DK))
#define CFG_TH 5000
#define CFG_PH 40
#define CFG_OST 2
#define CFG_OTH AUX_SYS_NT
#define CFG_IST 2
#define DBG_JF 5
#define DBG_OOB 0u
#define CFG_FO 5
constexpr int FS = 5;
#define CFG_HT1 150
#define CFG_HT0 0
#define CFG_ILD 2
#define CFG_PST AUX_AGENT
#define CFG_LDA 0
#define CFG_RLD AUX_SYS
#define CFG_TCH 3
#define CFG_GS 16
#define CFG_AGG 24
static_assert(CFG_AGG % 8 == 0 && CFG_AGG >= NS && NS <= 8 && NIN * 4 <= 512, "groups");
#define CFG_PWND 3
#define CFG_EH 1
#define CFG_MIX 1
static_assert(4 * NRANK * NL <= NPK && (NL - NPUSH) * 64 / PU <= NW && NL * 64 <= 512, "leading units");
#define CFG_PACE0 40
#define CFG_P3 320
#define CFG_P3N 2
#define CFG_LOOK 104
#define CFG_TAIL NW
#define CFG_EVEN 280
#define CFG_EVEND 200
#define CFG_ERATE CFG_RATE
#define CFG_ZB 112
#define CFG_LOOKP 100
#define CFG_PLAN (CFG_ZB + CFG_LOOKP + 50)
#define CFG_DEC (CFG_LOOKP + 50)
#define CFG_QR 0.915f
#define CFG_QH 0.68f
#define CFG_LT 1712
#define CFG_MARG 500.0f
#define CFG_BAL 8
constexpr int ZB = CFG_ZB;
static_assert(ZB <= 127 && ZB <= BPB && ZB % 16 == 0 && NTURN < 504 && NW == PW * 8, "plan");
#define CFG_PWIN 4u
#define CFG_PS 2
#define CFG_RATE 1680
#define CFG_CRED 41
#define CFG_GRID 36
#define CFG_EPS 0
#define CFG_SPIN 8
constexpr unsigned PACE16 = CFG_RATE * CFG_WIN / 64;
#define CFG_PAT 4
#define CFG_ROT 1
#define CFG_DRY 0
#define CFG_GD 8
#define CFG_GS2 8
#define CFG_PRE 1
constexpr int ops_after(int k, int shallow) { return (k == 0 ? 5 : k == 2 ? 2 : 1) + shallow; }
constexpr int ops_since(int k, int back, int shallow) { int n = back - 1; for (int i = 1; i <= back; ++i) n += ops_after((k - i) & 3, shallow); return n; }
static_assert(ops_since(0, 3, 0) <= ops_since(0, 2, 1), "early announcements");

#define CFG_EA 8
#define CFG_EB 4
#define CFG_ER 80
#define CFG_ES 17
#define CFG_EMIN 16
constexpr int NTAB = 4 * (G0 + GM);
static_assert(NTAB <= 256, "byte tables");
struct Order {
    unsigned char from[256], to[256];
    static constexpr int due(int o) {
        const int d = o >= 4 * G0 ? CFG_ER + (o - 4 * G0) / GM * CFG_ES + ((o - 4 * G0) % GM & 1) * CFG_EB
                                  : o / G0 * CFG_ES + (o % G0 < GW ? o % G0 / (GW / 2) * CFG_EB : (o % G0 - GW) / 4 * CFG_EA);
        return d < CFG_EMIN ? CFG_EMIN : d;
    }
    constexpr Order() : from(), to() {
        for (int o = 0; o < NTAB; ++o) {
            int n = 0;
            for (int k = 0; k < NTAB; ++k) n += due(k) < due(o) || (due(k) == due(o) && k < o);
            to[o] = (unsigned char)n; from[n] = (unsigned char)o;
        }
    }
};
constexpr Order ORDER;
static_assert(ORDER.to[NS - 1] == NS - 1, "the leading units come first");

struct Params {
    const void* x; const void* w;
    char* wsl; char* outl;
    int rank, pad[7];
    char* ws[NRANK]; char* out[NRANK];
    Order order;
};
__device__ __forceinline__ int look_up(unsigned tab, int o) { return o < NTAB ? (unsigned)__builtin_amdgcn_readlane(tab, o >> 2) >> (o & 3) * 8 & 255u : o; }

__device__ __forceinline__ rsrc_t mk_rsrc(const void* p, unsigned bytes) {
    const unsigned long long v = (unsigned long long)p;
    const unsigned lo = __builtin_amdgcn_readfirstlane((unsigned)v), hi = __builtin_amdgcn_readfirstlane((unsigned)(v >> 32));
    return __builtin_amdgcn_make_buffer_rsrc((void*)(((unsigned long long)hi << 32) | lo), 0, bytes, 0x00020000);
}
__device__ __forceinline__ u32x4 mk_words(const void* p, unsigned bytes) {
    const unsigned long long v = (unsigned long long)p;
    return (u32x4){(unsigned)__builtin_amdgcn_readfirstlane((unsigned)v), (unsigned)__builtin_amdgcn_readfirstlane((unsigned)(v >> 32)) & 0xffffu, bytes, 0x00020000u};
}
__device__ u32x4 buffer_load16(u32x4 rsrc, unsigned voff, unsigned soff, int aux) __asm("llvm.amdgcn.raw.buffer.load.v4i32");
__device__ __forceinline__ unsigned ld_sys(const unsigned* p) { return __hip_atomic_load(p, __ATOMIC_RELAXED, __HIP_MEMORY_SCOPE_SYSTEM); }
__device__ __forceinline__ unsigned ld_agent(const unsigned* p) { return __hip_atomic_load(p, __ATOMIC_RELAXED, __HIP_MEMORY_SCOPE_AGENT); }
__device__ __forceinline__ void st_sys(unsigned* p, unsigned v) { __hip_atomic_store(p, v, __ATOMIC_RELAXED, __HIP_MEMORY_SCOPE_SYSTEM); }
__device__ __forceinline__ void st_agent(unsigned* p, unsigned v) { __hip_atomic_store(p, v, __ATOMIC_RELAXED, __HIP_MEMORY_SCOPE_AGENT); }
__device__ __forceinline__ void add_agent(unsigned* p, unsigned v) { (void)__hip_atomic_fetch_add(p, v, __ATOMIC_RELAXED, __HIP_MEMORY_SCOPE_AGENT); }
#define STORE16(v, rs, vo, so, aux) do { const u32x4 _d = (v); __builtin_amdgcn_raw_buffer_store_b128(_d, rs, vo, so, aux); asm volatile("s_nop 1" :: "v"(_d) : "memory"); } while (0)
#define WAIT_VM0() asm volatile("s_waitcnt vmcnt(0)" ::: "memory")
#define WAIT_VMN(n) do { asm volatile("" ::: "memory"); __builtin_amdgcn_s_waitcnt(0x0F70 | (n)); asm volatile("" ::: "memory"); } while (0)

__device__ __forceinline__ bool give_up(unsigned* tmo, unsigned call, unsigned long long t0) {
    const unsigned long long dt = __builtin_amdgcn_s_memrealtime() - t0;
    if (dt < 2000ull) return false;
    if ((unsigned)__builtin_amdgcn_readfirstlane(ld_agent(tmo)) == call) return true;
    if (dt > DEADLINE) { st_agent(tmo, call); return true; }
    return false;
}

__device__ __forceinline__ int gpan_of(int v, int consumer) { return v < NS ? consumer * NS + v : (NRANK - 1) * NS + v; }
__device__ __forceinline__ int opan_of(int y, int consumer) { return ((consumer + 1 + y % 7) & 7) * NS + NS - 1 - y / 7; }
__device__ __forceinline__ int col_of(int v, int consumer) { return (v < NWP ? gpan_of(v, consumer) : opan_of(v - NWP, consumer)) * 256; }
__device__ __forceinline__ int band_of(int e, int consumer) { return e < 7 ? (consumer + 1 + e) & 7 : e < 15 ? 8 + ((consumer + e) & 7) : consumer; }
__device__ __forceinline__ int unit_of(int o, int consumer, int& q, int& pos) {
    if (o < 4 * G0) { q = o / G0; const int m = o % G0; pos = q * U0 + gpan_of(m, consumer); return m < GW ? NTM + m : m - GW; }
    pos = o + 4 * (NRANK - 1) * NS;
    o -= 4 * G0;
    const int g = o / (4 * GM), r = o % (4 * GM);
    int i;
    if (GL == 0 || g < NGF) { q = r / GM; i = r % GM; } else { q = r / (GL ? GL : 1); i = r % (GL ? GL : 1); }
    return NTM + GW + GM * g + i;
}

__device__ __forceinline__ unsigned cvt_pk_bf16(float lo, float hi) { unsigned r; asm("v_cvt_pk_bf16_f32 %0, %1, %2" : "=v"(r) : "v"(lo), "v"(hi)); return r; }
__device__ __forceinline__ float silu(float y) {
    const float c = __builtin_amdgcn_fmed3f(y, -60.0f, 60.0f);
    const float e = __builtin_amdgcn_exp2f(c * -1.44269504088896f);
    return y * __builtin_amdgcn_rcpf(1.0f + e);
}

typedef float f32x2 __attribute__((ext_vector_type(2)));
__device__ __forceinline__ f32x2 silu2(f32x2 y) {
    const f32x2 t = __builtin_elementwise_fma(y, (f32x2){-1.44269504088896f, -1.44269504088896f}, (f32x2){-86.5617024533378f, -86.5617024533378f});
    const f32x2 e = {__builtin_amdgcn_fmed3f(__builtin_amdgcn_exp2f(t[0]), 0.0f, 1.0f), __builtin_amdgcn_fmed3f(__builtin_amdgcn_exp2f(t[1]), 0.0f, 1.0f)};
    const f32x2 d = __builtin_elementwise_fma(e, (f32x2){1.1420073898156842e26f, 1.1420073898156842e26f}, (f32x2){1.0f, 1.0f});
    return y * (f32x2){__builtin_amdgcn_rcpf(d[0]), __builtin_amdgcn_rcpf(d[1])};
}

#define STAMP1(P, slot) do {} while (0)
#define STAMP(P, slot) do {} while (0)

template <class T> __device__ __forceinline__ T now_s(T x) { asm volatile("" : "+s"(x)); return x; }
__device__ __forceinline__ int now_v(int x) { asm volatile("" : "+v"(x)); return x; }
__device__ __forceinline__ void pull_role(const Params& P, unsigned tk, unsigned* tkp, int peer, int widx, int lane, int from, LAS u32x4* tab) {
    const int rank = P.rank;
    char* wsl = P.wsl;
    const rsrc_t rwl = mk_rsrc(wsl, WS_END), rol = mk_rsrc(P.outl, OUTB);
    const u32x4 rwr = mk_words(P.ws[peer], WS_END), ror = mk_words(P.out[peer], OUTB);
    {
        const int y = threadIdx.x, x = y - NO * 32, r = x % (2 * NB), band = band_of(x / (2 * NB), rank);
        const int v = x < 0 ? NS - 1 - y / 32 : r % NB, half = x < 0 ? y % 32 : band * 2 + r / NB, at = x < 0 ? y : NO * 32 + (band * NB + v) * 2 + r / NB;
        tab[NPIN / 16 + y] = (u32x4){WS_OSN + (unsigned)at * UNB, (unsigned)half * (128u * NN * 2u) + (unsigned)(peer * NS + v) * 512u,
                                     x >= 0 && x < (NRANK - 1) * 2 * NB && r >= 2 * NB - ZB / 16 ? WS_ZC + (unsigned)(band * NRANK + peer) * 128u : 0u, 0u};
    }
    auto input = [&](int pan, int q, int pos, unsigned c) {
        u32x4 t = {WS_SND + (unsigned)pos * UNB, WS_PAN + (unsigned)pan * PANB + (unsigned)peer * CHB + (unsigned)q * UNB, WS_CNT + (unsigned)(pan * NRANK + peer) * 128u + (unsigned)q * 4u, 0u};
        asm volatile("" : "+v"(t), "+v"(c));
        t[0] += (~c & 1u) * SNDB;
        return t;
    };
    if (threadIdx.x < NPIN / 16) {
        int q, pos;
        const int e = now_v(threadIdx.x), pan = unit_of(e + NPUSH < NTAB ? from : e + NPUSH, rank, q, pos);
        tab[e] = input(pan, q, pos, tk);
    }
    const unsigned call = __builtin_amdgcn_readfirstlane(now_v(tk)) + 1u;
    st_agent(tkp, call);
    unsigned* tmo = (unsigned*)(wsl + WS_TMO);
    constexpr unsigned OOB = 0xfffffff0u;
    const unsigned lane0 = lane == 0 ? 0u : OOB;
    const unsigned hoff = WS_PH + (call & 1u) * (NRANK * 128u);
    const unsigned head = lane == 0 ? hoff + (unsigned)peer * 128u : OOB;
    unsigned dvo = CFG_DRY ? OOB : head;
    auto draw = [&]() { return (unsigned)__builtin_amdgcn_raw_ptr_buffer_atomic_add_i32(1, rwl, dvo, 0, 0); };
    __builtin_amdgcn_raw_buffer_store_b32(0u, rwl, widx ? OOB : lane == 0 ? WS_PH + (~call & 1u) * (NRANK * 128u) + (unsigned)peer * 128u : lane == 1 ? WS_HEAD + (~call & 1u) * 128u : OOB, 0, AUX_SYS);
    auto arrive = [&](unsigned vo, unsigned cnt) { (void)__builtin_amdgcn_raw_ptr_buffer_atomic_add_i32(PU, rwl, vo, cnt, 0); };
    const int wrk = (widx & 7) * (NW / 8) + (widx >> 3);
    const bool deep0 = wrk < DEEP;
    const unsigned per0 = deep0 ? PACE16 / (BASE + 1) : PACE16 / BASE;
    asm volatile("s_waitcnt lgkmcnt(0)" ::: "memory");
    __builtin_amdgcn_s_barrier();
    bool bad = false; int stamped = 0; (void)stamped;
    int skip = NO ? -1 : 0;
    auto index_of = [&](unsigned v) { const int i = __builtin_amdgcn_readfirstlane(v) + NW; return i >= NPIN && skip > 0 ? i + skip : i; };
    int cur = index_of((unsigned)(widx - NW));
    const unsigned fso = WS_FLAG + (unsigned)peer * FLAGB;
    auto first_of = [&](int i) { return i < NPIN ? (i >> (6 - PSH)) + NPUSH : NTURN; };
    int lim = 0, flim = 0, fin = NPC;
    bool planned = ZB == 0;
    unsigned long long plan = 0;
    auto moved = [&](int j) { return (int)(signed char)(plan >> (j * 8)); };
    auto update = [&](const u32x4& f0, const u32x4& f1) {
        const int first = first_of(cur) - lane * 8;
        unsigned un = 0;
#pragma unroll
        for (int j = 0; j < 8; ++j) un |= ((j < 4 ? f0[j & 3] : f1[j & 3]) - call) >> 31 << j;
        un &= 255u << (first < 0 ? 0 : first > 8 ? 8 : first);
        const int l = __builtin_ctzll(__ballot(un != 0u) | (1ull << 63));
        const int u = l * 8 + __builtin_ctz(__builtin_amdgcn_readlane(un, l) | 256u);
        const int fl = u <= NTURN ? (u - NPUSH) << (6 - PSH) : NPC + 128;
        flim = fl > flim ? fl : flim;
        if (!planned && (unsigned)__builtin_amdgcn_readlane(f0[0], 63) == call && (unsigned)__builtin_amdgcn_readlane(f0[2], 63) == call) {
            plan = (unsigned long long)(unsigned)__builtin_amdgcn_readlane(f0[3], 63) << 32 | (unsigned)__builtin_amdgcn_readlane(f0[1], 63);
            planned = true; fin = NPC - moved(peer);
        }
        if (skip < 0 && (unsigned)__builtin_amdgcn_readlane(f1[0], 63) - call * 4u < 4u) {
            skip = (__builtin_amdgcn_readlane(f1[0], 63) & 3) * OSK;
            if (cur >= NPIN) cur += skip;
            const int e = now_v(lane), q = e / 7 & 3, y = e / 28 * 7 + e % 7;
            if (e < skip * 7 / 16) tab[(NPIN + skip) / 16 + e] = input(NIN + y, q, q * U0 + opan_of(y, rank), call - 1u);
        }
        const int pl = skip < 0 ? NPIN : planned ? fin : NPC - ZB;
        lim = flim < pl ? flim : pl;
    };
    auto make_plan = [&](int lane, int rank, unsigned call, char* wsl) {
        unsigned old = 0;
        if (lane == 0) old = __hip_atomic_exchange((gptr_t)(wsl + WS_PH + 2 * NRANK * 128), call, __ATOMIC_RELAXED, __HIP_MEMORY_SCOPE_AGENT);
        if ((unsigned)__builtin_amdgcn_readfirstlane(old) == call) return;
        const unsigned hd = __builtin_amdgcn_raw_buffer_load_b32(rwl, lane < NRANK ? (unsigned)lane * 128u : OOB, hoff, AUX_SYS);
        const int t = lane >> 3, s = lane & 7;
        const unsigned zc = ld_sys((const unsigned*)(P.ws[t] + WS_ZC + (rank * NRANK + s) * 128));
        const bool ok = __all(t == rank || s == rank || s == t || (int)(zc - call * (unsigned)(ZB * PU)) >= 0);
        int h[NRANK], sum = 0, give = 0, take = 0;
#pragma unroll
        for (int j = 0; j < NRANK; ++j) { h[j] = __builtin_amdgcn_readlane(hd, j); sum += j == rank ? 0 : h[j]; }
#pragma unroll
        for (int j = 0; j < NRANK; ++j) {
            const int d = j == rank ? 0 : (sum - (NRANK - 1) * h[j]) * CFG_BAL / (8 * (NRANK - 1));
            h[j] = d > ZB ? ZB : d < -127 ? -127 : d;
            if (h[j] > 0) give += h[j]; else take -= h[j];
        }
        give = take = !ok ? 0 : give < take ? give : take;
        unsigned long long pl = 0;
#pragma unroll
        for (int j = 0; j < NRANK; ++j) {
            int d = h[j];
            if (d > 0) { d = d < give ? d : give; give -= d; } else { d = -d < take ? d : -take; take += d; }
            pl |= (unsigned long long)(d & 255) << (j * 8);
        }
        __builtin_amdgcn_raw_buffer_store_b128((u32x4){call, (unsigned)pl, call, (unsigned)(pl >> 32)}, rwl, lane < NRANK ? WS_FLAG + 63 * 32 + (unsigned)lane * FLAGB : OOB, 0, AUX_SYS);
    };
    auto decide = [&](int lane, int rank, unsigned call, char* wsl) {
        unsigned old = 0;
        if (lane == 0) old = __hip_atomic_exchange((gptr_t)(wsl + WS_PH + 2 * NRANK * 128 + 64), call, __ATOMIC_RELAXED, __HIP_MEMORY_SCOPE_AGENT);
        if ((unsigned)__builtin_amdgcn_readfirstlane(old) == call) return;
        const int c = lane >> 3, r = lane & 7, np = CFG_NP >> c * 4 & 15;
        const u32x4 st = __builtin_amdgcn_raw_buffer_load_b128(rwl, (unsigned)(c * 128 + r * 8), WS_COL, AUX_SYS);
        const unsigned now = (unsigned)__builtin_amdgcn_s_memrealtime();
        const unsigned long long have = __ballot(r >= 1 && r < RS - 1 && st[0] == call && st[2] == call);
        auto shorter = [&](int to) { const int n = to - PR0 + 1; return n < 0 ? 0 : n > np ? np : n; };
        const float tau = (float)(st[3] - st[1]) * (shorter(r + 1) != shorter(r) ? 1.0f / CFG_QR : 1.0f);
        float arr = (float)(int)(st[3] - now) + tau * (shorter(r + 2) != shorter(r + 1) ? CFG_QR : 1.0f);
        arr = (arr < 0.f ? 0.f : arr) + tau * ((float)(RS - 3 - r) - (1.0f - CFG_QR) * (float)(np - shorter(r + 2)));
        if (r + 2 == RS) arr = (float)(int)(st[3] - now);
        const bool last = (have >> lane & 1) && !(have >> lane & 2);
        int m = 0;
        if (__builtin_popcountll(__ballot(last)) == 8) {
            int before = 0, after = 0;
            unsigned long long b = __ballot(last);
#pragma unroll
            for (int k = 0; k < 8; ++k) {
                const int l = __builtin_ctzll(b); b &= b - 1;
                const float a = __builtin_bit_cast(float, __builtin_amdgcn_readlane(__builtin_bit_cast(int, arr), l)), t = __builtin_bit_cast(float, __builtin_amdgcn_readlane(__builtin_bit_cast(int, tau), l));
                before += a < arr ? NG : 0; after += a + t < arr + tau ? NG : 0;
            }
#pragma unroll
            for (int k = 1; k <= NO; ++k) {
                const float end = arr + tau * (before < 8 * full_of(k) ? (after < 8 * (items_of(k) - NG) ? 1.0f + CFG_QH : 1.0f) : before < 8 * items_of(k) ? CFG_QH : 0.0f);
                if (!__any(last && end > (float)(CFG_DEC + NPC - NPIN - (k - 1) * OSK) * (CFG_LT / 256.0f) - CFG_MARG)) m = k;
            }
        }
        const unsigned full = call * CNT_FULL;
#pragma unroll
        for (int k = 0; k < NO; ++k)
            __builtin_amdgcn_raw_buffer_store_b128((u32x4){full, full, full, full}, rwl, k >= m && lane < 56 && r != rank ? (unsigned)lane * 128u : OOB, WS_CNT + (unsigned)(NIN + 7 * k) * (NRANK * 128u), AUX_SYS);
        __builtin_amdgcn_raw_buffer_store_b32(call * 4u + (unsigned)m, rwl, lane < NRANK ? WS_FLAG + 63 * 32 + 16 + (unsigned)lane * FLAGB : OOB, 0, AUX_SYS);
    };
    rsrc_t rd = rwl;
    u32x4 rs = rwr;
    const int rot = CFG_ROT == 1 ? wrk : CFG_ROT == 2 ? widx : CFG_ROT == 3 ? widx >> 1 : 0;
    auto piece = [&](int k) -> unsigned {
        k = k + rot & 3;
        const int q = lane >> 4;
        return CFG_PAT == 0 ? lane * 16 + k * 1024 : CFG_PAT == 2 ? q * 1024 + (k + q & 3) * 256 + (lane & 15) * 16
             : CFG_PAT == 4 ? (lane >> 3) * 512 + k * 128 + (lane & 7) * 16
             : CFG_PAT == 8 ? (lane >> 2) * 256 + k * 64 + (lane & 3) * 16
             : CFG_PAT == 9 ? (lane >> 3) * 512 + (k + (lane >> 3) & 3) * 128 + (lane & 7) * 16
             : (lane >> 3) * 512 + ((k & 1) * 2 + (k >> 1)) * 128 + (lane & 7) * 16;
    };
    unsigned o[PU], in[PU], rows[PU], src, dst, cb, dsh = 12, dadd = 0, per = per0, fvo = CFG_DRY ? OOB - 16u : (unsigned)lane * 32u;
#pragma unroll
    for (int k = 0; k < PU; ++k) { o[k] = in[k] = CFG_DRY ? OOB : piece(k); rows[k] = (piece(k) & 511u) + (piece(k) >> 9) * (NN * 2u); }
    bool tail = CFG_DRY, deep = deep0, look = true;
    int next = CFG_DRY ? NPC : 0, ti;
    auto places = [&]() {
        const u32x4 t = tab[ti >> 4];
        dst = __builtin_amdgcn_readfirstlane(t[1]) + ((ti & 15) << dsh) + dadd;
        src = ZB && cur >= NPC ? dst : __builtin_amdgcn_readfirstlane(t[0]) + ((ti & 15) << 12);
        cb = __builtin_amdgcn_readfirstlane(t[2]);
    };
    auto turn = [&]() {
        if (ZB && cur == NPC - CFG_PLAN) make_plan(now_v(lane), now_s(rank), now_s(call), now_s(wsl));
        if (NO && cur == NPIN - CFG_DEC) decide(now_v(lane), now_s(rank), now_s(call), now_s(wsl));
        const int even = fin - (deep0 ? CFG_EVEND : CFG_EVEN), nout = NPIN + (skip < 0 ? 0 : skip) * NRANK;
        tail = cur >= fin - CFG_TAIL; deep = deep0 || tail || cur >= even;
        per = cur >= even ? CFG_ERATE * NW / 64 : per0;
        if (cur >= nout) {
            rd = rol; dsh = 19;
#pragma unroll
            for (int k = 0; k < PU; ++k) o[k] = rows[k];
        }
        const int all = flim < NPC ? flim - CFG_LOOK : 2 * NPC, one = skip < 0 ? NPIN - CFG_LOOKP : planned ? 2 * NPC : NPC - ZB - CFG_LOOKP;
        look = cur >= all || cur >= one;
        fvo = cur >= all || (look && lane == 63) ? (unsigned)lane * 32u : OOB - 16u;
        next = all < one ? all : one;
        next = next < lim ? next : lim;
        const int at[6] = {NPC - CFG_PLAN, NPIN - CFG_DEC, even, fin - CFG_TAIL, nout, NPC};
#pragma unroll
        for (int j = 0; j < 6; ++j) next = at[j] > cur && at[j] < next ? at[j] : next;
        if (ZB && cur >= NPC) {
            int x = cur - NPC;
            const int me = now_s(peer);
            for (int j = 0; j < me; ++j) x -= moved(j) < 0 ? moved(j) : 0;
            for (int j = 0; j < NRANK; ++j) { const int g = moved(j) > 0 ? moved(j) : 0; if ((unsigned)x < (unsigned)g) { dadd = (unsigned)(j - me) * (SCOL * 2u); ti = NPC - g + x; } x -= g; }
            rs = ror; next = 0; ti = __builtin_amdgcn_readfirstlane(ti); dadd = __builtin_amdgcn_readfirstlane(dadd);
#pragma unroll
            for (int k = 0; k < PU; ++k) in[k] = rows[k];
        }
    };
    int first = CFG_DRY ? 2 : 1;
    while (!planned || cur < fin) {
        const bool lead = first == 1 && cur < (NL - NPUSH) << (6 - PSH);
        if (first == 2 || lead) lim = flim = cur + 1;
        if (!bad && cur >= lim) {
            const unsigned long long t0 = __builtin_amdgcn_s_memrealtime();
            const unsigned pvo = (unsigned)(lane - (first_of(cur) >> 3)) < CFG_PWIN || (lane == 63 && cur >= NPIN) ? (unsigned)lane * 32u : OOB - 16u;
            for (;;) {
                const u32x4 f0 = __builtin_amdgcn_raw_buffer_load_b128(rwl, pvo, fso, AUX_SYS), f1 = __builtin_amdgcn_raw_buffer_load_b128(rwl, pvo + 16u, fso, AUX_SYS);
                update(f0, f1);
                if (cur < lim || (planned && cur >= fin)) break;
                if (give_up(tmo, call, t0)) { bad = true; planned = true; plan = 0; fin = NPC; skip = skip < 0 ? 0 : skip; break; }
                __builtin_amdgcn_s_sleep(CFG_PS);
            }
            asm volatile("" ::: "memory");
            if (cur >= fin && planned) break;
        }
        ti = cur;
        if (bad) { dsh = 19; dadd = 0; }
        else if (!first) turn();
        if (first == 2) ti = NPC;
        places();
        if (lead) {
            const unsigned long long t0 = __builtin_amdgcn_s_memrealtime();
            const unsigned fo = fso + 2048u + (unsigned)(((cur >> 3) + 2 * NPUSH) * 2 + (cur >> 1 & 1)) * 8u;
            for (;;) {
                const unsigned f = __builtin_amdgcn_raw_buffer_load_b32(rwl, (unsigned)(lane & 1) * 4u, fo, AUX_SYS);
                if (__all((int)(f - call) >= 0)) break;
                if (give_up(tmo, call, t0)) { bad = true; planned = true; skip = 0; break; }
                __builtin_amdgcn_s_sleep(CFG_PS);
            }
            asm volatile("" ::: "memory");
        }
        if (bad) {
#pragma unroll
            for (int k = 0; k < PU; ++k) STORE16(((u32x4){0x7fc07fc0u, 0x7fc07fc0u, 0x7fc07fc0u, 0x7fc07fc0u}), rol, cur >= NPIN + skip * NRANK ? rows[k] : OOB, dst, AUX_AGENT);
            cur = index_of(draw()); continue;
        }
        if (first < 2) STAMP1(P, 1);
        unsigned a[PU] = {OOB, OOB, OOB, OOB}, cprev = 0, cold = 0, dp = 0;
        rsrc_t rp = rwl;
        u32x4 v[PU] = {};
        auto put = [&](int k) { STORE16(v[k & 3], k < 0 ? rp : rd, a[k & 3], k < 0 ? dp : dst, CFG_PST); };
        unsigned tick = (unsigned)__builtin_amdgcn_s_memrealtime();
        unsigned slot = (tick << 4) - (tick << 4 & 0xffffffu) % PACE16 + (unsigned)wrk * (PACE16 / NW) - PACE16;
        tick += CFG_GRID - tick % CFG_GRID;
        bool early = false;
        do {
            unsigned nv; u32x4 f0, f1;
            auto step = [&](auto kc) {
                constexpr int k = kc;
                if (!tail) {
                    slot += per;
                    if ((int)((tick << 4) - slot) < 0) tick += CFG_GRID;
                    if ((int)((tick << 4) - slot) < 0) tick += CFG_GRID;
                    asm volatile("" : "+s"(tick), "+s"(slot));
                    WAIT_VMN(ops_since(k, BASE + 1, 0));
                    if (!deep) WAIT_VMN(ops_since(k, BASE, 1));
                    asm volatile("s_memrealtime vcc\n\ts_waitcnt lgkmcnt(0)\n\ts_branch 2f\n"
                                 "1:\ts_add_u32 %0, %0, %3\n2:\ts_sub_u32 vcc_hi, %0, vcc_lo\n\ts_cmp_lt_i32 vcc_hi, %5\n\ts_cbranch_scc1 1b\n"
                                 "3:\ts_memrealtime vcc\n\ts_waitcnt lgkmcnt(0)\n\ts_sub_u32 vcc_lo, %0, vcc_lo\n\ts_cmp_le_i32 vcc_lo, 0\n\ts_cbranch_scc1 4f\n\t"
                                 "s_cmp_le_i32 vcc_lo, %4\n\ts_cbranch_scc1 3b\n\ts_sleep 1\n\ts_branch 3b\n4:"
                                 : "+s"(tick), "+s"(rs), "+s"(src) : "n"(CFG_GRID), "n"(CFG_SPIN), "n"(CFG_PRE) : "scc", "vcc", "memory");
                }
                WAIT_VMN(ops_since(k, BASE + 1, 0));
                v[k] = buffer_load16(rs, in[k], src, CFG_RLD);
                __builtin_amdgcn_sched_barrier(0);
                a[k] = o[k];
                if (k == 0) {
                    nv = draw();
                    f0 = __builtin_amdgcn_raw_buffer_load_b128(rwl, fvo, fso, AUX_SYS); f1 = __builtin_amdgcn_raw_buffer_load_b128(rwl, fvo + 16u, fso, AUX_SYS);
                    arrive(early && cold ? lane0 : OOB, cold);
                    if (early) cold = 0;
                }
                if (k == 2) arrive(cold ? lane0 : OOB, cold);
                put(k - BASE - 1);
                if (!deep) { put(k - BASE); a[(k - BASE) & 3] = OOB; }
                const int d = (int)((tick << 4) - slot);
                if (d > CFG_CRED * 16) slot += (unsigned)(d - CFG_CRED * 16);
                tick += (deep ? CFG_GD : CFG_GS2) >> k & 1 ? 2 * CFG_GRID : CFG_GRID;
            };
            step(std::integral_constant<int, 0>()); step(std::integral_constant<int, 1>()); step(std::integral_constant<int, 2>()); step(std::integral_constant<int, 3>());
            asm volatile("" ::: "memory");
            early = !deep; cold = cprev; cprev = cb; rp = rd; dp = dst;
            ti = cur = index_of(nv);
            if (__builtin_expect(cur >= next, 0)) {
                if (look) update(f0, f1);
                ti = cur;
                if (cur < lim) turn();
            }
            places();
        } while (cur < lim);
#pragma unroll
        for (int k = -1 - BASE; k < 0; ++k) put(k);
        if (cprev | cold) { WAIT_VM0(); arrive(cprev ? lane0 : OOB, cprev); arrive(cold ? lane0 : OOB, cold); }
        if (first == 2) {
            cur = widx; lim = flim = next = 0; dvo = head; tail = false; fvo = (unsigned)lane * 32u;
#pragma unroll
            for (int k = 0; k < PU; ++k) o[k] = in[k] = piece(k);
        }
        first >>= 1;
    }
    STAMP(P, 4);
}

__device__ __forceinline__ void load_w(u32x4 (&v)[8], const rsrc_t rw, int col, int q, int job, int lane) {
    const unsigned vo = (unsigned)((lane >> 4) * 8) * (NN * 2) + (unsigned)(lane & 15) * 16u;
    const unsigned so = (unsigned)(q * 128 + (job >> 1) * 32) * (NN * 2) + (unsigned)(col + (job & 1) * 128) * 2u;
#pragma unroll
    for (int j = 0; j < 8; ++j) v[j] = __builtin_amdgcn_raw_buffer_load_b128(rw, vo, so + (unsigned)j * (NN * 2), 2);
}
__device__ __forceinline__ void store_w(const u32x4 (&v)[8], const rsrc_t rd, unsigned vo, unsigned unit, int job, bool slow) {
    unit += (unsigned)(job >> 2) * KTB + (unsigned)(job & 1) * 8192u + (unsigned)(job >> 1 & 1) * 1024u;
#pragma unroll
    for (int i = 0; i < 8; ++i) {
        u32x4 o;
#pragma unroll
        for (int e = 0; e < 4; ++e) o[e] = __builtin_amdgcn_perm(v[2 * e + 1][i >> 1], v[2 * e][i >> 1], (i & 1) ? 0x07060302u : 0x05040100u);
        STORE16(o, rd, vo, unit + (unsigned)(i >> 2) * HTB + (unsigned)(i & 3) * 2048u, AUX_SYS);
        if (CFG_PWND && i >= CFG_PWND - 1 && i < 7 && slow) WAIT_VMN(CFG_PWND ? CFG_PWND - 1 : 0);
    }
}

__device__ __forceinline__ void lead_role(const Params& P, int rank, unsigned long long wsp, unsigned tk, unsigned t0, int g, int wid, int lane) {
    const int o = NL - 1 - (g >> 5), c = (g >> 3) & 3, job = (c >> 1) * 4 + wid * 2 + (c & 1);
    const int dest = (rank + 1 + g + (o < NPUSH ? CFG_MIX * (2 * c + o) : 0)) & (NRANK - 1);
    u32x4 v[8];
    load_w(v, mk_rsrc(P.w, (unsigned)KSH * NN * 2), dest * SCOL + o * 256, 0, job, lane);
    asm volatile("" : "+v"(tk), "+v"(wsp) :: "memory");
    const bool push = o < NPUSH && dest != rank, count = o < NPUSH || dest == rank;
    char* wsd = (char*)((unsigned long long)(unsigned)__builtin_amdgcn_readlane((unsigned)(wsp >> 32), dest) << 32 | (unsigned)__builtin_amdgcn_readlane((unsigned)wsp, dest));
    gptr_t fp = (gptr_t)(wsd + (count ? WS_CNT + (unsigned)((NTM + o) * NRANK + rank) * 128u : WS_FLAG + (unsigned)rank * FLAGB + 2048u + (unsigned)((o * 4 + c) * 2 + wid) * 4u));
    while (CFG_P3 && push && o >= CFG_P3N && (int)((unsigned)__builtin_amdgcn_s_memrealtime() - t0) < CFG_P3 + (o - CFG_P3N) * 110) __builtin_amdgcn_s_sleep(1);
    store_w(v, mk_rsrc(push ? wsd : P.wsl, WS_END), (unsigned)lane * 16u + (push ? 0u : (~tk & 1u) * SNDB),
            push ? WS_PAN + (unsigned)(NTM + o) * PANB + (unsigned)rank * CHB : WS_SND + (unsigned)(dest * NS + o) * UNB, job, push);
    WAIT_VMN(0);
    STAMP(P, 6);
    if (lane == 0) {
        if (!count) __hip_atomic_store(fp, tk + 1u, __ATOMIC_RELAXED, __HIP_MEMORY_SCOPE_SYSTEM);
        else if ((__hip_atomic_fetch_add((gptr_t)(P.wsl + WS_LEAD + (o * NRANK + dest) * 64), 1u, __ATOMIC_RELAXED, __HIP_MEMORY_SCOPE_AGENT) & 7u) == 7u)
            __hip_atomic_store(fp, (tk + 1u) * CNT_FULL, __ATOMIC_RELAXED, __HIP_MEMORY_SCOPE_SYSTEM);
    }
}

__device__ __forceinline__ void pack_role(const Params& P, unsigned long long wsp, unsigned tk, unsigned t0, int g, int wid, int lane) {
    const int rank = P.rank;
    const rsrc_t rx = mk_rsrc(P.x, (unsigned)MM * KSH * 2), rw = mk_rsrc(P.w, (unsigned)KSH * NN * 2), rwl = mk_rsrc(P.wsl, WS_END);
    const unsigned sv = (unsigned)lane * 16u + (~tk & 1u) * SNDB;
    const unsigned to = ((const unsigned*)P.order.to)[lane];
    constexpr int NH = CFG_UH < NUNIT ? 8 * DK : 0, UH = NH ? CFG_UH : NUNIT;
    const bool late = g >= NPK - NH;
    constexpr int U1 = NRANK * NL, NE = CFG_EH ? NH : 0;
    static_assert(CFG_TH < (UH - U1 - NH) * CFG_PH, "unpaced");
    for (int u = late ? (NE ? U1 : UH) + g - (NPK - NH) : U1 + NE + g; u < (late ? NUNIT : UH); u += !late ? NPK - NH : u < UH ? UH - U1 : NH) {
        int o, dest = -1;
        if (u < 4 * U0) {
            const int r = u % U0;
            if (r < NRANK * NS) { o = r >> 3; dest = (rank + 1 + r) & (NRANK - 1); } else o = r - (NRANK - 1) * NS;
            o += u / U0 * G0;
        } else o = u - 4 * (NRANK - 1) * NS;
        int q, pos; const int pan = unit_of(o, dest < 0 ? 0 : dest, q, pos);
        while ((int)((unsigned)__builtin_amdgcn_s_memrealtime() - t0) < (late ? CFG_TH + (u - UH) * CFG_PH : (u - CFG_PACE0) * CFG_PACE / 10)) __builtin_amdgcn_s_sleep(1);
        const unsigned dst = WS_SND + (unsigned)pos * UNB;
        u32x4 v[8];
        if (pan < NTM) {
            const int rb0 = (wid & 3) * 4;
            const unsigned vo = (unsigned)(lane & 15) * (KSH * 2) + (unsigned)(lane >> 4) * 16u;
            const unsigned so = (unsigned)(pan * 256 + rb0 * 16) * (KSH * 2) + (unsigned)(q * 2 + (wid >> 2)) * 128u;
#pragma unroll
            for (int i = 0; i < 8; ++i) v[i] = __builtin_amdgcn_raw_buffer_load_b128(rx, vo + (i & 1) * 64, so + (unsigned)(i >> 1) * (16 * KSH * 2), 0);
#pragma unroll
            for (int i = 0; i < 8; ++i) STORE16(v[i], rwl, sv + (i & 1) * 1024, dst + (unsigned)(wid >> 2) * KTB + (unsigned)(rb0 + (i >> 1)) * 2048u, AUX_SYS);
        } else {
            load_w(v, rw, gpan_of(pan - NTM, dest < 0 ? 0 : dest) * 256, q, wid, lane);
            store_w(v, rwl, sv, dst, wid, false);
        }
        o = look_up(to, o);
        const bool own = lane == rank;
        unsigned long long fp = wsp + (own ? WS_CNT + (unsigned)((pan * NRANK + rank) * 32 + q) * 4u : WS_FLAG + (unsigned)rank * FLAGB + (unsigned)o * 4u);
        unsigned fv = own ? (tk + 1u) * CNT_FULL : tk + 1u;
        asm volatile("" : "+v"(fp), "+v"(fv));
        WAIT_VMN(0);
        __builtin_amdgcn_s_barrier();
        const bool mine = lane < NRANK && (dest < 0 || dest == lane);
        if (wid == 0 && mine && (own || o < CFG_AGG || o >= NTURN / 8 * 8)) __hip_atomic_store((gptr_t)fp, fv, __ATOMIC_RELAXED, __HIP_MEMORY_SCOPE_SYSTEM);
        if (wid == 1 && o >= CFG_AGG && o < NTURN / 8 * 8) {
            unsigned old = 0;
            if (mine && !own) old = __hip_atomic_fetch_add((gptr_t)(P.wsl + WS_GRP) + (o >> 3) * NRANK + lane, 1u, __ATOMIC_RELAXED, __HIP_MEMORY_SCOPE_AGENT);
            const unsigned full = (unsigned)__ballot((old & 7u) == 7u);
            const unsigned long long wsq = (unsigned long long)(unsigned)__builtin_amdgcn_ds_bpermute(lane >> 3 << 2, (int)(wsp >> 32)) << 32 | (unsigned)__builtin_amdgcn_ds_bpermute(lane >> 3 << 2, (int)wsp);
            if (full >> (lane >> 3) & 1) __hip_atomic_store((gptr_t)(wsq + WS_FLAG + (unsigned)rank * FLAGB) + (o & ~7) + (lane & 7), tk + 1u, __ATOMIC_RELAXED, __HIP_MEMORY_SCOPE_SYSTEM);
        }
        if (wid == 2 && dest >= 0 && pan >= NTM + NB) {
            if (lane == 0 && dest != rank) st_sys((unsigned*)(P.wsl + WS_CNT) + ((NIN + (NTM + NS - 1 - pan) * 7 + ((dest - rank - 1) & 7)) * NRANK + rank) * 32 + q, (tk + 1u) * CNT_FULL);
            unsigned old = 0;
            if (lane == 0) old = __hip_atomic_fetch_add((gptr_t)(P.wsl + WS_BAND), 1u, __ATOMIC_RELAXED, __HIP_MEMORY_SCOPE_AGENT);
            if ((unsigned)__builtin_amdgcn_readfirstlane(old) + 1u == (tk + 1u) * NLAST && lane < NRANK && !own)
                __hip_atomic_store((gptr_t)(wsp + WS_FLAG + (unsigned)rank * FLAGB) + NTURN, tk + 1u, __ATOMIC_RELAXED, __HIP_MEMORY_SCOPE_SYSTEM);
        }
    }
}

struct Tile { int tm, v, half; unsigned sA, stA; };
enum { WHOLE = -1, HEAD = -2, REST = -3 };
__device__ __forceinline__ void tile_of(int idx, int col, int rank, unsigned snd, Tile& t) {
    const int v = idx < 2 * GW ? (idx >> 2) + (col >> 2) * (GW / 2) : 2 * (idx >> 2) + (col >> 2);
    t.tm = 4 * (col & 3) + (idx & 3); t.v = v; t.half = -1;
    const int g = (v - GW) / GM;
    t.sA = snd + (unsigned)(v < GW ? gpan_of(v, rank) : 4 * U0 + g * 3 * GM + v - GW) * UNB;
    t.stA = (unsigned)(v < GW ? U0 : g < NGF ? GM : GL) * UNB;
}
__device__ __forceinline__ void item_of(int q, int full, int rank, unsigned snd, Tile& t) {
    const int n = q < full ? q : full + ((q - full) >> 1);
    int at = 0;
    auto from = [&](auto c) { constexpr int l = left_of(c), a = (nt_of(c) - left_of(c + 1)) * 8 + c; at = n >= l ? a : at; };
#define C(i) std::integral_constant<int, i>()
    from(C(0)); from(C(1)); from(C(2)); from(C(3)); from(C(4)); from(C(5)); from(C(6)); from(C(7));
#undef C
    tile_of((at >> 3) + n, at & 7, rank, snd, t);
    if (n >= PN) {
        const int g = (n - PN) / 28, w = (n - PN) % 28, y = (g >> 2) * 7 + (w >> 2);
        t.tm = 4 * (g & 3) + (w & 3); t.v = NWP + y; t.sA = snd + (unsigned)opan_of(y, rank) * UNB; t.stA = U0 * UNB;
    }
    if (q >= full) t.half = (q - full) & 1;
}

template <int wr>
__device__ __forceinline__ void gemm_role(const Params& P, unsigned call, int col, int slot, int kx, unsigned t0, LAS unsigned char* lds) {
    const int tid = threadIdx.x, wid = __builtin_amdgcn_readfirstlane(tid >> 6), lane = tid & 63, wc = wid & 3;
    const int rank = P.rank;
    char* wsl = P.wsl;
    const rsrc_t rws = mk_rsrc(wsl, WS_END);
    LAS unsigned* ctl = (LAS unsigned*)(lds + STAGE_BYTES);
    const unsigned voff = (unsigned)tid * 16u;
    const unsigned ldsw = (unsigned)wid * 1024u;
    const int aoff = wr * 8192 + lane * 16, boff = wc * 4096 + lane * 16;

    auto lane_now = [&]() { int tz = threadIdx.x; asm volatile("" : "+v"(tz)); return tz & 63; };
    auto cnt_of = [&](const Tile& t) { const int ln = lane_now(); return WS_CNT + (unsigned)((((ln & 32) ? NTM + t.v : t.tm) * NRANK + ((ln >> 2) & 7)) * 32 + (ln & 3)) * 4u; };
    auto level_of = [&](unsigned c) -> int {
        const unsigned long long miss = __ballot((int)(c - call * CNT_FULL) < 0);
        unsigned m = (unsigned)miss | (unsigned)(miss >> 32);
        const bool own = ((m >> (rank * 4)) & 15u) == 0u;
        m |= m >> 16; m |= m >> 8; m |= m >> 4;
        return own ? 1 + __builtin_ctz(m | 16u) : 0;
    };
    auto ready = [&](const Tile& t, int want, bool join = false, bool first = false) -> int {
        if (wr == 0 && wid == 0) {
            const unsigned* c = (const unsigned*)(wsl + cnt_of(t));
            unsigned* tmo = (unsigned*)(wsl + WS_TMO);
            const unsigned long long t0 = __builtin_amdgcn_s_memrealtime();
            if (join)
                while (__builtin_amdgcn_readfirstlane(ld_agent((const unsigned*)(wsl + WS_COL + col * 128))) != call && !give_up(tmo, call, t0)) __builtin_amdgcn_s_sleep(16);
            int got;
            for (;;) {
                got = level_of(ld_sys(c));
                if (got >= want) break;
                if (give_up(tmo, call, t0)) { ctl[0] = 1u; got = 5; break; }
                __builtin_amdgcn_s_sleep(CFG_GS);
            }
            ctl[2] = (unsigned)got;
            if (first) __builtin_amdgcn_fence(__ATOMIC_ACQUIRE, "agent");
            asm volatile("s_waitcnt vmcnt(0) lgkmcnt(0)" ::: "memory");
        }
        asm volatile("" ::: "memory"); __builtin_amdgcn_s_barrier(); asm volatile("" ::: "memory");
        const int r = __builtin_amdgcn_readfirstlane(ctl[2]);
        asm volatile("s_waitcnt lgkmcnt(0)" ::: "memory");
        return r;
    };

#define SA(b, h) (((b) * 2 + (h)) * HTB)
#define SB(b, h) ((4 + (b) * 2 + (h)) * HTB)
#define STAGE(bufoff, soff) do { _Pragma("unroll") for (int _i = 0; _i < 2; ++_i) \
        __builtin_amdgcn_raw_ptr_buffer_load_lds(rws, (LAS unsigned*)(lds + (bufoff) + ldsw + _i * 8192), 16, voff, (soff) + _i * 8192u, 0, CFG_LDA); } while (0)
#define LDA(dst, b, h) do { _Pragma("unroll") for (int m = 0; m < 4; ++m) _Pragma("unroll") for (int k = 0; k < 2; ++k) dst[m][k] = *(const LAS bf16x8*)(lds + SA(b, h) + aoff + m * 2048 + k * 1024); } while (0)
#define LDB(dst, b, h) do { _Pragma("unroll") for (int n = 0; n < 2; ++n) _Pragma("unroll") for (int k = 0; k < 2; ++k) dst[n][k] = *(const LAS bf16x8*)(ldb + (SB(b, h) - SB(0, 0)) + n * 2048 + k * 1024); } while (0)
#define MMA(ai, first, prio, extra) do { __builtin_amdgcn_s_setprio(prio); \
        _Pragma("unroll") for (int i = 0; i < 16; ++i) { \
            const int bj = ((i >> 1) ^ (i >> 2)) & 1, m = i >> 2, n = (i ^ (i >> 2)) & 1, k = i & 1; \
            if (first) asm volatile("v_mfma_f32_16x16x32_bf16 %0, %1, %2, 0\n\tv_mfma_f32_16x16x32_bf16 %0, %3, %4, %0" : "=&v"(acc[ai][bj][m][n]) \
                         : "v"(bj ? B1[n][k] : B0[n][k]), "v"(At[m][k]), "v"(bj ? B1[n][k ^ 1] : B0[n][k ^ 1]), "v"(At[m][k ^ 1])); \
            else asm volatile("v_mfma_f32_16x16x32_bf16 %0, %1, %2, %0\n\tv_mfma_f32_16x16x32_bf16 %0, %3, %4, %0" : "+v"(acc[ai][bj][m][n]) \
                         : "v"(bj ? B1[n][k] : B0[n][k]), "v"(At[m][k]), "v"(bj ? B1[n][k ^ 1] : B0[n][k ^ 1]), "v"(At[m][k ^ 1])); } \
        extra; \
        __builtin_amdgcn_s_setprio(0); } while (0)
#define WAIT_L0() do { asm volatile("" ::: "memory"); __builtin_amdgcn_s_waitcnt(0xC07F); asm volatile("" ::: "memory"); } while (0)
#define WAIT_V(n) asm volatile("s_waitcnt vmcnt(" #n ")" ::: "memory")
#define WAIT_L(n) asm volatile("s_waitcnt lgkmcnt(" #n ")" ::: "memory")
#define BAR __builtin_amdgcn_s_barrier()
#define SCHED __builtin_amdgcn_sched_barrier(0)

    LAS unsigned char* ldb = lds + SB(0, 0) + boff;
    asm volatile("" : "+v"(ldb));
    unsigned long long outp = (unsigned long long)P.outl;
    asm volatile("" : "+v"(outp));
    const int np = CFG_NP >> col * 4 & 15;
    const unsigned snd = WS_SND + (call & 1u) * SNDB;
    auto after = [&](int r) { return r == -1 && slot >= R0 ? 1 : r + 1; };
    auto fixed = [&](int r, Tile& t) {
        const int n = r < 0 ? r + np + PR0 : r;
        tile_of(n ? R0 + (n - 1) * kx + slot : slot, col, rank, snd, t);
        t.half = r < 0 ? HEAD : (unsigned)(r - PR0) < (unsigned)np ? REST : WHOLE;
    };
    Tile cur, nxt; int rnd = after(-np - 1);
    fixed(rnd, cur);
    bf16x8 At[4][2], B0[2][2], B1[2][2];
    constexpr unsigned SB0 = ((NRANK - 1) * NS + GW) * UNB, STB = U0 * UNB;
    const int l4 = lane < 4 ? 0 : lane - 4;
    const unsigned kov = WS_PAN + (unsigned)((rank + 1 + l4 % 7) & (NRANK - 1)) * CHB + (unsigned)(l4 / 7 & 3) * UNB;
    unsigned tabA, tabB;
    auto tables = [&](const Tile& t) {
        const int ln = lane_now();
        const unsigned la = t.sA + (unsigned)(ln & 3) * t.stA, ra = (unsigned)(NTM + t.v) * PANB + kov;
        const unsigned lb = SB0 + snd + (unsigned)t.tm * UNB + (unsigned)(ln & 3) * STB, rb = (unsigned)t.tm * PANB + kov;
        const int nloc = t.half == HEAD ? 64 : t.half == REST ? 0 : 4;
        tabA = ln < nloc ? la : ra;
        tabB = (ln < nloc ? lb : rb) + (t.half == 1 ? HTB : 0u);
    };
    auto pick = [&]() {
        Tile t = cur; bool ok = true;
        if (rnd + 1 < RS) fixed(after(rnd), t);
        else {
            const unsigned d = (unsigned)__builtin_amdgcn_raw_ptr_buffer_atomic_add_i32(1, rws, lane_now() == 0 ? 0u : 0xfffffff0u, WS_HEAD + (call & 1u) * 128u, 0);
            int m = 0;
            if (NO) {
                const unsigned long long t0 = __builtin_amdgcn_s_memrealtime();
                for (;;) {
                    m = __builtin_amdgcn_readfirstlane(__builtin_amdgcn_raw_buffer_load_b32(rws, 0, WS_FLAG + 63 * 32 + 16, AUX_SYS) - call * 4u);
                    if ((unsigned)m < 4u) break;
                    m = NO;
                    if (give_up((unsigned*)(wsl + WS_TMO), call, t0)) { ctl[0] = 1u; break; }
                    __builtin_amdgcn_s_sleep(CFG_GS);
                }
            }
            const int q = __builtin_amdgcn_readfirstlane(d);
            ok = q < items_of(m);
            if (ok) item_of(q, full_of(m), rank, snd, t);
        }
        *(LAS u32x4*)(ctl + 4) = (u32x4){ok ? (unsigned)(t.tm | t.v << 8 | (t.half + 4) << 16) : 0u, t.sA, t.stA, 0u};
        __builtin_amdgcn_raw_ptr_buffer_load_lds(rws, ctl + 64, 4, cnt_of(t), 0, 0, AUX_SYS);
    };
    auto picked = [&](int& level) -> bool {
        const u32x4 f = *(const LAS u32x4*)(ctl + 4);
        level = level_of(ctl[64 + lane_now()]);
        const unsigned w = __builtin_amdgcn_readfirstlane(f[0]);
        nxt.tm = w & 255u; nxt.v = (w >> 8) & 255u; nxt.half = (int)(w >> 16) - 4;
        nxt.sA = __builtin_amdgcn_readfirstlane(f[1]); nxt.stA = __builtin_amdgcn_readfirstlane(f[2]);
        return w != 0u;
    };
    constexpr int JP = KT / 2 - 3, JL = KT / 2 - 1;
    const int jend = wr == 0 && wid == 0 ? JP : JL;
    while (CFG_HT0 && (int)((unsigned)__builtin_amdgcn_s_memrealtime() - t0) < CFG_HT0 + ((col & 3) * 2 + (col >> 2)) * CFG_HT1) __builtin_amdgcn_s_sleep(8);
    int lvl = ready(cur, 1, rnd == 1, true);
    STAMP(P, 2);
    tables(cur);
    unsigned a2 = cur.sA, b2 = SB0 + snd + (unsigned)cur.tm * UNB;
    STAGE(SB(0, 0), b2); STAGE(SA(0, 0), a2); STAGE(SB(0, 1), b2 + HTB); STAGE(SA(0, 1), a2 + HTB);
    STAGE(SB(1, 0), b2 + KTB); STAGE(SA(1, 0), a2 + KTB); STAGE(SB(1, 1), b2 + HTB + KTB);
    WAIT_V(0); BAR;
#define ROUND_DONE() do {} while (0)
    bool has_next = false;
    const unsigned one = (lane >> 4) == ((lane >> 2) & 3) ? (lane & 1 ? 0x3f800000u : 0x3f80u) : 0u;
    u32x4 sel0 = {lane & 2 ? 0u : one, lane & 2 ? one : 0u, 0u, 0u}, sel1 = {0u, 0u, sel0[0], sel0[1]};
    asm volatile("" : "+v"(sel0), "+v"(sel1));
#pragma unroll 1
    for (;;) {
        f32x4 acc[2][2][4][2];
#define ACC(i) acc[(i) >> 4][((i) >> 3) & 1][((i) >> 1) & 3][(i) & 1]
        unsigned prk = WS_PRK + (unsigned)((rnd < 0 ? rnd + np : rnd - PR0) * GRID + (int)blockIdx.x) * (2 * UNB);
        u32x4 pf[4];
#pragma unroll
        for (int i = 0; i < 4; ++i) asm volatile("" : "=v"(pf[i]));
        int jf = cur.half == REST ? DBG_JF + slot % CFG_FO : 99, je = -1, fe = 0;
#define FOLD2(a, b, p) "v_mfma_f32_16x16x32_bf16 %" #a ", %8, %" #p ", %" #a "\n\tv_mfma_f32_16x16x32_bf16 %" #b ", %9, %" #p ", %" #b "\n\t"
#define FOLD() do { \
            _Pragma("unroll") for (int c = 0; c < 4; ++c) \
                asm volatile("s_cmp_lg_u32 %14, %15\n\ts_cbranch_scc1 1f\n\t" FOLD2(0, 1, 10) FOLD2(2, 3, 11) FOLD2(4, 5, 12) FOLD2(6, 7, 13) "\n1:" \
                    : "+v"(ACC(c * 8)), "+v"(ACC(c * 8 + 1)), "+v"(ACC(c * 8 + 2)), "+v"(ACC(c * 8 + 3)), "+v"(ACC(c * 8 + 4)), "+v"(ACC(c * 8 + 5)), "+v"(ACC(c * 8 + 6)), "+v"(ACC(c * 8 + 7)) \
                    : "v"(sel0), "v"(sel1), "v"(pf[0]), "v"(pf[1]), "v"(pf[2]), "v"(pf[3]), "s"(fe), "n"(c + 1) : "scc"); \
            if (fe < 4) { \
                _Pragma("unroll") for (int i = 0; i < 4; ++i) pf[i] = __builtin_amdgcn_raw_buffer_load_b128(rws, voff + DBG_OOB, prk + i * 8192u, CFG_ILD); \
                prk += 32768u; je = j; } \
            jf += FS; ++fe; } while (0)
        int jchk = lvl < 5 && cur.half != HEAD ? 7 * lvl - 4 : jend;
#define SYNC_MMA(ai, first, wv, extra) do { if (wv) WAIT_V(8); WAIT_L0(); if (wr == 1) BAR; MMA(ai, first, wr + 1, extra); if (wr == 0) BAR; SCHED; } while (0)
#define WAIT_X() asm volatile("s_cmp_eq_u32 %0, %1\n\ts_waitcnt vmcnt(%2)\n\ts_cbranch_scc1 1f\n\ts_waitcnt vmcnt(8)\n1:" :: "s"(j), "s"(je), "n"(12) : "scc", "memory")
#define PASS(first, check, extra) do { \
            LDB(B0, 0, 0); LDA(At, 0, 0); LDB(B1, 0, 1); STAGE(SA(1, 1), a2 + KTB + HTB); SCHED; \
            check; \
            a2 = __builtin_amdgcn_readlane(tabA, (j + 1) & 31); b2 = __builtin_amdgcn_readlane(tabB, (j + 1) & 31); \
            SYNC_MMA(0, first, !first, ); \
            LDA(At, 0, 1); STAGE(SB(0, 0), b2); STAGE(SA(0, 0), a2); STAGE(SB(0, 1), b2 + HTB); \
            SYNC_MMA(1, first, !first, extra); \
            LDB(B0, 1, 0); LDA(At, 1, 0); LDB(B1, 1, 1); STAGE(SA(0, 1), a2 + HTB); \
            if (!first) WAIT_X(); \
            SYNC_MMA(0, false, first, ); \
            LDA(At, 1, 1); STAGE(SB(1, 0), b2 + KTB); STAGE(SA(1, 0), a2 + KTB); STAGE(SB(1, 1), b2 + KTB + HTB); \
            if (!first) WAIT_X(); \
            SYNC_MMA(1, false, first, ); } while (0)
        int j = cur.half == HEAD ? KT / 2 - 4 : cur.half == REST ? 4 : 0;
        PASS(true, , );
#pragma unroll 1
        for (++j; j < KT / 2; ++j)
            PASS(false, if (__builtin_expect(j == jchk, 0)) {
                    if (j < JP) { lvl = ready(cur, lvl + 1); jchk = lvl < 5 ? 7 * lvl - 4 : jend; }
                    else if (wr == 0 && j == JP) { pick(); jchk = JL; }
                    else {
                        has_next = picked(lvl);
                        if (has_next) {
                            const int want = nxt.half >= 0 ? 5 : nxt.half == REST ? 2 : 1;
                            const bool join = rnd == -1 && slot >= R0;
                            if (lvl < want || join) lvl = ready(nxt, want, join);
                            tables(nxt);
                        }
                    }
                }, if (__builtin_expect(j == jf, 0)) FOLD());
        asm volatile("s_nop 7" ::: "memory");
        SCHED;
        if (has_next && nxt.half >= 0) STAGE(SA(1, 1), a2 + KTB + HTB);
        {
            const int ln = lane_now(), fr = ln & 15, fq = ln >> 4;
            const rsrc_t rol = mk_rsrc((const void*)outp, OUTB);
            const bool shared = cur.v < NS;
            if (__builtin_amdgcn_readfirstlane(ctl[0]) != 0u) {
#pragma unroll
                for (int i = 0; i < 32; ++i) acc[i >> 4][(i >> 3) & 1][(i >> 1) & 3][i & 1] = __builtin_bit_cast(f32x4, (u32x4){0x7fc00000u, 0x7fc00000u, 0x7fc00000u, 0x7fc00000u});
            }
            if ((unsigned)rnd < (unsigned)RS && wr == 0 && slot == 0 && wid == 0)
                __builtin_amdgcn_raw_buffer_store_b64((u32x2){call, (unsigned)__builtin_amdgcn_s_memrealtime()}, rws, ln == 0 ? 0u : 0xfffffff0u, WS_COL + (unsigned)(col * 128 + rnd * 8), AUX_SYS);
            if (CFG_TCH && (unsigned)(rnd - CFG_TCH) < 2u && slot * 8 + col < SCOL / 8) {
                unsigned lines = voff; asm volatile("" : "+v"(lines)); lines <<= 2;
                __builtin_amdgcn_raw_ptr_buffer_load_lds(rws, ctl + 128, 4, lines, WS_OSN + (unsigned)((rnd - CFG_TCH) * (SCOL / 8) + slot * 8 + col) * 32768u, 0, 0);
            }
            const bool thru = rnd >= RS;
            const unsigned vo = (unsigned)(wc * 32 + fq * 4) * (NN * 2) + (unsigned)(wr * 256 + fr * 16);
            const unsigned so = (unsigned)(cur.tm * 256) * (NN * 2) + (unsigned)col_of(cur.v, rank) * 2u;
            const unsigned vs = (unsigned)(wc * 32 + fq * 4) * 512u + (unsigned)(wr * 256 + fr * 16);
            const unsigned ss = WS_OSN + (unsigned)(cur.v < NB ? NO * NTM + cur.tm * NB + cur.v : (NS - 1 - cur.v) * NTM + cur.tm) * (2 * UNB);
            auto finish = [&](auto mode) {
#pragma unroll
                for (int bj = 0; bj < 2; ++bj)
#pragma unroll
                    for (int n = 0; n < 2; ++n)
#pragma unroll
                        for (int r = 0; r < 4; ++r) {
                            u32x4 o;
                            if ((r & 1) == 0) {
#pragma unroll
                                for (int i = 0; i < 8; ++i) { f32x4& a = acc[i >> 2][bj][i & 3][n]; const f32x2 z = silu2((f32x2){a[r], a[r + 1]}); a[r] = z[0]; a[r + 1] = z[1]; }
                            }
#pragma unroll
                            for (int i = 0; i < 4; ++i) o[i] = cvt_pk_bf16(acc[i >> 1][bj][(i & 1) * 2][n][r], acc[i >> 1][bj][(i & 1) * 2 + 1][n][r]);
                            if (mode == 1) STORE16(o, rws, vs, ss + (unsigned)(bj * 128 + n * 16 + r) * 512u, AUX_SYS);
                            STORE16(o, rol, vo, so + (unsigned)(bj * 128 + n * 16 + r) * (NN * 2), mode == 2 ? CFG_OTH : CFG_OST);
                        }
            };
            if (rnd < 0) {
#pragma unroll
                for (int i = 0; i < 16; ++i) {
                    const f32x4 a = ACC(2 * i), b = ACC(2 * i + 1);
                    STORE16(((u32x4){cvt_pk_bf16(a[0], a[1]), cvt_pk_bf16(a[2], a[3]), cvt_pk_bf16(b[0], b[1]), cvt_pk_bf16(b[2], b[3])}), rws, voff, prk + i * 8192u, CFG_IST);
                }
            }
            else if (shared) finish(std::integral_constant<int, 1>());
            else if (thru) finish(std::integral_constant<int, 2>());
            else finish(std::integral_constant<int, 0>());
            WAIT_V(16);
            if (shared) {
                WAIT_VM0();
                unsigned old = 0;
                if (ln == 0) old = __hip_atomic_fetch_add(ctl + 1, 1u, __ATOMIC_RELAXED, __HIP_MEMORY_SCOPE_WORKGROUP);
                old = __builtin_amdgcn_readfirstlane(old);
                if ((old & 7u) == 7u) {
                    const unsigned n = (unsigned)__builtin_amdgcn_raw_ptr_buffer_atomic_add_i32(1, rws, ln == 0 ? 0u : 0xfffffff0u, WS_BAND, 0);
                    if ((unsigned)__builtin_amdgcn_readfirstlane(n) + 1u == call * NLAST && ln < NRANK && ln != rank)
                        st_sys((unsigned*)(P.ws[ln] + WS_FLAG + rank * FLAGB) + NTURN, call);
                }
            }
        }
        ROUND_DONE();
        if (!has_next) break;
        cur = nxt; rnd = after(rnd);
        if (cur.half >= 0) break;
    }
#define HB(b, i) ((b) < 2 ? ((i) < 2 ? SA(b, i) : SB(b, 0)) : (i) < 2 ? SB(i, 1) : 8 * HTB)
#define HSTAGE(b, t) do { const unsigned _o = ((t) & 1) * KTB, _a = __builtin_amdgcn_readlane(tabA, ((t) >> 1) & 31) + _o, _b = __builtin_amdgcn_readlane(tabB, ((t) >> 1) & 31) + _o; \
        STAGE(HB(b, 0), _a); STAGE(HB(b, 1), _a + HTB); STAGE(HB(b, 2), _b); } while (0)
#define HLOAD(b) do { _Pragma("unroll") for (int i = 0; i < 4; ++i) B0[i >> 1][i & 1] = *(const LAS bf16x8*)(lds + HB(b, 2) + boff + i * 1024); \
        _Pragma("unroll") for (int i = 0; i < 8; ++i) At[i >> 1][i & 1] = *(const LAS bf16x8*)(lds + HB(b, 0) + aoff + i * 1024); \
        _Pragma("unroll") for (int i = 0; i < 8; ++i) Au[i >> 1][i & 1] = *(const LAS bf16x8*)(lds + HB(b, 1) + aoff + i * 1024); } while (0)
#define HPASS(b, t, first) do { \
        if (wr == 0) { HLOAD(b); HSTAGE((b + 2) % 3, (t) + 2); } else { HSTAGE((b + 2) % 3, (t) + 2); SCHED; HLOAD(b); if (!first) WAIT_V(6); } \
        WAIT_L0(); if (wr == 1) BAR; \
        __builtin_amdgcn_s_setprio(wr + 1); \
        _Pragma("unroll") for (int i = 0; i < 32; ++i) { const int ai = i >> 4, m = (i >> 2) & 3, n = (i >> 1) & 1, k = i & 1; \
            acc[ai][m][n] = __builtin_amdgcn_mfma_f32_16x16x32_bf16(B0[n][k], ai ? Au[m][k] : At[m][k], first && k == 0 ? (f32x4){0.f, 0.f, 0.f, 0.f} : acc[ai][m][n], 0, 0, 0); } \
        __builtin_amdgcn_s_setprio(0); \
        if (wr == 0) { if (!first) WAIT_V(6); BAR; } SCHED; } while (0)
#pragma unroll 1
    while (has_next) {
        f32x4 acc[2][4][2];
        bf16x8 Au[4][2];
        BAR;
        HPASS(0, 0, true);
#pragma unroll 1
        for (int t = 1; t < KT; t += 3) {
            if (wr == 0 && wid == 0 && t == KT - 9) pick();
            HPASS(1, t, false); HPASS(2, t + 1, false); HPASS(0, t + 2, false);
        }
        has_next = picked(lvl);
        if (has_next) {
            (void)ready(nxt, 5);
            tables(nxt);
            HSTAGE(0, 0); HSTAGE(1, 1);
        }
        const int ln = lane_now(), fr = ln & 15, fq = ln >> 4;
        const rsrc_t rol = mk_rsrc((const void*)outp, OUTB);
        if (__builtin_amdgcn_readfirstlane(ctl[0]) != 0u) {
#pragma unroll
            for (int i = 0; i < 16; ++i) acc[i >> 3][(i >> 1) & 3][i & 1] = __builtin_bit_cast(f32x4, (u32x4){0x7fc00000u, 0x7fc00000u, 0x7fc00000u, 0x7fc00000u});
        }
        const unsigned vo = (unsigned)(wc * 32 + fq * 4) * (NN * 2) + (unsigned)(wr * 256 + fr * 16);
        const unsigned so = (unsigned)(cur.tm * 256 + cur.half * 128) * (NN * 2) + (unsigned)col_of(cur.v, rank) * 2u;
#pragma unroll
        for (int n = 0; n < 2; ++n)
#pragma unroll
            for (int r = 0; r < 4; ++r) {
                u32x4 o;
                if ((r & 1) == 0) {
#pragma unroll
                    for (int i = 0; i < 8; ++i) { f32x4& a = acc[i >> 2][i & 3][n]; const f32x2 z = silu2((f32x2){a[r], a[r + 1]}); a[r] = z[0]; a[r + 1] = z[1]; }
                }
#pragma unroll
                for (int i = 0; i < 4; ++i) o[i] = cvt_pk_bf16(acc[i >> 1][(i & 1) * 2][n][r], acc[i >> 1][(i & 1) * 2 + 1][n][r]);
                STORE16(o, rol, vo, so + (unsigned)(n * 16 + r) * (NN * 2), CFG_OTH);
            }
        WAIT_V(8);
        ROUND_DONE();
        cur = nxt; ++rnd;
    }
    WAIT_V(0);
}

__global__ __launch_bounds__(512, 2) void fused_gemm_ar(Params P) {
    extern __shared__ __attribute__((aligned(16))) unsigned char shm[];
    LAS unsigned char* lds = (LAS unsigned char*)shm;
    const int wid = __builtin_amdgcn_readfirstlane(threadIdx.x >> 6), lane = threadIdx.x & 63;
    const int b = __builtin_amdgcn_readfirstlane(blockIdx.x);
    const unsigned long long wsp = (unsigned long long)P.ws[lane & (NRANK - 1)];
    const int from = P.order.from[(threadIdx.x + NPUSH) & 255];
    const unsigned t0 = (unsigned)__builtin_amdgcn_s_memrealtime();
    int rk = P.rank;
    asm volatile("" : "+s"(rk) : "s"(P.x), "s"(P.w), "s"(P.wsl), "s"(P.rank), "s"(t0) : "memory");
    unsigned* tkp = (unsigned*)(P.wsl + WS_TICKET + b * 128 + wid * 16);
    const unsigned tk = ld_agent(tkp);
    if (b < NPULL) {
        pull_role(P, tk, tkp, (P.rank + 1 + b / PW) & (NRANK - 1), b % PW * 8 + wid, lane, from, (LAS u32x4*)lds);
    } else {
        const int g = b - NPULL;
        if (g < 4 * NRANK * NL && wid < 2) lead_role(P, rk, wsp, tk, t0, g, wid, lane);
        ((LAS unsigned*)(lds + STAGE_BYTES))[lane & 3] = 0u;
        pack_role(P, wsp, tk, t0, g, wid, lane);
        const unsigned call = __builtin_amdgcn_readfirstlane(tk) + 1u;
        st_agent(tkp, call);
        STAMP(P, 1);
        asm volatile("s_waitcnt lgkmcnt(0)" ::: "memory");
        __builtin_amdgcn_s_barrier();
        if (b >= GRID - NG) {
            const int col = (CFG_SWAP >> (b & 7) * 4) & 7;
            const int slot = b < NPS ? KX : (b - NPS) >> 3, kx = KX + ((b & 7) >= 8 - (NG - 8 * KX));
            if (wid < 4) gemm_role<0>(P, call, col, slot, kx, t0, lds);
            else gemm_role<1>(P, call, col, slot, kx, t0, lds);
        }
    }
}
}

extern "C" void kernel_launch(int rank, int nranks, ncclComm_t comm,
                              hipStream_t stream,
                              void* const* d_in, const int* in_sizes, int n_in,
                              void* d_out, int out_size,
                              const int* in_dtypes, int out_dtype,
                              void* const* d_out_all,
                              void* const* d_ws_all, size_t ws_bytes) {
    if (nranks != NRANK || (unsigned)rank >= (unsigned)NRANK || ws_bytes < (size_t)WS_END || n_in < 2 ||
        in_sizes[0] != MM * KSH || in_sizes[1] != KSH * NN || out_size != MM * NN) {
        fprintf(stderr, "unexpected problem geometry\n");
        return;
    }
    static bool attr_done[NRANK] = {};
    if (!attr_done[rank]) {
        HIP_CHECK(hipFuncSetAttribute((const void*)fused_gemm_ar, hipFuncAttributeMaxDynamicSharedMemorySize, LDS_BYTES));
        attr_done[rank] = true;
    }
    Params P = {};
    P.order = ORDER;
    P.x = d_in[0]; P.w = d_in[1]; P.rank = rank; P.wsl = (char*)d_ws_all[rank]; P.outl = (char*)d_out_all[rank];
    for (int p = 0; p < NRANK; ++p) { P.out[p] = (char*)d_out_all[p]; P.ws[p] = (char*)d_ws_all[p]; }
    hipLaunchKernelGGL(fused_gemm_ar, dim3(GRID), dim3(512), LDS_BYTES, stream, P);
    HIP_CHECK(hipGetLastError());
}
```

```cpp
#include <hip/hip_runtime.h>
#include <rccl/rccl.h>
#include <cstdio>
#include <type_traits>

#define HIP_CHECK(expr) do { hipError_t _e = (expr); if (_e != hipSuccess) { \
    fprintf(stderr, "HIP error %s at %s:%d\n", hipGetErrorString(_e), __FILE__, __LINE__); \
    return; } } while (0)

#define LAS __attribute__((address_space(3)))
typedef short bf16x8 __attribute__((ext_vector_type(8)));
typedef float f32x4 __attribute__((ext_vector_type(4)));
typedef unsigned u32x4 __attribute__((ext_vector_type(4)));
typedef unsigned u32x2 __attribute__((ext_vector_type(2)));
typedef __amdgpu_buffer_rsrc_t rsrc_t;
typedef __attribute__((address_space(1))) unsigned* gptr_t;

namespace {
constexpr int NRANK = 8;
constexpr int MM = 4096, KK = 4096, NN = 32768, KSH = KK / NRANK;
constexpr int KT = KK / 64, KTS = KSH / 64, NTM = MM / 256, NPAN = NN / 256;
#define CFG_NS 6
#define CFG_PW 5
constexpr int NS = CFG_NS;
#define CFG_NO 1
constexpr int NO = CFG_NO, NB = NS - NO;
constexpr int SCOL = NS * 256;
constexpr int NRP = NPAN - NRANK * NS;
constexpr int NWP = NS + NRP;
constexpr int NIN = NTM + NWP;
constexpr int NPL = NIN + (NRANK - 1) * NO;
constexpr int NTURN = NIN * 4;
constexpr int NUNIT = (NTM + NRANK * NS + NRP) * 4;
constexpr int GRID = 256;
constexpr int PW = CFG_PW;
#define CFG_NPULL 0
#define CFG_NW (PW * 8)
constexpr int NPULL = CFG_NPULL ? CFG_NPULL : (CFG_NW * (NRANK - 1) + 7) / 8;
constexpr int NW = CFG_NPULL ? NPULL * 8 / (NRANK - 1) : CFG_NW;
constexpr int NPK = GRID - NPULL;
constexpr int NPS = (NPULL + 7) / 8 * 8;
constexpr int KX = (GRID - NPS) / 8;
#define CFG_GW (2 * (KX / 4))
constexpr int GW = CFG_GW;
constexpr int G0 = GW + NTM;
#define CFG_SKIP 1
#define CFG_GM (GW + 2)
constexpr int GM = CFG_GM;
constexpr int NGF = (NWP - GW) / GM, GL = (NWP - GW) % GM;
static_assert(NS <= GW && NWP > GW, "shared panels sit in the first group");
constexpr int PU = 4, PSH = 2;
#define CFG_PUSH 2
constexpr int NPUSH = CFG_PUSH;
constexpr int NPIN = (NTURN - NPUSH) * 64 / PU;
#define CFG_NL (NPUSH + NW * PU / 64 < NB ? NPUSH + NW * PU / 64 : NB)
constexpr int NL = CFG_NL;
static_assert(NPUSH <= NL && NL <= NB, "leading units have a single consumer");
constexpr int OPB = MM / 2 / PU, OSK = OPB / NRANK;
constexpr int NPO = NPIN + NO * OPB;
constexpr int BPB = NB * 128 / PU;
constexpr int NPC = NPO + NTM * BPB;
constexpr unsigned CNT_FULL = 64;

#define CFG_SWAP 0x57461302u
#define CFG_XTRA 1
constexpr int NG = CFG_XTRA ? NPK : 8 * KX;
constexpr int DK = CFG_SKIP ? KX - 2 * GW : 0, R0 = KX - DK;
constexpr int nt_of(int col) { return 4 * ((NWP + 1 - (col >> 2)) / 2); }
constexpr int kx_of(int col) { int n = 0; for (int b = GRID - NG; b < GRID; ++b) n += (int)(CFG_SWAP >> (b & 7) * 4 & 7) == col; return n; }
#define CFG_RS ((nt_of(7) - 1 - R0) / (KX + (NG > 8 * KX)) + 1)
constexpr int RS = CFG_RS;
#define CFG_NP 0x22112211u
#define CFG_PR0 2
constexpr int PR0 = CFG_PR0;
constexpr int np_max() { int n = 0; for (int c = 0; c < 8; ++c) n = (int)(CFG_NP >> c * 4 & 15) > n ? CFG_NP >> c * 4 & 15 : n; return n; }
static_assert(PR0 >= 1 && PR0 + np_max() <= RS, "only statically assigned tiles are started early");
constexpr int left_of(int col) { int n = 0; for (int c = 0; c < col; ++c) n += nt_of(c) - R0 - (RS - 1) * kx_of(c); return n; }
constexpr int PN = left_of(8);
#define CFG_FD 0
constexpr int OT = (NRANK - 1) * NTM;
__device__ constexpr int full_of(int m) { const int f = 2 * (PN + m * OT) - NG - CFG_FD; return f < 0 ? 0 : f > NG ? NG : f; }
__device__ constexpr int items_of(int m) { return 2 * (PN + m * OT) - full_of(m); }
constexpr unsigned KTB = 32768, HTB = 16384;
constexpr unsigned PANB = KT * KTB;
constexpr unsigned CHB = KTS * KTB;
constexpr unsigned UNB = 2 * KTB;
constexpr unsigned WS_TICKET = 0;
constexpr unsigned WS_TMO = WS_TICKET + GRID * 128;
constexpr unsigned WS_FLAG = WS_TMO + 128;
constexpr unsigned FLAGB = 4096;
constexpr unsigned WS_LEAD = WS_FLAG + NRANK * FLAGB;
constexpr unsigned WS_HEAD = WS_LEAD + NRANK * 512;
constexpr unsigned WS_COL = WS_HEAD + NRANK * 128;
#define CFG_PHOFF 0
constexpr unsigned WS_PH = WS_COL + 8 * 128 + CFG_PHOFF;
constexpr unsigned WS_ZC = WS_PH + 2 * NRANK * 128 + 128;
constexpr unsigned WS_GRP = WS_ZC + NRANK * NRANK * 128;
constexpr unsigned WS_BAND = WS_GRP + 64 * NRANK * 4;
constexpr unsigned WS_CNT = WS_BAND + 128;
constexpr unsigned WS_CTRL_END = WS_CNT + NPL * NRANK * 128;
constexpr unsigned WS_DBG = 512u << 10;
constexpr unsigned WS_SND = 1u << 20, SNDB = NUNIT * UNB;
constexpr unsigned WS_OSN = WS_SND + NUNIT * UNB + SNDB;
constexpr unsigned WS_PAN = (WS_OSN + MM * SCOL * 2 + PANB - 1) / PANB * PANB;
constexpr unsigned WS_PRK = WS_PAN + NPL * PANB;
constexpr unsigned WS_END = WS_PRK + np_max() * GRID * 2 * UNB;
constexpr int U0 = G0 + (NRANK - 1) * NS;
constexpr int UQ = U0 - NRANK * NO;
constexpr unsigned OUTB = (unsigned)MM * NN * 2;
static_assert(WS_CTRL_END <= WS_DBG, "ctrl region");
constexpr unsigned NLAST = NS * NTM + NO * NRANK * 4;
#define CFG_WIN 88
constexpr int BASE = (CFG_WIN - 1) / NW, DEEP = CFG_WIN - BASE * NW;
static_assert(BASE == 2, "window");
constexpr unsigned STAGE_BYTES = 9 * HTB;
constexpr unsigned LDS_BYTES = STAGE_BYTES + 768;
constexpr unsigned long long DEADLINE = 50000000ull;
constexpr int AUX_SYS = 17, AUX_SYS_NT = 19, AUX_AGENT = 16;
#define CFG_PACE 110
#define CFG_UK 1000
#define CFG_PACE2 40
#define CFG_UH (2 * (NPK - 8 * DK))
#define CFG_TH 4000
#define CFG_PH 24
#define CFG_OST 2
#define CFG_OTH AUX_SYS_NT
#define CFG_IST 2
#define DBG_JF 5
#define DBG_OOB 0u
#define CFG_FO 5
constexpr int FS = 5;
#define CFG_HT1 150
#define CFG_HT0 0
#define CFG_ILD 2
#define CFG_PST AUX_AGENT
#define CFG_LDA 0
#define CFG_RLD AUX_SYS
#define CFG_TCH 3
#define CFG_GS 16
#define CFG_AGG 24
static_assert(CFG_AGG % 8 == 0 && CFG_AGG >= NS && NS <= 8 && NIN * 4 <= 512, "groups");
#define CFG_PWND 3
#define CFG_EH 1
#define CFG_MIX 1
static_assert(4 * NRANK * NL <= NPK && (NL - NPUSH) * 64 / PU <= NW && NL * 64 <= 512, "leading units");
#define CFG_PACE0 40
#define CFG_P3 320
#define CFG_P3N 2
#define CFG_LOOK 104
#define CFG_TAIL NW
#define CFG_EVEN 280
#define CFG_EVEND 200
#define CFG_ERATE CFG_RATE
#define CFG_ZB 112
#define CFG_LOOKP 100
#define CFG_PLAN (CFG_ZB + CFG_LOOKP + 50)
#define CFG_DEC (CFG_LOOKP + 50)
#define CFG_QR 0.915f
#define CFG_QH 0.68f
#define CFG_LT 1712
#define CFG_MARG 500.0f
#define CFG_BAL 8
constexpr int ZB = CFG_ZB;
static_assert(ZB <= 127 && ZB <= BPB && ZB % 16 == 0 && NTURN < 504 && NW == PW * 8, "plan");
#define CFG_PWIN 4u
#define CFG_PS 2
#define CFG_RATE 1680
#define CFG_CRED 41
#define CFG_GRID 36
#define CFG_EPS 0
#define CFG_SPIN 8
constexpr unsigned PACE16 = CFG_RATE * CFG_WIN / 64;
#define CFG_OAL 1
#define CFG_OTC 0
#define CFG_PAT 4
#define CFG_ROT 1
#define CFG_DRY 0
#define CFG_GD 8
#define CFG_GS2 8
#define CFG_PRE 1
constexpr int ops_after(int k, int shallow) { return (k == 0 ? 5 : k == 2 ? 2 : 1) + shallow; }
constexpr int ops_since(int k, int back, int shallow) { int n = back - 1; for (int i = 1; i <= back; ++i) n += ops_after((k - i) & 3, shallow); return n; }
static_assert(ops_since(0, 3, 0) <= ops_since(0, 2, 1), "early announcements");

#define CFG_EA 8
#define CFG_EB 4
#define CFG_ER 80
#define CFG_ES 17
#define CFG_EMIN 16
constexpr int NTAB = 4 * (G0 + GM);
static_assert(NTAB <= 256, "byte tables");
struct Order {
    unsigned char from[256], to[256];
    static constexpr int due(int o) {
        const int d = o >= 4 * G0 ? CFG_ER + (o - 4 * G0) / GM * CFG_ES + ((o - 4 * G0) % GM & 1) * CFG_EB
                                  : o / G0 * CFG_ES + (o % G0 < GW ? o % G0 / (GW / 2) * CFG_EB : (o % G0 - GW) / 4 * CFG_EA);
        return d < CFG_EMIN ? CFG_EMIN : d;
    }
    constexpr Order() : from(), to() {
        for (int o = 0; o < NTAB; ++o) {
            int n = 0;
            for (int k = 0; k < NTAB; ++k) n += due(k) < due(o) || (due(k) == due(o) && k < o);
            to[o] = (unsigned char)n; from[n] = (unsigned char)o;
        }
    }
};
constexpr Order ORDER;
static_assert(ORDER.to[NS - 1] == NS - 1, "the leading units come first");

struct Params {
    const void* x; const void* w;
    char* wsl; char* outl;
    int rank, pad[7];
    char* ws[NRANK]; char* out[NRANK];
    Order order;
};
__device__ __forceinline__ int look_up(unsigned tab, int o) { return o < NTAB ? (unsigned)__builtin_amdgcn_readlane(tab, o >> 2) >> (o & 3) * 8 & 255u : o; }

__device__ __forceinline__ rsrc_t mk_rsrc(const void* p, unsigned bytes) {
    const unsigned long long v = (unsigned long long)p;
    const unsigned lo = __builtin_amdgcn_readfirstlane((unsigned)v), hi = __builtin_amdgcn_readfirstlane((unsigned)(v >> 32));
    return __builtin_amdgcn_make_buffer_rsrc((void*)(((unsigned long long)hi << 32) | lo), 0, bytes, 0x00020000);
}
__device__ __forceinline__ u32x4 mk_words(const void* p, unsigned bytes) {
    const unsigned long long v = (unsigned long long)p;
    return (u32x4){(unsigned)__builtin_amdgcn_readfirstlane((unsigned)v), (unsigned)__builtin_amdgcn_readfirstlane((unsigned)(v >> 32)) & 0xffffu, bytes, 0x00020000u};
}
__device__ u32x4 buffer_load16(u32x4 rsrc, unsigned voff, unsigned soff, int aux) __asm("llvm.amdgcn.raw.buffer.load.v4i32");
__device__ __forceinline__ unsigned ld_sys(const unsigned* p) { return __hip_atomic_load(p, __ATOMIC_RELAXED, __HIP_MEMORY_SCOPE_SYSTEM); }
__device__ __forceinline__ unsigned ld_agent(const unsigned* p) { return __hip_atomic_load(p, __ATOMIC_RELAXED, __HIP_MEMORY_SCOPE_AGENT); }
__device__ __forceinline__ void st_sys(unsigned* p, unsigned v) { __hip_atomic_store(p, v, __ATOMIC_RELAXED, __HIP_MEMORY_SCOPE_SYSTEM); }
__device__ __forceinline__ void st_agent(unsigned* p, unsigned v) { __hip_atomic_store(p, v, __ATOMIC_RELAXED, __HIP_MEMORY_SCOPE_AGENT); }
__device__ __forceinline__ void add_agent(unsigned* p, unsigned v) { (void)__hip_atomic_fetch_add(p, v, __ATOMIC_RELAXED, __HIP_MEMORY_SCOPE_AGENT); }
#define STORE16(v, rs, vo, so, aux) do { const u32x4 _d = (v); __builtin_amdgcn_raw_buffer_store_b128(_d, rs, vo, so, aux); asm volatile("s_nop 1" :: "v"(_d) : "memory"); } while (0)
#define WAIT_VM0() asm volatile("s_waitcnt vmcnt(0)" ::: "memory")
#define WAIT_VMN(n) do { asm volatile("" ::: "memory"); __builtin_amdgcn_s_waitcnt(0x0F70 | (n)); asm volatile("" ::: "memory"); } while (0)

__device__ __forceinline__ bool give_up(unsigned* tmo, unsigned call, unsigned long long t0) {
    const unsigned long long dt = __builtin_amdgcn_s_memrealtime() - t0;
    if (dt < 2000ull) return false;
    if ((unsigned)__builtin_amdgcn_readfirstlane(ld_agent(tmo)) == call) return true;
    if (dt > DEADLINE) { st_agent(tmo, call); return true; }
    return false;
}

__device__ __forceinline__ int gpan_of(int v, int consumer) { return v < NS ? consumer * NS + v : (NRANK - 1) * NS + v; }
__device__ __forceinline__ int opan_of(int y, int consumer) { return ((consumer + 1 + y % 7) & 7) * NS + NS - 1 - y / 7; }
__device__ __forceinline__ int spos_of(int v, int consumer) { return v < NB ? consumer * NB + v : v < NS ? 4 * UQ + (NS - 1 - v) * 4 * NRANK + consumer : NRANK * NB - NS + v; }
__device__ __forceinline__ int step_of(int v) { return v >= NB && v < NS ? NRANK : UQ; }
__device__ __forceinline__ int opos_of(int y, int consumer) { return 4 * UQ + y / 7 * 4 * NRANK + ((consumer + 1 + y % 7) & 7); }
__device__ __forceinline__ int col_of(int v, int consumer) { return (v < NWP ? gpan_of(v, consumer) : opan_of(v - NWP, consumer)) * 256; }
__device__ __forceinline__ int band_of(int e, int consumer) { return e < 7 ? (consumer + 1 + e) & 7 : e < 15 ? 8 + ((consumer + e) & 7) : consumer; }
__device__ __forceinline__ int unit_of(int o, int consumer, int& q, int& pos) {
    if (o < 4 * G0) { q = o / G0; const int m = o % G0; pos = q * step_of(m) + spos_of(m, consumer); return m < GW ? NTM + m : m - GW; }
    pos = o + 4 * (NRANK - 1) * NS;
    o -= 4 * G0;
    const int g = o / (4 * GM), r = o % (4 * GM);
    int i;
    if (GL == 0 || g < NGF) { q = r / GM; i = r % GM; } else { q = r / (GL ? GL : 1); i = r % (GL ? GL : 1); }
    return NTM + GW + GM * g + i;
}

__device__ __forceinline__ unsigned cvt_pk_bf16(float lo, float hi) { unsigned r; asm("v_cvt_pk_bf16_f32 %0, %1, %2" : "=v"(r) : "v"(lo), "v"(hi)); return r; }
__device__ __forceinline__ float silu(float y) {
    const float c = __builtin_amdgcn_fmed3f(y, -60.0f, 60.0f);
    const float e = __builtin_amdgcn_exp2f(c * -1.44269504088896f);
    return y * __builtin_amdgcn_rcpf(1.0f + e);
}

typedef float f32x2 __attribute__((ext_vector_type(2)));
__device__ __forceinline__ f32x2 silu2(f32x2 y) {
    const f32x2 t = __builtin_elementwise_fma(y, (f32x2){-1.44269504088896f, -1.44269504088896f}, (f32x2){-86.5617024533378f, -86.5617024533378f});
    const f32x2 e = {__builtin_amdgcn_fmed3f(__builtin_amdgcn_exp2f(t[0]), 0.0f, 1.0f), __builtin_amdgcn_fmed3f(__builtin_amdgcn_exp2f(t[1]), 0.0f, 1.0f)};
    const f32x2 d = __builtin_elementwise_fma(e, (f32x2){1.1420073898156842e26f, 1.1420073898156842e26f}, (f32x2){1.0f, 1.0f});
    return y * (f32x2){__builtin_amdgcn_rcpf(d[0]), __builtin_amdgcn_rcpf(d[1])};
}

#define STAMP1(P, slot) do {} while (0)
#define STAMP(P, slot) do {} while (0)

template <class T> __device__ __forceinline__ T now_s(T x) { asm volatile("" : "+s"(x)); return x; }
__device__ __forceinline__ int now_v(int x) { asm volatile("" : "+v"(x)); return x; }
__device__ __forceinline__ void pull_role(const Params& P, unsigned tk, unsigned* tkp, int peer, int widx, int lane, int from, LAS u32x4* tab) {
    const int rank = P.rank;
    char* wsl = P.wsl;
    const rsrc_t rwl = mk_rsrc(wsl, WS_END), rol = mk_rsrc(P.outl, OUTB);
    const u32x4 rwr = mk_words(P.ws[peer], WS_END), ror = mk_words(P.out[peer], OUTB);
    {
        const int y = threadIdx.x, x = y - NO * 32, r = x % (2 * NB), band = band_of(x / (2 * NB), rank);
        const int v = x < 0 ? NS - 1 - y / 32 : r % NB, half = x < 0 ? y % 32 : band * 2 + r / NB, at = x < 0 ? y : NO * 32 + (band * NB + v) * 2 + r / NB;
        tab[NPIN / 16 + y] = (u32x4){WS_OSN + (unsigned)at * UNB, (unsigned)half * (128u * NN * 2u) + (unsigned)(peer * NS + v) * 512u,
                                     x >= 0 && x < (NRANK - 1) * 2 * NB && r >= 2 * NB - ZB / 16 ? WS_ZC + (unsigned)(band * NRANK + peer) * 128u : 0u, 0u};
    }
    auto input = [&](int pan, int q, int pos, unsigned c) {
        u32x4 t = {WS_SND + (unsigned)pos * UNB, WS_PAN + (unsigned)pan * PANB + (unsigned)peer * CHB + (unsigned)q * UNB, WS_CNT + (unsigned)(pan * NRANK + peer) * 128u + (unsigned)q * 4u, 0u};
        asm volatile("" : "+v"(t), "+v"(c));
        t[0] += (~c & 1u) * SNDB;
        return t;
    };
    if (threadIdx.x < NPIN / 16) {
        int q, pos;
        const int e = now_v(threadIdx.x), pan = unit_of(e + NPUSH < NTAB ? from : e + NPUSH, rank, q, pos);
        tab[e] = input(pan, q, pos, tk);
    }
    const unsigned call = __builtin_amdgcn_readfirstlane(now_v(tk)) + 1u;
    st_agent(tkp, call);
    unsigned* tmo = (unsigned*)(wsl + WS_TMO);
    constexpr unsigned OOB = 0xfffffff0u;
    const unsigned lane0 = lane == 0 ? 0u : OOB;
    const unsigned hoff = WS_PH + (call & 1u) * (NRANK * 128u);
    const unsigned head = lane == 0 ? hoff + (unsigned)peer * 128u : OOB;
    unsigned dvo = CFG_DRY ? OOB : head;
    auto draw = [&]() { return (unsigned)__builtin_amdgcn_raw_ptr_buffer_atomic_add_i32(1, rwl, dvo, 0, 0); };
    __builtin_amdgcn_raw_buffer_store_b32(0u, rwl, widx ? OOB : lane == 0 ? WS_PH + (~call & 1u) * (NRANK * 128u) + (unsigned)peer * 128u : lane == 1 ? WS_HEAD + (~call & 1u) * 128u : OOB, 0, AUX_SYS);
    auto arrive = [&](unsigned vo, unsigned cnt) { (void)__builtin_amdgcn_raw_ptr_buffer_atomic_add_i32(PU, rwl, vo, cnt, 0); };
    const int wrk = (widx & 7) * (NW / 8) + (widx >> 3);
    const bool deep0 = wrk < DEEP;
    const unsigned per0 = deep0 ? PACE16 / (BASE + 1) : PACE16 / BASE;
    asm volatile("s_waitcnt lgkmcnt(0)" ::: "memory");
    __builtin_amdgcn_s_barrier();
    bool bad = false; int stamped = 0; (void)stamped;
    int skip = NO ? -1 : 0;
    auto index_of = [&](unsigned v) { const int i = __builtin_amdgcn_readfirstlane(v) + NW; return i >= NPIN && skip > 0 ? i + skip : i; };
    int cur = index_of((unsigned)(widx - NW));
    const unsigned fso = WS_FLAG + (unsigned)peer * FLAGB;
    auto first_of = [&](int i) { return i < NPIN ? (i >> (6 - PSH)) + NPUSH : NTURN; };
    int lim = 0, flim = 0, fin = NPC;
    bool planned = ZB == 0;
    unsigned long long plan = 0;
    auto moved = [&](int j) { return (int)(signed char)(plan >> (j * 8)); };
    auto update = [&](const u32x4& f0, const u32x4& f1) {
        const int first = first_of(cur) - lane * 8;
        unsigned un = 0;
#pragma unroll
        for (int j = 0; j < 8; ++j) un |= ((j < 4 ? f0[j & 3] : f1[j & 3]) - call) >> 31 << j;
        un &= 255u << (first < 0 ? 0 : first > 8 ? 8 : first);
        const int l = __builtin_ctzll(__ballot(un != 0u) | (1ull << 63));
        const int u = l * 8 + __builtin_ctz(__builtin_amdgcn_readlane(un, l) | 256u);
        const int fl = u <= NTURN ? (u - NPUSH) << (6 - PSH) : NPC + 128;
        flim = fl > flim ? fl : flim;
        if (!planned && (unsigned)__builtin_amdgcn_readlane(f0[0], 63) == call && (unsigned)__builtin_amdgcn_readlane(f0[2], 63) == call) {
            plan = (unsigned long long)(unsigned)__builtin_amdgcn_readlane(f0[3], 63) << 32 | (unsigned)__builtin_amdgcn_readlane(f0[1], 63);
            planned = true; fin = NPC - moved(peer);
        }
        if (skip < 0 && (unsigned)__builtin_amdgcn_readlane(f1[0], 63) - call * 4u < 4u) {
            skip = (__builtin_amdgcn_readlane(f1[0], 63) & 3) * OSK;
            if (cur >= NPIN) cur += skip;
            const int e = now_v(lane), q = e / 7 & 3, y = e / 28 * 7 + (CFG_OAL ? (e % 7 + 7 - rank) % 7 : e % 7);
            if (e < skip * 7 / 16) tab[(NPIN + skip) / 16 + e] = input(NIN + y, q, q * NRANK + opos_of(y, rank), call - 1u);
        }
        const int pl = skip < 0 ? NPIN : planned ? fin : NPC - ZB;
        lim = flim < pl ? flim : pl;
    };
    auto make_plan = [&](int lane, int rank, unsigned call, char* wsl) {
        unsigned old = 0;
        if (lane == 0) old = __hip_atomic_exchange((gptr_t)(wsl + WS_PH + 2 * NRANK * 128), call, __ATOMIC_RELAXED, __HIP_MEMORY_SCOPE_AGENT);
        if ((unsigned)__builtin_amdgcn_readfirstlane(old) == call) return;
        const unsigned hd = __builtin_amdgcn_raw_buffer_load_b32(rwl, lane < NRANK ? (unsigned)lane * 128u : OOB, hoff, AUX_SYS);
        const int t = lane >> 3, s = lane & 7;
        const unsigned zc = ld_sys((const unsigned*)(P.ws[t] + WS_ZC + (rank * NRANK + s) * 128));
        const bool ok = __all(t == rank || s == rank || s == t || (int)(zc - call * (unsigned)(ZB * PU)) >= 0);
        int h[NRANK], sum = 0, give = 0, take = 0;
#pragma unroll
        for (int j = 0; j < NRANK; ++j) { h[j] = __builtin_amdgcn_readlane(hd, j); sum += j == rank ? 0 : h[j]; }
#pragma unroll
        for (int j = 0; j < NRANK; ++j) {
            const int d = j == rank ? 0 : (sum - (NRANK - 1) * h[j]) * CFG_BAL / (8 * (NRANK - 1));
            h[j] = d > ZB ? ZB : d < -127 ? -127 : d;
            if (h[j] > 0) give += h[j]; else take -= h[j];
        }
        give = take = !ok ? 0 : give < take ? give : take;
        unsigned long long pl = 0;
#pragma unroll
        for (int j = 0; j < NRANK; ++j) {
            int d = h[j];
            if (d > 0) { d = d < give ? d : give; give -= d; } else { d = -d < take ? d : -take; take += d; }
            pl |= (unsigned long long)(d & 255) << (j * 8);
        }
        __builtin_amdgcn_raw_buffer_store_b128((u32x4){call, (unsigned)pl, call, (unsigned)(pl >> 32)}, rwl, lane < NRANK ? WS_FLAG + 63 * 32 + (unsigned)lane * FLAGB : OOB, 0, AUX_SYS);
    };
    auto decide = [&](int lane, int rank, unsigned call, char* wsl) {
        unsigned old = 0;
        if (lane == 0) old = __hip_atomic_exchange((gptr_t)(wsl + WS_PH + 2 * NRANK * 128 + 64), call, __ATOMIC_RELAXED, __HIP_MEMORY_SCOPE_AGENT);
        if ((unsigned)__builtin_amdgcn_readfirstlane(old) == call) return;
        const int c = lane >> 3, r = lane & 7, np = CFG_NP >> c * 4 & 15;
        const u32x4 st = __builtin_amdgcn_raw_buffer_load_b128(rwl, (unsigned)(c * 128 + r * 8), WS_COL, AUX_SYS);
        const unsigned now = (unsigned)__builtin_amdgcn_s_memrealtime();
        const unsigned long long have = __ballot(r >= 1 && r < RS - 1 && st[0] == call && st[2] == call);
        auto shorter = [&](int to) { const int n = to - PR0 + 1; return n < 0 ? 0 : n > np ? np : n; };
        const float tau = (float)(st[3] - st[1]) * (shorter(r + 1) != shorter(r) ? 1.0f / CFG_QR : 1.0f);
        float arr = (float)(int)(st[3] - now) + tau * (shorter(r + 2) != shorter(r + 1) ? CFG_QR : 1.0f);
        arr = (arr < 0.f ? 0.f : arr) + tau * ((float)(RS - 3 - r) - (1.0f - CFG_QR) * (float)(np - shorter(r + 2)));
        if (r + 2 == RS) arr = (float)(int)(st[3] - now);
        const bool last = (have >> lane & 1) && !(have >> lane & 2);
        int m = 0;
        if (__builtin_popcountll(__ballot(last)) == 8) {
            int before = 0, after = 0;
            unsigned long long b = __ballot(last);
#pragma unroll
            for (int k = 0; k < 8; ++k) {
                const int l = __builtin_ctzll(b); b &= b - 1;
                const float a = __builtin_bit_cast(float, __builtin_amdgcn_readlane(__builtin_bit_cast(int, arr), l)), t = __builtin_bit_cast(float, __builtin_amdgcn_readlane(__builtin_bit_cast(int, tau), l));
                before += a < arr ? NG : 0; after += a + t < arr + tau ? NG : 0;
            }
#pragma unroll
            for (int k = 1; k <= NO; ++k) {
                const float end = arr + tau * (before < 8 * full_of(k) ? (after < 8 * (items_of(k) - NG) ? 1.0f + CFG_QH : 1.0f) : before < 8 * items_of(k) ? CFG_QH : 0.0f);
                if (!__any(last && end > (float)(CFG_DEC + NPC - NPIN - (k - 1) * OSK) * (CFG_LT / 256.0f) - CFG_MARG)) m = k;
            }
        }
        const unsigned full = call * CNT_FULL;
#pragma unroll
        for (int k = 0; k < NO; ++k)
            __builtin_amdgcn_raw_buffer_store_b128((u32x4){full, full, full, full}, rwl, k >= m && lane < 56 && r != rank ? (unsigned)lane * 128u : OOB, WS_CNT + (unsigned)(NIN + 7 * k) * (NRANK * 128u), AUX_SYS);
        __builtin_amdgcn_raw_buffer_store_b32(call * 4u + (unsigned)m, rwl, lane < NRANK ? WS_FLAG + 63 * 32 + 16 + (unsigned)lane * FLAGB : OOB, 0, AUX_SYS);
    };
    rsrc_t rd = rwl;
    u32x4 rs = rwr;
    const int rot = CFG_ROT == 1 ? wrk : CFG_ROT == 2 ? widx : CFG_ROT == 3 ? widx >> 1 : 0;
    auto piece = [&](int k) -> unsigned {
        k = k + rot & 3;
        const int q = lane >> 4;
        return CFG_PAT == 0 ? lane * 16 + k * 1024 : CFG_PAT == 2 ? q * 1024 + (k + q & 3) * 256 + (lane & 15) * 16
             : CFG_PAT == 4 ? (lane >> 3) * 512 + k * 128 + (lane & 7) * 16
             : CFG_PAT == 8 ? (lane >> 2) * 256 + k * 64 + (lane & 3) * 16
             : CFG_PAT == 9 ? (lane >> 3) * 512 + (k + (lane >> 3) & 3) * 128 + (lane & 7) * 16
             : (lane >> 3) * 512 + ((k & 1) * 2 + (k >> 1)) * 128 + (lane & 7) * 16;
    };
    unsigned o[PU], in[PU], rows[PU], src, dst, cb, dsh = 12, dadd = 0, per = per0, fvo = CFG_DRY ? OOB - 16u : (unsigned)lane * 32u;
#pragma unroll
    for (int k = 0; k < PU; ++k) { o[k] = in[k] = CFG_DRY ? OOB : piece(k); rows[k] = (piece(k) & 511u) + (piece(k) >> 9) * (NN * 2u); }
    bool tail = CFG_DRY, deep = deep0, look = true;
    int next = CFG_DRY ? NPC : 0, ti;
    auto places = [&]() {
        const u32x4 t = tab[ti >> 4];
        dst = __builtin_amdgcn_readfirstlane(t[1]) + ((ti & 15) << dsh) + dadd;
        src = ZB && cur >= NPC ? dst : __builtin_amdgcn_readfirstlane(t[0]) + ((ti & 15) << 12);
        cb = __builtin_amdgcn_readfirstlane(t[2]);
    };
    auto turn = [&]() {
        if (ZB && cur == NPC - CFG_PLAN) make_plan(now_v(lane), now_s(rank), now_s(call), now_s(wsl));
        if (NO && cur == NPIN - CFG_DEC) decide(now_v(lane), now_s(rank), now_s(call), now_s(wsl));
        const int even = fin - (deep0 ? CFG_EVEND : CFG_EVEN), nout = NPIN + (skip < 0 ? 0 : skip) * NRANK;
        tail = cur >= fin - CFG_TAIL; deep = deep0 || tail || cur >= even;
        per = cur >= even ? CFG_ERATE * NW / 64 : per0;
        if (cur >= nout) {
            rd = rol; dsh = 19;
#pragma unroll
            for (int k = 0; k < PU; ++k) o[k] = rows[k];
        }
        const int all = flim < NPC ? flim - CFG_LOOK : 2 * NPC, one = skip < 0 ? NPIN - CFG_LOOKP : planned ? 2 * NPC : NPC - ZB - CFG_LOOKP;
        look = cur >= all || cur >= one;
        fvo = cur >= all || (look && lane == 63) ? (unsigned)lane * 32u : OOB - 16u;
        next = all < one ? all : one;
        next = next < lim ? next : lim;
        const int at[6] = {NPC - CFG_PLAN, NPIN - CFG_DEC, even, fin - CFG_TAIL, nout, NPC};
#pragma unroll
        for (int j = 0; j < 6; ++j) next = at[j] > cur && at[j] < next ? at[j] : next;
        if (ZB && cur >= NPC) {
            int x = cur - NPC;
            const int me = now_s(peer);
            for (int j = 0; j < me; ++j) x -= moved(j) < 0 ? moved(j) : 0;
            for (int j = 0; j < NRANK; ++j) { const int g = moved(j) > 0 ? moved(j) : 0; if ((unsigned)x < (unsigned)g) { dadd = (unsigned)(j - me) * (SCOL * 2u); ti = NPC - g + x; } x -= g; }
            rs = ror; next = 0; ti = __builtin_amdgcn_readfirstlane(ti); dadd = __builtin_amdgcn_readfirstlane(dadd);
#pragma unroll
            for (int k = 0; k < PU; ++k) in[k] = rows[k];
        }
    };
    int first = CFG_DRY ? 2 : 1;
    while (!planned || cur < fin) {
        const bool lead = first == 1 && cur < (NL - NPUSH) << (6 - PSH);
        if (first == 2 || lead) lim = flim = cur + 1;
        if (!bad && cur >= lim) {
            const unsigned long long t0 = __builtin_amdgcn_s_memrealtime();
            const unsigned pvo = (unsigned)(lane - (first_of(cur) >> 3)) < CFG_PWIN || (lane == 63 && cur >= NPIN) ? (unsigned)lane * 32u : OOB - 16u;
            for (;;) {
                const u32x4 f0 = __builtin_amdgcn_raw_buffer_load_b128(rwl, pvo, fso, AUX_SYS), f1 = __builtin_amdgcn_raw_buffer_load_b128(rwl, pvo + 16u, fso, AUX_SYS);
                update(f0, f1);
                if (cur < lim || (planned && cur >= fin)) break;
                if (give_up(tmo, call, t0)) { bad = true; planned = true; plan = 0; fin = NPC; skip = skip < 0 ? 0 : skip; break; }
                __builtin_amdgcn_s_sleep(CFG_PS);
            }
            asm volatile("" ::: "memory");
            if (cur >= fin && planned) break;
        }
        ti = cur;
        if (bad) { dsh = 19; dadd = 0; }
        else if (!first) turn();
        if (first == 2) ti = NPC;
        places();
        if (lead) {
            const unsigned long long t0 = __builtin_amdgcn_s_memrealtime();
            const unsigned fo = fso + 2048u + (unsigned)(((cur >> 3) + 2 * NPUSH) * 2 + (cur >> 1 & 1)) * 8u;
            for (;;) {
                const unsigned f = __builtin_amdgcn_raw_buffer_load_b32(rwl, (unsigned)(lane & 1) * 4u, fo, AUX_SYS);
                if (__all((int)(f - call) >= 0)) break;
                if (give_up(tmo, call, t0)) { bad = true; planned = true; skip = 0; break; }
                __builtin_amdgcn_s_sleep(CFG_PS);
            }
            asm volatile("" ::: "memory");
        }
        if (bad) {
#pragma unroll
            for (int k = 0; k < PU; ++k) STORE16(((u32x4){0x7fc07fc0u, 0x7fc07fc0u, 0x7fc07fc0u, 0x7fc07fc0u}), rol, cur >= NPIN + skip * NRANK ? rows[k] : OOB, dst, AUX_AGENT);
            cur = index_of(draw()); continue;
        }
        if (first < 2) STAMP1(P, 1);
        unsigned a[PU] = {OOB, OOB, OOB, OOB}, cprev = 0, cold = 0, dp = 0;
        rsrc_t rp = rwl;
        u32x4 v[PU] = {};
        auto put = [&](int k) { STORE16(v[k & 3], k < 0 ? rp : rd, a[k & 3], k < 0 ? dp : dst, CFG_PST); };
        unsigned tick = (unsigned)__builtin_amdgcn_s_memrealtime();
        unsigned slot = (tick << 4) - (tick << 4 & 0xffffffu) % PACE16 + (unsigned)wrk * (PACE16 / NW) - PACE16;
        tick += CFG_GRID - tick % CFG_GRID;
        bool early = false;
        do {
            unsigned nv; u32x4 f0, f1;
            auto step = [&](auto kc) {
                constexpr int k = kc;
                if (!tail) {
                    slot += per;
                    if ((int)((tick << 4) - slot) < 0) tick += CFG_GRID;
                    if ((int)((tick << 4) - slot) < 0) tick += CFG_GRID;
                    asm volatile("" : "+s"(tick), "+s"(slot));
                    WAIT_VMN(ops_since(k, BASE + 1, 0));
                    if (!deep) WAIT_VMN(ops_since(k, BASE, 1));
                    asm volatile("s_memrealtime vcc\n\ts_waitcnt lgkmcnt(0)\n\ts_branch 2f\n"
                                 "1:\ts_add_u32 %0, %0, %3\n2:\ts_sub_u32 vcc_hi, %0, vcc_lo\n\ts_cmp_lt_i32 vcc_hi, %5\n\ts_cbranch_scc1 1b\n"
                                 "3:\ts_memrealtime vcc\n\ts_waitcnt lgkmcnt(0)\n\ts_sub_u32 vcc_lo, %0, vcc_lo\n\ts_cmp_le_i32 vcc_lo, 0\n\ts_cbranch_scc1 4f\n\t"
                                 "s_cmp_le_i32 vcc_lo, %4\n\ts_cbranch_scc1 3b\n\ts_sleep 1\n\ts_branch 3b\n4:"
                                 : "+s"(tick), "+s"(rs), "+s"(src) : "n"(CFG_GRID), "n"(CFG_SPIN), "n"(CFG_PRE) : "scc", "vcc", "memory");
                }
                WAIT_VMN(ops_since(k, BASE + 1, 0));
                v[k] = buffer_load16(rs, in[k], src, CFG_RLD);
                __builtin_amdgcn_sched_barrier(0);
                a[k] = o[k];
                if (k == 0) {
                    nv = draw();
                    f0 = __builtin_amdgcn_raw_buffer_load_b128(rwl, fvo, fso, AUX_SYS); f1 = __builtin_amdgcn_raw_buffer_load_b128(rwl, fvo + 16u, fso, AUX_SYS);
                    arrive(early && cold ? lane0 : OOB, cold);
                    if (early) cold = 0;
                }
                if (k == 2) arrive(cold ? lane0 : OOB, cold);
                put(k - BASE - 1);
                if (!deep) { put(k - BASE); a[(k - BASE) & 3] = OOB; }
                const int d = (int)((tick << 4) - slot);
                if (d > CFG_CRED * 16) slot += (unsigned)(d - CFG_CRED * 16);
                tick += (deep ? CFG_GD : CFG_GS2) >> k & 1 ? 2 * CFG_GRID : CFG_GRID;
            };
            step(std::integral_constant<int, 0>()); step(std::integral_constant<int, 1>()); step(std::integral_constant<int, 2>()); step(std::integral_constant<int, 3>());
            asm volatile("" ::: "memory");
            early = !deep; cold = cprev; cprev = cb; rp = rd; dp = dst;
            ti = cur = index_of(nv);
            if (__builtin_expect(cur >= next, 0)) {
                if (look) update(f0, f1);
                ti = cur;
                if (cur < lim) turn();
            }
            places();
        } while (cur < lim);
#pragma unroll
        for (int k = -1 - BASE; k < 0; ++k) put(k);
        if (cprev | cold) { WAIT_VM0(); arrive(cprev ? lane0 : OOB, cprev); arrive(cold ? lane0 : OOB, cold); }
        if (first == 2) {
            cur = widx; lim = flim = next = 0; dvo = head; tail = false; fvo = (unsigned)lane * 32u;
#pragma unroll
            for (int k = 0; k < PU; ++k) o[k] = in[k] = piece(k);
        }
        first >>= 1;
    }
    STAMP(P, 4);
}

__device__ __forceinline__ void load_w(u32x4 (&v)[8], const rsrc_t rw, int col, int q, int job, int lane) {
    const unsigned vo = (unsigned)((lane >> 4) * 8) * (NN * 2) + (unsigned)(lane & 15) * 16u;
    const unsigned so = (unsigned)(q * 128 + (job >> 1) * 32) * (NN * 2) + (unsigned)(col + (job & 1) * 128) * 2u;
#pragma unroll
    for (int j = 0; j < 8; ++j) v[j] = __builtin_amdgcn_raw_buffer_load_b128(rw, vo, so + (unsigned)j * (NN * 2), 2);
}
__device__ __forceinline__ void store_w(const u32x4 (&v)[8], const rsrc_t rd, unsigned vo, unsigned unit, int job, bool slow) {
    unit += (unsigned)(job >> 2) * KTB + (unsigned)(job & 1) * 8192u + (unsigned)(job >> 1 & 1) * 1024u;
#pragma unroll
    for (int i = 0; i < 8; ++i) {
        u32x4 o;
#pragma unroll
        for (int e = 0; e < 4; ++e) o[e] = __builtin_amdgcn_perm(v[2 * e + 1][i >> 1], v[2 * e][i >> 1], (i & 1) ? 0x07060302u : 0x05040100u);
        STORE16(o, rd, vo, unit + (unsigned)(i >> 2) * HTB + (unsigned)(i & 3) * 2048u, AUX_SYS);
        if (CFG_PWND && i >= CFG_PWND - 1 && i < 7 && slow) WAIT_VMN(CFG_PWND ? CFG_PWND - 1 : 0);
    }
}

__device__ __forceinline__ void lead_role(const Params& P, int rank, unsigned long long wsp, unsigned tk, unsigned t0, int g, int wid, int lane) {
    const int o = NL - 1 - (g >> 5), c = (g >> 3) & 3, job = (c >> 1) * 4 + wid * 2 + (c & 1);
    const int dest = (rank + 1 + g + (o < NPUSH ? CFG_MIX * (2 * c + o) : 0)) & (NRANK - 1);
    u32x4 v[8];
    load_w(v, mk_rsrc(P.w, (unsigned)KSH * NN * 2), dest * SCOL + o * 256, 0, job, lane);
    asm volatile("" : "+v"(tk), "+v"(wsp) :: "memory");
    const bool push = o < NPUSH && dest != rank, count = o < NPUSH || dest == rank;
    char* wsd = (char*)((unsigned long long)(unsigned)__builtin_amdgcn_readlane((unsigned)(wsp >> 32), dest) << 32 | (unsigned)__builtin_amdgcn_readlane((unsigned)wsp, dest));
    gptr_t fp = (gptr_t)(wsd + (count ? WS_CNT + (unsigned)((NTM + o) * NRANK + rank) * 128u : WS_FLAG + (unsigned)rank * FLAGB + 2048u + (unsigned)((o * 4 + c) * 2 + wid) * 4u));
    while (CFG_P3 && push && o >= CFG_P3N && (int)((unsigned)__builtin_amdgcn_s_memrealtime() - t0) < CFG_P3 + (o - CFG_P3N) * 110) __builtin_amdgcn_s_sleep(1);
    store_w(v, mk_rsrc(push ? wsd : P.wsl, WS_END), (unsigned)lane * 16u + (push ? 0u : (~tk & 1u) * SNDB),
            push ? WS_PAN + (unsigned)(NTM + o) * PANB + (unsigned)rank * CHB : WS_SND + (unsigned)(dest * NB + o) * UNB, job, push);
    WAIT_VMN(0);
    STAMP(P, 6);
    if (lane == 0) {
        if (!count) __hip_atomic_store(fp, tk + 1u, __ATOMIC_RELAXED, __HIP_MEMORY_SCOPE_SYSTEM);
        else if ((__hip_atomic_fetch_add((gptr_t)(P.wsl + WS_LEAD + (o * NRANK + dest) * 64), 1u, __ATOMIC_RELAXED, __HIP_MEMORY_SCOPE_AGENT) & 7u) == 7u)
            __hip_atomic_store(fp, (tk + 1u) * CNT_FULL, __ATOMIC_RELAXED, __HIP_MEMORY_SCOPE_SYSTEM);
    }
}

__device__ __forceinline__ void pack_role(const Params& P, unsigned long long wsp, unsigned tk, unsigned t0, int g, int wid, int lane) {
    const int rank = P.rank;
    const rsrc_t rx = mk_rsrc(P.x, (unsigned)MM * KSH * 2), rw = mk_rsrc(P.w, (unsigned)KSH * NN * 2), rwl = mk_rsrc(P.wsl, WS_END);
    const unsigned sv = (unsigned)lane * 16u + (~tk & 1u) * SNDB;
    const unsigned to = ((const unsigned*)P.order.to)[lane];
    constexpr int NH = CFG_UH < NUNIT ? 8 * DK : 0, UH = NH ? CFG_UH : NUNIT;
    const bool late = g >= NPK - NH;
    constexpr int U1 = NRANK * NL, NE = CFG_EH ? NH : 0;
    static_assert(CFG_TH < (UH - U1 - NH) * CFG_PH, "unpaced");
    for (int u = late ? (NE ? U1 : UH) + g - (NPK - NH) : U1 + NE + g; u < (late ? NUNIT : UH); u += !late ? NPK - NH : u < UH ? UH - U1 : NH) {
        int o, dest = -1;
        if (u < 4 * U0) {
            const int r = u % U0;
            if (r < NRANK * NS) { o = r >> 3; dest = (rank + 1 + r) & (NRANK - 1); } else o = r - (NRANK - 1) * NS;
            o += u / U0 * G0;
        } else o = u - 4 * (NRANK - 1) * NS;
        int q, pos; const int pan = unit_of(o, dest < 0 ? 0 : dest, q, pos);
        while ((int)((unsigned)__builtin_amdgcn_s_memrealtime() - t0) < (late ? CFG_TH + (u - UH) * CFG_PH : ((u - CFG_PACE0) * CFG_PACE - (u > CFG_UK ? (u - CFG_UK) * (CFG_PACE - CFG_PACE2) : 0)) / 10)) __builtin_amdgcn_s_sleep(1);
        const unsigned dst = WS_SND + (unsigned)pos * UNB;
        u32x4 v[8];
        if (pan < NTM) {
            const int rb0 = (wid & 3) * 4;
            const unsigned vo = (unsigned)(lane & 15) * (KSH * 2) + (unsigned)(lane >> 4) * 16u;
            const unsigned so = (unsigned)(pan * 256 + rb0 * 16) * (KSH * 2) + (unsigned)(q * 2 + (wid >> 2)) * 128u;
#pragma unroll
            for (int i = 0; i < 8; ++i) v[i] = __builtin_amdgcn_raw_buffer_load_b128(rx, vo + (i & 1) * 64, so + (unsigned)(i >> 1) * (16 * KSH * 2), 0);
#pragma unroll
            for (int i = 0; i < 8; ++i) STORE16(v[i], rwl, sv + (i & 1) * 1024, dst + (unsigned)(wid >> 2) * KTB + (unsigned)(rb0 + (i >> 1)) * 2048u, AUX_SYS);
        } else {
            load_w(v, rw, gpan_of(pan - NTM, dest < 0 ? 0 : dest) * 256, q, wid, lane);
            store_w(v, rwl, sv, dst, wid, false);
        }
        o = look_up(to, o);
        const bool own = lane == rank;
        unsigned long long fp = wsp + (own ? WS_CNT + (unsigned)((pan * NRANK + rank) * 32 + q) * 4u : WS_FLAG + (unsigned)rank * FLAGB + (unsigned)o * 4u);
        unsigned fv = own ? (tk + 1u) * CNT_FULL : tk + 1u;
        asm volatile("" : "+v"(fp), "+v"(fv));
        WAIT_VMN(0);
        __builtin_amdgcn_s_barrier();
        const bool mine = lane < NRANK && (dest < 0 || dest == lane);
        if (wid == 0 && mine && (own || o < CFG_AGG || o >= NTURN / 8 * 8)) __hip_atomic_store((gptr_t)fp, fv, __ATOMIC_RELAXED, __HIP_MEMORY_SCOPE_SYSTEM);
        if (wid == 1 && o >= CFG_AGG && o < NTURN / 8 * 8) {
            unsigned old = 0;
            if (mine && !own) old = __hip_atomic_fetch_add((gptr_t)(P.wsl + WS_GRP) + (o >> 3) * NRANK + lane, 1u, __ATOMIC_RELAXED, __HIP_MEMORY_SCOPE_AGENT);
            const unsigned full = (unsigned)__ballot((old & 7u) == 7u);
            const unsigned long long wsq = (unsigned long long)(unsigned)__builtin_amdgcn_ds_bpermute(lane >> 3 << 2, (int)(wsp >> 32)) << 32 | (unsigned)__builtin_amdgcn_ds_bpermute(lane >> 3 << 2, (int)wsp);
            if (full >> (lane >> 3) & 1) __hip_atomic_store((gptr_t)(wsq + WS_FLAG + (unsigned)rank * FLAGB) + (o & ~7) + (lane & 7), tk + 1u, __ATOMIC_RELAXED, __HIP_MEMORY_SCOPE_SYSTEM);
        }
        if (wid == 2 && dest >= 0 && pan >= NTM + NB) {
            if (lane == 0 && dest != rank) st_sys((unsigned*)(P.wsl + WS_CNT) + ((NIN + (NTM + NS - 1 - pan) * 7 + ((dest - rank - 1) & 7)) * NRANK + rank) * 32 + q, (tk + 1u) * CNT_FULL);
            unsigned old = 0;
            if (lane == 0) old = __hip_atomic_fetch_add((gptr_t)(P.wsl + WS_BAND), 1u, __ATOMIC_RELAXED, __HIP_MEMORY_SCOPE_AGENT);
            if ((unsigned)__builtin_amdgcn_readfirstlane(old) + 1u == (tk + 1u) * NLAST && lane < NRANK && !own)
                __hip_atomic_store((gptr_t)(wsp + WS_FLAG + (unsigned)rank * FLAGB) + NTURN, tk + 1u, __ATOMIC_RELAXED, __HIP_MEMORY_SCOPE_SYSTEM);
        }
    }
}

struct Tile { int tm, v, half; unsigned sA, stA; };
enum { WHOLE = -1, HEAD = -2, REST = -3 };
__device__ __forceinline__ void tile_of(int idx, int col, int rank, unsigned snd, Tile& t) {
    const int v = idx < 2 * GW ? (idx >> 2) + (col >> 2) * (GW / 2) : 2 * (idx >> 2) + (col >> 2);
    t.tm = 4 * (col & 3) + (idx & 3); t.v = v; t.half = -1;
    const int g = (v - GW) / GM;
    t.sA = snd + (unsigned)(v < GW ? spos_of(v, rank) : 4 * U0 + g * 3 * GM + v - GW) * UNB;
    t.stA = (unsigned)(v < GW ? step_of(v) : g < NGF ? GM : GL) * UNB;
}
__device__ __forceinline__ void item_of(int q, int full, int rank, unsigned snd, Tile& t) {
    const int n = q < full ? q : full + ((q - full) >> 1);
    int at = 0;
    auto from = [&](auto c) { constexpr int l = left_of(c), a = (nt_of(c) - left_of(c + 1)) * 8 + c; at = n >= l ? a : at; };
#define C(i) std::integral_constant<int, i>()
    from(C(0)); from(C(1)); from(C(2)); from(C(3)); from(C(4)); from(C(5)); from(C(6)); from(C(7));
#undef C
    tile_of((at >> 3) + n, at & 7, rank, snd, t);
    if (n >= PN) {
        const int g = (n - PN) / 28, w = (n - PN) % 28, y = (g >> 2) * 7 + (w >> 2);
        t.tm = 4 * (g & 3) + (w & 3); t.v = NWP + y; t.sA = snd + (unsigned)opos_of(y, rank) * UNB; t.stA = NRANK * UNB;
    }
    if (q >= full) t.half = (q - full) & 1;
}

template <int wr>
__device__ __forceinline__ void gemm_role(const Params& P, unsigned call, int col, int slot, int kx, unsigned t0, LAS unsigned char* lds) {
    const int tid = threadIdx.x, wid = __builtin_amdgcn_readfirstlane(tid >> 6), lane = tid & 63, wc = wid & 3;
    const int rank = P.rank;
    char* wsl = P.wsl;
    const rsrc_t rws = mk_rsrc(wsl, WS_END);
    LAS unsigned* ctl = (LAS unsigned*)(lds + STAGE_BYTES);
    const unsigned voff = (unsigned)tid * 16u;
    const unsigned ldsw = (unsigned)wid * 1024u;
    const int aoff = wr * 8192 + lane * 16, boff = wc * 4096 + lane * 16;

    auto lane_now = [&]() { int tz = threadIdx.x; asm volatile("" : "+v"(tz)); return tz & 63; };
    auto cnt_of = [&](const Tile& t) { const int ln = lane_now(); return WS_CNT + (unsigned)((((ln & 32) ? NTM + t.v : t.tm) * NRANK + ((ln >> 2) & 7)) * 32 + (ln & 3)) * 4u; };
    auto level_of = [&](unsigned c) -> int {
        const unsigned long long miss = __ballot((int)(c - call * CNT_FULL) < 0);
        unsigned m = (unsigned)miss | (unsigned)(miss >> 32);
        const bool own = ((m >> (rank * 4)) & 15u) == 0u;
        m |= m >> 16; m |= m >> 8; m |= m >> 4;
        return own ? 1 + __builtin_ctz(m | 16u) : 0;
    };
    auto ready = [&](const Tile& t, int want, bool join = false, bool first = false) -> int {
        if (wr == 0 && wid == 0) {
            const unsigned* c = (const unsigned*)(wsl + cnt_of(t));
            unsigned* tmo = (unsigned*)(wsl + WS_TMO);
            const unsigned long long t0 = __builtin_amdgcn_s_memrealtime();
            if (join)
                while (__builtin_amdgcn_readfirstlane(ld_agent((const unsigned*)(wsl + WS_COL + col * 128))) != call && !give_up(tmo, call, t0)) __builtin_amdgcn_s_sleep(16);
            int got;
            for (;;) {
                got = level_of(ld_sys(c));
                if (got >= want) break;
                if (give_up(tmo, call, t0)) { ctl[0] = 1u; got = 5; break; }
                __builtin_amdgcn_s_sleep(CFG_GS);
            }
            ctl[2] = (unsigned)got;
            if (first) __builtin_amdgcn_fence(__ATOMIC_ACQUIRE, "agent");
            asm volatile("s_waitcnt vmcnt(0) lgkmcnt(0)" ::: "memory");
        }
        asm volatile("" ::: "memory"); __builtin_amdgcn_s_barrier(); asm volatile("" ::: "memory");
        const int r = __builtin_amdgcn_readfirstlane(ctl[2]);
        asm volatile("s_waitcnt lgkmcnt(0)" ::: "memory");
        return r;
    };

#define SA(b, h) (((b) * 2 + (h)) * HTB)
#define SB(b, h) ((4 + (b) * 2 + (h)) * HTB)
#define STAGE(bufoff, soff) do { _Pragma("unroll") for (int _i = 0; _i < 2; ++_i) \
        __builtin_amdgcn_raw_ptr_buffer_load_lds(rws, (LAS unsigned*)(lds + (bufoff) + ldsw + _i * 8192), 16, voff, (soff) + _i * 8192u, 0, CFG_LDA); } while (0)
#define LDA(dst, b, h) do { _Pragma("unroll") for (int m = 0; m < 4; ++m) _Pragma("unroll") for (int k = 0; k < 2; ++k) dst[m][k] = *(const LAS bf16x8*)(lds + SA(b, h) + aoff + m * 2048 + k * 1024); } while (0)
#define LDB(dst, b, h) do { _Pragma("unroll") for (int n = 0; n < 2; ++n) _Pragma("unroll") for (int k = 0; k < 2; ++k) dst[n][k] = *(const LAS bf16x8*)(ldb + (SB(b, h) - SB(0, 0)) + n * 2048 + k * 1024); } while (0)
#define MMA(ai, first, prio, extra) do { __builtin_amdgcn_s_setprio(prio); \
        _Pragma("unroll") for (int i = 0; i < 16; ++i) { \
            const int bj = ((i >> 1) ^ (i >> 2)) & 1, m = i >> 2, n = (i ^ (i >> 2)) & 1, k = i & 1; \
            if (first) asm volatile("v_mfma_f32_16x16x32_bf16 %0, %1, %2, 0\n\tv_mfma_f32_16x16x32_bf16 %0, %3, %4, %0" : "=&v"(acc[ai][bj][m][n]) \
                         : "v"(bj ? B1[n][k] : B0[n][k]), "v"(At[m][k]), "v"(bj ? B1[n][k ^ 1] : B0[n][k ^ 1]), "v"(At[m][k ^ 1])); \
            else asm volatile("v_mfma_f32_16x16x32_bf16 %0, %1, %2, %0\n\tv_mfma_f32_16x16x32_bf16 %0, %3, %4, %0" : "+v"(acc[ai][bj][m][n]) \
                         : "v"(bj ? B1[n][k] : B0[n][k]), "v"(At[m][k]), "v"(bj ? B1[n][k ^ 1] : B0[n][k ^ 1]), "v"(At[m][k ^ 1])); } \
        extra; \
        __builtin_amdgcn_s_setprio(0); } while (0)
#define WAIT_L0() do { asm volatile("" ::: "memory"); __builtin_amdgcn_s_waitcnt(0xC07F); asm volatile("" ::: "memory"); } while (0)
#define WAIT_V(n) asm volatile("s_waitcnt vmcnt(" #n ")" ::: "memory")
#define WAIT_L(n) asm volatile("s_waitcnt lgkmcnt(" #n ")" ::: "memory")
#define BAR __builtin_amdgcn_s_barrier()
#define SCHED __builtin_amdgcn_sched_barrier(0)

    LAS unsigned char* ldb = lds + SB(0, 0) + boff;
    asm volatile("" : "+v"(ldb));
    unsigned long long outp = (unsigned long long)P.outl;
    asm volatile("" : "+v"(outp));
    const int np = CFG_NP >> col * 4 & 15;
    const unsigned snd = WS_SND + (call & 1u) * SNDB;
    auto after = [&](int r) { return r == -1 && slot >= R0 ? 1 : r + 1; };
    auto fixed = [&](int r, Tile& t) {
        const int n = r < 0 ? r + np + PR0 : r;
        tile_of(n ? R0 + (n - 1) * kx + slot : slot, col, rank, snd, t);
        t.half = r < 0 ? HEAD : (unsigned)(r - PR0) < (unsigned)np ? REST : WHOLE;
    };
    Tile cur, nxt; int rnd = after(-np - 1);
    fixed(rnd, cur);
    bf16x8 At[4][2], B0[2][2], B1[2][2];
    constexpr unsigned SB0 = (NRANK * NB - NS + GW) * UNB, STB = UQ * UNB;
    const int l4 = lane < 4 ? 0 : lane - 4;
    const unsigned kov = WS_PAN + (unsigned)((rank + 1 + l4 % 7) & (NRANK - 1)) * CHB + (unsigned)(l4 / 7 & 3) * UNB;
    unsigned tabA, tabB;
    auto tables = [&](const Tile& t) {
        const int ln = lane_now();
        const unsigned la = t.sA + (unsigned)(ln & 3) * t.stA, ra = (unsigned)(NTM + t.v) * PANB + kov;
        const unsigned lb = SB0 + snd + (unsigned)t.tm * UNB + (unsigned)(ln & 3) * STB, rb = (unsigned)t.tm * PANB + kov;
        const int nloc = t.half == HEAD ? 64 : t.half == REST ? 0 : 4;
        tabA = ln < nloc ? la : ra;
        tabB = (ln < nloc ? lb : rb) + (t.half == 1 ? HTB : 0u);
    };
    auto pick = [&]() {
        Tile t = cur; bool ok = true;
        if (rnd + 1 < RS) fixed(after(rnd), t);
        else {
            const unsigned d = (unsigned)__builtin_amdgcn_raw_ptr_buffer_atomic_add_i32(1, rws, lane_now() == 0 ? 0u : 0xfffffff0u, WS_HEAD + (call & 1u) * 128u, 0);
            int m = 0;
            if (NO) {
                const unsigned long long t0 = __builtin_amdgcn_s_memrealtime();
                for (;;) {
                    m = __builtin_amdgcn_readfirstlane(__builtin_amdgcn_raw_buffer_load_b32(rws, 0, WS_FLAG + 63 * 32 + 16, AUX_SYS) - call * 4u);
                    if ((unsigned)m < 4u) break;
                    m = NO;
                    if (give_up((unsigned*)(wsl + WS_TMO), call, t0)) { ctl[0] = 1u; break; }
                    __builtin_amdgcn_s_sleep(CFG_GS);
                }
            }
            const int q = __builtin_amdgcn_readfirstlane(d);
            ok = q < items_of(m);
            if (ok) item_of(q, full_of(m), rank, snd, t);
        }
        *(LAS u32x4*)(ctl + 4) = (u32x4){ok ? (unsigned)(t.tm | t.v << 8 | (t.half + 4) << 16) : 0u, t.sA, t.stA, 0u};
        __builtin_amdgcn_raw_ptr_buffer_load_lds(rws, ctl + 64, 4, cnt_of(t), 0, 0, AUX_SYS);
    };
    auto picked = [&](int& level) -> bool {
        const u32x4 f = *(const LAS u32x4*)(ctl + 4);
        level = level_of(ctl[64 + lane_now()]);
        const unsigned w = __builtin_amdgcn_readfirstlane(f[0]);
        nxt.tm = w & 255u; nxt.v = (w >> 8) & 255u; nxt.half = (int)(w >> 16) - 4;
        nxt.sA = __builtin_amdgcn_readfirstlane(f[1]); nxt.stA = __builtin_amdgcn_readfirstlane(f[2]);
        return w != 0u;
    };
    constexpr int JP = KT / 2 - 3, JL = KT / 2 - 1;
    const int jend = wr == 0 && wid == 0 ? JP : JL;
    while (CFG_HT0 && (int)((unsigned)__builtin_amdgcn_s_memrealtime() - t0) < CFG_HT0 + ((col & 3) * 2 + (col >> 2)) * CFG_HT1) __builtin_amdgcn_s_sleep(8);
    int lvl = ready(cur, 1, rnd == 1, true);
    STAMP(P, 2);
    tables(cur);
    unsigned a2 = cur.sA, b2 = SB0 + snd + (unsigned)cur.tm * UNB;
    STAGE(SB(0, 0), b2); STAGE(SA(0, 0), a2); STAGE(SB(0, 1), b2 + HTB); STAGE(SA(0, 1), a2 + HTB);
    STAGE(SB(1, 0), b2 + KTB); STAGE(SA(1, 0), a2 + KTB); STAGE(SB(1, 1), b2 + HTB + KTB);
    WAIT_V(0); BAR;
#define ROUND_DONE() do {} while (0)
    bool has_next = false;
    const unsigned one = (lane >> 4) == ((lane >> 2) & 3) ? (lane & 1 ? 0x3f800000u : 0x3f80u) : 0u;
    u32x4 sel0 = {lane & 2 ? 0u : one, lane & 2 ? one : 0u, 0u, 0u}, sel1 = {0u, 0u, sel0[0], sel0[1]};
    asm volatile("" : "+v"(sel0), "+v"(sel1));
#pragma unroll 1
    for (;;) {
        f32x4 acc[2][2][4][2];
#define ACC(i) acc[(i) >> 4][((i) >> 3) & 1][((i) >> 1) & 3][(i) & 1]
        unsigned prk = WS_PRK + (unsigned)((rnd < 0 ? rnd + np : rnd - PR0) * GRID + (int)blockIdx.x) * (2 * UNB);
        u32x4 pf[4];
#pragma unroll
        for (int i = 0; i < 4; ++i) asm volatile("" : "=v"(pf[i]));
        int jf = cur.half == REST ? DBG_JF + slot % CFG_FO : 99, je = -1, fe = 0;
#define FOLD2(a, b, p) "v_mfma_f32_16x16x32_bf16 %" #a ", %8, %" #p ", %" #a "\n\tv_mfma_f32_16x16x32_bf16 %" #b ", %9, %" #p ", %" #b "\n\t"
#define FOLD() do { \
            _Pragma("unroll") for (int c = 0; c < 4; ++c) \
                asm volatile("s_cmp_lg_u32 %14, %15\n\ts_cbranch_scc1 1f\n\t" FOLD2(0, 1, 10) FOLD2(2, 3, 11) FOLD2(4, 5, 12) FOLD2(6, 7, 13) "\n1:" \
                    : "+v"(ACC(c * 8)), "+v"(ACC(c * 8 + 1)), "+v"(ACC(c * 8 + 2)), "+v"(ACC(c * 8 + 3)), "+v"(ACC(c * 8 + 4)), "+v"(ACC(c * 8 + 5)), "+v"(ACC(c * 8 + 6)), "+v"(ACC(c * 8 + 7)) \
                    : "v"(sel0), "v"(sel1), "v"(pf[0]), "v"(pf[1]), "v"(pf[2]), "v"(pf[3]), "s"(fe), "n"(c + 1) : "scc"); \
            if (fe < 4) { \
                _Pragma("unroll") for (int i = 0; i < 4; ++i) pf[i] = __builtin_amdgcn_raw_buffer_load_b128(rws, voff + DBG_OOB, prk + i * 8192u, CFG_ILD); \
                prk += 32768u; je = j; } \
            jf += FS; ++fe; } while (0)
        int jchk = lvl < 5 && cur.half != HEAD ? 7 * lvl - 4 : jend;
#define SYNC_MMA(ai, first, wv, extra) do { if (wv) WAIT_V(8); WAIT_L0(); if (wr == 1) BAR; MMA(ai, first, wr + 1, extra); if (wr == 0) BAR; SCHED; } while (0)
#define WAIT_X() asm volatile("s_cmp_eq_u32 %0, %1\n\ts_waitcnt vmcnt(%2)\n\ts_cbranch_scc1 1f\n\ts_waitcnt vmcnt(8)\n1:" :: "s"(j), "s"(je), "n"(12) : "scc", "memory")
#define PASS(first, check, extra) do { \
            LDB(B0, 0, 0); LDA(At, 0, 0); LDB(B1, 0, 1); STAGE(SA(1, 1), a2 + KTB + HTB); SCHED; \
            check; \
            a2 = __builtin_amdgcn_readlane(tabA, (j + 1) & 31); b2 = __builtin_amdgcn_readlane(tabB, (j + 1) & 31); \
            SYNC_MMA(0, first, !first, ); \
            LDA(At, 0, 1); STAGE(SB(0, 0), b2); STAGE(SA(0, 0), a2); STAGE(SB(0, 1), b2 + HTB); \
            SYNC_MMA(1, first, !first, extra); \
            LDB(B0, 1, 0); LDA(At, 1, 0); LDB(B1, 1, 1); STAGE(SA(0, 1), a2 + HTB); \
            if (!first) WAIT_X(); \
            SYNC_MMA(0, false, first, ); \
            LDA(At, 1, 1); STAGE(SB(1, 0), b2 + KTB); STAGE(SA(1, 0), a2 + KTB); STAGE(SB(1, 1), b2 + KTB + HTB); \
            if (!first) WAIT_X(); \
            SYNC_MMA(1, false, first, ); } while (0)
        int j = cur.half == HEAD ? KT / 2 - 4 : cur.half == REST ? 4 : 0;
        PASS(true, , );
#pragma unroll 1
        for (++j; j < KT / 2; ++j)
            PASS(false, if (__builtin_expect(j == jchk, 0)) {
                    if (j < JP) { lvl = ready(cur, lvl + 1); jchk = lvl < 5 ? 7 * lvl - 4 : jend; }
                    else if (wr == 0 && j == JP) { pick(); jchk = JL; }
                    else {
                        has_next = picked(lvl);
                        if (has_next) {
                            const int want = nxt.half >= 0 ? 5 : nxt.half == REST ? 2 : 1;
                            const bool join = rnd == -1 && slot >= R0;
                            if (lvl < want || join) lvl = ready(nxt, want, join);
                            tables(nxt);
                        }
                    }
                }, if (__builtin_expect(j == jf, 0)) FOLD());
        asm volatile("s_nop 7" ::: "memory");
        SCHED;
        if (has_next && nxt.half >= 0) STAGE(SA(1, 1), a2 + KTB + HTB);
        {
            const int ln = lane_now(), fr = ln & 15, fq = ln >> 4;
            const rsrc_t rol = mk_rsrc((const void*)outp, OUTB);
            const bool shared = cur.v < NS;
            if (__builtin_amdgcn_readfirstlane(ctl[0]) != 0u) {
#pragma unroll
                for (int i = 0; i < 32; ++i) acc[i >> 4][(i >> 3) & 1][(i >> 1) & 3][i & 1] = __builtin_bit_cast(f32x4, (u32x4){0x7fc00000u, 0x7fc00000u, 0x7fc00000u, 0x7fc00000u});
            }
            if ((unsigned)rnd < (unsigned)RS && wr == 0 && slot == 0 && wid == 0)
                __builtin_amdgcn_raw_buffer_store_b64((u32x2){call, (unsigned)__builtin_amdgcn_s_memrealtime()}, rws, ln == 0 ? 0u : 0xfffffff0u, WS_COL + (unsigned)(col * 128 + rnd * 8), AUX_SYS);
            if (CFG_TCH && (unsigned)(rnd - CFG_TCH) < 2u && slot * 8 + col < SCOL / 8) {
                unsigned lines = voff; asm volatile("" : "+v"(lines)); lines <<= 2;
                __builtin_amdgcn_raw_ptr_buffer_load_lds(rws, ctl + 128, 4, lines, WS_OSN + (unsigned)((rnd - CFG_TCH) * (SCOL / 8) + slot * 8 + col) * 32768u, 0, 0);
            }
            if (CFG_OTC && rnd == CFG_OTC && slot * 8 + col < NO * 64) {
                unsigned lines = voff; asm volatile("" : "+v"(lines)); lines <<= 2;
                __builtin_amdgcn_raw_ptr_buffer_load_lds(rws, ctl + 128, 4, lines, snd + 4 * UQ * UNB + (unsigned)(slot * 8 + col) * 32768u, 0, 0);
            }
            const bool thru = rnd >= RS;
            const unsigned vo = (unsigned)(wc * 32 + fq * 4) * (NN * 2) + (unsigned)(wr * 256 + fr * 16);
            const unsigned so = (unsigned)(cur.tm * 256) * (NN * 2) + (unsigned)col_of(cur.v, rank) * 2u;
            const unsigned vs = (unsigned)(wc * 32 + fq * 4) * 512u + (unsigned)(wr * 256 + fr * 16);
            const unsigned ss = WS_OSN + (unsigned)(cur.v < NB ? NO * NTM + cur.tm * NB + cur.v : (NS - 1 - cur.v) * NTM + cur.tm) * (2 * UNB);
            auto finish = [&](auto mode) {
#pragma unroll
                for (int bj = 0; bj < 2; ++bj)
#pragma unroll
                    for (int n = 0; n < 2; ++n)
#pragma unroll
                        for (int r = 0; r < 4; ++r) {
                            u32x4 o;
                            if ((r & 1) == 0) {
#pragma unroll
                                for (int i = 0; i < 8; ++i) { f32x4& a = acc[i >> 2][bj][i & 3][n]; const f32x2 z = silu2((f32x2){a[r], a[r + 1]}); a[r] = z[0]; a[r + 1] = z[1]; }
                            }
#pragma unroll
                            for (int i = 0; i < 4; ++i) o[i] = cvt_pk_bf16(acc[i >> 1][bj][(i & 1) * 2][n][r], acc[i >> 1][bj][(i & 1) * 2 + 1][n][r]);
                            if (mode == 1) STORE16(o, rws, vs, ss + (unsigned)(bj * 128 + n * 16 + r) * 512u, AUX_SYS);
                            STORE16(o, rol, vo, so + (unsigned)(bj * 128 + n * 16 + r) * (NN * 2), mode == 2 ? CFG_OTH : CFG_OST);
                        }
            };
            if (rnd < 0) {
#pragma unroll
                for (int i = 0; i < 16; ++i) {
                    const f32x4 a = ACC(2 * i), b = ACC(2 * i + 1);
                    STORE16(((u32x4){cvt_pk_bf16(a[0], a[1]), cvt_pk_bf16(a[2], a[3]), cvt_pk_bf16(b[0], b[1]), cvt_pk_bf16(b[2], b[3])}), rws, voff, prk + i * 8192u, CFG_IST);
                }
            }
            else if (shared) finish(std::integral_constant<int, 1>());
            else if (thru) finish(std::integral_constant<int, 2>());
            else finish(std::integral_constant<int, 0>());
            WAIT_V(16);
            if (shared) {
                WAIT_VM0();
                unsigned old = 0;
                if (ln == 0) old = __hip_atomic_fetch_add(ctl + 1, 1u, __ATOMIC_RELAXED, __HIP_MEMORY_SCOPE_WORKGROUP);
                old = __builtin_amdgcn_readfirstlane(old);
                if ((old & 7u) == 7u) {
                    const unsigned n = (unsigned)__builtin_amdgcn_raw_ptr_buffer_atomic_add_i32(1, rws, ln == 0 ? 0u : 0xfffffff0u, WS_BAND, 0);
                    if ((unsigned)__builtin_amdgcn_readfirstlane(n) + 1u == call * NLAST && ln < NRANK && ln != rank)
                        st_sys((unsigned*)(P.ws[ln] + WS_FLAG + rank * FLAGB) + NTURN, call);
                }
            }
        }
        ROUND_DONE();
        if (!has_next) break;
        cur = nxt; rnd = after(rnd);
        if (cur.half >= 0) break;
    }
#define HB(b, i) ((b) < 2 ? ((i) < 2 ? SA(b, i) : SB(b, 0)) : (i) < 2 ? SB(i, 1) : 8 * HTB)
#define HSTAGE(b, t) do { const unsigned _o = ((t) & 1) * KTB, _a = __builtin_amdgcn_readlane(tabA, ((t) >> 1) & 31) + _o, _b = __builtin_amdgcn_readlane(tabB, ((t) >> 1) & 31) + _o; \
        STAGE(HB(b, 0), _a); STAGE(HB(b, 1), _a + HTB); STAGE(HB(b, 2), _b); } while (0)
#define HLOAD(b) do { _Pragma("unroll") for (int i = 0; i < 4; ++i) B0[i >> 1][i & 1] = *(const LAS bf16x8*)(lds + HB(b, 2) + boff + i * 1024); \
        _Pragma("unroll") for (int i = 0; i < 8; ++i) At[i >> 1][i & 1] = *(const LAS bf16x8*)(lds + HB(b, 0) + aoff + i * 1024); \
        _Pragma("unroll") for (int i = 0; i < 8; ++i) Au[i >> 1][i & 1] = *(const LAS bf16x8*)(lds + HB(b, 1) + aoff + i * 1024); } while (0)
#define HPASS(b, t, first) do { \
        if (wr == 0) { HLOAD(b); HSTAGE((b + 2) % 3, (t) + 2); } else { HSTAGE((b + 2) % 3, (t) + 2); SCHED; HLOAD(b); if (!first) WAIT_V(6); } \
        WAIT_L0(); if (wr == 1) BAR; \
        __builtin_amdgcn_s_setprio(wr + 1); \
        _Pragma("unroll") for (int i = 0; i < 32; ++i) { const int ai = i >> 4, m = (i >> 2) & 3, n = (i >> 1) & 1, k = i & 1; \
            acc[ai][m][n] = __builtin_amdgcn_mfma_f32_16x16x32_bf16(B0[n][k], ai ? Au[m][k] : At[m][k], first && k == 0 ? (f32x4){0.f, 0.f, 0.f, 0.f} : acc[ai][m][n], 0, 0, 0); } \
        __builtin_amdgcn_s_setprio(0); \
        if (wr == 0) { if (!first) WAIT_V(6); BAR; } SCHED; } while (0)
#pragma unroll 1
    while (has_next) {
        f32x4 acc[2][4][2];
        bf16x8 Au[4][2];
        BAR;
        HPASS(0, 0, true);
#pragma unroll 1
        for (int t = 1; t < KT; t += 3) {
            if (wr == 0 && wid == 0 && t == KT - 9) pick();
            HPASS(1, t, false); HPASS(2, t + 1, false); HPASS(0, t + 2, false);
        }
        has_next = picked(lvl);
        if (has_next) {
            (void)ready(nxt, 5);
            tables(nxt);
            HSTAGE(0, 0); HSTAGE(1, 1);
        }
        const int ln = lane_now(), fr = ln & 15, fq = ln >> 4;
        const rsrc_t rol = mk_rsrc((const void*)outp, OUTB);
        if (__builtin_amdgcn_readfirstlane(ctl[0]) != 0u) {
#pragma unroll
            for (int i = 0; i < 16; ++i) acc[i >> 3][(i >> 1) & 3][i & 1] = __builtin_bit_cast(f32x4, (u32x4){0x7fc00000u, 0x7fc00000u, 0x7fc00000u, 0x7fc00000u});
        }
        const unsigned vo = (unsigned)(wc * 32 + fq * 4) * (NN * 2) + (unsigned)(wr * 256 + fr * 16);
        const unsigned so = (unsigned)(cur.tm * 256 + cur.half * 128) * (NN * 2) + (unsigned)col_of(cur.v, rank) * 2u;
#pragma unroll
        for (int n = 0; n < 2; ++n)
#pragma unroll
            for (int r = 0; r < 4; ++r) {
                u32x4 o;
                if ((r & 1) == 0) {
#pragma unroll
                    for (int i = 0; i < 8; ++i) { f32x4& a = acc[i >> 2][i & 3][n]; const f32x2 z = silu2((f32x2){a[r], a[r + 1]}); a[r] = z[0]; a[r + 1] = z[1]; }
                }
#pragma unroll
                for (int i = 0; i < 4; ++i) o[i] = cvt_pk_bf16(acc[i >> 1][(i & 1) * 2][n][r], acc[i >> 1][(i & 1) * 2 + 1][n][r]);
                STORE16(o, rol, vo, so + (unsigned)(n * 16 + r) * (NN * 2), CFG_OTH);
            }
        WAIT_V(8);
        ROUND_DONE();
        cur = nxt; ++rnd;
    }
    WAIT_V(0);
}

__global__ __launch_bounds__(512, 2) void fused_gemm_ar(Params P) {
    extern __shared__ __attribute__((aligned(16))) unsigned char shm[];
    LAS unsigned char* lds = (LAS unsigned char*)shm;
    const int wid = __builtin_amdgcn_readfirstlane(threadIdx.x >> 6), lane = threadIdx.x & 63;
    const int b = __builtin_amdgcn_readfirstlane(blockIdx.x);
    const unsigned long long wsp = (unsigned long long)P.ws[lane & (NRANK - 1)];
    const int from = P.order.from[(threadIdx.x + NPUSH) & 255];
    const unsigned t0 = (unsigned)__builtin_amdgcn_s_memrealtime();
    int rk = P.rank;
    asm volatile("" : "+s"(rk) : "s"(P.x), "s"(P.w), "s"(P.wsl), "s"(P.rank), "s"(t0) : "memory");
    unsigned* tkp = (unsigned*)(P.wsl + WS_TICKET + b * 128 + wid * 16);
    const unsigned tk = ld_agent(tkp);
    if (b < NPULL) {
        pull_role(P, tk, tkp, (P.rank + 1 + b / PW) & (NRANK - 1), b % PW * 8 + wid, lane, from, (LAS u32x4*)lds);
    } else {
        const int g = b - NPULL;
        if (g < 4 * NRANK * NL && wid < 2) lead_role(P, rk, wsp, tk, t0, g, wid, lane);
        ((LAS unsigned*)(lds + STAGE_BYTES))[lane & 3] = 0u;
        pack_role(P, wsp, tk, t0, g, wid, lane);
        const unsigned call = __builtin_amdgcn_readfirstlane(tk) + 1u;
        st_agent(tkp, call);
        STAMP(P, 1);
        asm volatile("s_waitcnt lgkmcnt(0)" ::: "memory");
        __builtin_amdgcn_s_barrier();
        if (b >= GRID - NG) {
            const int col = (CFG_SWAP >> (b & 7) * 4) & 7;
            const int slot = b < NPS ? KX : (b - NPS) >> 3, kx = KX + ((b & 7) >= 8 - (NG - 8 * KX));
            if (wid < 4) gemm_role<0>(P, call, col, slot, kx, t0, lds);
            else gemm_role<1>(P, call, col, slot, kx, t0, lds);
        }
    }
}
}

extern "C" void kernel_launch(int rank, int nranks, ncclComm_t comm,
                              hipStream_t stream,
                              void* const* d_in, const int* in_sizes, int n_in,
                              void* d_out, int out_size,
                              const int* in_dtypes, int out_dtype,
                              void* const* d_out_all,
                              void* const* d_ws_all, size_t ws_bytes) {
    if (nranks != NRANK || (unsigned)rank >= (unsigned)NRANK || ws_bytes < (size_t)WS_END || n_in < 2 ||
        in_sizes[0] != MM * KSH || in_sizes[1] != KSH * NN || out_size != MM * NN) {
        fprintf(stderr, "unexpected problem geometry\n");
        return;
    }
    static bool attr_done[NRANK] = {};
    if (!attr_done[rank]) {
        HIP_CHECK(hipFuncSetAttribute((const void*)fused_gemm_ar, hipFuncAttributeMaxDynamicSharedMemorySize, LDS_BYTES));
        attr_done[rank] = true;
    }
    Params P = {};
    P.order = ORDER;
    P.x = d_in[0]; P.w = d_in[1]; P.rank = rank; P.wsl = (char*)d_ws_all[rank]; P.outl = (char*)d_out_all[rank];
    for (int p = 0; p < NRANK; ++p) { P.out[p] = (char*)d_out_all[p]; P.ws[p] = (char*)d_ws_all[p]; }
    hipLaunchKernelGGL(fused_gemm_ar, dim3(GRID), dim3(512), LDS_BYTES, stream, P);
    HIP_CHECK(hipGetLastError());
}
```

```cpp
#include <hip/hip_runtime.h>
#include <rccl/rccl.h>
#include <cstdio>
#include <type_traits>

#define HIP_CHECK(expr) do { hipError_t _e = (expr); if (_e != hipSuccess) { \
    fprintf(stderr, "HIP error %s at %s:%d\n", hipGetErrorString(_e), __FILE__, __LINE__); \
    return; } } while (0)

#define LAS __attribute__((address_space(3)))
typedef short bf16x8 __attribute__((ext_vector_type(8)));
typedef float f32x4 __attribute__((ext_vector_type(4)));
typedef unsigned u32x4 __attribute__((ext_vector_type(4)));
typedef unsigned u32x2 __attribute__((ext_vector_type(2)));
typedef __amdgpu_buffer_rsrc_t rsrc_t;
typedef __attribute__((address_space(1))) unsigned* gptr_t;

namespace {
constexpr int NRANK = 8;
constexpr int MM = 4096, KK = 4096, NN = 32768, KSH = KK / NRANK;
constexpr int KT = KK / 64, KTS = KSH / 64, NTM = MM / 256, NPAN = NN / 256;
#define CFG_NS 6
#define CFG_PW 5
constexpr int NS = CFG_NS;
#define CFG_NO 2
constexpr int NO = CFG_NO, NB = NS - NO;
constexpr int SCOL = NS * 256;
constexpr int NRP = NPAN - NRANK * NS;
constexpr int NWP = NS + NRP;
constexpr int NIN = NTM + NWP;
constexpr int NPL = NIN + (NRANK - 1) * NO;
constexpr int NTURN = NIN * 4;
constexpr int NUNIT = (NTM + NRANK * NS + NRP) * 4;
constexpr int GRID = 256;
constexpr int PW = CFG_PW;
#define CFG_NPULL 0
#define CFG_NW (PW * 8)
constexpr int NPULL = CFG_NPULL ? CFG_NPULL : (CFG_NW * (NRANK - 1) + 7) / 8;
constexpr int NW = CFG_NPULL ? NPULL * 8 / (NRANK - 1) : CFG_NW;
constexpr int NPK = GRID - NPULL;
constexpr int NPS = (NPULL + 7) / 8 * 8;
constexpr int KX = (GRID - NPS) / 8;
#define CFG_GW (2 * (KX / 4))
constexpr int GW = CFG_GW;
constexpr int G0 = GW + NTM;
#define CFG_SKIP 1
#define CFG_GM (GW + 2)
constexpr int GM = CFG_GM;
constexpr int NGF = (NWP - GW) / GM, GL = (NWP - GW) % GM;
static_assert(NS <= GW && NWP > GW, "shared panels sit in the first group");
constexpr int PU = 4, PSH = 2;
#define CFG_PUSH 2
constexpr int NPUSH = CFG_PUSH;
constexpr int NPIN = (NTURN - NPUSH) * 64 / PU;
#define CFG_NL (NPUSH + NW * PU / 64 < NB ? NPUSH + NW * PU / 64 : NB)
constexpr int NL = CFG_NL;
static_assert(NPUSH <= NL && NL <= NB, "leading units have a single consumer");
constexpr int OPB = MM / 2 / PU, OSK = OPB / NRANK;
constexpr int OIB = OPB - OSK, NPI2 = NPIN + NO * OIB, NPO = NPI2 + NO * OPB;
constexpr int BPB = NB * 128 / PU;
constexpr int NPC = NPO + NTM * BPB;
constexpr unsigned CNT_FULL = 64;

#define CFG_SWAP 0x57461302u
#define CFG_XTRA 1
constexpr int NG = CFG_XTRA ? NPK : 8 * KX;
constexpr int DK = CFG_SKIP ? KX - 2 * GW : 0, R0 = KX - DK;
constexpr int nt_of(int col) { return 4 * ((NWP + 1 - (col >> 2)) / 2); }
constexpr int kx_of(int col) { int n = 0; for (int b = GRID - NG; b < GRID; ++b) n += (int)(CFG_SWAP >> (b & 7) * 4 & 7) == col; return n; }
#define CFG_RS ((nt_of(7) - 1 - R0) / (KX + (NG > 8 * KX)) + 1)
constexpr int RS = CFG_RS;
#define CFG_NP 0x55435543u
#define CFG_PR0 2
constexpr int PR0 = CFG_PR0;
constexpr int np_max() { int n = 0; for (int c = 0; c < 8; ++c) n = (int)(CFG_NP >> c * 4 & 15) > n ? CFG_NP >> c * 4 & 15 : n; return n; }
static_assert(PR0 >= 1 && np_max() < RS, "only statically assigned tiles are started early");
__device__ constexpr int pr0_of(int np) { return RS - np < PR0 ? RS - np : PR0; }
constexpr int left_of(int col) { int n = 0; for (int c = 0; c < col; ++c) n += nt_of(c) - R0 - (RS - 1) * kx_of(c); return n; }
constexpr int PN = left_of(8);
#define CFG_L2 17
#define CFG_L3A 14
#define CFG_L3D 1
#define CFG_L3B 11
constexpr int OT = (NRANK - 1) * NTM;
__device__ constexpr int last_of(int m) { return (PN + m * OT) % NG; }
__device__ constexpr int cut_of(int m) { return 2 * last_of(m) > NG ? NG - last_of(m) : last_of(m); }
__device__ constexpr int full_of(int m) { return PN + m * OT - cut_of(m); }
__device__ constexpr int three_of(int m) { return 2 * last_of(m) > NG ? 0 : 3 * last_of(m) > NG ? NG - 2 * last_of(m) : last_of(m); }
__device__ constexpr int two_of(int m) { return cut_of(m) - three_of(m); }
__device__ constexpr int items_of(int m) { return full_of(m) + 2 * two_of(m) + 3 * three_of(m); }
__device__ __forceinline__ int piece_of(int z, int two, int three, int& u0, int& u1, int& p) {
    const int y = z - 2 * two, g = y >= 2 * three ? 2 : y >= three, w = g == 2 ? 3 * three - 1 - y : y - g * three;
    const int mid = CFG_L3B + CFG_L3A + (w <= KX ? CFG_L3D : w > 2 * KX ? -CFG_L3D : 0);
    p = y >= 0 ? g + (g >> 1) : z >= two ? 2 : 0;
    u0 = p == 2 ? CFG_L2 : p == 3 ? mid : p == 0 && y >= 0 ? CFG_L3B : 0;
    u1 = p >= 2 ? KT / 2 : p == 1 ? CFG_L3B : y >= 0 ? mid : CFG_L2;
    return y >= 0 ? two + w : z >= two ? z - two : z;
}
static_assert(PN + NO * OT < 2 * NG && CFG_L2 >= 4 && CFG_L2 <= 28 && CFG_L3A - CFG_L3D >= 4 && CFG_L3B >= 4 && CFG_L3A + CFG_L3B + CFG_L3D <= 28, "pieces");
constexpr unsigned KTB = 32768, HTB = 16384;
constexpr unsigned PANB = KT * KTB;
constexpr unsigned CHB = KTS * KTB;
constexpr unsigned UNB = 2 * KTB;
constexpr unsigned WS_TICKET = 0;
constexpr unsigned WS_TMO = WS_TICKET + GRID * 128;
constexpr unsigned WS_FLAG = WS_TMO + 128;
constexpr unsigned FLAGB = 4096;
constexpr unsigned WS_LEAD = WS_FLAG + NRANK * FLAGB;
constexpr unsigned WS_HEAD = WS_LEAD + NRANK * 512;
constexpr unsigned WS_COL = WS_HEAD + NRANK * 128;
#define CFG_PHOFF 0
constexpr unsigned WS_PH = WS_COL + 8 * 128 + CFG_PHOFF;
constexpr unsigned WS_ZC = WS_PH + 2 * NRANK * 128 + 128;
constexpr unsigned WS_GRP = WS_ZC + NRANK * NRANK * 128;
constexpr unsigned WS_BAND = WS_GRP + 64 * NRANK * 4;
constexpr unsigned WS_PCF = WS_BAND + 128;
constexpr unsigned WS_CNT = WS_PCF + GRID * 4;
constexpr unsigned WS_CTRL_END = WS_CNT + NPL * NRANK * 128;
constexpr unsigned WS_DBG = 512u << 10;
constexpr unsigned WS_SND = 1u << 20, SNDB = NUNIT * UNB;
constexpr unsigned WS_OSN = WS_SND + NUNIT * UNB + SNDB;
constexpr unsigned WS_PAN = (WS_OSN + MM * SCOL * 2 + PANB - 1) / PANB * PANB;
constexpr unsigned WS_PRK = WS_PAN + NPL * PANB;
constexpr unsigned WS_END = WS_PRK + (np_max() + 1) * NG * 2 * UNB;
constexpr int U0 = G0 + (NRANK - 1) * NS;
constexpr int UQ = U0 - NRANK * NO;
constexpr unsigned OUTB = (unsigned)MM * NN * 2;
static_assert(WS_CTRL_END <= WS_DBG, "ctrl region");
constexpr unsigned NLAST = NS * NTM + NO * NRANK * 4;
#define CFG_WIN 88
constexpr int BASE = (CFG_WIN - 1) / NW, DEEP = CFG_WIN - BASE * NW;
static_assert(BASE == 2, "window");
constexpr unsigned STAGE_BYTES = 9 * HTB;
constexpr unsigned LDS_BYTES = STAGE_BYTES + 768;
constexpr unsigned long long DEADLINE = 50000000ull;
constexpr int AUX_SYS = 17, AUX_SYS_NT = 19, AUX_AGENT = 16;
#define CFG_PACE 40
#define CFG_UK 1000
#define CFG_PACE2 40
#define CFG_UH (2 * (NPK - 8 * DK))
#define CFG_TH 1200
#define CFG_PH 12
#define CFG_OST 2
#define CFG_OTH AUX_SYS_NT
#define CFG_IST 2
#define DBG_JF 5
#define DBG_OOB 0u
#define CFG_FO 5
constexpr int FS = 5;
#define CFG_HT1 150
#define CFG_HT0 0
#define CFG_ILD 2
#define CFG_PST AUX_AGENT
#define CFG_LDA 0
#define CFG_RLD AUX_SYS
#define CFG_TCH 3
#define CFG_GS 16
#define CFG_AGG 24
static_assert(CFG_AGG % 8 == 0 && CFG_AGG >= NS && NS <= 8 && NPI2 / 16 <= 512 && NIN * 4 <= 512, "groups");
#define CFG_PWND 3
#define CFG_EH 1
#define CFG_MIX 1
static_assert(4 * NRANK * NL <= NPK && (NL - NPUSH) * 64 / PU <= NW && NL * 64 <= 512, "leading units");
#define CFG_PACE0 40
#define CFG_P3 320
#define CFG_P3N 2
#define CFG_LOOK 104
#define CFG_TAIL NW
#define CFG_EVEN 280
#define CFG_EVEND 200
#define CFG_ERATE CFG_RATE
#define CFG_ZB 112
#define CFG_LOOKP 100
#define CFG_LSP 0
#define CFG_LSP2 0
#define CFG_PLAN (CFG_ZB + CFG_LOOKP + NW * CFG_LSP2 + 50)
#define CFG_DEC (CFG_LOOKP + NW * CFG_LSP + 80)
#define CFG_QR 0.915f
#define CFG_QH 0.16f
#define CFG_LT 1712
#define CFG_MARG 550.0f
#define CFG_BAL 8
constexpr int ZB = CFG_ZB;
static_assert(ZB <= 127 && ZB <= BPB && ZB % 16 == 0 && NTURN < 504 && NW == PW * 8, "plan");
#define CFG_PWIN 4u
#define CFG_PS 2
#define CFG_RATE 1680
#define CFG_CRED 41
#define CFG_GRID 36
#define CFG_EPS 0
#define CFG_SPIN 8
constexpr unsigned PACE16 = CFG_RATE * CFG_WIN / 64;
#define CFG_NZ 16
#define CFG_OTN 1
#define CFG_OTC 3
#define CFG_PAT 4
#define CFG_ROT 1
#define CFG_DRY 0
#define CFG_GD 8
#define CFG_GS2 8
#define CFG_PRE 1
constexpr int ops_after(int k, int shallow) { return (k == 0 ? 5 : k == 2 ? 2 : 1) + shallow; }
constexpr int ops_since(int k, int back, int shallow) { int n = back - 1; for (int i = 1; i <= back; ++i) n += ops_after((k - i) & 3, shallow); return n; }
static_assert(ops_since(0, 3, 0) <= ops_since(0, 2, 1), "early announcements");

#define CFG_EA 8
#define CFG_EB 4
#define CFG_ER 80
#define CFG_ES 17
#define CFG_EMIN 16
constexpr int NTAB = 4 * (G0 + GM);
static_assert(NTAB <= 256, "byte tables");
struct Order {
    unsigned char from[256], to[256];
    static constexpr int due(int o) {
        const int d = o >= 4 * G0 ? CFG_ER + (o - 4 * G0) / GM * CFG_ES + ((o - 4 * G0) % GM & 1) * CFG_EB
                                  : o / G0 * CFG_ES + (o % G0 < GW ? o % G0 / (GW / 2) * CFG_EB : (o % G0 - GW) / 4 * CFG_EA);
        return d < CFG_EMIN ? CFG_EMIN : d;
    }
    constexpr Order() : from(), to() {
        for (int o = 0; o < NTAB; ++o) {
            int n = 0;
            for (int k = 0; k < NTAB; ++k) n += due(k) < due(o) || (due(k) == due(o) && k < o);
            to[o] = (unsigned char)n; from[n] = (unsigned char)o;
        }
    }
};
constexpr Order ORDER;
static_assert(ORDER.to[NS - 1] == NS - 1, "the leading units come first");

struct Params {
    const void* x; const void* w;
    char* wsl; char* outl;
    int rank, pad[7];
    char* ws[NRANK]; char* out[NRANK];
    Order order;
};
__device__ __forceinline__ int look_up(unsigned tab, int o) { return o < NTAB ? (unsigned)__builtin_amdgcn_readlane(tab, o >> 2) >> (o & 3) * 8 & 255u : o; }

__device__ __forceinline__ rsrc_t mk_rsrc(const void* p, unsigned bytes) {
    const unsigned long long v = (unsigned long long)p;
    const unsigned lo = __builtin_amdgcn_readfirstlane((unsigned)v), hi = __builtin_amdgcn_readfirstlane((unsigned)(v >> 32));
    return __builtin_amdgcn_make_buffer_rsrc((void*)(((unsigned long long)hi << 32) | lo), 0, bytes, 0x00020000);
}
__device__ __forceinline__ u32x4 mk_words(const void* p, unsigned bytes) {
    const unsigned long long v = (unsigned long long)p;
    return (u32x4){(unsigned)__builtin_amdgcn_readfirstlane((unsigned)v), (unsigned)__builtin_amdgcn_readfirstlane((unsigned)(v >> 32)) & 0xffffu, bytes, 0x00020000u};
}
__device__ u32x4 buffer_load16(u32x4 rsrc, unsigned voff, unsigned soff, int aux) __asm("llvm.amdgcn.raw.buffer.load.v4i32");
__device__ unsigned buffer_swap(unsigned v, rsrc_t rsrc, unsigned voff, unsigned soff, int aux) __asm("llvm.amdgcn.raw.ptr.buffer.atomic.swap.i32");
__device__ __forceinline__ unsigned ld_sys(const unsigned* p) { return __hip_atomic_load(p, __ATOMIC_RELAXED, __HIP_MEMORY_SCOPE_SYSTEM); }
__device__ __forceinline__ unsigned ld_agent(const unsigned* p) { return __hip_atomic_load(p, __ATOMIC_RELAXED, __HIP_MEMORY_SCOPE_AGENT); }
__device__ __forceinline__ void st_sys(unsigned* p, unsigned v) { __hip_atomic_store(p, v, __ATOMIC_RELAXED, __HIP_MEMORY_SCOPE_SYSTEM); }
__device__ __forceinline__ void st_agent(unsigned* p, unsigned v) { __hip_atomic_store(p, v, __ATOMIC_RELAXED, __HIP_MEMORY_SCOPE_AGENT); }
__device__ __forceinline__ void add_agent(unsigned* p, unsigned v) { (void)__hip_atomic_fetch_add(p, v, __ATOMIC_RELAXED, __HIP_MEMORY_SCOPE_AGENT); }
#define STORE16(v, rs, vo, so, aux) do { const u32x4 _d = (v); __builtin_amdgcn_raw_buffer_store_b128(_d, rs, vo, so, aux); asm volatile("s_nop 1" :: "v"(_d) : "memory"); } while (0)
#define WAIT_VM0() asm volatile("s_waitcnt vmcnt(0)" ::: "memory")
#define WAIT_VMN(n) do { asm volatile("" ::: "memory"); __builtin_amdgcn_s_waitcnt(0x0F70 | (n)); asm volatile("" ::: "memory"); } while (0)

__device__ __forceinline__ bool give_up(unsigned* tmo, unsigned call, unsigned long long t0) {
    const unsigned long long dt = __builtin_amdgcn_s_memrealtime() - t0;
    if (dt < 2000ull) return false;
    if ((unsigned)__builtin_amdgcn_readfirstlane(ld_agent(tmo)) == call) return true;
    if (dt > DEADLINE) { st_agent(tmo, call); return true; }
    return false;
}

__device__ __forceinline__ int gpan_of(int v, int consumer) { return v < NS ? consumer * NS + v : (NRANK - 1) * NS + v; }
__device__ __forceinline__ int opan_of(int y, int consumer) { return ((consumer + 1 + y % 7) & 7) * NS + NS - 1 - y / 7; }
__device__ __forceinline__ int spos_of(int v, int consumer) { return v < NB ? consumer * NB + v : v < NS ? 4 * UQ + (NS - 1 - v) * 4 * NRANK + consumer : NRANK * NB - NS + v; }
__device__ __forceinline__ int step_of(int v) { return v >= NB && v < NS ? NRANK : UQ; }
__device__ __forceinline__ int opos_of(int y, int consumer) { return 4 * UQ + y / 7 * 4 * NRANK + ((consumer + 1 + y % 7) & 7); }
__device__ __forceinline__ int col_of(int v, int consumer) { return (v < NWP ? gpan_of(v, consumer) : opan_of(v - NWP, consumer)) * 256; }
__device__ __forceinline__ int band_of(int e, int consumer) { return e < 7 ? (consumer + 1 + e) & 7 : e < 15 ? 8 + ((consumer + e) & 7) : consumer; }
__device__ __forceinline__ int unit_of(int o, int consumer, int& q, int& pos) {
    if (o < 4 * G0) { q = o / G0; const int m = o % G0; pos = q * step_of(m) + spos_of(m, consumer); return m < GW ? NTM + m : m - GW; }
    pos = o + 4 * (NRANK - 1) * NS;
    o -= 4 * G0;
    const int g = o / (4 * GM), r = o % (4 * GM);
    int i;
    if (GL == 0 || g < NGF) { q = r / GM; i = r % GM; } else { q = r / (GL ? GL : 1); i = r % (GL ? GL : 1); }
    return NTM + GW + GM * g + i;
}

__device__ __forceinline__ unsigned cvt_pk_bf16(float lo, float hi) { unsigned r; asm("v_cvt_pk_bf16_f32 %0, %1, %2" : "=v"(r) : "v"(lo), "v"(hi)); return r; }
__device__ __forceinline__ float silu(float y) {
    const float c = __builtin_amdgcn_fmed3f(y, -60.0f, 60.0f);
    const float e = __builtin_amdgcn_exp2f(c * -1.44269504088896f);
    return y * __builtin_amdgcn_rcpf(1.0f + e);
}

typedef float f32x2 __attribute__((ext_vector_type(2)));
__device__ __forceinline__ f32x2 silu2(f32x2 y) {
    const f32x2 t = __builtin_elementwise_fma(y, (f32x2){-1.44269504088896f, -1.44269504088896f}, (f32x2){-86.5617024533378f, -86.5617024533378f});
    const f32x2 e = {__builtin_amdgcn_fmed3f(__builtin_amdgcn_exp2f(t[0]), 0.0f, 1.0f), __builtin_amdgcn_fmed3f(__builtin_amdgcn_exp2f(t[1]), 0.0f, 1.0f)};
    const f32x2 d = __builtin_elementwise_fma(e, (f32x2){1.1420073898156842e26f, 1.1420073898156842e26f}, (f32x2){1.0f, 1.0f});
    return y * (f32x2){__builtin_amdgcn_rcpf(d[0]), __builtin_amdgcn_rcpf(d[1])};
}

#define STAMP1(P, slot) do {} while (0)
#define STAMP(P, slot) do {} while (0)

template <class T> __device__ __forceinline__ T now_s(T x) { asm volatile("" : "+s"(x)); return x; }
__device__ __forceinline__ int now_v(int x) { asm volatile("" : "+v"(x)); return x; }
__device__ __forceinline__ void pull_role(const Params& P, unsigned tk, unsigned* tkp, int peer, int widx, int lane, int from, LAS u32x4* tab) {
    const int rank = P.rank;
    char* wsl = P.wsl;
    const rsrc_t rwl = mk_rsrc(wsl, WS_END), rol = mk_rsrc(P.outl, OUTB);
    const u32x4 rwr = mk_words(P.ws[peer], WS_END), ror = mk_words(P.out[peer], OUTB);
    {
        const int y = threadIdx.x, x = y - NO * 32, r = x % (2 * NB), band = band_of(x / (2 * NB), rank);
        const int v = x < 0 ? NS - 1 - y / 32 : r % NB, half = x < 0 ? y % 32 : band * 2 + r / NB, at = x < 0 ? y : NO * 32 + (band * NB + v) * 2 + r / NB;
        tab[NPI2 / 16 + y] = (u32x4){WS_OSN + (unsigned)at * UNB, (unsigned)half * (128u * NN * 2u) + (unsigned)(peer * NS + v) * 512u,
                                     x >= 0 && x < (NRANK - 1) * 2 * NB && r >= 2 * NB - ZB / 16 ? WS_ZC + (unsigned)(band * NRANK + peer) * 128u : 0u, 0u};
    }
    auto input = [&](int pan, int q, int pos, unsigned c) {
        u32x4 t = {WS_SND + (unsigned)pos * UNB, WS_PAN + (unsigned)pan * PANB + (unsigned)peer * CHB + (unsigned)q * UNB, WS_CNT + (unsigned)(pan * NRANK + peer) * 128u + (unsigned)q * 4u, 0u};
        asm volatile("" : "+v"(t), "+v"(c));
        t[0] += (~c & 1u) * SNDB;
        return t;
    };
    if (threadIdx.x < NPI2 / 16) {
        int q, pos;
        const int e = now_v(threadIdx.x);
        int pan = unit_of(e + NPUSH < NTAB ? from : e + NPUSH, rank, q, pos);
        if (NO && e >= NPIN / 16) {
            const int z = e - NPIN / 16, y = z / 28 * 7 + (z % 7 + 7 - rank) % 7;
            q = z / 7 & 3; pan = NIN + y; pos = q * NRANK + opos_of(y, rank);
        }
        tab[e] = input(pan, q, pos, tk);
    }
    const unsigned call = __builtin_amdgcn_readfirstlane(now_v(tk)) + 1u;
    st_agent(tkp, call);
    unsigned* tmo = (unsigned*)(wsl + WS_TMO);
    constexpr unsigned OOB = 0xfffffff0u;
    const unsigned lane0 = lane == 0 ? 0u : OOB;
    const unsigned hoff = WS_PH + (call & 1u) * (NRANK * 128u);
    const unsigned head = lane == 0 ? hoff + (unsigned)peer * 128u : OOB;
    unsigned dvo = CFG_DRY ? OOB : head;
    auto draw = [&]() { return (unsigned)__builtin_amdgcn_raw_ptr_buffer_atomic_add_i32(1, rwl, dvo, 0, 0); };
    __builtin_amdgcn_raw_buffer_store_b32(0u, rwl, widx ? OOB : lane == 0 ? WS_PH + (~call & 1u) * (NRANK * 128u) + (unsigned)peer * 128u : lane == 1 ? WS_HEAD + (~call & 1u) * 128u : OOB, 0, AUX_SYS);
    auto arrive = [&](unsigned vo, unsigned cnt) { (void)__builtin_amdgcn_raw_ptr_buffer_atomic_add_i32(PU, rwl, vo, cnt, 0); };
    const int wrk = (widx & 7) * (NW / 8) + (widx >> 3);
    const bool deep0 = wrk < DEEP;
    const unsigned per0 = deep0 ? PACE16 / (BASE + 1) : PACE16 / BASE;
    asm volatile("s_waitcnt lgkmcnt(0)" ::: "memory");
    __builtin_amdgcn_s_barrier();
    bool bad = false; int stamped = 0; (void)stamped;
    int skip = NO ? -1 : 0, nout = NPIN;
    auto index_of = [&](unsigned v) { const int i = __builtin_amdgcn_readfirstlane(v) + NW; return i >= nout && skip > 0 ? i + skip : i; };
    int cur = index_of((unsigned)(widx - NW));
    const unsigned fso = WS_FLAG + (unsigned)peer * FLAGB;
    auto first_of = [&](int i) { return i < NPIN ? (i >> (6 - PSH)) + NPUSH : NTURN; };
    int lim = 0, flim = 0, fin = NPC;
    bool planned = ZB == 0;
    unsigned long long plan = 0;
    auto moved = [&](int j) { return (int)(signed char)(plan >> (j * 8)); };
    auto update = [&](const u32x4& f0, const u32x4& f1) {
        const int first = first_of(cur) - lane * 8;
        unsigned un = 0;
#pragma unroll
        for (int j = 0; j < 8; ++j) un |= ((j < 4 ? f0[j & 3] : f1[j & 3]) - call) >> 31 << j;
        un &= 255u << (first < 0 ? 0 : first > 8 ? 8 : first);
        const int l = __builtin_ctzll(__ballot(un != 0u) | (1ull << 63));
        const int u = l * 8 + __builtin_ctz(__builtin_amdgcn_readlane(un, l) | 256u);
        const int fl = u <= NTURN ? (u - NPUSH) << (6 - PSH) : NPC + 128;
        flim = fl > flim ? fl : flim;
        if (!planned && (unsigned)__builtin_amdgcn_readlane(f0[0], 63) == call && (unsigned)__builtin_amdgcn_readlane(f0[2], 63) == call) {
            plan = (unsigned long long)(unsigned)__builtin_amdgcn_readlane(f0[3], 63) << 32 | (unsigned)__builtin_amdgcn_readlane(f0[1], 63);
            planned = true; fin = NPC - moved(peer);
        }
        if (skip < 0 && (unsigned)__builtin_amdgcn_readlane(f1[0], 63) - call * 4u < 4u) {
            const int m = __builtin_amdgcn_readlane(f1[0], 63) & 3;
            skip = NO * OIB + m * OSK; nout = NPIN + m * OIB;
            if (cur >= nout) cur += skip;
        }
        const int pl = skip < 0 ? NPIN : planned ? fin : NPC - ZB;
        lim = flim < pl ? flim : pl;
    };
    auto make_plan = [&](int lane, int rank, unsigned call, char* wsl) {
        unsigned old = 0;
        if (lane == 0) old = __hip_atomic_exchange((gptr_t)(wsl + WS_PH + 2 * NRANK * 128), call, __ATOMIC_RELAXED, __HIP_MEMORY_SCOPE_AGENT);
        if ((unsigned)__builtin_amdgcn_readfirstlane(old) == call) return;
        const unsigned hd = __builtin_amdgcn_raw_buffer_load_b32(rwl, lane < NRANK ? (unsigned)lane * 128u : OOB, hoff, AUX_SYS);
        const int t = lane >> 3, s = lane & 7;
        const unsigned zc = ld_sys((const unsigned*)(P.ws[t] + WS_ZC + (rank * NRANK + s) * 128));
        const bool ok = __all(t == rank || s == rank || s == t || (int)(zc - call * (unsigned)(ZB * PU)) >= 0);
        int h[NRANK], sum = 0, give = 0, take = 0;
#pragma unroll
        for (int j = 0; j < NRANK; ++j) { h[j] = __builtin_amdgcn_readlane(hd, j); sum += j == rank ? 0 : h[j]; }
#pragma unroll
        for (int j = 0; j < NRANK; ++j) {
            const int d = j == rank ? 0 : (sum - (NRANK - 1) * h[j]) * CFG_BAL / (8 * (NRANK - 1));
            h[j] = d > ZB ? ZB : d < -127 ? -127 : d;
            if (h[j] > 0) give += h[j]; else take -= h[j];
        }
        give = take = !ok ? 0 : give < take ? give : take;
        unsigned long long pl = 0;
#pragma unroll
        for (int j = 0; j < NRANK; ++j) {
            int d = h[j];
            if (d > 0) { d = d < give ? d : give; give -= d; } else { d = -d < take ? d : -take; take += d; }
            pl |= (unsigned long long)(d & 255) << (j * 8);
        }
        __builtin_amdgcn_raw_buffer_store_b128((u32x4){call, (unsigned)pl, call, (unsigned)(pl >> 32)}, rwl, lane < NRANK ? WS_FLAG + 63 * 32 + (unsigned)lane * FLAGB : OOB, 0, AUX_SYS);
    };
    auto decide = [&](int lane, int rank, unsigned call, char* wsl) {
        unsigned old = 0;
        if (lane == 0) old = __hip_atomic_exchange((gptr_t)(wsl + WS_PH + 2 * NRANK * 128 + 64), call, __ATOMIC_RELAXED, __HIP_MEMORY_SCOPE_AGENT);
        if ((unsigned)__builtin_amdgcn_readfirstlane(old) == call) return;
        const int c = lane >> 3, r = lane & 7, np = CFG_NP >> c * 4 & 15;
        const u32x4 st = __builtin_amdgcn_raw_buffer_load_b128(rwl, (unsigned)(c * 128 + r * 8), WS_COL, AUX_SYS);
        const unsigned now = (unsigned)__builtin_amdgcn_s_memrealtime();
        const unsigned long long have = __ballot(r >= 1 && r < RS - 1 && st[0] == call && st[2] == call);
        auto shorter = [&](int to) { const int n = to - pr0_of(np) + 1; return n < 0 ? 0 : n > np ? np : n; };
        const float tau = (float)(st[3] - st[1]) * (shorter(r + 1) != shorter(r) ? 1.0f / CFG_QR : 1.0f);
        float arr = (float)(int)(st[3] - now) + tau * (shorter(r + 2) != shorter(r + 1) ? CFG_QR : 1.0f);
        arr = (arr < 0.f ? 0.f : arr) + tau * ((float)(RS - 3 - r) - (1.0f - CFG_QR) * (float)(np - shorter(r + 2)));
        if (r + 2 == RS) arr = (float)(int)(st[3] - now);
        const bool last = (have >> lane & 1) && !(have >> lane & 2);
        int m = 0;
        if (__builtin_popcountll(__ballot(last)) == 8) {
            int before = 0, after = 0;
            unsigned long long b = __ballot(last);
#pragma unroll
            for (int k = 0; k < 8; ++k) {
                const int l = __builtin_ctzll(b); b &= b - 1;
                const float a = __builtin_bit_cast(float, __builtin_amdgcn_readlane(__builtin_bit_cast(int, arr), l)), t = __builtin_bit_cast(float, __builtin_amdgcn_readlane(__builtin_bit_cast(int, tau), l));
                before += a < arr ? NG : 0; after += a + t < arr + tau ? NG : 0;
            }
#pragma unroll
            for (int k = 1; k <= NO; ++k) {
                const bool whole = before < 8 * full_of(k);
                const int at = ((whole ? after + 8 * NG : before) >> 3) - full_of(k);
                int u0, u1, p;
                (void)piece_of(at, two_of(k), three_of(k), u0, u1, p);
                const float end = arr + tau * ((whole ? 1.0f : 0.0f) + (at < items_of(k) - full_of(k) ? (float)(u1 - u0) * (0.95f / 32) + CFG_QH : 0.0f));
                if (!__any(last && end > (float)(CFG_DEC + NPC - NPI2 - (k - 1) * OSK) * (CFG_LT / 256.0f) - CFG_MARG)) m = k;
            }
        }
        const unsigned full = call * CNT_FULL;
#pragma unroll
        for (int k = 0; k < NO; ++k)
            __builtin_amdgcn_raw_buffer_store_b128((u32x4){full, full, full, full}, rwl, k >= m && lane < 56 && r != rank ? (unsigned)lane * 128u : OOB, WS_CNT + (unsigned)(NIN + 7 * k) * (NRANK * 128u), AUX_SYS);
        __builtin_amdgcn_raw_buffer_store_b32(call * 4u + (unsigned)m, rwl, lane < NRANK ? WS_FLAG + 63 * 32 + 16 + (unsigned)lane * FLAGB : OOB, 0, AUX_SYS);
    };
    rsrc_t rd = rwl;
    u32x4 rs = rwr;
    const int rot = CFG_ROT == 1 ? wrk : CFG_ROT == 2 ? widx : CFG_ROT == 3 ? widx >> 1 : 0;
    auto piece = [&](int k) -> unsigned {
        k = k + rot & 3;
        const int q = lane >> 4;
        return CFG_PAT == 0 ? lane * 16 + k * 1024 : CFG_PAT == 2 ? q * 1024 + (k + q & 3) * 256 + (lane & 15) * 16
             : CFG_PAT == 4 ? (lane >> 3) * 512 + k * 128 + (lane & 7) * 16
             : CFG_PAT == 8 ? (lane >> 2) * 256 + k * 64 + (lane & 3) * 16
             : CFG_PAT == 9 ? (lane >> 3) * 512 + (k + (lane >> 3) & 3) * 128 + (lane & 7) * 16
             : (lane >> 3) * 512 + ((k & 1) * 2 + (k >> 1)) * 128 + (lane & 7) * 16;
    };
    unsigned o[PU], in[PU], rows[PU], src, dst, cb, dsh = 12, dadd = 0, per = per0, fvo = CFG_DRY ? OOB - 16u : (unsigned)lane * 32u;
#pragma unroll
    for (int k = 0; k < PU; ++k) { o[k] = in[k] = CFG_DRY ? OOB : piece(k); rows[k] = (piece(k) & 511u) + (piece(k) >> 9) * (NN * 2u); }
    bool tail = CFG_DRY, deep = deep0, look = true;
    int next = CFG_DRY ? NPC : 0, ti;
    auto places = [&]() {
        const u32x4 t = tab[ti >> 4];
        dst = __builtin_amdgcn_readfirstlane(t[1]) + ((ti & 15) << dsh) + dadd;
        src = ZB && cur >= NPC ? dst : __builtin_amdgcn_readfirstlane(t[0]) + ((ti & 15) << 12);
        cb = __builtin_amdgcn_readfirstlane(t[2]);
    };
    auto turn = [&]() {
        if (ZB && cur == NPC - CFG_PLAN) make_plan(now_v(lane), now_s(rank), now_s(call), now_s(wsl));
        if (NO && cur == NPIN - CFG_DEC) decide(now_v(lane), now_s(rank), now_s(call), now_s(wsl));
        const int even = fin - (deep0 ? CFG_EVEND : CFG_EVEN);
        tail = cur >= fin - CFG_TAIL; deep = deep0 || tail || cur >= even;
        per = cur >= even ? CFG_ERATE * NW / 64 : per0;
        if (cur >= nout) {
            rd = rol; dsh = 19;
#pragma unroll
            for (int k = 0; k < PU; ++k) o[k] = rows[k];
        }
        const int all = flim < NPC ? flim - CFG_LOOK : 2 * NPC, one = skip < 0 ? NPIN - CFG_LOOKP - wrk * CFG_LSP : planned ? 2 * NPC : NPC - ZB - CFG_LOOKP - wrk * CFG_LSP2;
        look = cur >= all || cur >= one;
        fvo = cur >= all || (look && lane == 63) ? (unsigned)lane * 32u : OOB - 16u;
        next = all < one ? all : one;
        next = next < lim ? next : lim;
        const int at[6] = {NPC - CFG_PLAN, NPIN - CFG_DEC, even, fin - CFG_TAIL, nout, NPC};
#pragma unroll
        for (int j = 0; j < 6; ++j) next = at[j] > cur && at[j] < next ? at[j] : next;
        if (ZB && cur >= NPC) {
            int x = cur - NPC;
            const int me = now_s(peer);
            for (int j = 0; j < me; ++j) x -= moved(j) < 0 ? moved(j) : 0;
            for (int j = 0; j < NRANK; ++j) { const int g = moved(j) > 0 ? moved(j) : 0; if ((unsigned)x < (unsigned)g) { dadd = (unsigned)(j - me) * (SCOL * 2u); ti = NPC - g + x; } x -= g; }
            rs = ror; next = 0; ti = __builtin_amdgcn_readfirstlane(ti); dadd = __builtin_amdgcn_readfirstlane(dadd);
#pragma unroll
            for (int k = 0; k < PU; ++k) in[k] = rows[k];
        }
    };
    int first = CFG_DRY ? 2 : 1;
    while (!planned || cur < fin) {
        const bool lead = first == 1 && cur < (NL - NPUSH) << (6 - PSH);
        if (first == 2 || lead) lim = flim = cur + 1;
        if (!bad && cur >= lim) {
            const unsigned long long t0 = __builtin_amdgcn_s_memrealtime();
            const unsigned pvo = (unsigned)(lane - (first_of(cur) >> 3)) < CFG_PWIN || (lane == 63 && cur >= NPIN) ? (unsigned)lane * 32u : OOB - 16u;
            for (;;) {
                const u32x4 f0 = __builtin_amdgcn_raw_buffer_load_b128(rwl, pvo, fso, AUX_SYS), f1 = __builtin_amdgcn_raw_buffer_load_b128(rwl, pvo + 16u, fso, AUX_SYS);
                update(f0, f1);
                if (cur < lim || (planned && cur >= fin)) break;
                if (give_up(tmo, call, t0)) { bad = true; planned = true; plan = 0; fin = NPC; skip = skip < 0 ? 0 : skip; break; }
                __builtin_amdgcn_s_sleep(CFG_PS);
            }
            asm volatile("" ::: "memory");
            if (cur >= fin && planned) break;
        }
        ti = cur;
        if (bad) { dsh = 19; dadd = 0; }
        else if (!first) turn();
        if (first == 2) ti = NPC;
        places();
        if (lead) {
            const unsigned long long t0 = __builtin_amdgcn_s_memrealtime();
            const unsigned fo = fso + 2048u + (unsigned)(((cur >> 3) + 2 * NPUSH) * 2 + (cur >> 1 & 1)) * 8u;
            for (;;) {
                const unsigned f = __builtin_amdgcn_raw_buffer_load_b32(rwl, (unsigned)(lane & 1) * 4u, fo, AUX_SYS);
                if (__all((int)(f - call) >= 0)) break;
                if (give_up(tmo, call, t0)) { bad = true; planned = true; skip = 0; break; }
                __builtin_amdgcn_s_sleep(CFG_PS);
            }
            asm volatile("" ::: "memory");
        }
        if (bad) {
#pragma unroll
            for (int k = 0; k < PU; ++k) STORE16(((u32x4){0x7fc07fc0u, 0x7fc07fc0u, 0x7fc07fc0u, 0x7fc07fc0u}), rol, cur >= NPI2 ? rows[k] : OOB, dst, AUX_AGENT);
            cur = index_of(draw()); continue;
        }
        if (first < 2) STAMP1(P, 1);
        unsigned a[PU] = {OOB, OOB, OOB, OOB}, cprev = 0, cold = 0, dp = 0;
        rsrc_t rp = rwl;
        u32x4 v[PU] = {};
        auto put = [&](int k) { STORE16(v[k & 3], k < 0 ? rp : rd, a[k & 3], k < 0 ? dp : dst, CFG_PST); };
        unsigned tick = (unsigned)__builtin_amdgcn_s_memrealtime();
        unsigned slot = (tick << 4) - (tick << 4 & 0xffffffu) % PACE16 + (unsigned)wrk * (PACE16 / NW) - PACE16;
        tick += CFG_GRID - tick % CFG_GRID;
        bool early = false;
        do {
            unsigned nv; u32x4 f0, f1;
            auto step = [&](auto kc) {
                constexpr int k = kc;
                if (!tail) {
                    slot += per;
                    if ((int)((tick << 4) - slot) < 0) tick += CFG_GRID;
                    if ((int)((tick << 4) - slot) < 0) tick += CFG_GRID;
                    asm volatile("" : "+s"(tick), "+s"(slot));
                    WAIT_VMN(ops_since(k, BASE + 1, 0));
                    if (!deep) WAIT_VMN(ops_since(k, BASE, 1));
                    asm volatile("s_memrealtime vcc\n\ts_waitcnt lgkmcnt(0)\n\ts_branch 2f\n"
                                 "1:\ts_add_u32 %0, %0, %3\n2:\ts_sub_u32 vcc_hi, %0, vcc_lo\n\ts_cmp_lt_i32 vcc_hi, %5\n\ts_cbranch_scc1 1b\n"
                                 "3:\ts_memrealtime vcc\n\ts_waitcnt lgkmcnt(0)\n\ts_sub_u32 vcc_lo, %0, vcc_lo\n\ts_cmp_le_i32 vcc_lo, 0\n\ts_cbranch_scc1 4f\n\t"
                                 "s_cmp_le_i32 vcc_lo, %4\n\ts_cbranch_scc1 3b\n\ts_sleep 1\n\ts_branch 3b\n4:"
                                 : "+s"(tick), "+s"(rs), "+s"(src) : "n"(CFG_GRID), "n"(CFG_SPIN), "n"(CFG_PRE) : "scc", "vcc", "memory");
                }
                WAIT_VMN(ops_since(k, BASE + 1, 0));
                v[k] = buffer_load16(rs, in[k], src, CFG_RLD);
                __builtin_amdgcn_sched_barrier(0);
                a[k] = o[k];
                if (k == 0) {
                    nv = draw();
                    f0 = __builtin_amdgcn_raw_buffer_load_b128(rwl, fvo, fso, AUX_SYS); f1 = __builtin_amdgcn_raw_buffer_load_b128(rwl, fvo + 16u, fso, AUX_SYS);
                    arrive(early && cold ? lane0 : OOB, cold);
                    if (early) cold = 0;
                }
                if (k == 2) arrive(cold ? lane0 : OOB, cold);
                put(k - BASE - 1);
                if (!deep) { put(k - BASE); a[(k - BASE) & 3] = OOB; }
                const int d = (int)((tick << 4) - slot);
                if (d > CFG_CRED * 16) slot += (unsigned)(d - CFG_CRED * 16);
                tick += (deep ? CFG_GD : CFG_GS2) >> k & 1 ? 2 * CFG_GRID : CFG_GRID;
            };
            step(std::integral_constant<int, 0>()); step(std::integral_constant<int, 1>()); step(std::integral_constant<int, 2>()); step(std::integral_constant<int, 3>());
            asm volatile("" ::: "memory");
            early = !deep; cold = cprev; cprev = cb; rp = rd; dp = dst;
            ti = cur = index_of(nv);
            if (__builtin_expect(cur >= next, 0)) {
                if (look) update(f0, f1);
                ti = cur;
                if (cur < lim) turn();
            }
            places();
        } while (cur < lim);
#pragma unroll
        for (int k = -1 - BASE; k < 0; ++k) put(k);
        if (cprev | cold) { WAIT_VM0(); arrive(cprev ? lane0 : OOB, cprev); arrive(cold ? lane0 : OOB, cold); }
        if (first == 2) {
            cur = widx; lim = flim = next = 0; dvo = head; tail = false; fvo = (unsigned)lane * 32u;
#pragma unroll
            for (int k = 0; k < PU; ++k) o[k] = in[k] = piece(k);
        }
        first >>= 1;
    }
    STAMP(P, 4);
}

__device__ __forceinline__ void load_w(u32x4 (&v)[8], const rsrc_t rw, int col, int q, int job, int lane) {
    const unsigned vo = (unsigned)((lane >> 4) * 8) * (NN * 2) + (unsigned)(lane & 15) * 16u;
    const unsigned so = (unsigned)(q * 128 + (job >> 1) * 32) * (NN * 2) + (unsigned)(col + (job & 1) * 128) * 2u;
#pragma unroll
    for (int j = 0; j < 8; ++j) v[j] = __builtin_amdgcn_raw_buffer_load_b128(rw, vo, so + (unsigned)j * (NN * 2), 2);
}
__device__ __forceinline__ void store_w(const u32x4 (&v)[8], const rsrc_t rd, unsigned vo, unsigned unit, int job, bool slow) {
    unit += (unsigned)(job >> 2) * KTB + (unsigned)(job & 1) * 8192u + (unsigned)(job >> 1 & 1) * 1024u;
#pragma unroll
    for (int i = 0; i < 8; ++i) {
        u32x4 o;
#pragma unroll
        for (int e = 0; e < 4; ++e) o[e] = __builtin_amdgcn_perm(v[2 * e + 1][i >> 1], v[2 * e][i >> 1], (i & 1) ? 0x07060302u : 0x05040100u);
        STORE16(o, rd, vo, unit + (unsigned)(i >> 2) * HTB + (unsigned)(i & 3) * 2048u, AUX_SYS);
        if (CFG_PWND && i >= CFG_PWND - 1 && i < 7 && slow) WAIT_VMN(CFG_PWND ? CFG_PWND - 1 : 0);
    }
}

__device__ __forceinline__ void lead_role(const Params& P, int rank, unsigned long long wsp, unsigned tk, unsigned t0, int g, int wid, int lane) {
    const int o = NL - 1 - (g >> 5), c = (g >> 3) & 3, job = (c >> 1) * 4 + wid * 2 + (c & 1);
    const int dest = (rank + 1 + g + (o < NPUSH ? CFG_MIX * (2 * c + o) : 0)) & (NRANK - 1);
    u32x4 v[8];
    load_w(v, mk_rsrc(P.w, (unsigned)KSH * NN * 2), dest * SCOL + o * 256, 0, job, lane);
    asm volatile("" : "+v"(tk), "+v"(wsp) :: "memory");
    const bool push = o < NPUSH && dest != rank, count = o < NPUSH || dest == rank;
    char* wsd = (char*)((unsigned long long)(unsigned)__builtin_amdgcn_readlane((unsigned)(wsp >> 32), dest) << 32 | (unsigned)__builtin_amdgcn_readlane((unsigned)wsp, dest));
    gptr_t fp = (gptr_t)(wsd + (count ? WS_CNT + (unsigned)((NTM + o) * NRANK + rank) * 128u : WS_FLAG + (unsigned)rank * FLAGB + 2048u + (unsigned)((o * 4 + c) * 2 + wid) * 4u));
    while (CFG_P3 && push && o >= CFG_P3N && (int)((unsigned)__builtin_amdgcn_s_memrealtime() - t0) < CFG_P3 + (o - CFG_P3N) * 110) __builtin_amdgcn_s_sleep(1);
    store_w(v, mk_rsrc(push ? wsd : P.wsl, WS_END), (unsigned)lane * 16u + (push ? 0u : (~tk & 1u) * SNDB),
            push ? WS_PAN + (unsigned)(NTM + o) * PANB + (unsigned)rank * CHB : WS_SND + (unsigned)(dest * NB + o) * UNB, job, push);
    WAIT_VMN(0);
    STAMP(P, 6);
    if (lane == 0) {
        if (!count) __hip_atomic_store(fp, tk + 1u, __ATOMIC_RELAXED, __HIP_MEMORY_SCOPE_SYSTEM);
        else if ((__hip_atomic_fetch_add((gptr_t)(P.wsl + WS_LEAD + (o * NRANK + dest) * 64), 1u, __ATOMIC_RELAXED, __HIP_MEMORY_SCOPE_AGENT) & 7u) == 7u)
            __hip_atomic_store(fp, (tk + 1u) * CNT_FULL, __ATOMIC_RELAXED, __HIP_MEMORY_SCOPE_SYSTEM);
    }
}

__device__ __forceinline__ void pack_role(const Params& P, unsigned long long wsp, unsigned tk, unsigned t0, int g, int wid, int lane) {
    const int rank = P.rank;
    const rsrc_t rx = mk_rsrc(P.x, (unsigned)MM * KSH * 2), rw = mk_rsrc(P.w, (unsigned)KSH * NN * 2), rwl = mk_rsrc(P.wsl, WS_END);
    const unsigned sv = (unsigned)lane * 16u + (~tk & 1u) * SNDB;
    const unsigned to = ((const unsigned*)P.order.to)[lane];
    constexpr int NH = CFG_UH < NUNIT ? 8 * DK : 0, UH = NH ? CFG_UH : NUNIT;
    const bool late = g >= NPK - NH;
    constexpr int U1 = NRANK * NL, NE = CFG_EH ? NH : 0;
    static_assert(CFG_TH < (UH - U1 - NH) * CFG_PH, "unpaced");
    for (int u = late ? (NE ? U1 : UH) + g - (NPK - NH) : U1 + NE + g; u < (late ? NUNIT : UH); u += !late ? NPK - NH : u < UH ? UH - U1 : NH) {
        int o, dest = -1;
        if (u < 4 * U0) {
            const int r = u % U0;
            if (r < NRANK * NS) { o = r >> 3; dest = (rank + 1 + r) & (NRANK - 1); } else o = r - (NRANK - 1) * NS;
            o += u / U0 * G0;
        } else o = u - 4 * (NRANK - 1) * NS;
        int q, pos; const int pan = unit_of(o, dest < 0 ? 0 : dest, q, pos);
        while ((int)((unsigned)__builtin_amdgcn_s_memrealtime() - t0) < (late ? CFG_TH + (u - UH) * CFG_PH : ((u - CFG_PACE0) * CFG_PACE - (u > CFG_UK ? (u - CFG_UK) * (CFG_PACE - CFG_PACE2) : 0)) / 10)) __builtin_amdgcn_s_sleep(1);
        const unsigned dst = WS_SND + (unsigned)pos * UNB;
        u32x4 v[8];
        if (pan < NTM) {
            const int rb0 = (wid & 3) * 4;
            const unsigned vo = (unsigned)(lane & 15) * (KSH * 2) + (unsigned)(lane >> 4) * 16u;
            const unsigned so = (unsigned)(pan * 256 + rb0 * 16) * (KSH * 2) + (unsigned)(q * 2 + (wid >> 2)) * 128u;
#pragma unroll
            for (int i = 0; i < 8; ++i) v[i] = __builtin_amdgcn_raw_buffer_load_b128(rx, vo + (i & 1) * 64, so + (unsigned)(i >> 1) * (16 * KSH * 2), 0);
#pragma unroll
            for (int i = 0; i < 8; ++i) STORE16(v[i], rwl, sv + (i & 1) * 1024, dst + (unsigned)(wid >> 2) * KTB + (unsigned)(rb0 + (i >> 1)) * 2048u, AUX_SYS);
        } else {
            load_w(v, rw, gpan_of(pan - NTM, dest < 0 ? 0 : dest) * 256, q, wid, lane);
            store_w(v, rwl, sv, dst, wid, false);
        }
        o = look_up(to, o);
        const bool own = lane == rank;
        unsigned long long fp = wsp + (own ? WS_CNT + (unsigned)((pan * NRANK + rank) * 32 + q) * 4u : WS_FLAG + (unsigned)rank * FLAGB + (unsigned)o * 4u);
        unsigned fv = own ? (tk + 1u) * CNT_FULL : tk + 1u;
        asm volatile("" : "+v"(fp), "+v"(fv));
        WAIT_VMN(0);
        __builtin_amdgcn_s_barrier();
        const bool mine = lane < NRANK && (dest < 0 || dest == lane);
        if (wid == 0 && mine && (own || o < CFG_AGG || o >= NTURN / 8 * 8)) __hip_atomic_store((gptr_t)fp, fv, __ATOMIC_RELAXED, __HIP_MEMORY_SCOPE_SYSTEM);
        if (wid == 1 && o >= CFG_AGG && o < NTURN / 8 * 8) {
            unsigned old = 0;
            if (mine && !own) old = __hip_atomic_fetch_add((gptr_t)(P.wsl + WS_GRP) + (o >> 3) * NRANK + lane, 1u, __ATOMIC_RELAXED, __HIP_MEMORY_SCOPE_AGENT);
            const unsigned full = (unsigned)__ballot((old & 7u) == 7u);
            const unsigned long long wsq = (unsigned long long)(unsigned)__builtin_amdgcn_ds_bpermute(lane >> 3 << 2, (int)(wsp >> 32)) << 32 | (unsigned)__builtin_amdgcn_ds_bpermute(lane >> 3 << 2, (int)wsp);
            if (full >> (lane >> 3) & 1) __hip_atomic_store((gptr_t)(wsq + WS_FLAG + (unsigned)rank * FLAGB) + (o & ~7) + (lane & 7), tk + 1u, __ATOMIC_RELAXED, __HIP_MEMORY_SCOPE_SYSTEM);
        }
        if (wid == 2 && dest >= 0 && pan >= NTM + NB) {
            if (lane == 0 && dest != rank) st_sys((unsigned*)(P.wsl + WS_CNT) + ((NIN + (NTM + NS - 1 - pan) * 7 + ((dest - rank - 1) & 7)) * NRANK + rank) * 32 + q, (tk + 1u) * CNT_FULL);
            unsigned old = 0;
            if (lane == 0) old = __hip_atomic_fetch_add((gptr_t)(P.wsl + WS_BAND), 1u, __ATOMIC_RELAXED, __HIP_MEMORY_SCOPE_AGENT);
            if ((unsigned)__builtin_amdgcn_readfirstlane(old) + 1u == (tk + 1u) * NLAST && lane < NRANK && !own)
                __hip_atomic_store((gptr_t)(wsp + WS_FLAG + (unsigned)rank * FLAGB) + NTURN, tk + 1u, __ATOMIC_RELAXED, __HIP_MEMORY_SCOPE_SYSTEM);
        }
    }
}

struct Tile { int tm, v, kind; unsigned sA, stA; };
enum { WHOLE = 0, HEAD = (KT / 2 - 4) << 5 | 1 << 10, REST = 4 | 4 << 5 | 2 << 10, PIECE = 3 << 10 };
__device__ __forceinline__ void tile_of(int idx, int col, int rank, unsigned snd, Tile& t) {
    const int v = idx < 2 * GW ? (idx >> 2) + (col >> 2) * (GW / 2) : 2 * (idx >> 2) + (col >> 2);
    t.tm = 4 * (col & 3) + (idx & 3); t.v = v; t.kind = WHOLE;
    const int g = (v - GW) / GM;
    t.sA = snd + (unsigned)(v < GW ? spos_of(v, rank) : 4 * U0 + g * 3 * GM + v - GW) * UNB;
    t.stA = (unsigned)(v < GW ? step_of(v) : g < NGF ? GM : GL) * UNB;
}
__device__ __forceinline__ void item_of(int q, int m, int rank, unsigned snd, Tile& t) {
    const int full = m == 0 ? full_of(0) : m == 1 ? full_of(1) : full_of(2), two = m == 0 ? two_of(0) : m == 1 ? two_of(1) : two_of(2);
    int u0, u1, p;
    const int n = full + piece_of(q - full, two, m == 0 ? three_of(0) : m == 1 ? three_of(1) : three_of(2), u0, u1, p);
    int at = 0;
    auto from = [&](auto c) { constexpr int l = left_of(c), a = (nt_of(c) - left_of(c + 1)) * 8 + c; at = n >= l ? a : at; };
#define C(i) std::integral_constant<int, i>()
    from(C(0)); from(C(1)); from(C(2)); from(C(3)); from(C(4)); from(C(5)); from(C(6)); from(C(7));
#undef C
    tile_of((at >> 3) + n, at & 7, rank, snd, t);
    if (n >= PN) {
        const int g = (n - PN) / 28, w = (n - PN) % 28, y = (g >> 2) * 7 + (w >> 2);
        t.tm = 4 * (g & 3) + (w & 3); t.v = NWP + y; t.sA = snd + (unsigned)opos_of(y, rank) * UNB; t.stA = NRANK * UNB;
    }
    if (q >= full) t.kind = u0 | (KT / 2 - u1 + u0) << 5 | (p < 2 ? 3 | (2 * (n - full) + p) << 3 : (2 + p) | 2 * (n - full) << 3) << 10;
}

template <int wr>
__device__ __forceinline__ void gemm_role(const Params& P, unsigned call, int col, int slot, int kx, unsigned t0, LAS unsigned char* lds) {
    const int tid = threadIdx.x, wid = __builtin_amdgcn_readfirstlane(tid >> 6), lane = tid & 63, wc = wid & 3;
    const int rank = P.rank;
    char* wsl = P.wsl;
    const rsrc_t rws = mk_rsrc(wsl, WS_END);
    LAS unsigned* ctl = (LAS unsigned*)(lds + STAGE_BYTES);
    const unsigned voff = (unsigned)tid * 16u;
    const unsigned ldsw = (unsigned)wid * 1024u;
    const int aoff = wr * 8192 + lane * 16, boff = wc * 4096 + lane * 16;

    auto lane_now = [&]() { int tz = threadIdx.x; asm volatile("" : "+v"(tz)); return tz & 63; };
    auto cnt_of = [&](const Tile& t) { const int ln = lane_now(); return WS_CNT + (unsigned)((((ln & 32) ? NTM + t.v : t.tm) * NRANK + ((ln >> 2) & 7)) * 32 + (ln & 3)) * 4u; };
    auto level_of = [&](unsigned c) -> int {
        const unsigned long long miss = __ballot((int)(c - call * CNT_FULL) < 0);
        unsigned m = (unsigned)miss | (unsigned)(miss >> 32);
        const bool own = ((m >> (rank * 4)) & 15u) == 0u;
        m |= m >> 16; m |= m >> 8; m |= m >> 4;
        return own ? 1 + __builtin_ctz(m | 16u) : 0;
    };
    auto ready = [&](const Tile& t, int want, bool join = false, bool first = false) -> int {
        if (wr == 0 && wid == 0) {
            const unsigned* c = (const unsigned*)(wsl + cnt_of(t));
            unsigned* tmo = (unsigned*)(wsl + WS_TMO);
            const unsigned long long t0 = __builtin_amdgcn_s_memrealtime();
            if (join)
                while (__builtin_amdgcn_readfirstlane(ld_agent((const unsigned*)(wsl + WS_COL + col * 128))) != call && !give_up(tmo, call, t0)) __builtin_amdgcn_s_sleep(16);
            int got;
            for (;;) {
                got = level_of(ld_sys(c));
                if (got >= want) break;
                if (give_up(tmo, call, t0)) { ctl[0] = 1u; got = 5; break; }
                __builtin_amdgcn_s_sleep(CFG_GS);
            }
            ctl[2] = (unsigned)got;
            if (first) __builtin_amdgcn_fence(__ATOMIC_ACQUIRE, "agent");
            asm volatile("s_waitcnt vmcnt(0) lgkmcnt(0)" ::: "memory");
        }
        asm volatile("" ::: "memory"); __builtin_amdgcn_s_barrier(); asm volatile("" ::: "memory");
        const int r = __builtin_amdgcn_readfirstlane(ctl[2]);
        asm volatile("s_waitcnt lgkmcnt(0)" ::: "memory");
        return r;
    };

#define SA(b, h) (((b) * 2 + (h)) * HTB)
#define SB(b, h) ((4 + (b) * 2 + (h)) * HTB)
#define STAGE(bufoff, soff) do { _Pragma("unroll") for (int _i = 0; _i < 2; ++_i) \
        __builtin_amdgcn_raw_ptr_buffer_load_lds(rws, (LAS unsigned*)(lds + (bufoff) + ldsw + _i * 8192), 16, voff, (soff) + _i * 8192u, 0, CFG_LDA); } while (0)
#define LDA(dst, b, h) do { _Pragma("unroll") for (int m = 0; m < 4; ++m) _Pragma("unroll") for (int k = 0; k < 2; ++k) dst[m][k] = *(const LAS bf16x8*)(lds + SA(b, h) + aoff + m * 2048 + k * 1024); } while (0)
#define LDB(dst, b, h) do { _Pragma("unroll") for (int n = 0; n < 2; ++n) _Pragma("unroll") for (int k = 0; k < 2; ++k) dst[n][k] = *(const LAS bf16x8*)(ldb + (SB(b, h) - SB(0, 0)) + n * 2048 + k * 1024); } while (0)
#define MMA(ai, first, prio, extra) do { __builtin_amdgcn_s_setprio(prio); \
        _Pragma("unroll") for (int i = 0; i < 16; ++i) { \
            const int bj = ((i >> 1) ^ (i >> 2)) & 1, m = i >> 2, n = (i ^ (i >> 2)) & 1, k = i & 1; \
            if (first) asm volatile("v_mfma_f32_16x16x32_bf16 %0, %1, %2, 0\n\tv_mfma_f32_16x16x32_bf16 %0, %3, %4, %0" : "=&v"(acc[ai][bj][m][n]) \
                         : "v"(bj ? B1[n][k] : B0[n][k]), "v"(At[m][k]), "v"(bj ? B1[n][k ^ 1] : B0[n][k ^ 1]), "v"(At[m][k ^ 1])); \
            else asm volatile("v_mfma_f32_16x16x32_bf16 %0, %1, %2, %0\n\tv_mfma_f32_16x16x32_bf16 %0, %3, %4, %0" : "+v"(acc[ai][bj][m][n]) \
                         : "v"(bj ? B1[n][k] : B0[n][k]), "v"(At[m][k]), "v"(bj ? B1[n][k ^ 1] : B0[n][k ^ 1]), "v"(At[m][k ^ 1])); } \
        extra; \
        __builtin_amdgcn_s_setprio(0); } while (0)
#define WAIT_L0() do { asm volatile("" ::: "memory"); __builtin_amdgcn_s_waitcnt(0xC07F); asm volatile("" ::: "memory"); } while (0)
#define WAIT_V(n) asm volatile("s_waitcnt vmcnt(" #n ")" ::: "memory")
#define WAIT_L(n) asm volatile("s_waitcnt lgkmcnt(" #n ")" ::: "memory")
#define BAR __builtin_amdgcn_s_barrier()
#define SCHED __builtin_amdgcn_sched_barrier(0)

    LAS unsigned char* ldb = lds + SB(0, 0) + boff;
    asm volatile("" : "+v"(ldb));
    unsigned long long outp = (unsigned long long)P.outl;
    asm volatile("" : "+v"(outp));
    const int np = CFG_NP >> col * 4 & 15;
    const unsigned snd = WS_SND + (call & 1u) * SNDB;
    auto after = [&](int r) { return r == -1 && slot >= R0 ? 1 : r + 1; };
    auto fixed = [&](int r, Tile& t) {
        const int n = r < 0 ? r + np + pr0_of(np) : r;
        tile_of(n ? R0 + (n - 1) * kx + slot : slot, col, rank, snd, t);
        t.kind = r < 0 ? HEAD : (unsigned)(r - pr0_of(np)) < (unsigned)np ? REST : WHOLE;
    };
    Tile cur, nxt; int rnd = after(-np - 1);
    fixed(rnd, cur);
    bf16x8 At[4][2], B0[2][2], B1[2][2];
    constexpr unsigned SB0 = (NRANK * NB - NS + GW) * UNB, STB = UQ * UNB;
    unsigned tabA, tabB;
    auto tables = [&](const Tile& t) {
        const int ln = lane_now() & 31, j0 = t.kind >> 5 & 31, u = (t.kind & 31) + (ln > j0 ? ln - j0 : 0);
        const int q = (u - 4) * 37 >> 8;
        const unsigned kov = WS_PAN + (unsigned)((rank - 3 + u - 7 * q) & (NRANK - 1)) * CHB + (unsigned)q * UNB;
        tabA = u < 4 ? t.sA + (unsigned)u * t.stA : (unsigned)(NTM + t.v) * PANB + kov;
        tabB = u < 4 ? SB0 + snd + (unsigned)t.tm * UNB + (unsigned)u * STB : (unsigned)t.tm * PANB + kov;
    };
    auto pick = [&]() {
        Tile t = cur; bool ok = true;
        if (rnd + 1 < RS) fixed(after(rnd), t);
        else if (rnd > RS) ok = false;
        else {
            const unsigned vo = lane_now() == 0 ? 0u : 0xfffffff0u, at = WS_COL + (unsigned)(col * 128 + 80 + (rnd + 1 - RS) * 4);
            if ((unsigned)__builtin_amdgcn_readfirstlane(buffer_swap(call, rws, vo, WS_HEAD + 256u + (unsigned)(col * 64 + (rnd + 1 - RS) * 4), 0)) != call) {
                const int d = __builtin_amdgcn_raw_ptr_buffer_atomic_add_i32(kx, rws, vo, WS_HEAD + (call & 1u) * 128u, 0);
                __builtin_amdgcn_raw_buffer_store_b32(call * 1024u + (unsigned)d, rws, vo, at, AUX_SYS);
            }
            int m, q;
            const unsigned long long t0 = __builtin_amdgcn_s_memrealtime();
            for (;;) {
                m = NO ? __builtin_amdgcn_raw_buffer_load_b32(rws, 0, WS_FLAG + 63 * 32 + 16, AUX_SYS) - call * 4u : 0;
                q = __builtin_amdgcn_readfirstlane(__builtin_amdgcn_raw_buffer_load_b32(rws, 0, at, AUX_SYS) - call * 1024u);
                m = __builtin_amdgcn_readfirstlane(m);
                if ((unsigned)m < 4u && (unsigned)q < 1024u) break;
                if (give_up((unsigned*)(wsl + WS_TMO), call, t0)) { ctl[0] = 1u; q = 1024; break; }
                __builtin_amdgcn_s_sleep(CFG_PS);
            }
            q += slot;
            ok = q < (m == 0 ? items_of(0) : m == 1 ? items_of(1) : items_of(2));
            if (ok) item_of(q, m, rank, snd, t);
        }
        *(LAS u32x4*)(ctl + 4) = (u32x4){ok ? (unsigned)(t.tm | t.v << 8 | 1 << 16) : 0u, t.sA, t.stA, (unsigned)t.kind};
        __builtin_amdgcn_raw_ptr_buffer_load_lds(rws, ctl + 64, 4, cnt_of(t), 0, 0, AUX_SYS);
    };
    auto picked = [&](int& level) -> bool {
        const u32x4 f = *(const LAS u32x4*)(ctl + 4);
        level = level_of(ctl[64 + lane_now()]);
        const unsigned w = __builtin_amdgcn_readfirstlane(f[0]);
        nxt.tm = w & 255u; nxt.v = (w >> 8) & 255u; nxt.kind = __builtin_amdgcn_readfirstlane(f[3]);
        nxt.sA = __builtin_amdgcn_readfirstlane(f[1]); nxt.stA = __builtin_amdgcn_readfirstlane(f[2]);
        return w != 0u;
    };
    constexpr int JP = KT / 2 - 3, JL = KT / 2 - 1;
    const int jend = wr == 0 && wid == 0 ? JP : JL;
    while (CFG_HT0 && (int)((unsigned)__builtin_amdgcn_s_memrealtime() - t0) < CFG_HT0 + ((col & 3) * 2 + (col >> 2)) * CFG_HT1) __builtin_amdgcn_s_sleep(8);
    int lvl = ready(cur, 1, rnd == 1, true);
    STAMP(P, 2);
    tables(cur);
    unsigned a2 = cur.sA, b2 = SB0 + snd + (unsigned)cur.tm * UNB;
    STAGE(SB(0, 0), b2); STAGE(SA(0, 0), a2); STAGE(SB(0, 1), b2 + HTB); STAGE(SA(0, 1), a2 + HTB);
    STAGE(SB(1, 0), b2 + KTB); STAGE(SA(1, 0), a2 + KTB); STAGE(SB(1, 1), b2 + HTB + KTB);
    WAIT_V(0); BAR;
#define ROUND_DONE() do {} while (0)
    bool has_next = false;
    const unsigned one = (lane >> 4) == ((lane >> 2) & 3) ? (lane & 1 ? 0x3f800000u : 0x3f80u) : 0u;
    u32x4 sel0 = {lane & 2 ? 0u : one, lane & 2 ? one : 0u, 0u, 0u}, sel1 = {0u, 0u, sel0[0], sel0[1]};
    asm volatile("" : "+v"(sel0), "+v"(sel1));
#pragma unroll 1
    for (;;) {
        f32x4 acc[2][2][4][2];
#define ACC(i) acc[(i) >> 4][((i) >> 3) & 1][((i) >> 1) & 3][(i) & 1]
        unsigned prk = WS_PRK + (unsigned)((rnd < 0 ? rnd + np : rnd - pr0_of(np)) * NG + (int)blockIdx.x - (GRID - NG)) * (2 * UNB);
        u32x4 pf[4];
#pragma unroll
        for (int i = 0; i < 4; ++i) asm volatile("" : "=v"(pf[i]));
        int jf = cur.kind == REST ? DBG_JF + slot % CFG_FO : 99, je = -1, fe = 0;
#define FOLD2(a, b, p) "v_mfma_f32_16x16x32_bf16 %" #a ", %8, %" #p ", %" #a "\n\tv_mfma_f32_16x16x32_bf16 %" #b ", %9, %" #p ", %" #b "\n\t"
#define FOLD() do { \
            _Pragma("unroll") for (int c = 0; c < 4; ++c) \
                asm volatile("s_cmp_lg_u32 %14, %15\n\ts_cbranch_scc1 1f\n\t" FOLD2(0, 1, 10) FOLD2(2, 3, 11) FOLD2(4, 5, 12) FOLD2(6, 7, 13) "\n1:" \
                    : "+v"(ACC(c * 8)), "+v"(ACC(c * 8 + 1)), "+v"(ACC(c * 8 + 2)), "+v"(ACC(c * 8 + 3)), "+v"(ACC(c * 8 + 4)), "+v"(ACC(c * 8 + 5)), "+v"(ACC(c * 8 + 6)), "+v"(ACC(c * 8 + 7)) \
                    : "v"(sel0), "v"(sel1), "v"(pf[0]), "v"(pf[1]), "v"(pf[2]), "v"(pf[3]), "s"(fe), "n"(c + 1) : "scc"); \
            if (fe < 4) { \
                _Pragma("unroll") for (int i = 0; i < 4; ++i) pf[i] = __builtin_amdgcn_raw_buffer_load_b128(rws, voff + DBG_OOB, prk + i * 8192u, CFG_ILD); \
                prk += 32768u; je = j; } \
            jf += FS; ++fe; } while (0)
        int jchk = lvl < 5 && cur.kind != HEAD ? 7 * lvl - 4 : jend;
#define SYNC_MMA(ai, first, wv, extra) do { if (wv) WAIT_V(8); WAIT_L0(); if (wr == 1) BAR; MMA(ai, first, wr + 1, extra); if (wr == 0) BAR; SCHED; } while (0)
#define WAIT_X() asm volatile("s_cmp_eq_u32 %0, %1\n\ts_waitcnt vmcnt(%2)\n\ts_cbranch_scc1 1f\n\ts_waitcnt vmcnt(8)\n1:" :: "s"(j), "s"(je), "n"(12) : "scc", "memory")
#define PASS(first, check, extra) do { \
            LDB(B0, 0, 0); LDA(At, 0, 0); LDB(B1, 0, 1); STAGE(SA(1, 1), a2 + KTB + HTB); SCHED; \
            check; \
            a2 = __builtin_amdgcn_readlane(tabA, (j + 1) & 31); b2 = __builtin_amdgcn_readlane(tabB, (j + 1) & 31); \
            SYNC_MMA(0, first, !first, ); \
            LDA(At, 0, 1); STAGE(SB(0, 0), b2); STAGE(SA(0, 0), a2); STAGE(SB(0, 1), b2 + HTB); \
            SYNC_MMA(1, first, !first, extra); \
            LDB(B0, 1, 0); LDA(At, 1, 0); LDB(B1, 1, 1); STAGE(SA(0, 1), a2 + HTB); \
            if (!first) WAIT_X(); \
            SYNC_MMA(0, false, first, ); \
            LDA(At, 1, 1); STAGE(SB(1, 0), b2 + KTB); STAGE(SA(1, 0), a2 + KTB); STAGE(SB(1, 1), b2 + KTB + HTB); \
            if (!first) WAIT_X(); \
            SYNC_MMA(1, false, first, ); } while (0)
        int j = cur.kind >> 5 & 31;
        PASS(true, , );
#pragma unroll 1
        for (++j; j < KT / 2; ++j)
            PASS(false, if (__builtin_expect(j == jchk, 0)) {
                    if (j < JP) { lvl = ready(cur, lvl + 1); jchk = lvl < 5 ? 7 * lvl - 4 : jend; }
                    else if (wr == 0 && j == JP) { pick(); jchk = JL; }
                    else {
                        has_next = picked(lvl);
                        if (has_next) {
                            const int want = nxt.kind >= PIECE ? 5 : nxt.kind == REST ? 2 : 1;
                            const bool join = rnd == -1 && slot >= R0;
                            if (lvl < want || join) lvl = ready(nxt, want, join);
                            tables(nxt);
                        }
                    }
                }, if (__builtin_expect(j == jf, 0)) FOLD());
        asm volatile("s_nop 7" ::: "memory");
        SCHED;
        const int type = cur.kind >> 10 & 7;
        const unsigned pcs = WS_PRK + (unsigned)(np_max() * NG + (cur.kind >> 13)) * (2 * UNB), pcf = WS_PCF + (unsigned)(cur.kind >> 13) * 4u;
        if (type > 3) {
            const unsigned long long t0 = __builtin_amdgcn_s_memrealtime();
            for (;;) {
                const u32x2 f = __builtin_amdgcn_raw_buffer_load_b64(rws, 0, pcf, AUX_SYS);
                if ((unsigned)__builtin_amdgcn_readfirstlane(f[0]) == call && (type == 4 || (unsigned)__builtin_amdgcn_readfirstlane(f[1]) == call)) break;
                if (give_up((unsigned*)(wsl + WS_TMO), call, t0)) { ctl[0] = 1u; break; }
                __builtin_amdgcn_s_sleep(CFG_PS);
            }
            asm volatile("" ::: "memory");
            const unsigned v2 = type == 4 ? 0xfffffff0u : voff;
            constexpr int NZ = CFG_NZ;
            u32x4 z[NZ];
#pragma unroll
            for (int i = 0; i < 32 + NZ; ++i) {
                if (i >= NZ) {
                    const int c = (i - NZ & 15) * 2;
                    asm volatile("v_mfma_f32_16x16x32_bf16 %0, %2, %4, %0\n\tv_mfma_f32_16x16x32_bf16 %1, %3, %4, %1\n\ts_nop 7" : "+v"(ACC(c)), "+v"(ACC(c + 1)) : "v"(sel0), "v"(sel1), "v"(z[i % NZ]));
                }
                if (i < 32) z[i % NZ] = __builtin_amdgcn_raw_buffer_load_b128(rws, i < 16 ? voff : v2, pcs + i * 8192u, AUX_SYS);
            }
            asm volatile("s_nop 7" ::: "memory");
        }
        {
            const int ln = lane_now(), fr = ln & 15, fq = ln >> 4;
            const rsrc_t rol = mk_rsrc((const void*)outp, OUTB);
            const bool shared = cur.v < NS;
            if (__builtin_amdgcn_readfirstlane(ctl[0]) != 0u) {
#pragma unroll
                for (int i = 0; i < 32; ++i) acc[i >> 4][(i >> 3) & 1][(i >> 1) & 3][i & 1] = __builtin_bit_cast(f32x4, (u32x4){0x7fc00000u, 0x7fc00000u, 0x7fc00000u, 0x7fc00000u});
            }
            if ((unsigned)rnd < (unsigned)RS && wr == 0 && slot == 0 && wid == 0)
                __builtin_amdgcn_raw_buffer_store_b64((u32x2){call, (unsigned)__builtin_amdgcn_s_memrealtime()}, rws, ln == 0 ? 0u : 0xfffffff0u, WS_COL + (unsigned)(col * 128 + rnd * 8), AUX_SYS);
            if (CFG_TCH && (unsigned)(rnd - CFG_TCH) < 2u && slot * 8 + col < SCOL / 8) {
                unsigned lines = voff; asm volatile("" : "+v"(lines)); lines <<= 2;
                __builtin_amdgcn_raw_ptr_buffer_load_lds(rws, ctl + 128, 4, lines, WS_OSN + (unsigned)((rnd - CFG_TCH) * (SCOL / 8) + slot * 8 + col) * 32768u, 0, 0);
            }
            if (CFG_OTC && rnd <= CFG_OTC && rnd > CFG_OTC - CFG_OTN && slot < KX) {
                unsigned lines = voff; asm volatile("" : "+v"(lines)); lines <<= 2;
                const int w = slot * 8 + col;
                __builtin_amdgcn_raw_ptr_buffer_load_lds(rws, ctl + 128, 4, lines, snd + (unsigned)(rnd == CFG_OTC && w < NO * 64 ? 8 * UQ + w : 2 * NUNIT - (CFG_OTC + 1 - rnd) * 8 * KX + (rnd < CFG_OTC) * NO * 64 + w) * 32768u, 0, 0);
            }
            const bool thru = rnd >= RS;
            const unsigned vo = (unsigned)(wc * 32 + fq * 4) * (NN * 2) + (unsigned)(wr * 256 + fr * 16);
            const unsigned so = (unsigned)(cur.tm * 256) * (NN * 2) + (unsigned)col_of(cur.v, rank) * 2u;
            const unsigned vs = (unsigned)(wc * 32 + fq * 4) * 512u + (unsigned)(wr * 256 + fr * 16);
            const unsigned ss = WS_OSN + (unsigned)(cur.v < NB ? NO * NTM + cur.tm * NB + cur.v : (NS - 1 - cur.v) * NTM + cur.tm) * (2 * UNB);
            auto finish = [&](auto mode) {
#pragma unroll
                for (int bj = 0; bj < 2; ++bj)
#pragma unroll
                    for (int n = 0; n < 2; ++n)
#pragma unroll
                        for (int r = 0; r < 4; ++r) {
                            u32x4 o;
                            if ((r & 1) == 0) {
#pragma unroll
                                for (int i = 0; i < 8; ++i) { f32x4& a = acc[i >> 2][bj][i & 3][n]; const f32x2 z = silu2((f32x2){a[r], a[r + 1]}); a[r] = z[0]; a[r + 1] = z[1]; }
                            }
#pragma unroll
                            for (int i = 0; i < 4; ++i) o[i] = cvt_pk_bf16(acc[i >> 1][bj][(i & 1) * 2][n][r], acc[i >> 1][bj][(i & 1) * 2 + 1][n][r]);
                            if (mode == 1) STORE16(o, rws, vs, ss + (unsigned)(bj * 128 + n * 16 + r) * 512u, AUX_SYS);
                            STORE16(o, rol, vo, so + (unsigned)(bj * 128 + n * 16 + r) * (NN * 2), mode == 2 ? CFG_OTH : CFG_OST);
                        }
            };
            auto leave = [&](unsigned to, auto aux) {
#pragma unroll
                for (int i = 0; i < 16; ++i) {
                    const f32x4 a = ACC(2 * i), b = ACC(2 * i + 1);
                    STORE16(((u32x4){cvt_pk_bf16(a[0], a[1]), cvt_pk_bf16(a[2], a[3]), cvt_pk_bf16(b[0], b[1]), cvt_pk_bf16(b[2], b[3])}), rws, voff, to + i * 8192u, aux);
                }
            };
            if (rnd < 0) leave(prk, std::integral_constant<int, CFG_IST>());
            else if (type == 3) leave(pcs, std::integral_constant<int, AUX_SYS>());
            else if (shared) finish(std::integral_constant<int, 1>());
            else if (thru) finish(std::integral_constant<int, 2>());
            else finish(std::integral_constant<int, 0>());
            WAIT_V(16);
            if (shared || type == 3) {
                WAIT_VM0();
                unsigned old = 0;
                if (ln == 0) old = __hip_atomic_fetch_add(ctl + 1, 1u, __ATOMIC_RELAXED, __HIP_MEMORY_SCOPE_WORKGROUP);
                old = __builtin_amdgcn_readfirstlane(old);
                if ((old & 7u) == 7u && !shared) __builtin_amdgcn_raw_buffer_store_b32(call, rws, ln == 0 ? 0u : 0xfffffff0u, pcf, AUX_SYS);
                else if ((old & 7u) == 7u) {
                    const unsigned n = (unsigned)__builtin_amdgcn_raw_ptr_buffer_atomic_add_i32(1, rws, ln == 0 ? 0u : 0xfffffff0u, WS_BAND, 0);
                    if ((unsigned)__builtin_amdgcn_readfirstlane(n) + 1u == call * NLAST && ln < NRANK && ln != rank)
                        st_sys((unsigned*)(P.ws[ln] + WS_FLAG + rank * FLAGB) + NTURN, call);
                }
            }
        }
        ROUND_DONE();
        if (!has_next) break;
        cur = nxt; rnd = after(rnd);
    }
    WAIT_V(0);
}

__global__ __launch_bounds__(512, 2) void fused_gemm_ar(Params P) {
    extern __shared__ __attribute__((aligned(16))) unsigned char shm[];
    LAS unsigned char* lds = (LAS unsigned char*)shm;
    const int wid = __builtin_amdgcn_readfirstlane(threadIdx.x >> 6), lane = threadIdx.x & 63;
    const int b = __builtin_amdgcn_readfirstlane(blockIdx.x);
    const unsigned long long wsp = (unsigned long long)P.ws[lane & (NRANK - 1)];
    const int from = P.order.from[(threadIdx.x + NPUSH) & 255];
    const unsigned t0 = (unsigned)__builtin_amdgcn_s_memrealtime();
    int rk = P.rank;
    asm volatile("" : "+s"(rk) : "s"(P.x), "s"(P.w), "s"(P.wsl), "s"(P.rank), "s"(t0) : "memory");
    unsigned* tkp = (unsigned*)(P.wsl + WS_TICKET + b * 128 + wid * 16);
    const unsigned tk = ld_agent(tkp);
    if (b < NPULL) {
        pull_role(P, tk, tkp, (P.rank + 1 + b / PW) & (NRANK - 1), b % PW * 8 + wid, lane, from, (LAS u32x4*)lds);
    } else {
        const int g = b - NPULL;
        if (g < 4 * NRANK * NL && wid < 2) lead_role(P, rk, wsp, tk, t0, g, wid, lane);
        ((LAS unsigned*)(lds + STAGE_BYTES))[lane & 3] = 0u;
        pack_role(P, wsp, tk, t0, g, wid, lane);
        const unsigned call = __builtin_amdgcn_readfirstlane(tk) + 1u;
        st_agent(tkp, call);
        STAMP(P, 1);
        asm volatile("s_waitcnt lgkmcnt(0)" ::: "memory");
        __builtin_amdgcn_s_barrier();
        if (b >= GRID - NG) {
            const int col = (CFG_SWAP >> (b & 7) * 4) & 7;
            const int slot = b < NPS ? KX : (b - NPS) >> 3, kx = KX + ((b & 7) >= 8 - (NG - 8 * KX));
            if (wid < 4) gemm_role<0>(P, call, col, slot, kx, t0, lds);
            else gemm_role<1>(P, call, col, slot, kx, t0, lds);
        }
    }
}
}

extern "C" void kernel_launch(int rank, int nranks, ncclComm_t comm,
                              hipStream_t stream,
                              void* const* d_in, const int* in_sizes, int n_in,
                              void* d_out, int out_size,
                              const int* in_dtypes, int out_dtype,
                              void* const* d_out_all,
                              void* const* d_ws_all, size_t ws_bytes) {
    if (nranks != NRANK || (unsigned)rank >= (unsigned)NRANK || ws_bytes < (size_t)WS_END || n_in < 2 ||
        in_sizes[0] != MM * KSH || in_sizes[1] != KSH * NN || out_size != MM * NN) {
        fprintf(stderr, "unexpected problem geometry\n");
        return;
    }
    static bool attr_done[NRANK] = {};
    if (!attr_done[rank]) {
        HIP_CHECK(hipFuncSetAttribute((const void*)fused_gemm_ar, hipFuncAttributeMaxDynamicSharedMemorySize, LDS_BYTES));
        attr_done[rank] = true;
    }
    Params P = {};
    P.order = ORDER;
    P.x = d_in[0]; P.w = d_in[1]; P.rank = rank; P.wsl = (char*)d_ws_all[rank]; P.outl = (char*)d_out_all[rank];
    for (int p = 0; p < NRANK; ++p) { P.out[p] = (char*)d_out_all[p]; P.ws[p] = (char*)d_ws_all[p]; }
    hipLaunchKernelGGL(fused_gemm_ar, dim3(GRID), dim3(512), LDS_BYTES, stream, P);
    HIP_CHECK(hipGetLastError());
}
```

```cpp
#include <hip/hip_runtime.h>
#include <rccl/rccl.h>
#include <cstdio>
#include <type_traits>

#define HIP_CHECK(expr) do { hipError_t _e = (expr); if (_e != hipSuccess) { \
    fprintf(stderr, "HIP error %s at %s:%d\n", hipGetErrorString(_e), __FILE__, __LINE__); \
    return; } } while (0)

#define LAS __attribute__((address_space(3)))
typedef short bf16x8 __attribute__((ext_vector_type(8)));
typedef float f32x4 __attribute__((ext_vector_type(4)));
typedef unsigned u32x4 __attribute__((ext_vector_type(4)));
typedef unsigned u32x2 __attribute__((ext_vector_type(2)));
typedef __amdgpu_buffer_rsrc_t rsrc_t;
typedef __attribute__((address_space(1))) unsigned* gptr_t;

namespace {
constexpr int NRANK = 8;
constexpr int MM = 4096, KK = 4096, NN = 32768, KSH = KK / NRANK;
constexpr int KT = KK / 64, KTS = KSH / 64, NTM = MM / 256, NPAN = NN / 256;
#define CFG_NS 6
#define CFG_PW 5
constexpr int NS = CFG_NS;
#define CFG_NO 2
constexpr int NO = CFG_NO, NB = NS - NO;
constexpr int SCOL = NS * 256;
constexpr int NRP = NPAN - NRANK * NS;
constexpr int NWP = NS + NRP;
constexpr int NIN = NTM + NWP;
constexpr int NPL = NIN + (NRANK - 1) * NO;
constexpr int NTURN = NIN * 4;
constexpr int NUNIT = (NTM + NRANK * NS + NRP) * 4;
constexpr int GRID = 256;
constexpr int PW = CFG_PW;
#define CFG_NPULL 0
#define CFG_NW (PW * 8)
constexpr int NPULL = CFG_NPULL ? CFG_NPULL : (CFG_NW * (NRANK - 1) + 7) / 8;
constexpr int NW = CFG_NPULL ? NPULL * 8 / (NRANK - 1) : CFG_NW;
constexpr int NPK = GRID - NPULL;
constexpr int NPS = (NPULL + 7) / 8 * 8;
constexpr int KX = (GRID - NPS) / 8;
#define CFG_GW (2 * (KX / 4))
constexpr int GW = CFG_GW;
constexpr int G0 = GW + NTM;
#define CFG_SKIP 1
#define CFG_GM (GW + 2)
constexpr int GM = CFG_GM;
constexpr int NGF = (NWP - GW) / GM, GL = (NWP - GW) % GM;
static_assert(NS <= GW && NWP > GW, "shared panels sit in the first group");
constexpr int PU = 4, PSH = 2;
#define CFG_PUSH 2
constexpr int NPUSH = CFG_PUSH;
constexpr int NPIN = (NTURN - NPUSH) * 64 / PU;
#define CFG_NL (NPUSH + NW * PU / 64 < NB ? NPUSH + NW * PU / 64 : NB)
constexpr int NL = CFG_NL;
static_assert(NPUSH <= NL && NL <= NB, "leading units have a single consumer");
constexpr int OPB = MM / 2 / PU, OSK = OPB / NRANK;
constexpr int OIB = OPB - OSK, NPI2 = NPIN + NO * OIB, NPO = NPI2 + NO * OPB;
constexpr int BPB = NB * 128 / PU;
constexpr int NPC = NPO + NTM * BPB;
constexpr unsigned CNT_FULL = 64;

#define CFG_SWAP 0x57461302u
#define CFG_XTRA 1
constexpr int NG = CFG_XTRA ? NPK : 8 * KX;
constexpr int DK = CFG_SKIP ? KX - 2 * GW : 0, R0 = KX - DK;
constexpr int nt_of(int col) { return 4 * ((NWP + 1 - (col >> 2)) / 2); }
constexpr int kx_of(int col) { int n = 0; for (int b = GRID - NG; b < GRID; ++b) n += (int)(CFG_SWAP >> (b & 7) * 4 & 7) == col; return n; }
#define CFG_RS ((nt_of(7) - 1 - R0) / (KX + (NG > 8 * KX)) + 1)
constexpr int RS = CFG_RS;
#define CFG_NP 0x55435543u
#define CFG_PR0 2
constexpr int PR0 = CFG_PR0;
constexpr int np_max() { int n = 0; for (int c = 0; c < 8; ++c) n = (int)(CFG_NP >> c * 4 & 15) > n ? CFG_NP >> c * 4 & 15 : n; return n; }
static_assert(PR0 >= 1 && np_max() < RS, "only statically assigned tiles are started early");
__device__ constexpr int pr0_of(int np) { return RS - np < PR0 ? RS - np : PR0; }
constexpr int left_of(int col) { int n = 0; for (int c = 0; c < col; ++c) n += nt_of(c) - R0 - (RS - 1) * kx_of(c); return n; }
constexpr int PN = left_of(8);
#define CFG_L2 16
#define CFG_L3A 14
#define CFG_L3D 1
#define CFG_L3B 11
constexpr int OT = (NRANK - 1) * NTM;
__device__ constexpr int last_of(int m) { return (PN + m * OT) % NG; }
__device__ constexpr int cut_of(int m) { return 2 * last_of(m) > NG ? NG - last_of(m) : last_of(m); }
__device__ constexpr int full_of(int m) { return PN + m * OT - cut_of(m); }
__device__ constexpr int three_of(int m) { return 2 * last_of(m) > NG ? 0 : 3 * last_of(m) > NG ? NG - 2 * last_of(m) : last_of(m); }
__device__ constexpr int two_of(int m) { return cut_of(m) - three_of(m); }
__device__ constexpr int items_of(int m) { return full_of(m) + 2 * two_of(m) + 3 * three_of(m); }
constexpr unsigned CUTS = (CFG_L3B | (CFG_L3B + CFG_L3A) << 5) * 0x100401u + (CFG_L3D << 5) - (CFG_L3D << 25);
__device__ __forceinline__ int piece_of(int z, int two, int three, unsigned cuts, int& u0, int& u1, int& p) {
    const int y = z - 2 * two, g = y >= 2 * three ? 2 : y >= three, w = g == 2 ? 3 * three - 1 - y : y - g * three;
    const unsigned c = cuts >> (w <= KX ? 0 : w > 2 * KX ? 20 : 10);
    const int low = c & 31, mid = c >> 5 & 31;
    p = y >= 0 ? (g ? 5 + g : 3) : z >= two ? 5 : 4;
    u0 = p == 5 ? CFG_L2 : p == 7 ? mid : p == 3 ? low : 0;
    u1 = p == 4 ? CFG_L2 : p == 6 ? low : p == 3 ? mid : KT / 2;
    return y >= 0 ? two + w : z >= two ? z - two : z;
}
static_assert(2 * (PN + NO * OT) <= 3 * NG && CFG_L2 >= 4 && CFG_L2 <= 28 && CFG_L3A - CFG_L3D >= 4 && CFG_L3B >= 4 && CFG_L3A + CFG_L3B + CFG_L3D <= 28, "pieces");
constexpr unsigned KTB = 32768, HTB = 16384;
constexpr unsigned PANB = KT * KTB;
constexpr unsigned CHB = KTS * KTB;
constexpr unsigned UNB = 2 * KTB;
constexpr unsigned WS_TICKET = 0;
constexpr unsigned WS_TMO = WS_TICKET + GRID * 128;
constexpr unsigned WS_FLAG = WS_TMO + 128;
constexpr unsigned FLAGB = 4096;
constexpr unsigned WS_LEAD = WS_FLAG + NRANK * FLAGB;
constexpr unsigned WS_HEAD = WS_LEAD + NRANK * 512;
constexpr unsigned WS_COL = WS_HEAD + NRANK * 128;
#define CFG_PHOFF 0
constexpr unsigned WS_PH = WS_COL + 8 * 128 + CFG_PHOFF;
constexpr unsigned WS_ZC = WS_PH + 2 * NRANK * 128 + 128;
constexpr unsigned WS_GRP = WS_ZC + NRANK * NRANK * 128;
constexpr unsigned WS_BAND = WS_GRP + 64 * NRANK * 4;
constexpr unsigned WS_PCF = WS_BAND + 128;
constexpr unsigned WS_CNT = WS_PCF + GRID * 8;
constexpr unsigned WS_CTRL_END = WS_CNT + NPL * NRANK * 128;
constexpr unsigned WS_DBG = 512u << 10;
constexpr unsigned WS_SND = 1u << 20, SNDB = NUNIT * UNB;
constexpr unsigned WS_OSN = WS_SND + NUNIT * UNB + SNDB;
constexpr unsigned WS_PAN = (WS_OSN + MM * SCOL * 2 + PANB - 1) / PANB * PANB;
constexpr unsigned WS_PRK = WS_PAN + NPL * PANB;
constexpr unsigned WS_END = WS_PRK + (np_max() + 1) * NG * 2 * UNB;
constexpr int U0 = G0 + (NRANK - 1) * NS;
constexpr int UQ = U0 - NRANK * NO;
constexpr unsigned OUTB = (unsigned)MM * NN * 2;
static_assert(WS_CTRL_END <= WS_DBG, "ctrl region");
constexpr unsigned NLAST = NS * NTM + NO * NRANK * 4;
#define CFG_WIN 88
constexpr int BASE = (CFG_WIN - 1) / NW, DEEP = CFG_WIN - BASE * NW;
static_assert(BASE == 2, "window");
constexpr unsigned STAGE_BYTES = 9 * HTB;
constexpr unsigned LDS_BYTES = STAGE_BYTES + 768;
constexpr unsigned long long DEADLINE = 50000000ull;
constexpr int AUX_SYS = 17, AUX_SYS_NT = 19, AUX_AGENT = 16;
#define CFG_PACE 40
#define CFG_UK 1000
#define CFG_PACE2 40
#define CFG_UH (2 * (NPK - 8 * DK))
#define CFG_TH 1200
#define CFG_PH 12
#define CFG_OST 2
#define CFG_OTH AUX_SYS_NT
#define CFG_IST 2
#define DBG_JF 5
#define DBG_OOB 0u
#define CFG_FO 5
constexpr int FS = 5;
#define CFG_HT1 150
#define CFG_HT0 0
#define CFG_ILD 2
#define CFG_PST AUX_AGENT
#define CFG_LDA 0
#define CFG_RLD AUX_SYS
#define CFG_TCH 3
#define CFG_GS 16
#define CFG_AGG 24
static_assert(CFG_AGG % 8 == 0 && CFG_AGG >= NS && NS <= 8 && NPI2 / 16 <= 512 && NIN * 4 <= 512, "groups");
#define CFG_PWND 3
#define CFG_EH 1
#define CFG_MIX 1
static_assert(4 * NRANK * NL <= NPK && (NL - NPUSH) * 64 / PU <= NW && NL * 64 <= 512, "leading units");
#define CFG_PACE0 40
#define CFG_P3 320
#define CFG_P3N 2
#define CFG_LOOK 104
#define CFG_TAIL NW
#define CFG_EVEN 280
#define CFG_EVEND 200
#define CFG_ERATE CFG_RATE
#define CFG_ZB 112
#define CFG_LOOKP 100
#define CFG_LSP 0
#define CFG_LSP2 0
#define CFG_PLAN (CFG_ZB + CFG_LOOKP + NW * CFG_LSP2 + 50)
#define CFG_DEC (CFG_LOOKP + NW * CFG_LSP + 80)
#define CFG_QR 0.915f
#define CFG_QH 0.16f
#define CFG_DYN 1
#define CFG_TP 450.0f
#define CFG_TC 300.0f
#define CFG_TF 720.0f
#define CFG_LT 1712
#define CFG_MARG 450.0f
#define CFG_BAL 8
constexpr int ZB = CFG_ZB;
static_assert(ZB <= 127 && ZB <= BPB && ZB % 16 == 0 && NTURN < 504 && NW == PW * 8, "plan");
#define CFG_PWIN 4u
#define CFG_PS 2
#define CFG_RATE 1680
#define CFG_CRED 41
#define CFG_GRID 36
#define CFG_EPS 0
#define CFG_SPIN 8
constexpr unsigned PACE16 = CFG_RATE * CFG_WIN / 64;
#define CFG_OTN 1
#define CFG_OTC 3
#define CFG_PAT 4
#define CFG_ROT 1
#define CFG_DRY 0
#define CFG_GD 8
#define CFG_GS2 8
#define CFG_PRE 1
constexpr int ops_after(int k, int shallow) { return (k == 0 ? 5 : k == 2 ? 2 : 1) + shallow; }
constexpr int ops_since(int k, int back, int shallow) { int n = back - 1; for (int i = 1; i <= back; ++i) n += ops_after((k - i) & 3, shallow); return n; }
static_assert(ops_since(0, 3, 0) <= ops_since(0, 2, 1), "early announcements");

#define CFG_EA 8
#define CFG_EB 4
#define CFG_ER 80
#define CFG_ES 17
#define CFG_EMIN 16
constexpr int NTAB = 4 * (G0 + GM);
static_assert(NTAB <= 256, "byte tables");
struct Order {
    unsigned char from[256], to[256];
    static constexpr int due(int o) {
        const int d = o >= 4 * G0 ? CFG_ER + (o - 4 * G0) / GM * CFG_ES + ((o - 4 * G0) % GM & 1) * CFG_EB
                                  : o / G0 * CFG_ES + (o % G0 < GW ? o % G0 / (GW / 2) * CFG_EB : (o % G0 - GW) / 4 * CFG_EA);
        return d < CFG_EMIN ? CFG_EMIN : d;
    }
    constexpr Order() : from(), to() {
        for (int o = 0; o < NTAB; ++o) {
            int n = 0;
            for (int k = 0; k < NTAB; ++k) n += due(k) < due(o) || (due(k) == due(o) && k < o);
            to[o] = (unsigned char)n; from[n] = (unsigned char)o;
        }
    }
};
constexpr Order ORDER;
static_assert(ORDER.to[NS - 1] == NS - 1, "the leading units come first");

struct Params {
    const void* x; const void* w;
    char* wsl; char* outl;
    int rank, pad[7];
    char* ws[NRANK]; char* out[NRANK];
    Order order;
};
__device__ __forceinline__ int look_up(unsigned tab, int o) { return o < NTAB ? (unsigned)__builtin_amdgcn_readlane(tab, o >> 2) >> (o & 3) * 8 & 255u : o; }

__device__ __forceinline__ rsrc_t mk_rsrc(const void* p, unsigned bytes) {
    const unsigned long long v = (unsigned long long)p;
    const unsigned lo = __builtin_amdgcn_readfirstlane((unsigned)v), hi = __builtin_amdgcn_readfirstlane((unsigned)(v >> 32));
    return __builtin_amdgcn_make_buffer_rsrc((void*)(((unsigned long long)hi << 32) | lo), 0, bytes, 0x00020000);
}
__device__ __forceinline__ u32x4 mk_words(const void* p, unsigned bytes) {
    const unsigned long long v = (unsigned long long)p;
    return (u32x4){(unsigned)__builtin_amdgcn_readfirstlane((unsigned)v), (unsigned)__builtin_amdgcn_readfirstlane((unsigned)(v >> 32)) & 0xffffu, bytes, 0x00020000u};
}
__device__ u32x4 buffer_load16(u32x4 rsrc, unsigned voff, unsigned soff, int aux) __asm("llvm.amdgcn.raw.buffer.load.v4i32");
__device__ unsigned buffer_swap(unsigned v, rsrc_t rsrc, unsigned voff, unsigned soff, int aux) __asm("llvm.amdgcn.raw.ptr.buffer.atomic.swap.i32");
__device__ __forceinline__ unsigned ld_sys(const unsigned* p) { return __hip_atomic_load(p, __ATOMIC_RELAXED, __HIP_MEMORY_SCOPE_SYSTEM); }
__device__ __forceinline__ unsigned ld_agent(const unsigned* p) { return __hip_atomic_load(p, __ATOMIC_RELAXED, __HIP_MEMORY_SCOPE_AGENT); }
__device__ __forceinline__ void st_sys(unsigned* p, unsigned v) { __hip_atomic_store(p, v, __ATOMIC_RELAXED, __HIP_MEMORY_SCOPE_SYSTEM); }
__device__ __forceinline__ void st_agent(unsigned* p, unsigned v) { __hip_atomic_store(p, v, __ATOMIC_RELAXED, __HIP_MEMORY_SCOPE_AGENT); }
__device__ __forceinline__ void add_agent(unsigned* p, unsigned v) { (void)__hip_atomic_fetch_add(p, v, __ATOMIC_RELAXED, __HIP_MEMORY_SCOPE_AGENT); }
#define STORE16(v, rs, vo, so, aux) do { const u32x4 _d = (v); __builtin_amdgcn_raw_buffer_store_b128(_d, rs, vo, so, aux); asm volatile("s_nop 1" :: "v"(_d) : "memory"); } while (0)
#define WAIT_VM0() asm volatile("s_waitcnt vmcnt(0)" ::: "memory")
#define WAIT_VMN(n) do { asm volatile("" ::: "memory"); __builtin_amdgcn_s_waitcnt(0x0F70 | (n)); asm volatile("" ::: "memory"); } while (0)

__device__ __forceinline__ bool give_up(unsigned* tmo, unsigned call, unsigned long long t0) {
    const unsigned long long dt = __builtin_amdgcn_s_memrealtime() - t0;
    if (dt < 2000ull) return false;
    if ((unsigned)__builtin_amdgcn_readfirstlane(ld_agent(tmo)) == call) return true;
    if (dt > DEADLINE) { st_agent(tmo, call); return true; }
    return false;
}

__device__ __forceinline__ int gpan_of(int v, int consumer) { return v < NS ? consumer * NS + v : (NRANK - 1) * NS + v; }
__device__ __forceinline__ int opan_of(int y, int consumer) { return ((consumer + 1 + y % 7) & 7) * NS + NS - 1 - y / 7; }
__device__ __forceinline__ int spos_of(int v, int consumer) { return v < NB ? consumer * NB + v : v < NS ? 4 * UQ + (NS - 1 - v) * 4 * NRANK + consumer : NRANK * NB - NS + v; }
__device__ __forceinline__ int step_of(int v) { return v >= NB && v < NS ? NRANK : UQ; }
__device__ __forceinline__ int opos_of(int y, int consumer) { return 4 * UQ + y / 7 * 4 * NRANK + ((consumer + 1 + y % 7) & 7); }
__device__ __forceinline__ int col_of(int v, int consumer) { return (v < NWP ? gpan_of(v, consumer) : opan_of(v - NWP, consumer)) * 256; }
__device__ __forceinline__ int band_of(int e, int consumer) { return e < 7 ? (consumer + 1 + e) & 7 : e < 15 ? 8 + ((consumer + e) & 7) : consumer; }
__device__ __forceinline__ int unit_of(int o, int consumer, int& q, int& pos) {
    if (o < 4 * G0) { q = o / G0; const int m = o % G0; pos = q * step_of(m) + spos_of(m, consumer); return m < GW ? NTM + m : m - GW; }
    pos = o + 4 * (NRANK - 1) * NS;
    o -= 4 * G0;
    const int g = o / (4 * GM), r = o % (4 * GM);
    int i;
    if (GL == 0 || g < NGF) { q = r / GM; i = r % GM; } else { q = r / (GL ? GL : 1); i = r % (GL ? GL : 1); }
    return NTM + GW + GM * g + i;
}

__device__ __forceinline__ unsigned cvt_pk_bf16(float lo, float hi) { unsigned r; asm("v_cvt_pk_bf16_f32 %0, %1, %2" : "=v"(r) : "v"(lo), "v"(hi)); return r; }
__device__ __forceinline__ float silu(float y) {
    const float c = __builtin_amdgcn_fmed3f(y, -60.0f, 60.0f);
    const float e = __builtin_amdgcn_exp2f(c * -1.44269504088896f);
    return y * __builtin_amdgcn_rcpf(1.0f + e);
}

typedef float f32x2 __attribute__((ext_vector_type(2)));
__device__ __forceinline__ f32x2 silu2(f32x2 y) {
    const f32x2 t = __builtin_elementwise_fma(y, (f32x2){-1.44269504088896f, -1.44269504088896f}, (f32x2){-86.5617024533378f, -86.5617024533378f});
    const f32x2 e = {__builtin_amdgcn_fmed3f(__builtin_amdgcn_exp2f(t[0]), 0.0f, 1.0f), __builtin_amdgcn_fmed3f(__builtin_amdgcn_exp2f(t[1]), 0.0f, 1.0f)};
    const f32x2 d = __builtin_elementwise_fma(e, (f32x2){1.1420073898156842e26f, 1.1420073898156842e26f}, (f32x2){1.0f, 1.0f});
    return y * (f32x2){__builtin_amdgcn_rcpf(d[0]), __builtin_amdgcn_rcpf(d[1])};
}

#define STAMP1(P, slot) do {} while (0)
#define STAMP(P, slot) do {} while (0)

template <class T> __device__ __forceinline__ T now_s(T x) { asm volatile("" : "+s"(x)); return x; }
__device__ __forceinline__ int now_v(int x) { asm volatile("" : "+v"(x)); return x; }
__device__ __forceinline__ void pull_role(const Params& P, unsigned tk, unsigned* tkp, int peer, int widx, int lane, int from, LAS u32x4* tab) {
    const int rank = P.rank;
    char* wsl = P.wsl;
    const rsrc_t rwl = mk_rsrc(wsl, WS_END), rol = mk_rsrc(P.outl, OUTB);
    const u32x4 rwr = mk_words(P.ws[peer], WS_END), ror = mk_words(P.out[peer], OUTB);
    {
        const int y = threadIdx.x, x = y - NO * 32, r = x % (2 * NB), band = band_of(x / (2 * NB), rank);
        const int v = x < 0 ? NS - 1 - y / 32 : r % NB, half = x < 0 ? y % 32 : band * 2 + r / NB, at = x < 0 ? y : NO * 32 + (band * NB + v) * 2 + r / NB;
        tab[NPI2 / 16 + y] = (u32x4){WS_OSN + (unsigned)at * UNB, (unsigned)half * (128u * NN * 2u) + (unsigned)(peer * NS + v) * 512u,
                                     x >= 0 && x < (NRANK - 1) * 2 * NB && r >= 2 * NB - ZB / 16 ? WS_ZC + (unsigned)(band * NRANK + peer) * 128u : 0u, 0u};
    }
    auto input = [&](int pan, int q, int pos, unsigned c) {
        u32x4 t = {WS_SND + (unsigned)pos * UNB, WS_PAN + (unsigned)pan * PANB + (unsigned)peer * CHB + (unsigned)q * UNB, WS_CNT + (unsigned)(pan * NRANK + peer) * 128u + (unsigned)q * 4u, 0u};
        asm volatile("" : "+v"(t), "+v"(c));
        t[0] += (~c & 1u) * SNDB;
        return t;
    };
    if (threadIdx.x < NPI2 / 16) {
        int q, pos;
        const int e = now_v(threadIdx.x);
        int pan = unit_of(e + NPUSH < NTAB ? from : e + NPUSH, rank, q, pos);
        if (NO && e >= NPIN / 16) {
            const int z = e - NPIN / 16, y = z / 28 * 7 + (z % 7 + 7 - rank) % 7;
            q = z / 7 & 3; pan = NIN + y; pos = q * NRANK + opos_of(y, rank);
        }
        tab[e] = input(pan, q, pos, tk);
    }
    const unsigned call = __builtin_amdgcn_readfirstlane(now_v(tk)) + 1u;
    st_agent(tkp, call);
    unsigned* tmo = (unsigned*)(wsl + WS_TMO);
    constexpr unsigned OOB = 0xfffffff0u;
    const unsigned lane0 = lane == 0 ? 0u : OOB;
    const unsigned hoff = WS_PH + (call & 1u) * (NRANK * 128u);
    const unsigned head = lane == 0 ? hoff + (unsigned)peer * 128u : OOB;
    unsigned dvo = CFG_DRY ? OOB : head;
    auto draw = [&]() { return (unsigned)__builtin_amdgcn_raw_ptr_buffer_atomic_add_i32(1, rwl, dvo, 0, 0); };
    __builtin_amdgcn_raw_buffer_store_b32(0u, rwl, widx ? OOB : lane == 0 ? WS_PH + (~call & 1u) * (NRANK * 128u) + (unsigned)peer * 128u : lane < 3 ? WS_HEAD + (~call & 1u) * 128u + (unsigned)(lane & 1) * 64u : OOB, 0, AUX_SYS);
    auto arrive = [&](unsigned vo, unsigned cnt) { (void)__builtin_amdgcn_raw_ptr_buffer_atomic_add_i32(PU, rwl, vo, cnt, 0); };
    const int wrk = (widx & 7) * (NW / 8) + (widx >> 3);
    const bool deep0 = wrk < DEEP;
    const unsigned per0 = deep0 ? PACE16 / (BASE + 1) : PACE16 / BASE;
    asm volatile("s_waitcnt lgkmcnt(0)" ::: "memory");
    __builtin_amdgcn_s_barrier();
    bool bad = false; int stamped = 0; (void)stamped;
    int skip = NO ? -1 : 0, nout = NPIN;
    auto index_of = [&](unsigned v) { const int i = __builtin_amdgcn_readfirstlane(v) + NW; return i >= nout && skip > 0 ? i + skip : i; };
    int cur = index_of((unsigned)(widx - NW));
    const unsigned fso = WS_FLAG + (unsigned)peer * FLAGB;
    auto first_of = [&](int i) { return i < NPIN ? (i >> (6 - PSH)) + NPUSH : NTURN; };
    int lim = 0, flim = 0, fin = NPC;
    bool planned = ZB == 0;
    unsigned long long plan = 0;
    auto moved = [&](int j) { return (int)(signed char)(plan >> (j * 8)); };
    auto update = [&](const u32x4& f0, const u32x4& f1) {
        const int first = first_of(cur) - lane * 8;
        unsigned un = 0;
#pragma unroll
        for (int j = 0; j < 8; ++j) un |= ((j < 4 ? f0[j & 3] : f1[j & 3]) - call) >> 31 << j;
        un &= 255u << (first < 0 ? 0 : first > 8 ? 8 : first);
        const int l = __builtin_ctzll(__ballot(un != 0u) | (1ull << 63));
        const int u = l * 8 + __builtin_ctz(__builtin_amdgcn_readlane(un, l) | 256u);
        const int fl = u <= NTURN ? (u - NPUSH) << (6 - PSH) : NPC + 128;
        flim = fl > flim ? fl : flim;
        if (!planned && (unsigned)__builtin_amdgcn_readlane(f0[0], 63) == call && (unsigned)__builtin_amdgcn_readlane(f0[2], 63) == call) {
            plan = (unsigned long long)(unsigned)__builtin_amdgcn_readlane(f0[3], 63) << 32 | (unsigned)__builtin_amdgcn_readlane(f0[1], 63);
            planned = true; fin = NPC - moved(peer);
        }
        if (skip < 0 && (unsigned)__builtin_amdgcn_readlane(f1[0], 63) - call * 4u < 4u) {
            const int m = __builtin_amdgcn_readlane(f1[0], 63) & 3;
            skip = NO * OIB + m * OSK; nout = NPIN + m * OIB;
            if (cur >= nout) cur += skip;
        }
        const int pl = skip < 0 ? NPIN : planned ? fin : NPC - ZB;
        lim = flim < pl ? flim : pl;
    };
    auto make_plan = [&](int lane, int rank, unsigned call, char* wsl) {
        unsigned old = 0;
        if (lane == 0) old = __hip_atomic_exchange((gptr_t)(wsl + WS_PH + 2 * NRANK * 128), call, __ATOMIC_RELAXED, __HIP_MEMORY_SCOPE_AGENT);
        if ((unsigned)__builtin_amdgcn_readfirstlane(old) == call) return;
        const unsigned hd = __builtin_amdgcn_raw_buffer_load_b32(rwl, lane < NRANK ? (unsigned)lane * 128u : OOB, hoff, AUX_SYS);
        const int t = lane >> 3, s = lane & 7;
        const unsigned zc = ld_sys((const unsigned*)(P.ws[t] + WS_ZC + (rank * NRANK + s) * 128));
        const bool ok = __all(t == rank || s == rank || s == t || (int)(zc - call * (unsigned)(ZB * PU)) >= 0);
        int h[NRANK], sum = 0, give = 0, take = 0;
#pragma unroll
        for (int j = 0; j < NRANK; ++j) { h[j] = __builtin_amdgcn_readlane(hd, j); sum += j == rank ? 0 : h[j]; }
#pragma unroll
        for (int j = 0; j < NRANK; ++j) {
            const int d = j == rank ? 0 : (sum - (NRANK - 1) * h[j]) * CFG_BAL / (8 * (NRANK - 1));
            h[j] = d > ZB ? ZB : d < -127 ? -127 : d;
            if (h[j] > 0) give += h[j]; else take -= h[j];
        }
        give = take = !ok ? 0 : give < take ? give : take;
        unsigned long long pl = 0;
#pragma unroll
        for (int j = 0; j < NRANK; ++j) {
            int d = h[j];
            if (d > 0) { d = d < give ? d : give; give -= d; } else { d = -d < take ? d : -take; take += d; }
            pl |= (unsigned long long)(d & 255) << (j * 8);
        }
        __builtin_amdgcn_raw_buffer_store_b128((u32x4){call, (unsigned)pl, call, (unsigned)(pl >> 32)}, rwl, lane < NRANK ? WS_FLAG + 63 * 32 + (unsigned)lane * FLAGB : OOB, 0, AUX_SYS);
    };
    auto decide = [&](int lane, int rank, unsigned call, char* wsl) {
        unsigned old = 0;
        if (lane == 0) old = __hip_atomic_exchange((gptr_t)(wsl + WS_PH + 2 * NRANK * 128 + 64), call, __ATOMIC_RELAXED, __HIP_MEMORY_SCOPE_AGENT);
        if ((unsigned)__builtin_amdgcn_readfirstlane(old) == call) return;
        const int c = lane >> 3, r = lane & 7, np = CFG_NP >> c * 4 & 15;
        const u32x4 st = __builtin_amdgcn_raw_buffer_load_b128(rwl, (unsigned)(c * 128 + r * 8), WS_COL, AUX_SYS);
        const unsigned now = (unsigned)__builtin_amdgcn_s_memrealtime();
        const unsigned long long have = __ballot(r >= 1 && r < RS - 1 && st[0] == call && st[2] == call);
        auto shorter = [&](int to) { const int n = to - pr0_of(np) + 1; return n < 0 ? 0 : n > np ? np : n; };
        const float tau = (float)(st[3] - st[1]) * (shorter(r + 1) != shorter(r) ? 1.0f / CFG_QR : 1.0f);
        float arr = (float)(int)(st[3] - now) + tau * (shorter(r + 2) != shorter(r + 1) ? CFG_QR : 1.0f);
        arr = (arr < 0.f ? 0.f : arr) + tau * ((float)(RS - 3 - r) - (1.0f - CFG_QR) * (float)(np - shorter(r + 2)));
        if (r + 2 == RS) arr = (float)(int)(st[3] - now);
        const bool last = (have >> lane & 1) && !(have >> lane & 2);
        int m = 0;
        unsigned cuts = CUTS;
        if (__builtin_popcountll(__ballot(last)) == 8) {
            int before = 0, after = 0;
            unsigned long long b = __ballot(last);
            float unit = 0.0f;
#pragma unroll
            for (int k = 0; k < 8; ++k) {
                const int l = __builtin_ctzll(b); b &= b - 1;
                const float a = __builtin_bit_cast(float, __builtin_amdgcn_readlane(__builtin_bit_cast(int, arr), l)), t = __builtin_bit_cast(float, __builtin_amdgcn_readlane(__builtin_bit_cast(int, tau), l));
                before += a < arr ? NG : 0; after += a + t < arr + tau || (a + t == arr + tau && k < c) ? NG : 0;
                unit += t * (0.95f / 256);
            }
            float ts[8];
#pragma unroll
            for (int k = 0; k < 8; ++k) ts[k] = __builtin_bit_cast(float, __builtin_amdgcn_readlane(__builtin_bit_cast(int, arr + tau), __builtin_ctzll(__ballot(last && after == k * NG) | 1ull << 63)));
            const float ta = (lane == 0 ? ts[1] : lane == 1 ? ts[2] : ts[3]) + CFG_TP, tb = (lane == 0 ? ts[4] : ts[5]) + CFG_TC, tc = (lane == 0 ? ts[7] : lane == 1 ? ts[6] : ts[5]) + CFG_TC;
            const float inv = 1.0f / unit;
            const float lc = __builtin_amdgcn_fmed3f(__builtin_rintf(((ta + tb - 2 * tc) * inv + 32.0f) * (1.0f / 3)), 4.0f, 14.0f);
            const float lb = __builtin_amdgcn_fmed3f(__builtin_rintf(((ta - tb) * inv + 32.0f - lc) * 0.5f), 4.0f, 14.0f);
            const float cend = fmaxf(fmaxf(ta + (32.0f - lc - lb) * unit, tb + lb * unit), fmaxf(tc + lc * unit, ts[0] + CFG_TC + (32 - CFG_L2) * unit)) + CFG_TF;
            const int cw = min(max((int)lb, 4), 14) | (32 - min(max((int)lc, 4), 14)) << 5;
#pragma unroll
            for (int k = 1; k <= NO; ++k) {
                const bool whole = before < 8 * full_of(k);
                const int at = ((whole ? after + 8 * NG : before) >> 3) - full_of(k);
                int u0, u1, p;
                (void)piece_of(at, two_of(k), three_of(k), CUTS, u0, u1, p);
                const bool dyn = CFG_DYN && full_of(k) == NG;
                const float end = dyn ? cend : arr + tau * ((whole ? 1.0f : 0.0f) + (at < items_of(k) - full_of(k) ? (float)(u1 - u0) * (0.95f / 32) + CFG_QH : 0.0f));
                if (!__any((dyn || last) && end > (float)(CFG_DEC + NPC - NPI2 - (k - 1) * OSK) * (CFG_LT / 256.0f) - CFG_MARG)) m = k;
            }
            if (CFG_DYN && full_of(NO) == NG && m == NO) cuts = __builtin_amdgcn_readlane(cw, 0) | __builtin_amdgcn_readlane(cw, 1) << 10 | __builtin_amdgcn_readlane(cw, 2) << 20;
        }
        const unsigned full = call * CNT_FULL;
#pragma unroll
        for (int k = 0; k < NO; ++k)
            __builtin_amdgcn_raw_buffer_store_b128((u32x4){full, full, full, full}, rwl, k >= m && lane < 56 && r != rank ? (unsigned)lane * 128u : OOB, WS_CNT + (unsigned)(NIN + 7 * k) * (NRANK * 128u), AUX_SYS);
        __builtin_amdgcn_raw_buffer_store_b64((u32x2){call * 4u + (unsigned)m, cuts}, rwl, lane < NRANK ? WS_FLAG + 63 * 32 + 16 + (unsigned)lane * FLAGB : OOB, 0, AUX_SYS);
    };
    rsrc_t rd = rwl;
    u32x4 rs = rwr;
    const int rot = CFG_ROT == 1 ? wrk : CFG_ROT == 2 ? widx : CFG_ROT == 3 ? widx >> 1 : 0;
    auto piece = [&](int k) -> unsigned {
        k = k + rot & 3;
        const int q = lane >> 4;
        return CFG_PAT == 0 ? lane * 16 + k * 1024 : CFG_PAT == 2 ? q * 1024 + (k + q & 3) * 256 + (lane & 15) * 16
             : CFG_PAT == 4 ? (lane >> 3) * 512 + k * 128 + (lane & 7) * 16
             : CFG_PAT == 8 ? (lane >> 2) * 256 + k * 64 + (lane & 3) * 16
             : CFG_PAT == 9 ? (lane >> 3) * 512 + (k + (lane >> 3) & 3) * 128 + (lane & 7) * 16
             : (lane >> 3) * 512 + ((k & 1) * 2 + (k >> 1)) * 128 + (lane & 7) * 16;
    };
    unsigned o[PU], in[PU], rows[PU], src, dst, cb, dsh = 12, dadd = 0, per = per0, fvo = CFG_DRY ? OOB - 16u : (unsigned)lane * 32u;
#pragma unroll
    for (int k = 0; k < PU; ++k) { o[k] = in[k] = CFG_DRY ? OOB : piece(k); rows[k] = (piece(k) & 511u) + (piece(k) >> 9) * (NN * 2u); }
    bool tail = CFG_DRY, deep = deep0, look = true;
    int next = CFG_DRY ? NPC : 0, ti;
    auto places = [&]() {
        const u32x4 t = tab[ti >> 4];
        dst = __builtin_amdgcn_readfirstlane(t[1]) + ((ti & 15) << dsh) + dadd;
        src = ZB && cur >= NPC ? dst : __builtin_amdgcn_readfirstlane(t[0]) + ((ti & 15) << 12);
        cb = __builtin_amdgcn_readfirstlane(t[2]);
    };
    auto turn = [&]() {
        if (ZB && cur == NPC - CFG_PLAN) make_plan(now_v(lane), now_s(rank), now_s(call), now_s(wsl));
        if (NO && cur == NPIN - CFG_DEC) decide(now_v(lane), now_s(rank), now_s(call), now_s(wsl));
        const int even = fin - (deep0 ? CFG_EVEND : CFG_EVEN);
        tail = cur >= fin - CFG_TAIL; deep = deep0 || tail || cur >= even;
        per = cur >= even ? CFG_ERATE * NW / 64 : per0;
        if (cur >= nout) {
            rd = rol; dsh = 19;
#pragma unroll
            for (int k = 0; k < PU; ++k) o[k] = rows[k];
        }
        const int all = flim < NPC ? flim - CFG_LOOK : 2 * NPC, one = skip < 0 ? NPIN - CFG_LOOKP - wrk * CFG_LSP : planned ? 2 * NPC : NPC - ZB - CFG_LOOKP - wrk * CFG_LSP2;
        look = cur >= all || cur >= one;
        fvo = cur >= all || (look && lane == 63) ? (unsigned)lane * 32u : OOB - 16u;
        next = all < one ? all : one;
        next = next < lim ? next : lim;
        const int at[6] = {NPC - CFG_PLAN, NPIN - CFG_DEC, even, fin - CFG_TAIL, nout, NPC};
#pragma unroll
        for (int j = 0; j < 6; ++j) next = at[j] > cur && at[j] < next ? at[j] : next;
        if (ZB && cur >= NPC) {
            int x = cur - NPC;
            const int me = now_s(peer);
            for (int j = 0; j < me; ++j) x -= moved(j) < 0 ? moved(j) : 0;
            for (int j = 0; j < NRANK; ++j) { const int g = moved(j) > 0 ? moved(j) : 0; if ((unsigned)x < (unsigned)g) { dadd = (unsigned)(j - me) * (SCOL * 2u); ti = NPC - g + x; } x -= g; }
            rs = ror; next = 0; ti = __builtin_amdgcn_readfirstlane(ti); dadd = __builtin_amdgcn_readfirstlane(dadd);
#pragma unroll
            for (int k = 0; k < PU; ++k) in[k] = rows[k];
        }
    };
    int first = CFG_DRY ? 2 : 1;
    while (!planned || cur < fin) {
        const bool lead = first == 1 && cur < (NL - NPUSH) << (6 - PSH);
        if (first == 2 || lead) lim = flim = cur + 1;
        if (!bad && cur >= lim) {
            const unsigned long long t0 = __builtin_amdgcn_s_memrealtime();
            const unsigned pvo = (unsigned)(lane - (first_of(cur) >> 3)) < CFG_PWIN || (lane == 63 && cur >= NPIN) ? (unsigned)lane * 32u : OOB - 16u;
            for (;;) {
                const u32x4 f0 = __builtin_amdgcn_raw_buffer_load_b128(rwl, pvo, fso, AUX_SYS), f1 = __builtin_amdgcn_raw_buffer_load_b128(rwl, pvo + 16u, fso, AUX_SYS);
                update(f0, f1);
                if (cur < lim || (planned && cur >= fin)) break;
                if (give_up(tmo, call, t0)) { bad = true; planned = true; plan = 0; fin = NPC; skip = skip < 0 ? 0 : skip; break; }
                __builtin_amdgcn_s_sleep(CFG_PS);
            }
            asm volatile("" ::: "memory");
            if (cur >= fin && planned) break;
        }
        ti = cur;
        if (bad) { dsh = 19; dadd = 0; }
        else if (!first) turn();
        if (first == 2) ti = NPC;
        places();
        if (lead) {
            const unsigned long long t0 = __builtin_amdgcn_s_memrealtime();
            const unsigned fo = fso + 2048u + (unsigned)(((cur >> 3) + 2 * NPUSH) * 2 + (cur >> 1 & 1)) * 8u;
            for (;;) {
                const unsigned f = __builtin_amdgcn_raw_buffer_load_b32(rwl, (unsigned)(lane & 1) * 4u, fo, AUX_SYS);
                if (__all((int)(f - call) >= 0)) break;
                if (give_up(tmo, call, t0)) { bad = true; planned = true; skip = 0; break; }
                __builtin_amdgcn_s_sleep(CFG_PS);
            }
            asm volatile("" ::: "memory");
        }
        if (bad) {
#pragma unroll
            for (int k = 0; k < PU; ++k) STORE16(((u32x4){0x7fc07fc0u, 0x7fc07fc0u, 0x7fc07fc0u, 0x7fc07fc0u}), rol, cur >= NPI2 ? rows[k] : OOB, dst, AUX_AGENT);
            cur = index_of(draw()); continue;
        }
        if (first < 2) STAMP1(P, 1);
        unsigned a[PU] = {OOB, OOB, OOB, OOB}, cprev = 0, cold = 0, dp = 0;
        rsrc_t rp = rwl;
        u32x4 v[PU] = {};
        auto put = [&](int k) { STORE16(v[k & 3], k < 0 ? rp : rd, a[k & 3], k < 0 ? dp : dst, CFG_PST); };
        unsigned tick = (unsigned)__builtin_amdgcn_s_memrealtime();
        unsigned slot = (tick << 4) - (tick << 4 & 0xffffffu) % PACE16 + (unsigned)wrk * (PACE16 / NW) - PACE16;
        tick += CFG_GRID - tick % CFG_GRID;
        bool early = false;
        do {
            unsigned nv; u32x4 f0, f1;
            auto step = [&](auto kc) {
                constexpr int k = kc;
                if (!tail) {
                    slot += per;
                    if ((int)((tick << 4) - slot) < 0) tick += CFG_GRID;
                    if ((int)((tick << 4) - slot) < 0) tick += CFG_GRID;
                    asm volatile("" : "+s"(tick), "+s"(slot));
                    WAIT_VMN(ops_since(k, BASE + 1, 0));
                    if (!deep) WAIT_VMN(ops_since(k, BASE, 1));
                    asm volatile("s_memrealtime vcc\n\ts_waitcnt lgkmcnt(0)\n\ts_branch 2f\n"
                                 "1:\ts_add_u32 %0, %0, %3\n2:\ts_sub_u32 vcc_hi, %0, vcc_lo\n\ts_cmp_lt_i32 vcc_hi, %5\n\ts_cbranch_scc1 1b\n"
                                 "3:\ts_memrealtime vcc\n\ts_waitcnt lgkmcnt(0)\n\ts_sub_u32 vcc_lo, %0, vcc_lo\n\ts_cmp_le_i32 vcc_lo, 0\n\ts_cbranch_scc1 4f\n\t"
                                 "s_cmp_le_i32 vcc_lo, %4\n\ts_cbranch_scc1 3b\n\ts_sleep 1\n\ts_branch 3b\n4:"
                                 : "+s"(tick), "+s"(rs), "+s"(src) : "n"(CFG_GRID), "n"(CFG_SPIN), "n"(CFG_PRE) : "scc", "vcc", "memory");
                }
                WAIT_VMN(ops_since(k, BASE + 1, 0));
                v[k] = buffer_load16(rs, in[k], src, CFG_RLD);
                __builtin_amdgcn_sched_barrier(0);
                a[k] = o[k];
                if (k == 0) {
                    nv = draw();
                    f0 = __builtin_amdgcn_raw_buffer_load_b128(rwl, fvo, fso, AUX_SYS); f1 = __builtin_amdgcn_raw_buffer_load_b128(rwl, fvo + 16u, fso, AUX_SYS);
                    arrive(early && cold ? lane0 : OOB, cold);
                    if (early) cold = 0;
                }
                if (k == 2) arrive(cold ? lane0 : OOB, cold);
                put(k - BASE - 1);
                if (!deep) { put(k - BASE); a[(k - BASE) & 3] = OOB; }
                const int d = (int)((tick << 4) - slot);
                if (d > CFG_CRED * 16) slot += (unsigned)(d - CFG_CRED * 16);
                tick += (deep ? CFG_GD : CFG_GS2) >> k & 1 ? 2 * CFG_GRID : CFG_GRID;
            };
            step(std::integral_constant<int, 0>()); step(std::integral_constant<int, 1>()); step(std::integral_constant<int, 2>()); step(std::integral_constant<int, 3>());
            asm volatile("" ::: "memory");
            early = !deep; cold = cprev; cprev = cb; rp = rd; dp = dst;
            ti = cur = index_of(nv);
            if (__builtin_expect(cur >= next, 0)) {
                if (look) update(f0, f1);
                ti = cur;
                if (cur < lim) turn();
            }
            places();
        } while (cur < lim);
#pragma unroll
        for (int k = -1 - BASE; k < 0; ++k) put(k);
        if (cprev | cold) { WAIT_VM0(); arrive(cprev ? lane0 : OOB, cprev); arrive(cold ? lane0 : OOB, cold); }
        if (first == 2) {
            cur = widx; lim = flim = next = 0; dvo = head; tail = false; fvo = (unsigned)lane * 32u;
#pragma unroll
            for (int k = 0; k < PU; ++k) o[k] = in[k] = piece(k);
        }
        first >>= 1;
    }
    STAMP(P, 4);
}

__device__ __forceinline__ void load_w(u32x4 (&v)[8], const rsrc_t rw, int col, int q, int job, int lane) {
    const unsigned vo = (unsigned)((lane >> 4) * 8) * (NN * 2) + (unsigned)(lane & 15) * 16u;
    const unsigned so = (unsigned)(q * 128 + (job >> 1) * 32) * (NN * 2) + (unsigned)(col + (job & 1) * 128) * 2u;
#pragma unroll
    for (int j = 0; j < 8; ++j) v[j] = __builtin_amdgcn_raw_buffer_load_b128(rw, vo, so + (unsigned)j * (NN * 2), 2);
}
__device__ __forceinline__ void store_w(const u32x4 (&v)[8], const rsrc_t rd, unsigned vo, unsigned unit, int job, bool slow) {
    unit += (unsigned)(job >> 2) * KTB + (unsigned)(job & 1) * 8192u + (unsigned)(job >> 1 & 1) * 1024u;
#pragma unroll
    for (int i = 0; i < 8; ++i) {
        u32x4 o;
#pragma unroll
        for (int e = 0; e < 4; ++e) o[e] = __builtin_amdgcn_perm(v[2 * e + 1][i >> 1], v[2 * e][i >> 1], (i & 1) ? 0x07060302u : 0x05040100u);
        STORE16(o, rd, vo, unit + (unsigned)(i >> 2) * HTB + (unsigned)(i & 3) * 2048u, AUX_SYS);
        if (CFG_PWND && i >= CFG_PWND - 1 && i < 7 && slow) WAIT_VMN(CFG_PWND ? CFG_PWND - 1 : 0);
    }
}

__device__ __forceinline__ void lead_role(const Params& P, int rank, unsigned long long wsp, unsigned tk, unsigned t0, int g, int wid, int lane) {
    const int o = NL - 1 - (g >> 5), c = (g >> 3) & 3, job = (c >> 1) * 4 + wid * 2 + (c & 1);
    const int dest = (rank + 1 + g + (o < NPUSH ? CFG_MIX * (2 * c + o) : 0)) & (NRANK - 1);
    u32x4 v[8];
    load_w(v, mk_rsrc(P.w, (unsigned)KSH * NN * 2), dest * SCOL + o * 256, 0, job, lane);
    asm volatile("" : "+v"(tk), "+v"(wsp) :: "memory");
    const bool push = o < NPUSH && dest != rank, count = o < NPUSH || dest == rank;
    char* wsd = (char*)((unsigned long long)(unsigned)__builtin_amdgcn_readlane((unsigned)(wsp >> 32), dest) << 32 | (unsigned)__builtin_amdgcn_readlane((unsigned)wsp, dest));
    gptr_t fp = (gptr_t)(wsd + (count ? WS_CNT + (unsigned)((NTM + o) * NRANK + rank) * 128u : WS_FLAG + (unsigned)rank * FLAGB + 2048u + (unsigned)((o * 4 + c) * 2 + wid) * 4u));
    while (CFG_P3 && push && o >= CFG_P3N && (int)((unsigned)__builtin_amdgcn_s_memrealtime() - t0) < CFG_P3 + (o - CFG_P3N) * 110) __builtin_amdgcn_s_sleep(1);
    store_w(v, mk_rsrc(push ? wsd : P.wsl, WS_END), (unsigned)lane * 16u + (push ? 0u : (~tk & 1u) * SNDB),
            push ? WS_PAN + (unsigned)(NTM + o) * PANB + (unsigned)rank * CHB : WS_SND + (unsigned)(dest * NB + o) * UNB, job, push);
    WAIT_VMN(0);
    STAMP(P, 6);
    if (lane == 0) {
        if (!count) __hip_atomic_store(fp, tk + 1u, __ATOMIC_RELAXED, __HIP_MEMORY_SCOPE_SYSTEM);
        else if ((__hip_atomic_fetch_add((gptr_t)(P.wsl + WS_LEAD + (o * NRANK + dest) * 64), 1u, __ATOMIC_RELAXED, __HIP_MEMORY_SCOPE_AGENT) & 7u) == 7u)
            __hip_atomic_store(fp, (tk + 1u) * CNT_FULL, __ATOMIC_RELAXED, __HIP_MEMORY_SCOPE_SYSTEM);
    }
}

__device__ __forceinline__ void pack_role(const Params& P, unsigned long long wsp, unsigned tk, unsigned t0, int g, int wid, int lane) {
    const int rank = P.rank;
    const rsrc_t rx = mk_rsrc(P.x, (unsigned)MM * KSH * 2), rw = mk_rsrc(P.w, (unsigned)KSH * NN * 2), rwl = mk_rsrc(P.wsl, WS_END);
    const unsigned sv = (unsigned)lane * 16u + (~tk & 1u) * SNDB;
    const unsigned to = ((const unsigned*)P.order.to)[lane];
    constexpr int NH = CFG_UH < NUNIT ? 8 * DK : 0, UH = NH ? CFG_UH : NUNIT;
    const bool late = g >= NPK - NH;
    constexpr int U1 = NRANK * NL, NE = CFG_EH ? NH : 0;
    static_assert(CFG_TH < (UH - U1 - NH) * CFG_PH, "unpaced");
    for (int u = late ? (NE ? U1 : UH) + g - (NPK - NH) : U1 + NE + g; u < (late ? NUNIT : UH); u += !late ? NPK - NH : u < UH ? UH - U1 : NH) {
        int o, dest = -1;
        if (u < 4 * U0) {
            const int r = u % U0;
            if (r < NRANK * NS) { o = r >> 3; dest = (rank + 1 + r) & (NRANK - 1); } else o = r - (NRANK - 1) * NS;
            o += u / U0 * G0;
        } else o = u - 4 * (NRANK - 1) * NS;
        int q, pos; const int pan = unit_of(o, dest < 0 ? 0 : dest, q, pos);
        while ((int)((unsigned)__builtin_amdgcn_s_memrealtime() - t0) < (late ? CFG_TH + (u - UH) * CFG_PH : ((u - CFG_PACE0) * CFG_PACE - (u > CFG_UK ? (u - CFG_UK) * (CFG_PACE - CFG_PACE2) : 0)) / 10)) __builtin_amdgcn_s_sleep(1);
        const unsigned dst = WS_SND + (unsigned)pos * UNB;
        u32x4 v[8];
        if (pan < NTM) {
            const int rb0 = (wid & 3) * 4;
            const unsigned vo = (unsigned)(lane & 15) * (KSH * 2) + (unsigned)(lane >> 4) * 16u;
            const unsigned so = (unsigned)(pan * 256 + rb0 * 16) * (KSH * 2) + (unsigned)(q * 2 + (wid >> 2)) * 128u;
#pragma unroll
            for (int i = 0; i < 8; ++i) v[i] = __builtin_amdgcn_raw_buffer_load_b128(rx, vo + (i & 1) * 64, so + (unsigned)(i >> 1) * (16 * KSH * 2), 0);
#pragma unroll
            for (int i = 0; i < 8; ++i) STORE16(v[i], rwl, sv + (i & 1) * 1024, dst + (unsigned)(wid >> 2) * KTB + (unsigned)(rb0 + (i >> 1)) * 2048u, AUX_SYS);
        } else {
            load_w(v, rw, gpan_of(pan - NTM, dest < 0 ? 0 : dest) * 256, q, wid, lane);
            store_w(v, rwl, sv, dst, wid, false);
        }
        o = look_up(to, o);
        const bool own = lane == rank;
        unsigned long long fp = wsp + (own ? WS_CNT + (unsigned)((pan * NRANK + rank) * 32 + q) * 4u : WS_FLAG + (unsigned)rank * FLAGB + (unsigned)o * 4u);
        unsigned fv = own ? (tk + 1u) * CNT_FULL : tk + 1u;
        asm volatile("" : "+v"(fp), "+v"(fv));
        WAIT_VMN(0);
        __builtin_amdgcn_s_barrier();
        const bool mine = lane < NRANK && (dest < 0 || dest == lane);
        if (wid == 0 && mine && (own || o < CFG_AGG || o >= NTURN / 8 * 8)) __hip_atomic_store((gptr_t)fp, fv, __ATOMIC_RELAXED, __HIP_MEMORY_SCOPE_SYSTEM);
        if (wid == 1 && o >= CFG_AGG && o < NTURN / 8 * 8) {
            unsigned old = 0;
            if (mine && !own) old = __hip_atomic_fetch_add((gptr_t)(P.wsl + WS_GRP) + (o >> 3) * NRANK + lane, 1u, __ATOMIC_RELAXED, __HIP_MEMORY_SCOPE_AGENT);
            const unsigned full = (unsigned)__ballot((old & 7u) == 7u);
            const unsigned long long wsq = (unsigned long long)(unsigned)__builtin_amdgcn_ds_bpermute(lane >> 3 << 2, (int)(wsp >> 32)) << 32 | (unsigned)__builtin_amdgcn_ds_bpermute(lane >> 3 << 2, (int)wsp);
            if (full >> (lane >> 3) & 1) __hip_atomic_store((gptr_t)(wsq + WS_FLAG + (unsigned)rank * FLAGB) + (o & ~7) + (lane & 7), tk + 1u, __ATOMIC_RELAXED, __HIP_MEMORY_SCOPE_SYSTEM);
        }
        if (wid == 2 && dest >= 0 && pan >= NTM + NB) {
            if (lane == 0 && dest != rank) st_sys((unsigned*)(P.wsl + WS_CNT) + ((NIN + (NTM + NS - 1 - pan) * 7 + ((dest - rank - 1) & 7)) * NRANK + rank) * 32 + q, (tk + 1u) * CNT_FULL);
            unsigned old = 0;
            if (lane == 0) old = __hip_atomic_fetch_add((gptr_t)(P.wsl + WS_BAND), 1u, __ATOMIC_RELAXED, __HIP_MEMORY_SCOPE_AGENT);
            if ((unsigned)__builtin_amdgcn_readfirstlane(old) + 1u == (tk + 1u) * NLAST && lane < NRANK && !own)
                __hip_atomic_store((gptr_t)(wsp + WS_FLAG + (unsigned)rank * FLAGB) + NTURN, tk + 1u, __ATOMIC_RELAXED, __HIP_MEMORY_SCOPE_SYSTEM);
        }
    }
}

struct Tile { int tm, v, kind; unsigned sA, stA; };
enum { WHOLE = 0, HEAD = (KT / 2 - 4) << 5 | 1 << 10, REST = 4 | 4 << 5 | 2 << 10, PIECE = 3 << 10 };
__device__ __forceinline__ void tile_of(int idx, int col, int rank, unsigned snd, Tile& t) {
    const int v = idx < 2 * GW ? (idx >> 2) + (col >> 2) * (GW / 2) : 2 * (idx >> 2) + (col >> 2);
    t.tm = 4 * (col & 3) + (idx & 3); t.v = v; t.kind = WHOLE;
    const int g = (v - GW) / GM;
    t.sA = snd + (unsigned)(v < GW ? spos_of(v, rank) : 4 * U0 + g * 3 * GM + v - GW) * UNB;
    t.stA = (unsigned)(v < GW ? step_of(v) : g < NGF ? GM : GL) * UNB;
}
__device__ __forceinline__ void item_of(int q, int m, unsigned cuts, int rank, unsigned snd, Tile& t) {
    const int full = m == 0 ? full_of(0) : m == 1 ? full_of(1) : full_of(2), two = m == 0 ? two_of(0) : m == 1 ? two_of(1) : two_of(2);
    int u0, u1, p;
    const int n = full + piece_of(q - full, two, m == 0 ? three_of(0) : m == 1 ? three_of(1) : three_of(2), cuts, u0, u1, p);
    int at = 0;
    auto from = [&](auto c) { constexpr int l = left_of(c), a = (nt_of(c) - left_of(c + 1)) * 8 + c; at = n >= l ? a : at; };
#define C(i) std::integral_constant<int, i>()
    from(C(0)); from(C(1)); from(C(2)); from(C(3)); from(C(4)); from(C(5)); from(C(6)); from(C(7));
#undef C
    tile_of((at >> 3) + n, at & 7, rank, snd, t);
    if (n >= PN) {
        const int g = (n - PN) / 28, w = (n - PN) % 28, y = (g >> 2) * 7 + (w >> 2);
        t.tm = 4 * (g & 3) + (w & 3); t.v = NWP + y; t.sA = snd + (unsigned)opos_of(y, rank) * UNB; t.stA = NRANK * UNB;
    }
    if (q >= full) t.kind = u0 | (KT / 2 - u1 + u0) << 5 | p << 10 | (n - full) << 13;
}

template <int wr>
__device__ __forceinline__ void gemm_role(const Params& P, unsigned call, int col, int slot, int kx, unsigned t0, LAS unsigned char* lds) {
    const int tid = threadIdx.x, wid = __builtin_amdgcn_readfirstlane(tid >> 6), lane = tid & 63, wc = wid & 3;
    const int rank = P.rank;
    char* wsl = P.wsl;
    const rsrc_t rws = mk_rsrc(wsl, WS_END);
    LAS unsigned* ctl = (LAS unsigned*)(lds + STAGE_BYTES);
    const unsigned voff = (unsigned)tid * 16u;
    const unsigned ldsw = (unsigned)wid * 1024u;
    const int aoff = wr * 8192 + lane * 16, boff = wc * 4096 + lane * 16;

    auto lane_now = [&]() { int tz = threadIdx.x; asm volatile("" : "+v"(tz)); return tz & 63; };
    auto cnt_of = [&](const Tile& t) { const int ln = lane_now(); return WS_CNT + (unsigned)((((ln & 32) ? NTM + t.v : t.tm) * NRANK + ((ln >> 2) & 7)) * 32 + (ln & 3)) * 4u; };
    auto level_of = [&](unsigned c) -> int {
        const unsigned long long miss = __ballot((int)(c - call * CNT_FULL) < 0);
        unsigned m = (unsigned)miss | (unsigned)(miss >> 32);
        const bool own = ((m >> (rank * 4)) & 15u) == 0u;
        m |= m >> 16; m |= m >> 8; m |= m >> 4;
        return own ? 1 + __builtin_ctz(m | 16u) : 0;
    };
    auto ready = [&](const Tile& t, int want, bool join = false, bool first = false) -> int {
        if (wr == 0 && wid == 0) {
            const unsigned* c = (const unsigned*)(wsl + cnt_of(t));
            unsigned* tmo = (unsigned*)(wsl + WS_TMO);
            const unsigned long long t0 = __builtin_amdgcn_s_memrealtime();
            if (join)
                while (__builtin_amdgcn_readfirstlane(ld_agent((const unsigned*)(wsl + WS_COL + col * 128))) != call && !give_up(tmo, call, t0)) __builtin_amdgcn_s_sleep(16);
            int got;
            for (;;) {
                got = level_of(ld_sys(c));
                if (got >= want) break;
                if (give_up(tmo, call, t0)) { ctl[0] = 1u; got = 5; break; }
                __builtin_amdgcn_s_sleep(CFG_GS);
            }
            ctl[2] = (unsigned)got;
            if (first) __builtin_amdgcn_fence(__ATOMIC_ACQUIRE, "agent");
            asm volatile("s_waitcnt vmcnt(0) lgkmcnt(0)" ::: "memory");
        }
        asm volatile("" ::: "memory"); __builtin_amdgcn_s_barrier(); asm volatile("" ::: "memory");
        const int r = __builtin_amdgcn_readfirstlane(ctl[2]);
        asm volatile("s_waitcnt lgkmcnt(0)" ::: "memory");
        return r;
    };

#define SA(b, h) (((b) * 2 + (h)) * HTB)
#define SB(b, h) ((4 + (b) * 2 + (h)) * HTB)
#define STAGE(bufoff, soff) do { _Pragma("unroll") for (int _i = 0; _i < 2; ++_i) \
        __builtin_amdgcn_raw_ptr_buffer_load_lds(rws, (LAS unsigned*)(lds + (bufoff) + ldsw + _i * 8192), 16, voff, (soff) + _i * 8192u, 0, CFG_LDA); } while (0)
#define LDA(dst, b, h) do { _Pragma("unroll") for (int m = 0; m < 4; ++m) _Pragma("unroll") for (int k = 0; k < 2; ++k) dst[m][k] = *(const LAS bf16x8*)(lds + SA(b, h) + aoff + m * 2048 + k * 1024); } while (0)
#define LDB(dst, b, h) do { _Pragma("unroll") for (int n = 0; n < 2; ++n) _Pragma("unroll") for (int k = 0; k < 2; ++k) dst[n][k] = *(const LAS bf16x8*)(ldb + (SB(b, h) - SB(0, 0)) + n * 2048 + k * 1024); } while (0)
#define MMA(ai, first, prio, extra) do { __builtin_amdgcn_s_setprio(prio); \
        _Pragma("unroll") for (int i = 0; i < 16; ++i) { \
            const int bj = ((i >> 1) ^ (i >> 2)) & 1, m = i >> 2, n = (i ^ (i >> 2)) & 1, k = i & 1; \
            if (first) asm volatile("v_mfma_f32_16x16x32_bf16 %0, %1, %2, 0\n\tv_mfma_f32_16x16x32_bf16 %0, %3, %4, %0" : "=&v"(acc[ai][bj][m][n]) \
                         : "v"(bj ? B1[n][k] : B0[n][k]), "v"(At[m][k]), "v"(bj ? B1[n][k ^ 1] : B0[n][k ^ 1]), "v"(At[m][k ^ 1])); \
            else asm volatile("v_mfma_f32_16x16x32_bf16 %0, %1, %2, %0\n\tv_mfma_f32_16x16x32_bf16 %0, %3, %4, %0" : "+v"(acc[ai][bj][m][n]) \
                         : "v"(bj ? B1[n][k] : B0[n][k]), "v"(At[m][k]), "v"(bj ? B1[n][k ^ 1] : B0[n][k ^ 1]), "v"(At[m][k ^ 1])); } \
        extra; \
        __builtin_amdgcn_s_setprio(0); } while (0)
#define WAIT_L0() do { asm volatile("" ::: "memory"); __builtin_amdgcn_s_waitcnt(0xC07F); asm volatile("" ::: "memory"); } while (0)
#define WAIT_V(n) asm volatile("s_waitcnt vmcnt(" #n ")" ::: "memory")
#define WAIT_L(n) asm volatile("s_waitcnt lgkmcnt(" #n ")" ::: "memory")
#define BAR __builtin_amdgcn_s_barrier()
#define SCHED __builtin_amdgcn_sched_barrier(0)

    LAS unsigned char* ldb = lds + SB(0, 0) + boff;
    asm volatile("" : "+v"(ldb));
    unsigned long long outp = (unsigned long long)P.outl;
    asm volatile("" : "+v"(outp));
    const int np = CFG_NP >> col * 4 & 15;
    const unsigned snd = WS_SND + (call & 1u) * SNDB;
    auto after = [&](int r) { return r == -1 && slot >= R0 ? 1 : r + 1; };
    auto fixed = [&](int r, Tile& t) {
        const int n = r < 0 ? r + np + pr0_of(np) : r;
        tile_of(n ? R0 + (n - 1) * kx + slot : slot, col, rank, snd, t);
        t.kind = r < 0 ? HEAD : (unsigned)(r - pr0_of(np)) < (unsigned)np ? REST : WHOLE;
    };
    Tile cur, nxt; int rnd = after(-np - 1);
    fixed(rnd, cur);
    bf16x8 At[4][2], B0[2][2], B1[2][2];
    constexpr unsigned SB0 = (NRANK * NB - NS + GW) * UNB, STB = UQ * UNB;
    unsigned tabA, tabB;
    auto tables = [&](const Tile& t) {
        const int ln = lane_now() & 31, j0 = t.kind >> 5 & 31, u = (t.kind & 31) + (ln > j0 ? ln - j0 : 0);
        const int q = (u - 4) * 37 >> 8;
        const unsigned kov = WS_PAN + (unsigned)((rank - 3 + u - 7 * q) & (NRANK - 1)) * CHB + (unsigned)q * UNB;
        tabA = u < 4 ? t.sA + (unsigned)u * t.stA : (unsigned)(NTM + t.v) * PANB + kov;
        tabB = u < 4 ? SB0 + snd + (unsigned)t.tm * UNB + (unsigned)u * STB : (unsigned)t.tm * PANB + kov;
    };
    auto pick = [&]() {
        Tile t = cur; bool ok = true;
        if (rnd + 1 < RS) fixed(after(rnd), t);
        else if (rnd > RS) ok = false;
        else {
            const unsigned vo = lane_now() == 0 ? 0u : 0xfffffff0u, at = WS_COL + (unsigned)(col * 128 + 80 + (rnd + 1 - RS) * 4);
            if ((unsigned)__builtin_amdgcn_readfirstlane(buffer_swap(call, rws, vo, WS_HEAD + 256u + (unsigned)(col * 64 + (rnd + 1 - RS) * 4), 0)) != call) {
                const int d = __builtin_amdgcn_raw_ptr_buffer_atomic_add_i32(kx, rws, vo, WS_HEAD + (call & 1u) * 128u + (unsigned)(rnd + 1 - RS) * 64u, 0);
                __builtin_amdgcn_raw_buffer_store_b32(call * 1024u + (unsigned)d, rws, vo, at, AUX_SYS);
            }
            int m, q;
            u32x2 f = {call * 4u, CUTS};
            const unsigned long long t0 = __builtin_amdgcn_s_memrealtime();
            for (;;) {
                if (NO) f = __builtin_amdgcn_raw_buffer_load_b64(rws, 0, WS_FLAG + 63 * 32 + 16, AUX_SYS);
                q = __builtin_amdgcn_readfirstlane(__builtin_amdgcn_raw_buffer_load_b32(rws, 0, at, AUX_SYS) - call * 1024u);
                m = __builtin_amdgcn_readfirstlane(f[0] - call * 4u);
                if ((unsigned)m < 4u && (unsigned)q < 1024u) break;
                if (give_up((unsigned*)(wsl + WS_TMO), call, t0)) { ctl[0] = 1u; q = 1024; break; }
                __builtin_amdgcn_s_sleep(CFG_PS);
            }
            q += slot + (rnd + 1 - RS) * NG;
            ok = q < (m == 0 ? items_of(0) : m == 1 ? items_of(1) : items_of(2));
            if (ok) item_of(q, m, __builtin_amdgcn_readfirstlane(f[1]), rank, snd, t);
        }
        *(LAS u32x4*)(ctl + 4) = (u32x4){ok ? (unsigned)(t.tm | t.v << 8 | 1 << 16) : 0u, t.sA, t.stA, (unsigned)t.kind};
        __builtin_amdgcn_raw_ptr_buffer_load_lds(rws, ctl + 64, 4, cnt_of(t), 0, 0, AUX_SYS);
    };
    auto picked = [&](int& level) -> bool {
        const u32x4 f = *(const LAS u32x4*)(ctl + 4);
        level = level_of(ctl[64 + lane_now()]);
        const unsigned w = __builtin_amdgcn_readfirstlane(f[0]);
        nxt.tm = w & 255u; nxt.v = (w >> 8) & 255u; nxt.kind = __builtin_amdgcn_readfirstlane(f[3]);
        nxt.sA = __builtin_amdgcn_readfirstlane(f[1]); nxt.stA = __builtin_amdgcn_readfirstlane(f[2]);
        return w != 0u;
    };
    constexpr int JP = KT / 2 - 3, JL = KT / 2 - 1;
    const int jend = wr == 0 && wid == 0 ? JP : JL;
    while (CFG_HT0 && (int)((unsigned)__builtin_amdgcn_s_memrealtime() - t0) < CFG_HT0 + ((col & 3) * 2 + (col >> 2)) * CFG_HT1) __builtin_amdgcn_s_sleep(8);
    int lvl = ready(cur, 1, rnd == 1, true);
    STAMP(P, 2);
    tables(cur);
    unsigned a2 = cur.sA, b2 = SB0 + snd + (unsigned)cur.tm * UNB;
    STAGE(SB(0, 0), b2); STAGE(SA(0, 0), a2); STAGE(SB(0, 1), b2 + HTB); STAGE(SA(0, 1), a2 + HTB);
    STAGE(SB(1, 0), b2 + KTB); STAGE(SA(1, 0), a2 + KTB); STAGE(SB(1, 1), b2 + HTB + KTB);
    WAIT_V(0); BAR;
#define ROUND_DONE() do {} while (0)
    bool has_next = false;
    const unsigned one = (lane >> 4) == ((lane >> 2) & 3) ? (lane & 1 ? 0x3f800000u : 0x3f80u) : 0u;
    u32x4 sel0 = {lane & 2 ? 0u : one, lane & 2 ? one : 0u, 0u, 0u}, sel1 = {0u, 0u, sel0[0], sel0[1]};
    asm volatile("" : "+v"(sel0), "+v"(sel1));
#pragma unroll 1
    for (;;) {
        f32x4 acc[2][2][4][2];
#define ACC(i) acc[(i) >> 4][((i) >> 3) & 1][((i) >> 1) & 3][(i) & 1]
        unsigned prk = WS_PRK + (unsigned)((rnd < 0 ? rnd + np : rnd - pr0_of(np)) * NG + (int)blockIdx.x - (GRID - NG)) * (2 * UNB);
        u32x4 pf[4];
#pragma unroll
        for (int i = 0; i < 4; ++i) asm volatile("" : "=v"(pf[i]));
        int jf = cur.kind == REST ? DBG_JF + slot % CFG_FO : 99, je = -1, fe = 0;
#define FOLD2(a, b, p) "v_mfma_f32_16x16x32_bf16 %" #a ", %8, %" #p ", %" #a "\n\tv_mfma_f32_16x16x32_bf16 %" #b ", %9, %" #p ", %" #b "\n\t"
#define FOLD() do { \
            _Pragma("unroll") for (int c = 0; c < 4; ++c) \
                asm volatile("s_cmp_lg_u32 %14, %15\n\ts_cbranch_scc1 1f\n\t" FOLD2(0, 1, 10) FOLD2(2, 3, 11) FOLD2(4, 5, 12) FOLD2(6, 7, 13) "\n1:" \
                    : "+v"(ACC(c * 8)), "+v"(ACC(c * 8 + 1)), "+v"(ACC(c * 8 + 2)), "+v"(ACC(c * 8 + 3)), "+v"(ACC(c * 8 + 4)), "+v"(ACC(c * 8 + 5)), "+v"(ACC(c * 8 + 6)), "+v"(ACC(c * 8 + 7)) \
                    : "v"(sel0), "v"(sel1), "v"(pf[0]), "v"(pf[1]), "v"(pf[2]), "v"(pf[3]), "s"(fe), "n"(c + 1) : "scc"); \
            if (fe < 4) { \
                _Pragma("unroll") for (int i = 0; i < 4; ++i) pf[i] = __builtin_amdgcn_raw_buffer_load_b128(rws, voff + DBG_OOB, prk + i * 8192u, CFG_ILD); \
                prk += 32768u; je = j; } \
            jf += FS; ++fe; } while (0)
        int jchk = lvl < 5 && cur.kind != HEAD ? 7 * lvl - 4 : jend;
#define SYNC_MMA(ai, first, wv, extra) do { if (wv) WAIT_V(8); WAIT_L0(); if (wr == 1) BAR; MMA(ai, first, wr + 1, extra); if (wr == 0) BAR; SCHED; } while (0)
#define WAIT_X() asm volatile("s_cmp_eq_u32 %0, %1\n\ts_waitcnt vmcnt(%2)\n\ts_cbranch_scc1 1f\n\ts_waitcnt vmcnt(8)\n1:" :: "s"(j), "s"(je), "n"(12) : "scc", "memory")
#define PASS(first, check, extra) do { \
            LDB(B0, 0, 0); LDA(At, 0, 0); LDB(B1, 0, 1); STAGE(SA(1, 1), a2 + KTB + HTB); SCHED; \
            check; \
            a2 = __builtin_amdgcn_readlane(tabA, (j + 1) & 31); b2 = __builtin_amdgcn_readlane(tabB, (j + 1) & 31); \
            SYNC_MMA(0, first, !first, ); \
            LDA(At, 0, 1); STAGE(SB(0, 0), b2); STAGE(SA(0, 0), a2); STAGE(SB(0, 1), b2 + HTB); \
            SYNC_MMA(1, first, !first, extra); \
            LDB(B0, 1, 0); LDA(At, 1, 0); LDB(B1, 1, 1); STAGE(SA(0, 1), a2 + HTB); \
            if (!first) WAIT_X(); \
            SYNC_MMA(0, false, first, ); \
            LDA(At, 1, 1); STAGE(SB(1, 0), b2 + KTB); STAGE(SA(1, 0), a2 + KTB); STAGE(SB(1, 1), b2 + KTB + HTB); \
            if (!first) WAIT_X(); \
            SYNC_MMA(1, false, first, ); } while (0)
        int j = cur.kind >> 5 & 31;
        PASS(true, , );
#pragma unroll 1
        for (++j; j < KT / 2; ++j)
            PASS(false, if (__builtin_expect(j == jchk, 0)) {
                    if (j < JP) { lvl = ready(cur, lvl + 1); jchk = lvl < 5 ? 7 * lvl - 4 : jend; }
                    else if (wr == 0 && j == JP) { pick(); jchk = JL; }
                    else {
                        has_next = picked(lvl);
                        if (has_next) {
                            const int want = nxt.kind >= PIECE ? 5 : nxt.kind == REST ? 2 : 1;
                            const bool join = rnd == -1 && slot >= R0;
                            if (lvl < want || join) lvl = ready(nxt, want, join);
                            tables(nxt);
                        }
                    }
                }, if (__builtin_expect(j == jf, 0)) FOLD());
        asm volatile("s_nop 7" ::: "memory");
        SCHED;
        const int type = cur.kind >> 10 & 7;
        const unsigned pcs = WS_PRK + (unsigned)(np_max() * NG + 2 * (cur.kind >> 13)) * (2 * UNB), pcf = WS_PCF + (unsigned)(cur.kind >> 13) * 16u;
        {
            const int ln = lane_now(), fr = ln & 15, fq = ln >> 4;
            const rsrc_t rol = mk_rsrc((const void*)outp, OUTB);
            const bool shared = cur.v < NS;
            if (__builtin_amdgcn_readfirstlane(ctl[0]) != 0u) {
#pragma unroll
                for (int i = 0; i < 32; ++i) acc[i >> 4][(i >> 3) & 1][(i >> 1) & 3][i & 1] = __builtin_bit_cast(f32x4, (u32x4){0x7fc00000u, 0x7fc00000u, 0x7fc00000u, 0x7fc00000u});
            }
            if ((unsigned)rnd < (unsigned)RS && wr == 0 && slot == 0 && wid == 0)
                __builtin_amdgcn_raw_buffer_store_b64((u32x2){call, (unsigned)__builtin_amdgcn_s_memrealtime()}, rws, ln == 0 ? 0u : 0xfffffff0u, WS_COL + (unsigned)(col * 128 + rnd * 8), AUX_SYS);
            if (CFG_TCH && (unsigned)(rnd - CFG_TCH) < 2u && slot * 8 + col < SCOL / 8) {
                unsigned lines = voff; asm volatile("" : "+v"(lines)); lines <<= 2;
                __builtin_amdgcn_raw_ptr_buffer_load_lds(rws, ctl + 128, 4, lines, WS_OSN + (unsigned)((rnd - CFG_TCH) * (SCOL / 8) + slot * 8 + col) * 32768u, 0, 0);
            }
            if (CFG_OTC && rnd <= CFG_OTC && rnd > CFG_OTC - CFG_OTN && slot < KX) {
                unsigned lines = voff; asm volatile("" : "+v"(lines)); lines <<= 2;
                const int w = slot * 8 + col;
                __builtin_amdgcn_raw_ptr_buffer_load_lds(rws, ctl + 128, 4, lines, snd + (unsigned)(rnd == CFG_OTC && w < NO * 64 ? 8 * UQ + w : 2 * NUNIT - (CFG_OTC + 1 - rnd) * 8 * KX + (rnd < CFG_OTC) * NO * 64 + w) * 32768u, 0, 0);
            }
            const bool thru = rnd >= RS;
            const unsigned vo = (unsigned)(wc * 32 + fq * 4) * (NN * 2) + (unsigned)(wr * 256 + fr * 16);
            const unsigned so = (unsigned)(cur.tm * 256) * (NN * 2) + (unsigned)col_of(cur.v, rank) * 2u;
            const unsigned vs = (unsigned)(wc * 32 + fq * 4) * 512u + (unsigned)(wr * 256 + fr * 16);
            const unsigned ss = WS_OSN + (unsigned)(cur.v < NB ? NO * NTM + cur.tm * NB + cur.v : (NS - 1 - cur.v) * NTM + cur.tm) * (2 * UNB);
            auto finish = [&](auto mode) {
#pragma unroll
                for (int bj = mode == 4; bj < 2 - (mode == 3); ++bj)
#pragma unroll
                    for (int n = 0; n < 2; ++n)
#pragma unroll
                        for (int r = 0; r < 4; ++r) {
                            u32x4 o;
                            if ((r & 1) == 0) {
#pragma unroll
                                for (int i = 0; i < 8; ++i) { f32x4& a = acc[i >> 2][bj][i & 3][n]; const f32x2 z = silu2((f32x2){a[r], a[r + 1]}); a[r] = z[0]; a[r + 1] = z[1]; }
                            }
#pragma unroll
                            for (int i = 0; i < 4; ++i) o[i] = cvt_pk_bf16(acc[i >> 1][bj][(i & 1) * 2][n][r], acc[i >> 1][bj][(i & 1) * 2 + 1][n][r]);
                            if (mode == 1) STORE16(o, rws, vs, ss + (unsigned)(bj * 128 + n * 16 + r) * 512u, AUX_SYS);
                            STORE16(o, rol, vo, so + (unsigned)(bj * 128 + n * 16 + r) * (NN * 2), mode >= 2 ? CFG_OTH : CFG_OST);
                        }
            };
            auto leave = [&](unsigned to, auto aux, int keep) {
#pragma unroll
                for (int i = 0; i < 16; ++i) {
                    const f32x4 a = ACC(2 * i), b = ACC(2 * i + 1);
                    if ((i >> 2 & 1) != keep) STORE16(((u32x4){cvt_pk_bf16(a[0], a[1]), cvt_pk_bf16(a[2], a[3]), cvt_pk_bf16(b[0], b[1]), cvt_pk_bf16(b[2], b[3])}), rws, voff, to + i * 8192u, aux);
                }
            };
            auto last_wave = [&]() {
                WAIT_VM0();
                unsigned old = 0;
                if (ln == 0) old = __hip_atomic_fetch_add(ctl + 1, 1u, __ATOMIC_RELAXED, __HIP_MEMORY_SCOPE_WORKGROUP);
                return (__builtin_amdgcn_readfirstlane(old) & 7) == 7;
            };
            auto swap = [&](auto half) {
                leave(pcs + 2 * UNB, std::integral_constant<int, AUX_SYS>(), half);
                if (last_wave()) __builtin_amdgcn_raw_buffer_store_b32(call, rws, ln == 0 ? 0u : 0xfffffff0u, pcf + 4u + half * 4u, AUX_SYS);
                u32x4 s0 = sel0, s1 = sel1, z[16];
                const unsigned long long t0 = __builtin_amdgcn_s_memrealtime();
                for (;;) {
                    const u32x4 f = __builtin_amdgcn_raw_buffer_load_b128(rws, 0, pcf, AUX_SYS);
                    if ((unsigned)__builtin_amdgcn_readfirstlane(f[2 - half]) == call && (type < 6 || (unsigned)__builtin_amdgcn_readfirstlane(f[0]) == call)) break;
                    if (give_up((unsigned*)(wsl + WS_TMO), call, t0)) { s0[3] = s1[0] = 0x7fc07fc0u; break; }
                    __builtin_amdgcn_s_sleep(CFG_PS);
                }
                asm volatile("" ::: "memory");
                const unsigned v2 = type < 6 ? 0xfffffff0u : voff;
#pragma unroll
                for (int i = 0; i < 16; ++i) z[i] = __builtin_amdgcn_raw_buffer_load_b128(rws, i < 8 ? v2 : voff, pcs + ((i & 3) + half * 4 + (i >> 2) * 8) * 8192u, AUX_SYS);
#pragma unroll
                for (int i = 0; i < 16; ++i) {
                    const int c = ((i & 3) + half * 4 + (i >> 2 & 1) * 8) * 2;
                    f32x4 &a = ACC(c), &b = ACC(c + 1);
                    asm volatile("s_nop 1\n\tv_mfma_f32_16x16x32_bf16 %0, %2, %4, %0\n\tv_mfma_f32_16x16x32_bf16 %1, %3, %4, %1" : "+v"(a), "+v"(b) : "v"(s0), "v"(s1), "v"(z[i]));
                }
                asm volatile("s_nop 7\n\ts_nop 7" ::: "memory");
            };
            if ((type | 2) == 6) swap(std::integral_constant<int, 0>());
            asm volatile("" ::: "memory");
            if ((now_s(type) | 2) == 7) swap(std::integral_constant<int, 1>());
            SCHED;
            const int ty = now_s(type);
            if (rnd < 0) leave(prk, std::integral_constant<int, CFG_IST>(), 2);
            else if (type == 3) leave(pcs, std::integral_constant<int, AUX_SYS>(), 2);
            else if (ty > 3 && (ty & 1)) finish(std::integral_constant<int, 4>());
            else if (ty > 3) finish(std::integral_constant<int, 3>());
            else if (shared) finish(std::integral_constant<int, 1>());
            else if (thru) finish(std::integral_constant<int, 2>());
            else finish(std::integral_constant<int, 0>());
            WAIT_V(16);
            if ((shared || type == 3) && last_wave()) {
                if (!shared) __builtin_amdgcn_raw_buffer_store_b32(call, rws, ln == 0 ? 0u : 0xfffffff0u, pcf, AUX_SYS);
                else {
                    const unsigned n = (unsigned)__builtin_amdgcn_raw_ptr_buffer_atomic_add_i32(1, rws, ln == 0 ? 0u : 0xfffffff0u, WS_BAND, 0);
                    if ((unsigned)__builtin_amdgcn_readfirstlane(n) + 1u == call * NLAST && ln < NRANK && ln != rank)
                        st_sys((unsigned*)(P.ws[ln] + WS_FLAG + rank * FLAGB) + NTURN, call);
                }
            }
        }
        ROUND_DONE();
        if (!has_next) break;
        cur = nxt; rnd = after(rnd);
    }
    WAIT_V(0);
}

__global__ __launch_bounds__(512, 2) void fused_gemm_ar(Params P) {
    extern __shared__ __attribute__((aligned(16))) unsigned char shm[];
    LAS unsigned char* lds = (LAS unsigned char*)shm;
    const int wid = __builtin_amdgcn_readfirstlane(threadIdx.x >> 6), lane = threadIdx.x & 63;
    const int b = __builtin_amdgcn_readfirstlane(blockIdx.x);
    const unsigned long long wsp = (unsigned long long)P.ws[lane & (NRANK - 1)];
    const int from = P.order.from[(threadIdx.x + NPUSH) & 255];
    const unsigned t0 = (unsigned)__builtin_amdgcn_s_memrealtime();
    int rk = P.rank;
    asm volatile("" : "+s"(rk) : "s"(P.x), "s"(P.w), "s"(P.wsl), "s"(P.rank), "s"(t0) : "memory");
    unsigned* tkp = (unsigned*)(P.wsl + WS_TICKET + b * 128 + wid * 16);
    const unsigned tk = ld_agent(tkp);
    if (b < NPULL) {
        pull_role(P, tk, tkp, (P.rank + 1 + b / PW) & (NRANK - 1), b % PW * 8 + wid, lane, from, (LAS u32x4*)lds);
    } else {
        const int g = b - NPULL;
        if (g < 4 * NRANK * NL && wid < 2) lead_role(P, rk, wsp, tk, t0, g, wid, lane);
        ((LAS unsigned*)(lds + STAGE_BYTES))[lane & 3] = 0u;
        pack_role(P, wsp, tk, t0, g, wid, lane);
        const unsigned call = __builtin_amdgcn_readfirstlane(tk) + 1u;
        st_agent(tkp, call);
        STAMP(P, 1);
        asm volatile("s_waitcnt lgkmcnt(0)" ::: "memory");
        __builtin_amdgcn_s_barrier();
        if (b >= GRID - NG) {
            const int col = (CFG_SWAP >> (b & 7) * 4) & 7;
            const int slot = b < NPS ? KX : (b - NPS) >> 3, kx = KX + ((b & 7) >= 8 - (NG - 8 * KX));
            if (wid < 4) gemm_role<0>(P, call, col, slot, kx, t0, lds);
            else gemm_role<1>(P, call, col, slot, kx, t0, lds);
        }
    }
}
}

extern "C" void kernel_launch(int rank, int nranks, ncclComm_t comm,
                              hipStream_t stream,
                              void* const* d_in, const int* in_sizes, int n_in,
                              void* d_out, int out_size,
                              const int* in_dtypes, int out_dtype,
                              void* const* d_out_all,
                              void* const* d_ws_all, size_t ws_bytes) {
    if (nranks != NRANK || (unsigned)rank >= (unsigned)NRANK || ws_bytes < (size_t)WS_END || n_in < 2 ||
        in_sizes[0] != MM * KSH || in_sizes[1] != KSH * NN || out_size != MM * NN) {
        fprintf(stderr, "unexpected problem geometry\n");
        return;
    }
    static bool attr_done[NRANK] = {};
    if (!attr_done[rank]) {
        HIP_CHECK(hipFuncSetAttribute((const void*)fused_gemm_ar, hipFuncAttributeMaxDynamicSharedMemorySize, LDS_BYTES));
        attr_done[rank] = true;
    }
    Params P = {};
    P.order = ORDER;
    P.x = d_in[0]; P.w = d_in[1]; P.rank = rank; P.wsl = (char*)d_ws_all[rank]; P.outl = (char*)d_out_all[rank];
    for (int p = 0; p < NRANK; ++p) { P.out[p] = (char*)d_out_all[p]; P.ws[p] = (char*)d_ws_all[p]; }
    hipLaunchKernelGGL(fused_gemm_ar, dim3(GRID), dim3(512), LDS_BYTES, stream, P);
    HIP_CHECK(hipGetLastError());
}
```
